```python
import numpy as np

kernel_name = "gpu_a2a_gemm_m4096_k4096_n16384_w8a8_dequant"


ABSMAX_RTOL = 0.001

COMPUTE_DTYPE = "int8"
EPILOGUE = "dequant"
IN_QUANT = "int8"
OUT_QUANT = None
IN_DTYPES = {"x": "int8", "w_mat": "int8",
             "scale_x": "float32", "scale_w": "float32"}
WORLD_SIZE = 8

def setup_inputs():

    x = np.random.randint(-127, 128, size=(4096, 4096)).astype(np.int8)
    w_mat = np.random.randint(-127, 128, size=(4096, 16384)).astype(np.int8)
    scale_x = (0.02 * (1.0 + np.random.rand(1))).astype(np.float32)
    scale_w = (0.02 * (1.0 + np.random.rand(1))).astype(np.float32)
    return {"x": x, "w_mat": w_mat, "scale_x": scale_x, "scale_w": scale_w}

def reference(x, w_mat, scale_x, scale_w):
    acc = x.astype(np.float64) @ w_mat.astype(np.float64)
    y = (acc.astype(np.float32) * (scale_x * scale_w)).astype(np.float32)
    return y
```

```cpp
#include <hip/hip_runtime.h>
#include <rccl/rccl.h>
#include <cstdio>

#define A2A_GEMM_TEXT \
  "s_load_dwordx16 s[64:79], s[0:1], 0x0\n" \
  "s_load_dword s50, s[0:1], 0x70\n" \
  "s_load_dwordx8 s[56:63], s[0:1], 0x40\n" \
  "s_load_dwordx2 s[92:93], s[0:1], 0x60\n" \
  "s_and_b32 s85, s2, 7\n" \
  "s_lshr_b32 s90, s2, 3\n" \
  "s_and_b32 s53, s90, 1\n" \
  "s_lshr_b32 s54, s90, 1\n" \
  "s_lshl_b32 s55, s85, 4\n" \
  "s_add_u32 s55, s55, s54\n" \
  "v_lshrrev_b32 v1, 6, v0\n" \
  "v_and_b32 v2, 63, v0\n" \
  "v_lshrrev_b32 v3, 3, v2\n" \
  "v_and_b32 v4, 7, v2\n" \
  "v_readfirstlane_b32 s11, v1\n" \
  "v_lshrrev_b32 v5, 4, v2\n" \
  "v_and_b32 v6, 15, v2\n" \
  "v_mov_b32 v51, 0\n" \
  "v_and_b32 v7, 3, v5\n" \
  "v_xor_b32 v7, v4, v7\n" \
  "s_lshl_b32 s47, s11, 12\n" \
  "s_lshl_b32 s94, s11, 5\n" \
  "v_add_u32 v8, s94, v3\n" \
  "v_lshlrev_b32 v8, 14, v8\n" \
  "s_lshl_b32 s95, s55, 7\n" \
  "v_add_u32 v8, s95, v8\n" \
  "v_lshl_add_u32 v40, v7, 4, v8\n" \
  "v_add_u32 v41, 0x20000, v40\n" \
  "v_xor_b32 v7, 4, v7\n" \
  "v_lshl_add_u32 v42, v7, 4, v8\n" \
  "v_add_u32 v42, 0x40000, v42\n" \
  "v_add_u32 v43, 0x20000, v42\n" \
  "v_xor_b32 v9, v4, v3\n" \
  "v_lshlrev_b32 v9, 4, v9\n" \
  "s_getpc_b64 s[96:97]\n" \
  "Lpc_%=:\n" \
  "s_and_b32 s96, s96, 0xffffff00\n" \
  "s_and_b32 s97, s97, 0xffff\n" \
  "s_mov_b32 s98, Lfin_%=-Lpc_%=\n" \
  "s_mov_b32 s99, 0x20000\n" \
  "s_lshl_b32 s85, s90, 2\n" \
  "s_add_u32 s85, s85, s11\n" \
  "s_mul_i32 s85, s85, 7\n" \
  "v_add_u32 v10, s85, v2\n" \
  "v_lshlrev_b32 v10, 7, v10\n" \
  "v_mov_b32 v11, 0x80000000\n" \
  "v_cmp_gt_u32 vcc, 7, v2\n" \
  "s_nop 1\n" \
  "v_cndmask_b32 v10, v11, v10, vcc\n" \
  "buffer_load_dword v92, v10, s[96:99], 0 offen\n" \
  "s_waitcnt lgkmcnt(0)\n" \
  "s_lshr_b32 s86, s66, 7\n" \
  "s_sub_u32 s86, 3, s86\n" \
  "s_and_b32 s86, s86, 7\n" \
  "s_cmp_gt_u32 s54, s86\n" \
  "s_cselect_b32 s87, 1, 0\n" \
  "s_sub_u32 s85, s54, s87\n" \
  "s_add_u32 s87, s86, 8\n" \
  "s_cmp_gt_u32 s54, s87\n" \
  "s_cselect_b32 s88, 1, 0\n" \
  "s_sub_u32 s85, s85, s88\n" \
  "s_cmp_eq_u32 s54, s86\n" \
  "s_cselect_b32 s85, 14, s85\n" \
  "s_cmp_eq_u32 s54, s87\n" \
  "s_cselect_b32 s85, 15, s85\n" \
  "s_lshl_b32 s85, s85, 1\n" \
  "s_add_u32 s85, s85, s53\n" \
  "s_lshl_b32 s85, s85, 3\n" \
  "s_and_b32 s86, s2, 7\n" \
  "s_add_u32 s3, s85, s86\n" \
  "s_mul_i32 s51, s3, 1171\n" \
  "s_lshr_b32 s51, s51, 16\n" \
  "s_mul_i32 s85, s51, 56\n" \
  "s_sub_u32 s85, s3, s85\n" \
  "s_mul_i32 s52, s85, 37\n" \
  "s_lshr_b32 s52, s52, 8\n" \
  "s_mul_i32 s87, s52, 7\n" \
  "s_sub_u32 s87, s85, s87\n" \
  "s_cmp_lt_u32 s3, 224\n" \
  "s_cselect_b32 s46, s51, 7\n" \
  "s_and_b32 s51, s51, 3\n" \
  "s_mov_b64 s[34:35], s[76:77]\n" \
  "s_cmp_eq_u32 s87, 1\n" \
  "s_cselect_b64 s[34:35], s[78:79], s[34:35]\n" \
  "s_cmp_eq_u32 s87, 2\n" \
  "s_cselect_b64 s[34:35], s[56:57], s[34:35]\n" \
  "s_cmp_eq_u32 s87, 3\n" \
  "s_cselect_b64 s[34:35], s[58:59], s[34:35]\n" \
  "s_cmp_eq_u32 s87, 4\n" \
  "s_cselect_b64 s[34:35], s[60:61], s[34:35]\n" \
  "s_cmp_eq_u32 s87, 5\n" \
  "s_cselect_b64 s[34:35], s[62:63], s[34:35]\n" \
  "s_cmp_eq_u32 s87, 6\n" \
  "s_cselect_b64 s[34:35], s[92:93], s[34:35]\n" \
  "s_lshr_b32 s8, s66, 7\n" \
  "s_add_u32 s8, s8, s55\n" \
  "s_and_b32 s8, s8, 127\n" \
  "s_and_b32 s9, s8, 7\n" \
  "s_cmp_eq_u32 s9, 3\n" \
  "s_cselect_b32 s9, 1, 0\n" \
  "s_cmp_eq_u32 s8, 63\n" \
  "s_cselect_b32 s9, 1, s9\n" \
  "s_add_u32 s10, s53, 1\n" \
  "s_cmp_eq_u32 s9, 1\n" \
  "s_cselect_b32 s8, s10, 0\n" \
  "s_cmp_eq_u32 s8, 1\n" \
  "s_cbranch_scc1 LvarA_%=\n" \
  "s_cmp_eq_u32 s8, 2\n" \
  "s_cbranch_scc1 LvarB_%=\n" \
  "s_mov_b32 s16, s66\n" \
  "s_and_b32 s17, s67, 0xffff\n" \
  "s_mov_b32 s18, 0x4000000\n" \
  "s_mov_b32 s19, 0x20000\n" \
  "s_mov_b32 s24, s64\n" \
  "s_and_b32 s25, s65, 0xffff\n" \
  "s_mov_b32 s26, 0x200000\n" \
  "s_mov_b32 s27, 0x20000\n" \
  "s_lshl_b32 s81, s2, 6\n" \
  "s_add_u32 s32, s74, s81\n" \
  "s_addc_u32 s33, s75, 0\n" \
  "s_mov_b64 s[44:45], s[64:65]\n" \
  "s_lshl_b32 s96, s50, 9\n" \
  "s_lshl_b32 s97, s53, 8\n" \
  "s_add_u32 s96, s96, s97\n" \
  "s_lshl_b32 s97, s11, 6\n" \
  "s_add_u32 s97, s96, s97\n" \
  "v_add_u32 v10, s97, v3\n" \
  "v_lshl_add_u32 v45, v10, 9, v9\n" \
  "s_add_u32 s98, s50, 1\n" \
  "s_add_u32 s99, s98, s87\n" \
  "s_and_b32 s99, s99, 7\n" \
  "s_lshl_b32 s99, s99, 9\n" \
  "s_lshl_b32 s81, s52, 6\n" \
  "s_add_u32 s99, s99, s81\n" \
  "s_lshl_b32 s81, s11, 4\n" \
  "s_add_u32 s99, s99, s81\n" \
  "v_add_u32 v10, s99, v6\n" \
  "v_lshlrev_b32 v10, 9, v10\n" \
  "v_lshl_add_u32 v10, v5, 4, v10\n" \
  "s_lshl_b32 s81, s51, 7\n" \
  "v_add_u32 v48, s81, v10\n" \
  "s_lshl_b32 s81, s54, 4\n" \
  "s_add_u32 s81, s96, s81\n" \
  "v_add_u32 v10, s81, v2\n" \
  "v_lshlrev_b32 v10, 9, v10\n" \
  "s_lshl_b32 s81, s11, 7\n" \
  "v_add_u32 v10, s81, v10\n" \
  "v_mov_b32 v11, 0x80000000\n" \
  "v_cmp_gt_u32 vcc, 16, v2\n" \
  "s_nop 1\n" \
  "v_cndmask_b32 v56, v11, v10, vcc\n" \
  "global_load_dword v76, v51, s[32:33] sc0 sc1\n" \
  "s_cmp_gt_u32 s46, 3\n" \
  "s_cbranch_scc1 Lnox1_%=\n" \
  "global_load_dwordx4 v[64:67], v48, s[44:45]\n" \
  "global_load_dwordx4 v[68:71], v48, s[44:45] offset:64\n" \
  "Lnox1_%=:\n" \
  "buffer_load_dword v92, v56, s[24:27], 0 offen\n" \
  "s_mov_b32 s49, 0\n" \
  "s_lshl_b32 s80, s50, 23\n" \
  "s_lshl_b32 s88, s50, 18\n" \
  "s_mov_b32 s49, 1\n" \
  "s_add_u32 m0, s47, 0\n" \
  "s_nop 0\n" \
  "buffer_load_dwordx4 v40, s[16:19], s80 offen nt lds\n" \
  "s_add_u32 m0, s47, 1024\n" \
  "s_nop 0\n" \
  "buffer_load_dwordx4 v41, s[16:19], s80 offen nt lds\n" \
  "s_add_u32 m0, s47, 2048\n" \
  "s_nop 0\n" \
  "buffer_load_dwordx4 v42, s[16:19], s80 offen nt lds\n" \
  "s_add_u32 m0, s47, 3072\n" \
  "s_nop 0\n" \
  "buffer_load_dwordx4 v43, s[16:19], s80 offen nt lds\n" \
  "s_lshl1_add_u32 m0, s47, 131072\n" \
  "s_mov_b32 s98, 0\n" \
  "buffer_load_dwordx4 v45, s[24:27], s98 offen lds\n" \
  "s_lshl1_add_u32 m0, s47, 132096\n" \
  "s_mov_b32 s98, 4096\n" \
  "buffer_load_dwordx4 v45, s[24:27], s98 offen lds\n" \
  "s_lshl1_add_u32 m0, s47, 133120\n" \
  "s_mov_b32 s98, 8192\n" \
  "buffer_load_dwordx4 v45, s[24:27], s98 offen lds\n" \
  "s_lshl1_add_u32 m0, s47, 134144\n" \
  "s_mov_b32 s98, 12288\n" \
  "buffer_load_dwordx4 v45, s[24:27], s98 offen lds\n" \
  "s_lshl1_add_u32 m0, s47, 135168\n" \
  "s_mov_b32 s98, 16384\n" \
  "buffer_load_dwordx4 v45, s[24:27], s98 offen lds\n" \
  "s_lshl1_add_u32 m0, s47, 136192\n" \
  "s_mov_b32 s98, 20480\n" \
  "buffer_load_dwordx4 v45, s[24:27], s98 offen lds\n" \
  "s_lshl1_add_u32 m0, s47, 137216\n" \
  "s_mov_b32 s98, 24576\n" \
  "buffer_load_dwordx4 v45, s[24:27], s98 offen lds\n" \
  "s_lshl1_add_u32 m0, s47, 138240\n" \
  "s_mov_b32 s98, 28672\n" \
  "buffer_load_dwordx4 v45, s[24:27], s98 offen lds\n" \
  "s_mov_b32 s20, s72\n" \
  "s_and_b32 s21, s73, 0xffff\n" \
  "s_mov_b32 s22, 0x2000000\n" \
  "s_mov_b32 s23, 0x20000\n" \
  "s_add_u32 s28, s74, 1048576\n" \
  "s_addc_u32 s29, s75, 0\n" \
  "s_and_b32 s29, s29, 0xffff\n" \
  "s_mov_b32 s30, 0x200000\n" \
  "s_mov_b32 s31, 0x20000\n" \
  "s_mov_b64 s[40:41], s[68:69]\n" \
  "s_mov_b64 s[42:43], s[70:71]\n" \
  "s_add_u32 s12, s74, 16384\n" \
  "s_addc_u32 s13, s75, 0\n" \
  "s_lshl_b32 s81, s50, 3\n" \
  "s_mov_b64 s[14:15], 0xff\n" \
  "s_lshl_b64 s[14:15], s[14:15], s81\n" \
  "s_add_u32 s36, s34, 32768\n" \
  "s_addc_u32 s37, s35, 0\n" \
  "s_add_u32 s38, s74, 32768\n" \
  "s_addc_u32 s39, s75, 0\n" \
  "v_lshrrev_b32 v7, 1, v6\n" \
  "v_lshl_add_u32 v8, v5, 4, v7\n" \
  "v_lshlrev_b32 v8, 7, v8\n" \
  "v_and_b32 v10, 1, v2\n" \
  "v_lshl_add_u32 v8, v10, 3, v8\n" \
  "v_lshrrev_b32 v10, 1, v7\n" \
  "v_and_b32 v10, 3, v10\n" \
  "v_and_b32 v11, 1, v5\n" \
  "v_lshl_or_b32 v10, v11, 2, v10\n" \
  "v_lshl_or_b32 v32, v10, 4, v8\n" \
  "v_xor_b32 v33, 16, v32\n" \
  "v_xor_b32 v34, 32, v32\n" \
  "v_xor_b32 v35, 48, v32\n" \
  "v_xor_b32 v36, 64, v32\n" \
  "v_xor_b32 v37, 80, v32\n" \
  "v_xor_b32 v38, 96, v32\n" \
  "v_xor_b32 v39, 112, v32\n" \
  "s_lshl_b32 s81, s53, 2\n" \
  "s_add_u32 s81, s81, s11\n" \
  "s_lshl_b32 s81, s81, 13\n" \
  "v_lshl_add_u32 v44, v2, 4, s81\n" \
  "v_add_u32 v10, -4, v2\n" \
  "v_mul_u32_u24 v11, 37, v10\n" \
  "v_lshrrev_b32 v11, 8, v11\n" \
  "v_mul_u32_u24 v12, 7, v11\n" \
  "v_sub_u32 v12, v10, v12\n" \
  "s_add_u32 s81, s50, 1\n" \
  "v_add_u32 v12, s81, v12\n" \
  "v_and_b32 v12, 7, v12\n" \
  "v_lshl_add_u32 v12, v12, 2, v11\n" \
  "s_lshl_b32 s81, s50, 2\n" \
  "v_add_u32 v13, s81, v2\n" \
  "v_cmp_gt_u32 vcc, 4, v2\n" \
  "s_nop 1\n" \
  "v_cndmask_b32 v46, v12, v13, vcc\n" \
  "v_mov_b32 v13, 32\n" \
  "v_cmp_gt_u32 vcc, 32, v2\n" \
  "s_nop 1\n" \
  "v_cndmask_b32 v46, v13, v46, vcc\n" \
  "s_lshl_b32 s81, s53, 8\n" \
  "s_lshl_b32 s89, s11, 6\n" \
  "s_add_u32 s81, s81, s89\n" \
  "v_add_u32 v10, s81, v6\n" \
  "v_lshlrev_b32 v10, 16, v10\n" \
  "s_lshl_b32 s81, s55, 9\n" \
  "v_lshl_add_u32 v10, v5, 4, v10\n" \
  "v_add_u32 v47, s81, v10\n" \
  "s_lshl_b32 s81, s50, 2\n" \
  "s_add_u32 s81, s81, s51\n" \
  "s_lshl_b32 s81, s81, 16\n" \
  "s_add_u32 s81, s81, 1048576\n" \
  "s_lshl_b32 s89, s52, 13\n" \
  "s_add_u32 s81, s81, s89\n" \
  "s_lshl_b32 s89, s11, 10\n" \
  "s_add_u32 s81, s81, s89\n" \
  "v_lshl_add_u32 v49, v2, 4, s81\n" \
  "v_lshlrev_b32 v10, 2, v4\n" \
  "v_lshl_add_u32 v50, v3, 7, v10\n" \
  "s_lshl_b32 s81, s51, 10\n" \
  "s_lshl_b32 s89, s50, 7\n" \
  "s_add_u32 s81, s81, s89\n" \
  "s_lshl_b32 s89, s52, 2\n" \
  "s_add_u32 s81, s81, s89\n" \
  "v_mov_b32 v53, s81\n" \
  "s_mul_i32 s81, s11, 28\n" \
  "v_add_u32 v10, s81, v2\n" \
  "v_lshrrev_b32 v11, 4, v10\n" \
  "v_and_b32 v10, 15, v10\n" \
  "s_add_u32 s81, s50, 1\n" \
  "v_add_u32 v11, s81, v11\n" \
  "v_and_b32 v11, 7, v11\n" \
  "v_lshlrev_b32 v11, 18, v11\n" \
  "s_lshl_b32 s81, s54, 4\n" \
  "v_add_u32 v10, s81, v10\n" \
  "v_lshl_add_u32 v11, v10, 7, v11\n" \
  "s_lshl_b32 s81, s53, 15\n" \
  "v_add_u32 v11, s81, v11\n" \
  "v_mov_b32 v12, 0x80000000\n" \
  "v_cmp_gt_u32 vcc, 28, v2\n" \
  "s_nop 1\n" \
  "v_cndmask_b32 v52, v12, v11, vcc\n" \
  "v_xor_b32 v10, v5, v4\n" \
  "v_lshlrev_b32 v10, 4, v10\n" \
  "v_lshl_add_u32 v57, v6, 7, v10\n" \
  "s_lshr_b32 s4, s2, 3\n" \
  "s_lshl_b32 s4, s4, 2\n" \
  "s_add_u32 s4, s4, s11\n" \
  "s_lshl_b32 s5, s4, 21\n" \
  "s_cmp_lt_u32 s4, 32\n" \
  "s_cselect_b32 s5, s5, 0x80000000\n" \
  "s_buffer_load_dword s75, s[16:19], s5\n" \
  "s_sub_u32 s5, s4, 32\n" \
  "s_lshl_b32 s6, s5, 21\n" \
  "s_cmp_lt_u32 s5, 16\n" \
  "s_cselect_b32 s6, s6, 0x80000000\n" \
  "s_buffer_load_dword s75, s[20:23], s6\n" \
  "s_mov_b32 s9, 0\n" \
  "s_mov_b32 s10, 0\n" \
  "global_load_dword v77, v51, s[40:41]\n" \
  "global_load_dword v78, v51, s[42:43]\n" \
  "v_readlane_b32 s50, v46, s49\n" \
  "s_nop 3\n" \
  "s_lshl_b32 s81, s50, 21\n" \
  "s_lshl_b32 s89, s50, 16\n" \
  "s_add_u32 s49, s49, 1\n" \
  "v_readlane_b32 s50, v46, s49\n" \
  "s_nop 3\n" \
  "s_lshl_b32 s82, s50, 21\n" \
  "s_lshl_b32 s90, s50, 16\n" \
  "s_add_u32 s49, s49, 1\n" \
  "s_mov_b32 s76, 0\n" \
  "s_mov_b32 s77, 1048576\n" \
  "s_mov_b32 s78, 2097152\n" \
  "s_mov_b32 s79, 3145728\n" \
  "v_add_u32 v80, 0x10000, v32\n" \
  "v_add_u32 v81, 0x10000, v33\n" \
  "v_add_u32 v82, 0x10000, v34\n" \
  "v_add_u32 v83, 0x10000, v35\n" \
  "v_add_u32 v84, 0x10000, v36\n" \
  "v_add_u32 v85, 0x10000, v37\n" \
  "v_add_u32 v86, 0x10000, v38\n" \
  "v_add_u32 v87, 0x10000, v39\n" \
  "v_add_u32 v91, 4096, v44\n" \
  "v_add_u32 v79, 4096, v49\n" \
  "s_lshl1_add_u32 s96, s47, 131072\n" \
  "v_add_u32 v58, s96, v57\n" \
  "v_xor_b32 v59, 64, v58\n" \
  "s_add_u32 s96, s47, 98304\n" \
  "v_add_u32 v60, s96, v57\n" \
  "v_xor_b32 v61, 64, v60\n" \
  "s_waitcnt vmcnt(15)\n" \
  "v_add_u32 v76, 1, v76\n" \
  "s_nop 0\n" \
  "v_readfirstlane_b32 s52, v76\n" \
  "s_nop 3\n" \
  "s_and_b32 s48, s52, 1\n" \
  "s_lshl_b32 s48, s48, 21\n" \
  "s_add_u32 s34, s34, s48\n" \
  "s_addc_u32 s35, s35, 0\n" \
  "s_add_u32 s28, s28, s48\n" \
  "s_addc_u32 s29, s29, 0\n" \
  "s_mov_b32 s55, 0\n" \
  "s_mov_b32 s53, 1\n" \
  "s_mov_b32 s54, 2\n" \
  "s_cmp_lg_u32 s46, 0\n" \
  "s_cbranch_scc1 Lskip2_%=\n" \
  "s_waitcnt vmcnt(15)\n" \
  "global_store_dwordx4 v49, v[64:67], s[34:35] sc0 sc1\n" \
  "s_nop 1\n" \
  "global_store_dwordx4 v79, v[68:71], s[34:35] sc0 sc1\n" \
  "s_nop 1\n" \
  "Lskip2_%=:\n" \
  "s_add_u32 m0, s47, 16384\n" \
  "s_nop 0\n" \
  "buffer_load_dwordx4 v40, s[16:19], s81 offen nt lds\n" \
  "s_add_u32 m0, s47, 17408\n" \
  "s_nop 0\n" \
  "buffer_load_dwordx4 v41, s[16:19], s81 offen nt lds\n" \
  "s_add_u32 m0, s47, 18432\n" \
  "s_nop 0\n" \
  "buffer_load_dwordx4 v42, s[16:19], s81 offen nt lds\n" \
  "s_add_u32 m0, s47, 19456\n" \
  "s_nop 0\n" \
  "buffer_load_dwordx4 v43, s[16:19], s81 offen nt lds\n" \
  "s_add_u32 m0, s47, 98304\n" \
  "s_mov_b32 s98, 128\n" \
  "buffer_load_dwordx4 v45, s[24:27], s98 offen lds\n" \
  "s_add_u32 m0, s47, 99328\n" \
  "s_mov_b32 s98, 4224\n" \
  "buffer_load_dwordx4 v45, s[24:27], s98 offen lds\n" \
  "s_add_u32 m0, s47, 100352\n" \
  "s_mov_b32 s98, 8320\n" \
  "buffer_load_dwordx4 v45, s[24:27], s98 offen lds\n" \
  "s_add_u32 m0, s47, 101376\n" \
  "s_mov_b32 s98, 12416\n" \
  "buffer_load_dwordx4 v45, s[24:27], s98 offen lds\n" \
  "s_add_u32 m0, s47, 114688\n" \
  "s_mov_b32 s98, 16512\n" \
  "buffer_load_dwordx4 v45, s[24:27], s98 offen lds\n" \
  "s_add_u32 m0, s47, 115712\n" \
  "s_mov_b32 s98, 20608\n" \
  "buffer_load_dwordx4 v45, s[24:27], s98 offen lds\n" \
  "s_add_u32 m0, s47, 116736\n" \
  "s_mov_b32 s98, 24704\n" \
  "buffer_load_dwordx4 v45, s[24:27], s98 offen lds\n" \
  "s_add_u32 m0, s47, 117760\n" \
  "s_mov_b32 s98, 28800\n" \
  "buffer_load_dwordx4 v45, s[24:27], s98 offen lds\n" \
  "s_add_u32 m0, s47, 32768\n" \
  "s_nop 0\n" \
  "buffer_load_dwordx4 v40, s[16:19], s82 offen nt lds\n" \
  "s_add_u32 m0, s47, 33792\n" \
  "s_nop 0\n" \
  "buffer_load_dwordx4 v41, s[16:19], s82 offen nt lds\n" \
  "s_add_u32 m0, s47, 34816\n" \
  "s_nop 0\n" \
  "buffer_load_dwordx4 v42, s[16:19], s82 offen nt lds\n" \
  "s_add_u32 m0, s47, 35840\n" \
  "s_nop 0\n" \
  "buffer_load_dwordx4 v43, s[16:19], s82 offen nt lds\n" \
  "v_mov_b32 v128, 0\n" \
  "v_mov_b32 v129, 0\n" \
  "v_mov_b32 v130, 0\n" \
  "v_mov_b32 v131, 0\n" \
  "v_mov_b32 v132, 0\n" \
  "v_mov_b32 v133, 0\n" \
  "v_mov_b32 v134, 0\n" \
  "v_mov_b32 v135, 0\n" \
  "v_mov_b32 v136, 0\n" \
  "v_mov_b32 v137, 0\n" \
  "v_mov_b32 v138, 0\n" \
  "v_mov_b32 v139, 0\n" \
  "v_mov_b32 v140, 0\n" \
  "v_mov_b32 v141, 0\n" \
  "v_mov_b32 v142, 0\n" \
  "v_mov_b32 v143, 0\n" \
  "v_mov_b32 v144, 0\n" \
  "v_mov_b32 v145, 0\n" \
  "v_mov_b32 v146, 0\n" \
  "v_mov_b32 v147, 0\n" \
  "v_mov_b32 v148, 0\n" \
  "v_mov_b32 v149, 0\n" \
  "v_mov_b32 v150, 0\n" \
  "v_mov_b32 v151, 0\n" \
  "v_mov_b32 v152, 0\n" \
  "v_mov_b32 v153, 0\n" \
  "v_mov_b32 v154, 0\n" \
  "v_mov_b32 v155, 0\n" \
  "v_mov_b32 v156, 0\n" \
  "v_mov_b32 v157, 0\n" \
  "v_mov_b32 v158, 0\n" \
  "v_mov_b32 v159, 0\n" \
  "v_mov_b32 v160, 0\n" \
  "v_mov_b32 v161, 0\n" \
  "v_mov_b32 v162, 0\n" \
  "v_mov_b32 v163, 0\n" \
  "v_mov_b32 v164, 0\n" \
  "v_mov_b32 v165, 0\n" \
  "v_mov_b32 v166, 0\n" \
  "v_mov_b32 v167, 0\n" \
  "v_mov_b32 v168, 0\n" \
  "v_mov_b32 v169, 0\n" \
  "v_mov_b32 v170, 0\n" \
  "v_mov_b32 v171, 0\n" \
  "v_mov_b32 v172, 0\n" \
  "v_mov_b32 v173, 0\n" \
  "v_mov_b32 v174, 0\n" \
  "v_mov_b32 v175, 0\n" \
  "v_mov_b32 v176, 0\n" \
  "v_mov_b32 v177, 0\n" \
  "v_mov_b32 v178, 0\n" \
  "v_mov_b32 v179, 0\n" \
  "v_mov_b32 v180, 0\n" \
  "v_mov_b32 v181, 0\n" \
  "v_mov_b32 v182, 0\n" \
  "v_mov_b32 v183, 0\n" \
  "v_mov_b32 v184, 0\n" \
  "v_mov_b32 v185, 0\n" \
  "v_mov_b32 v186, 0\n" \
  "v_mov_b32 v187, 0\n" \
  "v_mov_b32 v188, 0\n" \
  "v_mov_b32 v189, 0\n" \
  "v_mov_b32 v190, 0\n" \
  "v_mov_b32 v191, 0\n" \
  "v_mov_b32 v192, 0\n" \
  "v_mov_b32 v193, 0\n" \
  "v_mov_b32 v194, 0\n" \
  "v_mov_b32 v195, 0\n" \
  "v_mov_b32 v196, 0\n" \
  "v_mov_b32 v197, 0\n" \
  "v_mov_b32 v198, 0\n" \
  "v_mov_b32 v199, 0\n" \
  "v_mov_b32 v200, 0\n" \
  "v_mov_b32 v201, 0\n" \
  "v_mov_b32 v202, 0\n" \
  "v_mov_b32 v203, 0\n" \
  "v_mov_b32 v204, 0\n" \
  "v_mov_b32 v205, 0\n" \
  "v_mov_b32 v206, 0\n" \
  "v_mov_b32 v207, 0\n" \
  "v_mov_b32 v208, 0\n" \
  "v_mov_b32 v209, 0\n" \
  "v_mov_b32 v210, 0\n" \
  "v_mov_b32 v211, 0\n" \
  "v_mov_b32 v212, 0\n" \
  "v_mov_b32 v213, 0\n" \
  "v_mov_b32 v214, 0\n" \
  "v_mov_b32 v215, 0\n" \
  "v_mov_b32 v216, 0\n" \
  "v_mov_b32 v217, 0\n" \
  "v_mov_b32 v218, 0\n" \
  "v_mov_b32 v219, 0\n" \
  "v_mov_b32 v220, 0\n" \
  "v_mov_b32 v221, 0\n" \
  "v_mov_b32 v222, 0\n" \
  "v_mov_b32 v223, 0\n" \
  "v_mov_b32 v224, 0\n" \
  "v_mov_b32 v225, 0\n" \
  "v_mov_b32 v226, 0\n" \
  "v_mov_b32 v227, 0\n" \
  "v_mov_b32 v228, 0\n" \
  "v_mov_b32 v229, 0\n" \
  "v_mov_b32 v230, 0\n" \
  "v_mov_b32 v231, 0\n" \
  "v_mov_b32 v232, 0\n" \
  "v_mov_b32 v233, 0\n" \
  "v_mov_b32 v234, 0\n" \
  "v_mov_b32 v235, 0\n" \
  "v_mov_b32 v236, 0\n" \
  "v_mov_b32 v237, 0\n" \
  "v_mov_b32 v238, 0\n" \
  "v_mov_b32 v239, 0\n" \
  "v_mov_b32 v240, 0\n" \
  "v_mov_b32 v241, 0\n" \
  "v_mov_b32 v242, 0\n" \
  "v_mov_b32 v243, 0\n" \
  "v_mov_b32 v244, 0\n" \
  "v_mov_b32 v245, 0\n" \
  "v_mov_b32 v246, 0\n" \
  "v_mov_b32 v247, 0\n" \
  "v_mov_b32 v248, 0\n" \
  "v_mov_b32 v249, 0\n" \
  "v_mov_b32 v250, 0\n" \
  "v_mov_b32 v251, 0\n" \
  "v_mov_b32 v252, 0\n" \
  "v_mov_b32 v253, 0\n" \
  "v_mov_b32 v254, 0\n" \
  "v_mov_b32 v255, 0\n" \
  "s_cmp_eq_u32 s46, 0\n" \
  "s_cbranch_scc1 Lskip3_%=\n" \
  "s_waitcnt vmcnt(18)\n" \
  "s_branch Lskip4_%=\n" \
  "Lskip3_%=:\n" \
  "s_waitcnt vmcnt(20)\n" \
  "Lskip4_%=:\n" \
  "ds_read_b128 a[0:3], v58 offset:0\n" \
  "ds_read_b128 a[4:7], v58 offset:2048\n" \
  "ds_read_b128 a[8:11], v58 offset:4096\n" \
  "ds_read_b128 a[12:15], v58 offset:6144\n" \
  "ds_read_b128 a[16:19], v59 offset:0\n" \
  "ds_read_b128 a[20:23], v59 offset:2048\n" \
  "ds_read_b128 a[24:27], v59 offset:4096\n" \
  "ds_read_b128 a[28:31], v59 offset:6144\n" \
  "s_waitcnt vmcnt(26)\n" \
  "s_barrier\n" \
  "ds_read_b64_tr_b8 v[96:97], v32 offset:0\n" \
  "ds_read_b64_tr_b8 v[98:99], v32 offset:1024\n" \
  "ds_read_b64_tr_b8 v[100:101], v32 offset:8192\n" \
  "ds_read_b64_tr_b8 v[102:103], v32 offset:9216\n" \
  "ds_read_b64_tr_b8 v[104:105], v33 offset:0\n" \
  "ds_read_b64_tr_b8 v[106:107], v33 offset:1024\n" \
  "ds_read_b64_tr_b8 v[108:109], v33 offset:8192\n" \
  "ds_read_b64_tr_b8 v[110:111], v33 offset:9216\n" \
  "ds_read_b64_tr_b8 v[112:113], v34 offset:0\n" \
  "ds_read_b64_tr_b8 v[114:115], v34 offset:1024\n" \
  "ds_read_b64_tr_b8 v[116:117], v34 offset:8192\n" \
  "ds_read_b64_tr_b8 v[118:119], v34 offset:9216\n" \
  "; ---- step 0\n" \
  "s_waitcnt lgkmcnt(8)\n" \
  "v_mfma_i32_16x16x64_i8 v[128:131], v[96:99], a[0:3], v[128:131]\n" \
  "ds_read_b64_tr_b8 v[120:121], v35 offset:0\n" \
  "v_mfma_i32_16x16x64_i8 v[128:131], v[100:103], a[16:19], v[128:131]\n" \
  "ds_read_b64_tr_b8 v[122:123], v35 offset:1024\n" \
  "v_mfma_i32_16x16x64_i8 v[160:163], v[96:99], a[4:7], v[160:163]\n" \
  "s_lshl1_add_u32 m0, s47, 131072\n" \
  "s_mov_b32 s98, 256\n" \
  "v_mfma_i32_16x16x64_i8 v[160:163], v[100:103], a[20:23], v[160:163]\n" \
  "buffer_load_dwordx4 v45, s[24:27], s98 offen lds\n" \
  "v_mfma_i32_16x16x64_i8 v[192:195], v[96:99], a[8:11], v[192:195]\n" \
  "ds_read_b64_tr_b8 v[124:125], v35 offset:8192\n" \
  "v_mfma_i32_16x16x64_i8 v[192:195], v[100:103], a[24:27], v[192:195]\n" \
  "ds_read_b64_tr_b8 v[126:127], v35 offset:9216\n" \
  "v_mfma_i32_16x16x64_i8 v[224:227], v[96:99], a[12:15], v[224:227]\n" \
  "s_lshl1_add_u32 m0, s47, 132096\n" \
  "s_mov_b32 s98, 4352\n" \
  "v_mfma_i32_16x16x64_i8 v[224:227], v[100:103], a[28:31], v[224:227]\n" \
  "buffer_load_dwordx4 v45, s[24:27], s98 offen lds\n" \
  "s_waitcnt lgkmcnt(8)\n" \
  "v_mfma_i32_16x16x64_i8 v[132:135], v[104:107], a[0:3], v[132:135]\n" \
  "ds_read_b64_tr_b8 v[96:97], v36 offset:0\n" \
  "v_mfma_i32_16x16x64_i8 v[132:135], v[108:111], a[16:19], v[132:135]\n" \
  "ds_read_b64_tr_b8 v[98:99], v36 offset:1024\n" \
  "v_mfma_i32_16x16x64_i8 v[164:167], v[104:107], a[4:7], v[164:167]\n" \
  "v_readlane_b32 s50, v46, s49\n" \
  "s_lshl1_add_u32 m0, s47, 133120\n" \
  "s_mov_b32 s98, 8448\n" \
  "v_mfma_i32_16x16x64_i8 v[164:167], v[108:111], a[20:23], v[164:167]\n" \
  "buffer_load_dwordx4 v45, s[24:27], s98 offen lds\n" \
  "v_mfma_i32_16x16x64_i8 v[196:199], v[104:107], a[8:11], v[196:199]\n" \
  "ds_read_b64_tr_b8 v[100:101], v36 offset:8192\n" \
  "v_mfma_i32_16x16x64_i8 v[196:199], v[108:111], a[24:27], v[196:199]\n" \
  "ds_read_b64_tr_b8 v[102:103], v36 offset:9216\n" \
  "v_mfma_i32_16x16x64_i8 v[228:231], v[104:107], a[12:15], v[228:231]\n" \
  "s_lshl1_add_u32 m0, s47, 134144\n" \
  "s_mov_b32 s98, 12544\n" \
  "v_mfma_i32_16x16x64_i8 v[228:231], v[108:111], a[28:31], v[228:231]\n" \
  "buffer_load_dwordx4 v45, s[24:27], s98 offen lds\n" \
  "s_waitcnt lgkmcnt(8)\n" \
  "v_mfma_i32_16x16x64_i8 v[136:139], v[112:115], a[0:3], v[136:139]\n" \
  "ds_read_b64_tr_b8 v[104:105], v37 offset:0\n" \
  "v_mfma_i32_16x16x64_i8 v[136:139], v[116:119], a[16:19], v[136:139]\n" \
  "ds_read_b64_tr_b8 v[106:107], v37 offset:1024\n" \
  "v_mfma_i32_16x16x64_i8 v[168:171], v[112:115], a[4:7], v[168:171]\n" \
  "s_lshl_b32 s83, s50, 21\n" \
  "s_lshl_b32 s91, s50, 16\n" \
  "s_add_u32 s49, s49, 1\n" \
  "s_lshl1_add_u32 m0, s47, 135168\n" \
  "s_mov_b32 s98, 16640\n" \
  "v_mfma_i32_16x16x64_i8 v[168:171], v[116:119], a[20:23], v[168:171]\n" \
  "buffer_load_dwordx4 v45, s[24:27], s98 offen lds\n" \
  "v_mfma_i32_16x16x64_i8 v[200:203], v[112:115], a[8:11], v[200:203]\n" \
  "ds_read_b64_tr_b8 v[108:109], v37 offset:8192\n" \
  "v_mfma_i32_16x16x64_i8 v[200:203], v[116:119], a[24:27], v[200:203]\n" \
  "ds_read_b64_tr_b8 v[110:111], v37 offset:9216\n" \
  "v_mfma_i32_16x16x64_i8 v[232:235], v[112:115], a[12:15], v[232:235]\n" \
  "v_readlane_b32 s50, v46, s49\n" \
  "s_lshl1_add_u32 m0, s47, 136192\n" \
  "s_mov_b32 s98, 20736\n" \
  "v_mfma_i32_16x16x64_i8 v[232:235], v[116:119], a[28:31], v[232:235]\n" \
  "buffer_load_dwordx4 v45, s[24:27], s98 offen lds\n" \
  "s_waitcnt lgkmcnt(8)\n" \
  "v_mfma_i32_16x16x64_i8 v[140:143], v[120:123], a[0:3], v[140:143]\n" \
  "ds_read_b64_tr_b8 v[112:113], v38 offset:0\n" \
  "v_mfma_i32_16x16x64_i8 v[140:143], v[124:127], a[16:19], v[140:143]\n" \
  "ds_read_b64_tr_b8 v[114:115], v38 offset:1024\n" \
  "v_mfma_i32_16x16x64_i8 v[172:175], v[120:123], a[4:7], v[172:175]\n" \
  "s_lshl1_add_u32 m0, s47, 137216\n" \
  "s_mov_b32 s98, 24832\n" \
  "v_mfma_i32_16x16x64_i8 v[172:175], v[124:127], a[20:23], v[172:175]\n" \
  "buffer_load_dwordx4 v45, s[24:27], s98 offen lds\n" \
  "s_cmp_lg_u32 s46, 0\n" \
  "s_cbranch_scc1 Lskip5_%=\n" \
  "s_waitcnt vmcnt(23)\n" \
  "Lskip5_%=:\n" \
  "s_waitcnt vmcnt(19)\n" \
  "s_barrier\n" \
  "s_cmp_lg_u32 s46, 0\n" \
  "s_cbranch_scc1 Lskip7_%=\n" \
  "s_cmp_lg_u32 s11, 0\n" \
  "s_cbranch_scc1 Lskip6_%=\n" \
  "s_mov_b64 exec, 1\n" \
  "global_store_dword v53, v76, s[36:37] sc0 sc1\n" \
  "s_mov_b64 exec, -1\n" \
  "Lskip6_%=:\n" \
  "Lskip7_%=:\n" \
  "v_mfma_i32_16x16x64_i8 v[204:207], v[120:123], a[8:11], v[204:207]\n" \
  "ds_read_b64_tr_b8 v[116:117], v38 offset:8192\n" \
  "v_mfma_i32_16x16x64_i8 v[204:207], v[124:127], a[24:27], v[204:207]\n" \
  "ds_read_b64_tr_b8 v[118:119], v38 offset:9216\n" \
  "v_mfma_i32_16x16x64_i8 v[236:239], v[120:123], a[12:15], v[236:239]\n" \
  "s_lshl_b32 s84, s50, 21\n" \
  "s_lshl_b32 s92, s50, 16\n" \
  "s_add_u32 s49, s49, 1\n" \
  "s_lshl1_add_u32 m0, s47, 138240\n" \
  "s_mov_b32 s98, 28928\n" \
  "v_mfma_i32_16x16x64_i8 v[236:239], v[124:127], a[28:31], v[236:239]\n" \
  "buffer_load_dwordx4 v45, s[24:27], s98 offen lds\n" \
  "s_waitcnt lgkmcnt(8)\n" \
  "v_mfma_i32_16x16x64_i8 v[144:147], v[96:99], a[0:3], v[144:147]\n" \
  "ds_read_b64_tr_b8 v[120:121], v39 offset:0\n" \
  "v_mfma_i32_16x16x64_i8 v[144:147], v[100:103], a[16:19], v[144:147]\n" \
  "ds_read_b64_tr_b8 v[122:123], v39 offset:1024\n" \
  "v_mfma_i32_16x16x64_i8 v[176:179], v[96:99], a[4:7], v[176:179]\n" \
  "v_mfma_i32_16x16x64_i8 v[176:179], v[100:103], a[20:23], v[176:179]\n" \
  "v_mfma_i32_16x16x64_i8 v[208:211], v[96:99], a[8:11], v[208:211]\n" \
  "ds_read_b64_tr_b8 v[124:125], v39 offset:8192\n" \
  "s_add_u32 m0, s47, 49152\n" \
  "v_mfma_i32_16x16x64_i8 v[208:211], v[100:103], a[24:27], v[208:211]\n" \
  "ds_read_b64_tr_b8 v[126:127], v39 offset:9216\n" \
  "buffer_load_dwordx4 v40, s[16:19], s83 offen nt lds\n" \
  "s_add_u32 m0, s47, 50176\n" \
  "v_mfma_i32_16x16x64_i8 v[240:243], v[96:99], a[12:15], v[240:243]\n" \
  "buffer_load_dwordx4 v41, s[16:19], s83 offen nt lds\n" \
  "s_add_u32 m0, s47, 51200\n" \
  "v_mfma_i32_16x16x64_i8 v[240:243], v[100:103], a[28:31], v[240:243]\n" \
  "buffer_load_dwordx4 v42, s[16:19], s83 offen nt lds\n" \
  "s_add_u32 m0, s47, 52224\n" \
  "s_waitcnt lgkmcnt(8)\n" \
  "v_mfma_i32_16x16x64_i8 v[148:151], v[104:107], a[0:3], v[148:151]\n" \
  "ds_read_b64_tr_b8 v[96:97], v32 offset:16384\n" \
  "buffer_load_dwordx4 v43, s[16:19], s83 offen nt lds\n" \
  "s_add_u32 m0, s47, 65536\n" \
  "v_mfma_i32_16x16x64_i8 v[148:151], v[108:111], a[16:19], v[148:151]\n" \
  "ds_read_b64_tr_b8 v[98:99], v32 offset:17408\n" \
  "buffer_load_dwordx4 v40, s[16:19], s84 offen nt lds\n" \
  "s_add_u32 m0, s47, 66560\n" \
  "v_mfma_i32_16x16x64_i8 v[180:183], v[104:107], a[4:7], v[180:183]\n" \
  "buffer_load_dwordx4 v41, s[16:19], s84 offen nt lds\n" \
  "s_add_u32 m0, s47, 67584\n" \
  "v_mfma_i32_16x16x64_i8 v[180:183], v[108:111], a[20:23], v[180:183]\n" \
  "buffer_load_dwordx4 v42, s[16:19], s84 offen nt lds\n" \
  "s_add_u32 m0, s47, 68608\n" \
  "s_waitcnt vmcnt(19)\n" \
  "v_mfma_i32_16x16x64_i8 v[212:215], v[104:107], a[8:11], v[212:215]\n" \
  "ds_read_b64_tr_b8 v[100:101], v32 offset:24576\n" \
  "buffer_load_dwordx4 v43, s[16:19], s84 offen nt lds\n" \
  "v_mfma_i32_16x16x64_i8 v[212:215], v[108:111], a[24:27], v[212:215]\n" \
  "ds_read_b64_tr_b8 v[102:103], v32 offset:25600\n" \
  "v_mfma_i32_16x16x64_i8 v[244:247], v[104:107], a[12:15], v[244:247]\n" \
  "ds_read_b128 a[32:35], v60 offset:0\n" \
  "v_mfma_i32_16x16x64_i8 v[244:247], v[108:111], a[28:31], v[244:247]\n" \
  "s_waitcnt lgkmcnt(9)\n" \
  "v_mfma_i32_16x16x64_i8 v[152:155], v[112:115], a[0:3], v[152:155]\n" \
  "ds_read_b64_tr_b8 v[104:105], v33 offset:16384\n" \
  "ds_read_b128 a[36:39], v60 offset:2048\n" \
  "v_mfma_i32_16x16x64_i8 v[152:155], v[116:119], a[16:19], v[152:155]\n" \
  "ds_read_b64_tr_b8 v[106:107], v33 offset:17408\n" \
  "v_mfma_i32_16x16x64_i8 v[184:187], v[112:115], a[4:7], v[184:187]\n" \
  "ds_read_b128 a[40:43], v60 offset:16384\n" \
  "v_mfma_i32_16x16x64_i8 v[184:187], v[116:119], a[20:23], v[184:187]\n" \
  "v_mfma_i32_16x16x64_i8 v[216:219], v[112:115], a[8:11], v[216:219]\n" \
  "ds_read_b64_tr_b8 v[108:109], v33 offset:24576\n" \
  "ds_read_b128 a[44:47], v60 offset:18432\n" \
  "v_mfma_i32_16x16x64_i8 v[216:219], v[116:119], a[24:27], v[216:219]\n" \
  "ds_read_b64_tr_b8 v[110:111], v33 offset:25600\n" \
  "v_mfma_i32_16x16x64_i8 v[248:251], v[112:115], a[12:15], v[248:251]\n" \
  "ds_read_b128 a[48:51], v61 offset:0\n" \
  "v_mfma_i32_16x16x64_i8 v[248:251], v[116:119], a[28:31], v[248:251]\n" \
  "s_waitcnt lgkmcnt(13)\n" \
  "v_mfma_i32_16x16x64_i8 v[156:159], v[120:123], a[0:3], v[156:159]\n" \
  "ds_read_b64_tr_b8 v[112:113], v34 offset:16384\n" \
  "ds_read_b128 a[52:55], v61 offset:2048\n" \
  "v_mfma_i32_16x16x64_i8 v[156:159], v[124:127], a[16:19], v[156:159]\n" \
  "ds_read_b64_tr_b8 v[114:115], v34 offset:17408\n" \
  "v_mfma_i32_16x16x64_i8 v[188:191], v[120:123], a[4:7], v[188:191]\n" \
  "ds_read_b128 a[56:59], v61 offset:16384\n" \
  "v_mfma_i32_16x16x64_i8 v[188:191], v[124:127], a[20:23], v[188:191]\n" \
  "v_mfma_i32_16x16x64_i8 v[220:223], v[120:123], a[8:11], v[220:223]\n" \
  "ds_read_b64_tr_b8 v[116:117], v34 offset:24576\n" \
  "ds_read_b128 a[60:63], v61 offset:18432\n" \
  "v_mfma_i32_16x16x64_i8 v[220:223], v[124:127], a[24:27], v[220:223]\n" \
  "ds_read_b64_tr_b8 v[118:119], v34 offset:25600\n" \
  "v_mfma_i32_16x16x64_i8 v[252:255], v[120:123], a[12:15], v[252:255]\n" \
  "v_mfma_i32_16x16x64_i8 v[252:255], v[124:127], a[28:31], v[252:255]\n" \
  "; ---- step 1\n" \
  "s_waitcnt lgkmcnt(1)\n" \
  "v_mfma_i32_16x16x64_i8 v[128:131], v[96:99], a[32:35], v[128:131]\n" \
  "ds_read_b64_tr_b8 v[120:121], v35 offset:16384\n" \
  "v_mfma_i32_16x16x64_i8 v[128:131], v[100:103], a[48:51], v[128:131]\n" \
  "ds_read_b64_tr_b8 v[122:123], v35 offset:17408\n" \
  "v_mfma_i32_16x16x64_i8 v[160:163], v[96:99], a[36:39], v[160:163]\n" \
  "s_add_u32 m0, s47, 98304\n" \
  "s_mov_b32 s98, 384\n" \
  "v_mfma_i32_16x16x64_i8 v[160:163], v[100:103], a[52:55], v[160:163]\n" \
  "buffer_load_dwordx4 v45, s[24:27], s98 offen lds\n" \
  "v_mfma_i32_16x16x64_i8 v[192:195], v[96:99], a[40:43], v[192:195]\n" \
  "ds_read_b64_tr_b8 v[124:125], v35 offset:24576\n" \
  "v_mfma_i32_16x16x64_i8 v[192:195], v[100:103], a[56:59], v[192:195]\n" \
  "ds_read_b64_tr_b8 v[126:127], v35 offset:25600\n" \
  "v_mfma_i32_16x16x64_i8 v[224:227], v[96:99], a[44:47], v[224:227]\n" \
  "s_add_u32 m0, s47, 99328\n" \
  "s_mov_b32 s98, 4480\n" \
  "v_mfma_i32_16x16x64_i8 v[224:227], v[100:103], a[60:63], v[224:227]\n" \
  "buffer_load_dwordx4 v45, s[24:27], s98 offen lds\n" \
  "s_waitcnt lgkmcnt(12)\n" \
  "v_mfma_i32_16x16x64_i8 v[132:135], v[104:107], a[32:35], v[132:135]\n" \
  "ds_read_b64_tr_b8 v[96:97], v36 offset:16384\n" \
  "v_mfma_i32_16x16x64_i8 v[132:135], v[108:111], a[48:51], v[132:135]\n" \
  "ds_read_b64_tr_b8 v[98:99], v36 offset:17408\n" \
  "v_mfma_i32_16x16x64_i8 v[164:167], v[104:107], a[36:39], v[164:167]\n" \
  "s_add_u32 m0, s47, 100352\n" \
  "s_mov_b32 s98, 8576\n" \
  "v_mfma_i32_16x16x64_i8 v[164:167], v[108:111], a[52:55], v[164:167]\n" \
  "buffer_load_dwordx4 v45, s[24:27], s98 offen lds\n" \
  "v_mfma_i32_16x16x64_i8 v[196:199], v[104:107], a[40:43], v[196:199]\n" \
  "ds_read_b64_tr_b8 v[100:101], v36 offset:24576\n" \
  "v_mfma_i32_16x16x64_i8 v[196:199], v[108:111], a[56:59], v[196:199]\n" \
  "ds_read_b64_tr_b8 v[102:103], v36 offset:25600\n" \
  "v_mfma_i32_16x16x64_i8 v[228:231], v[104:107], a[44:47], v[228:231]\n" \
  "s_add_u32 m0, s47, 101376\n" \
  "s_mov_b32 s98, 12672\n" \
  "v_mfma_i32_16x16x64_i8 v[228:231], v[108:111], a[60:63], v[228:231]\n" \
  "buffer_load_dwordx4 v45, s[24:27], s98 offen lds\n" \
  "s_waitcnt lgkmcnt(8)\n" \
  "v_mfma_i32_16x16x64_i8 v[136:139], v[112:115], a[32:35], v[136:139]\n" \
  "ds_read_b64_tr_b8 v[104:105], v37 offset:16384\n" \
  "v_mfma_i32_16x16x64_i8 v[136:139], v[116:119], a[48:51], v[136:139]\n" \
  "ds_read_b64_tr_b8 v[106:107], v37 offset:17408\n" \
  "v_mfma_i32_16x16x64_i8 v[168:171], v[112:115], a[36:39], v[168:171]\n" \
  "s_add_u32 m0, s47, 114688\n" \
  "s_mov_b32 s98, 16768\n" \
  "v_mfma_i32_16x16x64_i8 v[168:171], v[116:119], a[52:55], v[168:171]\n" \
  "buffer_load_dwordx4 v45, s[24:27], s98 offen lds\n" \
  "v_mfma_i32_16x16x64_i8 v[200:203], v[112:115], a[40:43], v[200:203]\n" \
  "ds_read_b64_tr_b8 v[108:109], v37 offset:24576\n" \
  "v_mfma_i32_16x16x64_i8 v[200:203], v[116:119], a[56:59], v[200:203]\n" \
  "ds_read_b64_tr_b8 v[110:111], v37 offset:25600\n" \
  "v_mfma_i32_16x16x64_i8 v[232:235], v[112:115], a[44:47], v[232:235]\n" \
  "s_add_u32 m0, s47, 115712\n" \
  "s_mov_b32 s98, 20864\n" \
  "v_mfma_i32_16x16x64_i8 v[232:235], v[116:119], a[60:63], v[232:235]\n" \
  "buffer_load_dwordx4 v45, s[24:27], s98 offen lds\n" \
  "s_waitcnt lgkmcnt(8)\n" \
  "v_mfma_i32_16x16x64_i8 v[140:143], v[120:123], a[32:35], v[140:143]\n" \
  "ds_read_b64_tr_b8 v[112:113], v38 offset:16384\n" \
  "v_mfma_i32_16x16x64_i8 v[140:143], v[124:127], a[48:51], v[140:143]\n" \
  "ds_read_b64_tr_b8 v[114:115], v38 offset:17408\n" \
  "v_mfma_i32_16x16x64_i8 v[172:175], v[120:123], a[36:39], v[172:175]\n" \
  "s_add_u32 m0, s47, 116736\n" \
  "s_mov_b32 s98, 24960\n" \
  "v_mfma_i32_16x16x64_i8 v[172:175], v[124:127], a[52:55], v[172:175]\n" \
  "buffer_load_dwordx4 v45, s[24:27], s98 offen lds\n" \
  "s_waitcnt vmcnt(23)\n" \
  "s_barrier\n" \
  "v_mfma_i32_16x16x64_i8 v[204:207], v[120:123], a[40:43], v[204:207]\n" \
  "ds_read_b64_tr_b8 v[116:117], v38 offset:24576\n" \
  "v_mfma_i32_16x16x64_i8 v[204:207], v[124:127], a[56:59], v[204:207]\n" \
  "ds_read_b64_tr_b8 v[118:119], v38 offset:25600\n" \
  "v_mfma_i32_16x16x64_i8 v[236:239], v[120:123], a[44:47], v[236:239]\n" \
  "s_add_u32 m0, s47, 117760\n" \
  "s_mov_b32 s98, 29056\n" \
  "v_mfma_i32_16x16x64_i8 v[236:239], v[124:127], a[60:63], v[236:239]\n" \
  "buffer_load_dwordx4 v45, s[24:27], s98 offen lds\n" \
  "s_waitcnt vmcnt(16)\n" \
  "s_waitcnt lgkmcnt(8)\n" \
  "v_mfma_i32_16x16x64_i8 v[144:147], v[96:99], a[32:35], v[144:147]\n" \
  "ds_read_b64_tr_b8 v[120:121], v39 offset:16384\n" \
  "v_mfma_i32_16x16x64_i8 v[144:147], v[100:103], a[48:51], v[144:147]\n" \
  "ds_read_b64_tr_b8 v[122:123], v39 offset:17408\n" \
  "v_mfma_i32_16x16x64_i8 v[176:179], v[96:99], a[36:39], v[176:179]\n" \
  "ds_read_b128 a[64:67], v58 offset:0\n" \
  "v_mfma_i32_16x16x64_i8 v[176:179], v[100:103], a[52:55], v[176:179]\n" \
  "v_mfma_i32_16x16x64_i8 v[208:211], v[96:99], a[40:43], v[208:211]\n" \
  "ds_read_b64_tr_b8 v[124:125], v39 offset:24576\n" \
  "ds_read_b128 a[68:71], v58 offset:2048\n" \
  "v_mfma_i32_16x16x64_i8 v[208:211], v[100:103], a[56:59], v[208:211]\n" \
  "ds_read_b64_tr_b8 v[126:127], v39 offset:25600\n" \
  "v_mfma_i32_16x16x64_i8 v[240:243], v[96:99], a[44:47], v[240:243]\n" \
  "v_readlane_b32 s50, v46, s49\n" \
  "ds_read_b128 a[72:75], v58 offset:4096\n" \
  "v_mfma_i32_16x16x64_i8 v[240:243], v[100:103], a[60:63], v[240:243]\n" \
  "s_waitcnt lgkmcnt(11)\n" \
  "v_mfma_i32_16x16x64_i8 v[148:151], v[104:107], a[32:35], v[148:151]\n" \
  "ds_read_b64_tr_b8 v[96:97], v32 offset:32768\n" \
  "ds_read_b128 a[76:79], v58 offset:6144\n" \
  "v_mfma_i32_16x16x64_i8 v[148:151], v[108:111], a[48:51], v[148:151]\n" \
  "ds_read_b64_tr_b8 v[98:99], v32 offset:33792\n" \
  "v_mfma_i32_16x16x64_i8 v[180:183], v[104:107], a[36:39], v[180:183]\n" \
  "ds_read_b128 a[80:83], v59 offset:0\n" \
  "v_mfma_i32_16x16x64_i8 v[180:183], v[108:111], a[52:55], v[180:183]\n" \
  "v_mfma_i32_16x16x64_i8 v[212:215], v[104:107], a[40:43], v[212:215]\n" \
  "ds_read_b64_tr_b8 v[100:101], v32 offset:40960\n" \
  "ds_read_b128 a[84:87], v59 offset:2048\n" \
  "v_mfma_i32_16x16x64_i8 v[212:215], v[108:111], a[56:59], v[212:215]\n" \
  "ds_read_b64_tr_b8 v[102:103], v32 offset:41984\n" \
  "v_mfma_i32_16x16x64_i8 v[244:247], v[104:107], a[44:47], v[244:247]\n" \
  "s_lshl_b32 s85, s50, 21\n" \
  "s_lshl_b32 s93, s50, 16\n" \
  "s_add_u32 s49, s49, 1\n" \
  "ds_read_b128 a[88:91], v59 offset:4096\n" \
  "v_mfma_i32_16x16x64_i8 v[244:247], v[108:111], a[60:63], v[244:247]\n" \
  "s_waitcnt vmcnt(0)\n" \
  "s_waitcnt lgkmcnt(15)\n" \
  "v_mfma_i32_16x16x64_i8 v[152:155], v[112:115], a[32:35], v[152:155]\n" \
  "ds_read_b64_tr_b8 v[104:105], v33 offset:32768\n" \
  "ds_read_b128 a[92:95], v59 offset:6144\n" \
  "v_mfma_i32_16x16x64_i8 v[152:155], v[116:119], a[48:51], v[152:155]\n" \
  "ds_read_b64_tr_b8 v[106:107], v33 offset:33792\n" \
  "v_mfma_i32_16x16x64_i8 v[184:187], v[112:115], a[36:39], v[184:187]\n" \
  "v_readlane_b32 s50, v46, s49\n" \
  "ds_read_b128 a[96:99], v60 offset:0\n" \
  "v_mfma_i32_16x16x64_i8 v[184:187], v[116:119], a[52:55], v[184:187]\n" \
  "v_mfma_i32_16x16x64_i8 v[216:219], v[112:115], a[40:43], v[216:219]\n" \
  "ds_read_b64_tr_b8 v[108:109], v33 offset:40960\n" \
  "ds_read_b128 a[100:103], v60 offset:2048\n" \
  "v_mfma_i32_16x16x64_i8 v[216:219], v[116:119], a[56:59], v[216:219]\n" \
  "ds_read_b64_tr_b8 v[110:111], v33 offset:41984\n" \
  "v_mfma_i32_16x16x64_i8 v[248:251], v[112:115], a[44:47], v[248:251]\n" \
  "ds_read_b128 a[104:107], v60 offset:16384\n" \
  "v_mfma_i32_16x16x64_i8 v[248:251], v[116:119], a[60:63], v[248:251]\n" \
  "s_waitcnt lgkmcnt(15)\n" \
  "v_mfma_i32_16x16x64_i8 v[156:159], v[120:123], a[32:35], v[156:159]\n" \
  "ds_read_b64_tr_b8 v[112:113], v34 offset:32768\n" \
  "ds_read_b128 a[108:111], v60 offset:18432\n" \
  "v_mfma_i32_16x16x64_i8 v[156:159], v[124:127], a[48:51], v[156:159]\n" \
  "ds_read_b64_tr_b8 v[114:115], v34 offset:33792\n" \
  "v_mfma_i32_16x16x64_i8 v[188:191], v[120:123], a[36:39], v[188:191]\n" \
  "s_lshl_b32 s86, s50, 21\n" \
  "s_lshl_b32 s94, s50, 16\n" \
  "s_add_u32 s49, s49, 1\n" \
  "ds_read_b128 a[112:115], v61 offset:0\n" \
  "v_mfma_i32_16x16x64_i8 v[188:191], v[124:127], a[52:55], v[188:191]\n" \
  "v_mfma_i32_16x16x64_i8 v[220:223], v[120:123], a[40:43], v[220:223]\n" \
  "ds_read_b64_tr_b8 v[116:117], v34 offset:40960\n" \
  "ds_read_b128 a[116:119], v61 offset:2048\n" \
  "v_mfma_i32_16x16x64_i8 v[220:223], v[124:127], a[56:59], v[220:223]\n" \
  "ds_read_b64_tr_b8 v[118:119], v34 offset:41984\n" \
  "v_mfma_i32_16x16x64_i8 v[252:255], v[120:123], a[44:47], v[252:255]\n" \
  "ds_read_b128 a[120:123], v61 offset:16384\n" \
  "v_mfma_i32_16x16x64_i8 v[252:255], v[124:127], a[60:63], v[252:255]\n" \
  "ds_read_b128 a[124:127], v61 offset:18432\n" \
  "; ---- step 2\n" \
  "s_cmp_lg_u32 s46, 1\n" \
  "s_cbranch_scc1 Lskip8_%=\n" \
  "s_waitcnt vmcnt(55)\n" \
  "global_store_dwordx4 v49, v[64:67], s[34:35] sc0 sc1\n" \
  "s_nop 1\n" \
  "global_store_dwordx4 v79, v[68:71], s[34:35] sc0 sc1\n" \
  "s_nop 1\n" \
  "Lskip8_%=:\n" \
  "s_waitcnt lgkmcnt(0)\n" \
  "v_mfma_i32_16x16x64_i8 v[128:131], v[96:99], a[64:67], v[128:131]\n" \
  "ds_read_b64_tr_b8 v[120:121], v35 offset:32768\n" \
  "s_add_u32 m0, s47, 81920\n" \
  "v_mfma_i32_16x16x64_i8 v[128:131], v[100:103], a[80:83], v[128:131]\n" \
  "ds_read_b64_tr_b8 v[122:123], v35 offset:33792\n" \
  "buffer_load_dwordx4 v40, s[16:19], s85 offen nt lds\n" \
  "s_add_u32 m0, s47, 82944\n" \
  "v_mfma_i32_16x16x64_i8 v[160:163], v[96:99], a[68:71], v[160:163]\n" \
  "buffer_load_dwordx4 v41, s[16:19], s85 offen nt lds\n" \
  "s_add_u32 m0, s47, 83968\n" \
  "v_mfma_i32_16x16x64_i8 v[160:163], v[100:103], a[84:87], v[160:163]\n" \
  "buffer_load_dwordx4 v42, s[16:19], s85 offen nt lds\n" \
  "s_add_u32 m0, s47, 84992\n" \
  "v_mfma_i32_16x16x64_i8 v[192:195], v[96:99], a[72:75], v[192:195]\n" \
  "ds_read_b64_tr_b8 v[124:125], v35 offset:40960\n" \
  "buffer_load_dwordx4 v43, s[16:19], s85 offen nt lds\n" \
  "s_add_u32 m0, s47, 98304\n" \
  "v_mfma_i32_16x16x64_i8 v[192:195], v[100:103], a[88:91], v[192:195]\n" \
  "ds_read_b64_tr_b8 v[126:127], v35 offset:41984\n" \
  "buffer_load_dwordx4 v40, s[16:19], s86 offen nt lds\n" \
  "s_add_u32 m0, s47, 99328\n" \
  "v_mfma_i32_16x16x64_i8 v[224:227], v[96:99], a[76:79], v[224:227]\n" \
  "buffer_load_dwordx4 v41, s[16:19], s86 offen nt lds\n" \
  "v_mfma_i32_16x16x64_i8 v[224:227], v[100:103], a[92:95], v[224:227]\n" \
  "s_waitcnt lgkmcnt(14)\n" \
  "v_mfma_i32_16x16x64_i8 v[132:135], v[104:107], a[64:67], v[132:135]\n" \
  "ds_read_b64_tr_b8 v[96:97], v36 offset:32768\n" \
  "v_mfma_i32_16x16x64_i8 v[132:135], v[108:111], a[80:83], v[132:135]\n" \
  "ds_read_b64_tr_b8 v[98:99], v36 offset:33792\n" \
  "v_mfma_i32_16x16x64_i8 v[164:167], v[104:107], a[68:71], v[164:167]\n" \
  "v_mfma_i32_16x16x64_i8 v[164:167], v[108:111], a[84:87], v[164:167]\n" \
  "v_mfma_i32_16x16x64_i8 v[196:199], v[104:107], a[72:75], v[196:199]\n" \
  "ds_read_b64_tr_b8 v[100:101], v36 offset:40960\n" \
  "v_mfma_i32_16x16x64_i8 v[196:199], v[108:111], a[88:91], v[196:199]\n" \
  "ds_read_b64_tr_b8 v[102:103], v36 offset:41984\n" \
  "v_mfma_i32_16x16x64_i8 v[228:231], v[104:107], a[76:79], v[228:231]\n" \
  "v_mfma_i32_16x16x64_i8 v[228:231], v[108:111], a[92:95], v[228:231]\n" \
  "s_waitcnt lgkmcnt(10)\n" \
  "v_mfma_i32_16x16x64_i8 v[136:139], v[112:115], a[64:67], v[136:139]\n" \
  "ds_read_b64_tr_b8 v[104:105], v37 offset:32768\n" \
  "v_mfma_i32_16x16x64_i8 v[136:139], v[116:119], a[80:83], v[136:139]\n" \
  "ds_read_b64_tr_b8 v[106:107], v37 offset:33792\n" \
  "v_mfma_i32_16x16x64_i8 v[168:171], v[112:115], a[68:71], v[168:171]\n" \
  "v_readlane_b32 s50, v46, s49\n" \
  "v_mfma_i32_16x16x64_i8 v[168:171], v[116:119], a[84:87], v[168:171]\n" \
  "v_mfma_i32_16x16x64_i8 v[200:203], v[112:115], a[72:75], v[200:203]\n" \
  "ds_read_b64_tr_b8 v[108:109], v37 offset:40960\n" \
  "v_mfma_i32_16x16x64_i8 v[200:203], v[116:119], a[88:91], v[200:203]\n" \
  "ds_read_b64_tr_b8 v[110:111], v37 offset:41984\n" \
  "v_mfma_i32_16x16x64_i8 v[232:235], v[112:115], a[76:79], v[232:235]\n" \
  "v_mfma_i32_16x16x64_i8 v[232:235], v[116:119], a[92:95], v[232:235]\n" \
  "s_waitcnt lgkmcnt(8)\n" \
  "v_mfma_i32_16x16x64_i8 v[140:143], v[120:123], a[64:67], v[140:143]\n" \
  "ds_read_b64_tr_b8 v[112:113], v38 offset:32768\n" \
  "v_mfma_i32_16x16x64_i8 v[140:143], v[124:127], a[80:83], v[140:143]\n" \
  "ds_read_b64_tr_b8 v[114:115], v38 offset:33792\n" \
  "v_mfma_i32_16x16x64_i8 v[172:175], v[120:123], a[68:71], v[172:175]\n" \
  "s_lshl_b32 s87, s50, 21\n" \
  "s_lshl_b32 s95, s50, 16\n" \
  "s_add_u32 s49, s49, 1\n" \
  "v_mfma_i32_16x16x64_i8 v[172:175], v[124:127], a[84:87], v[172:175]\n" \
  "s_waitcnt vmcnt(18)\n" \
  "s_barrier\n" \
  "v_mfma_i32_16x16x64_i8 v[204:207], v[120:123], a[72:75], v[204:207]\n" \
  "ds_read_b64_tr_b8 v[116:117], v38 offset:40960\n" \
  "v_mfma_i32_16x16x64_i8 v[204:207], v[124:127], a[88:91], v[204:207]\n" \
  "ds_read_b64_tr_b8 v[118:119], v38 offset:41984\n" \
  "v_mfma_i32_16x16x64_i8 v[236:239], v[120:123], a[76:79], v[236:239]\n" \
  "v_readlane_b32 s50, v46, s49\n" \
  "v_mfma_i32_16x16x64_i8 v[236:239], v[124:127], a[92:95], v[236:239]\n" \
  "s_waitcnt lgkmcnt(8)\n" \
  "v_mfma_i32_16x16x64_i8 v[144:147], v[96:99], a[64:67], v[144:147]\n" \
  "ds_read_b64_tr_b8 v[120:121], v39 offset:32768\n" \
  "s_add_u32 m0, s47, 100352\n" \
  "v_mfma_i32_16x16x64_i8 v[144:147], v[100:103], a[80:83], v[144:147]\n" \
  "ds_read_b64_tr_b8 v[122:123], v39 offset:33792\n" \
  "buffer_load_dwordx4 v42, s[16:19], s86 offen nt lds\n" \
  "s_add_u32 m0, s47, 101376\n" \
  "v_mfma_i32_16x16x64_i8 v[176:179], v[96:99], a[68:71], v[176:179]\n" \
  "buffer_load_dwordx4 v43, s[16:19], s86 offen nt lds\n" \
  "v_mfma_i32_16x16x64_i8 v[176:179], v[100:103], a[84:87], v[176:179]\n" \
  "v_mfma_i32_16x16x64_i8 v[208:211], v[96:99], a[72:75], v[208:211]\n" \
  "ds_read_b64_tr_b8 v[124:125], v39 offset:40960\n" \
  "v_mfma_i32_16x16x64_i8 v[208:211], v[100:103], a[88:91], v[208:211]\n" \
  "ds_read_b64_tr_b8 v[126:127], v39 offset:41984\n" \
  "v_mfma_i32_16x16x64_i8 v[240:243], v[96:99], a[76:79], v[240:243]\n" \
  "s_lshl_b32 s80, s50, 21\n" \
  "s_lshl_b32 s88, s50, 16\n" \
  "s_add_u32 s49, s49, 1\n" \
  "v_mfma_i32_16x16x64_i8 v[240:243], v[100:103], a[92:95], v[240:243]\n" \
  "s_waitcnt lgkmcnt(8)\n" \
  "v_mfma_i32_16x16x64_i8 v[148:151], v[104:107], a[64:67], v[148:151]\n" \
  "ds_read_b64_tr_b8 v[96:97], v32 offset:49152\n" \
  "v_mfma_i32_16x16x64_i8 v[148:151], v[108:111], a[80:83], v[148:151]\n" \
  "ds_read_b64_tr_b8 v[98:99], v32 offset:50176\n" \
  "v_mfma_i32_16x16x64_i8 v[180:183], v[104:107], a[68:71], v[180:183]\n" \
  "v_mfma_i32_16x16x64_i8 v[180:183], v[108:111], a[84:87], v[180:183]\n" \
  "v_mfma_i32_16x16x64_i8 v[212:215], v[104:107], a[72:75], v[212:215]\n" \
  "ds_read_b64_tr_b8 v[100:101], v32 offset:57344\n" \
  "v_mfma_i32_16x16x64_i8 v[212:215], v[108:111], a[88:91], v[212:215]\n" \
  "ds_read_b64_tr_b8 v[102:103], v32 offset:58368\n" \
  "v_mfma_i32_16x16x64_i8 v[244:247], v[104:107], a[76:79], v[244:247]\n" \
  "v_mfma_i32_16x16x64_i8 v[244:247], v[108:111], a[92:95], v[244:247]\n" \
  "s_waitcnt lgkmcnt(8)\n" \
  "v_mfma_i32_16x16x64_i8 v[152:155], v[112:115], a[64:67], v[152:155]\n" \
  "ds_read_b64_tr_b8 v[104:105], v33 offset:49152\n" \
  "s_add_u32 m0, s47, 114688\n" \
  "v_mfma_i32_16x16x64_i8 v[152:155], v[116:119], a[80:83], v[152:155]\n" \
  "ds_read_b64_tr_b8 v[106:107], v33 offset:50176\n" \
  "buffer_load_dwordx4 v40, s[16:19], s87 offen nt lds\n" \
  "s_add_u32 m0, s47, 115712\n" \
  "v_mfma_i32_16x16x64_i8 v[184:187], v[112:115], a[68:71], v[184:187]\n" \
  "buffer_load_dwordx4 v41, s[16:19], s87 offen nt lds\n" \
  "s_add_u32 m0, s47, 116736\n" \
  "v_mfma_i32_16x16x64_i8 v[184:187], v[116:119], a[84:87], v[184:187]\n" \
  "buffer_load_dwordx4 v42, s[16:19], s87 offen nt lds\n" \
  "s_add_u32 m0, s47, 117760\n" \
  "v_mfma_i32_16x16x64_i8 v[216:219], v[112:115], a[72:75], v[216:219]\n" \
  "ds_read_b64_tr_b8 v[108:109], v33 offset:57344\n" \
  "buffer_load_dwordx4 v43, s[16:19], s87 offen nt lds\n" \
  "s_add_u32 m0, s47, 0\n" \
  "v_mfma_i32_16x16x64_i8 v[216:219], v[116:119], a[88:91], v[216:219]\n" \
  "ds_read_b64_tr_b8 v[110:111], v33 offset:58368\n" \
  "buffer_load_dwordx4 v40, s[16:19], s80 offen nt lds\n" \
  "s_add_u32 m0, s47, 1024\n" \
  "v_mfma_i32_16x16x64_i8 v[248:251], v[112:115], a[76:79], v[248:251]\n" \
  "buffer_load_dwordx4 v41, s[16:19], s80 offen nt lds\n" \
  "s_add_u32 m0, s47, 2048\n" \
  "v_mfma_i32_16x16x64_i8 v[248:251], v[116:119], a[92:95], v[248:251]\n" \
  "buffer_load_dwordx4 v42, s[16:19], s80 offen nt lds\n" \
  "s_add_u32 m0, s47, 3072\n" \
  "s_waitcnt lgkmcnt(8)\n" \
  "v_mfma_i32_16x16x64_i8 v[156:159], v[120:123], a[64:67], v[156:159]\n" \
  "ds_read_b64_tr_b8 v[112:113], v34 offset:49152\n" \
  "buffer_load_dwordx4 v43, s[16:19], s80 offen nt lds\n" \
  "v_mfma_i32_16x16x64_i8 v[156:159], v[124:127], a[80:83], v[156:159]\n" \
  "ds_read_b64_tr_b8 v[114:115], v34 offset:50176\n" \
  "v_mfma_i32_16x16x64_i8 v[188:191], v[120:123], a[68:71], v[188:191]\n" \
  "v_mfma_i32_16x16x64_i8 v[188:191], v[124:127], a[84:87], v[188:191]\n" \
  "v_mfma_i32_16x16x64_i8 v[220:223], v[120:123], a[72:75], v[220:223]\n" \
  "ds_read_b64_tr_b8 v[116:117], v34 offset:57344\n" \
  "v_mfma_i32_16x16x64_i8 v[220:223], v[124:127], a[88:91], v[220:223]\n" \
  "ds_read_b64_tr_b8 v[118:119], v34 offset:58368\n" \
  "v_mfma_i32_16x16x64_i8 v[252:255], v[120:123], a[76:79], v[252:255]\n" \
  "v_mfma_i32_16x16x64_i8 v[252:255], v[124:127], a[92:95], v[252:255]\n" \
  "; ---- step 3\n" \
  "s_waitcnt lgkmcnt(8)\n" \
  "v_mfma_i32_16x16x64_i8 v[128:131], v[96:99], a[96:99], v[128:131]\n" \
  "ds_read_b64_tr_b8 v[120:121], v35 offset:49152\n" \
  "v_mfma_i32_16x16x64_i8 v[128:131], v[100:103], a[112:115], v[128:131]\n" \
  "ds_read_b64_tr_b8 v[122:123], v35 offset:50176\n" \
  "v_mfma_i32_16x16x64_i8 v[160:163], v[96:99], a[100:103], v[160:163]\n" \
  "v_mfma_i32_16x16x64_i8 v[160:163], v[100:103], a[116:119], v[160:163]\n" \
  "v_mfma_i32_16x16x64_i8 v[192:195], v[96:99], a[104:107], v[192:195]\n" \
  "ds_read_b64_tr_b8 v[124:125], v35 offset:57344\n" \
  "v_mfma_i32_16x16x64_i8 v[192:195], v[100:103], a[120:123], v[192:195]\n" \
  "ds_read_b64_tr_b8 v[126:127], v35 offset:58368\n" \
  "v_mfma_i32_16x16x64_i8 v[224:227], v[96:99], a[108:111], v[224:227]\n" \
  "v_mfma_i32_16x16x64_i8 v[224:227], v[100:103], a[124:127], v[224:227]\n" \
  "s_waitcnt lgkmcnt(8)\n" \
  "v_mfma_i32_16x16x64_i8 v[132:135], v[104:107], a[96:99], v[132:135]\n" \
  "ds_read_b64_tr_b8 v[96:97], v36 offset:49152\n" \
  "v_mfma_i32_16x16x64_i8 v[132:135], v[108:111], a[112:115], v[132:135]\n" \
  "ds_read_b64_tr_b8 v[98:99], v36 offset:50176\n" \
  "v_mfma_i32_16x16x64_i8 v[164:167], v[104:107], a[100:103], v[164:167]\n" \
  "v_mfma_i32_16x16x64_i8 v[164:167], v[108:111], a[116:119], v[164:167]\n" \
  "v_mfma_i32_16x16x64_i8 v[196:199], v[104:107], a[104:107], v[196:199]\n" \
  "ds_read_b64_tr_b8 v[100:101], v36 offset:57344\n" \
  "v_mfma_i32_16x16x64_i8 v[196:199], v[108:111], a[120:123], v[196:199]\n" \
  "ds_read_b64_tr_b8 v[102:103], v36 offset:58368\n" \
  "v_mfma_i32_16x16x64_i8 v[228:231], v[104:107], a[108:111], v[228:231]\n" \
  "v_mfma_i32_16x16x64_i8 v[228:231], v[108:111], a[124:127], v[228:231]\n" \
  "s_waitcnt lgkmcnt(8)\n" \
  "v_mfma_i32_16x16x64_i8 v[136:139], v[112:115], a[96:99], v[136:139]\n" \
  "ds_read_b64_tr_b8 v[104:105], v37 offset:49152\n" \
  "v_mfma_i32_16x16x64_i8 v[136:139], v[116:119], a[112:115], v[136:139]\n" \
  "ds_read_b64_tr_b8 v[106:107], v37 offset:50176\n" \
  "v_mfma_i32_16x16x64_i8 v[168:171], v[112:115], a[100:103], v[168:171]\n" \
  "v_mfma_i32_16x16x64_i8 v[168:171], v[116:119], a[116:119], v[168:171]\n" \
  "v_mfma_i32_16x16x64_i8 v[200:203], v[112:115], a[104:107], v[200:203]\n" \
  "ds_read_b64_tr_b8 v[108:109], v37 offset:57344\n" \
  "v_mfma_i32_16x16x64_i8 v[200:203], v[116:119], a[120:123], v[200:203]\n" \
  "ds_read_b64_tr_b8 v[110:111], v37 offset:58368\n" \
  "v_mfma_i32_16x16x64_i8 v[232:235], v[112:115], a[108:111], v[232:235]\n" \
  "v_mfma_i32_16x16x64_i8 v[232:235], v[116:119], a[124:127], v[232:235]\n" \
  "s_waitcnt lgkmcnt(8)\n" \
  "v_mfma_i32_16x16x64_i8 v[140:143], v[120:123], a[96:99], v[140:143]\n" \
  "ds_read_b64_tr_b8 v[112:113], v38 offset:49152\n" \
  "v_mfma_i32_16x16x64_i8 v[140:143], v[124:127], a[112:115], v[140:143]\n" \
  "ds_read_b64_tr_b8 v[114:115], v38 offset:50176\n" \
  "v_mfma_i32_16x16x64_i8 v[172:175], v[120:123], a[100:103], v[172:175]\n" \
  "v_mfma_i32_16x16x64_i8 v[172:175], v[124:127], a[116:119], v[172:175]\n" \
  "s_cmp_lg_u32 s46, 1\n" \
  "s_cbranch_scc1 Lskip9_%=\n" \
  "s_waitcnt vmcnt(16)\n" \
  "Lskip9_%=:\n" \
  "s_waitcnt vmcnt(24)\n" \
  "s_barrier\n" \
  "s_cmp_lg_u32 s46, 1\n" \
  "s_cbranch_scc1 Lskip11_%=\n" \
  "s_cmp_lg_u32 s11, 0\n" \
  "s_cbranch_scc1 Lskip10_%=\n" \
  "s_mov_b64 exec, 1\n" \
  "global_store_dword v53, v76, s[36:37] sc0 sc1\n" \
  "s_mov_b64 exec, -1\n" \
  "Lskip10_%=:\n" \
  "Lskip11_%=:\n" \
  "v_mfma_i32_16x16x64_i8 v[204:207], v[120:123], a[104:107], v[204:207]\n" \
  "ds_read_b64_tr_b8 v[116:117], v38 offset:57344\n" \
  "v_mfma_i32_16x16x64_i8 v[204:207], v[124:127], a[120:123], v[204:207]\n" \
  "ds_read_b64_tr_b8 v[118:119], v38 offset:58368\n" \
  "v_mfma_i32_16x16x64_i8 v[236:239], v[120:123], a[108:111], v[236:239]\n" \
  "v_mfma_i32_16x16x64_i8 v[236:239], v[124:127], a[124:127], v[236:239]\n" \
  "s_waitcnt lgkmcnt(8)\n" \
  "v_mfma_i32_16x16x64_i8 v[144:147], v[96:99], a[96:99], v[144:147]\n" \
  "ds_read_b64_tr_b8 v[120:121], v39 offset:49152\n" \
  "v_mfma_i32_16x16x64_i8 v[144:147], v[100:103], a[112:115], v[144:147]\n" \
  "ds_read_b64_tr_b8 v[122:123], v39 offset:50176\n" \
  "v_mfma_i32_16x16x64_i8 v[176:179], v[96:99], a[100:103], v[176:179]\n" \
  "v_mfma_i32_16x16x64_i8 v[176:179], v[100:103], a[116:119], v[176:179]\n" \
  "v_mfma_i32_16x16x64_i8 v[208:211], v[96:99], a[104:107], v[208:211]\n" \
  "ds_read_b64_tr_b8 v[124:125], v39 offset:57344\n" \
  "v_mfma_i32_16x16x64_i8 v[208:211], v[100:103], a[120:123], v[208:211]\n" \
  "ds_read_b64_tr_b8 v[126:127], v39 offset:58368\n" \
  "v_mfma_i32_16x16x64_i8 v[240:243], v[96:99], a[108:111], v[240:243]\n" \
  "v_mfma_i32_16x16x64_i8 v[240:243], v[100:103], a[124:127], v[240:243]\n" \
  "s_waitcnt lgkmcnt(8)\n" \
  "v_mfma_i32_16x16x64_i8 v[148:151], v[104:107], a[96:99], v[148:151]\n" \
  "ds_read_b64_tr_b8 v[96:97], v80 offset:0\n" \
  "v_mfma_i32_16x16x64_i8 v[148:151], v[108:111], a[112:115], v[148:151]\n" \
  "ds_read_b64_tr_b8 v[98:99], v80 offset:1024\n" \
  "v_mfma_i32_16x16x64_i8 v[180:183], v[104:107], a[100:103], v[180:183]\n" \
  "v_mfma_i32_16x16x64_i8 v[180:183], v[108:111], a[116:119], v[180:183]\n" \
  "v_mfma_i32_16x16x64_i8 v[212:215], v[104:107], a[104:107], v[212:215]\n" \
  "ds_read_b64_tr_b8 v[100:101], v80 offset:8192\n" \
  "v_mfma_i32_16x16x64_i8 v[212:215], v[108:111], a[120:123], v[212:215]\n" \
  "ds_read_b64_tr_b8 v[102:103], v80 offset:9216\n" \
  "v_mfma_i32_16x16x64_i8 v[244:247], v[104:107], a[108:111], v[244:247]\n" \
  "v_mfma_i32_16x16x64_i8 v[244:247], v[108:111], a[124:127], v[244:247]\n" \
  "s_waitcnt lgkmcnt(8)\n" \
  "v_mfma_i32_16x16x64_i8 v[152:155], v[112:115], a[96:99], v[152:155]\n" \
  "ds_read_b64_tr_b8 v[104:105], v81 offset:0\n" \
  "v_mfma_i32_16x16x64_i8 v[152:155], v[116:119], a[112:115], v[152:155]\n" \
  "ds_read_b64_tr_b8 v[106:107], v81 offset:1024\n" \
  "v_mfma_i32_16x16x64_i8 v[184:187], v[112:115], a[100:103], v[184:187]\n" \
  "v_mfma_i32_16x16x64_i8 v[184:187], v[116:119], a[116:119], v[184:187]\n" \
  "v_mfma_i32_16x16x64_i8 v[216:219], v[112:115], a[104:107], v[216:219]\n" \
  "ds_read_b64_tr_b8 v[108:109], v81 offset:8192\n" \
  "v_mfma_i32_16x16x64_i8 v[216:219], v[116:119], a[120:123], v[216:219]\n" \
  "ds_read_b64_tr_b8 v[110:111], v81 offset:9216\n" \
  "v_mfma_i32_16x16x64_i8 v[248:251], v[112:115], a[108:111], v[248:251]\n" \
  "v_mfma_i32_16x16x64_i8 v[248:251], v[116:119], a[124:127], v[248:251]\n" \
  "s_waitcnt lgkmcnt(8)\n" \
  "v_mfma_i32_16x16x64_i8 v[156:159], v[120:123], a[96:99], v[156:159]\n" \
  "ds_read_b64_tr_b8 v[112:113], v82 offset:0\n" \
  "v_mfma_i32_16x16x64_i8 v[156:159], v[124:127], a[112:115], v[156:159]\n" \
  "ds_read_b64_tr_b8 v[114:115], v82 offset:1024\n" \
  "v_mfma_i32_16x16x64_i8 v[188:191], v[120:123], a[100:103], v[188:191]\n" \
  "v_mfma_i32_16x16x64_i8 v[188:191], v[124:127], a[116:119], v[188:191]\n" \
  "v_mfma_i32_16x16x64_i8 v[220:223], v[120:123], a[104:107], v[220:223]\n" \
  "ds_read_b64_tr_b8 v[116:117], v82 offset:8192\n" \
  "v_mfma_i32_16x16x64_i8 v[220:223], v[124:127], a[120:123], v[220:223]\n" \
  "ds_read_b64_tr_b8 v[118:119], v82 offset:9216\n" \
  "v_mfma_i32_16x16x64_i8 v[252:255], v[120:123], a[108:111], v[252:255]\n" \
  "v_mfma_i32_16x16x64_i8 v[252:255], v[124:127], a[124:127], v[252:255]\n" \
  "s_cmp_lg_u32 s11, 0\n" \
  "s_cbranch_scc1 Lskip14_%=\n" \
  "global_load_dword v72, v50, s[38:39] sc0 sc1\n" \
  "s_waitcnt vmcnt(0)\n" \
  "v_sub_u32 v72, v72, v76\n" \
  "v_cmp_gt_i32 vcc, 0, v72\n" \
  "s_nop 1\n" \
  "s_andn2_b64 vcc, vcc, s[14:15]\n" \
  "s_cbranch_scc1 Lslow12_%=\n" \
  "Lback13_%=:\n" \
  "v_add_u32 v50, 1024, v50\n" \
  "Lskip14_%=:\n" \
  "s_barrier\n" \
  "buffer_load_dwordx4 a[128:131], v44, s[28:31], s92 offen offset:0 sc0 sc1\n" \
  "buffer_load_dwordx4 a[132:135], v44, s[28:31], s92 offen offset:1024 sc0 sc1\n" \
  "buffer_load_dwordx4 a[136:139], v44, s[28:31], s92 offen offset:2048 sc0 sc1\n" \
  "buffer_load_dwordx4 a[140:143], v44, s[28:31], s92 offen offset:3072 sc0 sc1\n" \
  "buffer_load_dword v92, v52, s[28:31], 0 offen sc0 sc1\n" \
  "v_add_u32 v52, 0x10000, v52\n" \
  "buffer_load_dwordx4 a[144:147], v91, s[28:31], s92 offen offset:0 sc0 sc1\n" \
  "buffer_load_dwordx4 a[148:151], v91, s[28:31], s92 offen offset:1024 sc0 sc1\n" \
  "buffer_load_dwordx4 a[152:155], v91, s[28:31], s92 offen offset:2048 sc0 sc1\n" \
  "buffer_load_dwordx4 a[156:159], v91, s[28:31], s92 offen offset:3072 sc0 sc1\n" \
  "; ---- step 4\n" \
  "s_waitcnt vmcnt(0)\n" \
  "s_waitcnt lgkmcnt(8)\n" \
  "v_mfma_i32_16x16x64_i8 v[128:131], v[96:99], a[128:131], v[128:131]\n" \
  "ds_read_b64_tr_b8 v[120:121], v83 offset:0\n" \
  "v_mfma_i32_16x16x64_i8 v[128:131], v[100:103], a[144:147], v[128:131]\n" \
  "ds_read_b64_tr_b8 v[122:123], v83 offset:1024\n" \
  "buffer_load_dwordx4 a[160:163], v44, s[28:31], s93 offen offset:0 sc0 sc1\n" \
  "v_mfma_i32_16x16x64_i8 v[160:163], v[96:99], a[132:135], v[160:163]\n" \
  "v_mfma_i32_16x16x64_i8 v[160:163], v[100:103], a[148:151], v[160:163]\n" \
  "buffer_load_dwordx4 a[164:167], v44, s[28:31], s93 offen offset:1024 sc0 sc1\n" \
  "v_mfma_i32_16x16x64_i8 v[192:195], v[96:99], a[136:139], v[192:195]\n" \
  "ds_read_b64_tr_b8 v[124:125], v83 offset:8192\n" \
  "v_mfma_i32_16x16x64_i8 v[192:195], v[100:103], a[152:155], v[192:195]\n" \
  "ds_read_b64_tr_b8 v[126:127], v83 offset:9216\n" \
  "buffer_load_dwordx4 a[168:171], v44, s[28:31], s93 offen offset:2048 sc0 sc1\n" \
  "v_mfma_i32_16x16x64_i8 v[224:227], v[96:99], a[140:143], v[224:227]\n" \
  "v_mfma_i32_16x16x64_i8 v[224:227], v[100:103], a[156:159], v[224:227]\n" \
  "s_waitcnt lgkmcnt(8)\n" \
  "v_mfma_i32_16x16x64_i8 v[132:135], v[104:107], a[128:131], v[132:135]\n" \
  "ds_read_b64_tr_b8 v[96:97], v84 offset:0\n" \
  "v_mfma_i32_16x16x64_i8 v[132:135], v[108:111], a[144:147], v[132:135]\n" \
  "ds_read_b64_tr_b8 v[98:99], v84 offset:1024\n" \
  "buffer_load_dwordx4 a[172:175], v44, s[28:31], s93 offen offset:3072 sc0 sc1\n" \
  "v_mfma_i32_16x16x64_i8 v[164:167], v[104:107], a[132:135], v[164:167]\n" \
  "v_mfma_i32_16x16x64_i8 v[164:167], v[108:111], a[148:151], v[164:167]\n" \
  "buffer_load_dwordx4 a[176:179], v91, s[28:31], s93 offen offset:0 sc0 sc1\n" \
  "v_mfma_i32_16x16x64_i8 v[196:199], v[104:107], a[136:139], v[196:199]\n" \
  "ds_read_b64_tr_b8 v[100:101], v84 offset:8192\n" \
  "v_mfma_i32_16x16x64_i8 v[196:199], v[108:111], a[152:155], v[196:199]\n" \
  "ds_read_b64_tr_b8 v[102:103], v84 offset:9216\n" \
  "buffer_load_dwordx4 a[180:183], v91, s[28:31], s93 offen offset:1024 sc0 sc1\n" \
  "v_mfma_i32_16x16x64_i8 v[228:231], v[104:107], a[140:143], v[228:231]\n" \
  "v_mfma_i32_16x16x64_i8 v[228:231], v[108:111], a[156:159], v[228:231]\n" \
  "s_waitcnt lgkmcnt(8)\n" \
  "v_mfma_i32_16x16x64_i8 v[136:139], v[112:115], a[128:131], v[136:139]\n" \
  "ds_read_b64_tr_b8 v[104:105], v85 offset:0\n" \
  "v_mfma_i32_16x16x64_i8 v[136:139], v[116:119], a[144:147], v[136:139]\n" \
  "ds_read_b64_tr_b8 v[106:107], v85 offset:1024\n" \
  "buffer_load_dwordx4 a[184:187], v91, s[28:31], s93 offen offset:2048 sc0 sc1\n" \
  "v_mfma_i32_16x16x64_i8 v[168:171], v[112:115], a[132:135], v[168:171]\n" \
  "v_mfma_i32_16x16x64_i8 v[168:171], v[116:119], a[148:151], v[168:171]\n" \
  "buffer_load_dwordx4 a[188:191], v91, s[28:31], s93 offen offset:3072 sc0 sc1\n" \
  "v_mfma_i32_16x16x64_i8 v[200:203], v[112:115], a[136:139], v[200:203]\n" \
  "ds_read_b64_tr_b8 v[108:109], v85 offset:8192\n" \
  "v_mfma_i32_16x16x64_i8 v[200:203], v[116:119], a[152:155], v[200:203]\n" \
  "ds_read_b64_tr_b8 v[110:111], v85 offset:9216\n" \
  "v_mfma_i32_16x16x64_i8 v[232:235], v[112:115], a[140:143], v[232:235]\n" \
  "v_readlane_b32 s50, v46, s49\n" \
  "v_mfma_i32_16x16x64_i8 v[232:235], v[116:119], a[156:159], v[232:235]\n" \
  "buffer_load_dwordx4 a[192:195], v44, s[28:31], s94 offen offset:0 sc0 sc1\n" \
  "s_waitcnt lgkmcnt(8)\n" \
  "v_mfma_i32_16x16x64_i8 v[140:143], v[120:123], a[128:131], v[140:143]\n" \
  "ds_read_b64_tr_b8 v[112:113], v86 offset:0\n" \
  "v_mfma_i32_16x16x64_i8 v[140:143], v[124:127], a[144:147], v[140:143]\n" \
  "ds_read_b64_tr_b8 v[114:115], v86 offset:1024\n" \
  "buffer_load_dwordx4 a[196:199], v44, s[28:31], s94 offen offset:1024 sc0 sc1\n" \
  "v_mfma_i32_16x16x64_i8 v[172:175], v[120:123], a[132:135], v[172:175]\n" \
  "v_mfma_i32_16x16x64_i8 v[172:175], v[124:127], a[148:151], v[172:175]\n" \
  "buffer_load_dwordx4 a[200:203], v44, s[28:31], s94 offen offset:2048 sc0 sc1\n" \
  "s_waitcnt vmcnt(32)\n" \
  "s_barrier\n" \
  "v_mfma_i32_16x16x64_i8 v[204:207], v[120:123], a[136:139], v[204:207]\n" \
  "ds_read_b64_tr_b8 v[116:117], v86 offset:8192\n" \
  "v_mfma_i32_16x16x64_i8 v[204:207], v[124:127], a[152:155], v[204:207]\n" \
  "ds_read_b64_tr_b8 v[118:119], v86 offset:9216\n" \
  "v_mfma_i32_16x16x64_i8 v[236:239], v[120:123], a[140:143], v[236:239]\n" \
  "s_lshl_b32 s81, s50, 21\n" \
  "s_lshl_b32 s89, s50, 16\n" \
  "s_add_u32 s49, s49, 1\n" \
  "v_mfma_i32_16x16x64_i8 v[236:239], v[124:127], a[156:159], v[236:239]\n" \
  "buffer_load_dwordx4 a[204:207], v44, s[28:31], s94 offen offset:3072 sc0 sc1\n" \
  "s_waitcnt lgkmcnt(8)\n" \
  "v_mfma_i32_16x16x64_i8 v[144:147], v[96:99], a[128:131], v[144:147]\n" \
  "ds_read_b64_tr_b8 v[120:121], v87 offset:0\n" \
  "v_mfma_i32_16x16x64_i8 v[144:147], v[100:103], a[144:147], v[144:147]\n" \
  "ds_read_b64_tr_b8 v[122:123], v87 offset:1024\n" \
  "buffer_load_dwordx4 a[208:211], v91, s[28:31], s94 offen offset:0 sc0 sc1\n" \
  "v_mfma_i32_16x16x64_i8 v[176:179], v[96:99], a[132:135], v[176:179]\n" \
  "v_readlane_b32 s50, v46, s49\n" \
  "v_mfma_i32_16x16x64_i8 v[176:179], v[100:103], a[148:151], v[176:179]\n" \
  "buffer_load_dwordx4 a[212:215], v91, s[28:31], s94 offen offset:1024 sc0 sc1\n" \
  "v_mfma_i32_16x16x64_i8 v[208:211], v[96:99], a[136:139], v[208:211]\n" \
  "ds_read_b64_tr_b8 v[124:125], v87 offset:8192\n" \
  "v_mfma_i32_16x16x64_i8 v[208:211], v[100:103], a[152:155], v[208:211]\n" \
  "ds_read_b64_tr_b8 v[126:127], v87 offset:9216\n" \
  "v_mfma_i32_16x16x64_i8 v[240:243], v[96:99], a[140:143], v[240:243]\n" \
  "v_mfma_i32_16x16x64_i8 v[240:243], v[100:103], a[156:159], v[240:243]\n" \
  "buffer_load_dwordx4 a[216:219], v91, s[28:31], s94 offen offset:2048 sc0 sc1\n" \
  "s_waitcnt lgkmcnt(8)\n" \
  "v_mfma_i32_16x16x64_i8 v[148:151], v[104:107], a[128:131], v[148:151]\n" \
  "ds_read_b64_tr_b8 v[96:97], v80 offset:16384\n" \
  "v_mfma_i32_16x16x64_i8 v[148:151], v[108:111], a[144:147], v[148:151]\n" \
  "ds_read_b64_tr_b8 v[98:99], v80 offset:17408\n" \
  "buffer_load_dwordx4 a[220:223], v91, s[28:31], s94 offen offset:3072 sc0 sc1\n" \
  "v_mfma_i32_16x16x64_i8 v[180:183], v[104:107], a[132:135], v[180:183]\n" \
  "s_lshl_b32 s82, s50, 21\n" \
  "s_lshl_b32 s90, s50, 16\n" \
  "s_add_u32 s49, s49, 1\n" \
  "v_mfma_i32_16x16x64_i8 v[180:183], v[108:111], a[148:151], v[180:183]\n" \
  "v_mfma_i32_16x16x64_i8 v[212:215], v[104:107], a[136:139], v[212:215]\n" \
  "ds_read_b64_tr_b8 v[100:101], v80 offset:24576\n" \
  "v_mfma_i32_16x16x64_i8 v[212:215], v[108:111], a[152:155], v[212:215]\n" \
  "ds_read_b64_tr_b8 v[102:103], v80 offset:25600\n" \
  "v_mfma_i32_16x16x64_i8 v[244:247], v[104:107], a[140:143], v[244:247]\n" \
  "v_mfma_i32_16x16x64_i8 v[244:247], v[108:111], a[156:159], v[244:247]\n" \
  "s_waitcnt lgkmcnt(8)\n" \
  "v_mfma_i32_16x16x64_i8 v[152:155], v[112:115], a[128:131], v[152:155]\n" \
  "ds_read_b64_tr_b8 v[104:105], v81 offset:16384\n" \
  "s_add_u32 m0, s47, 16384\n" \
  "v_mfma_i32_16x16x64_i8 v[152:155], v[116:119], a[144:147], v[152:155]\n" \
  "ds_read_b64_tr_b8 v[106:107], v81 offset:17408\n" \
  "buffer_load_dwordx4 v40, s[16:19], s81 offen nt lds\n" \
  "s_add_u32 m0, s47, 17408\n" \
  "v_mfma_i32_16x16x64_i8 v[184:187], v[112:115], a[132:135], v[184:187]\n" \
  "buffer_load_dwordx4 v41, s[16:19], s81 offen nt lds\n" \
  "s_add_u32 m0, s47, 18432\n" \
  "v_mfma_i32_16x16x64_i8 v[184:187], v[116:119], a[148:151], v[184:187]\n" \
  "buffer_load_dwordx4 v42, s[16:19], s81 offen nt lds\n" \
  "s_add_u32 m0, s47, 19456\n" \
  "v_mfma_i32_16x16x64_i8 v[216:219], v[112:115], a[136:139], v[216:219]\n" \
  "ds_read_b64_tr_b8 v[108:109], v81 offset:24576\n" \
  "buffer_load_dwordx4 v43, s[16:19], s81 offen nt lds\n" \
  "s_add_u32 m0, s47, 32768\n" \
  "v_mfma_i32_16x16x64_i8 v[216:219], v[116:119], a[152:155], v[216:219]\n" \
  "ds_read_b64_tr_b8 v[110:111], v81 offset:25600\n" \
  "buffer_load_dwordx4 v40, s[16:19], s82 offen nt lds\n" \
  "s_add_u32 m0, s47, 33792\n" \
  "v_mfma_i32_16x16x64_i8 v[248:251], v[112:115], a[140:143], v[248:251]\n" \
  "buffer_load_dwordx4 v41, s[16:19], s82 offen nt lds\n" \
  "s_add_u32 m0, s47, 34816\n" \
  "v_mfma_i32_16x16x64_i8 v[248:251], v[116:119], a[156:159], v[248:251]\n" \
  "buffer_load_dwordx4 v42, s[16:19], s82 offen nt lds\n" \
  "s_add_u32 m0, s47, 35840\n" \
  "s_waitcnt lgkmcnt(8)\n" \
  "v_mfma_i32_16x16x64_i8 v[156:159], v[120:123], a[128:131], v[156:159]\n" \
  "ds_read_b64_tr_b8 v[112:113], v82 offset:16384\n" \
  "buffer_load_dwordx4 v43, s[16:19], s82 offen nt lds\n" \
  "v_mfma_i32_16x16x64_i8 v[156:159], v[124:127], a[144:147], v[156:159]\n" \
  "ds_read_b64_tr_b8 v[114:115], v82 offset:17408\n" \
  "v_mfma_i32_16x16x64_i8 v[188:191], v[120:123], a[132:135], v[188:191]\n" \
  "v_mfma_i32_16x16x64_i8 v[188:191], v[124:127], a[148:151], v[188:191]\n" \
  "v_mfma_i32_16x16x64_i8 v[220:223], v[120:123], a[136:139], v[220:223]\n" \
  "ds_read_b64_tr_b8 v[116:117], v82 offset:24576\n" \
  "v_mfma_i32_16x16x64_i8 v[220:223], v[124:127], a[152:155], v[220:223]\n" \
  "ds_read_b64_tr_b8 v[118:119], v82 offset:25600\n" \
  "v_mfma_i32_16x16x64_i8 v[252:255], v[120:123], a[140:143], v[252:255]\n" \
  "v_mfma_i32_16x16x64_i8 v[252:255], v[124:127], a[156:159], v[252:255]\n" \
  "; ---- step 5\n" \
  "s_waitcnt vmcnt(16)\n" \
  "s_waitcnt lgkmcnt(8)\n" \
  "v_mfma_i32_16x16x64_i8 v[128:131], v[96:99], a[160:163], v[128:131]\n" \
  "ds_read_b64_tr_b8 v[120:121], v83 offset:16384\n" \
  "v_mfma_i32_16x16x64_i8 v[128:131], v[100:103], a[176:179], v[128:131]\n" \
  "ds_read_b64_tr_b8 v[122:123], v83 offset:17408\n" \
  "v_mfma_i32_16x16x64_i8 v[160:163], v[96:99], a[164:167], v[160:163]\n" \
  "v_mfma_i32_16x16x64_i8 v[160:163], v[100:103], a[180:183], v[160:163]\n" \
  "v_mfma_i32_16x16x64_i8 v[192:195], v[96:99], a[168:171], v[192:195]\n" \
  "ds_read_b64_tr_b8 v[124:125], v83 offset:24576\n" \
  "v_mfma_i32_16x16x64_i8 v[192:195], v[100:103], a[184:187], v[192:195]\n" \
  "ds_read_b64_tr_b8 v[126:127], v83 offset:25600\n" \
  "v_mfma_i32_16x16x64_i8 v[224:227], v[96:99], a[172:175], v[224:227]\n" \
  "v_mfma_i32_16x16x64_i8 v[224:227], v[100:103], a[188:191], v[224:227]\n" \
  "s_waitcnt lgkmcnt(8)\n" \
  "v_mfma_i32_16x16x64_i8 v[132:135], v[104:107], a[160:163], v[132:135]\n" \
  "ds_read_b64_tr_b8 v[96:97], v84 offset:16384\n" \
  "v_mfma_i32_16x16x64_i8 v[132:135], v[108:111], a[176:179], v[132:135]\n" \
  "ds_read_b64_tr_b8 v[98:99], v84 offset:17408\n" \
  "v_mfma_i32_16x16x64_i8 v[164:167], v[104:107], a[164:167], v[164:167]\n" \
  "v_mfma_i32_16x16x64_i8 v[164:167], v[108:111], a[180:183], v[164:167]\n" \
  "v_mfma_i32_16x16x64_i8 v[196:199], v[104:107], a[168:171], v[196:199]\n" \
  "ds_read_b64_tr_b8 v[100:101], v84 offset:24576\n" \
  "v_mfma_i32_16x16x64_i8 v[196:199], v[108:111], a[184:187], v[196:199]\n" \
  "ds_read_b64_tr_b8 v[102:103], v84 offset:25600\n" \
  "v_mfma_i32_16x16x64_i8 v[228:231], v[104:107], a[172:175], v[228:231]\n" \
  "v_mfma_i32_16x16x64_i8 v[228:231], v[108:111], a[188:191], v[228:231]\n" \
  "s_waitcnt lgkmcnt(8)\n" \
  "v_mfma_i32_16x16x64_i8 v[136:139], v[112:115], a[160:163], v[136:139]\n" \
  "ds_read_b64_tr_b8 v[104:105], v85 offset:16384\n" \
  "v_mfma_i32_16x16x64_i8 v[136:139], v[116:119], a[176:179], v[136:139]\n" \
  "ds_read_b64_tr_b8 v[106:107], v85 offset:17408\n" \
  "buffer_load_dwordx4 a[224:227], v44, s[28:31], s95 offen offset:0 sc0 sc1\n" \
  "v_mfma_i32_16x16x64_i8 v[168:171], v[112:115], a[164:167], v[168:171]\n" \
  "v_mfma_i32_16x16x64_i8 v[168:171], v[116:119], a[180:183], v[168:171]\n" \
  "buffer_load_dwordx4 a[228:231], v44, s[28:31], s95 offen offset:1024 sc0 sc1\n" \
  "v_mfma_i32_16x16x64_i8 v[200:203], v[112:115], a[168:171], v[200:203]\n" \
  "ds_read_b64_tr_b8 v[108:109], v85 offset:24576\n" \
  "v_mfma_i32_16x16x64_i8 v[200:203], v[116:119], a[184:187], v[200:203]\n" \
  "ds_read_b64_tr_b8 v[110:111], v85 offset:25600\n" \
  "v_mfma_i32_16x16x64_i8 v[232:235], v[112:115], a[172:175], v[232:235]\n" \
  "v_mfma_i32_16x16x64_i8 v[232:235], v[116:119], a[188:191], v[232:235]\n" \
  "buffer_load_dwordx4 a[232:235], v44, s[28:31], s95 offen offset:2048 sc0 sc1\n" \
  "s_waitcnt lgkmcnt(8)\n" \
  "v_mfma_i32_16x16x64_i8 v[140:143], v[120:123], a[160:163], v[140:143]\n" \
  "ds_read_b64_tr_b8 v[112:113], v86 offset:16384\n" \
  "v_mfma_i32_16x16x64_i8 v[140:143], v[124:127], a[176:179], v[140:143]\n" \
  "ds_read_b64_tr_b8 v[114:115], v86 offset:17408\n" \
  "buffer_load_dwordx4 a[236:239], v44, s[28:31], s95 offen offset:3072 sc0 sc1\n" \
  "v_mfma_i32_16x16x64_i8 v[172:175], v[120:123], a[164:167], v[172:175]\n" \
  "v_mfma_i32_16x16x64_i8 v[172:175], v[124:127], a[180:183], v[172:175]\n" \
  "s_waitcnt vmcnt(45)\n" \
  "s_barrier\n" \
  "v_mfma_i32_16x16x64_i8 v[204:207], v[120:123], a[168:171], v[204:207]\n" \
  "ds_read_b64_tr_b8 v[116:117], v86 offset:24576\n" \
  "v_mfma_i32_16x16x64_i8 v[204:207], v[124:127], a[184:187], v[204:207]\n" \
  "ds_read_b64_tr_b8 v[118:119], v86 offset:25600\n" \
  "buffer_load_dwordx4 a[240:243], v91, s[28:31], s95 offen offset:0 sc0 sc1\n" \
  "v_mfma_i32_16x16x64_i8 v[236:239], v[120:123], a[172:175], v[236:239]\n" \
  "v_mfma_i32_16x16x64_i8 v[236:239], v[124:127], a[188:191], v[236:239]\n" \
  "buffer_load_dwordx4 a[244:247], v91, s[28:31], s95 offen offset:1024 sc0 sc1\n" \
  "s_waitcnt lgkmcnt(8)\n" \
  "v_mfma_i32_16x16x64_i8 v[144:147], v[96:99], a[160:163], v[144:147]\n" \
  "ds_read_b64_tr_b8 v[120:121], v87 offset:16384\n" \
  "v_mfma_i32_16x16x64_i8 v[144:147], v[100:103], a[176:179], v[144:147]\n" \
  "ds_read_b64_tr_b8 v[122:123], v87 offset:17408\n" \
  "v_mfma_i32_16x16x64_i8 v[176:179], v[96:99], a[164:167], v[176:179]\n" \
  "v_mfma_i32_16x16x64_i8 v[176:179], v[100:103], a[180:183], v[176:179]\n" \
  "buffer_load_dwordx4 a[248:251], v91, s[28:31], s95 offen offset:2048 sc0 sc1\n" \
  "v_mfma_i32_16x16x64_i8 v[208:211], v[96:99], a[168:171], v[208:211]\n" \
  "ds_read_b64_tr_b8 v[124:125], v87 offset:24576\n" \
  "v_mfma_i32_16x16x64_i8 v[208:211], v[100:103], a[184:187], v[208:211]\n" \
  "ds_read_b64_tr_b8 v[126:127], v87 offset:25600\n" \
  "buffer_load_dwordx4 a[252:255], v91, s[28:31], s95 offen offset:3072 sc0 sc1\n" \
  "v_mfma_i32_16x16x64_i8 v[240:243], v[96:99], a[172:175], v[240:243]\n" \
  "v_readlane_b32 s50, v46, s49\n" \
  "v_mfma_i32_16x16x64_i8 v[240:243], v[100:103], a[188:191], v[240:243]\n" \
  "s_waitcnt lgkmcnt(8)\n" \
  "v_mfma_i32_16x16x64_i8 v[148:151], v[104:107], a[160:163], v[148:151]\n" \
  "ds_read_b64_tr_b8 v[96:97], v80 offset:32768\n" \
  "v_mfma_i32_16x16x64_i8 v[148:151], v[108:111], a[176:179], v[148:151]\n" \
  "ds_read_b64_tr_b8 v[98:99], v80 offset:33792\n" \
  "buffer_load_dwordx4 a[0:3], v44, s[28:31], s88 offen offset:0 sc0 sc1\n" \
  "v_mfma_i32_16x16x64_i8 v[180:183], v[104:107], a[164:167], v[180:183]\n" \
  "v_mfma_i32_16x16x64_i8 v[180:183], v[108:111], a[180:183], v[180:183]\n" \
  "buffer_load_dwordx4 a[4:7], v44, s[28:31], s88 offen offset:1024 sc0 sc1\n" \
  "v_mfma_i32_16x16x64_i8 v[212:215], v[104:107], a[168:171], v[212:215]\n" \
  "ds_read_b64_tr_b8 v[100:101], v80 offset:40960\n" \
  "v_mfma_i32_16x16x64_i8 v[212:215], v[108:111], a[184:187], v[212:215]\n" \
  "ds_read_b64_tr_b8 v[102:103], v80 offset:41984\n" \
  "v_mfma_i32_16x16x64_i8 v[244:247], v[104:107], a[172:175], v[244:247]\n" \
  "s_lshl_b32 s83, s50, 21\n" \
  "s_lshl_b32 s91, s50, 16\n" \
  "s_add_u32 s49, s49, 1\n" \
  "v_mfma_i32_16x16x64_i8 v[244:247], v[108:111], a[188:191], v[244:247]\n" \
  "buffer_load_dwordx4 a[8:11], v44, s[28:31], s88 offen offset:2048 sc0 sc1\n" \
  "s_waitcnt lgkmcnt(8)\n" \
  "v_mfma_i32_16x16x64_i8 v[152:155], v[112:115], a[160:163], v[152:155]\n" \
  "ds_read_b64_tr_b8 v[104:105], v81 offset:32768\n" \
  "v_mfma_i32_16x16x64_i8 v[152:155], v[116:119], a[176:179], v[152:155]\n" \
  "ds_read_b64_tr_b8 v[106:107], v81 offset:33792\n" \
  "buffer_load_dwordx4 a[12:15], v44, s[28:31], s88 offen offset:3072 sc0 sc1\n" \
  "v_mfma_i32_16x16x64_i8 v[184:187], v[112:115], a[164:167], v[184:187]\n" \
  "v_readlane_b32 s50, v46, s49\n" \
  "v_mfma_i32_16x16x64_i8 v[184:187], v[116:119], a[180:183], v[184:187]\n" \
  "v_mfma_i32_16x16x64_i8 v[216:219], v[112:115], a[168:171], v[216:219]\n" \
  "ds_read_b64_tr_b8 v[108:109], v81 offset:40960\n" \
  "v_mfma_i32_16x16x64_i8 v[216:219], v[116:119], a[184:187], v[216:219]\n" \
  "ds_read_b64_tr_b8 v[110:111], v81 offset:41984\n" \
  "buffer_load_dwordx4 a[16:19], v91, s[28:31], s88 offen offset:0 sc0 sc1\n" \
  "v_mfma_i32_16x16x64_i8 v[248:251], v[112:115], a[172:175], v[248:251]\n" \
  "v_mfma_i32_16x16x64_i8 v[248:251], v[116:119], a[188:191], v[248:251]\n" \
  "buffer_load_dwordx4 a[20:23], v91, s[28:31], s88 offen offset:1024 sc0 sc1\n" \
  "s_waitcnt lgkmcnt(8)\n" \
  "v_mfma_i32_16x16x64_i8 v[156:159], v[120:123], a[160:163], v[156:159]\n" \
  "ds_read_b64_tr_b8 v[112:113], v82 offset:32768\n" \
  "v_mfma_i32_16x16x64_i8 v[156:159], v[124:127], a[176:179], v[156:159]\n" \
  "ds_read_b64_tr_b8 v[114:115], v82 offset:33792\n" \
  "v_mfma_i32_16x16x64_i8 v[188:191], v[120:123], a[164:167], v[188:191]\n" \
  "s_lshl_b32 s84, s50, 21\n" \
  "s_lshl_b32 s92, s50, 16\n" \
  "s_add_u32 s49, s49, 1\n" \
  "v_mfma_i32_16x16x64_i8 v[188:191], v[124:127], a[180:183], v[188:191]\n" \
  "buffer_load_dwordx4 a[24:27], v91, s[28:31], s88 offen offset:2048 sc0 sc1\n" \
  "v_mfma_i32_16x16x64_i8 v[220:223], v[120:123], a[168:171], v[220:223]\n" \
  "ds_read_b64_tr_b8 v[116:117], v82 offset:40960\n" \
  "v_mfma_i32_16x16x64_i8 v[220:223], v[124:127], a[184:187], v[220:223]\n" \
  "ds_read_b64_tr_b8 v[118:119], v82 offset:41984\n" \
  "buffer_load_dwordx4 a[28:31], v91, s[28:31], s88 offen offset:3072 sc0 sc1\n" \
  "v_mfma_i32_16x16x64_i8 v[252:255], v[120:123], a[172:175], v[252:255]\n" \
  "v_mfma_i32_16x16x64_i8 v[252:255], v[124:127], a[188:191], v[252:255]\n" \
  "; ---- step 6\n" \
  "s_cmp_lg_u32 s11, 0\n" \
  "s_cbranch_scc1 Lskip16_%=\n" \
  "global_load_dword v72, v50, s[38:39] sc0 sc1\n" \
  "Lskip16_%=:\n" \
  "s_cmp_lg_u32 s46, 2\n" \
  "s_cbranch_scc1 Lskip15_%=\n" \
  "s_waitcnt vmcnt(63)\n" \
  "global_store_dwordx4 v49, v[64:67], s[34:35] sc0 sc1\n" \
  "s_nop 1\n" \
  "global_store_dwordx4 v79, v[68:71], s[34:35] sc0 sc1\n" \
  "s_nop 1\n" \
  "Lskip15_%=:\n" \
  "s_waitcnt vmcnt(24)\n" \
  "s_waitcnt lgkmcnt(8)\n" \
  "v_mfma_i32_16x16x64_i8 v[128:131], v[96:99], a[192:195], v[128:131]\n" \
  "ds_read_b64_tr_b8 v[120:121], v83 offset:32768\n" \
  "s_add_u32 m0, s47, 49152\n" \
  "v_mfma_i32_16x16x64_i8 v[128:131], v[100:103], a[208:211], v[128:131]\n" \
  "ds_read_b64_tr_b8 v[122:123], v83 offset:33792\n" \
  "buffer_load_dwordx4 v40, s[16:19], s83 offen nt lds\n" \
  "s_add_u32 m0, s47, 50176\n" \
  "v_mfma_i32_16x16x64_i8 v[160:163], v[96:99], a[196:199], v[160:163]\n" \
  "buffer_load_dwordx4 v41, s[16:19], s83 offen nt lds\n" \
  "s_add_u32 m0, s47, 51200\n" \
  "v_mfma_i32_16x16x64_i8 v[160:163], v[100:103], a[212:215], v[160:163]\n" \
  "buffer_load_dwordx4 v42, s[16:19], s83 offen nt lds\n" \
  "s_add_u32 m0, s47, 52224\n" \
  "v_mfma_i32_16x16x64_i8 v[192:195], v[96:99], a[200:203], v[192:195]\n" \
  "ds_read_b64_tr_b8 v[124:125], v83 offset:40960\n" \
  "buffer_load_dwordx4 v43, s[16:19], s83 offen nt lds\n" \
  "s_add_u32 m0, s47, 65536\n" \
  "v_mfma_i32_16x16x64_i8 v[192:195], v[100:103], a[216:219], v[192:195]\n" \
  "ds_read_b64_tr_b8 v[126:127], v83 offset:41984\n" \
  "buffer_load_dwordx4 v40, s[16:19], s84 offen nt lds\n" \
  "s_add_u32 m0, s47, 66560\n" \
  "v_mfma_i32_16x16x64_i8 v[224:227], v[96:99], a[204:207], v[224:227]\n" \
  "buffer_load_dwordx4 v41, s[16:19], s84 offen nt lds\n" \
  "v_mfma_i32_16x16x64_i8 v[224:227], v[100:103], a[220:223], v[224:227]\n" \
  "s_waitcnt lgkmcnt(8)\n" \
  "v_mfma_i32_16x16x64_i8 v[132:135], v[104:107], a[192:195], v[132:135]\n" \
  "ds_read_b64_tr_b8 v[96:97], v84 offset:32768\n" \
  "v_mfma_i32_16x16x64_i8 v[132:135], v[108:111], a[208:211], v[132:135]\n" \
  "ds_read_b64_tr_b8 v[98:99], v84 offset:33792\n" \
  "v_mfma_i32_16x16x64_i8 v[164:167], v[104:107], a[196:199], v[164:167]\n" \
  "v_mfma_i32_16x16x64_i8 v[164:167], v[108:111], a[212:215], v[164:167]\n" \
  "v_mfma_i32_16x16x64_i8 v[196:199], v[104:107], a[200:203], v[196:199]\n" \
  "ds_read_b64_tr_b8 v[100:101], v84 offset:40960\n" \
  "v_mfma_i32_16x16x64_i8 v[196:199], v[108:111], a[216:219], v[196:199]\n" \
  "ds_read_b64_tr_b8 v[102:103], v84 offset:41984\n" \
  "v_mfma_i32_16x16x64_i8 v[228:231], v[104:107], a[204:207], v[228:231]\n" \
  "v_mfma_i32_16x16x64_i8 v[228:231], v[108:111], a[220:223], v[228:231]\n" \
  "s_waitcnt lgkmcnt(8)\n" \
  "v_mfma_i32_16x16x64_i8 v[136:139], v[112:115], a[192:195], v[136:139]\n" \
  "ds_read_b64_tr_b8 v[104:105], v85 offset:32768\n" \
  "v_mfma_i32_16x16x64_i8 v[136:139], v[116:119], a[208:211], v[136:139]\n" \
  "ds_read_b64_tr_b8 v[106:107], v85 offset:33792\n" \
  "v_mfma_i32_16x16x64_i8 v[168:171], v[112:115], a[196:199], v[168:171]\n" \
  "v_mfma_i32_16x16x64_i8 v[168:171], v[116:119], a[212:215], v[168:171]\n" \
  "v_mfma_i32_16x16x64_i8 v[200:203], v[112:115], a[200:203], v[200:203]\n" \
  "ds_read_b64_tr_b8 v[108:109], v85 offset:40960\n" \
  "v_mfma_i32_16x16x64_i8 v[200:203], v[116:119], a[216:219], v[200:203]\n" \
  "ds_read_b64_tr_b8 v[110:111], v85 offset:41984\n" \
  "v_mfma_i32_16x16x64_i8 v[232:235], v[112:115], a[204:207], v[232:235]\n" \
  "v_mfma_i32_16x16x64_i8 v[232:235], v[116:119], a[220:223], v[232:235]\n" \
  "s_waitcnt lgkmcnt(8)\n" \
  "v_mfma_i32_16x16x64_i8 v[140:143], v[120:123], a[192:195], v[140:143]\n" \
  "ds_read_b64_tr_b8 v[112:113], v86 offset:32768\n" \
  "v_mfma_i32_16x16x64_i8 v[140:143], v[124:127], a[208:211], v[140:143]\n" \
  "ds_read_b64_tr_b8 v[114:115], v86 offset:33792\n" \
  "v_mfma_i32_16x16x64_i8 v[172:175], v[120:123], a[196:199], v[172:175]\n" \
  "v_mfma_i32_16x16x64_i8 v[172:175], v[124:127], a[212:215], v[172:175]\n" \
  "s_waitcnt vmcnt(59)\n" \
  "s_barrier\n" \
  "v_mfma_i32_16x16x64_i8 v[204:207], v[120:123], a[200:203], v[204:207]\n" \
  "ds_read_b64_tr_b8 v[116:117], v86 offset:40960\n" \
  "v_mfma_i32_16x16x64_i8 v[204:207], v[124:127], a[216:219], v[204:207]\n" \
  "ds_read_b64_tr_b8 v[118:119], v86 offset:41984\n" \
  "v_mfma_i32_16x16x64_i8 v[236:239], v[120:123], a[204:207], v[236:239]\n" \
  "v_mfma_i32_16x16x64_i8 v[236:239], v[124:127], a[220:223], v[236:239]\n" \
  "s_waitcnt lgkmcnt(8)\n" \
  "v_mfma_i32_16x16x64_i8 v[144:147], v[96:99], a[192:195], v[144:147]\n" \
  "ds_read_b64_tr_b8 v[120:121], v87 offset:32768\n" \
  "v_mfma_i32_16x16x64_i8 v[144:147], v[100:103], a[208:211], v[144:147]\n" \
  "ds_read_b64_tr_b8 v[122:123], v87 offset:33792\n" \
  "v_mfma_i32_16x16x64_i8 v[176:179], v[96:99], a[196:199], v[176:179]\n" \
  "v_mfma_i32_16x16x64_i8 v[176:179], v[100:103], a[212:215], v[176:179]\n" \
  "v_mfma_i32_16x16x64_i8 v[208:211], v[96:99], a[200:203], v[208:211]\n" \
  "ds_read_b64_tr_b8 v[124:125], v87 offset:40960\n" \
  "v_mfma_i32_16x16x64_i8 v[208:211], v[100:103], a[216:219], v[208:211]\n" \
  "ds_read_b64_tr_b8 v[126:127], v87 offset:41984\n" \
  "v_mfma_i32_16x16x64_i8 v[240:243], v[96:99], a[204:207], v[240:243]\n" \
  "v_mfma_i32_16x16x64_i8 v[240:243], v[100:103], a[220:223], v[240:243]\n" \
  "s_waitcnt lgkmcnt(8)\n" \
  "v_mfma_i32_16x16x64_i8 v[148:151], v[104:107], a[192:195], v[148:151]\n" \
  "ds_read_b64_tr_b8 v[96:97], v80 offset:49152\n" \
  "s_add_u32 m0, s47, 67584\n" \
  "v_mfma_i32_16x16x64_i8 v[148:151], v[108:111], a[208:211], v[148:151]\n" \
  "ds_read_b64_tr_b8 v[98:99], v80 offset:50176\n" \
  "buffer_load_dwordx4 v42, s[16:19], s84 offen nt lds\n" \
  "s_add_u32 m0, s47, 68608\n" \
  "v_mfma_i32_16x16x64_i8 v[180:183], v[104:107], a[196:199], v[180:183]\n" \
  "buffer_load_dwordx4 v43, s[16:19], s84 offen nt lds\n" \
  "v_mfma_i32_16x16x64_i8 v[180:183], v[108:111], a[212:215], v[180:183]\n" \
  "v_mfma_i32_16x16x64_i8 v[212:215], v[104:107], a[200:203], v[212:215]\n" \
  "ds_read_b64_tr_b8 v[100:101], v80 offset:57344\n" \
  "v_mfma_i32_16x16x64_i8 v[212:215], v[108:111], a[216:219], v[212:215]\n" \
  "ds_read_b64_tr_b8 v[102:103], v80 offset:58368\n" \
  "v_mfma_i32_16x16x64_i8 v[244:247], v[104:107], a[204:207], v[244:247]\n" \
  "v_mfma_i32_16x16x64_i8 v[244:247], v[108:111], a[220:223], v[244:247]\n" \
  "s_waitcnt lgkmcnt(8)\n" \
  "v_mfma_i32_16x16x64_i8 v[152:155], v[112:115], a[192:195], v[152:155]\n" \
  "ds_read_b64_tr_b8 v[104:105], v81 offset:49152\n" \
  "v_mfma_i32_16x16x64_i8 v[152:155], v[116:119], a[208:211], v[152:155]\n" \
  "ds_read_b64_tr_b8 v[106:107], v81 offset:50176\n" \
  "v_mfma_i32_16x16x64_i8 v[184:187], v[112:115], a[196:199], v[184:187]\n" \
  "v_mfma_i32_16x16x64_i8 v[184:187], v[116:119], a[212:215], v[184:187]\n" \
  "v_mfma_i32_16x16x64_i8 v[216:219], v[112:115], a[200:203], v[216:219]\n" \
  "ds_read_b64_tr_b8 v[108:109], v81 offset:57344\n" \
  "v_mfma_i32_16x16x64_i8 v[216:219], v[116:119], a[216:219], v[216:219]\n" \
  "ds_read_b64_tr_b8 v[110:111], v81 offset:58368\n" \
  "v_mfma_i32_16x16x64_i8 v[248:251], v[112:115], a[204:207], v[248:251]\n" \
  "v_mfma_i32_16x16x64_i8 v[248:251], v[116:119], a[220:223], v[248:251]\n" \
  "s_waitcnt lgkmcnt(8)\n" \
  "v_mfma_i32_16x16x64_i8 v[156:159], v[120:123], a[192:195], v[156:159]\n" \
  "ds_read_b64_tr_b8 v[112:113], v82 offset:49152\n" \
  "v_mfma_i32_16x16x64_i8 v[156:159], v[124:127], a[208:211], v[156:159]\n" \
  "ds_read_b64_tr_b8 v[114:115], v82 offset:50176\n" \
  "v_mfma_i32_16x16x64_i8 v[188:191], v[120:123], a[196:199], v[188:191]\n" \
  "v_mfma_i32_16x16x64_i8 v[188:191], v[124:127], a[212:215], v[188:191]\n" \
  "v_mfma_i32_16x16x64_i8 v[220:223], v[120:123], a[200:203], v[220:223]\n" \
  "ds_read_b64_tr_b8 v[116:117], v82 offset:57344\n" \
  "v_mfma_i32_16x16x64_i8 v[220:223], v[124:127], a[216:219], v[220:223]\n" \
  "ds_read_b64_tr_b8 v[118:119], v82 offset:58368\n" \
  "v_mfma_i32_16x16x64_i8 v[252:255], v[120:123], a[204:207], v[252:255]\n" \
  "v_mfma_i32_16x16x64_i8 v[252:255], v[124:127], a[220:223], v[252:255]\n" \
  "; ---- step 7\n" \
  "s_waitcnt vmcnt(16)\n" \
  "s_waitcnt lgkmcnt(8)\n" \
  "v_mfma_i32_16x16x64_i8 v[128:131], v[96:99], a[224:227], v[128:131]\n" \
  "ds_read_b64_tr_b8 v[120:121], v83 offset:49152\n" \
  "v_mfma_i32_16x16x64_i8 v[128:131], v[100:103], a[240:243], v[128:131]\n" \
  "ds_read_b64_tr_b8 v[122:123], v83 offset:50176\n" \
  "v_mfma_i32_16x16x64_i8 v[160:163], v[96:99], a[228:231], v[160:163]\n" \
  "v_mfma_i32_16x16x64_i8 v[160:163], v[100:103], a[244:247], v[160:163]\n" \
  "v_mfma_i32_16x16x64_i8 v[192:195], v[96:99], a[232:235], v[192:195]\n" \
  "ds_read_b64_tr_b8 v[124:125], v83 offset:57344\n" \
  "v_mfma_i32_16x16x64_i8 v[192:195], v[100:103], a[248:251], v[192:195]\n" \
  "ds_read_b64_tr_b8 v[126:127], v83 offset:58368\n" \
  "v_mfma_i32_16x16x64_i8 v[224:227], v[96:99], a[236:239], v[224:227]\n" \
  "v_mfma_i32_16x16x64_i8 v[224:227], v[100:103], a[252:255], v[224:227]\n" \
  "s_waitcnt lgkmcnt(8)\n" \
  "v_mfma_i32_16x16x64_i8 v[132:135], v[104:107], a[224:227], v[132:135]\n" \
  "ds_read_b64_tr_b8 v[96:97], v84 offset:49152\n" \
  "v_mfma_i32_16x16x64_i8 v[132:135], v[108:111], a[240:243], v[132:135]\n" \
  "ds_read_b64_tr_b8 v[98:99], v84 offset:50176\n" \
  "v_mfma_i32_16x16x64_i8 v[164:167], v[104:107], a[228:231], v[164:167]\n" \
  "v_mfma_i32_16x16x64_i8 v[164:167], v[108:111], a[244:247], v[164:167]\n" \
  "v_mfma_i32_16x16x64_i8 v[196:199], v[104:107], a[232:235], v[196:199]\n" \
  "ds_read_b64_tr_b8 v[100:101], v84 offset:57344\n" \
  "v_mfma_i32_16x16x64_i8 v[196:199], v[108:111], a[248:251], v[196:199]\n" \
  "ds_read_b64_tr_b8 v[102:103], v84 offset:58368\n" \
  "v_mfma_i32_16x16x64_i8 v[228:231], v[104:107], a[236:239], v[228:231]\n" \
  "v_mfma_i32_16x16x64_i8 v[228:231], v[108:111], a[252:255], v[228:231]\n" \
  "s_waitcnt lgkmcnt(8)\n" \
  "v_mfma_i32_16x16x64_i8 v[136:139], v[112:115], a[224:227], v[136:139]\n" \
  "ds_read_b64_tr_b8 v[104:105], v85 offset:49152\n" \
  "v_mfma_i32_16x16x64_i8 v[136:139], v[116:119], a[240:243], v[136:139]\n" \
  "ds_read_b64_tr_b8 v[106:107], v85 offset:50176\n" \
  "v_mfma_i32_16x16x64_i8 v[168:171], v[112:115], a[228:231], v[168:171]\n" \
  "v_mfma_i32_16x16x64_i8 v[168:171], v[116:119], a[244:247], v[168:171]\n" \
  "v_mfma_i32_16x16x64_i8 v[200:203], v[112:115], a[232:235], v[200:203]\n" \
  "ds_read_b64_tr_b8 v[108:109], v85 offset:57344\n" \
  "v_mfma_i32_16x16x64_i8 v[200:203], v[116:119], a[248:251], v[200:203]\n" \
  "ds_read_b64_tr_b8 v[110:111], v85 offset:58368\n" \
  "buffer_load_dwordx4 a[32:35], v44, s[28:31], s89 offen offset:0 sc0 sc1\n" \
  "v_mfma_i32_16x16x64_i8 v[232:235], v[112:115], a[236:239], v[232:235]\n" \
  "v_mfma_i32_16x16x64_i8 v[232:235], v[116:119], a[252:255], v[232:235]\n" \
  "buffer_load_dwordx4 a[36:39], v44, s[28:31], s89 offen offset:1024 sc0 sc1\n" \
  "s_waitcnt lgkmcnt(8)\n" \
  "v_mfma_i32_16x16x64_i8 v[140:143], v[120:123], a[224:227], v[140:143]\n" \
  "ds_read_b64_tr_b8 v[112:113], v86 offset:49152\n" \
  "v_mfma_i32_16x16x64_i8 v[140:143], v[124:127], a[240:243], v[140:143]\n" \
  "ds_read_b64_tr_b8 v[114:115], v86 offset:50176\n" \
  "buffer_load_dwordx4 a[40:43], v44, s[28:31], s89 offen offset:2048 sc0 sc1\n" \
  "v_mfma_i32_16x16x64_i8 v[172:175], v[120:123], a[228:231], v[172:175]\n" \
  "v_mfma_i32_16x16x64_i8 v[172:175], v[124:127], a[244:247], v[172:175]\n" \
  "s_cmp_lg_u32 s46, 2\n" \
  "s_cbranch_scc1 Lskip17_%=\n" \
  "s_waitcnt vmcnt(11)\n" \
  "Lskip17_%=:\n" \
  "s_waitcnt vmcnt(60)\n" \
  "s_barrier\n" \
  "s_cmp_lg_u32 s46, 2\n" \
  "s_cbranch_scc1 Lskip19_%=\n" \
  "s_cmp_lg_u32 s11, 0\n" \
  "s_cbranch_scc1 Lskip18_%=\n" \
  "s_mov_b64 exec, 1\n" \
  "global_store_dword v53, v76, s[36:37] sc0 sc1\n" \
  "s_mov_b64 exec, -1\n" \
  "Lskip18_%=:\n" \
  "Lskip19_%=:\n" \
  "v_mfma_i32_16x16x64_i8 v[204:207], v[120:123], a[232:235], v[204:207]\n" \
  "ds_read_b64_tr_b8 v[116:117], v86 offset:57344\n" \
  "v_mfma_i32_16x16x64_i8 v[204:207], v[124:127], a[248:251], v[204:207]\n" \
  "ds_read_b64_tr_b8 v[118:119], v86 offset:58368\n" \
  "buffer_load_dwordx4 a[44:47], v44, s[28:31], s89 offen offset:3072 sc0 sc1\n" \
  "v_mfma_i32_16x16x64_i8 v[236:239], v[120:123], a[236:239], v[236:239]\n" \
  "v_mfma_i32_16x16x64_i8 v[236:239], v[124:127], a[252:255], v[236:239]\n" \
  "buffer_load_dwordx4 a[48:51], v91, s[28:31], s89 offen offset:0 sc0 sc1\n" \
  "s_waitcnt lgkmcnt(8)\n" \
  "v_mfma_i32_16x16x64_i8 v[144:147], v[96:99], a[224:227], v[144:147]\n" \
  "ds_read_b64_tr_b8 v[120:121], v87 offset:49152\n" \
  "v_mfma_i32_16x16x64_i8 v[144:147], v[100:103], a[240:243], v[144:147]\n" \
  "ds_read_b64_tr_b8 v[122:123], v87 offset:50176\n" \
  "buffer_load_dwordx4 a[52:55], v91, s[28:31], s89 offen offset:1024 sc0 sc1\n" \
  "v_mfma_i32_16x16x64_i8 v[176:179], v[96:99], a[228:231], v[176:179]\n" \
  "v_mfma_i32_16x16x64_i8 v[176:179], v[100:103], a[244:247], v[176:179]\n" \
  "v_mfma_i32_16x16x64_i8 v[208:211], v[96:99], a[232:235], v[208:211]\n" \
  "ds_read_b64_tr_b8 v[124:125], v87 offset:57344\n" \
  "v_mfma_i32_16x16x64_i8 v[208:211], v[100:103], a[248:251], v[208:211]\n" \
  "ds_read_b64_tr_b8 v[126:127], v87 offset:58368\n" \
  "buffer_load_dwordx4 a[56:59], v91, s[28:31], s89 offen offset:2048 sc0 sc1\n" \
  "v_mfma_i32_16x16x64_i8 v[240:243], v[96:99], a[236:239], v[240:243]\n" \
  "v_readlane_b32 s50, v46, s49\n" \
  "v_mfma_i32_16x16x64_i8 v[240:243], v[100:103], a[252:255], v[240:243]\n" \
  "buffer_load_dwordx4 a[60:63], v91, s[28:31], s89 offen offset:3072 sc0 sc1\n" \
  "s_waitcnt lgkmcnt(8)\n" \
  "v_mfma_i32_16x16x64_i8 v[148:151], v[104:107], a[224:227], v[148:151]\n" \
  "ds_read_b64_tr_b8 v[96:97], v32 offset:0\n" \
  "v_mfma_i32_16x16x64_i8 v[148:151], v[108:111], a[240:243], v[148:151]\n" \
  "ds_read_b64_tr_b8 v[98:99], v32 offset:1024\n" \
  "v_mfma_i32_16x16x64_i8 v[180:183], v[104:107], a[228:231], v[180:183]\n" \
  "v_mfma_i32_16x16x64_i8 v[180:183], v[108:111], a[244:247], v[180:183]\n" \
  "buffer_load_dwordx4 a[64:67], v44, s[28:31], s90 offen offset:0 sc0 sc1\n" \
  "v_mfma_i32_16x16x64_i8 v[212:215], v[104:107], a[232:235], v[212:215]\n" \
  "ds_read_b64_tr_b8 v[100:101], v32 offset:8192\n" \
  "v_mfma_i32_16x16x64_i8 v[212:215], v[108:111], a[248:251], v[212:215]\n" \
  "ds_read_b64_tr_b8 v[102:103], v32 offset:9216\n" \
  "buffer_load_dwordx4 a[68:71], v44, s[28:31], s90 offen offset:1024 sc0 sc1\n" \
  "v_mfma_i32_16x16x64_i8 v[244:247], v[104:107], a[236:239], v[244:247]\n" \
  "s_lshl_b32 s85, s50, 21\n" \
  "s_lshl_b32 s93, s50, 16\n" \
  "s_add_u32 s49, s49, 1\n" \
  "v_mfma_i32_16x16x64_i8 v[244:247], v[108:111], a[252:255], v[244:247]\n" \
  "buffer_load_dwordx4 a[72:75], v44, s[28:31], s90 offen offset:2048 sc0 sc1\n" \
  "s_waitcnt lgkmcnt(8)\n" \
  "v_mfma_i32_16x16x64_i8 v[152:155], v[112:115], a[224:227], v[152:155]\n" \
  "ds_read_b64_tr_b8 v[104:105], v33 offset:0\n" \
  "v_mfma_i32_16x16x64_i8 v[152:155], v[116:119], a[240:243], v[152:155]\n" \
  "ds_read_b64_tr_b8 v[106:107], v33 offset:1024\n" \
  "v_mfma_i32_16x16x64_i8 v[184:187], v[112:115], a[228:231], v[184:187]\n" \
  "v_readlane_b32 s50, v46, s49\n" \
  "v_mfma_i32_16x16x64_i8 v[184:187], v[116:119], a[244:247], v[184:187]\n" \
  "buffer_load_dwordx4 a[76:79], v44, s[28:31], s90 offen offset:3072 sc0 sc1\n" \
  "s_cmp_lg_u32 s11, 0\n" \
  "s_cbranch_scc1 Lskip22_%=\n" \
  "s_waitcnt vmcnt(20)\n" \
  "v_sub_u32 v72, v72, v76\n" \
  "v_cmp_gt_i32 vcc, 0, v72\n" \
  "s_nop 1\n" \
  "s_andn2_b64 vcc, vcc, s[14:15]\n" \
  "s_cbranch_scc1 Lslow20_%=\n" \
  "Lback21_%=:\n" \
  "v_add_u32 v50, 1024, v50\n" \
  "Lskip22_%=:\n" \
  "v_mfma_i32_16x16x64_i8 v[216:219], v[112:115], a[232:235], v[216:219]\n" \
  "ds_read_b64_tr_b8 v[108:109], v33 offset:8192\n" \
  "v_mfma_i32_16x16x64_i8 v[216:219], v[116:119], a[248:251], v[216:219]\n" \
  "ds_read_b64_tr_b8 v[110:111], v33 offset:9216\n" \
  "buffer_load_dwordx4 a[80:83], v91, s[28:31], s90 offen offset:0 sc0 sc1\n" \
  "v_mfma_i32_16x16x64_i8 v[248:251], v[112:115], a[236:239], v[248:251]\n" \
  "v_mfma_i32_16x16x64_i8 v[248:251], v[116:119], a[252:255], v[248:251]\n" \
  "buffer_load_dwordx4 a[84:87], v91, s[28:31], s90 offen offset:1024 sc0 sc1\n" \
  "s_waitcnt lgkmcnt(8)\n" \
  "v_mfma_i32_16x16x64_i8 v[156:159], v[120:123], a[224:227], v[156:159]\n" \
  "ds_read_b64_tr_b8 v[112:113], v34 offset:0\n" \
  "v_mfma_i32_16x16x64_i8 v[156:159], v[124:127], a[240:243], v[156:159]\n" \
  "ds_read_b64_tr_b8 v[114:115], v34 offset:1024\n" \
  "v_mfma_i32_16x16x64_i8 v[188:191], v[120:123], a[228:231], v[188:191]\n" \
  "s_lshl_b32 s86, s50, 21\n" \
  "s_lshl_b32 s94, s50, 16\n" \
  "s_add_u32 s49, s49, 1\n" \
  "v_mfma_i32_16x16x64_i8 v[188:191], v[124:127], a[244:247], v[188:191]\n" \
  "buffer_load_dwordx4 a[88:91], v91, s[28:31], s90 offen offset:2048 sc0 sc1\n" \
  "v_mfma_i32_16x16x64_i8 v[220:223], v[120:123], a[232:235], v[220:223]\n" \
  "ds_read_b64_tr_b8 v[116:117], v34 offset:8192\n" \
  "v_mfma_i32_16x16x64_i8 v[220:223], v[124:127], a[248:251], v[220:223]\n" \
  "ds_read_b64_tr_b8 v[118:119], v34 offset:9216\n" \
  "buffer_load_dwordx4 a[92:95], v91, s[28:31], s90 offen offset:3072 sc0 sc1\n" \
  "v_mfma_i32_16x16x64_i8 v[252:255], v[120:123], a[236:239], v[252:255]\n" \
  "v_mfma_i32_16x16x64_i8 v[252:255], v[124:127], a[252:255], v[252:255]\n" \
  "s_barrier\n" \
  "buffer_load_dword v92, v52, s[28:31], 0 offen sc0 sc1\n" \
  "v_add_u32 v52, 0x10000, v52\n" \
  "s_add_u32 s53, s53, 1\n" \
  "s_add_u32 s54, s54, 1\n" \
  "s_mov_b32 s51, 2\n" \
  "L_loop_%=:\n" \
  "; ---- step 0\n" \
  "s_waitcnt vmcnt(25)\n" \
  "s_waitcnt lgkmcnt(8)\n" \
  "v_mfma_i32_16x16x64_i8 v[128:131], v[96:99], a[0:3], v[128:131]\n" \
  "ds_read_b64_tr_b8 v[120:121], v35 offset:0\n" \
  "s_add_u32 m0, s47, 81920\n" \
  "v_mfma_i32_16x16x64_i8 v[128:131], v[100:103], a[16:19], v[128:131]\n" \
  "ds_read_b64_tr_b8 v[122:123], v35 offset:1024\n" \
  "buffer_load_dwordx4 v40, s[16:19], s85 offen nt lds\n" \
  "s_add_u32 m0, s47, 82944\n" \
  "v_mfma_i32_16x16x64_i8 v[160:163], v[96:99], a[4:7], v[160:163]\n" \
  "buffer_load_dwordx4 v41, s[16:19], s85 offen nt lds\n" \
  "s_add_u32 m0, s47, 83968\n" \
  "v_mfma_i32_16x16x64_i8 v[160:163], v[100:103], a[20:23], v[160:163]\n" \
  "buffer_load_dwordx4 v42, s[16:19], s85 offen nt lds\n" \
  "s_add_u32 m0, s47, 84992\n" \
  "v_mfma_i32_16x16x64_i8 v[192:195], v[96:99], a[8:11], v[192:195]\n" \
  "ds_read_b64_tr_b8 v[124:125], v35 offset:8192\n" \
  "buffer_load_dwordx4 v43, s[16:19], s85 offen nt lds\n" \
  "s_add_u32 m0, s47, 98304\n" \
  "v_mfma_i32_16x16x64_i8 v[192:195], v[100:103], a[24:27], v[192:195]\n" \
  "ds_read_b64_tr_b8 v[126:127], v35 offset:9216\n" \
  "buffer_load_dwordx4 v40, s[16:19], s86 offen nt lds\n" \
  "s_add_u32 m0, s47, 99328\n" \
  "v_mfma_i32_16x16x64_i8 v[224:227], v[96:99], a[12:15], v[224:227]\n" \
  "buffer_load_dwordx4 v41, s[16:19], s86 offen nt lds\n" \
  "s_add_u32 m0, s47, 100352\n" \
  "v_mfma_i32_16x16x64_i8 v[224:227], v[100:103], a[28:31], v[224:227]\n" \
  "buffer_load_dwordx4 v42, s[16:19], s86 offen nt lds\n" \
  "s_add_u32 m0, s47, 101376\n" \
  "s_waitcnt lgkmcnt(8)\n" \
  "v_mfma_i32_16x16x64_i8 v[132:135], v[104:107], a[0:3], v[132:135]\n" \
  "ds_read_b64_tr_b8 v[96:97], v36 offset:0\n" \
  "buffer_load_dwordx4 v43, s[16:19], s86 offen nt lds\n" \
  "v_mfma_i32_16x16x64_i8 v[132:135], v[108:111], a[16:19], v[132:135]\n" \
  "ds_read_b64_tr_b8 v[98:99], v36 offset:1024\n" \
  "v_mfma_i32_16x16x64_i8 v[164:167], v[104:107], a[4:7], v[164:167]\n" \
  "v_mfma_i32_16x16x64_i8 v[164:167], v[108:111], a[20:23], v[164:167]\n" \
  "v_mfma_i32_16x16x64_i8 v[196:199], v[104:107], a[8:11], v[196:199]\n" \
  "ds_read_b64_tr_b8 v[100:101], v36 offset:8192\n" \
  "v_mfma_i32_16x16x64_i8 v[196:199], v[108:111], a[24:27], v[196:199]\n" \
  "ds_read_b64_tr_b8 v[102:103], v36 offset:9216\n" \
  "v_mfma_i32_16x16x64_i8 v[228:231], v[104:107], a[12:15], v[228:231]\n" \
  "v_mfma_i32_16x16x64_i8 v[228:231], v[108:111], a[28:31], v[228:231]\n" \
  "s_waitcnt lgkmcnt(8)\n" \
  "v_mfma_i32_16x16x64_i8 v[136:139], v[112:115], a[0:3], v[136:139]\n" \
  "ds_read_b64_tr_b8 v[104:105], v37 offset:0\n" \
  "v_mfma_i32_16x16x64_i8 v[136:139], v[116:119], a[16:19], v[136:139]\n" \
  "ds_read_b64_tr_b8 v[106:107], v37 offset:1024\n" \
  "v_mfma_i32_16x16x64_i8 v[168:171], v[112:115], a[4:7], v[168:171]\n" \
  "v_mfma_i32_16x16x64_i8 v[168:171], v[116:119], a[20:23], v[168:171]\n" \
  "v_mfma_i32_16x16x64_i8 v[200:203], v[112:115], a[8:11], v[200:203]\n" \
  "ds_read_b64_tr_b8 v[108:109], v37 offset:8192\n" \
  "v_mfma_i32_16x16x64_i8 v[200:203], v[116:119], a[24:27], v[200:203]\n" \
  "ds_read_b64_tr_b8 v[110:111], v37 offset:9216\n" \
  "v_mfma_i32_16x16x64_i8 v[232:235], v[112:115], a[12:15], v[232:235]\n" \
  "v_mfma_i32_16x16x64_i8 v[232:235], v[116:119], a[28:31], v[232:235]\n" \
  "s_waitcnt lgkmcnt(8)\n" \
  "v_mfma_i32_16x16x64_i8 v[140:143], v[120:123], a[0:3], v[140:143]\n" \
  "ds_read_b64_tr_b8 v[112:113], v38 offset:0\n" \
  "v_mfma_i32_16x16x64_i8 v[140:143], v[124:127], a[16:19], v[140:143]\n" \
  "ds_read_b64_tr_b8 v[114:115], v38 offset:1024\n" \
  "v_mfma_i32_16x16x64_i8 v[172:175], v[120:123], a[4:7], v[172:175]\n" \
  "v_mfma_i32_16x16x64_i8 v[172:175], v[124:127], a[20:23], v[172:175]\n" \
  "s_waitcnt vmcnt(53)\n" \
  "s_barrier\n" \
  "v_mfma_i32_16x16x64_i8 v[204:207], v[120:123], a[8:11], v[204:207]\n" \
  "ds_read_b64_tr_b8 v[116:117], v38 offset:8192\n" \
  "v_mfma_i32_16x16x64_i8 v[204:207], v[124:127], a[24:27], v[204:207]\n" \
  "ds_read_b64_tr_b8 v[118:119], v38 offset:9216\n" \
  "v_mfma_i32_16x16x64_i8 v[236:239], v[120:123], a[12:15], v[236:239]\n" \
  "v_mfma_i32_16x16x64_i8 v[236:239], v[124:127], a[28:31], v[236:239]\n" \
  "s_waitcnt lgkmcnt(8)\n" \
  "v_mfma_i32_16x16x64_i8 v[144:147], v[96:99], a[0:3], v[144:147]\n" \
  "ds_read_b64_tr_b8 v[120:121], v39 offset:0\n" \
  "v_mfma_i32_16x16x64_i8 v[144:147], v[100:103], a[16:19], v[144:147]\n" \
  "ds_read_b64_tr_b8 v[122:123], v39 offset:1024\n" \
  "v_mfma_i32_16x16x64_i8 v[176:179], v[96:99], a[4:7], v[176:179]\n" \
  "v_mfma_i32_16x16x64_i8 v[176:179], v[100:103], a[20:23], v[176:179]\n" \
  "v_mfma_i32_16x16x64_i8 v[208:211], v[96:99], a[8:11], v[208:211]\n" \
  "ds_read_b64_tr_b8 v[124:125], v39 offset:8192\n" \
  "v_mfma_i32_16x16x64_i8 v[208:211], v[100:103], a[24:27], v[208:211]\n" \
  "ds_read_b64_tr_b8 v[126:127], v39 offset:9216\n" \
  "v_mfma_i32_16x16x64_i8 v[240:243], v[96:99], a[12:15], v[240:243]\n" \
  "v_mfma_i32_16x16x64_i8 v[240:243], v[100:103], a[28:31], v[240:243]\n" \
  "s_waitcnt lgkmcnt(8)\n" \
  "v_mfma_i32_16x16x64_i8 v[148:151], v[104:107], a[0:3], v[148:151]\n" \
  "ds_read_b64_tr_b8 v[96:97], v32 offset:16384\n" \
  "v_mfma_i32_16x16x64_i8 v[148:151], v[108:111], a[16:19], v[148:151]\n" \
  "ds_read_b64_tr_b8 v[98:99], v32 offset:17408\n" \
  "v_mfma_i32_16x16x64_i8 v[180:183], v[104:107], a[4:7], v[180:183]\n" \
  "v_mfma_i32_16x16x64_i8 v[180:183], v[108:111], a[20:23], v[180:183]\n" \
  "v_mfma_i32_16x16x64_i8 v[212:215], v[104:107], a[8:11], v[212:215]\n" \
  "ds_read_b64_tr_b8 v[100:101], v32 offset:24576\n" \
  "v_mfma_i32_16x16x64_i8 v[212:215], v[108:111], a[24:27], v[212:215]\n" \
  "ds_read_b64_tr_b8 v[102:103], v32 offset:25600\n" \
  "v_mfma_i32_16x16x64_i8 v[244:247], v[104:107], a[12:15], v[244:247]\n" \
  "v_mfma_i32_16x16x64_i8 v[244:247], v[108:111], a[28:31], v[244:247]\n" \
  "s_waitcnt lgkmcnt(8)\n" \
  "v_mfma_i32_16x16x64_i8 v[152:155], v[112:115], a[0:3], v[152:155]\n" \
  "ds_read_b64_tr_b8 v[104:105], v33 offset:16384\n" \
  "v_mfma_i32_16x16x64_i8 v[152:155], v[116:119], a[16:19], v[152:155]\n" \
  "ds_read_b64_tr_b8 v[106:107], v33 offset:17408\n" \
  "v_mfma_i32_16x16x64_i8 v[184:187], v[112:115], a[4:7], v[184:187]\n" \
  "v_mfma_i32_16x16x64_i8 v[184:187], v[116:119], a[20:23], v[184:187]\n" \
  "v_mfma_i32_16x16x64_i8 v[216:219], v[112:115], a[8:11], v[216:219]\n" \
  "ds_read_b64_tr_b8 v[108:109], v33 offset:24576\n" \
  "v_mfma_i32_16x16x64_i8 v[216:219], v[116:119], a[24:27], v[216:219]\n" \
  "ds_read_b64_tr_b8 v[110:111], v33 offset:25600\n" \
  "v_mfma_i32_16x16x64_i8 v[248:251], v[112:115], a[12:15], v[248:251]\n" \
  "v_mfma_i32_16x16x64_i8 v[248:251], v[116:119], a[28:31], v[248:251]\n" \
  "s_waitcnt lgkmcnt(8)\n" \
  "v_mfma_i32_16x16x64_i8 v[156:159], v[120:123], a[0:3], v[156:159]\n" \
  "ds_read_b64_tr_b8 v[112:113], v34 offset:16384\n" \
  "v_mfma_i32_16x16x64_i8 v[156:159], v[124:127], a[16:19], v[156:159]\n" \
  "ds_read_b64_tr_b8 v[114:115], v34 offset:17408\n" \
  "v_mfma_i32_16x16x64_i8 v[188:191], v[120:123], a[4:7], v[188:191]\n" \
  "v_mfma_i32_16x16x64_i8 v[188:191], v[124:127], a[20:23], v[188:191]\n" \
  "v_mfma_i32_16x16x64_i8 v[220:223], v[120:123], a[8:11], v[220:223]\n" \
  "ds_read_b64_tr_b8 v[116:117], v34 offset:24576\n" \
  "v_mfma_i32_16x16x64_i8 v[220:223], v[124:127], a[24:27], v[220:223]\n" \
  "ds_read_b64_tr_b8 v[118:119], v34 offset:25600\n" \
  "v_mfma_i32_16x16x64_i8 v[252:255], v[120:123], a[12:15], v[252:255]\n" \
  "v_mfma_i32_16x16x64_i8 v[252:255], v[124:127], a[28:31], v[252:255]\n" \
  "; ---- step 1\n" \
  "s_waitcnt vmcnt(16)\n" \
  "s_waitcnt lgkmcnt(8)\n" \
  "v_mfma_i32_16x16x64_i8 v[128:131], v[96:99], a[32:35], v[128:131]\n" \
  "ds_read_b64_tr_b8 v[120:121], v35 offset:16384\n" \
  "v_mfma_i32_16x16x64_i8 v[128:131], v[100:103], a[48:51], v[128:131]\n" \
  "ds_read_b64_tr_b8 v[122:123], v35 offset:17408\n" \
  "v_mfma_i32_16x16x64_i8 v[160:163], v[96:99], a[36:39], v[160:163]\n" \
  "v_mfma_i32_16x16x64_i8 v[160:163], v[100:103], a[52:55], v[160:163]\n" \
  "v_mfma_i32_16x16x64_i8 v[192:195], v[96:99], a[40:43], v[192:195]\n" \
  "ds_read_b64_tr_b8 v[124:125], v35 offset:24576\n" \
  "v_mfma_i32_16x16x64_i8 v[192:195], v[100:103], a[56:59], v[192:195]\n" \
  "ds_read_b64_tr_b8 v[126:127], v35 offset:25600\n" \
  "v_mfma_i32_16x16x64_i8 v[224:227], v[96:99], a[44:47], v[224:227]\n" \
  "v_mfma_i32_16x16x64_i8 v[224:227], v[100:103], a[60:63], v[224:227]\n" \
  "s_waitcnt lgkmcnt(8)\n" \
  "v_mfma_i32_16x16x64_i8 v[132:135], v[104:107], a[32:35], v[132:135]\n" \
  "ds_read_b64_tr_b8 v[96:97], v36 offset:16384\n" \
  "v_mfma_i32_16x16x64_i8 v[132:135], v[108:111], a[48:51], v[132:135]\n" \
  "ds_read_b64_tr_b8 v[98:99], v36 offset:17408\n" \
  "v_mfma_i32_16x16x64_i8 v[164:167], v[104:107], a[36:39], v[164:167]\n" \
  "v_mfma_i32_16x16x64_i8 v[164:167], v[108:111], a[52:55], v[164:167]\n" \
  "v_mfma_i32_16x16x64_i8 v[196:199], v[104:107], a[40:43], v[196:199]\n" \
  "ds_read_b64_tr_b8 v[100:101], v36 offset:24576\n" \
  "v_mfma_i32_16x16x64_i8 v[196:199], v[108:111], a[56:59], v[196:199]\n" \
  "ds_read_b64_tr_b8 v[102:103], v36 offset:25600\n" \
  "v_mfma_i32_16x16x64_i8 v[228:231], v[104:107], a[44:47], v[228:231]\n" \
  "v_mfma_i32_16x16x64_i8 v[228:231], v[108:111], a[60:63], v[228:231]\n" \
  "s_waitcnt lgkmcnt(8)\n" \
  "v_mfma_i32_16x16x64_i8 v[136:139], v[112:115], a[32:35], v[136:139]\n" \
  "ds_read_b64_tr_b8 v[104:105], v37 offset:16384\n" \
  "v_mfma_i32_16x16x64_i8 v[136:139], v[116:119], a[48:51], v[136:139]\n" \
  "ds_read_b64_tr_b8 v[106:107], v37 offset:17408\n" \
  "v_mfma_i32_16x16x64_i8 v[168:171], v[112:115], a[36:39], v[168:171]\n" \
  "v_mfma_i32_16x16x64_i8 v[168:171], v[116:119], a[52:55], v[168:171]\n" \
  "v_mfma_i32_16x16x64_i8 v[200:203], v[112:115], a[40:43], v[200:203]\n" \
  "ds_read_b64_tr_b8 v[108:109], v37 offset:24576\n" \
  "v_mfma_i32_16x16x64_i8 v[200:203], v[116:119], a[56:59], v[200:203]\n" \
  "ds_read_b64_tr_b8 v[110:111], v37 offset:25600\n" \
  "buffer_load_dwordx4 a[96:99], v44, s[28:31], s91 offen offset:0 sc0 sc1\n" \
  "v_mfma_i32_16x16x64_i8 v[232:235], v[112:115], a[44:47], v[232:235]\n" \
  "v_mfma_i32_16x16x64_i8 v[232:235], v[116:119], a[60:63], v[232:235]\n" \
  "buffer_load_dwordx4 a[100:103], v44, s[28:31], s91 offen offset:1024 sc0 sc1\n" \
  "s_waitcnt lgkmcnt(8)\n" \
  "v_mfma_i32_16x16x64_i8 v[140:143], v[120:123], a[32:35], v[140:143]\n" \
  "ds_read_b64_tr_b8 v[112:113], v38 offset:16384\n" \
  "v_mfma_i32_16x16x64_i8 v[140:143], v[124:127], a[48:51], v[140:143]\n" \
  "ds_read_b64_tr_b8 v[114:115], v38 offset:17408\n" \
  "buffer_load_dwordx4 a[104:107], v44, s[28:31], s91 offen offset:2048 sc0 sc1\n" \
  "v_mfma_i32_16x16x64_i8 v[172:175], v[120:123], a[36:39], v[172:175]\n" \
  "v_mfma_i32_16x16x64_i8 v[172:175], v[124:127], a[52:55], v[172:175]\n" \
  "s_waitcnt vmcnt(52)\n" \
  "s_barrier\n" \
  "v_mfma_i32_16x16x64_i8 v[204:207], v[120:123], a[40:43], v[204:207]\n" \
  "ds_read_b64_tr_b8 v[116:117], v38 offset:24576\n" \
  "v_mfma_i32_16x16x64_i8 v[204:207], v[124:127], a[56:59], v[204:207]\n" \
  "ds_read_b64_tr_b8 v[118:119], v38 offset:25600\n" \
  "buffer_load_dwordx4 a[108:111], v44, s[28:31], s91 offen offset:3072 sc0 sc1\n" \
  "v_mfma_i32_16x16x64_i8 v[236:239], v[120:123], a[44:47], v[236:239]\n" \
  "v_mfma_i32_16x16x64_i8 v[236:239], v[124:127], a[60:63], v[236:239]\n" \
  "buffer_load_dwordx4 a[112:115], v91, s[28:31], s91 offen offset:0 sc0 sc1\n" \
  "s_waitcnt lgkmcnt(8)\n" \
  "v_mfma_i32_16x16x64_i8 v[144:147], v[96:99], a[32:35], v[144:147]\n" \
  "ds_read_b64_tr_b8 v[120:121], v39 offset:16384\n" \
  "v_mfma_i32_16x16x64_i8 v[144:147], v[100:103], a[48:51], v[144:147]\n" \
  "ds_read_b64_tr_b8 v[122:123], v39 offset:17408\n" \
  "buffer_load_dwordx4 a[116:119], v91, s[28:31], s91 offen offset:1024 sc0 sc1\n" \
  "v_mfma_i32_16x16x64_i8 v[176:179], v[96:99], a[36:39], v[176:179]\n" \
  "v_mfma_i32_16x16x64_i8 v[176:179], v[100:103], a[52:55], v[176:179]\n" \
  "v_mfma_i32_16x16x64_i8 v[208:211], v[96:99], a[40:43], v[208:211]\n" \
  "ds_read_b64_tr_b8 v[124:125], v39 offset:24576\n" \
  "v_mfma_i32_16x16x64_i8 v[208:211], v[100:103], a[56:59], v[208:211]\n" \
  "ds_read_b64_tr_b8 v[126:127], v39 offset:25600\n" \
  "buffer_load_dwordx4 a[120:123], v91, s[28:31], s91 offen offset:2048 sc0 sc1\n" \
  "v_mfma_i32_16x16x64_i8 v[240:243], v[96:99], a[44:47], v[240:243]\n" \
  "v_readlane_b32 s50, v46, s49\n" \
  "v_mfma_i32_16x16x64_i8 v[240:243], v[100:103], a[60:63], v[240:243]\n" \
  "buffer_load_dwordx4 a[124:127], v91, s[28:31], s91 offen offset:3072 sc0 sc1\n" \
  "s_waitcnt lgkmcnt(8)\n" \
  "v_mfma_i32_16x16x64_i8 v[148:151], v[104:107], a[32:35], v[148:151]\n" \
  "ds_read_b64_tr_b8 v[96:97], v32 offset:32768\n" \
  "v_mfma_i32_16x16x64_i8 v[148:151], v[108:111], a[48:51], v[148:151]\n" \
  "ds_read_b64_tr_b8 v[98:99], v32 offset:33792\n" \
  "v_mfma_i32_16x16x64_i8 v[180:183], v[104:107], a[36:39], v[180:183]\n" \
  "v_mfma_i32_16x16x64_i8 v[180:183], v[108:111], a[52:55], v[180:183]\n" \
  "buffer_load_dwordx4 a[128:131], v44, s[28:31], s92 offen offset:0 sc0 sc1\n" \
  "v_mfma_i32_16x16x64_i8 v[212:215], v[104:107], a[40:43], v[212:215]\n" \
  "ds_read_b64_tr_b8 v[100:101], v32 offset:40960\n" \
  "v_mfma_i32_16x16x64_i8 v[212:215], v[108:111], a[56:59], v[212:215]\n" \
  "ds_read_b64_tr_b8 v[102:103], v32 offset:41984\n" \
  "buffer_load_dwordx4 a[132:135], v44, s[28:31], s92 offen offset:1024 sc0 sc1\n" \
  "v_mfma_i32_16x16x64_i8 v[244:247], v[104:107], a[44:47], v[244:247]\n" \
  "s_lshl_b32 s87, s50, 21\n" \
  "s_lshl_b32 s95, s50, 16\n" \
  "s_add_u32 s49, s49, 1\n" \
  "v_mfma_i32_16x16x64_i8 v[244:247], v[108:111], a[60:63], v[244:247]\n" \
  "buffer_load_dwordx4 a[136:139], v44, s[28:31], s92 offen offset:2048 sc0 sc1\n" \
  "s_waitcnt lgkmcnt(8)\n" \
  "v_mfma_i32_16x16x64_i8 v[152:155], v[112:115], a[32:35], v[152:155]\n" \
  "ds_read_b64_tr_b8 v[104:105], v33 offset:32768\n" \
  "v_mfma_i32_16x16x64_i8 v[152:155], v[116:119], a[48:51], v[152:155]\n" \
  "ds_read_b64_tr_b8 v[106:107], v33 offset:33792\n" \
  "v_mfma_i32_16x16x64_i8 v[184:187], v[112:115], a[36:39], v[184:187]\n" \
  "v_readlane_b32 s50, v46, s49\n" \
  "v_mfma_i32_16x16x64_i8 v[184:187], v[116:119], a[52:55], v[184:187]\n" \
  "buffer_load_dwordx4 a[140:143], v44, s[28:31], s92 offen offset:3072 sc0 sc1\n" \
  "v_mfma_i32_16x16x64_i8 v[216:219], v[112:115], a[40:43], v[216:219]\n" \
  "ds_read_b64_tr_b8 v[108:109], v33 offset:40960\n" \
  "v_mfma_i32_16x16x64_i8 v[216:219], v[116:119], a[56:59], v[216:219]\n" \
  "ds_read_b64_tr_b8 v[110:111], v33 offset:41984\n" \
  "buffer_load_dwordx4 a[144:147], v91, s[28:31], s92 offen offset:0 sc0 sc1\n" \
  "v_mfma_i32_16x16x64_i8 v[248:251], v[112:115], a[44:47], v[248:251]\n" \
  "v_mfma_i32_16x16x64_i8 v[248:251], v[116:119], a[60:63], v[248:251]\n" \
  "buffer_load_dwordx4 a[148:151], v91, s[28:31], s92 offen offset:1024 sc0 sc1\n" \
  "s_waitcnt lgkmcnt(8)\n" \
  "v_mfma_i32_16x16x64_i8 v[156:159], v[120:123], a[32:35], v[156:159]\n" \
  "ds_read_b64_tr_b8 v[112:113], v34 offset:32768\n" \
  "v_mfma_i32_16x16x64_i8 v[156:159], v[124:127], a[48:51], v[156:159]\n" \
  "ds_read_b64_tr_b8 v[114:115], v34 offset:33792\n" \
  "v_mfma_i32_16x16x64_i8 v[188:191], v[120:123], a[36:39], v[188:191]\n" \
  "s_lshl_b32 s80, s50, 21\n" \
  "s_lshl_b32 s88, s50, 16\n" \
  "s_add_u32 s49, s49, 1\n" \
  "v_mfma_i32_16x16x64_i8 v[188:191], v[124:127], a[52:55], v[188:191]\n" \
  "buffer_load_dwordx4 a[152:155], v91, s[28:31], s92 offen offset:2048 sc0 sc1\n" \
  "v_mfma_i32_16x16x64_i8 v[220:223], v[120:123], a[40:43], v[220:223]\n" \
  "ds_read_b64_tr_b8 v[116:117], v34 offset:40960\n" \
  "v_mfma_i32_16x16x64_i8 v[220:223], v[124:127], a[56:59], v[220:223]\n" \
  "ds_read_b64_tr_b8 v[118:119], v34 offset:41984\n" \
  "buffer_load_dwordx4 a[156:159], v91, s[28:31], s92 offen offset:3072 sc0 sc1\n" \
  "v_mfma_i32_16x16x64_i8 v[252:255], v[120:123], a[44:47], v[252:255]\n" \
  "v_mfma_i32_16x16x64_i8 v[252:255], v[124:127], a[60:63], v[252:255]\n" \
  "; ---- step 2\n" \
  "s_cmp_lg_u32 s46, s54\n" \
  "s_cbranch_scc1 Lskip23_%=\n" \
  "s_waitcnt vmcnt(63)\n" \
  "global_store_dwordx4 v49, v[64:67], s[34:35] sc0 sc1\n" \
  "s_nop 1\n" \
  "global_store_dwordx4 v79, v[68:71], s[34:35] sc0 sc1\n" \
  "s_nop 1\n" \
  "Lskip23_%=:\n" \
  "s_waitcnt vmcnt(24)\n" \
  "s_waitcnt lgkmcnt(8)\n" \
  "v_mfma_i32_16x16x64_i8 v[128:131], v[96:99], a[64:67], v[128:131]\n" \
  "ds_read_b64_tr_b8 v[120:121], v35 offset:32768\n" \
  "s_add_u32 m0, s47, 114688\n" \
  "v_mfma_i32_16x16x64_i8 v[128:131], v[100:103], a[80:83], v[128:131]\n" \
  "ds_read_b64_tr_b8 v[122:123], v35 offset:33792\n" \
  "buffer_load_dwordx4 v40, s[16:19], s87 offen nt lds\n" \
  "s_add_u32 m0, s47, 115712\n" \
  "v_mfma_i32_16x16x64_i8 v[160:163], v[96:99], a[68:71], v[160:163]\n" \
  "buffer_load_dwordx4 v41, s[16:19], s87 offen nt lds\n" \
  "s_add_u32 m0, s47, 116736\n" \
  "v_mfma_i32_16x16x64_i8 v[160:163], v[100:103], a[84:87], v[160:163]\n" \
  "buffer_load_dwordx4 v42, s[16:19], s87 offen nt lds\n" \
  "s_add_u32 m0, s47, 117760\n" \
  "v_mfma_i32_16x16x64_i8 v[192:195], v[96:99], a[72:75], v[192:195]\n" \
  "ds_read_b64_tr_b8 v[124:125], v35 offset:40960\n" \
  "buffer_load_dwordx4 v43, s[16:19], s87 offen nt lds\n" \
  "s_add_u32 m0, s47, 0\n" \
  "v_mfma_i32_16x16x64_i8 v[192:195], v[100:103], a[88:91], v[192:195]\n" \
  "ds_read_b64_tr_b8 v[126:127], v35 offset:41984\n" \
  "buffer_load_dwordx4 v40, s[16:19], s80 offen nt lds\n" \
  "s_add_u32 m0, s47, 1024\n" \
  "v_mfma_i32_16x16x64_i8 v[224:227], v[96:99], a[76:79], v[224:227]\n" \
  "buffer_load_dwordx4 v41, s[16:19], s80 offen nt lds\n" \
  "v_mfma_i32_16x16x64_i8 v[224:227], v[100:103], a[92:95], v[224:227]\n" \
  "s_waitcnt lgkmcnt(8)\n" \
  "v_mfma_i32_16x16x64_i8 v[132:135], v[104:107], a[64:67], v[132:135]\n" \
  "ds_read_b64_tr_b8 v[96:97], v36 offset:32768\n" \
  "v_mfma_i32_16x16x64_i8 v[132:135], v[108:111], a[80:83], v[132:135]\n" \
  "ds_read_b64_tr_b8 v[98:99], v36 offset:33792\n" \
  "v_mfma_i32_16x16x64_i8 v[164:167], v[104:107], a[68:71], v[164:167]\n" \
  "v_mfma_i32_16x16x64_i8 v[164:167], v[108:111], a[84:87], v[164:167]\n" \
  "v_mfma_i32_16x16x64_i8 v[196:199], v[104:107], a[72:75], v[196:199]\n" \
  "ds_read_b64_tr_b8 v[100:101], v36 offset:40960\n" \
  "v_mfma_i32_16x16x64_i8 v[196:199], v[108:111], a[88:91], v[196:199]\n" \
  "ds_read_b64_tr_b8 v[102:103], v36 offset:41984\n" \
  "v_mfma_i32_16x16x64_i8 v[228:231], v[104:107], a[76:79], v[228:231]\n" \
  "v_mfma_i32_16x16x64_i8 v[228:231], v[108:111], a[92:95], v[228:231]\n" \
  "s_waitcnt lgkmcnt(8)\n" \
  "v_mfma_i32_16x16x64_i8 v[136:139], v[112:115], a[64:67], v[136:139]\n" \
  "ds_read_b64_tr_b8 v[104:105], v37 offset:32768\n" \
  "v_mfma_i32_16x16x64_i8 v[136:139], v[116:119], a[80:83], v[136:139]\n" \
  "ds_read_b64_tr_b8 v[106:107], v37 offset:33792\n" \
  "v_mfma_i32_16x16x64_i8 v[168:171], v[112:115], a[68:71], v[168:171]\n" \
  "v_mfma_i32_16x16x64_i8 v[168:171], v[116:119], a[84:87], v[168:171]\n" \
  "v_mfma_i32_16x16x64_i8 v[200:203], v[112:115], a[72:75], v[200:203]\n" \
  "ds_read_b64_tr_b8 v[108:109], v37 offset:40960\n" \
  "v_mfma_i32_16x16x64_i8 v[200:203], v[116:119], a[88:91], v[200:203]\n" \
  "ds_read_b64_tr_b8 v[110:111], v37 offset:41984\n" \
  "v_mfma_i32_16x16x64_i8 v[232:235], v[112:115], a[76:79], v[232:235]\n" \
  "v_mfma_i32_16x16x64_i8 v[232:235], v[116:119], a[92:95], v[232:235]\n" \
  "s_waitcnt lgkmcnt(8)\n" \
  "v_mfma_i32_16x16x64_i8 v[140:143], v[120:123], a[64:67], v[140:143]\n" \
  "ds_read_b64_tr_b8 v[112:113], v38 offset:32768\n" \
  "v_mfma_i32_16x16x64_i8 v[140:143], v[124:127], a[80:83], v[140:143]\n" \
  "ds_read_b64_tr_b8 v[114:115], v38 offset:33792\n" \
  "v_mfma_i32_16x16x64_i8 v[172:175], v[120:123], a[68:71], v[172:175]\n" \
  "v_mfma_i32_16x16x64_i8 v[172:175], v[124:127], a[84:87], v[172:175]\n" \
  "s_waitcnt vmcnt(50)\n" \
  "s_barrier\n" \
  "v_mfma_i32_16x16x64_i8 v[204:207], v[120:123], a[72:75], v[204:207]\n" \
  "ds_read_b64_tr_b8 v[116:117], v38 offset:40960\n" \
  "v_mfma_i32_16x16x64_i8 v[204:207], v[124:127], a[88:91], v[204:207]\n" \
  "ds_read_b64_tr_b8 v[118:119], v38 offset:41984\n" \
  "v_mfma_i32_16x16x64_i8 v[236:239], v[120:123], a[76:79], v[236:239]\n" \
  "v_mfma_i32_16x16x64_i8 v[236:239], v[124:127], a[92:95], v[236:239]\n" \
  "s_waitcnt lgkmcnt(8)\n" \
  "v_mfma_i32_16x16x64_i8 v[144:147], v[96:99], a[64:67], v[144:147]\n" \
  "ds_read_b64_tr_b8 v[120:121], v39 offset:32768\n" \
  "v_mfma_i32_16x16x64_i8 v[144:147], v[100:103], a[80:83], v[144:147]\n" \
  "ds_read_b64_tr_b8 v[122:123], v39 offset:33792\n" \
  "v_mfma_i32_16x16x64_i8 v[176:179], v[96:99], a[68:71], v[176:179]\n" \
  "v_mfma_i32_16x16x64_i8 v[176:179], v[100:103], a[84:87], v[176:179]\n" \
  "v_mfma_i32_16x16x64_i8 v[208:211], v[96:99], a[72:75], v[208:211]\n" \
  "ds_read_b64_tr_b8 v[124:125], v39 offset:40960\n" \
  "v_mfma_i32_16x16x64_i8 v[208:211], v[100:103], a[88:91], v[208:211]\n" \
  "ds_read_b64_tr_b8 v[126:127], v39 offset:41984\n" \
  "v_mfma_i32_16x16x64_i8 v[240:243], v[96:99], a[76:79], v[240:243]\n" \
  "v_mfma_i32_16x16x64_i8 v[240:243], v[100:103], a[92:95], v[240:243]\n" \
  "s_waitcnt lgkmcnt(8)\n" \
  "v_mfma_i32_16x16x64_i8 v[148:151], v[104:107], a[64:67], v[148:151]\n" \
  "ds_read_b64_tr_b8 v[96:97], v32 offset:49152\n" \
  "s_add_u32 m0, s47, 2048\n" \
  "v_mfma_i32_16x16x64_i8 v[148:151], v[108:111], a[80:83], v[148:151]\n" \
  "ds_read_b64_tr_b8 v[98:99], v32 offset:50176\n" \
  "buffer_load_dwordx4 v42, s[16:19], s80 offen nt lds\n" \
  "s_add_u32 m0, s47, 3072\n" \
  "v_mfma_i32_16x16x64_i8 v[180:183], v[104:107], a[68:71], v[180:183]\n" \
  "buffer_load_dwordx4 v43, s[16:19], s80 offen nt lds\n" \
  "v_mfma_i32_16x16x64_i8 v[180:183], v[108:111], a[84:87], v[180:183]\n" \
  "v_mfma_i32_16x16x64_i8 v[212:215], v[104:107], a[72:75], v[212:215]\n" \
  "ds_read_b64_tr_b8 v[100:101], v32 offset:57344\n" \
  "v_mfma_i32_16x16x64_i8 v[212:215], v[108:111], a[88:91], v[212:215]\n" \
  "ds_read_b64_tr_b8 v[102:103], v32 offset:58368\n" \
  "v_mfma_i32_16x16x64_i8 v[244:247], v[104:107], a[76:79], v[244:247]\n" \
  "v_mfma_i32_16x16x64_i8 v[244:247], v[108:111], a[92:95], v[244:247]\n" \
  "s_waitcnt lgkmcnt(8)\n" \
  "v_mfma_i32_16x16x64_i8 v[152:155], v[112:115], a[64:67], v[152:155]\n" \
  "ds_read_b64_tr_b8 v[104:105], v33 offset:49152\n" \
  "v_mfma_i32_16x16x64_i8 v[152:155], v[116:119], a[80:83], v[152:155]\n" \
  "ds_read_b64_tr_b8 v[106:107], v33 offset:50176\n" \
  "v_mfma_i32_16x16x64_i8 v[184:187], v[112:115], a[68:71], v[184:187]\n" \
  "v_mfma_i32_16x16x64_i8 v[184:187], v[116:119], a[84:87], v[184:187]\n" \
  "v_mfma_i32_16x16x64_i8 v[216:219], v[112:115], a[72:75], v[216:219]\n" \
  "ds_read_b64_tr_b8 v[108:109], v33 offset:57344\n" \
  "v_mfma_i32_16x16x64_i8 v[216:219], v[116:119], a[88:91], v[216:219]\n" \
  "ds_read_b64_tr_b8 v[110:111], v33 offset:58368\n" \
  "v_mfma_i32_16x16x64_i8 v[248:251], v[112:115], a[76:79], v[248:251]\n" \
  "v_mfma_i32_16x16x64_i8 v[248:251], v[116:119], a[92:95], v[248:251]\n" \
  "s_waitcnt lgkmcnt(8)\n" \
  "v_mfma_i32_16x16x64_i8 v[156:159], v[120:123], a[64:67], v[156:159]\n" \
  "ds_read_b64_tr_b8 v[112:113], v34 offset:49152\n" \
  "v_mfma_i32_16x16x64_i8 v[156:159], v[124:127], a[80:83], v[156:159]\n" \
  "ds_read_b64_tr_b8 v[114:115], v34 offset:50176\n" \
  "v_mfma_i32_16x16x64_i8 v[188:191], v[120:123], a[68:71], v[188:191]\n" \
  "v_mfma_i32_16x16x64_i8 v[188:191], v[124:127], a[84:87], v[188:191]\n" \
  "v_mfma_i32_16x16x64_i8 v[220:223], v[120:123], a[72:75], v[220:223]\n" \
  "ds_read_b64_tr_b8 v[116:117], v34 offset:57344\n" \
  "v_mfma_i32_16x16x64_i8 v[220:223], v[124:127], a[88:91], v[220:223]\n" \
  "ds_read_b64_tr_b8 v[118:119], v34 offset:58368\n" \
  "v_mfma_i32_16x16x64_i8 v[252:255], v[120:123], a[76:79], v[252:255]\n" \
  "v_mfma_i32_16x16x64_i8 v[252:255], v[124:127], a[92:95], v[252:255]\n" \
  "; ---- step 3\n" \
  "s_waitcnt vmcnt(16)\n" \
  "s_waitcnt lgkmcnt(8)\n" \
  "v_mfma_i32_16x16x64_i8 v[128:131], v[96:99], a[96:99], v[128:131]\n" \
  "ds_read_b64_tr_b8 v[120:121], v35 offset:49152\n" \
  "v_mfma_i32_16x16x64_i8 v[128:131], v[100:103], a[112:115], v[128:131]\n" \
  "ds_read_b64_tr_b8 v[122:123], v35 offset:50176\n" \
  "v_mfma_i32_16x16x64_i8 v[160:163], v[96:99], a[100:103], v[160:163]\n" \
  "v_mfma_i32_16x16x64_i8 v[160:163], v[100:103], a[116:119], v[160:163]\n" \
  "v_mfma_i32_16x16x64_i8 v[192:195], v[96:99], a[104:107], v[192:195]\n" \
  "ds_read_b64_tr_b8 v[124:125], v35 offset:57344\n" \
  "v_mfma_i32_16x16x64_i8 v[192:195], v[100:103], a[120:123], v[192:195]\n" \
  "ds_read_b64_tr_b8 v[126:127], v35 offset:58368\n" \
  "v_mfma_i32_16x16x64_i8 v[224:227], v[96:99], a[108:111], v[224:227]\n" \
  "v_mfma_i32_16x16x64_i8 v[224:227], v[100:103], a[124:127], v[224:227]\n" \
  "s_waitcnt lgkmcnt(8)\n" \
  "v_mfma_i32_16x16x64_i8 v[132:135], v[104:107], a[96:99], v[132:135]\n" \
  "ds_read_b64_tr_b8 v[96:97], v36 offset:49152\n" \
  "v_mfma_i32_16x16x64_i8 v[132:135], v[108:111], a[112:115], v[132:135]\n" \
  "ds_read_b64_tr_b8 v[98:99], v36 offset:50176\n" \
  "v_mfma_i32_16x16x64_i8 v[164:167], v[104:107], a[100:103], v[164:167]\n" \
  "v_mfma_i32_16x16x64_i8 v[164:167], v[108:111], a[116:119], v[164:167]\n" \
  "v_mfma_i32_16x16x64_i8 v[196:199], v[104:107], a[104:107], v[196:199]\n" \
  "ds_read_b64_tr_b8 v[100:101], v36 offset:57344\n" \
  "v_mfma_i32_16x16x64_i8 v[196:199], v[108:111], a[120:123], v[196:199]\n" \
  "ds_read_b64_tr_b8 v[102:103], v36 offset:58368\n" \
  "v_mfma_i32_16x16x64_i8 v[228:231], v[104:107], a[108:111], v[228:231]\n" \
  "v_mfma_i32_16x16x64_i8 v[228:231], v[108:111], a[124:127], v[228:231]\n" \
  "s_waitcnt lgkmcnt(8)\n" \
  "v_mfma_i32_16x16x64_i8 v[136:139], v[112:115], a[96:99], v[136:139]\n" \
  "ds_read_b64_tr_b8 v[104:105], v37 offset:49152\n" \
  "v_mfma_i32_16x16x64_i8 v[136:139], v[116:119], a[112:115], v[136:139]\n" \
  "ds_read_b64_tr_b8 v[106:107], v37 offset:50176\n" \
  "v_mfma_i32_16x16x64_i8 v[168:171], v[112:115], a[100:103], v[168:171]\n" \
  "v_mfma_i32_16x16x64_i8 v[168:171], v[116:119], a[116:119], v[168:171]\n" \
  "v_mfma_i32_16x16x64_i8 v[200:203], v[112:115], a[104:107], v[200:203]\n" \
  "ds_read_b64_tr_b8 v[108:109], v37 offset:57344\n" \
  "v_mfma_i32_16x16x64_i8 v[200:203], v[116:119], a[120:123], v[200:203]\n" \
  "ds_read_b64_tr_b8 v[110:111], v37 offset:58368\n" \
  "buffer_load_dwordx4 a[160:163], v44, s[28:31], s93 offen offset:0 sc0 sc1\n" \
  "v_mfma_i32_16x16x64_i8 v[232:235], v[112:115], a[108:111], v[232:235]\n" \
  "v_mfma_i32_16x16x64_i8 v[232:235], v[116:119], a[124:127], v[232:235]\n" \
  "buffer_load_dwordx4 a[164:167], v44, s[28:31], s93 offen offset:1024 sc0 sc1\n" \
  "s_waitcnt lgkmcnt(8)\n" \
  "v_mfma_i32_16x16x64_i8 v[140:143], v[120:123], a[96:99], v[140:143]\n" \
  "ds_read_b64_tr_b8 v[112:113], v38 offset:49152\n" \
  "v_mfma_i32_16x16x64_i8 v[140:143], v[124:127], a[112:115], v[140:143]\n" \
  "ds_read_b64_tr_b8 v[114:115], v38 offset:50176\n" \
  "buffer_load_dwordx4 a[168:171], v44, s[28:31], s93 offen offset:2048 sc0 sc1\n" \
  "v_mfma_i32_16x16x64_i8 v[172:175], v[120:123], a[100:103], v[172:175]\n" \
  "v_mfma_i32_16x16x64_i8 v[172:175], v[124:127], a[116:119], v[172:175]\n" \
  "s_cmp_lg_u32 s46, s54\n" \
  "s_cbranch_scc1 Lskip24_%=\n" \
  "s_waitcnt vmcnt(11)\n" \
  "Lskip24_%=:\n" \
  "s_waitcnt vmcnt(51)\n" \
  "s_barrier\n" \
  "s_cmp_lg_u32 s46, s54\n" \
  "s_cbranch_scc1 Lskip26_%=\n" \
  "s_cmp_lg_u32 s11, 0\n" \
  "s_cbranch_scc1 Lskip25_%=\n" \
  "s_mov_b64 exec, 1\n" \
  "global_store_dword v53, v76, s[36:37] sc0 sc1\n" \
  "s_mov_b64 exec, -1\n" \
  "Lskip25_%=:\n" \
  "Lskip26_%=:\n" \
  "v_mfma_i32_16x16x64_i8 v[204:207], v[120:123], a[104:107], v[204:207]\n" \
  "ds_read_b64_tr_b8 v[116:117], v38 offset:57344\n" \
  "v_mfma_i32_16x16x64_i8 v[204:207], v[124:127], a[120:123], v[204:207]\n" \
  "ds_read_b64_tr_b8 v[118:119], v38 offset:58368\n" \
  "buffer_load_dwordx4 a[172:175], v44, s[28:31], s93 offen offset:3072 sc0 sc1\n" \
  "v_mfma_i32_16x16x64_i8 v[236:239], v[120:123], a[108:111], v[236:239]\n" \
  "v_mfma_i32_16x16x64_i8 v[236:239], v[124:127], a[124:127], v[236:239]\n" \
  "buffer_load_dwordx4 a[176:179], v91, s[28:31], s93 offen offset:0 sc0 sc1\n" \
  "s_waitcnt lgkmcnt(8)\n" \
  "v_mfma_i32_16x16x64_i8 v[144:147], v[96:99], a[96:99], v[144:147]\n" \
  "ds_read_b64_tr_b8 v[120:121], v39 offset:49152\n" \
  "v_mfma_i32_16x16x64_i8 v[144:147], v[100:103], a[112:115], v[144:147]\n" \
  "ds_read_b64_tr_b8 v[122:123], v39 offset:50176\n" \
  "buffer_load_dwordx4 a[180:183], v91, s[28:31], s93 offen offset:1024 sc0 sc1\n" \
  "v_mfma_i32_16x16x64_i8 v[176:179], v[96:99], a[100:103], v[176:179]\n" \
  "v_mfma_i32_16x16x64_i8 v[176:179], v[100:103], a[116:119], v[176:179]\n" \
  "v_mfma_i32_16x16x64_i8 v[208:211], v[96:99], a[104:107], v[208:211]\n" \
  "ds_read_b64_tr_b8 v[124:125], v39 offset:57344\n" \
  "v_mfma_i32_16x16x64_i8 v[208:211], v[100:103], a[120:123], v[208:211]\n" \
  "ds_read_b64_tr_b8 v[126:127], v39 offset:58368\n" \
  "buffer_load_dwordx4 a[184:187], v91, s[28:31], s93 offen offset:2048 sc0 sc1\n" \
  "v_mfma_i32_16x16x64_i8 v[240:243], v[96:99], a[108:111], v[240:243]\n" \
  "v_readlane_b32 s50, v46, s49\n" \
  "v_mfma_i32_16x16x64_i8 v[240:243], v[100:103], a[124:127], v[240:243]\n" \
  "buffer_load_dwordx4 a[188:191], v91, s[28:31], s93 offen offset:3072 sc0 sc1\n" \
  "s_waitcnt lgkmcnt(8)\n" \
  "v_mfma_i32_16x16x64_i8 v[148:151], v[104:107], a[96:99], v[148:151]\n" \
  "ds_read_b64_tr_b8 v[96:97], v80 offset:0\n" \
  "v_mfma_i32_16x16x64_i8 v[148:151], v[108:111], a[112:115], v[148:151]\n" \
  "ds_read_b64_tr_b8 v[98:99], v80 offset:1024\n" \
  "v_mfma_i32_16x16x64_i8 v[180:183], v[104:107], a[100:103], v[180:183]\n" \
  "v_mfma_i32_16x16x64_i8 v[180:183], v[108:111], a[116:119], v[180:183]\n" \
  "buffer_load_dwordx4 a[192:195], v44, s[28:31], s94 offen offset:0 sc0 sc1\n" \
  "v_mfma_i32_16x16x64_i8 v[212:215], v[104:107], a[104:107], v[212:215]\n" \
  "ds_read_b64_tr_b8 v[100:101], v80 offset:8192\n" \
  "v_mfma_i32_16x16x64_i8 v[212:215], v[108:111], a[120:123], v[212:215]\n" \
  "ds_read_b64_tr_b8 v[102:103], v80 offset:9216\n" \
  "buffer_load_dwordx4 a[196:199], v44, s[28:31], s94 offen offset:1024 sc0 sc1\n" \
  "v_mfma_i32_16x16x64_i8 v[244:247], v[104:107], a[108:111], v[244:247]\n" \
  "s_lshl_b32 s81, s50, 21\n" \
  "s_lshl_b32 s89, s50, 16\n" \
  "s_add_u32 s49, s49, 1\n" \
  "v_mfma_i32_16x16x64_i8 v[244:247], v[108:111], a[124:127], v[244:247]\n" \
  "buffer_load_dwordx4 a[200:203], v44, s[28:31], s94 offen offset:2048 sc0 sc1\n" \
  "s_waitcnt lgkmcnt(8)\n" \
  "v_mfma_i32_16x16x64_i8 v[152:155], v[112:115], a[96:99], v[152:155]\n" \
  "ds_read_b64_tr_b8 v[104:105], v81 offset:0\n" \
  "v_mfma_i32_16x16x64_i8 v[152:155], v[116:119], a[112:115], v[152:155]\n" \
  "ds_read_b64_tr_b8 v[106:107], v81 offset:1024\n" \
  "v_mfma_i32_16x16x64_i8 v[184:187], v[112:115], a[100:103], v[184:187]\n" \
  "v_readlane_b32 s50, v46, s49\n" \
  "v_mfma_i32_16x16x64_i8 v[184:187], v[116:119], a[116:119], v[184:187]\n" \
  "buffer_load_dwordx4 a[204:207], v44, s[28:31], s94 offen offset:3072 sc0 sc1\n" \
  "v_mfma_i32_16x16x64_i8 v[216:219], v[112:115], a[104:107], v[216:219]\n" \
  "ds_read_b64_tr_b8 v[108:109], v81 offset:8192\n" \
  "v_mfma_i32_16x16x64_i8 v[216:219], v[116:119], a[120:123], v[216:219]\n" \
  "ds_read_b64_tr_b8 v[110:111], v81 offset:9216\n" \
  "buffer_load_dwordx4 a[208:211], v91, s[28:31], s94 offen offset:0 sc0 sc1\n" \
  "v_mfma_i32_16x16x64_i8 v[248:251], v[112:115], a[108:111], v[248:251]\n" \
  "v_mfma_i32_16x16x64_i8 v[248:251], v[116:119], a[124:127], v[248:251]\n" \
  "buffer_load_dwordx4 a[212:215], v91, s[28:31], s94 offen offset:1024 sc0 sc1\n" \
  "s_waitcnt lgkmcnt(8)\n" \
  "v_mfma_i32_16x16x64_i8 v[156:159], v[120:123], a[96:99], v[156:159]\n" \
  "ds_read_b64_tr_b8 v[112:113], v82 offset:0\n" \
  "v_mfma_i32_16x16x64_i8 v[156:159], v[124:127], a[112:115], v[156:159]\n" \
  "ds_read_b64_tr_b8 v[114:115], v82 offset:1024\n" \
  "v_mfma_i32_16x16x64_i8 v[188:191], v[120:123], a[100:103], v[188:191]\n" \
  "s_lshl_b32 s82, s50, 21\n" \
  "s_lshl_b32 s90, s50, 16\n" \
  "s_add_u32 s49, s49, 1\n" \
  "v_mfma_i32_16x16x64_i8 v[188:191], v[124:127], a[116:119], v[188:191]\n" \
  "buffer_load_dwordx4 a[216:219], v91, s[28:31], s94 offen offset:2048 sc0 sc1\n" \
  "v_mfma_i32_16x16x64_i8 v[220:223], v[120:123], a[104:107], v[220:223]\n" \
  "ds_read_b64_tr_b8 v[116:117], v82 offset:8192\n" \
  "v_mfma_i32_16x16x64_i8 v[220:223], v[124:127], a[120:123], v[220:223]\n" \
  "ds_read_b64_tr_b8 v[118:119], v82 offset:9216\n" \
  "buffer_load_dwordx4 a[220:223], v91, s[28:31], s94 offen offset:3072 sc0 sc1\n" \
  "v_mfma_i32_16x16x64_i8 v[252:255], v[120:123], a[108:111], v[252:255]\n" \
  "v_mfma_i32_16x16x64_i8 v[252:255], v[124:127], a[124:127], v[252:255]\n" \
  "; ---- step 4\n" \
  "s_cmp_lg_u32 s11, 0\n" \
  "s_cbranch_scc1 Lskip27_%=\n" \
  "global_load_dword v72, v50, s[38:39] sc0 sc1\n" \
  "Lskip27_%=:\n" \
  "s_waitcnt vmcnt(24)\n" \
  "s_waitcnt lgkmcnt(8)\n" \
  "v_mfma_i32_16x16x64_i8 v[128:131], v[96:99], a[128:131], v[128:131]\n" \
  "ds_read_b64_tr_b8 v[120:121], v83 offset:0\n" \
  "s_add_u32 m0, s47, 16384\n" \
  "v_mfma_i32_16x16x64_i8 v[128:131], v[100:103], a[144:147], v[128:131]\n" \
  "ds_read_b64_tr_b8 v[122:123], v83 offset:1024\n" \
  "buffer_load_dwordx4 v40, s[16:19], s81 offen nt lds\n" \
  "s_add_u32 m0, s47, 17408\n" \
  "v_mfma_i32_16x16x64_i8 v[160:163], v[96:99], a[132:135], v[160:163]\n" \
  "buffer_load_dwordx4 v41, s[16:19], s81 offen nt lds\n" \
  "s_add_u32 m0, s47, 18432\n" \
  "v_mfma_i32_16x16x64_i8 v[160:163], v[100:103], a[148:151], v[160:163]\n" \
  "buffer_load_dwordx4 v42, s[16:19], s81 offen nt lds\n" \
  "s_add_u32 m0, s47, 19456\n" \
  "v_mfma_i32_16x16x64_i8 v[192:195], v[96:99], a[136:139], v[192:195]\n" \
  "ds_read_b64_tr_b8 v[124:125], v83 offset:8192\n" \
  "buffer_load_dwordx4 v43, s[16:19], s81 offen nt lds\n" \
  "s_add_u32 m0, s47, 32768\n" \
  "v_mfma_i32_16x16x64_i8 v[192:195], v[100:103], a[152:155], v[192:195]\n" \
  "ds_read_b64_tr_b8 v[126:127], v83 offset:9216\n" \
  "buffer_load_dwordx4 v40, s[16:19], s82 offen nt lds\n" \
  "s_add_u32 m0, s47, 33792\n" \
  "v_mfma_i32_16x16x64_i8 v[224:227], v[96:99], a[140:143], v[224:227]\n" \
  "buffer_load_dwordx4 v41, s[16:19], s82 offen nt lds\n" \
  "s_add_u32 m0, s47, 34816\n" \
  "v_mfma_i32_16x16x64_i8 v[224:227], v[100:103], a[156:159], v[224:227]\n" \
  "buffer_load_dwordx4 v42, s[16:19], s82 offen nt lds\n" \
  "s_add_u32 m0, s47, 35840\n" \
  "s_waitcnt lgkmcnt(8)\n" \
  "v_mfma_i32_16x16x64_i8 v[132:135], v[104:107], a[128:131], v[132:135]\n" \
  "ds_read_b64_tr_b8 v[96:97], v84 offset:0\n" \
  "buffer_load_dwordx4 v43, s[16:19], s82 offen nt lds\n" \
  "v_mfma_i32_16x16x64_i8 v[132:135], v[108:111], a[144:147], v[132:135]\n" \
  "ds_read_b64_tr_b8 v[98:99], v84 offset:1024\n" \
  "v_mfma_i32_16x16x64_i8 v[164:167], v[104:107], a[132:135], v[164:167]\n" \
  "v_mfma_i32_16x16x64_i8 v[164:167], v[108:111], a[148:151], v[164:167]\n" \
  "v_mfma_i32_16x16x64_i8 v[196:199], v[104:107], a[136:139], v[196:199]\n" \
  "ds_read_b64_tr_b8 v[100:101], v84 offset:8192\n" \
  "v_mfma_i32_16x16x64_i8 v[196:199], v[108:111], a[152:155], v[196:199]\n" \
  "ds_read_b64_tr_b8 v[102:103], v84 offset:9216\n" \
  "v_mfma_i32_16x16x64_i8 v[228:231], v[104:107], a[140:143], v[228:231]\n" \
  "v_mfma_i32_16x16x64_i8 v[228:231], v[108:111], a[156:159], v[228:231]\n" \
  "s_waitcnt lgkmcnt(8)\n" \
  "v_mfma_i32_16x16x64_i8 v[136:139], v[112:115], a[128:131], v[136:139]\n" \
  "ds_read_b64_tr_b8 v[104:105], v85 offset:0\n" \
  "v_mfma_i32_16x16x64_i8 v[136:139], v[116:119], a[144:147], v[136:139]\n" \
  "ds_read_b64_tr_b8 v[106:107], v85 offset:1024\n" \
  "v_mfma_i32_16x16x64_i8 v[168:171], v[112:115], a[132:135], v[168:171]\n" \
  "v_mfma_i32_16x16x64_i8 v[168:171], v[116:119], a[148:151], v[168:171]\n" \
  "v_mfma_i32_16x16x64_i8 v[200:203], v[112:115], a[136:139], v[200:203]\n" \
  "ds_read_b64_tr_b8 v[108:109], v85 offset:8192\n" \
  "v_mfma_i32_16x16x64_i8 v[200:203], v[116:119], a[152:155], v[200:203]\n" \
  "ds_read_b64_tr_b8 v[110:111], v85 offset:9216\n" \
  "v_mfma_i32_16x16x64_i8 v[232:235], v[112:115], a[140:143], v[232:235]\n" \
  "v_mfma_i32_16x16x64_i8 v[232:235], v[116:119], a[156:159], v[232:235]\n" \
  "s_waitcnt lgkmcnt(8)\n" \
  "v_mfma_i32_16x16x64_i8 v[140:143], v[120:123], a[128:131], v[140:143]\n" \
  "ds_read_b64_tr_b8 v[112:113], v86 offset:0\n" \
  "v_mfma_i32_16x16x64_i8 v[140:143], v[124:127], a[144:147], v[140:143]\n" \
  "ds_read_b64_tr_b8 v[114:115], v86 offset:1024\n" \
  "v_mfma_i32_16x16x64_i8 v[172:175], v[120:123], a[132:135], v[172:175]\n" \
  "v_mfma_i32_16x16x64_i8 v[172:175], v[124:127], a[148:151], v[172:175]\n" \
  "s_waitcnt vmcnt(52)\n" \
  "s_barrier\n" \
  "v_mfma_i32_16x16x64_i8 v[204:207], v[120:123], a[136:139], v[204:207]\n" \
  "ds_read_b64_tr_b8 v[116:117], v86 offset:8192\n" \
  "v_mfma_i32_16x16x64_i8 v[204:207], v[124:127], a[152:155], v[204:207]\n" \
  "ds_read_b64_tr_b8 v[118:119], v86 offset:9216\n" \
  "v_mfma_i32_16x16x64_i8 v[236:239], v[120:123], a[140:143], v[236:239]\n" \
  "v_mfma_i32_16x16x64_i8 v[236:239], v[124:127], a[156:159], v[236:239]\n" \
  "s_waitcnt lgkmcnt(8)\n" \
  "v_mfma_i32_16x16x64_i8 v[144:147], v[96:99], a[128:131], v[144:147]\n" \
  "ds_read_b64_tr_b8 v[120:121], v87 offset:0\n" \
  "v_mfma_i32_16x16x64_i8 v[144:147], v[100:103], a[144:147], v[144:147]\n" \
  "ds_read_b64_tr_b8 v[122:123], v87 offset:1024\n" \
  "v_mfma_i32_16x16x64_i8 v[176:179], v[96:99], a[132:135], v[176:179]\n" \
  "v_mfma_i32_16x16x64_i8 v[176:179], v[100:103], a[148:151], v[176:179]\n" \
  "v_mfma_i32_16x16x64_i8 v[208:211], v[96:99], a[136:139], v[208:211]\n" \
  "ds_read_b64_tr_b8 v[124:125], v87 offset:8192\n" \
  "v_mfma_i32_16x16x64_i8 v[208:211], v[100:103], a[152:155], v[208:211]\n" \
  "ds_read_b64_tr_b8 v[126:127], v87 offset:9216\n" \
  "v_mfma_i32_16x16x64_i8 v[240:243], v[96:99], a[140:143], v[240:243]\n" \
  "v_mfma_i32_16x16x64_i8 v[240:243], v[100:103], a[156:159], v[240:243]\n" \
  "s_waitcnt lgkmcnt(8)\n" \
  "v_mfma_i32_16x16x64_i8 v[148:151], v[104:107], a[128:131], v[148:151]\n" \
  "ds_read_b64_tr_b8 v[96:97], v80 offset:16384\n" \
  "v_mfma_i32_16x16x64_i8 v[148:151], v[108:111], a[144:147], v[148:151]\n" \
  "ds_read_b64_tr_b8 v[98:99], v80 offset:17408\n" \
  "v_mfma_i32_16x16x64_i8 v[180:183], v[104:107], a[132:135], v[180:183]\n" \
  "v_mfma_i32_16x16x64_i8 v[180:183], v[108:111], a[148:151], v[180:183]\n" \
  "v_mfma_i32_16x16x64_i8 v[212:215], v[104:107], a[136:139], v[212:215]\n" \
  "ds_read_b64_tr_b8 v[100:101], v80 offset:24576\n" \
  "v_mfma_i32_16x16x64_i8 v[212:215], v[108:111], a[152:155], v[212:215]\n" \
  "ds_read_b64_tr_b8 v[102:103], v80 offset:25600\n" \
  "v_mfma_i32_16x16x64_i8 v[244:247], v[104:107], a[140:143], v[244:247]\n" \
  "v_mfma_i32_16x16x64_i8 v[244:247], v[108:111], a[156:159], v[244:247]\n" \
  "s_waitcnt lgkmcnt(8)\n" \
  "v_mfma_i32_16x16x64_i8 v[152:155], v[112:115], a[128:131], v[152:155]\n" \
  "ds_read_b64_tr_b8 v[104:105], v81 offset:16384\n" \
  "v_mfma_i32_16x16x64_i8 v[152:155], v[116:119], a[144:147], v[152:155]\n" \
  "ds_read_b64_tr_b8 v[106:107], v81 offset:17408\n" \
  "v_mfma_i32_16x16x64_i8 v[184:187], v[112:115], a[132:135], v[184:187]\n" \
  "v_mfma_i32_16x16x64_i8 v[184:187], v[116:119], a[148:151], v[184:187]\n" \
  "v_mfma_i32_16x16x64_i8 v[216:219], v[112:115], a[136:139], v[216:219]\n" \
  "ds_read_b64_tr_b8 v[108:109], v81 offset:24576\n" \
  "v_mfma_i32_16x16x64_i8 v[216:219], v[116:119], a[152:155], v[216:219]\n" \
  "ds_read_b64_tr_b8 v[110:111], v81 offset:25600\n" \
  "v_mfma_i32_16x16x64_i8 v[248:251], v[112:115], a[140:143], v[248:251]\n" \
  "v_mfma_i32_16x16x64_i8 v[248:251], v[116:119], a[156:159], v[248:251]\n" \
  "s_waitcnt lgkmcnt(8)\n" \
  "v_mfma_i32_16x16x64_i8 v[156:159], v[120:123], a[128:131], v[156:159]\n" \
  "ds_read_b64_tr_b8 v[112:113], v82 offset:16384\n" \
  "v_mfma_i32_16x16x64_i8 v[156:159], v[124:127], a[144:147], v[156:159]\n" \
  "ds_read_b64_tr_b8 v[114:115], v82 offset:17408\n" \
  "v_mfma_i32_16x16x64_i8 v[188:191], v[120:123], a[132:135], v[188:191]\n" \
  "v_mfma_i32_16x16x64_i8 v[188:191], v[124:127], a[148:151], v[188:191]\n" \
  "v_mfma_i32_16x16x64_i8 v[220:223], v[120:123], a[136:139], v[220:223]\n" \
  "ds_read_b64_tr_b8 v[116:117], v82 offset:24576\n" \
  "v_mfma_i32_16x16x64_i8 v[220:223], v[124:127], a[152:155], v[220:223]\n" \
  "ds_read_b64_tr_b8 v[118:119], v82 offset:25600\n" \
  "v_mfma_i32_16x16x64_i8 v[252:255], v[120:123], a[140:143], v[252:255]\n" \
  "v_mfma_i32_16x16x64_i8 v[252:255], v[124:127], a[156:159], v[252:255]\n" \
  "; ---- step 5\n" \
  "s_waitcnt vmcnt(16)\n" \
  "s_waitcnt lgkmcnt(8)\n" \
  "v_mfma_i32_16x16x64_i8 v[128:131], v[96:99], a[160:163], v[128:131]\n" \
  "ds_read_b64_tr_b8 v[120:121], v83 offset:16384\n" \
  "v_mfma_i32_16x16x64_i8 v[128:131], v[100:103], a[176:179], v[128:131]\n" \
  "ds_read_b64_tr_b8 v[122:123], v83 offset:17408\n" \
  "v_mfma_i32_16x16x64_i8 v[160:163], v[96:99], a[164:167], v[160:163]\n" \
  "v_mfma_i32_16x16x64_i8 v[160:163], v[100:103], a[180:183], v[160:163]\n" \
  "v_mfma_i32_16x16x64_i8 v[192:195], v[96:99], a[168:171], v[192:195]\n" \
  "ds_read_b64_tr_b8 v[124:125], v83 offset:24576\n" \
  "v_mfma_i32_16x16x64_i8 v[192:195], v[100:103], a[184:187], v[192:195]\n" \
  "ds_read_b64_tr_b8 v[126:127], v83 offset:25600\n" \
  "v_mfma_i32_16x16x64_i8 v[224:227], v[96:99], a[172:175], v[224:227]\n" \
  "v_mfma_i32_16x16x64_i8 v[224:227], v[100:103], a[188:191], v[224:227]\n" \
  "s_waitcnt lgkmcnt(8)\n" \
  "v_mfma_i32_16x16x64_i8 v[132:135], v[104:107], a[160:163], v[132:135]\n" \
  "ds_read_b64_tr_b8 v[96:97], v84 offset:16384\n" \
  "v_mfma_i32_16x16x64_i8 v[132:135], v[108:111], a[176:179], v[132:135]\n" \
  "ds_read_b64_tr_b8 v[98:99], v84 offset:17408\n" \
  "v_mfma_i32_16x16x64_i8 v[164:167], v[104:107], a[164:167], v[164:167]\n" \
  "v_mfma_i32_16x16x64_i8 v[164:167], v[108:111], a[180:183], v[164:167]\n" \
  "v_mfma_i32_16x16x64_i8 v[196:199], v[104:107], a[168:171], v[196:199]\n" \
  "ds_read_b64_tr_b8 v[100:101], v84 offset:24576\n" \
  "v_mfma_i32_16x16x64_i8 v[196:199], v[108:111], a[184:187], v[196:199]\n" \
  "ds_read_b64_tr_b8 v[102:103], v84 offset:25600\n" \
  "v_mfma_i32_16x16x64_i8 v[228:231], v[104:107], a[172:175], v[228:231]\n" \
  "v_mfma_i32_16x16x64_i8 v[228:231], v[108:111], a[188:191], v[228:231]\n" \
  "s_waitcnt lgkmcnt(8)\n" \
  "v_mfma_i32_16x16x64_i8 v[136:139], v[112:115], a[160:163], v[136:139]\n" \
  "ds_read_b64_tr_b8 v[104:105], v85 offset:16384\n" \
  "v_mfma_i32_16x16x64_i8 v[136:139], v[116:119], a[176:179], v[136:139]\n" \
  "ds_read_b64_tr_b8 v[106:107], v85 offset:17408\n" \
  "v_mfma_i32_16x16x64_i8 v[168:171], v[112:115], a[164:167], v[168:171]\n" \
  "v_mfma_i32_16x16x64_i8 v[168:171], v[116:119], a[180:183], v[168:171]\n" \
  "v_mfma_i32_16x16x64_i8 v[200:203], v[112:115], a[168:171], v[200:203]\n" \
  "ds_read_b64_tr_b8 v[108:109], v85 offset:24576\n" \
  "v_mfma_i32_16x16x64_i8 v[200:203], v[116:119], a[184:187], v[200:203]\n" \
  "ds_read_b64_tr_b8 v[110:111], v85 offset:25600\n" \
  "buffer_load_dwordx4 a[224:227], v44, s[28:31], s95 offen offset:0 sc0 sc1\n" \
  "v_mfma_i32_16x16x64_i8 v[232:235], v[112:115], a[172:175], v[232:235]\n" \
  "v_mfma_i32_16x16x64_i8 v[232:235], v[116:119], a[188:191], v[232:235]\n" \
  "buffer_load_dwordx4 a[228:231], v44, s[28:31], s95 offen offset:1024 sc0 sc1\n" \
  "s_waitcnt lgkmcnt(8)\n" \
  "v_mfma_i32_16x16x64_i8 v[140:143], v[120:123], a[160:163], v[140:143]\n" \
  "ds_read_b64_tr_b8 v[112:113], v86 offset:16384\n" \
  "v_mfma_i32_16x16x64_i8 v[140:143], v[124:127], a[176:179], v[140:143]\n" \
  "ds_read_b64_tr_b8 v[114:115], v86 offset:17408\n" \
  "buffer_load_dwordx4 a[232:235], v44, s[28:31], s95 offen offset:2048 sc0 sc1\n" \
  "v_mfma_i32_16x16x64_i8 v[172:175], v[120:123], a[164:167], v[172:175]\n" \
  "v_mfma_i32_16x16x64_i8 v[172:175], v[124:127], a[180:183], v[172:175]\n" \
  "s_waitcnt vmcnt(51)\n" \
  "s_barrier\n" \
  "v_mfma_i32_16x16x64_i8 v[204:207], v[120:123], a[168:171], v[204:207]\n" \
  "ds_read_b64_tr_b8 v[116:117], v86 offset:24576\n" \
  "v_mfma_i32_16x16x64_i8 v[204:207], v[124:127], a[184:187], v[204:207]\n" \
  "ds_read_b64_tr_b8 v[118:119], v86 offset:25600\n" \
  "buffer_load_dwordx4 a[236:239], v44, s[28:31], s95 offen offset:3072 sc0 sc1\n" \
  "v_mfma_i32_16x16x64_i8 v[236:239], v[120:123], a[172:175], v[236:239]\n" \
  "v_mfma_i32_16x16x64_i8 v[236:239], v[124:127], a[188:191], v[236:239]\n" \
  "buffer_load_dwordx4 a[240:243], v91, s[28:31], s95 offen offset:0 sc0 sc1\n" \
  "s_waitcnt lgkmcnt(8)\n" \
  "v_mfma_i32_16x16x64_i8 v[144:147], v[96:99], a[160:163], v[144:147]\n" \
  "ds_read_b64_tr_b8 v[120:121], v87 offset:16384\n" \
  "v_mfma_i32_16x16x64_i8 v[144:147], v[100:103], a[176:179], v[144:147]\n" \
  "ds_read_b64_tr_b8 v[122:123], v87 offset:17408\n" \
  "buffer_load_dwordx4 a[244:247], v91, s[28:31], s95 offen offset:1024 sc0 sc1\n" \
  "v_mfma_i32_16x16x64_i8 v[176:179], v[96:99], a[164:167], v[176:179]\n" \
  "v_mfma_i32_16x16x64_i8 v[176:179], v[100:103], a[180:183], v[176:179]\n" \
  "v_mfma_i32_16x16x64_i8 v[208:211], v[96:99], a[168:171], v[208:211]\n" \
  "ds_read_b64_tr_b8 v[124:125], v87 offset:24576\n" \
  "v_mfma_i32_16x16x64_i8 v[208:211], v[100:103], a[184:187], v[208:211]\n" \
  "ds_read_b64_tr_b8 v[126:127], v87 offset:25600\n" \
  "buffer_load_dwordx4 a[248:251], v91, s[28:31], s95 offen offset:2048 sc0 sc1\n" \
  "v_mfma_i32_16x16x64_i8 v[240:243], v[96:99], a[172:175], v[240:243]\n" \
  "v_readlane_b32 s50, v46, s49\n" \
  "v_mfma_i32_16x16x64_i8 v[240:243], v[100:103], a[188:191], v[240:243]\n" \
  "buffer_load_dwordx4 a[252:255], v91, s[28:31], s95 offen offset:3072 sc0 sc1\n" \
  "s_waitcnt lgkmcnt(8)\n" \
  "v_mfma_i32_16x16x64_i8 v[148:151], v[104:107], a[160:163], v[148:151]\n" \
  "ds_read_b64_tr_b8 v[96:97], v80 offset:32768\n" \
  "v_mfma_i32_16x16x64_i8 v[148:151], v[108:111], a[176:179], v[148:151]\n" \
  "ds_read_b64_tr_b8 v[98:99], v80 offset:33792\n" \
  "v_mfma_i32_16x16x64_i8 v[180:183], v[104:107], a[164:167], v[180:183]\n" \
  "v_mfma_i32_16x16x64_i8 v[180:183], v[108:111], a[180:183], v[180:183]\n" \
  "buffer_load_dwordx4 a[0:3], v44, s[28:31], s88 offen offset:0 sc0 sc1\n" \
  "v_mfma_i32_16x16x64_i8 v[212:215], v[104:107], a[168:171], v[212:215]\n" \
  "ds_read_b64_tr_b8 v[100:101], v80 offset:40960\n" \
  "v_mfma_i32_16x16x64_i8 v[212:215], v[108:111], a[184:187], v[212:215]\n" \
  "ds_read_b64_tr_b8 v[102:103], v80 offset:41984\n" \
  "buffer_load_dwordx4 a[4:7], v44, s[28:31], s88 offen offset:1024 sc0 sc1\n" \
  "v_mfma_i32_16x16x64_i8 v[244:247], v[104:107], a[172:175], v[244:247]\n" \
  "s_lshl_b32 s83, s50, 21\n" \
  "s_lshl_b32 s91, s50, 16\n" \
  "s_add_u32 s49, s49, 1\n" \
  "v_mfma_i32_16x16x64_i8 v[244:247], v[108:111], a[188:191], v[244:247]\n" \
  "buffer_load_dwordx4 a[8:11], v44, s[28:31], s88 offen offset:2048 sc0 sc1\n" \
  "s_waitcnt lgkmcnt(8)\n" \
  "v_mfma_i32_16x16x64_i8 v[152:155], v[112:115], a[160:163], v[152:155]\n" \
  "ds_read_b64_tr_b8 v[104:105], v81 offset:32768\n" \
  "v_mfma_i32_16x16x64_i8 v[152:155], v[116:119], a[176:179], v[152:155]\n" \
  "ds_read_b64_tr_b8 v[106:107], v81 offset:33792\n" \
  "v_mfma_i32_16x16x64_i8 v[184:187], v[112:115], a[164:167], v[184:187]\n" \
  "v_readlane_b32 s50, v46, s49\n" \
  "v_mfma_i32_16x16x64_i8 v[184:187], v[116:119], a[180:183], v[184:187]\n" \
  "buffer_load_dwordx4 a[12:15], v44, s[28:31], s88 offen offset:3072 sc0 sc1\n" \
  "s_cmp_lg_u32 s11, 0\n" \
  "s_cbranch_scc1 Lskip30_%=\n" \
  "s_waitcnt vmcnt(20)\n" \
  "v_sub_u32 v72, v72, v76\n" \
  "v_cmp_gt_i32 vcc, 0, v72\n" \
  "s_nop 1\n" \
  "s_andn2_b64 vcc, vcc, s[14:15]\n" \
  "s_cbranch_scc1 Lslow28_%=\n" \
  "Lback29_%=:\n" \
  "v_add_u32 v50, 1024, v50\n" \
  "Lskip30_%=:\n" \
  "v_mfma_i32_16x16x64_i8 v[216:219], v[112:115], a[168:171], v[216:219]\n" \
  "ds_read_b64_tr_b8 v[108:109], v81 offset:40960\n" \
  "v_mfma_i32_16x16x64_i8 v[216:219], v[116:119], a[184:187], v[216:219]\n" \
  "ds_read_b64_tr_b8 v[110:111], v81 offset:41984\n" \
  "buffer_load_dwordx4 a[16:19], v91, s[28:31], s88 offen offset:0 sc0 sc1\n" \
  "v_mfma_i32_16x16x64_i8 v[248:251], v[112:115], a[172:175], v[248:251]\n" \
  "v_mfma_i32_16x16x64_i8 v[248:251], v[116:119], a[188:191], v[248:251]\n" \
  "buffer_load_dwordx4 a[20:23], v91, s[28:31], s88 offen offset:1024 sc0 sc1\n" \
  "s_waitcnt lgkmcnt(8)\n" \
  "v_mfma_i32_16x16x64_i8 v[156:159], v[120:123], a[160:163], v[156:159]\n" \
  "ds_read_b64_tr_b8 v[112:113], v82 offset:32768\n" \
  "v_mfma_i32_16x16x64_i8 v[156:159], v[124:127], a[176:179], v[156:159]\n" \
  "ds_read_b64_tr_b8 v[114:115], v82 offset:33792\n" \
  "v_mfma_i32_16x16x64_i8 v[188:191], v[120:123], a[164:167], v[188:191]\n" \
  "s_lshl_b32 s84, s50, 21\n" \
  "s_lshl_b32 s92, s50, 16\n" \
  "s_add_u32 s49, s49, 1\n" \
  "v_mfma_i32_16x16x64_i8 v[188:191], v[124:127], a[180:183], v[188:191]\n" \
  "buffer_load_dwordx4 a[24:27], v91, s[28:31], s88 offen offset:2048 sc0 sc1\n" \
  "v_mfma_i32_16x16x64_i8 v[220:223], v[120:123], a[168:171], v[220:223]\n" \
  "ds_read_b64_tr_b8 v[116:117], v82 offset:40960\n" \
  "v_mfma_i32_16x16x64_i8 v[220:223], v[124:127], a[184:187], v[220:223]\n" \
  "ds_read_b64_tr_b8 v[118:119], v82 offset:41984\n" \
  "buffer_load_dwordx4 a[28:31], v91, s[28:31], s88 offen offset:3072 sc0 sc1\n" \
  "v_mfma_i32_16x16x64_i8 v[252:255], v[120:123], a[172:175], v[252:255]\n" \
  "v_mfma_i32_16x16x64_i8 v[252:255], v[124:127], a[188:191], v[252:255]\n" \
  "s_barrier\n" \
  "buffer_load_dword v92, v52, s[28:31], 0 offen sc0 sc1\n" \
  "v_add_u32 v52, 0x10000, v52\n" \
  "; ---- step 6\n" \
  "s_waitcnt vmcnt(25)\n" \
  "s_waitcnt lgkmcnt(8)\n" \
  "v_mfma_i32_16x16x64_i8 v[128:131], v[96:99], a[192:195], v[128:131]\n" \
  "ds_read_b64_tr_b8 v[120:121], v83 offset:32768\n" \
  "s_add_u32 m0, s47, 49152\n" \
  "v_mfma_i32_16x16x64_i8 v[128:131], v[100:103], a[208:211], v[128:131]\n" \
  "ds_read_b64_tr_b8 v[122:123], v83 offset:33792\n" \
  "buffer_load_dwordx4 v40, s[16:19], s83 offen nt lds\n" \
  "s_add_u32 m0, s47, 50176\n" \
  "v_mfma_i32_16x16x64_i8 v[160:163], v[96:99], a[196:199], v[160:163]\n" \
  "buffer_load_dwordx4 v41, s[16:19], s83 offen nt lds\n" \
  "s_add_u32 m0, s47, 51200\n" \
  "v_mfma_i32_16x16x64_i8 v[160:163], v[100:103], a[212:215], v[160:163]\n" \
  "buffer_load_dwordx4 v42, s[16:19], s83 offen nt lds\n" \
  "s_add_u32 m0, s47, 52224\n" \
  "v_mfma_i32_16x16x64_i8 v[192:195], v[96:99], a[200:203], v[192:195]\n" \
  "ds_read_b64_tr_b8 v[124:125], v83 offset:40960\n" \
  "buffer_load_dwordx4 v43, s[16:19], s83 offen nt lds\n" \
  "s_add_u32 m0, s47, 65536\n" \
  "v_mfma_i32_16x16x64_i8 v[192:195], v[100:103], a[216:219], v[192:195]\n" \
  "ds_read_b64_tr_b8 v[126:127], v83 offset:41984\n" \
  "buffer_load_dwordx4 v40, s[16:19], s84 offen nt lds\n" \
  "s_add_u32 m0, s47, 66560\n" \
  "v_mfma_i32_16x16x64_i8 v[224:227], v[96:99], a[204:207], v[224:227]\n" \
  "buffer_load_dwordx4 v41, s[16:19], s84 offen nt lds\n" \
  "s_add_u32 m0, s47, 67584\n" \
  "v_mfma_i32_16x16x64_i8 v[224:227], v[100:103], a[220:223], v[224:227]\n" \
  "buffer_load_dwordx4 v42, s[16:19], s84 offen nt lds\n" \
  "s_add_u32 m0, s47, 68608\n" \
  "s_waitcnt lgkmcnt(8)\n" \
  "v_mfma_i32_16x16x64_i8 v[132:135], v[104:107], a[192:195], v[132:135]\n" \
  "ds_read_b64_tr_b8 v[96:97], v84 offset:32768\n" \
  "buffer_load_dwordx4 v43, s[16:19], s84 offen nt lds\n" \
  "v_mfma_i32_16x16x64_i8 v[132:135], v[108:111], a[208:211], v[132:135]\n" \
  "ds_read_b64_tr_b8 v[98:99], v84 offset:33792\n" \
  "v_mfma_i32_16x16x64_i8 v[164:167], v[104:107], a[196:199], v[164:167]\n" \
  "v_mfma_i32_16x16x64_i8 v[164:167], v[108:111], a[212:215], v[164:167]\n" \
  "v_mfma_i32_16x16x64_i8 v[196:199], v[104:107], a[200:203], v[196:199]\n" \
  "ds_read_b64_tr_b8 v[100:101], v84 offset:40960\n" \
  "v_mfma_i32_16x16x64_i8 v[196:199], v[108:111], a[216:219], v[196:199]\n" \
  "ds_read_b64_tr_b8 v[102:103], v84 offset:41984\n" \
  "v_mfma_i32_16x16x64_i8 v[228:231], v[104:107], a[204:207], v[228:231]\n" \
  "v_mfma_i32_16x16x64_i8 v[228:231], v[108:111], a[220:223], v[228:231]\n" \
  "s_waitcnt lgkmcnt(8)\n" \
  "v_mfma_i32_16x16x64_i8 v[136:139], v[112:115], a[192:195], v[136:139]\n" \
  "ds_read_b64_tr_b8 v[104:105], v85 offset:32768\n" \
  "v_mfma_i32_16x16x64_i8 v[136:139], v[116:119], a[208:211], v[136:139]\n" \
  "ds_read_b64_tr_b8 v[106:107], v85 offset:33792\n" \
  "v_mfma_i32_16x16x64_i8 v[168:171], v[112:115], a[196:199], v[168:171]\n" \
  "v_mfma_i32_16x16x64_i8 v[168:171], v[116:119], a[212:215], v[168:171]\n" \
  "v_mfma_i32_16x16x64_i8 v[200:203], v[112:115], a[200:203], v[200:203]\n" \
  "ds_read_b64_tr_b8 v[108:109], v85 offset:40960\n" \
  "v_mfma_i32_16x16x64_i8 v[200:203], v[116:119], a[216:219], v[200:203]\n" \
  "ds_read_b64_tr_b8 v[110:111], v85 offset:41984\n" \
  "v_mfma_i32_16x16x64_i8 v[232:235], v[112:115], a[204:207], v[232:235]\n" \
  "v_mfma_i32_16x16x64_i8 v[232:235], v[116:119], a[220:223], v[232:235]\n" \
  "s_waitcnt lgkmcnt(8)\n" \
  "v_mfma_i32_16x16x64_i8 v[140:143], v[120:123], a[192:195], v[140:143]\n" \
  "ds_read_b64_tr_b8 v[112:113], v86 offset:32768\n" \
  "v_mfma_i32_16x16x64_i8 v[140:143], v[124:127], a[208:211], v[140:143]\n" \
  "ds_read_b64_tr_b8 v[114:115], v86 offset:33792\n" \
  "v_mfma_i32_16x16x64_i8 v[172:175], v[120:123], a[196:199], v[172:175]\n" \
  "v_mfma_i32_16x16x64_i8 v[172:175], v[124:127], a[212:215], v[172:175]\n" \
  "s_waitcnt vmcnt(53)\n" \
  "s_barrier\n" \
  "v_mfma_i32_16x16x64_i8 v[204:207], v[120:123], a[200:203], v[204:207]\n" \
  "ds_read_b64_tr_b8 v[116:117], v86 offset:40960\n" \
  "v_mfma_i32_16x16x64_i8 v[204:207], v[124:127], a[216:219], v[204:207]\n" \
  "ds_read_b64_tr_b8 v[118:119], v86 offset:41984\n" \
  "v_mfma_i32_16x16x64_i8 v[236:239], v[120:123], a[204:207], v[236:239]\n" \
  "v_mfma_i32_16x16x64_i8 v[236:239], v[124:127], a[220:223], v[236:239]\n" \
  "s_waitcnt lgkmcnt(8)\n" \
  "v_mfma_i32_16x16x64_i8 v[144:147], v[96:99], a[192:195], v[144:147]\n" \
  "ds_read_b64_tr_b8 v[120:121], v87 offset:32768\n" \
  "v_mfma_i32_16x16x64_i8 v[144:147], v[100:103], a[208:211], v[144:147]\n" \
  "ds_read_b64_tr_b8 v[122:123], v87 offset:33792\n" \
  "v_mfma_i32_16x16x64_i8 v[176:179], v[96:99], a[196:199], v[176:179]\n" \
  "v_mfma_i32_16x16x64_i8 v[176:179], v[100:103], a[212:215], v[176:179]\n" \
  "v_mfma_i32_16x16x64_i8 v[208:211], v[96:99], a[200:203], v[208:211]\n" \
  "ds_read_b64_tr_b8 v[124:125], v87 offset:40960\n" \
  "v_mfma_i32_16x16x64_i8 v[208:211], v[100:103], a[216:219], v[208:211]\n" \
  "ds_read_b64_tr_b8 v[126:127], v87 offset:41984\n" \
  "v_mfma_i32_16x16x64_i8 v[240:243], v[96:99], a[204:207], v[240:243]\n" \
  "v_mfma_i32_16x16x64_i8 v[240:243], v[100:103], a[220:223], v[240:243]\n" \
  "s_waitcnt lgkmcnt(8)\n" \
  "v_mfma_i32_16x16x64_i8 v[148:151], v[104:107], a[192:195], v[148:151]\n" \
  "ds_read_b64_tr_b8 v[96:97], v80 offset:49152\n" \
  "v_mfma_i32_16x16x64_i8 v[148:151], v[108:111], a[208:211], v[148:151]\n" \
  "ds_read_b64_tr_b8 v[98:99], v80 offset:50176\n" \
  "v_mfma_i32_16x16x64_i8 v[180:183], v[104:107], a[196:199], v[180:183]\n" \
  "v_mfma_i32_16x16x64_i8 v[180:183], v[108:111], a[212:215], v[180:183]\n" \
  "v_mfma_i32_16x16x64_i8 v[212:215], v[104:107], a[200:203], v[212:215]\n" \
  "ds_read_b64_tr_b8 v[100:101], v80 offset:57344\n" \
  "v_mfma_i32_16x16x64_i8 v[212:215], v[108:111], a[216:219], v[212:215]\n" \
  "ds_read_b64_tr_b8 v[102:103], v80 offset:58368\n" \
  "v_mfma_i32_16x16x64_i8 v[244:247], v[104:107], a[204:207], v[244:247]\n" \
  "v_mfma_i32_16x16x64_i8 v[244:247], v[108:111], a[220:223], v[244:247]\n" \
  "s_waitcnt lgkmcnt(8)\n" \
  "v_mfma_i32_16x16x64_i8 v[152:155], v[112:115], a[192:195], v[152:155]\n" \
  "ds_read_b64_tr_b8 v[104:105], v81 offset:49152\n" \
  "v_mfma_i32_16x16x64_i8 v[152:155], v[116:119], a[208:211], v[152:155]\n" \
  "ds_read_b64_tr_b8 v[106:107], v81 offset:50176\n" \
  "v_mfma_i32_16x16x64_i8 v[184:187], v[112:115], a[196:199], v[184:187]\n" \
  "v_mfma_i32_16x16x64_i8 v[184:187], v[116:119], a[212:215], v[184:187]\n" \
  "v_mfma_i32_16x16x64_i8 v[216:219], v[112:115], a[200:203], v[216:219]\n" \
  "ds_read_b64_tr_b8 v[108:109], v81 offset:57344\n" \
  "v_mfma_i32_16x16x64_i8 v[216:219], v[116:119], a[216:219], v[216:219]\n" \
  "ds_read_b64_tr_b8 v[110:111], v81 offset:58368\n" \
  "v_mfma_i32_16x16x64_i8 v[248:251], v[112:115], a[204:207], v[248:251]\n" \
  "v_mfma_i32_16x16x64_i8 v[248:251], v[116:119], a[220:223], v[248:251]\n" \
  "s_waitcnt lgkmcnt(8)\n" \
  "v_mfma_i32_16x16x64_i8 v[156:159], v[120:123], a[192:195], v[156:159]\n" \
  "ds_read_b64_tr_b8 v[112:113], v82 offset:49152\n" \
  "v_mfma_i32_16x16x64_i8 v[156:159], v[124:127], a[208:211], v[156:159]\n" \
  "ds_read_b64_tr_b8 v[114:115], v82 offset:50176\n" \
  "v_mfma_i32_16x16x64_i8 v[188:191], v[120:123], a[196:199], v[188:191]\n" \
  "v_mfma_i32_16x16x64_i8 v[188:191], v[124:127], a[212:215], v[188:191]\n" \
  "v_mfma_i32_16x16x64_i8 v[220:223], v[120:123], a[200:203], v[220:223]\n" \
  "ds_read_b64_tr_b8 v[116:117], v82 offset:57344\n" \
  "v_mfma_i32_16x16x64_i8 v[220:223], v[124:127], a[216:219], v[220:223]\n" \
  "ds_read_b64_tr_b8 v[118:119], v82 offset:58368\n" \
  "v_mfma_i32_16x16x64_i8 v[252:255], v[120:123], a[204:207], v[252:255]\n" \
  "v_mfma_i32_16x16x64_i8 v[252:255], v[124:127], a[220:223], v[252:255]\n" \
  "; ---- step 7\n" \
  "s_waitcnt vmcnt(17)\n" \
  "s_waitcnt lgkmcnt(8)\n" \
  "v_mfma_i32_16x16x64_i8 v[128:131], v[96:99], a[224:227], v[128:131]\n" \
  "ds_read_b64_tr_b8 v[120:121], v83 offset:49152\n" \
  "v_mfma_i32_16x16x64_i8 v[128:131], v[100:103], a[240:243], v[128:131]\n" \
  "ds_read_b64_tr_b8 v[122:123], v83 offset:50176\n" \
  "v_mfma_i32_16x16x64_i8 v[160:163], v[96:99], a[228:231], v[160:163]\n" \
  "v_mfma_i32_16x16x64_i8 v[160:163], v[100:103], a[244:247], v[160:163]\n" \
  "v_mfma_i32_16x16x64_i8 v[192:195], v[96:99], a[232:235], v[192:195]\n" \
  "ds_read_b64_tr_b8 v[124:125], v83 offset:57344\n" \
  "v_mfma_i32_16x16x64_i8 v[192:195], v[100:103], a[248:251], v[192:195]\n" \
  "ds_read_b64_tr_b8 v[126:127], v83 offset:58368\n" \
  "v_mfma_i32_16x16x64_i8 v[224:227], v[96:99], a[236:239], v[224:227]\n" \
  "v_mfma_i32_16x16x64_i8 v[224:227], v[100:103], a[252:255], v[224:227]\n" \
  "s_waitcnt lgkmcnt(8)\n" \
  "v_mfma_i32_16x16x64_i8 v[132:135], v[104:107], a[224:227], v[132:135]\n" \
  "ds_read_b64_tr_b8 v[96:97], v84 offset:49152\n" \
  "v_mfma_i32_16x16x64_i8 v[132:135], v[108:111], a[240:243], v[132:135]\n" \
  "ds_read_b64_tr_b8 v[98:99], v84 offset:50176\n" \
  "v_mfma_i32_16x16x64_i8 v[164:167], v[104:107], a[228:231], v[164:167]\n" \
  "v_mfma_i32_16x16x64_i8 v[164:167], v[108:111], a[244:247], v[164:167]\n" \
  "v_mfma_i32_16x16x64_i8 v[196:199], v[104:107], a[232:235], v[196:199]\n" \
  "ds_read_b64_tr_b8 v[100:101], v84 offset:57344\n" \
  "v_mfma_i32_16x16x64_i8 v[196:199], v[108:111], a[248:251], v[196:199]\n" \
  "ds_read_b64_tr_b8 v[102:103], v84 offset:58368\n" \
  "v_mfma_i32_16x16x64_i8 v[228:231], v[104:107], a[236:239], v[228:231]\n" \
  "v_mfma_i32_16x16x64_i8 v[228:231], v[108:111], a[252:255], v[228:231]\n" \
  "s_waitcnt lgkmcnt(8)\n" \
  "v_mfma_i32_16x16x64_i8 v[136:139], v[112:115], a[224:227], v[136:139]\n" \
  "ds_read_b64_tr_b8 v[104:105], v85 offset:49152\n" \
  "v_mfma_i32_16x16x64_i8 v[136:139], v[116:119], a[240:243], v[136:139]\n" \
  "ds_read_b64_tr_b8 v[106:107], v85 offset:50176\n" \
  "v_mfma_i32_16x16x64_i8 v[168:171], v[112:115], a[228:231], v[168:171]\n" \
  "v_mfma_i32_16x16x64_i8 v[168:171], v[116:119], a[244:247], v[168:171]\n" \
  "v_mfma_i32_16x16x64_i8 v[200:203], v[112:115], a[232:235], v[200:203]\n" \
  "ds_read_b64_tr_b8 v[108:109], v85 offset:57344\n" \
  "v_mfma_i32_16x16x64_i8 v[200:203], v[116:119], a[248:251], v[200:203]\n" \
  "ds_read_b64_tr_b8 v[110:111], v85 offset:58368\n" \
  "buffer_load_dwordx4 a[32:35], v44, s[28:31], s89 offen offset:0 sc0 sc1\n" \
  "v_mfma_i32_16x16x64_i8 v[232:235], v[112:115], a[236:239], v[232:235]\n" \
  "v_mfma_i32_16x16x64_i8 v[232:235], v[116:119], a[252:255], v[232:235]\n" \
  "buffer_load_dwordx4 a[36:39], v44, s[28:31], s89 offen offset:1024 sc0 sc1\n" \
  "s_waitcnt lgkmcnt(8)\n" \
  "v_mfma_i32_16x16x64_i8 v[140:143], v[120:123], a[224:227], v[140:143]\n" \
  "ds_read_b64_tr_b8 v[112:113], v86 offset:49152\n" \
  "v_mfma_i32_16x16x64_i8 v[140:143], v[124:127], a[240:243], v[140:143]\n" \
  "ds_read_b64_tr_b8 v[114:115], v86 offset:50176\n" \
  "buffer_load_dwordx4 a[40:43], v44, s[28:31], s89 offen offset:2048 sc0 sc1\n" \
  "v_mfma_i32_16x16x64_i8 v[172:175], v[120:123], a[228:231], v[172:175]\n" \
  "v_mfma_i32_16x16x64_i8 v[172:175], v[124:127], a[244:247], v[172:175]\n" \
  "s_waitcnt vmcnt(52)\n" \
  "s_barrier\n" \
  "v_mfma_i32_16x16x64_i8 v[204:207], v[120:123], a[232:235], v[204:207]\n" \
  "ds_read_b64_tr_b8 v[116:117], v86 offset:57344\n" \
  "v_mfma_i32_16x16x64_i8 v[204:207], v[124:127], a[248:251], v[204:207]\n" \
  "ds_read_b64_tr_b8 v[118:119], v86 offset:58368\n" \
  "buffer_load_dwordx4 a[44:47], v44, s[28:31], s89 offen offset:3072 sc0 sc1\n" \
  "v_mfma_i32_16x16x64_i8 v[236:239], v[120:123], a[236:239], v[236:239]\n" \
  "v_mfma_i32_16x16x64_i8 v[236:239], v[124:127], a[252:255], v[236:239]\n" \
  "buffer_load_dwordx4 a[48:51], v91, s[28:31], s89 offen offset:0 sc0 sc1\n" \
  "s_waitcnt lgkmcnt(8)\n" \
  "v_mfma_i32_16x16x64_i8 v[144:147], v[96:99], a[224:227], v[144:147]\n" \
  "ds_read_b64_tr_b8 v[120:121], v87 offset:49152\n" \
  "v_mfma_i32_16x16x64_i8 v[144:147], v[100:103], a[240:243], v[144:147]\n" \
  "ds_read_b64_tr_b8 v[122:123], v87 offset:50176\n" \
  "buffer_load_dwordx4 a[52:55], v91, s[28:31], s89 offen offset:1024 sc0 sc1\n" \
  "v_mfma_i32_16x16x64_i8 v[176:179], v[96:99], a[228:231], v[176:179]\n" \
  "v_mfma_i32_16x16x64_i8 v[176:179], v[100:103], a[244:247], v[176:179]\n" \
  "v_mfma_i32_16x16x64_i8 v[208:211], v[96:99], a[232:235], v[208:211]\n" \
  "ds_read_b64_tr_b8 v[124:125], v87 offset:57344\n" \
  "v_mfma_i32_16x16x64_i8 v[208:211], v[100:103], a[248:251], v[208:211]\n" \
  "ds_read_b64_tr_b8 v[126:127], v87 offset:58368\n" \
  "buffer_load_dwordx4 a[56:59], v91, s[28:31], s89 offen offset:2048 sc0 sc1\n" \
  "v_mfma_i32_16x16x64_i8 v[240:243], v[96:99], a[236:239], v[240:243]\n" \
  "v_readlane_b32 s50, v46, s49\n" \
  "v_mfma_i32_16x16x64_i8 v[240:243], v[100:103], a[252:255], v[240:243]\n" \
  "buffer_load_dwordx4 a[60:63], v91, s[28:31], s89 offen offset:3072 sc0 sc1\n" \
  "s_waitcnt lgkmcnt(8)\n" \
  "v_mfma_i32_16x16x64_i8 v[148:151], v[104:107], a[224:227], v[148:151]\n" \
  "ds_read_b64_tr_b8 v[96:97], v32 offset:0\n" \
  "v_mfma_i32_16x16x64_i8 v[148:151], v[108:111], a[240:243], v[148:151]\n" \
  "ds_read_b64_tr_b8 v[98:99], v32 offset:1024\n" \
  "v_mfma_i32_16x16x64_i8 v[180:183], v[104:107], a[228:231], v[180:183]\n" \
  "v_mfma_i32_16x16x64_i8 v[180:183], v[108:111], a[244:247], v[180:183]\n" \
  "buffer_load_dwordx4 a[64:67], v44, s[28:31], s90 offen offset:0 sc0 sc1\n" \
  "v_mfma_i32_16x16x64_i8 v[212:215], v[104:107], a[232:235], v[212:215]\n" \
  "ds_read_b64_tr_b8 v[100:101], v32 offset:8192\n" \
  "v_mfma_i32_16x16x64_i8 v[212:215], v[108:111], a[248:251], v[212:215]\n" \
  "ds_read_b64_tr_b8 v[102:103], v32 offset:9216\n" \
  "buffer_load_dwordx4 a[68:71], v44, s[28:31], s90 offen offset:1024 sc0 sc1\n" \
  "v_mfma_i32_16x16x64_i8 v[244:247], v[104:107], a[236:239], v[244:247]\n" \
  "s_lshl_b32 s85, s50, 21\n" \
  "s_lshl_b32 s93, s50, 16\n" \
  "s_add_u32 s49, s49, 1\n" \
  "v_mfma_i32_16x16x64_i8 v[244:247], v[108:111], a[252:255], v[244:247]\n" \
  "buffer_load_dwordx4 a[72:75], v44, s[28:31], s90 offen offset:2048 sc0 sc1\n" \
  "s_waitcnt lgkmcnt(8)\n" \
  "v_mfma_i32_16x16x64_i8 v[152:155], v[112:115], a[224:227], v[152:155]\n" \
  "ds_read_b64_tr_b8 v[104:105], v33 offset:0\n" \
  "v_mfma_i32_16x16x64_i8 v[152:155], v[116:119], a[240:243], v[152:155]\n" \
  "ds_read_b64_tr_b8 v[106:107], v33 offset:1024\n" \
  "v_mfma_i32_16x16x64_i8 v[184:187], v[112:115], a[228:231], v[184:187]\n" \
  "v_readlane_b32 s50, v46, s49\n" \
  "v_mfma_i32_16x16x64_i8 v[184:187], v[116:119], a[244:247], v[184:187]\n" \
  "buffer_load_dwordx4 a[76:79], v44, s[28:31], s90 offen offset:3072 sc0 sc1\n" \
  "v_mfma_i32_16x16x64_i8 v[216:219], v[112:115], a[232:235], v[216:219]\n" \
  "ds_read_b64_tr_b8 v[108:109], v33 offset:8192\n" \
  "v_mfma_i32_16x16x64_i8 v[216:219], v[116:119], a[248:251], v[216:219]\n" \
  "ds_read_b64_tr_b8 v[110:111], v33 offset:9216\n" \
  "buffer_load_dwordx4 a[80:83], v91, s[28:31], s90 offen offset:0 sc0 sc1\n" \
  "v_mfma_i32_16x16x64_i8 v[248:251], v[112:115], a[236:239], v[248:251]\n" \
  "v_mfma_i32_16x16x64_i8 v[248:251], v[116:119], a[252:255], v[248:251]\n" \
  "buffer_load_dwordx4 a[84:87], v91, s[28:31], s90 offen offset:1024 sc0 sc1\n" \
  "s_waitcnt lgkmcnt(8)\n" \
  "v_mfma_i32_16x16x64_i8 v[156:159], v[120:123], a[224:227], v[156:159]\n" \
  "ds_read_b64_tr_b8 v[112:113], v34 offset:0\n" \
  "v_mfma_i32_16x16x64_i8 v[156:159], v[124:127], a[240:243], v[156:159]\n" \
  "ds_read_b64_tr_b8 v[114:115], v34 offset:1024\n" \
  "v_mfma_i32_16x16x64_i8 v[188:191], v[120:123], a[228:231], v[188:191]\n" \
  "s_lshl_b32 s86, s50, 21\n" \
  "s_lshl_b32 s94, s50, 16\n" \
  "s_add_u32 s49, s49, 1\n" \
  "v_mfma_i32_16x16x64_i8 v[188:191], v[124:127], a[244:247], v[188:191]\n" \
  "buffer_load_dwordx4 a[88:91], v91, s[28:31], s90 offen offset:2048 sc0 sc1\n" \
  "v_mfma_i32_16x16x64_i8 v[220:223], v[120:123], a[232:235], v[220:223]\n" \
  "ds_read_b64_tr_b8 v[116:117], v34 offset:8192\n" \
  "v_mfma_i32_16x16x64_i8 v[220:223], v[124:127], a[248:251], v[220:223]\n" \
  "ds_read_b64_tr_b8 v[118:119], v34 offset:9216\n" \
  "buffer_load_dwordx4 a[92:95], v91, s[28:31], s90 offen offset:3072 sc0 sc1\n" \
  "v_mfma_i32_16x16x64_i8 v[252:255], v[120:123], a[236:239], v[252:255]\n" \
  "v_mfma_i32_16x16x64_i8 v[252:255], v[124:127], a[252:255], v[252:255]\n" \
  "s_add_u32 s53, s53, 1\n" \
  "s_add_u32 s54, s54, 1\n" \
  "s_sub_u32 s51, s51, 1\n" \
  "s_cmp_lg_u32 s51, 0\n" \
  "s_cbranch_scc1 L_loop_%=\n" \
  "; ---- step 0\n" \
  "global_load_dword v90, v51, s[12:13] sc1\n" \
  "s_waitcnt vmcnt(26)\n" \
  "s_waitcnt lgkmcnt(8)\n" \
  "v_mfma_i32_16x16x64_i8 v[128:131], v[96:99], a[0:3], v[128:131]\n" \
  "ds_read_b64_tr_b8 v[120:121], v35 offset:0\n" \
  "s_add_u32 m0, s47, 81920\n" \
  "v_mfma_i32_16x16x64_i8 v[128:131], v[100:103], a[16:19], v[128:131]\n" \
  "ds_read_b64_tr_b8 v[122:123], v35 offset:1024\n" \
  "buffer_load_dwordx4 v40, s[16:19], s85 offen nt lds\n" \
  "s_add_u32 m0, s47, 82944\n" \
  "v_mfma_i32_16x16x64_i8 v[160:163], v[96:99], a[4:7], v[160:163]\n" \
  "buffer_load_dwordx4 v41, s[16:19], s85 offen nt lds\n" \
  "s_add_u32 m0, s47, 83968\n" \
  "v_mfma_i32_16x16x64_i8 v[160:163], v[100:103], a[20:23], v[160:163]\n" \
  "buffer_load_dwordx4 v42, s[16:19], s85 offen nt lds\n" \
  "s_add_u32 m0, s47, 84992\n" \
  "v_mfma_i32_16x16x64_i8 v[192:195], v[96:99], a[8:11], v[192:195]\n" \
  "ds_read_b64_tr_b8 v[124:125], v35 offset:8192\n" \
  "buffer_load_dwordx4 v43, s[16:19], s85 offen nt lds\n" \
  "s_add_u32 m0, s47, 98304\n" \
  "v_mfma_i32_16x16x64_i8 v[192:195], v[100:103], a[24:27], v[192:195]\n" \
  "ds_read_b64_tr_b8 v[126:127], v35 offset:9216\n" \
  "buffer_load_dwordx4 v40, s[16:19], s86 offen nt lds\n" \
  "s_add_u32 m0, s47, 99328\n" \
  "v_mfma_i32_16x16x64_i8 v[224:227], v[96:99], a[12:15], v[224:227]\n" \
  "buffer_load_dwordx4 v41, s[16:19], s86 offen nt lds\n" \
  "s_add_u32 m0, s47, 100352\n" \
  "v_mfma_i32_16x16x64_i8 v[224:227], v[100:103], a[28:31], v[224:227]\n" \
  "buffer_load_dwordx4 v42, s[16:19], s86 offen nt lds\n" \
  "s_add_u32 m0, s47, 101376\n" \
  "s_waitcnt lgkmcnt(8)\n" \
  "v_mfma_i32_16x16x64_i8 v[132:135], v[104:107], a[0:3], v[132:135]\n" \
  "ds_read_b64_tr_b8 v[96:97], v36 offset:0\n" \
  "buffer_load_dwordx4 v43, s[16:19], s86 offen nt lds\n" \
  "v_mfma_i32_16x16x64_i8 v[132:135], v[108:111], a[16:19], v[132:135]\n" \
  "ds_read_b64_tr_b8 v[98:99], v36 offset:1024\n" \
  "v_mfma_i32_16x16x64_i8 v[164:167], v[104:107], a[4:7], v[164:167]\n" \
  "v_mfma_i32_16x16x64_i8 v[164:167], v[108:111], a[20:23], v[164:167]\n" \
  "v_mfma_i32_16x16x64_i8 v[196:199], v[104:107], a[8:11], v[196:199]\n" \
  "ds_read_b64_tr_b8 v[100:101], v36 offset:8192\n" \
  "v_mfma_i32_16x16x64_i8 v[196:199], v[108:111], a[24:27], v[196:199]\n" \
  "ds_read_b64_tr_b8 v[102:103], v36 offset:9216\n" \
  "v_mfma_i32_16x16x64_i8 v[228:231], v[104:107], a[12:15], v[228:231]\n" \
  "v_mfma_i32_16x16x64_i8 v[228:231], v[108:111], a[28:31], v[228:231]\n" \
  "s_waitcnt lgkmcnt(8)\n" \
  "v_mfma_i32_16x16x64_i8 v[136:139], v[112:115], a[0:3], v[136:139]\n" \
  "ds_read_b64_tr_b8 v[104:105], v37 offset:0\n" \
  "v_mfma_i32_16x16x64_i8 v[136:139], v[116:119], a[16:19], v[136:139]\n" \
  "ds_read_b64_tr_b8 v[106:107], v37 offset:1024\n" \
  "v_mfma_i32_16x16x64_i8 v[168:171], v[112:115], a[4:7], v[168:171]\n" \
  "v_mfma_i32_16x16x64_i8 v[168:171], v[116:119], a[20:23], v[168:171]\n" \
  "v_mfma_i32_16x16x64_i8 v[200:203], v[112:115], a[8:11], v[200:203]\n" \
  "ds_read_b64_tr_b8 v[108:109], v37 offset:8192\n" \
  "v_mfma_i32_16x16x64_i8 v[200:203], v[116:119], a[24:27], v[200:203]\n" \
  "ds_read_b64_tr_b8 v[110:111], v37 offset:9216\n" \
  "v_mfma_i32_16x16x64_i8 v[232:235], v[112:115], a[12:15], v[232:235]\n" \
  "v_mfma_i32_16x16x64_i8 v[232:235], v[116:119], a[28:31], v[232:235]\n" \
  "s_waitcnt lgkmcnt(8)\n" \
  "v_mfma_i32_16x16x64_i8 v[140:143], v[120:123], a[0:3], v[140:143]\n" \
  "ds_read_b64_tr_b8 v[112:113], v38 offset:0\n" \
  "v_mfma_i32_16x16x64_i8 v[140:143], v[124:127], a[16:19], v[140:143]\n" \
  "ds_read_b64_tr_b8 v[114:115], v38 offset:1024\n" \
  "v_mfma_i32_16x16x64_i8 v[172:175], v[120:123], a[4:7], v[172:175]\n" \
  "v_mfma_i32_16x16x64_i8 v[172:175], v[124:127], a[20:23], v[172:175]\n" \
  "s_waitcnt vmcnt(54)\n" \
  "s_barrier\n" \
  "v_mfma_i32_16x16x64_i8 v[204:207], v[120:123], a[8:11], v[204:207]\n" \
  "ds_read_b64_tr_b8 v[116:117], v38 offset:8192\n" \
  "v_mfma_i32_16x16x64_i8 v[204:207], v[124:127], a[24:27], v[204:207]\n" \
  "ds_read_b64_tr_b8 v[118:119], v38 offset:9216\n" \
  "v_mfma_i32_16x16x64_i8 v[236:239], v[120:123], a[12:15], v[236:239]\n" \
  "v_mfma_i32_16x16x64_i8 v[236:239], v[124:127], a[28:31], v[236:239]\n" \
  "s_waitcnt lgkmcnt(8)\n" \
  "v_mfma_i32_16x16x64_i8 v[144:147], v[96:99], a[0:3], v[144:147]\n" \
  "ds_read_b64_tr_b8 v[120:121], v39 offset:0\n" \
  "v_mfma_i32_16x16x64_i8 v[144:147], v[100:103], a[16:19], v[144:147]\n" \
  "ds_read_b64_tr_b8 v[122:123], v39 offset:1024\n" \
  "v_mfma_i32_16x16x64_i8 v[176:179], v[96:99], a[4:7], v[176:179]\n" \
  "v_mfma_i32_16x16x64_i8 v[176:179], v[100:103], a[20:23], v[176:179]\n" \
  "v_mfma_i32_16x16x64_i8 v[208:211], v[96:99], a[8:11], v[208:211]\n" \
  "ds_read_b64_tr_b8 v[124:125], v39 offset:8192\n" \
  "v_mfma_i32_16x16x64_i8 v[208:211], v[100:103], a[24:27], v[208:211]\n" \
  "ds_read_b64_tr_b8 v[126:127], v39 offset:9216\n" \
  "v_mfma_i32_16x16x64_i8 v[240:243], v[96:99], a[12:15], v[240:243]\n" \
  "v_mfma_i32_16x16x64_i8 v[240:243], v[100:103], a[28:31], v[240:243]\n" \
  "s_waitcnt lgkmcnt(8)\n" \
  "v_mfma_i32_16x16x64_i8 v[148:151], v[104:107], a[0:3], v[148:151]\n" \
  "ds_read_b64_tr_b8 v[96:97], v32 offset:16384\n" \
  "v_mfma_i32_16x16x64_i8 v[148:151], v[108:111], a[16:19], v[148:151]\n" \
  "ds_read_b64_tr_b8 v[98:99], v32 offset:17408\n" \
  "v_mfma_i32_16x16x64_i8 v[180:183], v[104:107], a[4:7], v[180:183]\n" \
  "v_mfma_i32_16x16x64_i8 v[180:183], v[108:111], a[20:23], v[180:183]\n" \
  "v_mfma_i32_16x16x64_i8 v[212:215], v[104:107], a[8:11], v[212:215]\n" \
  "ds_read_b64_tr_b8 v[100:101], v32 offset:24576\n" \
  "v_mfma_i32_16x16x64_i8 v[212:215], v[108:111], a[24:27], v[212:215]\n" \
  "ds_read_b64_tr_b8 v[102:103], v32 offset:25600\n" \
  "v_mfma_i32_16x16x64_i8 v[244:247], v[104:107], a[12:15], v[244:247]\n" \
  "v_mfma_i32_16x16x64_i8 v[244:247], v[108:111], a[28:31], v[244:247]\n" \
  "s_waitcnt lgkmcnt(8)\n" \
  "v_mfma_i32_16x16x64_i8 v[152:155], v[112:115], a[0:3], v[152:155]\n" \
  "ds_read_b64_tr_b8 v[104:105], v33 offset:16384\n" \
  "v_mfma_i32_16x16x64_i8 v[152:155], v[116:119], a[16:19], v[152:155]\n" \
  "ds_read_b64_tr_b8 v[106:107], v33 offset:17408\n" \
  "v_mfma_i32_16x16x64_i8 v[184:187], v[112:115], a[4:7], v[184:187]\n" \
  "v_mfma_i32_16x16x64_i8 v[184:187], v[116:119], a[20:23], v[184:187]\n" \
  "v_mfma_i32_16x16x64_i8 v[216:219], v[112:115], a[8:11], v[216:219]\n" \
  "ds_read_b64_tr_b8 v[108:109], v33 offset:24576\n" \
  "v_mfma_i32_16x16x64_i8 v[216:219], v[116:119], a[24:27], v[216:219]\n" \
  "ds_read_b64_tr_b8 v[110:111], v33 offset:25600\n" \
  "v_mfma_i32_16x16x64_i8 v[248:251], v[112:115], a[12:15], v[248:251]\n" \
  "v_mfma_i32_16x16x64_i8 v[248:251], v[116:119], a[28:31], v[248:251]\n" \
  "s_waitcnt lgkmcnt(8)\n" \
  "v_mfma_i32_16x16x64_i8 v[156:159], v[120:123], a[0:3], v[156:159]\n" \
  "ds_read_b64_tr_b8 v[112:113], v34 offset:16384\n" \
  "v_mfma_i32_16x16x64_i8 v[156:159], v[124:127], a[16:19], v[156:159]\n" \
  "ds_read_b64_tr_b8 v[114:115], v34 offset:17408\n" \
  "v_mfma_i32_16x16x64_i8 v[188:191], v[120:123], a[4:7], v[188:191]\n" \
  "v_mfma_i32_16x16x64_i8 v[188:191], v[124:127], a[20:23], v[188:191]\n" \
  "v_mfma_i32_16x16x64_i8 v[220:223], v[120:123], a[8:11], v[220:223]\n" \
  "ds_read_b64_tr_b8 v[116:117], v34 offset:24576\n" \
  "v_mfma_i32_16x16x64_i8 v[220:223], v[124:127], a[24:27], v[220:223]\n" \
  "ds_read_b64_tr_b8 v[118:119], v34 offset:25600\n" \
  "v_mfma_i32_16x16x64_i8 v[252:255], v[120:123], a[12:15], v[252:255]\n" \
  "v_mfma_i32_16x16x64_i8 v[252:255], v[124:127], a[28:31], v[252:255]\n" \
  "; ---- step 1\n" \
  "s_waitcnt vmcnt(17)\n" \
  "s_waitcnt lgkmcnt(8)\n" \
  "v_mfma_i32_16x16x64_i8 v[128:131], v[96:99], a[32:35], v[128:131]\n" \
  "ds_read_b64_tr_b8 v[120:121], v35 offset:16384\n" \
  "v_mfma_i32_16x16x64_i8 v[128:131], v[100:103], a[48:51], v[128:131]\n" \
  "ds_read_b64_tr_b8 v[122:123], v35 offset:17408\n" \
  "v_mfma_i32_16x16x64_i8 v[160:163], v[96:99], a[36:39], v[160:163]\n" \
  "v_mfma_i32_16x16x64_i8 v[160:163], v[100:103], a[52:55], v[160:163]\n" \
  "v_mfma_i32_16x16x64_i8 v[192:195], v[96:99], a[40:43], v[192:195]\n" \
  "ds_read_b64_tr_b8 v[124:125], v35 offset:24576\n" \
  "v_mfma_i32_16x16x64_i8 v[192:195], v[100:103], a[56:59], v[192:195]\n" \
  "ds_read_b64_tr_b8 v[126:127], v35 offset:25600\n" \
  "v_mfma_i32_16x16x64_i8 v[224:227], v[96:99], a[44:47], v[224:227]\n" \
  "v_mfma_i32_16x16x64_i8 v[224:227], v[100:103], a[60:63], v[224:227]\n" \
  "s_waitcnt lgkmcnt(8)\n" \
  "v_mfma_i32_16x16x64_i8 v[132:135], v[104:107], a[32:35], v[132:135]\n" \
  "ds_read_b64_tr_b8 v[96:97], v36 offset:16384\n" \
  "v_mfma_i32_16x16x64_i8 v[132:135], v[108:111], a[48:51], v[132:135]\n" \
  "ds_read_b64_tr_b8 v[98:99], v36 offset:17408\n" \
  "v_mfma_i32_16x16x64_i8 v[164:167], v[104:107], a[36:39], v[164:167]\n" \
  "v_mfma_i32_16x16x64_i8 v[164:167], v[108:111], a[52:55], v[164:167]\n" \
  "v_mfma_i32_16x16x64_i8 v[196:199], v[104:107], a[40:43], v[196:199]\n" \
  "ds_read_b64_tr_b8 v[100:101], v36 offset:24576\n" \
  "v_mfma_i32_16x16x64_i8 v[196:199], v[108:111], a[56:59], v[196:199]\n" \
  "ds_read_b64_tr_b8 v[102:103], v36 offset:25600\n" \
  "v_mfma_i32_16x16x64_i8 v[228:231], v[104:107], a[44:47], v[228:231]\n" \
  "v_mfma_i32_16x16x64_i8 v[228:231], v[108:111], a[60:63], v[228:231]\n" \
  "s_waitcnt lgkmcnt(8)\n" \
  "v_mfma_i32_16x16x64_i8 v[136:139], v[112:115], a[32:35], v[136:139]\n" \
  "ds_read_b64_tr_b8 v[104:105], v37 offset:16384\n" \
  "v_mfma_i32_16x16x64_i8 v[136:139], v[116:119], a[48:51], v[136:139]\n" \
  "ds_read_b64_tr_b8 v[106:107], v37 offset:17408\n" \
  "v_mfma_i32_16x16x64_i8 v[168:171], v[112:115], a[36:39], v[168:171]\n" \
  "v_mfma_i32_16x16x64_i8 v[168:171], v[116:119], a[52:55], v[168:171]\n" \
  "v_mfma_i32_16x16x64_i8 v[200:203], v[112:115], a[40:43], v[200:203]\n" \
  "ds_read_b64_tr_b8 v[108:109], v37 offset:24576\n" \
  "v_mfma_i32_16x16x64_i8 v[200:203], v[116:119], a[56:59], v[200:203]\n" \
  "ds_read_b64_tr_b8 v[110:111], v37 offset:25600\n" \
  "buffer_load_dwordx4 a[96:99], v44, s[28:31], s91 offen offset:0 sc0 sc1\n" \
  "v_mfma_i32_16x16x64_i8 v[232:235], v[112:115], a[44:47], v[232:235]\n" \
  "v_mfma_i32_16x16x64_i8 v[232:235], v[116:119], a[60:63], v[232:235]\n" \
  "buffer_load_dwordx4 a[100:103], v44, s[28:31], s91 offen offset:1024 sc0 sc1\n" \
  "s_waitcnt lgkmcnt(8)\n" \
  "v_mfma_i32_16x16x64_i8 v[140:143], v[120:123], a[32:35], v[140:143]\n" \
  "ds_read_b64_tr_b8 v[112:113], v38 offset:16384\n" \
  "v_mfma_i32_16x16x64_i8 v[140:143], v[124:127], a[48:51], v[140:143]\n" \
  "ds_read_b64_tr_b8 v[114:115], v38 offset:17408\n" \
  "buffer_load_dwordx4 a[104:107], v44, s[28:31], s91 offen offset:2048 sc0 sc1\n" \
  "v_mfma_i32_16x16x64_i8 v[172:175], v[120:123], a[36:39], v[172:175]\n" \
  "v_mfma_i32_16x16x64_i8 v[172:175], v[124:127], a[52:55], v[172:175]\n" \
  "s_waitcnt vmcnt(53)\n" \
  "s_barrier\n" \
  "v_mfma_i32_16x16x64_i8 v[204:207], v[120:123], a[40:43], v[204:207]\n" \
  "ds_read_b64_tr_b8 v[116:117], v38 offset:24576\n" \
  "v_mfma_i32_16x16x64_i8 v[204:207], v[124:127], a[56:59], v[204:207]\n" \
  "ds_read_b64_tr_b8 v[118:119], v38 offset:25600\n" \
  "buffer_load_dwordx4 a[108:111], v44, s[28:31], s91 offen offset:3072 sc0 sc1\n" \
  "v_mfma_i32_16x16x64_i8 v[236:239], v[120:123], a[44:47], v[236:239]\n" \
  "v_mfma_i32_16x16x64_i8 v[236:239], v[124:127], a[60:63], v[236:239]\n" \
  "buffer_load_dwordx4 a[112:115], v91, s[28:31], s91 offen offset:0 sc0 sc1\n" \
  "s_waitcnt lgkmcnt(8)\n" \
  "v_mfma_i32_16x16x64_i8 v[144:147], v[96:99], a[32:35], v[144:147]\n" \
  "ds_read_b64_tr_b8 v[120:121], v39 offset:16384\n" \
  "v_mfma_i32_16x16x64_i8 v[144:147], v[100:103], a[48:51], v[144:147]\n" \
  "ds_read_b64_tr_b8 v[122:123], v39 offset:17408\n" \
  "buffer_load_dwordx4 a[116:119], v91, s[28:31], s91 offen offset:1024 sc0 sc1\n" \
  "v_mfma_i32_16x16x64_i8 v[176:179], v[96:99], a[36:39], v[176:179]\n" \
  "v_mfma_i32_16x16x64_i8 v[176:179], v[100:103], a[52:55], v[176:179]\n" \
  "v_mfma_i32_16x16x64_i8 v[208:211], v[96:99], a[40:43], v[208:211]\n" \
  "ds_read_b64_tr_b8 v[124:125], v39 offset:24576\n" \
  "v_mfma_i32_16x16x64_i8 v[208:211], v[100:103], a[56:59], v[208:211]\n" \
  "ds_read_b64_tr_b8 v[126:127], v39 offset:25600\n" \
  "buffer_load_dwordx4 a[120:123], v91, s[28:31], s91 offen offset:2048 sc0 sc1\n" \
  "v_mfma_i32_16x16x64_i8 v[240:243], v[96:99], a[44:47], v[240:243]\n" \
  "v_mfma_i32_16x16x64_i8 v[240:243], v[100:103], a[60:63], v[240:243]\n" \
  "buffer_load_dwordx4 a[124:127], v91, s[28:31], s91 offen offset:3072 sc0 sc1\n" \
  "s_waitcnt lgkmcnt(8)\n" \
  "v_mfma_i32_16x16x64_i8 v[148:151], v[104:107], a[32:35], v[148:151]\n" \
  "ds_read_b64_tr_b8 v[96:97], v32 offset:32768\n" \
  "v_mfma_i32_16x16x64_i8 v[148:151], v[108:111], a[48:51], v[148:151]\n" \
  "ds_read_b64_tr_b8 v[98:99], v32 offset:33792\n" \
  "v_mfma_i32_16x16x64_i8 v[180:183], v[104:107], a[36:39], v[180:183]\n" \
  "v_mfma_i32_16x16x64_i8 v[180:183], v[108:111], a[52:55], v[180:183]\n" \
  "buffer_load_dwordx4 a[128:131], v44, s[28:31], s92 offen offset:0 sc0 sc1\n" \
  "v_mfma_i32_16x16x64_i8 v[212:215], v[104:107], a[40:43], v[212:215]\n" \
  "ds_read_b64_tr_b8 v[100:101], v32 offset:40960\n" \
  "v_mfma_i32_16x16x64_i8 v[212:215], v[108:111], a[56:59], v[212:215]\n" \
  "ds_read_b64_tr_b8 v[102:103], v32 offset:41984\n" \
  "buffer_load_dwordx4 a[132:135], v44, s[28:31], s92 offen offset:1024 sc0 sc1\n" \
  "v_mfma_i32_16x16x64_i8 v[244:247], v[104:107], a[44:47], v[244:247]\n" \
  "v_mfma_i32_16x16x64_i8 v[244:247], v[108:111], a[60:63], v[244:247]\n" \
  "buffer_load_dwordx4 a[136:139], v44, s[28:31], s92 offen offset:2048 sc0 sc1\n" \
  "s_waitcnt lgkmcnt(8)\n" \
  "v_mfma_i32_16x16x64_i8 v[152:155], v[112:115], a[32:35], v[152:155]\n" \
  "ds_read_b64_tr_b8 v[104:105], v33 offset:32768\n" \
  "v_mfma_i32_16x16x64_i8 v[152:155], v[116:119], a[48:51], v[152:155]\n" \
  "ds_read_b64_tr_b8 v[106:107], v33 offset:33792\n" \
  "v_mfma_i32_16x16x64_i8 v[184:187], v[112:115], a[36:39], v[184:187]\n" \
  "v_readlane_b32 s50, v46, s49\n" \
  "v_mfma_i32_16x16x64_i8 v[184:187], v[116:119], a[52:55], v[184:187]\n" \
  "buffer_load_dwordx4 a[140:143], v44, s[28:31], s92 offen offset:3072 sc0 sc1\n" \
  "v_mfma_i32_16x16x64_i8 v[216:219], v[112:115], a[40:43], v[216:219]\n" \
  "ds_read_b64_tr_b8 v[108:109], v33 offset:40960\n" \
  "v_mfma_i32_16x16x64_i8 v[216:219], v[116:119], a[56:59], v[216:219]\n" \
  "ds_read_b64_tr_b8 v[110:111], v33 offset:41984\n" \
  "buffer_load_dwordx4 a[144:147], v91, s[28:31], s92 offen offset:0 sc0 sc1\n" \
  "v_mfma_i32_16x16x64_i8 v[248:251], v[112:115], a[44:47], v[248:251]\n" \
  "v_mfma_i32_16x16x64_i8 v[248:251], v[116:119], a[60:63], v[248:251]\n" \
  "buffer_load_dwordx4 a[148:151], v91, s[28:31], s92 offen offset:1024 sc0 sc1\n" \
  "s_waitcnt lgkmcnt(8)\n" \
  "v_mfma_i32_16x16x64_i8 v[156:159], v[120:123], a[32:35], v[156:159]\n" \
  "ds_read_b64_tr_b8 v[112:113], v34 offset:32768\n" \
  "v_mfma_i32_16x16x64_i8 v[156:159], v[124:127], a[48:51], v[156:159]\n" \
  "ds_read_b64_tr_b8 v[114:115], v34 offset:33792\n" \
  "v_mfma_i32_16x16x64_i8 v[188:191], v[120:123], a[36:39], v[188:191]\n" \
  "s_lshl_b32 s87, s50, 21\n" \
  "s_lshl_b32 s95, s50, 16\n" \
  "s_add_u32 s49, s49, 1\n" \
  "v_mfma_i32_16x16x64_i8 v[188:191], v[124:127], a[52:55], v[188:191]\n" \
  "buffer_load_dwordx4 a[152:155], v91, s[28:31], s92 offen offset:2048 sc0 sc1\n" \
  "v_mfma_i32_16x16x64_i8 v[220:223], v[120:123], a[40:43], v[220:223]\n" \
  "ds_read_b64_tr_b8 v[116:117], v34 offset:40960\n" \
  "v_mfma_i32_16x16x64_i8 v[220:223], v[124:127], a[56:59], v[220:223]\n" \
  "ds_read_b64_tr_b8 v[118:119], v34 offset:41984\n" \
  "buffer_load_dwordx4 a[156:159], v91, s[28:31], s92 offen offset:3072 sc0 sc1\n" \
  "v_mfma_i32_16x16x64_i8 v[252:255], v[120:123], a[44:47], v[252:255]\n" \
  "v_mfma_i32_16x16x64_i8 v[252:255], v[124:127], a[60:63], v[252:255]\n" \
  "; ---- step 2\n" \
  "s_waitcnt vmcnt(24)\n" \
  "v_mul_f32 v88, v77, v78\n" \
  "v_cmp_eq_u32 vcc, v90, v76\n" \
  "s_nop 1\n" \
  "s_cmp_lg_u32 s55, 0\n" \
  "s_cselect_b64 vcc, -1, vcc\n" \
  "v_mov_b32 v89, 0x7fc00000\n" \
  "s_nop 1\n" \
  "v_cndmask_b32 v88, v88, v89, vcc\n" \
  "v_mov_b32 v89, v88\n" \
  "s_waitcnt vmcnt(25)\n" \
  "s_waitcnt lgkmcnt(8)\n" \
  "v_mfma_i32_16x16x64_i8 v[128:131], v[96:99], a[64:67], v[128:131]\n" \
  "ds_read_b64_tr_b8 v[120:121], v35 offset:32768\n" \
  "s_add_u32 m0, s47, 114688\n" \
  "v_mfma_i32_16x16x64_i8 v[128:131], v[100:103], a[80:83], v[128:131]\n" \
  "ds_read_b64_tr_b8 v[122:123], v35 offset:33792\n" \
  "buffer_load_dwordx4 v40, s[16:19], s87 offen nt lds\n" \
  "s_add_u32 m0, s47, 115712\n" \
  "v_mfma_i32_16x16x64_i8 v[160:163], v[96:99], a[68:71], v[160:163]\n" \
  "buffer_load_dwordx4 v41, s[16:19], s87 offen nt lds\n" \
  "s_add_u32 m0, s47, 116736\n" \
  "v_mfma_i32_16x16x64_i8 v[160:163], v[100:103], a[84:87], v[160:163]\n" \
  "buffer_load_dwordx4 v42, s[16:19], s87 offen nt lds\n" \
  "s_add_u32 m0, s47, 117760\n" \
  "v_mfma_i32_16x16x64_i8 v[192:195], v[96:99], a[72:75], v[192:195]\n" \
  "ds_read_b64_tr_b8 v[124:125], v35 offset:40960\n" \
  "buffer_load_dwordx4 v43, s[16:19], s87 offen nt lds\n" \
  "v_mfma_i32_16x16x64_i8 v[192:195], v[100:103], a[88:91], v[192:195]\n" \
  "ds_read_b64_tr_b8 v[126:127], v35 offset:41984\n" \
  "v_mfma_i32_16x16x64_i8 v[224:227], v[96:99], a[76:79], v[224:227]\n" \
  "v_mfma_i32_16x16x64_i8 v[224:227], v[100:103], a[92:95], v[224:227]\n" \
  "s_waitcnt lgkmcnt(8)\n" \
  "v_mfma_i32_16x16x64_i8 v[132:135], v[104:107], a[64:67], v[132:135]\n" \
  "ds_read_b64_tr_b8 v[96:97], v36 offset:32768\n" \
  "v_mfma_i32_16x16x64_i8 v[132:135], v[108:111], a[80:83], v[132:135]\n" \
  "ds_read_b64_tr_b8 v[98:99], v36 offset:33792\n" \
  "v_mfma_i32_16x16x64_i8 v[164:167], v[104:107], a[68:71], v[164:167]\n" \
  "v_mfma_i32_16x16x64_i8 v[164:167], v[108:111], a[84:87], v[164:167]\n" \
  "v_mfma_i32_16x16x64_i8 v[196:199], v[104:107], a[72:75], v[196:199]\n" \
  "ds_read_b64_tr_b8 v[100:101], v36 offset:40960\n" \
  "v_mfma_i32_16x16x64_i8 v[196:199], v[108:111], a[88:91], v[196:199]\n" \
  "ds_read_b64_tr_b8 v[102:103], v36 offset:41984\n" \
  "v_mfma_i32_16x16x64_i8 v[228:231], v[104:107], a[76:79], v[228:231]\n" \
  "v_mfma_i32_16x16x64_i8 v[228:231], v[108:111], a[92:95], v[228:231]\n" \
  "s_waitcnt lgkmcnt(8)\n" \
  "v_mfma_i32_16x16x64_i8 v[136:139], v[112:115], a[64:67], v[136:139]\n" \
  "ds_read_b64_tr_b8 v[104:105], v37 offset:32768\n" \
  "v_mfma_i32_16x16x64_i8 v[136:139], v[116:119], a[80:83], v[136:139]\n" \
  "ds_read_b64_tr_b8 v[106:107], v37 offset:33792\n" \
  "v_mfma_i32_16x16x64_i8 v[168:171], v[112:115], a[68:71], v[168:171]\n" \
  "v_mfma_i32_16x16x64_i8 v[168:171], v[116:119], a[84:87], v[168:171]\n" \
  "v_mfma_i32_16x16x64_i8 v[200:203], v[112:115], a[72:75], v[200:203]\n" \
  "ds_read_b64_tr_b8 v[108:109], v37 offset:40960\n" \
  "v_mfma_i32_16x16x64_i8 v[200:203], v[116:119], a[88:91], v[200:203]\n" \
  "ds_read_b64_tr_b8 v[110:111], v37 offset:41984\n" \
  "v_mfma_i32_16x16x64_i8 v[232:235], v[112:115], a[76:79], v[232:235]\n" \
  "v_mfma_i32_16x16x64_i8 v[232:235], v[116:119], a[92:95], v[232:235]\n" \
  "s_waitcnt lgkmcnt(8)\n" \
  "v_mfma_i32_16x16x64_i8 v[140:143], v[120:123], a[64:67], v[140:143]\n" \
  "ds_read_b64_tr_b8 v[112:113], v38 offset:32768\n" \
  "v_mfma_i32_16x16x64_i8 v[140:143], v[124:127], a[80:83], v[140:143]\n" \
  "ds_read_b64_tr_b8 v[114:115], v38 offset:33792\n" \
  "v_mfma_i32_16x16x64_i8 v[172:175], v[120:123], a[68:71], v[172:175]\n" \
  "v_mfma_i32_16x16x64_i8 v[172:175], v[124:127], a[84:87], v[172:175]\n" \
  "s_waitcnt vmcnt(20)\n" \
  "s_barrier\n" \
  "v_mfma_i32_16x16x64_i8 v[204:207], v[120:123], a[72:75], v[204:207]\n" \
  "ds_read_b64_tr_b8 v[116:117], v38 offset:40960\n" \
  "v_mfma_i32_16x16x64_i8 v[204:207], v[124:127], a[88:91], v[204:207]\n" \
  "ds_read_b64_tr_b8 v[118:119], v38 offset:41984\n" \
  "v_mfma_i32_16x16x64_i8 v[236:239], v[120:123], a[76:79], v[236:239]\n" \
  "v_mfma_i32_16x16x64_i8 v[236:239], v[124:127], a[92:95], v[236:239]\n" \
  "s_waitcnt lgkmcnt(8)\n" \
  "v_mfma_i32_16x16x64_i8 v[144:147], v[96:99], a[64:67], v[144:147]\n" \
  "ds_read_b64_tr_b8 v[120:121], v39 offset:32768\n" \
  "v_mfma_i32_16x16x64_i8 v[144:147], v[100:103], a[80:83], v[144:147]\n" \
  "ds_read_b64_tr_b8 v[122:123], v39 offset:33792\n" \
  "v_mfma_i32_16x16x64_i8 v[176:179], v[96:99], a[68:71], v[176:179]\n" \
  "v_mfma_i32_16x16x64_i8 v[176:179], v[100:103], a[84:87], v[176:179]\n" \
  "buffer_load_dwordx4 a[160:163], v44, s[28:31], s93 offen offset:0 sc0 sc1\n" \
  "v_mfma_i32_16x16x64_i8 v[208:211], v[96:99], a[72:75], v[208:211]\n" \
  "ds_read_b64_tr_b8 v[124:125], v39 offset:40960\n" \
  "v_mfma_i32_16x16x64_i8 v[208:211], v[100:103], a[88:91], v[208:211]\n" \
  "ds_read_b64_tr_b8 v[126:127], v39 offset:41984\n" \
  "v_mfma_i32_16x16x64_i8 v[240:243], v[96:99], a[76:79], v[240:243]\n" \
  "v_mfma_i32_16x16x64_i8 v[240:243], v[100:103], a[92:95], v[240:243]\n" \
  "buffer_load_dwordx4 a[164:167], v44, s[28:31], s93 offen offset:1024 sc0 sc1\n" \
  "s_waitcnt lgkmcnt(8)\n" \
  "v_mfma_i32_16x16x64_i8 v[148:151], v[104:107], a[64:67], v[148:151]\n" \
  "ds_read_b64_tr_b8 v[96:97], v32 offset:49152\n" \
  "v_mfma_i32_16x16x64_i8 v[148:151], v[108:111], a[80:83], v[148:151]\n" \
  "ds_read_b64_tr_b8 v[98:99], v32 offset:50176\n" \
  "v_mfma_i32_16x16x64_i8 v[180:183], v[104:107], a[68:71], v[180:183]\n" \
  "v_mfma_i32_16x16x64_i8 v[180:183], v[108:111], a[84:87], v[180:183]\n" \
  "buffer_load_dwordx4 a[168:171], v44, s[28:31], s93 offen offset:2048 sc0 sc1\n" \
  "v_mfma_i32_16x16x64_i8 v[212:215], v[104:107], a[72:75], v[212:215]\n" \
  "ds_read_b64_tr_b8 v[100:101], v32 offset:57344\n" \
  "v_mfma_i32_16x16x64_i8 v[212:215], v[108:111], a[88:91], v[212:215]\n" \
  "ds_read_b64_tr_b8 v[102:103], v32 offset:58368\n" \
  "v_mfma_i32_16x16x64_i8 v[244:247], v[104:107], a[76:79], v[244:247]\n" \
  "v_mfma_i32_16x16x64_i8 v[244:247], v[108:111], a[92:95], v[244:247]\n" \
  "buffer_load_dwordx4 a[172:175], v44, s[28:31], s93 offen offset:3072 sc0 sc1\n" \
  "s_waitcnt lgkmcnt(8)\n" \
  "v_mfma_i32_16x16x64_i8 v[152:155], v[112:115], a[64:67], v[152:155]\n" \
  "ds_read_b64_tr_b8 v[104:105], v33 offset:49152\n" \
  "v_mfma_i32_16x16x64_i8 v[152:155], v[116:119], a[80:83], v[152:155]\n" \
  "ds_read_b64_tr_b8 v[106:107], v33 offset:50176\n" \
  "v_mfma_i32_16x16x64_i8 v[184:187], v[112:115], a[68:71], v[184:187]\n" \
  "v_mfma_i32_16x16x64_i8 v[184:187], v[116:119], a[84:87], v[184:187]\n" \
  "buffer_load_dwordx4 a[176:179], v91, s[28:31], s93 offen offset:0 sc0 sc1\n" \
  "v_mfma_i32_16x16x64_i8 v[216:219], v[112:115], a[72:75], v[216:219]\n" \
  "ds_read_b64_tr_b8 v[108:109], v33 offset:57344\n" \
  "v_mfma_i32_16x16x64_i8 v[216:219], v[116:119], a[88:91], v[216:219]\n" \
  "ds_read_b64_tr_b8 v[110:111], v33 offset:58368\n" \
  "v_mfma_i32_16x16x64_i8 v[248:251], v[112:115], a[76:79], v[248:251]\n" \
  "v_mfma_i32_16x16x64_i8 v[248:251], v[116:119], a[92:95], v[248:251]\n" \
  "buffer_load_dwordx4 a[180:183], v91, s[28:31], s93 offen offset:1024 sc0 sc1\n" \
  "s_waitcnt lgkmcnt(8)\n" \
  "v_mfma_i32_16x16x64_i8 v[156:159], v[120:123], a[64:67], v[156:159]\n" \
  "ds_read_b64_tr_b8 v[112:113], v80 offset:0\n" \
  "v_mfma_i32_16x16x64_i8 v[156:159], v[124:127], a[80:83], v[156:159]\n" \
  "ds_read_b64_tr_b8 v[114:115], v80 offset:1024\n" \
  "v_mfma_i32_16x16x64_i8 v[188:191], v[120:123], a[68:71], v[188:191]\n" \
  "v_mfma_i32_16x16x64_i8 v[188:191], v[124:127], a[84:87], v[188:191]\n" \
  "buffer_load_dwordx4 a[184:187], v91, s[28:31], s93 offen offset:2048 sc0 sc1\n" \
  "v_mfma_i32_16x16x64_i8 v[220:223], v[120:123], a[72:75], v[220:223]\n" \
  "ds_read_b64_tr_b8 v[116:117], v80 offset:8192\n" \
  "v_mfma_i32_16x16x64_i8 v[220:223], v[124:127], a[88:91], v[220:223]\n" \
  "ds_read_b64_tr_b8 v[118:119], v80 offset:9216\n" \
  "v_mfma_i32_16x16x64_i8 v[252:255], v[120:123], a[76:79], v[252:255]\n" \
  "v_mfma_i32_16x16x64_i8 v[252:255], v[124:127], a[92:95], v[252:255]\n" \
  "buffer_load_dwordx4 a[188:191], v91, s[28:31], s93 offen offset:3072 sc0 sc1\n" \
  "; ---- step 3\n" \
  "s_waitcnt vmcnt(20)\n" \
  "s_waitcnt lgkmcnt(8)\n" \
  "v_mfma_i32_16x16x64_i8 v[128:131], v[96:99], a[96:99], v[128:131]\n" \
  "ds_read_b64_tr_b8 v[120:121], v81 offset:0\n" \
  "v_mfma_i32_16x16x64_i8 v[128:131], v[100:103], a[112:115], v[128:131]\n" \
  "ds_read_b64_tr_b8 v[122:123], v81 offset:1024\n" \
  "buffer_load_dwordx4 a[192:195], v44, s[28:31], s94 offen offset:0 sc0 sc1\n" \
  "v_mfma_i32_16x16x64_i8 v[160:163], v[96:99], a[100:103], v[160:163]\n" \
  "v_mfma_i32_16x16x64_i8 v[160:163], v[100:103], a[116:119], v[160:163]\n" \
  "buffer_load_dwordx4 a[196:199], v44, s[28:31], s94 offen offset:1024 sc0 sc1\n" \
  "v_mfma_i32_16x16x64_i8 v[192:195], v[96:99], a[104:107], v[192:195]\n" \
  "ds_read_b64_tr_b8 v[124:125], v81 offset:8192\n" \
  "v_mfma_i32_16x16x64_i8 v[192:195], v[100:103], a[120:123], v[192:195]\n" \
  "ds_read_b64_tr_b8 v[126:127], v81 offset:9216\n" \
  "buffer_load_dwordx4 a[200:203], v44, s[28:31], s94 offen offset:2048 sc0 sc1\n" \
  "v_mfma_i32_16x16x64_i8 v[224:227], v[96:99], a[108:111], v[224:227]\n" \
  "v_mfma_i32_16x16x64_i8 v[224:227], v[100:103], a[124:127], v[224:227]\n" \
  "buffer_load_dwordx4 a[204:207], v44, s[28:31], s94 offen offset:3072 sc0 sc1\n" \
  "s_waitcnt lgkmcnt(8)\n" \
  "v_mfma_i32_16x16x64_i8 v[132:135], v[104:107], a[96:99], v[132:135]\n" \
  "ds_read_b64_tr_b8 v[96:97], v80 offset:16384\n" \
  "v_mfma_i32_16x16x64_i8 v[132:135], v[108:111], a[112:115], v[132:135]\n" \
  "ds_read_b64_tr_b8 v[98:99], v80 offset:17408\n" \
  "buffer_load_dwordx4 a[208:211], v91, s[28:31], s94 offen offset:0 sc0 sc1\n" \
  "v_mfma_i32_16x16x64_i8 v[164:167], v[104:107], a[100:103], v[164:167]\n" \
  "v_mfma_i32_16x16x64_i8 v[164:167], v[108:111], a[116:119], v[164:167]\n" \
  "buffer_load_dwordx4 a[212:215], v91, s[28:31], s94 offen offset:1024 sc0 sc1\n" \
  "v_mfma_i32_16x16x64_i8 v[196:199], v[104:107], a[104:107], v[196:199]\n" \
  "ds_read_b64_tr_b8 v[100:101], v80 offset:24576\n" \
  "v_mfma_i32_16x16x64_i8 v[196:199], v[108:111], a[120:123], v[196:199]\n" \
  "ds_read_b64_tr_b8 v[102:103], v80 offset:25600\n" \
  "buffer_load_dwordx4 a[216:219], v91, s[28:31], s94 offen offset:2048 sc0 sc1\n" \
  "v_mfma_i32_16x16x64_i8 v[228:231], v[104:107], a[108:111], v[228:231]\n" \
  "v_mfma_i32_16x16x64_i8 v[228:231], v[108:111], a[124:127], v[228:231]\n" \
  "buffer_load_dwordx4 a[220:223], v91, s[28:31], s94 offen offset:3072 sc0 sc1\n" \
  "s_waitcnt vmcnt(20)\n" \
  "s_waitcnt lgkmcnt(8)\n" \
  "v_mfma_i32_16x16x64_i8 v[128:131], v[112:115], a[128:131], v[128:131]\n" \
  "ds_read_b64_tr_b8 v[104:105], v81 offset:16384\n" \
  "v_mfma_i32_16x16x64_i8 v[128:131], v[116:119], a[144:147], v[128:131]\n" \
  "ds_read_b64_tr_b8 v[106:107], v81 offset:17408\n" \
  "buffer_load_dwordx4 a[224:227], v44, s[28:31], s95 offen offset:0 sc0 sc1\n" \
  "v_mfma_i32_16x16x64_i8 v[160:163], v[112:115], a[132:135], v[160:163]\n" \
  "v_mfma_i32_16x16x64_i8 v[160:163], v[116:119], a[148:151], v[160:163]\n" \
  "buffer_load_dwordx4 a[228:231], v44, s[28:31], s95 offen offset:1024 sc0 sc1\n" \
  "v_mfma_i32_16x16x64_i8 v[192:195], v[112:115], a[136:139], v[192:195]\n" \
  "ds_read_b64_tr_b8 v[108:109], v81 offset:24576\n" \
  "v_mfma_i32_16x16x64_i8 v[192:195], v[116:119], a[152:155], v[192:195]\n" \
  "ds_read_b64_tr_b8 v[110:111], v81 offset:25600\n" \
  "buffer_load_dwordx4 a[232:235], v44, s[28:31], s95 offen offset:2048 sc0 sc1\n" \
  "v_mfma_i32_16x16x64_i8 v[224:227], v[112:115], a[140:143], v[224:227]\n" \
  "v_mfma_i32_16x16x64_i8 v[224:227], v[116:119], a[156:159], v[224:227]\n" \
  "buffer_load_dwordx4 a[236:239], v44, s[28:31], s95 offen offset:3072 sc0 sc1\n" \
  "s_waitcnt lgkmcnt(8)\n" \
  "v_mfma_i32_16x16x64_i8 v[132:135], v[120:123], a[128:131], v[132:135]\n" \
  "ds_read_b64_tr_b8 v[112:113], v80 offset:32768\n" \
  "v_mfma_i32_16x16x64_i8 v[132:135], v[124:127], a[144:147], v[132:135]\n" \
  "ds_read_b64_tr_b8 v[114:115], v80 offset:33792\n" \
  "buffer_load_dwordx4 a[240:243], v91, s[28:31], s95 offen offset:0 sc0 sc1\n" \
  "v_mfma_i32_16x16x64_i8 v[164:167], v[120:123], a[132:135], v[164:167]\n" \
  "v_mfma_i32_16x16x64_i8 v[164:167], v[124:127], a[148:151], v[164:167]\n" \
  "buffer_load_dwordx4 a[244:247], v91, s[28:31], s95 offen offset:1024 sc0 sc1\n" \
  "v_mfma_i32_16x16x64_i8 v[196:199], v[120:123], a[136:139], v[196:199]\n" \
  "ds_read_b64_tr_b8 v[116:117], v80 offset:40960\n" \
  "v_mfma_i32_16x16x64_i8 v[196:199], v[124:127], a[152:155], v[196:199]\n" \
  "ds_read_b64_tr_b8 v[118:119], v80 offset:41984\n" \
  "buffer_load_dwordx4 a[248:251], v91, s[28:31], s95 offen offset:2048 sc0 sc1\n" \
  "v_mfma_i32_16x16x64_i8 v[228:231], v[120:123], a[140:143], v[228:231]\n" \
  "v_mfma_i32_16x16x64_i8 v[228:231], v[124:127], a[156:159], v[228:231]\n" \
  "buffer_load_dwordx4 a[252:255], v91, s[28:31], s95 offen offset:3072 sc0 sc1\n" \
  "s_waitcnt vmcnt(16)\n" \
  "s_waitcnt lgkmcnt(8)\n" \
  "v_mfma_i32_16x16x64_i8 v[128:131], v[96:99], a[160:163], v[128:131]\n" \
  "ds_read_b64_tr_b8 v[120:121], v81 offset:32768\n" \
  "v_mfma_i32_16x16x64_i8 v[128:131], v[100:103], a[176:179], v[128:131]\n" \
  "ds_read_b64_tr_b8 v[122:123], v81 offset:33792\n" \
  "v_mfma_i32_16x16x64_i8 v[160:163], v[96:99], a[164:167], v[160:163]\n" \
  "v_mfma_i32_16x16x64_i8 v[160:163], v[100:103], a[180:183], v[160:163]\n" \
  "v_mfma_i32_16x16x64_i8 v[192:195], v[96:99], a[168:171], v[192:195]\n" \
  "ds_read_b64_tr_b8 v[124:125], v81 offset:40960\n" \
  "v_mfma_i32_16x16x64_i8 v[192:195], v[100:103], a[184:187], v[192:195]\n" \
  "ds_read_b64_tr_b8 v[126:127], v81 offset:41984\n" \
  "v_mfma_i32_16x16x64_i8 v[224:227], v[96:99], a[172:175], v[224:227]\n" \
  "v_mfma_i32_16x16x64_i8 v[224:227], v[100:103], a[188:191], v[224:227]\n" \
  "s_waitcnt vmcnt(24)\n" \
  "s_barrier\n" \
  "s_waitcnt lgkmcnt(8)\n" \
  "v_mfma_i32_16x16x64_i8 v[132:135], v[104:107], a[160:163], v[132:135]\n" \
  "ds_read_b64_tr_b8 v[96:97], v80 offset:49152\n" \
  "v_mfma_i32_16x16x64_i8 v[132:135], v[108:111], a[176:179], v[132:135]\n" \
  "ds_read_b64_tr_b8 v[98:99], v80 offset:50176\n" \
  "v_mfma_i32_16x16x64_i8 v[164:167], v[104:107], a[164:167], v[164:167]\n" \
  "v_mfma_i32_16x16x64_i8 v[164:167], v[108:111], a[180:183], v[164:167]\n" \
  "v_mfma_i32_16x16x64_i8 v[196:199], v[104:107], a[168:171], v[196:199]\n" \
  "ds_read_b64_tr_b8 v[100:101], v80 offset:57344\n" \
  "v_mfma_i32_16x16x64_i8 v[196:199], v[108:111], a[184:187], v[196:199]\n" \
  "ds_read_b64_tr_b8 v[102:103], v80 offset:58368\n" \
  "v_mfma_i32_16x16x64_i8 v[228:231], v[104:107], a[172:175], v[228:231]\n" \
  "v_mfma_i32_16x16x64_i8 v[228:231], v[108:111], a[188:191], v[228:231]\n" \
  "s_waitcnt vmcnt(8)\n" \
  "s_waitcnt lgkmcnt(8)\n" \
  "v_mfma_i32_16x16x64_i8 v[128:131], v[112:115], a[192:195], v[128:131]\n" \
  "ds_read_b64_tr_b8 v[104:105], v81 offset:49152\n" \
  "v_mfma_i32_16x16x64_i8 v[128:131], v[116:119], a[208:211], v[128:131]\n" \
  "ds_read_b64_tr_b8 v[106:107], v81 offset:50176\n" \
  "v_mfma_i32_16x16x64_i8 v[160:163], v[112:115], a[196:199], v[160:163]\n" \
  "v_mfma_i32_16x16x64_i8 v[160:163], v[116:119], a[212:215], v[160:163]\n" \
  "v_mfma_i32_16x16x64_i8 v[192:195], v[112:115], a[200:203], v[192:195]\n" \
  "ds_read_b64_tr_b8 v[108:109], v81 offset:57344\n" \
  "v_mfma_i32_16x16x64_i8 v[192:195], v[116:119], a[216:219], v[192:195]\n" \
  "ds_read_b64_tr_b8 v[110:111], v81 offset:58368\n" \
  "v_mfma_i32_16x16x64_i8 v[224:227], v[112:115], a[204:207], v[224:227]\n" \
  "v_mfma_i32_16x16x64_i8 v[224:227], v[116:119], a[220:223], v[224:227]\n" \
  "s_waitcnt lgkmcnt(8)\n" \
  "v_mfma_i32_16x16x64_i8 v[132:135], v[120:123], a[192:195], v[132:135]\n" \
  "ds_read_b64_tr_b8 v[112:113], v34 offset:49152\n" \
  "v_mfma_i32_16x16x64_i8 v[132:135], v[124:127], a[208:211], v[132:135]\n" \
  "ds_read_b64_tr_b8 v[114:115], v34 offset:50176\n" \
  "v_mfma_i32_16x16x64_i8 v[164:167], v[120:123], a[196:199], v[164:167]\n" \
  "v_mfma_i32_16x16x64_i8 v[164:167], v[124:127], a[212:215], v[164:167]\n" \
  "v_mfma_i32_16x16x64_i8 v[196:199], v[120:123], a[200:203], v[196:199]\n" \
  "ds_read_b64_tr_b8 v[116:117], v34 offset:57344\n" \
  "v_mfma_i32_16x16x64_i8 v[196:199], v[124:127], a[216:219], v[196:199]\n" \
  "ds_read_b64_tr_b8 v[118:119], v34 offset:58368\n" \
  "v_mfma_i32_16x16x64_i8 v[228:231], v[120:123], a[204:207], v[228:231]\n" \
  "v_mfma_i32_16x16x64_i8 v[228:231], v[124:127], a[220:223], v[228:231]\n" \
  "; ---- step 4\n" \
  "s_waitcnt vmcnt(0)\n" \
  "s_waitcnt lgkmcnt(8)\n" \
  "v_mfma_i32_16x16x64_i8 v[128:131], v[96:99], a[224:227], v[128:131]\n" \
  "ds_read_b64_tr_b8 v[120:121], v35 offset:49152\n" \
  "v_mfma_i32_16x16x64_i8 v[128:131], v[100:103], a[240:243], v[128:131]\n" \
  "ds_read_b64_tr_b8 v[122:123], v35 offset:50176\n" \
  "v_mfma_i32_16x16x64_i8 v[160:163], v[96:99], a[228:231], v[160:163]\n" \
  "v_mfma_i32_16x16x64_i8 v[160:163], v[100:103], a[244:247], v[160:163]\n" \
  "v_mfma_i32_16x16x64_i8 v[192:195], v[96:99], a[232:235], v[192:195]\n" \
  "ds_read_b64_tr_b8 v[124:125], v35 offset:57344\n" \
  "v_mfma_i32_16x16x64_i8 v[192:195], v[100:103], a[248:251], v[192:195]\n" \
  "ds_read_b64_tr_b8 v[126:127], v35 offset:58368\n" \
  "v_mfma_i32_16x16x64_i8 v[224:227], v[96:99], a[236:239], v[224:227]\n" \
  "v_mfma_i32_16x16x64_i8 v[224:227], v[100:103], a[252:255], v[224:227]\n" \
  "s_waitcnt lgkmcnt(8)\n" \
  "v_mfma_i32_16x16x64_i8 v[132:135], v[104:107], a[224:227], v[132:135]\n" \
  "ds_read_b64_tr_b8 v[96:97], v82 offset:0\n" \
  "v_mfma_i32_16x16x64_i8 v[132:135], v[108:111], a[240:243], v[132:135]\n" \
  "ds_read_b64_tr_b8 v[98:99], v82 offset:1024\n" \
  "v_mfma_i32_16x16x64_i8 v[164:167], v[104:107], a[228:231], v[164:167]\n" \
  "v_mfma_i32_16x16x64_i8 v[164:167], v[108:111], a[244:247], v[164:167]\n" \
  "v_mfma_i32_16x16x64_i8 v[196:199], v[104:107], a[232:235], v[196:199]\n" \
  "ds_read_b64_tr_b8 v[100:101], v82 offset:8192\n" \
  "v_mfma_i32_16x16x64_i8 v[196:199], v[108:111], a[248:251], v[196:199]\n" \
  "ds_read_b64_tr_b8 v[102:103], v82 offset:9216\n" \
  "v_mfma_i32_16x16x64_i8 v[228:231], v[104:107], a[236:239], v[228:231]\n" \
  "v_mfma_i32_16x16x64_i8 v[228:231], v[108:111], a[252:255], v[228:231]\n" \
  "s_waitcnt lgkmcnt(8)\n" \
  "v_mfma_i32_16x16x64_i8 v[136:139], v[112:115], a[96:99], v[136:139]\n" \
  "ds_read_b64_tr_b8 v[104:105], v83 offset:0\n" \
  "v_mfma_i32_16x16x64_i8 v[136:139], v[116:119], a[112:115], v[136:139]\n" \
  "ds_read_b64_tr_b8 v[106:107], v83 offset:1024\n" \
  "v_mfma_i32_16x16x64_i8 v[168:171], v[112:115], a[100:103], v[168:171]\n" \
  "v_mfma_i32_16x16x64_i8 v[168:171], v[116:119], a[116:119], v[168:171]\n" \
  "v_mfma_i32_16x16x64_i8 v[200:203], v[112:115], a[104:107], v[200:203]\n" \
  "ds_read_b64_tr_b8 v[108:109], v83 offset:8192\n" \
  "v_mfma_i32_16x16x64_i8 v[200:203], v[116:119], a[120:123], v[200:203]\n" \
  "ds_read_b64_tr_b8 v[110:111], v83 offset:9216\n" \
  "v_mfma_i32_16x16x64_i8 v[232:235], v[112:115], a[108:111], v[232:235]\n" \
  "v_mfma_i32_16x16x64_i8 v[232:235], v[116:119], a[124:127], v[232:235]\n" \
  "s_waitcnt lgkmcnt(8)\n" \
  "v_mfma_i32_16x16x64_i8 v[140:143], v[120:123], a[96:99], v[140:143]\n" \
  "ds_read_b64_tr_b8 v[112:113], v82 offset:16384\n" \
  "v_cvt_f32_i32 v128, v128\n" \
  "v_mfma_i32_16x16x64_i8 v[140:143], v[124:127], a[112:115], v[140:143]\n" \
  "ds_read_b64_tr_b8 v[114:115], v82 offset:17408\n" \
  "v_cvt_f32_i32 v129, v129\n" \
  "v_mfma_i32_16x16x64_i8 v[172:175], v[120:123], a[100:103], v[172:175]\n" \
  "v_cvt_f32_i32 v130, v130\n" \
  "v_mfma_i32_16x16x64_i8 v[172:175], v[124:127], a[116:119], v[172:175]\n" \
  "v_cvt_f32_i32 v131, v131\n" \
  "v_mfma_i32_16x16x64_i8 v[204:207], v[120:123], a[104:107], v[204:207]\n" \
  "ds_read_b64_tr_b8 v[116:117], v82 offset:24576\n" \
  "v_mfma_i32_16x16x64_i8 v[204:207], v[124:127], a[120:123], v[204:207]\n" \
  "ds_read_b64_tr_b8 v[118:119], v82 offset:25600\n" \
  "v_pk_mul_f32 v[128:129], v[128:129], v[88:89]\n" \
  "v_mfma_i32_16x16x64_i8 v[236:239], v[120:123], a[108:111], v[236:239]\n" \
  "v_pk_mul_f32 v[130:131], v[130:131], v[88:89]\n" \
  "v_mfma_i32_16x16x64_i8 v[236:239], v[124:127], a[124:127], v[236:239]\n" \
  "buffer_store_dwordx4 v[128:131], v47, s[20:23], s76 offen offset:0 sc0 sc1\n" \
  "s_waitcnt lgkmcnt(8)\n" \
  "v_mfma_i32_16x16x64_i8 v[136:139], v[96:99], a[128:131], v[136:139]\n" \
  "ds_read_b64_tr_b8 v[120:121], v83 offset:16384\n" \
  "v_mfma_i32_16x16x64_i8 v[136:139], v[100:103], a[144:147], v[136:139]\n" \
  "ds_read_b64_tr_b8 v[122:123], v83 offset:17408\n" \
  "v_cvt_f32_i32 v132, v132\n" \
  "v_mfma_i32_16x16x64_i8 v[168:171], v[96:99], a[132:135], v[168:171]\n" \
  "v_cvt_f32_i32 v133, v133\n" \
  "v_mfma_i32_16x16x64_i8 v[168:171], v[100:103], a[148:151], v[168:171]\n" \
  "v_cvt_f32_i32 v134, v134\n" \
  "v_mfma_i32_16x16x64_i8 v[200:203], v[96:99], a[136:139], v[200:203]\n" \
  "ds_read_b64_tr_b8 v[124:125], v83 offset:24576\n" \
  "v_cvt_f32_i32 v135, v135\n" \
  "v_mfma_i32_16x16x64_i8 v[200:203], v[100:103], a[152:155], v[200:203]\n" \
  "ds_read_b64_tr_b8 v[126:127], v83 offset:25600\n" \
  "v_mfma_i32_16x16x64_i8 v[232:235], v[96:99], a[140:143], v[232:235]\n" \
  "v_pk_mul_f32 v[132:133], v[132:133], v[88:89]\n" \
  "v_mfma_i32_16x16x64_i8 v[232:235], v[100:103], a[156:159], v[232:235]\n" \
  "v_pk_mul_f32 v[134:135], v[134:135], v[88:89]\n" \
  "s_waitcnt lgkmcnt(8)\n" \
  "v_mfma_i32_16x16x64_i8 v[140:143], v[104:107], a[128:131], v[140:143]\n" \
  "ds_read_b64_tr_b8 v[96:97], v82 offset:32768\n" \
  "buffer_store_dwordx4 v[132:135], v47, s[20:23], s76 offen offset:64 sc0 sc1\n" \
  "v_mfma_i32_16x16x64_i8 v[140:143], v[108:111], a[144:147], v[140:143]\n" \
  "ds_read_b64_tr_b8 v[98:99], v82 offset:33792\n" \
  "v_mfma_i32_16x16x64_i8 v[172:175], v[104:107], a[132:135], v[172:175]\n" \
  "v_cvt_f32_i32 v160, v160\n" \
  "v_mfma_i32_16x16x64_i8 v[172:175], v[108:111], a[148:151], v[172:175]\n" \
  "v_cvt_f32_i32 v161, v161\n" \
  "v_mfma_i32_16x16x64_i8 v[204:207], v[104:107], a[136:139], v[204:207]\n" \
  "ds_read_b64_tr_b8 v[100:101], v82 offset:40960\n" \
  "v_cvt_f32_i32 v162, v162\n" \
  "v_mfma_i32_16x16x64_i8 v[204:207], v[108:111], a[152:155], v[204:207]\n" \
  "ds_read_b64_tr_b8 v[102:103], v82 offset:41984\n" \
  "v_cvt_f32_i32 v163, v163\n" \
  "v_mfma_i32_16x16x64_i8 v[236:239], v[104:107], a[140:143], v[236:239]\n" \
  "v_mfma_i32_16x16x64_i8 v[236:239], v[108:111], a[156:159], v[236:239]\n" \
  "v_pk_mul_f32 v[160:161], v[160:161], v[88:89]\n" \
  "s_waitcnt lgkmcnt(8)\n" \
  "v_mfma_i32_16x16x64_i8 v[136:139], v[112:115], a[160:163], v[136:139]\n" \
  "ds_read_b64_tr_b8 v[104:105], v83 offset:32768\n" \
  "v_pk_mul_f32 v[162:163], v[162:163], v[88:89]\n" \
  "v_mfma_i32_16x16x64_i8 v[136:139], v[116:119], a[176:179], v[136:139]\n" \
  "ds_read_b64_tr_b8 v[106:107], v83 offset:33792\n" \
  "buffer_store_dwordx4 v[160:163], v47, s[20:23], s77 offen offset:0 sc0 sc1\n" \
  "v_mfma_i32_16x16x64_i8 v[168:171], v[112:115], a[164:167], v[168:171]\n" \
  "v_mfma_i32_16x16x64_i8 v[168:171], v[116:119], a[180:183], v[168:171]\n" \
  "v_cvt_f32_i32 v164, v164\n" \
  "v_mfma_i32_16x16x64_i8 v[200:203], v[112:115], a[168:171], v[200:203]\n" \
  "ds_read_b64_tr_b8 v[108:109], v83 offset:40960\n" \
  "v_cvt_f32_i32 v165, v165\n" \
  "v_mfma_i32_16x16x64_i8 v[200:203], v[116:119], a[184:187], v[200:203]\n" \
  "ds_read_b64_tr_b8 v[110:111], v83 offset:41984\n" \
  "v_cvt_f32_i32 v166, v166\n" \
  "v_mfma_i32_16x16x64_i8 v[232:235], v[112:115], a[172:175], v[232:235]\n" \
  "v_cvt_f32_i32 v167, v167\n" \
  "v_mfma_i32_16x16x64_i8 v[232:235], v[116:119], a[188:191], v[232:235]\n" \
  "s_waitcnt lgkmcnt(8)\n" \
  "v_mfma_i32_16x16x64_i8 v[140:143], v[120:123], a[160:163], v[140:143]\n" \
  "ds_read_b64_tr_b8 v[112:113], v82 offset:49152\n" \
  "v_pk_mul_f32 v[164:165], v[164:165], v[88:89]\n" \
  "v_mfma_i32_16x16x64_i8 v[140:143], v[124:127], a[176:179], v[140:143]\n" \
  "ds_read_b64_tr_b8 v[114:115], v82 offset:50176\n" \
  "v_pk_mul_f32 v[166:167], v[166:167], v[88:89]\n" \
  "v_mfma_i32_16x16x64_i8 v[172:175], v[120:123], a[164:167], v[172:175]\n" \
  "buffer_store_dwordx4 v[164:167], v47, s[20:23], s77 offen offset:64 sc0 sc1\n" \
  "v_mfma_i32_16x16x64_i8 v[172:175], v[124:127], a[180:183], v[172:175]\n" \
  "v_mfma_i32_16x16x64_i8 v[204:207], v[120:123], a[168:171], v[204:207]\n" \
  "ds_read_b64_tr_b8 v[116:117], v82 offset:57344\n" \
  "v_cvt_f32_i32 v192, v192\n" \
  "v_mfma_i32_16x16x64_i8 v[204:207], v[124:127], a[184:187], v[204:207]\n" \
  "ds_read_b64_tr_b8 v[118:119], v82 offset:58368\n" \
  "v_cvt_f32_i32 v193, v193\n" \
  "v_mfma_i32_16x16x64_i8 v[236:239], v[120:123], a[172:175], v[236:239]\n" \
  "v_cvt_f32_i32 v194, v194\n" \
  "v_mfma_i32_16x16x64_i8 v[236:239], v[124:127], a[188:191], v[236:239]\n" \
  "v_cvt_f32_i32 v195, v195\n" \
  "; ---- step 5\n" \
  "s_waitcnt lgkmcnt(8)\n" \
  "v_mfma_i32_16x16x64_i8 v[136:139], v[96:99], a[192:195], v[136:139]\n" \
  "ds_read_b64_tr_b8 v[120:121], v83 offset:49152\n" \
  "v_mfma_i32_16x16x64_i8 v[136:139], v[100:103], a[208:211], v[136:139]\n" \
  "ds_read_b64_tr_b8 v[122:123], v83 offset:50176\n" \
  "v_pk_mul_f32 v[192:193], v[192:193], v[88:89]\n" \
  "v_mfma_i32_16x16x64_i8 v[168:171], v[96:99], a[196:199], v[168:171]\n" \
  "v_pk_mul_f32 v[194:195], v[194:195], v[88:89]\n" \
  "v_mfma_i32_16x16x64_i8 v[168:171], v[100:103], a[212:215], v[168:171]\n" \
  "buffer_store_dwordx4 v[192:195], v47, s[20:23], s78 offen offset:0 sc0 sc1\n" \
  "v_mfma_i32_16x16x64_i8 v[200:203], v[96:99], a[200:203], v[200:203]\n" \
  "ds_read_b64_tr_b8 v[124:125], v83 offset:57344\n" \
  "v_mfma_i32_16x16x64_i8 v[200:203], v[100:103], a[216:219], v[200:203]\n" \
  "ds_read_b64_tr_b8 v[126:127], v83 offset:58368\n" \
  "v_cvt_f32_i32 v196, v196\n" \
  "v_mfma_i32_16x16x64_i8 v[232:235], v[96:99], a[204:207], v[232:235]\n" \
  "v_cvt_f32_i32 v197, v197\n" \
  "v_mfma_i32_16x16x64_i8 v[232:235], v[100:103], a[220:223], v[232:235]\n" \
  "v_cvt_f32_i32 v198, v198\n" \
  "s_waitcnt lgkmcnt(8)\n" \
  "v_mfma_i32_16x16x64_i8 v[140:143], v[104:107], a[192:195], v[140:143]\n" \
  "ds_read_b64_tr_b8 v[96:97], v36 offset:49152\n" \
  "v_cvt_f32_i32 v199, v199\n" \
  "v_mfma_i32_16x16x64_i8 v[140:143], v[108:111], a[208:211], v[140:143]\n" \
  "ds_read_b64_tr_b8 v[98:99], v36 offset:50176\n" \
  "v_mfma_i32_16x16x64_i8 v[172:175], v[104:107], a[196:199], v[172:175]\n" \
  "v_pk_mul_f32 v[196:197], v[196:197], v[88:89]\n" \
  "v_mfma_i32_16x16x64_i8 v[172:175], v[108:111], a[212:215], v[172:175]\n" \
  "v_pk_mul_f32 v[198:199], v[198:199], v[88:89]\n" \
  "v_mfma_i32_16x16x64_i8 v[204:207], v[104:107], a[200:203], v[204:207]\n" \
  "ds_read_b64_tr_b8 v[100:101], v36 offset:57344\n" \
  "buffer_store_dwordx4 v[196:199], v47, s[20:23], s78 offen offset:64 sc0 sc1\n" \
  "v_mfma_i32_16x16x64_i8 v[204:207], v[108:111], a[216:219], v[204:207]\n" \
  "ds_read_b64_tr_b8 v[102:103], v36 offset:58368\n" \
  "v_mfma_i32_16x16x64_i8 v[236:239], v[104:107], a[204:207], v[236:239]\n" \
  "v_cvt_f32_i32 v224, v224\n" \
  "v_mfma_i32_16x16x64_i8 v[236:239], v[108:111], a[220:223], v[236:239]\n" \
  "v_cvt_f32_i32 v225, v225\n" \
  "s_waitcnt lgkmcnt(8)\n" \
  "v_mfma_i32_16x16x64_i8 v[136:139], v[112:115], a[224:227], v[136:139]\n" \
  "ds_read_b64_tr_b8 v[104:105], v37 offset:49152\n" \
  "v_cvt_f32_i32 v226, v226\n" \
  "v_mfma_i32_16x16x64_i8 v[136:139], v[116:119], a[240:243], v[136:139]\n" \
  "ds_read_b64_tr_b8 v[106:107], v37 offset:50176\n" \
  "v_cvt_f32_i32 v227, v227\n" \
  "v_mfma_i32_16x16x64_i8 v[168:171], v[112:115], a[228:231], v[168:171]\n" \
  "v_mfma_i32_16x16x64_i8 v[168:171], v[116:119], a[244:247], v[168:171]\n" \
  "v_pk_mul_f32 v[224:225], v[224:225], v[88:89]\n" \
  "v_mfma_i32_16x16x64_i8 v[200:203], v[112:115], a[232:235], v[200:203]\n" \
  "ds_read_b64_tr_b8 v[108:109], v37 offset:57344\n" \
  "v_pk_mul_f32 v[226:227], v[226:227], v[88:89]\n" \
  "v_mfma_i32_16x16x64_i8 v[200:203], v[116:119], a[248:251], v[200:203]\n" \
  "ds_read_b64_tr_b8 v[110:111], v37 offset:58368\n" \
  "buffer_store_dwordx4 v[224:227], v47, s[20:23], s79 offen offset:0 sc0 sc1\n" \
  "v_mfma_i32_16x16x64_i8 v[232:235], v[112:115], a[236:239], v[232:235]\n" \
  "v_mfma_i32_16x16x64_i8 v[232:235], v[116:119], a[252:255], v[232:235]\n" \
  "v_cvt_f32_i32 v228, v228\n" \
  "s_waitcnt lgkmcnt(8)\n" \
  "v_mfma_i32_16x16x64_i8 v[140:143], v[120:123], a[224:227], v[140:143]\n" \
  "ds_read_b64_tr_b8 v[112:113], v84 offset:0\n" \
  "v_cvt_f32_i32 v229, v229\n" \
  "v_mfma_i32_16x16x64_i8 v[140:143], v[124:127], a[240:243], v[140:143]\n" \
  "ds_read_b64_tr_b8 v[114:115], v84 offset:1024\n" \
  "v_cvt_f32_i32 v230, v230\n" \
  "v_mfma_i32_16x16x64_i8 v[172:175], v[120:123], a[228:231], v[172:175]\n" \
  "v_cvt_f32_i32 v231, v231\n" \
  "v_mfma_i32_16x16x64_i8 v[172:175], v[124:127], a[244:247], v[172:175]\n" \
  "v_mfma_i32_16x16x64_i8 v[204:207], v[120:123], a[232:235], v[204:207]\n" \
  "ds_read_b64_tr_b8 v[116:117], v84 offset:8192\n" \
  "v_pk_mul_f32 v[228:229], v[228:229], v[88:89]\n" \
  "v_mfma_i32_16x16x64_i8 v[204:207], v[124:127], a[248:251], v[204:207]\n" \
  "ds_read_b64_tr_b8 v[118:119], v84 offset:9216\n" \
  "v_pk_mul_f32 v[230:231], v[230:231], v[88:89]\n" \
  "v_mfma_i32_16x16x64_i8 v[236:239], v[120:123], a[236:239], v[236:239]\n" \
  "buffer_store_dwordx4 v[228:231], v47, s[20:23], s79 offen offset:64 sc0 sc1\n" \
  "v_mfma_i32_16x16x64_i8 v[236:239], v[124:127], a[252:255], v[236:239]\n" \
  "s_waitcnt lgkmcnt(8)\n" \
  "v_mfma_i32_16x16x64_i8 v[144:147], v[96:99], a[96:99], v[144:147]\n" \
  "ds_read_b64_tr_b8 v[120:121], v85 offset:0\n" \
  "v_mfma_i32_16x16x64_i8 v[144:147], v[100:103], a[112:115], v[144:147]\n" \
  "ds_read_b64_tr_b8 v[122:123], v85 offset:1024\n" \
  "v_mfma_i32_16x16x64_i8 v[176:179], v[96:99], a[100:103], v[176:179]\n" \
  "v_mfma_i32_16x16x64_i8 v[176:179], v[100:103], a[116:119], v[176:179]\n" \
  "v_mfma_i32_16x16x64_i8 v[208:211], v[96:99], a[104:107], v[208:211]\n" \
  "ds_read_b64_tr_b8 v[124:125], v85 offset:8192\n" \
  "v_mfma_i32_16x16x64_i8 v[208:211], v[100:103], a[120:123], v[208:211]\n" \
  "ds_read_b64_tr_b8 v[126:127], v85 offset:9216\n" \
  "v_mfma_i32_16x16x64_i8 v[240:243], v[96:99], a[108:111], v[240:243]\n" \
  "v_mfma_i32_16x16x64_i8 v[240:243], v[100:103], a[124:127], v[240:243]\n" \
  "s_waitcnt lgkmcnt(8)\n" \
  "v_mfma_i32_16x16x64_i8 v[148:151], v[104:107], a[96:99], v[148:151]\n" \
  "ds_read_b64_tr_b8 v[96:97], v84 offset:16384\n" \
  "v_cvt_f32_i32 v136, v136\n" \
  "v_mfma_i32_16x16x64_i8 v[148:151], v[108:111], a[112:115], v[148:151]\n" \
  "ds_read_b64_tr_b8 v[98:99], v84 offset:17408\n" \
  "v_cvt_f32_i32 v137, v137\n" \
  "v_mfma_i32_16x16x64_i8 v[180:183], v[104:107], a[100:103], v[180:183]\n" \
  "v_cvt_f32_i32 v138, v138\n" \
  "v_mfma_i32_16x16x64_i8 v[180:183], v[108:111], a[116:119], v[180:183]\n" \
  "v_cvt_f32_i32 v139, v139\n" \
  "v_mfma_i32_16x16x64_i8 v[212:215], v[104:107], a[104:107], v[212:215]\n" \
  "ds_read_b64_tr_b8 v[100:101], v84 offset:24576\n" \
  "v_mfma_i32_16x16x64_i8 v[212:215], v[108:111], a[120:123], v[212:215]\n" \
  "ds_read_b64_tr_b8 v[102:103], v84 offset:25600\n" \
  "v_pk_mul_f32 v[136:137], v[136:137], v[88:89]\n" \
  "v_mfma_i32_16x16x64_i8 v[244:247], v[104:107], a[108:111], v[244:247]\n" \
  "v_pk_mul_f32 v[138:139], v[138:139], v[88:89]\n" \
  "v_mfma_i32_16x16x64_i8 v[244:247], v[108:111], a[124:127], v[244:247]\n" \
  "buffer_store_dwordx4 v[136:139], v47, s[20:23], s76 offen offset:128 sc0 sc1\n" \
  "s_waitcnt lgkmcnt(8)\n" \
  "v_mfma_i32_16x16x64_i8 v[144:147], v[112:115], a[128:131], v[144:147]\n" \
  "ds_read_b64_tr_b8 v[104:105], v85 offset:16384\n" \
  "v_mfma_i32_16x16x64_i8 v[144:147], v[116:119], a[144:147], v[144:147]\n" \
  "ds_read_b64_tr_b8 v[106:107], v85 offset:17408\n" \
  "v_cvt_f32_i32 v140, v140\n" \
  "v_mfma_i32_16x16x64_i8 v[176:179], v[112:115], a[132:135], v[176:179]\n" \
  "v_cvt_f32_i32 v141, v141\n" \
  "v_mfma_i32_16x16x64_i8 v[176:179], v[116:119], a[148:151], v[176:179]\n" \
  "v_cvt_f32_i32 v142, v142\n" \
  "v_mfma_i32_16x16x64_i8 v[208:211], v[112:115], a[136:139], v[208:211]\n" \
  "ds_read_b64_tr_b8 v[108:109], v85 offset:24576\n" \
  "v_cvt_f32_i32 v143, v143\n" \
  "v_mfma_i32_16x16x64_i8 v[208:211], v[116:119], a[152:155], v[208:211]\n" \
  "ds_read_b64_tr_b8 v[110:111], v85 offset:25600\n" \
  "v_mfma_i32_16x16x64_i8 v[240:243], v[112:115], a[140:143], v[240:243]\n" \
  "v_pk_mul_f32 v[140:141], v[140:141], v[88:89]\n" \
  "v_mfma_i32_16x16x64_i8 v[240:243], v[116:119], a[156:159], v[240:243]\n" \
  "v_pk_mul_f32 v[142:143], v[142:143], v[88:89]\n" \
  "s_waitcnt lgkmcnt(8)\n" \
  "v_mfma_i32_16x16x64_i8 v[148:151], v[120:123], a[128:131], v[148:151]\n" \
  "ds_read_b64_tr_b8 v[112:113], v84 offset:32768\n" \
  "buffer_store_dwordx4 v[140:143], v47, s[20:23], s76 offen offset:192 sc0 sc1\n" \
  "v_mfma_i32_16x16x64_i8 v[148:151], v[124:127], a[144:147], v[148:151]\n" \
  "ds_read_b64_tr_b8 v[114:115], v84 offset:33792\n" \
  "v_mfma_i32_16x16x64_i8 v[180:183], v[120:123], a[132:135], v[180:183]\n" \
  "v_cvt_f32_i32 v168, v168\n" \
  "v_mfma_i32_16x16x64_i8 v[180:183], v[124:127], a[148:151], v[180:183]\n" \
  "v_cvt_f32_i32 v169, v169\n" \
  "v_mfma_i32_16x16x64_i8 v[212:215], v[120:123], a[136:139], v[212:215]\n" \
  "ds_read_b64_tr_b8 v[116:117], v84 offset:40960\n" \
  "v_cvt_f32_i32 v170, v170\n" \
  "v_mfma_i32_16x16x64_i8 v[212:215], v[124:127], a[152:155], v[212:215]\n" \
  "ds_read_b64_tr_b8 v[118:119], v84 offset:41984\n" \
  "v_cvt_f32_i32 v171, v171\n" \
  "v_mfma_i32_16x16x64_i8 v[244:247], v[120:123], a[140:143], v[244:247]\n" \
  "v_mfma_i32_16x16x64_i8 v[244:247], v[124:127], a[156:159], v[244:247]\n" \
  "v_pk_mul_f32 v[168:169], v[168:169], v[88:89]\n" \
  "; ---- step 6\n" \
  "s_waitcnt lgkmcnt(8)\n" \
  "v_mfma_i32_16x16x64_i8 v[144:147], v[96:99], a[160:163], v[144:147]\n" \
  "ds_read_b64_tr_b8 v[120:121], v85 offset:32768\n" \
  "v_pk_mul_f32 v[170:171], v[170:171], v[88:89]\n" \
  "v_mfma_i32_16x16x64_i8 v[144:147], v[100:103], a[176:179], v[144:147]\n" \
  "ds_read_b64_tr_b8 v[122:123], v85 offset:33792\n" \
  "buffer_store_dwordx4 v[168:171], v47, s[20:23], s77 offen offset:128 sc0 sc1\n" \
  "v_mfma_i32_16x16x64_i8 v[176:179], v[96:99], a[164:167], v[176:179]\n" \
  "v_mfma_i32_16x16x64_i8 v[176:179], v[100:103], a[180:183], v[176:179]\n" \
  "v_cvt_f32_i32 v172, v172\n" \
  "v_mfma_i32_16x16x64_i8 v[208:211], v[96:99], a[168:171], v[208:211]\n" \
  "ds_read_b64_tr_b8 v[124:125], v85 offset:40960\n" \
  "v_cvt_f32_i32 v173, v173\n" \
  "v_mfma_i32_16x16x64_i8 v[208:211], v[100:103], a[184:187], v[208:211]\n" \
  "ds_read_b64_tr_b8 v[126:127], v85 offset:41984\n" \
  "v_cvt_f32_i32 v174, v174\n" \
  "v_mfma_i32_16x16x64_i8 v[240:243], v[96:99], a[172:175], v[240:243]\n" \
  "v_cvt_f32_i32 v175, v175\n" \
  "v_mfma_i32_16x16x64_i8 v[240:243], v[100:103], a[188:191], v[240:243]\n" \
  "s_waitcnt lgkmcnt(8)\n" \
  "v_mfma_i32_16x16x64_i8 v[148:151], v[104:107], a[160:163], v[148:151]\n" \
  "ds_read_b64_tr_b8 v[96:97], v84 offset:49152\n" \
  "v_pk_mul_f32 v[172:173], v[172:173], v[88:89]\n" \
  "v_mfma_i32_16x16x64_i8 v[148:151], v[108:111], a[176:179], v[148:151]\n" \
  "ds_read_b64_tr_b8 v[98:99], v84 offset:50176\n" \
  "v_pk_mul_f32 v[174:175], v[174:175], v[88:89]\n" \
  "v_mfma_i32_16x16x64_i8 v[180:183], v[104:107], a[164:167], v[180:183]\n" \
  "buffer_store_dwordx4 v[172:175], v47, s[20:23], s77 offen offset:192 sc0 sc1\n" \
  "v_mfma_i32_16x16x64_i8 v[180:183], v[108:111], a[180:183], v[180:183]\n" \
  "v_mfma_i32_16x16x64_i8 v[212:215], v[104:107], a[168:171], v[212:215]\n" \
  "ds_read_b64_tr_b8 v[100:101], v84 offset:57344\n" \
  "v_cvt_f32_i32 v200, v200\n" \
  "v_mfma_i32_16x16x64_i8 v[212:215], v[108:111], a[184:187], v[212:215]\n" \
  "ds_read_b64_tr_b8 v[102:103], v84 offset:58368\n" \
  "v_cvt_f32_i32 v201, v201\n" \
  "v_mfma_i32_16x16x64_i8 v[244:247], v[104:107], a[172:175], v[244:247]\n" \
  "v_cvt_f32_i32 v202, v202\n" \
  "v_mfma_i32_16x16x64_i8 v[244:247], v[108:111], a[188:191], v[244:247]\n" \
  "v_cvt_f32_i32 v203, v203\n" \
  "s_waitcnt lgkmcnt(8)\n" \
  "v_mfma_i32_16x16x64_i8 v[144:147], v[112:115], a[192:195], v[144:147]\n" \
  "ds_read_b64_tr_b8 v[104:105], v85 offset:49152\n" \
  "v_mfma_i32_16x16x64_i8 v[144:147], v[116:119], a[208:211], v[144:147]\n" \
  "ds_read_b64_tr_b8 v[106:107], v85 offset:50176\n" \
  "v_pk_mul_f32 v[200:201], v[200:201], v[88:89]\n" \
  "v_mfma_i32_16x16x64_i8 v[176:179], v[112:115], a[196:199], v[176:179]\n" \
  "v_pk_mul_f32 v[202:203], v[202:203], v[88:89]\n" \
  "v_mfma_i32_16x16x64_i8 v[176:179], v[116:119], a[212:215], v[176:179]\n" \
  "buffer_store_dwordx4 v[200:203], v47, s[20:23], s78 offen offset:128 sc0 sc1\n" \
  "v_mfma_i32_16x16x64_i8 v[208:211], v[112:115], a[200:203], v[208:211]\n" \
  "ds_read_b64_tr_b8 v[108:109], v85 offset:57344\n" \
  "v_mfma_i32_16x16x64_i8 v[208:211], v[116:119], a[216:219], v[208:211]\n" \
  "ds_read_b64_tr_b8 v[110:111], v85 offset:58368\n" \
  "v_cvt_f32_i32 v204, v204\n" \
  "v_mfma_i32_16x16x64_i8 v[240:243], v[112:115], a[204:207], v[240:243]\n" \
  "v_cvt_f32_i32 v205, v205\n" \
  "v_mfma_i32_16x16x64_i8 v[240:243], v[116:119], a[220:223], v[240:243]\n" \
  "v_cvt_f32_i32 v206, v206\n" \
  "s_waitcnt lgkmcnt(8)\n" \
  "v_mfma_i32_16x16x64_i8 v[148:151], v[120:123], a[192:195], v[148:151]\n" \
  "ds_read_b64_tr_b8 v[112:113], v38 offset:49152\n" \
  "v_cvt_f32_i32 v207, v207\n" \
  "v_mfma_i32_16x16x64_i8 v[148:151], v[124:127], a[208:211], v[148:151]\n" \
  "ds_read_b64_tr_b8 v[114:115], v38 offset:50176\n" \
  "v_mfma_i32_16x16x64_i8 v[180:183], v[120:123], a[196:199], v[180:183]\n" \
  "v_pk_mul_f32 v[204:205], v[204:205], v[88:89]\n" \
  "v_mfma_i32_16x16x64_i8 v[180:183], v[124:127], a[212:215], v[180:183]\n" \
  "v_pk_mul_f32 v[206:207], v[206:207], v[88:89]\n" \
  "v_mfma_i32_16x16x64_i8 v[212:215], v[120:123], a[200:203], v[212:215]\n" \
  "ds_read_b64_tr_b8 v[116:117], v38 offset:57344\n" \
  "buffer_store_dwordx4 v[204:207], v47, s[20:23], s78 offen offset:192 sc0 sc1\n" \
  "v_mfma_i32_16x16x64_i8 v[212:215], v[124:127], a[216:219], v[212:215]\n" \
  "ds_read_b64_tr_b8 v[118:119], v38 offset:58368\n" \
  "v_mfma_i32_16x16x64_i8 v[244:247], v[120:123], a[204:207], v[244:247]\n" \
  "v_cvt_f32_i32 v232, v232\n" \
  "v_mfma_i32_16x16x64_i8 v[244:247], v[124:127], a[220:223], v[244:247]\n" \
  "v_cvt_f32_i32 v233, v233\n" \
  "s_waitcnt lgkmcnt(8)\n" \
  "v_mfma_i32_16x16x64_i8 v[144:147], v[96:99], a[224:227], v[144:147]\n" \
  "ds_read_b64_tr_b8 v[120:121], v39 offset:49152\n" \
  "v_cvt_f32_i32 v234, v234\n" \
  "v_mfma_i32_16x16x64_i8 v[144:147], v[100:103], a[240:243], v[144:147]\n" \
  "ds_read_b64_tr_b8 v[122:123], v39 offset:50176\n" \
  "v_cvt_f32_i32 v235, v235\n" \
  "v_mfma_i32_16x16x64_i8 v[176:179], v[96:99], a[228:231], v[176:179]\n" \
  "v_mfma_i32_16x16x64_i8 v[176:179], v[100:103], a[244:247], v[176:179]\n" \
  "v_pk_mul_f32 v[232:233], v[232:233], v[88:89]\n" \
  "v_mfma_i32_16x16x64_i8 v[208:211], v[96:99], a[232:235], v[208:211]\n" \
  "ds_read_b64_tr_b8 v[124:125], v39 offset:57344\n" \
  "v_pk_mul_f32 v[234:235], v[234:235], v[88:89]\n" \
  "v_mfma_i32_16x16x64_i8 v[208:211], v[100:103], a[248:251], v[208:211]\n" \
  "ds_read_b64_tr_b8 v[126:127], v39 offset:58368\n" \
  "buffer_store_dwordx4 v[232:235], v47, s[20:23], s79 offen offset:128 sc0 sc1\n" \
  "v_mfma_i32_16x16x64_i8 v[240:243], v[96:99], a[236:239], v[240:243]\n" \
  "v_mfma_i32_16x16x64_i8 v[240:243], v[100:103], a[252:255], v[240:243]\n" \
  "v_cvt_f32_i32 v236, v236\n" \
  "s_waitcnt lgkmcnt(8)\n" \
  "v_mfma_i32_16x16x64_i8 v[148:151], v[104:107], a[224:227], v[148:151]\n" \
  "ds_read_b64_tr_b8 v[96:97], v86 offset:0\n" \
  "v_cvt_f32_i32 v237, v237\n" \
  "v_mfma_i32_16x16x64_i8 v[148:151], v[108:111], a[240:243], v[148:151]\n" \
  "ds_read_b64_tr_b8 v[98:99], v86 offset:1024\n" \
  "v_cvt_f32_i32 v238, v238\n" \
  "v_mfma_i32_16x16x64_i8 v[180:183], v[104:107], a[228:231], v[180:183]\n" \
  "v_cvt_f32_i32 v239, v239\n" \
  "v_mfma_i32_16x16x64_i8 v[180:183], v[108:111], a[244:247], v[180:183]\n" \
  "v_mfma_i32_16x16x64_i8 v[212:215], v[104:107], a[232:235], v[212:215]\n" \
  "ds_read_b64_tr_b8 v[100:101], v86 offset:8192\n" \
  "v_pk_mul_f32 v[236:237], v[236:237], v[88:89]\n" \
  "v_mfma_i32_16x16x64_i8 v[212:215], v[108:111], a[248:251], v[212:215]\n" \
  "ds_read_b64_tr_b8 v[102:103], v86 offset:9216\n" \
  "v_pk_mul_f32 v[238:239], v[238:239], v[88:89]\n" \
  "v_mfma_i32_16x16x64_i8 v[244:247], v[104:107], a[236:239], v[244:247]\n" \
  "buffer_store_dwordx4 v[236:239], v47, s[20:23], s79 offen offset:192 sc0 sc1\n" \
  "v_mfma_i32_16x16x64_i8 v[244:247], v[108:111], a[252:255], v[244:247]\n" \
  "s_waitcnt lgkmcnt(8)\n" \
  "v_mfma_i32_16x16x64_i8 v[152:155], v[112:115], a[96:99], v[152:155]\n" \
  "ds_read_b64_tr_b8 v[104:105], v87 offset:0\n" \
  "v_mfma_i32_16x16x64_i8 v[152:155], v[116:119], a[112:115], v[152:155]\n" \
  "ds_read_b64_tr_b8 v[106:107], v87 offset:1024\n" \
  "v_mfma_i32_16x16x64_i8 v[184:187], v[112:115], a[100:103], v[184:187]\n" \
  "v_mfma_i32_16x16x64_i8 v[184:187], v[116:119], a[116:119], v[184:187]\n" \
  "v_mfma_i32_16x16x64_i8 v[216:219], v[112:115], a[104:107], v[216:219]\n" \
  "ds_read_b64_tr_b8 v[108:109], v87 offset:8192\n" \
  "v_mfma_i32_16x16x64_i8 v[216:219], v[116:119], a[120:123], v[216:219]\n" \
  "ds_read_b64_tr_b8 v[110:111], v87 offset:9216\n" \
  "v_mfma_i32_16x16x64_i8 v[248:251], v[112:115], a[108:111], v[248:251]\n" \
  "v_mfma_i32_16x16x64_i8 v[248:251], v[116:119], a[124:127], v[248:251]\n" \
  "s_waitcnt lgkmcnt(8)\n" \
  "v_mfma_i32_16x16x64_i8 v[156:159], v[120:123], a[96:99], v[156:159]\n" \
  "ds_read_b64_tr_b8 v[112:113], v86 offset:16384\n" \
  "v_cvt_f32_i32 v144, v144\n" \
  "v_mfma_i32_16x16x64_i8 v[156:159], v[124:127], a[112:115], v[156:159]\n" \
  "ds_read_b64_tr_b8 v[114:115], v86 offset:17408\n" \
  "v_cvt_f32_i32 v145, v145\n" \
  "v_mfma_i32_16x16x64_i8 v[188:191], v[120:123], a[100:103], v[188:191]\n" \
  "v_cvt_f32_i32 v146, v146\n" \
  "v_mfma_i32_16x16x64_i8 v[188:191], v[124:127], a[116:119], v[188:191]\n" \
  "v_cvt_f32_i32 v147, v147\n" \
  "v_mfma_i32_16x16x64_i8 v[220:223], v[120:123], a[104:107], v[220:223]\n" \
  "ds_read_b64_tr_b8 v[116:117], v86 offset:24576\n" \
  "v_mfma_i32_16x16x64_i8 v[220:223], v[124:127], a[120:123], v[220:223]\n" \
  "ds_read_b64_tr_b8 v[118:119], v86 offset:25600\n" \
  "v_pk_mul_f32 v[144:145], v[144:145], v[88:89]\n" \
  "v_mfma_i32_16x16x64_i8 v[252:255], v[120:123], a[108:111], v[252:255]\n" \
  "v_pk_mul_f32 v[146:147], v[146:147], v[88:89]\n" \
  "v_mfma_i32_16x16x64_i8 v[252:255], v[124:127], a[124:127], v[252:255]\n" \
  "buffer_store_dwordx4 v[144:147], v47, s[20:23], s76 offen offset:256 sc0 sc1\n" \
  "; ---- step 7\n" \
  "s_waitcnt lgkmcnt(8)\n" \
  "v_mfma_i32_16x16x64_i8 v[152:155], v[96:99], a[128:131], v[152:155]\n" \
  "ds_read_b64_tr_b8 v[120:121], v87 offset:16384\n" \
  "v_mfma_i32_16x16x64_i8 v[152:155], v[100:103], a[144:147], v[152:155]\n" \
  "ds_read_b64_tr_b8 v[122:123], v87 offset:17408\n" \
  "v_cvt_f32_i32 v148, v148\n" \
  "v_mfma_i32_16x16x64_i8 v[184:187], v[96:99], a[132:135], v[184:187]\n" \
  "v_cvt_f32_i32 v149, v149\n" \
  "v_mfma_i32_16x16x64_i8 v[184:187], v[100:103], a[148:151], v[184:187]\n" \
  "v_cvt_f32_i32 v150, v150\n" \
  "v_mfma_i32_16x16x64_i8 v[216:219], v[96:99], a[136:139], v[216:219]\n" \
  "ds_read_b64_tr_b8 v[124:125], v87 offset:24576\n" \
  "v_cvt_f32_i32 v151, v151\n" \
  "v_mfma_i32_16x16x64_i8 v[216:219], v[100:103], a[152:155], v[216:219]\n" \
  "ds_read_b64_tr_b8 v[126:127], v87 offset:25600\n" \
  "v_mfma_i32_16x16x64_i8 v[248:251], v[96:99], a[140:143], v[248:251]\n" \
  "v_pk_mul_f32 v[148:149], v[148:149], v[88:89]\n" \
  "v_mfma_i32_16x16x64_i8 v[248:251], v[100:103], a[156:159], v[248:251]\n" \
  "v_pk_mul_f32 v[150:151], v[150:151], v[88:89]\n" \
  "s_waitcnt lgkmcnt(8)\n" \
  "v_mfma_i32_16x16x64_i8 v[156:159], v[104:107], a[128:131], v[156:159]\n" \
  "ds_read_b64_tr_b8 v[96:97], v86 offset:32768\n" \
  "buffer_store_dwordx4 v[148:151], v47, s[20:23], s76 offen offset:320 sc0 sc1\n" \
  "v_mfma_i32_16x16x64_i8 v[156:159], v[108:111], a[144:147], v[156:159]\n" \
  "ds_read_b64_tr_b8 v[98:99], v86 offset:33792\n" \
  "v_mfma_i32_16x16x64_i8 v[188:191], v[104:107], a[132:135], v[188:191]\n" \
  "v_cvt_f32_i32 v176, v176\n" \
  "v_mfma_i32_16x16x64_i8 v[188:191], v[108:111], a[148:151], v[188:191]\n" \
  "v_cvt_f32_i32 v177, v177\n" \
  "v_mfma_i32_16x16x64_i8 v[220:223], v[104:107], a[136:139], v[220:223]\n" \
  "ds_read_b64_tr_b8 v[100:101], v86 offset:40960\n" \
  "v_cvt_f32_i32 v178, v178\n" \
  "v_mfma_i32_16x16x64_i8 v[220:223], v[108:111], a[152:155], v[220:223]\n" \
  "ds_read_b64_tr_b8 v[102:103], v86 offset:41984\n" \
  "v_cvt_f32_i32 v179, v179\n" \
  "v_mfma_i32_16x16x64_i8 v[252:255], v[104:107], a[140:143], v[252:255]\n" \
  "v_mfma_i32_16x16x64_i8 v[252:255], v[108:111], a[156:159], v[252:255]\n" \
  "v_pk_mul_f32 v[176:177], v[176:177], v[88:89]\n" \
  "s_waitcnt lgkmcnt(8)\n" \
  "v_mfma_i32_16x16x64_i8 v[152:155], v[112:115], a[160:163], v[152:155]\n" \
  "ds_read_b64_tr_b8 v[104:105], v87 offset:32768\n" \
  "v_pk_mul_f32 v[178:179], v[178:179], v[88:89]\n" \
  "v_mfma_i32_16x16x64_i8 v[152:155], v[116:119], a[176:179], v[152:155]\n" \
  "ds_read_b64_tr_b8 v[106:107], v87 offset:33792\n" \
  "buffer_store_dwordx4 v[176:179], v47, s[20:23], s77 offen offset:256 sc0 sc1\n" \
  "v_mfma_i32_16x16x64_i8 v[184:187], v[112:115], a[164:167], v[184:187]\n" \
  "v_mfma_i32_16x16x64_i8 v[184:187], v[116:119], a[180:183], v[184:187]\n" \
  "v_cvt_f32_i32 v180, v180\n" \
  "v_mfma_i32_16x16x64_i8 v[216:219], v[112:115], a[168:171], v[216:219]\n" \
  "ds_read_b64_tr_b8 v[108:109], v87 offset:40960\n" \
  "v_cvt_f32_i32 v181, v181\n" \
  "v_mfma_i32_16x16x64_i8 v[216:219], v[116:119], a[184:187], v[216:219]\n" \
  "ds_read_b64_tr_b8 v[110:111], v87 offset:41984\n" \
  "v_cvt_f32_i32 v182, v182\n" \
  "v_mfma_i32_16x16x64_i8 v[248:251], v[112:115], a[172:175], v[248:251]\n" \
  "v_cvt_f32_i32 v183, v183\n" \
  "v_mfma_i32_16x16x64_i8 v[248:251], v[116:119], a[188:191], v[248:251]\n" \
  "s_waitcnt lgkmcnt(8)\n" \
  "v_mfma_i32_16x16x64_i8 v[156:159], v[120:123], a[160:163], v[156:159]\n" \
  "ds_read_b64_tr_b8 v[112:113], v86 offset:49152\n" \
  "v_pk_mul_f32 v[180:181], v[180:181], v[88:89]\n" \
  "v_mfma_i32_16x16x64_i8 v[156:159], v[124:127], a[176:179], v[156:159]\n" \
  "ds_read_b64_tr_b8 v[114:115], v86 offset:50176\n" \
  "v_pk_mul_f32 v[182:183], v[182:183], v[88:89]\n" \
  "v_mfma_i32_16x16x64_i8 v[188:191], v[120:123], a[164:167], v[188:191]\n" \
  "buffer_store_dwordx4 v[180:183], v47, s[20:23], s77 offen offset:320 sc0 sc1\n" \
  "v_mfma_i32_16x16x64_i8 v[188:191], v[124:127], a[180:183], v[188:191]\n" \
  "v_mfma_i32_16x16x64_i8 v[220:223], v[120:123], a[168:171], v[220:223]\n" \
  "ds_read_b64_tr_b8 v[116:117], v86 offset:57344\n" \
  "v_cvt_f32_i32 v208, v208\n" \
  "v_mfma_i32_16x16x64_i8 v[220:223], v[124:127], a[184:187], v[220:223]\n" \
  "ds_read_b64_tr_b8 v[118:119], v86 offset:58368\n" \
  "v_cvt_f32_i32 v209, v209\n" \
  "v_mfma_i32_16x16x64_i8 v[252:255], v[120:123], a[172:175], v[252:255]\n" \
  "v_cvt_f32_i32 v210, v210\n" \
  "v_mfma_i32_16x16x64_i8 v[252:255], v[124:127], a[188:191], v[252:255]\n" \
  "v_cvt_f32_i32 v211, v211\n" \
  "s_waitcnt lgkmcnt(8)\n" \
  "v_mfma_i32_16x16x64_i8 v[152:155], v[96:99], a[192:195], v[152:155]\n" \
  "ds_read_b64_tr_b8 v[120:121], v87 offset:49152\n" \
  "v_mfma_i32_16x16x64_i8 v[152:155], v[100:103], a[208:211], v[152:155]\n" \
  "ds_read_b64_tr_b8 v[122:123], v87 offset:50176\n" \
  "v_pk_mul_f32 v[208:209], v[208:209], v[88:89]\n" \
  "v_mfma_i32_16x16x64_i8 v[184:187], v[96:99], a[196:199], v[184:187]\n" \
  "v_pk_mul_f32 v[210:211], v[210:211], v[88:89]\n" \
  "v_mfma_i32_16x16x64_i8 v[184:187], v[100:103], a[212:215], v[184:187]\n" \
  "buffer_store_dwordx4 v[208:211], v47, s[20:23], s78 offen offset:256 sc0 sc1\n" \
  "v_mfma_i32_16x16x64_i8 v[216:219], v[96:99], a[200:203], v[216:219]\n" \
  "ds_read_b64_tr_b8 v[124:125], v87 offset:57344\n" \
  "v_mfma_i32_16x16x64_i8 v[216:219], v[100:103], a[216:219], v[216:219]\n" \
  "ds_read_b64_tr_b8 v[126:127], v87 offset:58368\n" \
  "v_cvt_f32_i32 v212, v212\n" \
  "v_mfma_i32_16x16x64_i8 v[248:251], v[96:99], a[204:207], v[248:251]\n" \
  "v_cvt_f32_i32 v213, v213\n" \
  "v_mfma_i32_16x16x64_i8 v[248:251], v[100:103], a[220:223], v[248:251]\n" \
  "v_cvt_f32_i32 v214, v214\n" \
  "s_waitcnt lgkmcnt(8)\n" \
  "v_mfma_i32_16x16x64_i8 v[156:159], v[104:107], a[192:195], v[156:159]\n" \
  "v_cvt_f32_i32 v215, v215\n" \
  "v_mfma_i32_16x16x64_i8 v[156:159], v[108:111], a[208:211], v[156:159]\n" \
  "v_mfma_i32_16x16x64_i8 v[188:191], v[104:107], a[196:199], v[188:191]\n" \
  "v_pk_mul_f32 v[212:213], v[212:213], v[88:89]\n" \
  "v_mfma_i32_16x16x64_i8 v[188:191], v[108:111], a[212:215], v[188:191]\n" \
  "v_pk_mul_f32 v[214:215], v[214:215], v[88:89]\n" \
  "v_mfma_i32_16x16x64_i8 v[220:223], v[104:107], a[200:203], v[220:223]\n" \
  "buffer_store_dwordx4 v[212:215], v47, s[20:23], s78 offen offset:320 sc0 sc1\n" \
  "v_mfma_i32_16x16x64_i8 v[220:223], v[108:111], a[216:219], v[220:223]\n" \
  "v_mfma_i32_16x16x64_i8 v[252:255], v[104:107], a[204:207], v[252:255]\n" \
  "v_cvt_f32_i32 v240, v240\n" \
  "v_mfma_i32_16x16x64_i8 v[252:255], v[108:111], a[220:223], v[252:255]\n" \
  "v_cvt_f32_i32 v241, v241\n" \
  "s_waitcnt lgkmcnt(4)\n" \
  "v_mfma_i32_16x16x64_i8 v[152:155], v[112:115], a[224:227], v[152:155]\n" \
  "v_cvt_f32_i32 v242, v242\n" \
  "v_mfma_i32_16x16x64_i8 v[152:155], v[116:119], a[240:243], v[152:155]\n" \
  "v_cvt_f32_i32 v243, v243\n" \
  "v_mfma_i32_16x16x64_i8 v[184:187], v[112:115], a[228:231], v[184:187]\n" \
  "v_mfma_i32_16x16x64_i8 v[184:187], v[116:119], a[244:247], v[184:187]\n" \
  "v_pk_mul_f32 v[240:241], v[240:241], v[88:89]\n" \
  "v_mfma_i32_16x16x64_i8 v[216:219], v[112:115], a[232:235], v[216:219]\n" \
  "v_pk_mul_f32 v[242:243], v[242:243], v[88:89]\n" \
  "v_mfma_i32_16x16x64_i8 v[216:219], v[116:119], a[248:251], v[216:219]\n" \
  "buffer_store_dwordx4 v[240:243], v47, s[20:23], s79 offen offset:256 sc0 sc1\n" \
  "v_mfma_i32_16x16x64_i8 v[248:251], v[112:115], a[236:239], v[248:251]\n" \
  "v_mfma_i32_16x16x64_i8 v[248:251], v[116:119], a[252:255], v[248:251]\n" \
  "v_cvt_f32_i32 v244, v244\n" \
  "s_waitcnt lgkmcnt(0)\n" \
  "v_mfma_i32_16x16x64_i8 v[156:159], v[120:123], a[224:227], v[156:159]\n" \
  "v_cvt_f32_i32 v245, v245\n" \
  "v_mfma_i32_16x16x64_i8 v[156:159], v[124:127], a[240:243], v[156:159]\n" \
  "v_cvt_f32_i32 v246, v246\n" \
  "v_mfma_i32_16x16x64_i8 v[188:191], v[120:123], a[228:231], v[188:191]\n" \
  "v_cvt_f32_i32 v247, v247\n" \
  "v_mfma_i32_16x16x64_i8 v[188:191], v[124:127], a[244:247], v[188:191]\n" \
  "v_mfma_i32_16x16x64_i8 v[220:223], v[120:123], a[232:235], v[220:223]\n" \
  "v_pk_mul_f32 v[244:245], v[244:245], v[88:89]\n" \
  "v_mfma_i32_16x16x64_i8 v[220:223], v[124:127], a[248:251], v[220:223]\n" \
  "v_pk_mul_f32 v[246:247], v[246:247], v[88:89]\n" \
  "v_mfma_i32_16x16x64_i8 v[252:255], v[120:123], a[236:239], v[252:255]\n" \
  "buffer_store_dwordx4 v[244:247], v47, s[20:23], s79 offen offset:320 sc0 sc1\n" \
  "v_mfma_i32_16x16x64_i8 v[252:255], v[124:127], a[252:255], v[252:255]\n" \
  "s_nop 15\n" \
  "v_cvt_f32_i32 v152, v152\n" \
  "v_cvt_f32_i32 v153, v153\n" \
  "v_cvt_f32_i32 v154, v154\n" \
  "v_cvt_f32_i32 v155, v155\n" \
  "v_pk_mul_f32 v[152:153], v[152:153], v[88:89]\n" \
  "v_pk_mul_f32 v[154:155], v[154:155], v[88:89]\n" \
  "buffer_store_dwordx4 v[152:155], v47, s[20:23], s76 offen offset:384 sc0 sc1\n" \
  "v_cvt_f32_i32 v156, v156\n" \
  "v_cvt_f32_i32 v157, v157\n" \
  "v_cvt_f32_i32 v158, v158\n" \
  "v_cvt_f32_i32 v159, v159\n" \
  "v_pk_mul_f32 v[156:157], v[156:157], v[88:89]\n" \
  "v_pk_mul_f32 v[158:159], v[158:159], v[88:89]\n" \
  "buffer_store_dwordx4 v[156:159], v47, s[20:23], s76 offen offset:448 sc0 sc1\n" \
  "v_cvt_f32_i32 v184, v184\n" \
  "v_cvt_f32_i32 v185, v185\n" \
  "v_cvt_f32_i32 v186, v186\n" \
  "v_cvt_f32_i32 v187, v187\n" \
  "v_pk_mul_f32 v[184:185], v[184:185], v[88:89]\n" \
  "v_pk_mul_f32 v[186:187], v[186:187], v[88:89]\n" \
  "buffer_store_dwordx4 v[184:187], v47, s[20:23], s77 offen offset:384 sc0 sc1\n" \
  "v_cvt_f32_i32 v188, v188\n" \
  "v_cvt_f32_i32 v189, v189\n" \
  "v_cvt_f32_i32 v190, v190\n" \
  "v_cvt_f32_i32 v191, v191\n" \
  "v_pk_mul_f32 v[188:189], v[188:189], v[88:89]\n" \
  "v_pk_mul_f32 v[190:191], v[190:191], v[88:89]\n" \
  "buffer_store_dwordx4 v[188:191], v47, s[20:23], s77 offen offset:448 sc0 sc1\n" \
  "v_cvt_f32_i32 v216, v216\n" \
  "v_cvt_f32_i32 v217, v217\n" \
  "v_cvt_f32_i32 v218, v218\n" \
  "v_cvt_f32_i32 v219, v219\n" \
  "v_pk_mul_f32 v[216:217], v[216:217], v[88:89]\n" \
  "v_pk_mul_f32 v[218:219], v[218:219], v[88:89]\n" \
  "buffer_store_dwordx4 v[216:219], v47, s[20:23], s78 offen offset:384 sc0 sc1\n" \
  "v_cvt_f32_i32 v220, v220\n" \
  "v_cvt_f32_i32 v221, v221\n" \
  "v_cvt_f32_i32 v222, v222\n" \
  "v_cvt_f32_i32 v223, v223\n" \
  "v_pk_mul_f32 v[220:221], v[220:221], v[88:89]\n" \
  "v_pk_mul_f32 v[222:223], v[222:223], v[88:89]\n" \
  "buffer_store_dwordx4 v[220:223], v47, s[20:23], s78 offen offset:448 sc0 sc1\n" \
  "v_cvt_f32_i32 v248, v248\n" \
  "v_cvt_f32_i32 v249, v249\n" \
  "v_cvt_f32_i32 v250, v250\n" \
  "v_cvt_f32_i32 v251, v251\n" \
  "v_pk_mul_f32 v[248:249], v[248:249], v[88:89]\n" \
  "v_pk_mul_f32 v[250:251], v[250:251], v[88:89]\n" \
  "buffer_store_dwordx4 v[248:251], v47, s[20:23], s79 offen offset:384 sc0 sc1\n" \
  "v_cvt_f32_i32 v252, v252\n" \
  "v_cvt_f32_i32 v253, v253\n" \
  "v_cvt_f32_i32 v254, v254\n" \
  "v_cvt_f32_i32 v255, v255\n" \
  "v_pk_mul_f32 v[252:253], v[252:253], v[88:89]\n" \
  "v_pk_mul_f32 v[254:255], v[254:255], v[88:89]\n" \
  "buffer_store_dwordx4 v[252:255], v47, s[20:23], s79 offen offset:448 sc0 sc1\n" \
  "s_mov_b64 exec, 1\n" \
  "global_store_dword v51, v76, s[32:33] sc0 sc1\n" \
  "s_mov_b64 exec, -1\n" \
  "s_waitcnt vmcnt(0)\n" \
  "s_branch L_end_%=\n" \
  "Lslow12_%=:\n" \
  "s_memrealtime s[96:97]\n" \
  "s_waitcnt lgkmcnt(0)\n" \
  "Lspin12_%=:\n" \
  "s_sleep 4\n" \
  "global_load_dword v72, v50, s[38:39] sc0 sc1\n" \
  "global_load_dword v90, v51, s[12:13] sc1\n" \
  "s_waitcnt vmcnt(0)\n" \
  "v_sub_u32 v72, v72, v76\n" \
  "v_cmp_gt_i32 vcc, 0, v72\n" \
  "s_nop 1\n" \
  "s_andn2_b64 vcc, vcc, s[14:15]\n" \
  "s_cbranch_scc0 Lback13_%=\n" \
  "v_cmp_eq_u32 vcc, v90, v76\n" \
  "s_nop 1\n" \
  "s_cmp_lg_u64 vcc, 0\n" \
  "s_cbranch_scc1 Ltmo12_%=\n" \
  "s_memrealtime s[98:99]\n" \
  "s_waitcnt lgkmcnt(0)\n" \
  "s_sub_u32 s98, s98, s96\n" \
  "s_cmp_lt_u32 s98, 50000000\n" \
  "s_cbranch_scc1 Lspin12_%=\n" \
  "Ltmo12_%=:\n" \
  "global_store_dword v51, v76, s[12:13] sc1\n" \
  "s_mov_b32 s55, 1\n" \
  "s_branch Lback13_%=\n" \
  "Lslow20_%=:\n" \
  "s_memrealtime s[96:97]\n" \
  "s_waitcnt lgkmcnt(0)\n" \
  "Lspin20_%=:\n" \
  "s_sleep 4\n" \
  "global_load_dword v72, v50, s[38:39] sc0 sc1\n" \
  "global_load_dword v90, v51, s[12:13] sc1\n" \
  "s_waitcnt vmcnt(0)\n" \
  "v_sub_u32 v72, v72, v76\n" \
  "v_cmp_gt_i32 vcc, 0, v72\n" \
  "s_nop 1\n" \
  "s_andn2_b64 vcc, vcc, s[14:15]\n" \
  "s_cbranch_scc0 Lback21_%=\n" \
  "v_cmp_eq_u32 vcc, v90, v76\n" \
  "s_nop 1\n" \
  "s_cmp_lg_u64 vcc, 0\n" \
  "s_cbranch_scc1 Ltmo20_%=\n" \
  "s_memrealtime s[98:99]\n" \
  "s_waitcnt lgkmcnt(0)\n" \
  "s_sub_u32 s98, s98, s96\n" \
  "s_cmp_lt_u32 s98, 50000000\n" \
  "s_cbranch_scc1 Lspin20_%=\n" \
  "Ltmo20_%=:\n" \
  "global_store_dword v51, v76, s[12:13] sc1\n" \
  "s_mov_b32 s55, 1\n" \
  "s_branch Lback21_%=\n" \
  "Lslow28_%=:\n" \
  "s_memrealtime s[96:97]\n" \
  "s_waitcnt lgkmcnt(0)\n" \
  "Lspin28_%=:\n" \
  "s_sleep 4\n" \
  "global_load_dword v72, v50, s[38:39] sc0 sc1\n" \
  "global_load_dword v90, v51, s[12:13] sc1\n" \
  "s_waitcnt vmcnt(0)\n" \
  "v_sub_u32 v72, v72, v76\n" \
  "v_cmp_gt_i32 vcc, 0, v72\n" \
  "s_nop 1\n" \
  "s_andn2_b64 vcc, vcc, s[14:15]\n" \
  "s_cbranch_scc0 Lback29_%=\n" \
  "v_cmp_eq_u32 vcc, v90, v76\n" \
  "s_nop 1\n" \
  "s_cmp_lg_u64 vcc, 0\n" \
  "s_cbranch_scc1 Ltmo28_%=\n" \
  "s_memrealtime s[98:99]\n" \
  "s_waitcnt lgkmcnt(0)\n" \
  "s_sub_u32 s98, s98, s96\n" \
  "s_cmp_lt_u32 s98, 50000000\n" \
  "s_cbranch_scc1 Lspin28_%=\n" \
  "Ltmo28_%=:\n" \
  "global_store_dword v51, v76, s[12:13] sc1\n" \
  "s_mov_b32 s55, 1\n" \
  "s_branch Lback29_%=\n" \
  "L_end_%=:\n" \
  "s_branch Lfin_%=\n" \
  "LvarA_%=:\n" \
  "s_mov_b32 s16, s66\n" \
  "s_and_b32 s17, s67, 0xffff\n" \
  "s_mov_b32 s18, 0x4000000\n" \
  "s_mov_b32 s19, 0x20000\n" \
  "s_mov_b32 s24, s64\n" \
  "s_and_b32 s25, s65, 0xffff\n" \
  "s_mov_b32 s26, 0x200000\n" \
  "s_mov_b32 s27, 0x20000\n" \
  "s_lshl_b32 s81, s2, 6\n" \
  "s_add_u32 s32, s74, s81\n" \
  "s_addc_u32 s33, s75, 0\n" \
  "s_mov_b64 s[44:45], s[64:65]\n" \
  "s_lshl_b32 s96, s50, 9\n" \
  "s_lshl_b32 s97, s53, 8\n" \
  "s_add_u32 s96, s96, s97\n" \
  "s_lshl_b32 s97, s11, 6\n" \
  "s_add_u32 s97, s96, s97\n" \
  "v_add_u32 v10, s97, v3\n" \
  "v_lshl_add_u32 v45, v10, 9, v9\n" \
  "s_add_u32 s98, s50, 1\n" \
  "s_add_u32 s99, s98, s87\n" \
  "s_and_b32 s99, s99, 7\n" \
  "s_lshl_b32 s99, s99, 9\n" \
  "s_lshl_b32 s81, s52, 6\n" \
  "s_add_u32 s99, s99, s81\n" \
  "s_lshl_b32 s81, s11, 4\n" \
  "s_add_u32 s99, s99, s81\n" \
  "v_add_u32 v10, s99, v6\n" \
  "v_lshlrev_b32 v10, 9, v10\n" \
  "v_lshl_add_u32 v10, v5, 4, v10\n" \
  "s_lshl_b32 s81, s51, 7\n" \
  "v_add_u32 v48, s81, v10\n" \
  "s_lshl_b32 s81, s54, 4\n" \
  "s_add_u32 s81, s96, s81\n" \
  "v_add_u32 v10, s81, v2\n" \
  "v_lshlrev_b32 v10, 9, v10\n" \
  "s_lshl_b32 s81, s11, 7\n" \
  "v_add_u32 v10, s81, v10\n" \
  "v_mov_b32 v11, 0x80000000\n" \
  "v_cmp_gt_u32 vcc, 16, v2\n" \
  "s_nop 1\n" \
  "v_cndmask_b32 v56, v11, v10, vcc\n" \
  "global_load_dword v76, v51, s[32:33] sc0 sc1\n" \
  "s_cmp_gt_u32 s46, 3\n" \
  "s_cbranch_scc1 Lnox1A_%=\n" \
  "global_load_dwordx4 v[64:67], v48, s[44:45]\n" \
  "global_load_dwordx4 v[68:71], v48, s[44:45] offset:64\n" \
  "Lnox1A_%=:\n" \
  "buffer_load_dword v92, v56, s[24:27], 0 offen\n" \
  "s_mov_b32 s49, 0\n" \
  "s_lshl_b32 s80, s50, 23\n" \
  "s_lshl_b32 s88, s50, 18\n" \
  "s_mov_b32 s49, 1\n" \
  "s_add_u32 m0, s47, 0\n" \
  "s_nop 0\n" \
  "buffer_load_dwordx4 v40, s[16:19], s80 offen nt lds\n" \
  "s_add_u32 m0, s47, 1024\n" \
  "s_nop 0\n" \
  "buffer_load_dwordx4 v41, s[16:19], s80 offen nt lds\n" \
  "s_add_u32 m0, s47, 2048\n" \
  "s_nop 0\n" \
  "buffer_load_dwordx4 v42, s[16:19], s80 offen nt lds\n" \
  "s_add_u32 m0, s47, 3072\n" \
  "s_nop 0\n" \
  "buffer_load_dwordx4 v43, s[16:19], s80 offen nt lds\n" \
  "s_lshl1_add_u32 m0, s47, 131072\n" \
  "s_mov_b32 s98, 0\n" \
  "buffer_load_dwordx4 v45, s[24:27], s98 offen lds\n" \
  "s_lshl1_add_u32 m0, s47, 132096\n" \
  "s_mov_b32 s98, 4096\n" \
  "buffer_load_dwordx4 v45, s[24:27], s98 offen lds\n" \
  "s_lshl1_add_u32 m0, s47, 133120\n" \
  "s_mov_b32 s98, 8192\n" \
  "buffer_load_dwordx4 v45, s[24:27], s98 offen lds\n" \
  "s_lshl1_add_u32 m0, s47, 134144\n" \
  "s_mov_b32 s98, 12288\n" \
  "buffer_load_dwordx4 v45, s[24:27], s98 offen lds\n" \
  "s_lshl1_add_u32 m0, s47, 135168\n" \
  "s_mov_b32 s98, 16384\n" \
  "buffer_load_dwordx4 v45, s[24:27], s98 offen lds\n" \
  "s_lshl1_add_u32 m0, s47, 136192\n" \
  "s_mov_b32 s98, 20480\n" \
  "buffer_load_dwordx4 v45, s[24:27], s98 offen lds\n" \
  "s_lshl1_add_u32 m0, s47, 137216\n" \
  "s_mov_b32 s98, 24576\n" \
  "buffer_load_dwordx4 v45, s[24:27], s98 offen lds\n" \
  "s_lshl1_add_u32 m0, s47, 138240\n" \
  "s_mov_b32 s98, 28672\n" \
  "buffer_load_dwordx4 v45, s[24:27], s98 offen lds\n" \
  "s_mov_b32 s20, s72\n" \
  "s_and_b32 s21, s73, 0xffff\n" \
  "s_mov_b32 s22, 0x2000000\n" \
  "s_mov_b32 s23, 0x20000\n" \
  "s_add_u32 s28, s74, 1048576\n" \
  "s_addc_u32 s29, s75, 0\n" \
  "s_and_b32 s29, s29, 0xffff\n" \
  "s_mov_b32 s30, 0x200000\n" \
  "s_mov_b32 s31, 0x20000\n" \
  "s_mov_b64 s[40:41], s[68:69]\n" \
  "s_mov_b64 s[42:43], s[70:71]\n" \
  "s_add_u32 s12, s74, 16384\n" \
  "s_addc_u32 s13, s75, 0\n" \
  "s_lshl_b32 s81, s50, 3\n" \
  "s_mov_b64 s[14:15], 0xff\n" \
  "s_lshl_b64 s[14:15], s[14:15], s81\n" \
  "s_add_u32 s36, s34, 32768\n" \
  "s_addc_u32 s37, s35, 0\n" \
  "s_add_u32 s38, s74, 32768\n" \
  "s_addc_u32 s39, s75, 0\n" \
  "v_lshrrev_b32 v7, 1, v6\n" \
  "v_lshl_add_u32 v8, v5, 4, v7\n" \
  "v_lshlrev_b32 v8, 7, v8\n" \
  "v_and_b32 v10, 1, v2\n" \
  "v_lshl_add_u32 v8, v10, 3, v8\n" \
  "v_lshrrev_b32 v10, 1, v7\n" \
  "v_and_b32 v10, 3, v10\n" \
  "v_and_b32 v11, 1, v5\n" \
  "v_lshl_or_b32 v10, v11, 2, v10\n" \
  "v_lshl_or_b32 v32, v10, 4, v8\n" \
  "v_xor_b32 v33, 16, v32\n" \
  "v_xor_b32 v34, 32, v32\n" \
  "v_xor_b32 v35, 48, v32\n" \
  "v_xor_b32 v36, 64, v32\n" \
  "v_xor_b32 v37, 80, v32\n" \
  "v_xor_b32 v38, 96, v32\n" \
  "v_xor_b32 v39, 112, v32\n" \
  "s_lshl_b32 s81, s53, 2\n" \
  "s_add_u32 s81, s81, s11\n" \
  "s_lshl_b32 s81, s81, 13\n" \
  "v_lshl_add_u32 v44, v2, 4, s81\n" \
  "v_add_u32 v10, -4, v2\n" \
  "v_mul_u32_u24 v11, 37, v10\n" \
  "v_lshrrev_b32 v11, 8, v11\n" \
  "v_mul_u32_u24 v12, 7, v11\n" \
  "v_sub_u32 v12, v10, v12\n" \
  "s_add_u32 s81, s50, 1\n" \
  "v_add_u32 v12, s81, v12\n" \
  "v_and_b32 v12, 7, v12\n" \
  "v_lshl_add_u32 v12, v12, 2, v11\n" \
  "s_lshl_b32 s81, s50, 2\n" \
  "v_add_u32 v13, s81, v2\n" \
  "v_cmp_gt_u32 vcc, 4, v2\n" \
  "s_nop 1\n" \
  "v_cndmask_b32 v46, v12, v13, vcc\n" \
  "v_mov_b32 v13, 32\n" \
  "v_cmp_gt_u32 vcc, 32, v2\n" \
  "s_nop 1\n" \
  "v_cndmask_b32 v46, v13, v46, vcc\n" \
  "s_lshl_b32 s81, s53, 8\n" \
  "s_lshl_b32 s89, s11, 6\n" \
  "s_add_u32 s81, s81, s89\n" \
  "v_add_u32 v10, s81, v6\n" \
  "v_lshlrev_b32 v10, 16, v10\n" \
  "s_lshl_b32 s81, s55, 9\n" \
  "v_lshl_add_u32 v10, v5, 4, v10\n" \
  "v_add_u32 v47, s81, v10\n" \
  "s_lshl_b32 s81, s50, 2\n" \
  "s_add_u32 s81, s81, s51\n" \
  "s_lshl_b32 s81, s81, 16\n" \
  "s_add_u32 s81, s81, 1048576\n" \
  "s_lshl_b32 s89, s52, 13\n" \
  "s_add_u32 s81, s81, s89\n" \
  "s_lshl_b32 s89, s11, 10\n" \
  "s_add_u32 s81, s81, s89\n" \
  "v_lshl_add_u32 v49, v2, 4, s81\n" \
  "v_lshlrev_b32 v10, 2, v4\n" \
  "v_lshl_add_u32 v50, v3, 7, v10\n" \
  "s_lshl_b32 s81, s51, 10\n" \
  "s_lshl_b32 s89, s50, 7\n" \
  "s_add_u32 s81, s81, s89\n" \
  "s_lshl_b32 s89, s52, 2\n" \
  "s_add_u32 s81, s81, s89\n" \
  "v_mov_b32 v53, s81\n" \
  "s_mul_i32 s81, s11, 28\n" \
  "v_add_u32 v10, s81, v2\n" \
  "v_lshrrev_b32 v11, 4, v10\n" \
  "v_and_b32 v10, 15, v10\n" \
  "s_add_u32 s81, s50, 1\n" \
  "v_add_u32 v11, s81, v11\n" \
  "v_and_b32 v11, 7, v11\n" \
  "v_lshlrev_b32 v11, 18, v11\n" \
  "s_lshl_b32 s81, s54, 4\n" \
  "v_add_u32 v10, s81, v10\n" \
  "v_lshl_add_u32 v11, v10, 7, v11\n" \
  "s_lshl_b32 s81, s53, 15\n" \
  "v_add_u32 v11, s81, v11\n" \
  "v_mov_b32 v12, 0x80000000\n" \
  "v_cmp_gt_u32 vcc, 28, v2\n" \
  "s_nop 1\n" \
  "v_cndmask_b32 v52, v12, v11, vcc\n" \
  "v_xor_b32 v10, v5, v4\n" \
  "v_lshlrev_b32 v10, 4, v10\n" \
  "v_lshl_add_u32 v57, v6, 7, v10\n" \
  "s_lshr_b32 s4, s2, 3\n" \
  "s_lshl_b32 s4, s4, 2\n" \
  "s_add_u32 s4, s4, s11\n" \
  "s_lshl_b32 s5, s4, 21\n" \
  "s_cmp_lt_u32 s4, 32\n" \
  "s_cselect_b32 s5, s5, 0x80000000\n" \
  "s_buffer_load_dword s75, s[16:19], s5\n" \
  "s_sub_u32 s5, s4, 32\n" \
  "s_lshl_b32 s6, s5, 21\n" \
  "s_cmp_lt_u32 s5, 16\n" \
  "s_cselect_b32 s6, s6, 0x80000000\n" \
  "s_buffer_load_dword s75, s[20:23], s6\n" \
  "s_mov_b32 s9, 0\n" \
  "s_mov_b32 s10, 0\n" \
  "global_load_dword v77, v51, s[40:41]\n" \
  "global_load_dword v78, v51, s[42:43]\n" \
  "v_readlane_b32 s50, v46, s49\n" \
  "s_nop 3\n" \
  "s_lshl_b32 s81, s50, 21\n" \
  "s_lshl_b32 s89, s50, 16\n" \
  "s_add_u32 s49, s49, 1\n" \
  "v_readlane_b32 s50, v46, s49\n" \
  "s_nop 3\n" \
  "s_lshl_b32 s82, s50, 21\n" \
  "s_lshl_b32 s90, s50, 16\n" \
  "s_add_u32 s49, s49, 1\n" \
  "s_mov_b32 s76, 0\n" \
  "s_mov_b32 s77, 1048576\n" \
  "s_mov_b32 s78, 2097152\n" \
  "s_mov_b32 s79, 3145728\n" \
  "v_add_u32 v80, 0x10000, v32\n" \
  "v_add_u32 v81, 0x10000, v33\n" \
  "v_add_u32 v82, 0x10000, v34\n" \
  "v_add_u32 v83, 0x10000, v35\n" \
  "v_add_u32 v84, 0x10000, v36\n" \
  "v_add_u32 v85, 0x10000, v37\n" \
  "v_add_u32 v86, 0x10000, v38\n" \
  "v_add_u32 v87, 0x10000, v39\n" \
  "v_add_u32 v91, 4096, v44\n" \
  "v_add_u32 v24, 0x20000, v32\n" \
  "v_add_u32 v25, 0x20000, v33\n" \
  "v_add_u32 v26, 0x20000, v34\n" \
  "v_add_u32 v27, 0x20000, v35\n" \
  "v_add_u32 v28, 0x20000, v36\n" \
  "v_add_u32 v29, 0x20000, v37\n" \
  "v_add_u32 v30, 0x20000, v38\n" \
  "v_add_u32 v31, 0x20000, v39\n" \
  "v_add_u32 v79, 4096, v49\n" \
  "s_lshl1_add_u32 s96, s47, 131072\n" \
  "v_add_u32 v58, s96, v57\n" \
  "v_xor_b32 v59, 64, v58\n" \
  "s_add_u32 s96, s47, 98304\n" \
  "v_add_u32 v60, s96, v57\n" \
  "v_xor_b32 v61, 64, v60\n" \
  "s_waitcnt vmcnt(15)\n" \
  "v_add_u32 v76, 1, v76\n" \
  "s_nop 0\n" \
  "v_readfirstlane_b32 s52, v76\n" \
  "s_nop 3\n" \
  "s_and_b32 s48, s52, 1\n" \
  "s_lshl_b32 s48, s48, 21\n" \
  "s_add_u32 s34, s34, s48\n" \
  "s_addc_u32 s35, s35, 0\n" \
  "s_add_u32 s28, s28, s48\n" \
  "s_addc_u32 s29, s29, 0\n" \
  "s_mov_b32 s55, 0\n" \
  "s_mov_b32 s53, 1\n" \
  "s_mov_b32 s54, 2\n" \
  "s_cmp_lg_u32 s46, 0\n" \
  "s_cbranch_scc1 Lskip2A_%=\n" \
  "s_waitcnt vmcnt(15)\n" \
  "global_store_dwordx4 v49, v[64:67], s[34:35] sc0 sc1\n" \
  "s_nop 1\n" \
  "global_store_dwordx4 v79, v[68:71], s[34:35] sc0 sc1\n" \
  "s_nop 1\n" \
  "Lskip2A_%=:\n" \
  "s_add_u32 m0, s47, 16384\n" \
  "s_nop 0\n" \
  "buffer_load_dwordx4 v40, s[16:19], s81 offen nt lds\n" \
  "s_add_u32 m0, s47, 17408\n" \
  "s_nop 0\n" \
  "buffer_load_dwordx4 v41, s[16:19], s81 offen nt lds\n" \
  "s_add_u32 m0, s47, 18432\n" \
  "s_nop 0\n" \
  "buffer_load_dwordx4 v42, s[16:19], s81 offen nt lds\n" \
  "s_add_u32 m0, s47, 19456\n" \
  "s_nop 0\n" \
  "buffer_load_dwordx4 v43, s[16:19], s81 offen nt lds\n" \
  "s_add_u32 m0, s47, 98304\n" \
  "s_mov_b32 s98, 128\n" \
  "buffer_load_dwordx4 v45, s[24:27], s98 offen lds\n" \
  "s_add_u32 m0, s47, 99328\n" \
  "s_mov_b32 s98, 4224\n" \
  "buffer_load_dwordx4 v45, s[24:27], s98 offen lds\n" \
  "s_add_u32 m0, s47, 100352\n" \
  "s_mov_b32 s98, 8320\n" \
  "buffer_load_dwordx4 v45, s[24:27], s98 offen lds\n" \
  "s_add_u32 m0, s47, 101376\n" \
  "s_mov_b32 s98, 12416\n" \
  "buffer_load_dwordx4 v45, s[24:27], s98 offen lds\n" \
  "s_add_u32 m0, s47, 114688\n" \
  "s_mov_b32 s98, 16512\n" \
  "buffer_load_dwordx4 v45, s[24:27], s98 offen lds\n" \
  "s_add_u32 m0, s47, 115712\n" \
  "s_mov_b32 s98, 20608\n" \
  "buffer_load_dwordx4 v45, s[24:27], s98 offen lds\n" \
  "s_add_u32 m0, s47, 116736\n" \
  "s_mov_b32 s98, 24704\n" \
  "buffer_load_dwordx4 v45, s[24:27], s98 offen lds\n" \
  "s_add_u32 m0, s47, 117760\n" \
  "s_mov_b32 s98, 28800\n" \
  "buffer_load_dwordx4 v45, s[24:27], s98 offen lds\n" \
  "s_add_u32 m0, s47, 32768\n" \
  "s_nop 0\n" \
  "buffer_load_dwordx4 v40, s[16:19], s82 offen nt lds\n" \
  "s_add_u32 m0, s47, 33792\n" \
  "s_nop 0\n" \
  "buffer_load_dwordx4 v41, s[16:19], s82 offen nt lds\n" \
  "s_add_u32 m0, s47, 34816\n" \
  "s_nop 0\n" \
  "buffer_load_dwordx4 v42, s[16:19], s82 offen nt lds\n" \
  "s_add_u32 m0, s47, 35840\n" \
  "s_nop 0\n" \
  "buffer_load_dwordx4 v43, s[16:19], s82 offen nt lds\n" \
  "v_mov_b32 v128, 0\n" \
  "v_mov_b32 v129, 0\n" \
  "v_mov_b32 v130, 0\n" \
  "v_mov_b32 v131, 0\n" \
  "v_mov_b32 v132, 0\n" \
  "v_mov_b32 v133, 0\n" \
  "v_mov_b32 v134, 0\n" \
  "v_mov_b32 v135, 0\n" \
  "v_mov_b32 v136, 0\n" \
  "v_mov_b32 v137, 0\n" \
  "v_mov_b32 v138, 0\n" \
  "v_mov_b32 v139, 0\n" \
  "v_mov_b32 v140, 0\n" \
  "v_mov_b32 v141, 0\n" \
  "v_mov_b32 v142, 0\n" \
  "v_mov_b32 v143, 0\n" \
  "v_mov_b32 v144, 0\n" \
  "v_mov_b32 v145, 0\n" \
  "v_mov_b32 v146, 0\n" \
  "v_mov_b32 v147, 0\n" \
  "v_mov_b32 v148, 0\n" \
  "v_mov_b32 v149, 0\n" \
  "v_mov_b32 v150, 0\n" \
  "v_mov_b32 v151, 0\n" \
  "v_mov_b32 v152, 0\n" \
  "v_mov_b32 v153, 0\n" \
  "v_mov_b32 v154, 0\n" \
  "v_mov_b32 v155, 0\n" \
  "v_mov_b32 v156, 0\n" \
  "v_mov_b32 v157, 0\n" \
  "v_mov_b32 v158, 0\n" \
  "v_mov_b32 v159, 0\n" \
  "v_mov_b32 v160, 0\n" \
  "v_mov_b32 v161, 0\n" \
  "v_mov_b32 v162, 0\n" \
  "v_mov_b32 v163, 0\n" \
  "v_mov_b32 v164, 0\n" \
  "v_mov_b32 v165, 0\n" \
  "v_mov_b32 v166, 0\n" \
  "v_mov_b32 v167, 0\n" \
  "v_mov_b32 v168, 0\n" \
  "v_mov_b32 v169, 0\n" \
  "v_mov_b32 v170, 0\n" \
  "v_mov_b32 v171, 0\n" \
  "v_mov_b32 v172, 0\n" \
  "v_mov_b32 v173, 0\n" \
  "v_mov_b32 v174, 0\n" \
  "v_mov_b32 v175, 0\n" \
  "v_mov_b32 v176, 0\n" \
  "v_mov_b32 v177, 0\n" \
  "v_mov_b32 v178, 0\n" \
  "v_mov_b32 v179, 0\n" \
  "v_mov_b32 v180, 0\n" \
  "v_mov_b32 v181, 0\n" \
  "v_mov_b32 v182, 0\n" \
  "v_mov_b32 v183, 0\n" \
  "v_mov_b32 v184, 0\n" \
  "v_mov_b32 v185, 0\n" \
  "v_mov_b32 v186, 0\n" \
  "v_mov_b32 v187, 0\n" \
  "v_mov_b32 v188, 0\n" \
  "v_mov_b32 v189, 0\n" \
  "v_mov_b32 v190, 0\n" \
  "v_mov_b32 v191, 0\n" \
  "v_mov_b32 v192, 0\n" \
  "v_mov_b32 v193, 0\n" \
  "v_mov_b32 v194, 0\n" \
  "v_mov_b32 v195, 0\n" \
  "v_mov_b32 v196, 0\n" \
  "v_mov_b32 v197, 0\n" \
  "v_mov_b32 v198, 0\n" \
  "v_mov_b32 v199, 0\n" \
  "v_mov_b32 v200, 0\n" \
  "v_mov_b32 v201, 0\n" \
  "v_mov_b32 v202, 0\n" \
  "v_mov_b32 v203, 0\n" \
  "v_mov_b32 v204, 0\n" \
  "v_mov_b32 v205, 0\n" \
  "v_mov_b32 v206, 0\n" \
  "v_mov_b32 v207, 0\n" \
  "v_mov_b32 v208, 0\n" \
  "v_mov_b32 v209, 0\n" \
  "v_mov_b32 v210, 0\n" \
  "v_mov_b32 v211, 0\n" \
  "v_mov_b32 v212, 0\n" \
  "v_mov_b32 v213, 0\n" \
  "v_mov_b32 v214, 0\n" \
  "v_mov_b32 v215, 0\n" \
  "v_mov_b32 v216, 0\n" \
  "v_mov_b32 v217, 0\n" \
  "v_mov_b32 v218, 0\n" \
  "v_mov_b32 v219, 0\n" \
  "v_mov_b32 v220, 0\n" \
  "v_mov_b32 v221, 0\n" \
  "v_mov_b32 v222, 0\n" \
  "v_mov_b32 v223, 0\n" \
  "v_mov_b32 v224, 0\n" \
  "v_mov_b32 v225, 0\n" \
  "v_mov_b32 v226, 0\n" \
  "v_mov_b32 v227, 0\n" \
  "v_mov_b32 v228, 0\n" \
  "v_mov_b32 v229, 0\n" \
  "v_mov_b32 v230, 0\n" \
  "v_mov_b32 v231, 0\n" \
  "v_mov_b32 v232, 0\n" \
  "v_mov_b32 v233, 0\n" \
  "v_mov_b32 v234, 0\n" \
  "v_mov_b32 v235, 0\n" \
  "v_mov_b32 v236, 0\n" \
  "v_mov_b32 v237, 0\n" \
  "v_mov_b32 v238, 0\n" \
  "v_mov_b32 v239, 0\n" \
  "v_mov_b32 v240, 0\n" \
  "v_mov_b32 v241, 0\n" \
  "v_mov_b32 v242, 0\n" \
  "v_mov_b32 v243, 0\n" \
  "v_mov_b32 v244, 0\n" \
  "v_mov_b32 v245, 0\n" \
  "v_mov_b32 v246, 0\n" \
  "v_mov_b32 v247, 0\n" \
  "v_mov_b32 v248, 0\n" \
  "v_mov_b32 v249, 0\n" \
  "v_mov_b32 v250, 0\n" \
  "v_mov_b32 v251, 0\n" \
  "v_mov_b32 v252, 0\n" \
  "v_mov_b32 v253, 0\n" \
  "v_mov_b32 v254, 0\n" \
  "v_mov_b32 v255, 0\n" \
  "s_cmp_eq_u32 s46, 0\n" \
  "s_cbranch_scc1 Lskip3A_%=\n" \
  "s_waitcnt vmcnt(18)\n" \
  "s_branch Lskip4A_%=\n" \
  "Lskip3A_%=:\n" \
  "s_waitcnt vmcnt(20)\n" \
  "Lskip4A_%=:\n" \
  "ds_read_b128 a[0:3], v58 offset:0\n" \
  "ds_read_b128 a[4:7], v58 offset:2048\n" \
  "ds_read_b128 a[8:11], v58 offset:4096\n" \
  "ds_read_b128 a[12:15], v58 offset:6144\n" \
  "ds_read_b128 a[16:19], v59 offset:0\n" \
  "ds_read_b128 a[20:23], v59 offset:2048\n" \
  "ds_read_b128 a[24:27], v59 offset:4096\n" \
  "ds_read_b128 a[28:31], v59 offset:6144\n" \
  "s_waitcnt vmcnt(26)\n" \
  "s_barrier\n" \
  "ds_read_b64_tr_b8 v[96:97], v32 offset:0\n" \
  "ds_read_b64_tr_b8 v[98:99], v32 offset:1024\n" \
  "ds_read_b64_tr_b8 v[100:101], v32 offset:8192\n" \
  "ds_read_b64_tr_b8 v[102:103], v32 offset:9216\n" \
  "ds_read_b64_tr_b8 v[104:105], v33 offset:0\n" \
  "ds_read_b64_tr_b8 v[106:107], v33 offset:1024\n" \
  "ds_read_b64_tr_b8 v[108:109], v33 offset:8192\n" \
  "ds_read_b64_tr_b8 v[110:111], v33 offset:9216\n" \
  "ds_read_b64_tr_b8 v[112:113], v34 offset:0\n" \
  "ds_read_b64_tr_b8 v[114:115], v34 offset:1024\n" \
  "ds_read_b64_tr_b8 v[116:117], v34 offset:8192\n" \
  "ds_read_b64_tr_b8 v[118:119], v34 offset:9216\n" \
  "; ---- step 0\n" \
  "s_waitcnt lgkmcnt(8)\n" \
  "v_mfma_i32_16x16x64_i8 v[128:131], v[96:99], a[0:3], v[128:131]\n" \
  "ds_read_b64_tr_b8 v[120:121], v35 offset:0\n" \
  "v_mfma_i32_16x16x64_i8 v[128:131], v[100:103], a[16:19], v[128:131]\n" \
  "ds_read_b64_tr_b8 v[122:123], v35 offset:1024\n" \
  "v_mfma_i32_16x16x64_i8 v[160:163], v[96:99], a[4:7], v[160:163]\n" \
  "s_lshl1_add_u32 m0, s47, 131072\n" \
  "s_mov_b32 s98, 256\n" \
  "v_mfma_i32_16x16x64_i8 v[160:163], v[100:103], a[20:23], v[160:163]\n" \
  "buffer_load_dwordx4 v45, s[24:27], s98 offen lds\n" \
  "v_mfma_i32_16x16x64_i8 v[192:195], v[96:99], a[8:11], v[192:195]\n" \
  "ds_read_b64_tr_b8 v[124:125], v35 offset:8192\n" \
  "v_mfma_i32_16x16x64_i8 v[192:195], v[100:103], a[24:27], v[192:195]\n" \
  "ds_read_b64_tr_b8 v[126:127], v35 offset:9216\n" \
  "v_mfma_i32_16x16x64_i8 v[224:227], v[96:99], a[12:15], v[224:227]\n" \
  "s_lshl1_add_u32 m0, s47, 132096\n" \
  "s_mov_b32 s98, 4352\n" \
  "v_mfma_i32_16x16x64_i8 v[224:227], v[100:103], a[28:31], v[224:227]\n" \
  "buffer_load_dwordx4 v45, s[24:27], s98 offen lds\n" \
  "s_waitcnt lgkmcnt(8)\n" \
  "v_mfma_i32_16x16x64_i8 v[132:135], v[104:107], a[0:3], v[132:135]\n" \
  "ds_read_b64_tr_b8 v[96:97], v36 offset:0\n" \
  "v_mfma_i32_16x16x64_i8 v[132:135], v[108:111], a[16:19], v[132:135]\n" \
  "ds_read_b64_tr_b8 v[98:99], v36 offset:1024\n" \
  "v_mfma_i32_16x16x64_i8 v[164:167], v[104:107], a[4:7], v[164:167]\n" \
  "v_readlane_b32 s50, v46, s49\n" \
  "s_lshl1_add_u32 m0, s47, 133120\n" \
  "s_mov_b32 s98, 8448\n" \
  "v_mfma_i32_16x16x64_i8 v[164:167], v[108:111], a[20:23], v[164:167]\n" \
  "buffer_load_dwordx4 v45, s[24:27], s98 offen lds\n" \
  "v_mfma_i32_16x16x64_i8 v[196:199], v[104:107], a[8:11], v[196:199]\n" \
  "ds_read_b64_tr_b8 v[100:101], v36 offset:8192\n" \
  "v_mfma_i32_16x16x64_i8 v[196:199], v[108:111], a[24:27], v[196:199]\n" \
  "ds_read_b64_tr_b8 v[102:103], v36 offset:9216\n" \
  "v_mfma_i32_16x16x64_i8 v[228:231], v[104:107], a[12:15], v[228:231]\n" \
  "s_lshl1_add_u32 m0, s47, 134144\n" \
  "s_mov_b32 s98, 12544\n" \
  "v_mfma_i32_16x16x64_i8 v[228:231], v[108:111], a[28:31], v[228:231]\n" \
  "buffer_load_dwordx4 v45, s[24:27], s98 offen lds\n" \
  "s_waitcnt lgkmcnt(8)\n" \
  "v_mfma_i32_16x16x64_i8 v[136:139], v[112:115], a[0:3], v[136:139]\n" \
  "ds_read_b64_tr_b8 v[104:105], v37 offset:0\n" \
  "v_mfma_i32_16x16x64_i8 v[136:139], v[116:119], a[16:19], v[136:139]\n" \
  "ds_read_b64_tr_b8 v[106:107], v37 offset:1024\n" \
  "v_mfma_i32_16x16x64_i8 v[168:171], v[112:115], a[4:7], v[168:171]\n" \
  "s_lshl_b32 s83, s50, 21\n" \
  "s_lshl_b32 s91, s50, 16\n" \
  "s_add_u32 s49, s49, 1\n" \
  "s_lshl1_add_u32 m0, s47, 135168\n" \
  "s_mov_b32 s98, 16640\n" \
  "v_mfma_i32_16x16x64_i8 v[168:171], v[116:119], a[20:23], v[168:171]\n" \
  "buffer_load_dwordx4 v45, s[24:27], s98 offen lds\n" \
  "v_mfma_i32_16x16x64_i8 v[200:203], v[112:115], a[8:11], v[200:203]\n" \
  "ds_read_b64_tr_b8 v[108:109], v37 offset:8192\n" \
  "v_mfma_i32_16x16x64_i8 v[200:203], v[116:119], a[24:27], v[200:203]\n" \
  "ds_read_b64_tr_b8 v[110:111], v37 offset:9216\n" \
  "v_mfma_i32_16x16x64_i8 v[232:235], v[112:115], a[12:15], v[232:235]\n" \
  "v_readlane_b32 s50, v46, s49\n" \
  "s_lshl1_add_u32 m0, s47, 136192\n" \
  "s_mov_b32 s98, 20736\n" \
  "v_mfma_i32_16x16x64_i8 v[232:235], v[116:119], a[28:31], v[232:235]\n" \
  "buffer_load_dwordx4 v45, s[24:27], s98 offen lds\n" \
  "s_waitcnt lgkmcnt(8)\n" \
  "v_mfma_i32_16x16x64_i8 v[140:143], v[120:123], a[0:3], v[140:143]\n" \
  "ds_read_b64_tr_b8 v[112:113], v38 offset:0\n" \
  "v_mfma_i32_16x16x64_i8 v[140:143], v[124:127], a[16:19], v[140:143]\n" \
  "ds_read_b64_tr_b8 v[114:115], v38 offset:1024\n" \
  "v_mfma_i32_16x16x64_i8 v[172:175], v[120:123], a[4:7], v[172:175]\n" \
  "s_lshl1_add_u32 m0, s47, 137216\n" \
  "s_mov_b32 s98, 24832\n" \
  "v_mfma_i32_16x16x64_i8 v[172:175], v[124:127], a[20:23], v[172:175]\n" \
  "buffer_load_dwordx4 v45, s[24:27], s98 offen lds\n" \
  "s_cmp_lg_u32 s46, 0\n" \
  "s_cbranch_scc1 Lskip5A_%=\n" \
  "s_waitcnt vmcnt(23)\n" \
  "Lskip5A_%=:\n" \
  "s_waitcnt vmcnt(19)\n" \
  "s_barrier\n" \
  "s_cmp_lg_u32 s46, 0\n" \
  "s_cbranch_scc1 Lskip7A_%=\n" \
  "s_cmp_lg_u32 s11, 0\n" \
  "s_cbranch_scc1 Lskip6A_%=\n" \
  "s_mov_b64 exec, 1\n" \
  "global_store_dword v53, v76, s[36:37] sc0 sc1\n" \
  "s_mov_b64 exec, -1\n" \
  "Lskip6A_%=:\n" \
  "Lskip7A_%=:\n" \
  "v_mfma_i32_16x16x64_i8 v[204:207], v[120:123], a[8:11], v[204:207]\n" \
  "ds_read_b64_tr_b8 v[116:117], v38 offset:8192\n" \
  "v_mfma_i32_16x16x64_i8 v[204:207], v[124:127], a[24:27], v[204:207]\n" \
  "ds_read_b64_tr_b8 v[118:119], v38 offset:9216\n" \
  "v_mfma_i32_16x16x64_i8 v[236:239], v[120:123], a[12:15], v[236:239]\n" \
  "s_lshl_b32 s84, s50, 21\n" \
  "s_lshl_b32 s92, s50, 16\n" \
  "s_add_u32 s49, s49, 1\n" \
  "s_lshl1_add_u32 m0, s47, 138240\n" \
  "s_mov_b32 s98, 28928\n" \
  "v_mfma_i32_16x16x64_i8 v[236:239], v[124:127], a[28:31], v[236:239]\n" \
  "buffer_load_dwordx4 v45, s[24:27], s98 offen lds\n" \
  "s_waitcnt lgkmcnt(8)\n" \
  "v_mfma_i32_16x16x64_i8 v[144:147], v[96:99], a[0:3], v[144:147]\n" \
  "ds_read_b64_tr_b8 v[120:121], v39 offset:0\n" \
  "v_mfma_i32_16x16x64_i8 v[144:147], v[100:103], a[16:19], v[144:147]\n" \
  "ds_read_b64_tr_b8 v[122:123], v39 offset:1024\n" \
  "v_mfma_i32_16x16x64_i8 v[176:179], v[96:99], a[4:7], v[176:179]\n" \
  "v_mfma_i32_16x16x64_i8 v[176:179], v[100:103], a[20:23], v[176:179]\n" \
  "v_mfma_i32_16x16x64_i8 v[208:211], v[96:99], a[8:11], v[208:211]\n" \
  "ds_read_b64_tr_b8 v[124:125], v39 offset:8192\n" \
  "s_add_u32 m0, s47, 49152\n" \
  "v_mfma_i32_16x16x64_i8 v[208:211], v[100:103], a[24:27], v[208:211]\n" \
  "ds_read_b64_tr_b8 v[126:127], v39 offset:9216\n" \
  "buffer_load_dwordx4 v40, s[16:19], s83 offen lds\n" \
  "s_add_u32 m0, s47, 50176\n" \
  "v_mfma_i32_16x16x64_i8 v[240:243], v[96:99], a[12:15], v[240:243]\n" \
  "buffer_load_dwordx4 v41, s[16:19], s83 offen lds\n" \
  "s_add_u32 m0, s47, 51200\n" \
  "v_mfma_i32_16x16x64_i8 v[240:243], v[100:103], a[28:31], v[240:243]\n" \
  "buffer_load_dwordx4 v42, s[16:19], s83 offen lds\n" \
  "s_add_u32 m0, s47, 52224\n" \
  "s_waitcnt lgkmcnt(8)\n" \
  "v_mfma_i32_16x16x64_i8 v[148:151], v[104:107], a[0:3], v[148:151]\n" \
  "ds_read_b64_tr_b8 v[96:97], v32 offset:16384\n" \
  "buffer_load_dwordx4 v43, s[16:19], s83 offen lds\n" \
  "s_add_u32 m0, s47, 65536\n" \
  "v_mfma_i32_16x16x64_i8 v[148:151], v[108:111], a[16:19], v[148:151]\n" \
  "ds_read_b64_tr_b8 v[98:99], v32 offset:17408\n" \
  "buffer_load_dwordx4 v40, s[16:19], s84 offen lds\n" \
  "s_add_u32 m0, s47, 66560\n" \
  "v_mfma_i32_16x16x64_i8 v[180:183], v[104:107], a[4:7], v[180:183]\n" \
  "buffer_load_dwordx4 v41, s[16:19], s84 offen lds\n" \
  "s_add_u32 m0, s47, 67584\n" \
  "v_mfma_i32_16x16x64_i8 v[180:183], v[108:111], a[20:23], v[180:183]\n" \
  "buffer_load_dwordx4 v42, s[16:19], s84 offen lds\n" \
  "s_add_u32 m0, s47, 68608\n" \
  "s_waitcnt vmcnt(19)\n" \
  "v_mfma_i32_16x16x64_i8 v[212:215], v[104:107], a[8:11], v[212:215]\n" \
  "ds_read_b64_tr_b8 v[100:101], v32 offset:24576\n" \
  "buffer_load_dwordx4 v43, s[16:19], s84 offen lds\n" \
  "v_mfma_i32_16x16x64_i8 v[212:215], v[108:111], a[24:27], v[212:215]\n" \
  "ds_read_b64_tr_b8 v[102:103], v32 offset:25600\n" \
  "v_mfma_i32_16x16x64_i8 v[244:247], v[104:107], a[12:15], v[244:247]\n" \
  "ds_read_b128 a[32:35], v60 offset:0\n" \
  "v_mfma_i32_16x16x64_i8 v[244:247], v[108:111], a[28:31], v[244:247]\n" \
  "s_waitcnt lgkmcnt(9)\n" \
  "v_mfma_i32_16x16x64_i8 v[152:155], v[112:115], a[0:3], v[152:155]\n" \
  "ds_read_b64_tr_b8 v[104:105], v33 offset:16384\n" \
  "ds_read_b128 a[36:39], v60 offset:2048\n" \
  "v_mfma_i32_16x16x64_i8 v[152:155], v[116:119], a[16:19], v[152:155]\n" \
  "ds_read_b64_tr_b8 v[106:107], v33 offset:17408\n" \
  "v_mfma_i32_16x16x64_i8 v[184:187], v[112:115], a[4:7], v[184:187]\n" \
  "ds_read_b128 a[40:43], v60 offset:16384\n" \
  "v_mfma_i32_16x16x64_i8 v[184:187], v[116:119], a[20:23], v[184:187]\n" \
  "v_mfma_i32_16x16x64_i8 v[216:219], v[112:115], a[8:11], v[216:219]\n" \
  "ds_read_b64_tr_b8 v[108:109], v33 offset:24576\n" \
  "ds_read_b128 a[44:47], v60 offset:18432\n" \
  "v_mfma_i32_16x16x64_i8 v[216:219], v[116:119], a[24:27], v[216:219]\n" \
  "ds_read_b64_tr_b8 v[110:111], v33 offset:25600\n" \
  "v_mfma_i32_16x16x64_i8 v[248:251], v[112:115], a[12:15], v[248:251]\n" \
  "ds_read_b128 a[48:51], v61 offset:0\n" \
  "v_mfma_i32_16x16x64_i8 v[248:251], v[116:119], a[28:31], v[248:251]\n" \
  "s_waitcnt lgkmcnt(13)\n" \
  "v_mfma_i32_16x16x64_i8 v[156:159], v[120:123], a[0:3], v[156:159]\n" \
  "ds_read_b64_tr_b8 v[112:113], v34 offset:16384\n" \
  "ds_read_b128 a[52:55], v61 offset:2048\n" \
  "v_mfma_i32_16x16x64_i8 v[156:159], v[124:127], a[16:19], v[156:159]\n" \
  "ds_read_b64_tr_b8 v[114:115], v34 offset:17408\n" \
  "v_mfma_i32_16x16x64_i8 v[188:191], v[120:123], a[4:7], v[188:191]\n" \
  "ds_read_b128 a[56:59], v61 offset:16384\n" \
  "v_mfma_i32_16x16x64_i8 v[188:191], v[124:127], a[20:23], v[188:191]\n" \
  "v_mfma_i32_16x16x64_i8 v[220:223], v[120:123], a[8:11], v[220:223]\n" \
  "ds_read_b64_tr_b8 v[116:117], v34 offset:24576\n" \
  "ds_read_b128 a[60:63], v61 offset:18432\n" \
  "v_mfma_i32_16x16x64_i8 v[220:223], v[124:127], a[24:27], v[220:223]\n" \
  "ds_read_b64_tr_b8 v[118:119], v34 offset:25600\n" \
  "v_mfma_i32_16x16x64_i8 v[252:255], v[120:123], a[12:15], v[252:255]\n" \
  "v_mfma_i32_16x16x64_i8 v[252:255], v[124:127], a[28:31], v[252:255]\n" \
  "; ---- step 1\n" \
  "s_waitcnt lgkmcnt(1)\n" \
  "v_mfma_i32_16x16x64_i8 v[128:131], v[96:99], a[32:35], v[128:131]\n" \
  "ds_read_b64_tr_b8 v[120:121], v35 offset:16384\n" \
  "v_mfma_i32_16x16x64_i8 v[128:131], v[100:103], a[48:51], v[128:131]\n" \
  "ds_read_b64_tr_b8 v[122:123], v35 offset:17408\n" \
  "v_mfma_i32_16x16x64_i8 v[160:163], v[96:99], a[36:39], v[160:163]\n" \
  "s_add_u32 m0, s47, 98304\n" \
  "s_mov_b32 s98, 384\n" \
  "v_mfma_i32_16x16x64_i8 v[160:163], v[100:103], a[52:55], v[160:163]\n" \
  "buffer_load_dwordx4 v45, s[24:27], s98 offen lds\n" \
  "v_mfma_i32_16x16x64_i8 v[192:195], v[96:99], a[40:43], v[192:195]\n" \
  "ds_read_b64_tr_b8 v[124:125], v35 offset:24576\n" \
  "v_mfma_i32_16x16x64_i8 v[192:195], v[100:103], a[56:59], v[192:195]\n" \
  "ds_read_b64_tr_b8 v[126:127], v35 offset:25600\n" \
  "v_mfma_i32_16x16x64_i8 v[224:227], v[96:99], a[44:47], v[224:227]\n" \
  "s_add_u32 m0, s47, 99328\n" \
  "s_mov_b32 s98, 4480\n" \
  "v_mfma_i32_16x16x64_i8 v[224:227], v[100:103], a[60:63], v[224:227]\n" \
  "buffer_load_dwordx4 v45, s[24:27], s98 offen lds\n" \
  "s_waitcnt lgkmcnt(12)\n" \
  "v_mfma_i32_16x16x64_i8 v[132:135], v[104:107], a[32:35], v[132:135]\n" \
  "ds_read_b64_tr_b8 v[96:97], v36 offset:16384\n" \
  "v_mfma_i32_16x16x64_i8 v[132:135], v[108:111], a[48:51], v[132:135]\n" \
  "ds_read_b64_tr_b8 v[98:99], v36 offset:17408\n" \
  "v_mfma_i32_16x16x64_i8 v[164:167], v[104:107], a[36:39], v[164:167]\n" \
  "s_add_u32 m0, s47, 100352\n" \
  "s_mov_b32 s98, 8576\n" \
  "v_mfma_i32_16x16x64_i8 v[164:167], v[108:111], a[52:55], v[164:167]\n" \
  "buffer_load_dwordx4 v45, s[24:27], s98 offen lds\n" \
  "v_mfma_i32_16x16x64_i8 v[196:199], v[104:107], a[40:43], v[196:199]\n" \
  "ds_read_b64_tr_b8 v[100:101], v36 offset:24576\n" \
  "v_mfma_i32_16x16x64_i8 v[196:199], v[108:111], a[56:59], v[196:199]\n" \
  "ds_read_b64_tr_b8 v[102:103], v36 offset:25600\n" \
  "v_mfma_i32_16x16x64_i8 v[228:231], v[104:107], a[44:47], v[228:231]\n" \
  "v_readlane_b32 s50, v46, s49\n" \
  "s_add_u32 m0, s47, 101376\n" \
  "s_mov_b32 s98, 12672\n" \
  "v_mfma_i32_16x16x64_i8 v[228:231], v[108:111], a[60:63], v[228:231]\n" \
  "buffer_load_dwordx4 v45, s[24:27], s98 offen lds\n" \
  "s_waitcnt lgkmcnt(8)\n" \
  "v_mfma_i32_16x16x64_i8 v[136:139], v[112:115], a[32:35], v[136:139]\n" \
  "ds_read_b64_tr_b8 v[104:105], v37 offset:16384\n" \
  "v_mfma_i32_16x16x64_i8 v[136:139], v[116:119], a[48:51], v[136:139]\n" \
  "ds_read_b64_tr_b8 v[106:107], v37 offset:17408\n" \
  "v_mfma_i32_16x16x64_i8 v[168:171], v[112:115], a[36:39], v[168:171]\n" \
  "s_add_u32 m0, s47, 114688\n" \
  "s_mov_b32 s98, 16768\n" \
  "v_mfma_i32_16x16x64_i8 v[168:171], v[116:119], a[52:55], v[168:171]\n" \
  "buffer_load_dwordx4 v45, s[24:27], s98 offen lds\n" \
  "v_mfma_i32_16x16x64_i8 v[200:203], v[112:115], a[40:43], v[200:203]\n" \
  "ds_read_b64_tr_b8 v[108:109], v37 offset:24576\n" \
  "v_mfma_i32_16x16x64_i8 v[200:203], v[116:119], a[56:59], v[200:203]\n" \
  "ds_read_b64_tr_b8 v[110:111], v37 offset:25600\n" \
  "v_mfma_i32_16x16x64_i8 v[232:235], v[112:115], a[44:47], v[232:235]\n" \
  "s_lshl_b32 s85, s50, 21\n" \
  "s_lshl_b32 s93, s50, 16\n" \
  "s_add_u32 s49, s49, 1\n" \
  "s_add_u32 m0, s47, 115712\n" \
  "s_mov_b32 s98, 20864\n" \
  "v_mfma_i32_16x16x64_i8 v[232:235], v[116:119], a[60:63], v[232:235]\n" \
  "buffer_load_dwordx4 v45, s[24:27], s98 offen lds\n" \
  "s_waitcnt lgkmcnt(8)\n" \
  "v_mfma_i32_16x16x64_i8 v[140:143], v[120:123], a[32:35], v[140:143]\n" \
  "ds_read_b64_tr_b8 v[112:113], v38 offset:16384\n" \
  "v_mfma_i32_16x16x64_i8 v[140:143], v[124:127], a[48:51], v[140:143]\n" \
  "ds_read_b64_tr_b8 v[114:115], v38 offset:17408\n" \
  "v_mfma_i32_16x16x64_i8 v[172:175], v[120:123], a[36:39], v[172:175]\n" \
  "v_readlane_b32 s50, v46, s49\n" \
  "s_add_u32 m0, s47, 116736\n" \
  "s_mov_b32 s98, 24960\n" \
  "v_mfma_i32_16x16x64_i8 v[172:175], v[124:127], a[52:55], v[172:175]\n" \
  "buffer_load_dwordx4 v45, s[24:27], s98 offen lds\n" \
  "s_waitcnt vmcnt(23)\n" \
  "s_barrier\n" \
  "v_mfma_i32_16x16x64_i8 v[204:207], v[120:123], a[40:43], v[204:207]\n" \
  "ds_read_b64_tr_b8 v[116:117], v38 offset:24576\n" \
  "v_mfma_i32_16x16x64_i8 v[204:207], v[124:127], a[56:59], v[204:207]\n" \
  "ds_read_b64_tr_b8 v[118:119], v38 offset:25600\n" \
  "v_mfma_i32_16x16x64_i8 v[236:239], v[120:123], a[44:47], v[236:239]\n" \
  "s_add_u32 m0, s47, 117760\n" \
  "s_mov_b32 s98, 29056\n" \
  "v_mfma_i32_16x16x64_i8 v[236:239], v[124:127], a[60:63], v[236:239]\n" \
  "buffer_load_dwordx4 v45, s[24:27], s98 offen lds\n" \
  "s_waitcnt vmcnt(16)\n" \
  "s_waitcnt lgkmcnt(8)\n" \
  "v_mfma_i32_16x16x64_i8 v[144:147], v[96:99], a[32:35], v[144:147]\n" \
  "ds_read_b64_tr_b8 v[120:121], v39 offset:16384\n" \
  "v_mfma_i32_16x16x64_i8 v[144:147], v[100:103], a[48:51], v[144:147]\n" \
  "ds_read_b64_tr_b8 v[122:123], v39 offset:17408\n" \
  "v_mfma_i32_16x16x64_i8 v[176:179], v[96:99], a[36:39], v[176:179]\n" \
  "s_lshl_b32 s86, s50, 21\n" \
  "s_lshl_b32 s94, s50, 16\n" \
  "s_add_u32 s49, s49, 1\n" \
  "ds_read_b128 a[64:67], v58 offset:0\n" \
  "v_mfma_i32_16x16x64_i8 v[176:179], v[100:103], a[52:55], v[176:179]\n" \
  "v_mfma_i32_16x16x64_i8 v[208:211], v[96:99], a[40:43], v[208:211]\n" \
  "ds_read_b64_tr_b8 v[124:125], v39 offset:24576\n" \
  "ds_read_b128 a[68:71], v58 offset:2048\n" \
  "v_mfma_i32_16x16x64_i8 v[208:211], v[100:103], a[56:59], v[208:211]\n" \
  "ds_read_b64_tr_b8 v[126:127], v39 offset:25600\n" \
  "v_mfma_i32_16x16x64_i8 v[240:243], v[96:99], a[44:47], v[240:243]\n" \
  "v_readlane_b32 s50, v46, s49\n" \
  "ds_read_b128 a[72:75], v58 offset:4096\n" \
  "v_mfma_i32_16x16x64_i8 v[240:243], v[100:103], a[60:63], v[240:243]\n" \
  "s_waitcnt lgkmcnt(11)\n" \
  "v_mfma_i32_16x16x64_i8 v[148:151], v[104:107], a[32:35], v[148:151]\n" \
  "ds_read_b64_tr_b8 v[96:97], v32 offset:32768\n" \
  "ds_read_b128 a[76:79], v58 offset:6144\n" \
  "v_mfma_i32_16x16x64_i8 v[148:151], v[108:111], a[48:51], v[148:151]\n" \
  "ds_read_b64_tr_b8 v[98:99], v32 offset:33792\n" \
  "v_mfma_i32_16x16x64_i8 v[180:183], v[104:107], a[36:39], v[180:183]\n" \
  "ds_read_b128 a[80:83], v59 offset:0\n" \
  "v_mfma_i32_16x16x64_i8 v[180:183], v[108:111], a[52:55], v[180:183]\n" \
  "v_mfma_i32_16x16x64_i8 v[212:215], v[104:107], a[40:43], v[212:215]\n" \
  "ds_read_b64_tr_b8 v[100:101], v32 offset:40960\n" \
  "ds_read_b128 a[84:87], v59 offset:2048\n" \
  "v_mfma_i32_16x16x64_i8 v[212:215], v[108:111], a[56:59], v[212:215]\n" \
  "ds_read_b64_tr_b8 v[102:103], v32 offset:41984\n" \
  "v_mfma_i32_16x16x64_i8 v[244:247], v[104:107], a[44:47], v[244:247]\n" \
  "s_lshl_b32 s87, s50, 21\n" \
  "s_lshl_b32 s95, s50, 16\n" \
  "s_add_u32 s49, s49, 1\n" \
  "ds_read_b128 a[88:91], v59 offset:4096\n" \
  "v_mfma_i32_16x16x64_i8 v[244:247], v[108:111], a[60:63], v[244:247]\n" \
  "s_waitcnt vmcnt(0)\n" \
  "s_waitcnt lgkmcnt(15)\n" \
  "v_mfma_i32_16x16x64_i8 v[152:155], v[112:115], a[32:35], v[152:155]\n" \
  "ds_read_b64_tr_b8 v[104:105], v33 offset:32768\n" \
  "ds_read_b128 a[92:95], v59 offset:6144\n" \
  "v_mfma_i32_16x16x64_i8 v[152:155], v[116:119], a[48:51], v[152:155]\n" \
  "ds_read_b64_tr_b8 v[106:107], v33 offset:33792\n" \
  "v_mfma_i32_16x16x64_i8 v[184:187], v[112:115], a[36:39], v[184:187]\n" \
  "v_readlane_b32 s50, v46, s49\n" \
  "ds_read_b128 a[96:99], v60 offset:0\n" \
  "v_mfma_i32_16x16x64_i8 v[184:187], v[116:119], a[52:55], v[184:187]\n" \
  "v_mfma_i32_16x16x64_i8 v[216:219], v[112:115], a[40:43], v[216:219]\n" \
  "ds_read_b64_tr_b8 v[108:109], v33 offset:40960\n" \
  "ds_read_b128 a[100:103], v60 offset:2048\n" \
  "v_mfma_i32_16x16x64_i8 v[216:219], v[116:119], a[56:59], v[216:219]\n" \
  "ds_read_b64_tr_b8 v[110:111], v33 offset:41984\n" \
  "v_mfma_i32_16x16x64_i8 v[248:251], v[112:115], a[44:47], v[248:251]\n" \
  "ds_read_b128 a[104:107], v60 offset:16384\n" \
  "v_mfma_i32_16x16x64_i8 v[248:251], v[116:119], a[60:63], v[248:251]\n" \
  "s_waitcnt lgkmcnt(15)\n" \
  "v_mfma_i32_16x16x64_i8 v[156:159], v[120:123], a[32:35], v[156:159]\n" \
  "ds_read_b64_tr_b8 v[112:113], v34 offset:32768\n" \
  "ds_read_b128 a[108:111], v60 offset:18432\n" \
  "v_mfma_i32_16x16x64_i8 v[156:159], v[124:127], a[48:51], v[156:159]\n" \
  "ds_read_b64_tr_b8 v[114:115], v34 offset:33792\n" \
  "v_mfma_i32_16x16x64_i8 v[188:191], v[120:123], a[36:39], v[188:191]\n" \
  "s_lshl_b32 s80, s50, 21\n" \
  "s_lshl_b32 s88, s50, 16\n" \
  "s_add_u32 s49, s49, 1\n" \
  "ds_read_b128 a[112:115], v61 offset:0\n" \
  "v_mfma_i32_16x16x64_i8 v[188:191], v[124:127], a[52:55], v[188:191]\n" \
  "v_mfma_i32_16x16x64_i8 v[220:223], v[120:123], a[40:43], v[220:223]\n" \
  "ds_read_b64_tr_b8 v[116:117], v34 offset:40960\n" \
  "ds_read_b128 a[116:119], v61 offset:2048\n" \
  "v_mfma_i32_16x16x64_i8 v[220:223], v[124:127], a[56:59], v[220:223]\n" \
  "ds_read_b64_tr_b8 v[118:119], v34 offset:41984\n" \
  "v_mfma_i32_16x16x64_i8 v[252:255], v[120:123], a[44:47], v[252:255]\n" \
  "ds_read_b128 a[120:123], v61 offset:16384\n" \
  "v_mfma_i32_16x16x64_i8 v[252:255], v[124:127], a[60:63], v[252:255]\n" \
  "ds_read_b128 a[124:127], v61 offset:18432\n" \
  "; ---- step 2\n" \
  "s_cmp_lg_u32 s46, 1\n" \
  "s_cbranch_scc1 Lskip8A_%=\n" \
  "s_waitcnt vmcnt(55)\n" \
  "global_store_dwordx4 v49, v[64:67], s[34:35] sc0 sc1\n" \
  "s_nop 1\n" \
  "global_store_dwordx4 v79, v[68:71], s[34:35] sc0 sc1\n" \
  "s_nop 1\n" \
  "Lskip8A_%=:\n" \
  "s_waitcnt lgkmcnt(0)\n" \
  "v_mfma_i32_16x16x64_i8 v[128:131], v[96:99], a[64:67], v[128:131]\n" \
  "ds_read_b64_tr_b8 v[120:121], v35 offset:32768\n" \
  "s_add_u32 m0, s47, 114688\n" \
  "v_mfma_i32_16x16x64_i8 v[128:131], v[100:103], a[80:83], v[128:131]\n" \
  "ds_read_b64_tr_b8 v[122:123], v35 offset:33792\n" \
  "buffer_load_dwordx4 v40, s[16:19], s87 offen lds\n" \
  "s_add_u32 m0, s47, 115712\n" \
  "v_mfma_i32_16x16x64_i8 v[160:163], v[96:99], a[68:71], v[160:163]\n" \
  "buffer_load_dwordx4 v41, s[16:19], s87 offen lds\n" \
  "s_add_u32 m0, s47, 116736\n" \
  "v_mfma_i32_16x16x64_i8 v[160:163], v[100:103], a[84:87], v[160:163]\n" \
  "buffer_load_dwordx4 v42, s[16:19], s87 offen lds\n" \
  "s_add_u32 m0, s47, 117760\n" \
  "v_mfma_i32_16x16x64_i8 v[192:195], v[96:99], a[72:75], v[192:195]\n" \
  "ds_read_b64_tr_b8 v[124:125], v35 offset:40960\n" \
  "buffer_load_dwordx4 v43, s[16:19], s87 offen lds\n" \
  "s_add_u32 m0, s47, 0\n" \
  "v_mfma_i32_16x16x64_i8 v[192:195], v[100:103], a[88:91], v[192:195]\n" \
  "ds_read_b64_tr_b8 v[126:127], v35 offset:41984\n" \
  "buffer_load_dwordx4 v40, s[16:19], s80 offen lds\n" \
  "s_add_u32 m0, s47, 1024\n" \
  "v_mfma_i32_16x16x64_i8 v[224:227], v[96:99], a[76:79], v[224:227]\n" \
  "buffer_load_dwordx4 v41, s[16:19], s80 offen lds\n" \
  "s_add_u32 m0, s47, 2048\n" \
  "v_mfma_i32_16x16x64_i8 v[224:227], v[100:103], a[92:95], v[224:227]\n" \
  "buffer_load_dwordx4 v42, s[16:19], s80 offen lds\n" \
  "s_add_u32 m0, s47, 3072\n" \
  "s_waitcnt lgkmcnt(14)\n" \
  "v_mfma_i32_16x16x64_i8 v[132:135], v[104:107], a[64:67], v[132:135]\n" \
  "ds_read_b64_tr_b8 v[96:97], v36 offset:32768\n" \
  "buffer_load_dwordx4 v43, s[16:19], s80 offen lds\n" \
  "v_mfma_i32_16x16x64_i8 v[132:135], v[108:111], a[80:83], v[132:135]\n" \
  "ds_read_b64_tr_b8 v[98:99], v36 offset:33792\n" \
  "v_mfma_i32_16x16x64_i8 v[164:167], v[104:107], a[68:71], v[164:167]\n" \
  "v_mfma_i32_16x16x64_i8 v[164:167], v[108:111], a[84:87], v[164:167]\n" \
  "v_mfma_i32_16x16x64_i8 v[196:199], v[104:107], a[72:75], v[196:199]\n" \
  "ds_read_b64_tr_b8 v[100:101], v36 offset:40960\n" \
  "v_mfma_i32_16x16x64_i8 v[196:199], v[108:111], a[88:91], v[196:199]\n" \
  "ds_read_b64_tr_b8 v[102:103], v36 offset:41984\n" \
  "v_mfma_i32_16x16x64_i8 v[228:231], v[104:107], a[76:79], v[228:231]\n" \
  "v_mfma_i32_16x16x64_i8 v[228:231], v[108:111], a[92:95], v[228:231]\n" \
  "s_waitcnt lgkmcnt(10)\n" \
  "v_mfma_i32_16x16x64_i8 v[136:139], v[112:115], a[64:67], v[136:139]\n" \
  "ds_read_b64_tr_b8 v[104:105], v37 offset:32768\n" \
  "v_readlane_b32 s98, v46, 30\n" \
  "v_readlane_b32 s99, v46, 31\n" \
  "v_mfma_i32_16x16x64_i8 v[136:139], v[116:119], a[80:83], v[136:139]\n" \
  "ds_read_b64_tr_b8 v[106:107], v37 offset:33792\n" \
  "v_mfma_i32_16x16x64_i8 v[168:171], v[112:115], a[68:71], v[168:171]\n" \
  "v_mfma_i32_16x16x64_i8 v[168:171], v[116:119], a[84:87], v[168:171]\n" \
  "v_mfma_i32_16x16x64_i8 v[200:203], v[112:115], a[72:75], v[200:203]\n" \
  "ds_read_b64_tr_b8 v[108:109], v37 offset:40960\n" \
  "v_mfma_i32_16x16x64_i8 v[200:203], v[116:119], a[88:91], v[200:203]\n" \
  "ds_read_b64_tr_b8 v[110:111], v37 offset:41984\n" \
  "v_mfma_i32_16x16x64_i8 v[232:235], v[112:115], a[76:79], v[232:235]\n" \
  "v_mfma_i32_16x16x64_i8 v[232:235], v[116:119], a[92:95], v[232:235]\n" \
  "s_waitcnt lgkmcnt(8)\n" \
  "v_mfma_i32_16x16x64_i8 v[140:143], v[120:123], a[64:67], v[140:143]\n" \
  "ds_read_b64_tr_b8 v[112:113], v38 offset:32768\n" \
  "v_mfma_i32_16x16x64_i8 v[140:143], v[124:127], a[80:83], v[140:143]\n" \
  "ds_read_b64_tr_b8 v[114:115], v38 offset:33792\n" \
  "v_mfma_i32_16x16x64_i8 v[172:175], v[120:123], a[68:71], v[172:175]\n" \
  "s_lshl_b32 s98, s98, 21\n" \
  "s_lshl_b32 s99, s99, 21\n" \
  "v_mfma_i32_16x16x64_i8 v[172:175], v[124:127], a[84:87], v[172:175]\n" \
  "s_waitcnt vmcnt(20)\n" \
  "s_barrier\n" \
  "v_mfma_i32_16x16x64_i8 v[204:207], v[120:123], a[72:75], v[204:207]\n" \
  "ds_read_b64_tr_b8 v[116:117], v38 offset:40960\n" \
  "v_mfma_i32_16x16x64_i8 v[204:207], v[124:127], a[88:91], v[204:207]\n" \
  "ds_read_b64_tr_b8 v[118:119], v38 offset:41984\n" \
  "v_mfma_i32_16x16x64_i8 v[236:239], v[120:123], a[76:79], v[236:239]\n" \
  "v_mfma_i32_16x16x64_i8 v[236:239], v[124:127], a[92:95], v[236:239]\n" \
  "s_waitcnt lgkmcnt(8)\n" \
  "v_mfma_i32_16x16x64_i8 v[144:147], v[96:99], a[64:67], v[144:147]\n" \
  "ds_read_b64_tr_b8 v[120:121], v39 offset:32768\n" \
  "v_mfma_i32_16x16x64_i8 v[144:147], v[100:103], a[80:83], v[144:147]\n" \
  "ds_read_b64_tr_b8 v[122:123], v39 offset:33792\n" \
  "v_mfma_i32_16x16x64_i8 v[176:179], v[96:99], a[68:71], v[176:179]\n" \
  "v_mfma_i32_16x16x64_i8 v[176:179], v[100:103], a[84:87], v[176:179]\n" \
  "v_mfma_i32_16x16x64_i8 v[208:211], v[96:99], a[72:75], v[208:211]\n" \
  "ds_read_b64_tr_b8 v[124:125], v39 offset:40960\n" \
  "v_mfma_i32_16x16x64_i8 v[208:211], v[100:103], a[88:91], v[208:211]\n" \
  "ds_read_b64_tr_b8 v[126:127], v39 offset:41984\n" \
  "v_mfma_i32_16x16x64_i8 v[240:243], v[96:99], a[76:79], v[240:243]\n" \
  "v_mfma_i32_16x16x64_i8 v[240:243], v[100:103], a[92:95], v[240:243]\n" \
  "s_waitcnt lgkmcnt(8)\n" \
  "v_mfma_i32_16x16x64_i8 v[148:151], v[104:107], a[64:67], v[148:151]\n" \
  "ds_read_b64_tr_b8 v[96:97], v32 offset:49152\n" \
  "v_mfma_i32_16x16x64_i8 v[148:151], v[108:111], a[80:83], v[148:151]\n" \
  "ds_read_b64_tr_b8 v[98:99], v32 offset:50176\n" \
  "v_mfma_i32_16x16x64_i8 v[180:183], v[104:107], a[68:71], v[180:183]\n" \
  "v_mfma_i32_16x16x64_i8 v[180:183], v[108:111], a[84:87], v[180:183]\n" \
  "v_mfma_i32_16x16x64_i8 v[212:215], v[104:107], a[72:75], v[212:215]\n" \
  "ds_read_b64_tr_b8 v[100:101], v32 offset:57344\n" \
  "v_mfma_i32_16x16x64_i8 v[212:215], v[108:111], a[88:91], v[212:215]\n" \
  "ds_read_b64_tr_b8 v[102:103], v32 offset:58368\n" \
  "v_mfma_i32_16x16x64_i8 v[244:247], v[104:107], a[76:79], v[244:247]\n" \
  "v_mfma_i32_16x16x64_i8 v[244:247], v[108:111], a[92:95], v[244:247]\n" \
  "s_waitcnt lgkmcnt(8)\n" \
  "v_mfma_i32_16x16x64_i8 v[152:155], v[112:115], a[64:67], v[152:155]\n" \
  "ds_read_b64_tr_b8 v[104:105], v33 offset:49152\n" \
  "v_mfma_i32_16x16x64_i8 v[152:155], v[116:119], a[80:83], v[152:155]\n" \
  "ds_read_b64_tr_b8 v[106:107], v33 offset:50176\n" \
  "v_mfma_i32_16x16x64_i8 v[184:187], v[112:115], a[68:71], v[184:187]\n" \
  "v_mfma_i32_16x16x64_i8 v[184:187], v[116:119], a[84:87], v[184:187]\n" \
  "v_mfma_i32_16x16x64_i8 v[216:219], v[112:115], a[72:75], v[216:219]\n" \
  "ds_read_b64_tr_b8 v[108:109], v33 offset:57344\n" \
  "v_mfma_i32_16x16x64_i8 v[216:219], v[116:119], a[88:91], v[216:219]\n" \
  "ds_read_b64_tr_b8 v[110:111], v33 offset:58368\n" \
  "v_mfma_i32_16x16x64_i8 v[248:251], v[112:115], a[76:79], v[248:251]\n" \
  "v_mfma_i32_16x16x64_i8 v[248:251], v[116:119], a[92:95], v[248:251]\n" \
  "s_waitcnt lgkmcnt(8)\n" \
  "v_mfma_i32_16x16x64_i8 v[156:159], v[120:123], a[64:67], v[156:159]\n" \
  "ds_read_b64_tr_b8 v[112:113], v34 offset:49152\n" \
  "v_mfma_i32_16x16x64_i8 v[156:159], v[124:127], a[80:83], v[156:159]\n" \
  "ds_read_b64_tr_b8 v[114:115], v34 offset:50176\n" \
  "v_mfma_i32_16x16x64_i8 v[188:191], v[120:123], a[68:71], v[188:191]\n" \
  "v_mfma_i32_16x16x64_i8 v[188:191], v[124:127], a[84:87], v[188:191]\n" \
  "v_mfma_i32_16x16x64_i8 v[220:223], v[120:123], a[72:75], v[220:223]\n" \
  "ds_read_b64_tr_b8 v[116:117], v34 offset:57344\n" \
  "v_mfma_i32_16x16x64_i8 v[220:223], v[124:127], a[88:91], v[220:223]\n" \
  "ds_read_b64_tr_b8 v[118:119], v34 offset:58368\n" \
  "v_mfma_i32_16x16x64_i8 v[252:255], v[120:123], a[76:79], v[252:255]\n" \
  "v_mfma_i32_16x16x64_i8 v[252:255], v[124:127], a[92:95], v[252:255]\n" \
  "; ---- step 3\n" \
  "s_waitcnt lgkmcnt(8)\n" \
  "v_mfma_i32_16x16x64_i8 v[128:131], v[96:99], a[96:99], v[128:131]\n" \
  "ds_read_b64_tr_b8 v[120:121], v35 offset:49152\n" \
  "v_mfma_i32_16x16x64_i8 v[128:131], v[100:103], a[112:115], v[128:131]\n" \
  "ds_read_b64_tr_b8 v[122:123], v35 offset:50176\n" \
  "v_mfma_i32_16x16x64_i8 v[160:163], v[96:99], a[100:103], v[160:163]\n" \
  "v_mfma_i32_16x16x64_i8 v[160:163], v[100:103], a[116:119], v[160:163]\n" \
  "v_mfma_i32_16x16x64_i8 v[192:195], v[96:99], a[104:107], v[192:195]\n" \
  "ds_read_b64_tr_b8 v[124:125], v35 offset:57344\n" \
  "s_add_u32 m0, s47, 131072\n" \
  "v_mfma_i32_16x16x64_i8 v[192:195], v[100:103], a[120:123], v[192:195]\n" \
  "ds_read_b64_tr_b8 v[126:127], v35 offset:58368\n" \
  "buffer_load_dwordx4 v40, s[16:19], s98 offen lds\n" \
  "s_add_u32 m0, s47, 132096\n" \
  "v_mfma_i32_16x16x64_i8 v[224:227], v[96:99], a[108:111], v[224:227]\n" \
  "buffer_load_dwordx4 v41, s[16:19], s98 offen lds\n" \
  "s_add_u32 m0, s47, 133120\n" \
  "v_mfma_i32_16x16x64_i8 v[224:227], v[100:103], a[124:127], v[224:227]\n" \
  "buffer_load_dwordx4 v42, s[16:19], s98 offen lds\n" \
  "s_add_u32 m0, s47, 134144\n" \
  "s_waitcnt lgkmcnt(8)\n" \
  "v_mfma_i32_16x16x64_i8 v[132:135], v[104:107], a[96:99], v[132:135]\n" \
  "ds_read_b64_tr_b8 v[96:97], v36 offset:49152\n" \
  "buffer_load_dwordx4 v43, s[16:19], s98 offen lds\n" \
  "s_add_u32 m0, s47, 147456\n" \
  "v_mfma_i32_16x16x64_i8 v[132:135], v[108:111], a[112:115], v[132:135]\n" \
  "ds_read_b64_tr_b8 v[98:99], v36 offset:50176\n" \
  "buffer_load_dwordx4 v40, s[16:19], s99 offen lds\n" \
  "s_add_u32 m0, s47, 148480\n" \
  "v_mfma_i32_16x16x64_i8 v[164:167], v[104:107], a[100:103], v[164:167]\n" \
  "buffer_load_dwordx4 v41, s[16:19], s99 offen lds\n" \
  "s_add_u32 m0, s47, 149504\n" \
  "v_mfma_i32_16x16x64_i8 v[164:167], v[108:111], a[116:119], v[164:167]\n" \
  "buffer_load_dwordx4 v42, s[16:19], s99 offen lds\n" \
  "s_add_u32 m0, s47, 150528\n" \
  "v_mfma_i32_16x16x64_i8 v[196:199], v[104:107], a[104:107], v[196:199]\n" \
  "ds_read_b64_tr_b8 v[100:101], v36 offset:57344\n" \
  "buffer_load_dwordx4 v43, s[16:19], s99 offen lds\n" \
  "v_mfma_i32_16x16x64_i8 v[196:199], v[108:111], a[120:123], v[196:199]\n" \
  "ds_read_b64_tr_b8 v[102:103], v36 offset:58368\n" \
  "v_mfma_i32_16x16x64_i8 v[228:231], v[104:107], a[108:111], v[228:231]\n" \
  "v_mfma_i32_16x16x64_i8 v[228:231], v[108:111], a[124:127], v[228:231]\n" \
  "s_waitcnt lgkmcnt(8)\n" \
  "v_mfma_i32_16x16x64_i8 v[136:139], v[112:115], a[96:99], v[136:139]\n" \
  "ds_read_b64_tr_b8 v[104:105], v37 offset:49152\n" \
  "v_mfma_i32_16x16x64_i8 v[136:139], v[116:119], a[112:115], v[136:139]\n" \
  "ds_read_b64_tr_b8 v[106:107], v37 offset:50176\n" \
  "v_mfma_i32_16x16x64_i8 v[168:171], v[112:115], a[100:103], v[168:171]\n" \
  "v_mfma_i32_16x16x64_i8 v[168:171], v[116:119], a[116:119], v[168:171]\n" \
  "v_mfma_i32_16x16x64_i8 v[200:203], v[112:115], a[104:107], v[200:203]\n" \
  "ds_read_b64_tr_b8 v[108:109], v37 offset:57344\n" \
  "v_mfma_i32_16x16x64_i8 v[200:203], v[116:119], a[120:123], v[200:203]\n" \
  "ds_read_b64_tr_b8 v[110:111], v37 offset:58368\n" \
  "v_mfma_i32_16x16x64_i8 v[232:235], v[112:115], a[108:111], v[232:235]\n" \
  "v_mfma_i32_16x16x64_i8 v[232:235], v[116:119], a[124:127], v[232:235]\n" \
  "s_waitcnt lgkmcnt(8)\n" \
  "v_mfma_i32_16x16x64_i8 v[140:143], v[120:123], a[96:99], v[140:143]\n" \
  "ds_read_b64_tr_b8 v[112:113], v38 offset:49152\n" \
  "v_mfma_i32_16x16x64_i8 v[140:143], v[124:127], a[112:115], v[140:143]\n" \
  "ds_read_b64_tr_b8 v[114:115], v38 offset:50176\n" \
  "v_mfma_i32_16x16x64_i8 v[172:175], v[120:123], a[100:103], v[172:175]\n" \
  "v_mfma_i32_16x16x64_i8 v[172:175], v[124:127], a[116:119], v[172:175]\n" \
  "s_cmp_lg_u32 s46, 1\n" \
  "s_cbranch_scc1 Lskip9A_%=\n" \
  "s_waitcnt vmcnt(16)\n" \
  "Lskip9A_%=:\n" \
  "s_waitcnt vmcnt(24)\n" \
  "s_barrier\n" \
  "s_cmp_lg_u32 s46, 1\n" \
  "s_cbranch_scc1 Lskip11A_%=\n" \
  "s_cmp_lg_u32 s11, 0\n" \
  "s_cbranch_scc1 Lskip10A_%=\n" \
  "s_mov_b64 exec, 1\n" \
  "global_store_dword v53, v76, s[36:37] sc0 sc1\n" \
  "s_mov_b64 exec, -1\n" \
  "Lskip10A_%=:\n" \
  "Lskip11A_%=:\n" \
  "v_mfma_i32_16x16x64_i8 v[204:207], v[120:123], a[104:107], v[204:207]\n" \
  "ds_read_b64_tr_b8 v[116:117], v38 offset:57344\n" \
  "v_mfma_i32_16x16x64_i8 v[204:207], v[124:127], a[120:123], v[204:207]\n" \
  "ds_read_b64_tr_b8 v[118:119], v38 offset:58368\n" \
  "v_mfma_i32_16x16x64_i8 v[236:239], v[120:123], a[108:111], v[236:239]\n" \
  "v_mfma_i32_16x16x64_i8 v[236:239], v[124:127], a[124:127], v[236:239]\n" \
  "s_waitcnt lgkmcnt(8)\n" \
  "v_mfma_i32_16x16x64_i8 v[144:147], v[96:99], a[96:99], v[144:147]\n" \
  "ds_read_b64_tr_b8 v[120:121], v39 offset:49152\n" \
  "v_mfma_i32_16x16x64_i8 v[144:147], v[100:103], a[112:115], v[144:147]\n" \
  "ds_read_b64_tr_b8 v[122:123], v39 offset:50176\n" \
  "s_add_u32 m0, s47, 81920\n" \
  "v_mfma_i32_16x16x64_i8 v[176:179], v[96:99], a[100:103], v[176:179]\n" \
  "buffer_load_dwordx4 v40, s[16:19], s85 offen nt lds\n" \
  "v_mfma_i32_16x16x64_i8 v[176:179], v[100:103], a[116:119], v[176:179]\n" \
  "v_mfma_i32_16x16x64_i8 v[208:211], v[96:99], a[104:107], v[208:211]\n" \
  "ds_read_b64_tr_b8 v[124:125], v39 offset:57344\n" \
  "v_mfma_i32_16x16x64_i8 v[208:211], v[100:103], a[120:123], v[208:211]\n" \
  "ds_read_b64_tr_b8 v[126:127], v39 offset:58368\n" \
  "s_add_u32 m0, s47, 82944\n" \
  "v_mfma_i32_16x16x64_i8 v[240:243], v[96:99], a[108:111], v[240:243]\n" \
  "buffer_load_dwordx4 v41, s[16:19], s85 offen nt lds\n" \
  "v_mfma_i32_16x16x64_i8 v[240:243], v[100:103], a[124:127], v[240:243]\n" \
  "s_waitcnt lgkmcnt(8)\n" \
  "v_mfma_i32_16x16x64_i8 v[148:151], v[104:107], a[96:99], v[148:151]\n" \
  "ds_read_b64_tr_b8 v[96:97], v80 offset:0\n" \
  "v_mfma_i32_16x16x64_i8 v[148:151], v[108:111], a[112:115], v[148:151]\n" \
  "ds_read_b64_tr_b8 v[98:99], v80 offset:1024\n" \
  "s_add_u32 m0, s47, 83968\n" \
  "v_mfma_i32_16x16x64_i8 v[180:183], v[104:107], a[100:103], v[180:183]\n" \
  "buffer_load_dwordx4 v42, s[16:19], s85 offen nt lds\n" \
  "v_mfma_i32_16x16x64_i8 v[180:183], v[108:111], a[116:119], v[180:183]\n" \
  "v_mfma_i32_16x16x64_i8 v[212:215], v[104:107], a[104:107], v[212:215]\n" \
  "ds_read_b64_tr_b8 v[100:101], v80 offset:8192\n" \
  "v_mfma_i32_16x16x64_i8 v[212:215], v[108:111], a[120:123], v[212:215]\n" \
  "ds_read_b64_tr_b8 v[102:103], v80 offset:9216\n" \
  "s_add_u32 m0, s47, 84992\n" \
  "v_mfma_i32_16x16x64_i8 v[244:247], v[104:107], a[108:111], v[244:247]\n" \
  "buffer_load_dwordx4 v43, s[16:19], s85 offen nt lds\n" \
  "v_mfma_i32_16x16x64_i8 v[244:247], v[108:111], a[124:127], v[244:247]\n" \
  "s_waitcnt lgkmcnt(8)\n" \
  "v_mfma_i32_16x16x64_i8 v[152:155], v[112:115], a[96:99], v[152:155]\n" \
  "ds_read_b64_tr_b8 v[104:105], v81 offset:0\n" \
  "v_mfma_i32_16x16x64_i8 v[152:155], v[116:119], a[112:115], v[152:155]\n" \
  "ds_read_b64_tr_b8 v[106:107], v81 offset:1024\n" \
  "s_add_u32 m0, s47, 98304\n" \
  "v_mfma_i32_16x16x64_i8 v[184:187], v[112:115], a[100:103], v[184:187]\n" \
  "buffer_load_dwordx4 v40, s[16:19], s86 offen nt lds\n" \
  "v_mfma_i32_16x16x64_i8 v[184:187], v[116:119], a[116:119], v[184:187]\n" \
  "v_mfma_i32_16x16x64_i8 v[216:219], v[112:115], a[104:107], v[216:219]\n" \
  "ds_read_b64_tr_b8 v[108:109], v81 offset:8192\n" \
  "v_mfma_i32_16x16x64_i8 v[216:219], v[116:119], a[120:123], v[216:219]\n" \
  "ds_read_b64_tr_b8 v[110:111], v81 offset:9216\n" \
  "s_add_u32 m0, s47, 99328\n" \
  "v_mfma_i32_16x16x64_i8 v[248:251], v[112:115], a[108:111], v[248:251]\n" \
  "buffer_load_dwordx4 v41, s[16:19], s86 offen nt lds\n" \
  "v_mfma_i32_16x16x64_i8 v[248:251], v[116:119], a[124:127], v[248:251]\n" \
  "s_waitcnt lgkmcnt(8)\n" \
  "v_mfma_i32_16x16x64_i8 v[156:159], v[120:123], a[96:99], v[156:159]\n" \
  "ds_read_b64_tr_b8 v[112:113], v82 offset:0\n" \
  "v_mfma_i32_16x16x64_i8 v[156:159], v[124:127], a[112:115], v[156:159]\n" \
  "ds_read_b64_tr_b8 v[114:115], v82 offset:1024\n" \
  "s_add_u32 m0, s47, 100352\n" \
  "v_mfma_i32_16x16x64_i8 v[188:191], v[120:123], a[100:103], v[188:191]\n" \
  "buffer_load_dwordx4 v42, s[16:19], s86 offen nt lds\n" \
  "v_mfma_i32_16x16x64_i8 v[188:191], v[124:127], a[116:119], v[188:191]\n" \
  "v_mfma_i32_16x16x64_i8 v[220:223], v[120:123], a[104:107], v[220:223]\n" \
  "ds_read_b64_tr_b8 v[116:117], v82 offset:8192\n" \
  "v_mfma_i32_16x16x64_i8 v[220:223], v[124:127], a[120:123], v[220:223]\n" \
  "ds_read_b64_tr_b8 v[118:119], v82 offset:9216\n" \
  "s_add_u32 m0, s47, 101376\n" \
  "v_mfma_i32_16x16x64_i8 v[252:255], v[120:123], a[108:111], v[252:255]\n" \
  "buffer_load_dwordx4 v43, s[16:19], s86 offen nt lds\n" \
  "v_mfma_i32_16x16x64_i8 v[252:255], v[124:127], a[124:127], v[252:255]\n" \
  "s_cmp_lg_u32 s11, 0\n" \
  "s_cbranch_scc1 Lskip14A_%=\n" \
  "global_load_dword v72, v50, s[38:39] sc0 sc1\n" \
  "s_waitcnt vmcnt(0)\n" \
  "v_sub_u32 v72, v72, v76\n" \
  "v_cmp_gt_i32 vcc, 0, v72\n" \
  "s_nop 1\n" \
  "s_andn2_b64 vcc, vcc, s[14:15]\n" \
  "s_cbranch_scc1 Lslow12A_%=\n" \
  "Lback13A_%=:\n" \
  "v_add_u32 v50, 1024, v50\n" \
  "Lskip14A_%=:\n" \
  "s_barrier\n" \
  "buffer_load_dwordx4 a[128:131], v44, s[28:31], s92 offen offset:0 sc0 sc1\n" \
  "buffer_load_dwordx4 a[132:135], v44, s[28:31], s92 offen offset:1024 sc0 sc1\n" \
  "buffer_load_dwordx4 a[136:139], v44, s[28:31], s92 offen offset:2048 sc0 sc1\n" \
  "buffer_load_dwordx4 a[140:143], v44, s[28:31], s92 offen offset:3072 sc0 sc1\n" \
  "buffer_load_dword v92, v52, s[28:31], 0 offen sc0 sc1\n" \
  "v_add_u32 v52, 0x10000, v52\n" \
  "buffer_load_dwordx4 a[144:147], v91, s[28:31], s92 offen offset:0 sc0 sc1\n" \
  "buffer_load_dwordx4 a[148:151], v91, s[28:31], s92 offen offset:1024 sc0 sc1\n" \
  "buffer_load_dwordx4 a[152:155], v91, s[28:31], s92 offen offset:2048 sc0 sc1\n" \
  "buffer_load_dwordx4 a[156:159], v91, s[28:31], s92 offen offset:3072 sc0 sc1\n" \
  "; ---- step 4\n" \
  "s_waitcnt vmcnt(0)\n" \
  "s_waitcnt lgkmcnt(8)\n" \
  "v_mfma_i32_16x16x64_i8 v[128:131], v[96:99], a[128:131], v[128:131]\n" \
  "ds_read_b64_tr_b8 v[120:121], v83 offset:0\n" \
  "v_mfma_i32_16x16x64_i8 v[128:131], v[100:103], a[144:147], v[128:131]\n" \
  "ds_read_b64_tr_b8 v[122:123], v83 offset:1024\n" \
  "buffer_load_dwordx4 a[160:163], v44, s[28:31], s93 offen offset:0 sc0 sc1\n" \
  "v_mfma_i32_16x16x64_i8 v[160:163], v[96:99], a[132:135], v[160:163]\n" \
  "v_mfma_i32_16x16x64_i8 v[160:163], v[100:103], a[148:151], v[160:163]\n" \
  "buffer_load_dwordx4 a[164:167], v44, s[28:31], s93 offen offset:1024 sc0 sc1\n" \
  "v_mfma_i32_16x16x64_i8 v[192:195], v[96:99], a[136:139], v[192:195]\n" \
  "ds_read_b64_tr_b8 v[124:125], v83 offset:8192\n" \
  "v_mfma_i32_16x16x64_i8 v[192:195], v[100:103], a[152:155], v[192:195]\n" \
  "ds_read_b64_tr_b8 v[126:127], v83 offset:9216\n" \
  "buffer_load_dwordx4 a[168:171], v44, s[28:31], s93 offen offset:2048 sc0 sc1\n" \
  "v_mfma_i32_16x16x64_i8 v[224:227], v[96:99], a[140:143], v[224:227]\n" \
  "v_mfma_i32_16x16x64_i8 v[224:227], v[100:103], a[156:159], v[224:227]\n" \
  "s_waitcnt lgkmcnt(8)\n" \
  "v_mfma_i32_16x16x64_i8 v[132:135], v[104:107], a[128:131], v[132:135]\n" \
  "ds_read_b64_tr_b8 v[96:97], v84 offset:0\n" \
  "v_mfma_i32_16x16x64_i8 v[132:135], v[108:111], a[144:147], v[132:135]\n" \
  "ds_read_b64_tr_b8 v[98:99], v84 offset:1024\n" \
  "buffer_load_dwordx4 a[172:175], v44, s[28:31], s93 offen offset:3072 sc0 sc1\n" \
  "v_mfma_i32_16x16x64_i8 v[164:167], v[104:107], a[132:135], v[164:167]\n" \
  "v_mfma_i32_16x16x64_i8 v[164:167], v[108:111], a[148:151], v[164:167]\n" \
  "buffer_load_dwordx4 a[176:179], v91, s[28:31], s93 offen offset:0 sc0 sc1\n" \
  "v_mfma_i32_16x16x64_i8 v[196:199], v[104:107], a[136:139], v[196:199]\n" \
  "ds_read_b64_tr_b8 v[100:101], v84 offset:8192\n" \
  "v_mfma_i32_16x16x64_i8 v[196:199], v[108:111], a[152:155], v[196:199]\n" \
  "ds_read_b64_tr_b8 v[102:103], v84 offset:9216\n" \
  "buffer_load_dwordx4 a[180:183], v91, s[28:31], s93 offen offset:1024 sc0 sc1\n" \
  "v_mfma_i32_16x16x64_i8 v[228:231], v[104:107], a[140:143], v[228:231]\n" \
  "v_mfma_i32_16x16x64_i8 v[228:231], v[108:111], a[156:159], v[228:231]\n" \
  "s_waitcnt lgkmcnt(8)\n" \
  "v_mfma_i32_16x16x64_i8 v[136:139], v[112:115], a[128:131], v[136:139]\n" \
  "ds_read_b64_tr_b8 v[104:105], v85 offset:0\n" \
  "v_mfma_i32_16x16x64_i8 v[136:139], v[116:119], a[144:147], v[136:139]\n" \
  "ds_read_b64_tr_b8 v[106:107], v85 offset:1024\n" \
  "buffer_load_dwordx4 a[184:187], v91, s[28:31], s93 offen offset:2048 sc0 sc1\n" \
  "v_mfma_i32_16x16x64_i8 v[168:171], v[112:115], a[132:135], v[168:171]\n" \
  "v_mfma_i32_16x16x64_i8 v[168:171], v[116:119], a[148:151], v[168:171]\n" \
  "buffer_load_dwordx4 a[188:191], v91, s[28:31], s93 offen offset:3072 sc0 sc1\n" \
  "v_mfma_i32_16x16x64_i8 v[200:203], v[112:115], a[136:139], v[200:203]\n" \
  "ds_read_b64_tr_b8 v[108:109], v85 offset:8192\n" \
  "v_mfma_i32_16x16x64_i8 v[200:203], v[116:119], a[152:155], v[200:203]\n" \
  "ds_read_b64_tr_b8 v[110:111], v85 offset:9216\n" \
  "v_mfma_i32_16x16x64_i8 v[232:235], v[112:115], a[140:143], v[232:235]\n" \
  "v_readlane_b32 s50, v46, s49\n" \
  "v_mfma_i32_16x16x64_i8 v[232:235], v[116:119], a[156:159], v[232:235]\n" \
  "buffer_load_dwordx4 a[192:195], v44, s[28:31], s94 offen offset:0 sc0 sc1\n" \
  "s_waitcnt lgkmcnt(8)\n" \
  "v_mfma_i32_16x16x64_i8 v[140:143], v[120:123], a[128:131], v[140:143]\n" \
  "ds_read_b64_tr_b8 v[112:113], v86 offset:0\n" \
  "v_mfma_i32_16x16x64_i8 v[140:143], v[124:127], a[144:147], v[140:143]\n" \
  "ds_read_b64_tr_b8 v[114:115], v86 offset:1024\n" \
  "buffer_load_dwordx4 a[196:199], v44, s[28:31], s94 offen offset:1024 sc0 sc1\n" \
  "v_mfma_i32_16x16x64_i8 v[172:175], v[120:123], a[132:135], v[172:175]\n" \
  "v_mfma_i32_16x16x64_i8 v[172:175], v[124:127], a[148:151], v[172:175]\n" \
  "buffer_load_dwordx4 a[200:203], v44, s[28:31], s94 offen offset:2048 sc0 sc1\n" \
  "s_waitcnt vmcnt(24)\n" \
  "s_barrier\n" \
  "v_mfma_i32_16x16x64_i8 v[204:207], v[120:123], a[136:139], v[204:207]\n" \
  "ds_read_b64_tr_b8 v[116:117], v86 offset:8192\n" \
  "v_mfma_i32_16x16x64_i8 v[204:207], v[124:127], a[152:155], v[204:207]\n" \
  "ds_read_b64_tr_b8 v[118:119], v86 offset:9216\n" \
  "v_mfma_i32_16x16x64_i8 v[236:239], v[120:123], a[140:143], v[236:239]\n" \
  "s_lshl_b32 s81, s50, 21\n" \
  "s_lshl_b32 s89, s50, 16\n" \
  "s_add_u32 s49, s49, 1\n" \
  "v_mfma_i32_16x16x64_i8 v[236:239], v[124:127], a[156:159], v[236:239]\n" \
  "buffer_load_dwordx4 a[204:207], v44, s[28:31], s94 offen offset:3072 sc0 sc1\n" \
  "s_waitcnt lgkmcnt(8)\n" \
  "v_mfma_i32_16x16x64_i8 v[144:147], v[96:99], a[128:131], v[144:147]\n" \
  "ds_read_b64_tr_b8 v[120:121], v87 offset:0\n" \
  "v_mfma_i32_16x16x64_i8 v[144:147], v[100:103], a[144:147], v[144:147]\n" \
  "ds_read_b64_tr_b8 v[122:123], v87 offset:1024\n" \
  "buffer_load_dwordx4 a[208:211], v91, s[28:31], s94 offen offset:0 sc0 sc1\n" \
  "v_mfma_i32_16x16x64_i8 v[176:179], v[96:99], a[132:135], v[176:179]\n" \
  "v_readlane_b32 s50, v46, s49\n" \
  "v_mfma_i32_16x16x64_i8 v[176:179], v[100:103], a[148:151], v[176:179]\n" \
  "buffer_load_dwordx4 a[212:215], v91, s[28:31], s94 offen offset:1024 sc0 sc1\n" \
  "v_mfma_i32_16x16x64_i8 v[208:211], v[96:99], a[136:139], v[208:211]\n" \
  "ds_read_b64_tr_b8 v[124:125], v87 offset:8192\n" \
  "v_mfma_i32_16x16x64_i8 v[208:211], v[100:103], a[152:155], v[208:211]\n" \
  "ds_read_b64_tr_b8 v[126:127], v87 offset:9216\n" \
  "v_mfma_i32_16x16x64_i8 v[240:243], v[96:99], a[140:143], v[240:243]\n" \
  "v_mfma_i32_16x16x64_i8 v[240:243], v[100:103], a[156:159], v[240:243]\n" \
  "buffer_load_dwordx4 a[216:219], v91, s[28:31], s94 offen offset:2048 sc0 sc1\n" \
  "s_waitcnt lgkmcnt(8)\n" \
  "v_mfma_i32_16x16x64_i8 v[148:151], v[104:107], a[128:131], v[148:151]\n" \
  "ds_read_b64_tr_b8 v[96:97], v80 offset:16384\n" \
  "v_mfma_i32_16x16x64_i8 v[148:151], v[108:111], a[144:147], v[148:151]\n" \
  "ds_read_b64_tr_b8 v[98:99], v80 offset:17408\n" \
  "buffer_load_dwordx4 a[220:223], v91, s[28:31], s94 offen offset:3072 sc0 sc1\n" \
  "v_mfma_i32_16x16x64_i8 v[180:183], v[104:107], a[132:135], v[180:183]\n" \
  "s_lshl_b32 s82, s50, 21\n" \
  "s_lshl_b32 s90, s50, 16\n" \
  "s_add_u32 s49, s49, 1\n" \
  "v_mfma_i32_16x16x64_i8 v[180:183], v[108:111], a[148:151], v[180:183]\n" \
  "v_mfma_i32_16x16x64_i8 v[212:215], v[104:107], a[136:139], v[212:215]\n" \
  "ds_read_b64_tr_b8 v[100:101], v80 offset:24576\n" \
  "v_mfma_i32_16x16x64_i8 v[212:215], v[108:111], a[152:155], v[212:215]\n" \
  "ds_read_b64_tr_b8 v[102:103], v80 offset:25600\n" \
  "v_mfma_i32_16x16x64_i8 v[244:247], v[104:107], a[140:143], v[244:247]\n" \
  "v_mfma_i32_16x16x64_i8 v[244:247], v[108:111], a[156:159], v[244:247]\n" \
  "s_waitcnt lgkmcnt(8)\n" \
  "v_mfma_i32_16x16x64_i8 v[152:155], v[112:115], a[128:131], v[152:155]\n" \
  "ds_read_b64_tr_b8 v[104:105], v81 offset:16384\n" \
  "s_add_u32 m0, s47, 16384\n" \
  "v_mfma_i32_16x16x64_i8 v[152:155], v[116:119], a[144:147], v[152:155]\n" \
  "ds_read_b64_tr_b8 v[106:107], v81 offset:17408\n" \
  "buffer_load_dwordx4 v40, s[16:19], s81 offen lds\n" \
  "s_add_u32 m0, s47, 17408\n" \
  "v_mfma_i32_16x16x64_i8 v[184:187], v[112:115], a[132:135], v[184:187]\n" \
  "buffer_load_dwordx4 v41, s[16:19], s81 offen lds\n" \
  "s_add_u32 m0, s47, 18432\n" \
  "v_mfma_i32_16x16x64_i8 v[184:187], v[116:119], a[148:151], v[184:187]\n" \
  "buffer_load_dwordx4 v42, s[16:19], s81 offen lds\n" \
  "s_add_u32 m0, s47, 19456\n" \
  "v_mfma_i32_16x16x64_i8 v[216:219], v[112:115], a[136:139], v[216:219]\n" \
  "ds_read_b64_tr_b8 v[108:109], v81 offset:24576\n" \
  "buffer_load_dwordx4 v43, s[16:19], s81 offen lds\n" \
  "s_add_u32 m0, s47, 32768\n" \
  "v_mfma_i32_16x16x64_i8 v[216:219], v[116:119], a[152:155], v[216:219]\n" \
  "ds_read_b64_tr_b8 v[110:111], v81 offset:25600\n" \
  "buffer_load_dwordx4 v40, s[16:19], s82 offen lds\n" \
  "s_add_u32 m0, s47, 33792\n" \
  "v_mfma_i32_16x16x64_i8 v[248:251], v[112:115], a[140:143], v[248:251]\n" \
  "buffer_load_dwordx4 v41, s[16:19], s82 offen lds\n" \
  "s_add_u32 m0, s47, 34816\n" \
  "v_mfma_i32_16x16x64_i8 v[248:251], v[116:119], a[156:159], v[248:251]\n" \
  "buffer_load_dwordx4 v42, s[16:19], s82 offen lds\n" \
  "s_add_u32 m0, s47, 35840\n" \
  "s_waitcnt lgkmcnt(8)\n" \
  "v_mfma_i32_16x16x64_i8 v[156:159], v[120:123], a[128:131], v[156:159]\n" \
  "ds_read_b64_tr_b8 v[112:113], v82 offset:16384\n" \
  "buffer_load_dwordx4 v43, s[16:19], s82 offen lds\n" \
  "v_mfma_i32_16x16x64_i8 v[156:159], v[124:127], a[144:147], v[156:159]\n" \
  "ds_read_b64_tr_b8 v[114:115], v82 offset:17408\n" \
  "v_mfma_i32_16x16x64_i8 v[188:191], v[120:123], a[132:135], v[188:191]\n" \
  "v_mfma_i32_16x16x64_i8 v[188:191], v[124:127], a[148:151], v[188:191]\n" \
  "v_mfma_i32_16x16x64_i8 v[220:223], v[120:123], a[136:139], v[220:223]\n" \
  "ds_read_b64_tr_b8 v[116:117], v82 offset:24576\n" \
  "v_mfma_i32_16x16x64_i8 v[220:223], v[124:127], a[152:155], v[220:223]\n" \
  "ds_read_b64_tr_b8 v[118:119], v82 offset:25600\n" \
  "v_mfma_i32_16x16x64_i8 v[252:255], v[120:123], a[140:143], v[252:255]\n" \
  "v_mfma_i32_16x16x64_i8 v[252:255], v[124:127], a[156:159], v[252:255]\n" \
  "; ---- step 5\n" \
  "s_waitcnt vmcnt(16)\n" \
  "s_waitcnt lgkmcnt(8)\n" \
  "v_mfma_i32_16x16x64_i8 v[128:131], v[96:99], a[160:163], v[128:131]\n" \
  "ds_read_b64_tr_b8 v[120:121], v83 offset:16384\n" \
  "v_mfma_i32_16x16x64_i8 v[128:131], v[100:103], a[176:179], v[128:131]\n" \
  "ds_read_b64_tr_b8 v[122:123], v83 offset:17408\n" \
  "v_mfma_i32_16x16x64_i8 v[160:163], v[96:99], a[164:167], v[160:163]\n" \
  "v_mfma_i32_16x16x64_i8 v[160:163], v[100:103], a[180:183], v[160:163]\n" \
  "v_mfma_i32_16x16x64_i8 v[192:195], v[96:99], a[168:171], v[192:195]\n" \
  "ds_read_b64_tr_b8 v[124:125], v83 offset:24576\n" \
  "v_mfma_i32_16x16x64_i8 v[192:195], v[100:103], a[184:187], v[192:195]\n" \
  "ds_read_b64_tr_b8 v[126:127], v83 offset:25600\n" \
  "v_mfma_i32_16x16x64_i8 v[224:227], v[96:99], a[172:175], v[224:227]\n" \
  "v_mfma_i32_16x16x64_i8 v[224:227], v[100:103], a[188:191], v[224:227]\n" \
  "s_waitcnt lgkmcnt(8)\n" \
  "v_mfma_i32_16x16x64_i8 v[132:135], v[104:107], a[160:163], v[132:135]\n" \
  "ds_read_b64_tr_b8 v[96:97], v84 offset:16384\n" \
  "v_mfma_i32_16x16x64_i8 v[132:135], v[108:111], a[176:179], v[132:135]\n" \
  "ds_read_b64_tr_b8 v[98:99], v84 offset:17408\n" \
  "v_mfma_i32_16x16x64_i8 v[164:167], v[104:107], a[164:167], v[164:167]\n" \
  "v_mfma_i32_16x16x64_i8 v[164:167], v[108:111], a[180:183], v[164:167]\n" \
  "v_mfma_i32_16x16x64_i8 v[196:199], v[104:107], a[168:171], v[196:199]\n" \
  "ds_read_b64_tr_b8 v[100:101], v84 offset:24576\n" \
  "v_mfma_i32_16x16x64_i8 v[196:199], v[108:111], a[184:187], v[196:199]\n" \
  "ds_read_b64_tr_b8 v[102:103], v84 offset:25600\n" \
  "v_mfma_i32_16x16x64_i8 v[228:231], v[104:107], a[172:175], v[228:231]\n" \
  "v_mfma_i32_16x16x64_i8 v[228:231], v[108:111], a[188:191], v[228:231]\n" \
  "s_waitcnt lgkmcnt(8)\n" \
  "v_mfma_i32_16x16x64_i8 v[136:139], v[112:115], a[160:163], v[136:139]\n" \
  "ds_read_b64_tr_b8 v[104:105], v85 offset:16384\n" \
  "v_mfma_i32_16x16x64_i8 v[136:139], v[116:119], a[176:179], v[136:139]\n" \
  "ds_read_b64_tr_b8 v[106:107], v85 offset:17408\n" \
  "buffer_load_dwordx4 a[224:227], v44, s[28:31], s95 offen offset:0 sc0 sc1\n" \
  "v_mfma_i32_16x16x64_i8 v[168:171], v[112:115], a[164:167], v[168:171]\n" \
  "v_mfma_i32_16x16x64_i8 v[168:171], v[116:119], a[180:183], v[168:171]\n" \
  "buffer_load_dwordx4 a[228:231], v44, s[28:31], s95 offen offset:1024 sc0 sc1\n" \
  "v_mfma_i32_16x16x64_i8 v[200:203], v[112:115], a[168:171], v[200:203]\n" \
  "ds_read_b64_tr_b8 v[108:109], v85 offset:24576\n" \
  "v_mfma_i32_16x16x64_i8 v[200:203], v[116:119], a[184:187], v[200:203]\n" \
  "ds_read_b64_tr_b8 v[110:111], v85 offset:25600\n" \
  "v_mfma_i32_16x16x64_i8 v[232:235], v[112:115], a[172:175], v[232:235]\n" \
  "v_mfma_i32_16x16x64_i8 v[232:235], v[116:119], a[188:191], v[232:235]\n" \
  "buffer_load_dwordx4 a[232:235], v44, s[28:31], s95 offen offset:2048 sc0 sc1\n" \
  "s_waitcnt lgkmcnt(8)\n" \
  "v_mfma_i32_16x16x64_i8 v[140:143], v[120:123], a[160:163], v[140:143]\n" \
  "ds_read_b64_tr_b8 v[112:113], v86 offset:16384\n" \
  "v_mfma_i32_16x16x64_i8 v[140:143], v[124:127], a[176:179], v[140:143]\n" \
  "ds_read_b64_tr_b8 v[114:115], v86 offset:17408\n" \
  "buffer_load_dwordx4 a[236:239], v44, s[28:31], s95 offen offset:3072 sc0 sc1\n" \
  "v_mfma_i32_16x16x64_i8 v[172:175], v[120:123], a[164:167], v[172:175]\n" \
  "v_mfma_i32_16x16x64_i8 v[172:175], v[124:127], a[180:183], v[172:175]\n" \
  "s_waitcnt vmcnt(37)\n" \
  "s_barrier\n" \
  "v_mfma_i32_16x16x64_i8 v[204:207], v[120:123], a[168:171], v[204:207]\n" \
  "ds_read_b64_tr_b8 v[116:117], v86 offset:24576\n" \
  "v_mfma_i32_16x16x64_i8 v[204:207], v[124:127], a[184:187], v[204:207]\n" \
  "ds_read_b64_tr_b8 v[118:119], v86 offset:25600\n" \
  "buffer_load_dwordx4 a[240:243], v91, s[28:31], s95 offen offset:0 sc0 sc1\n" \
  "v_mfma_i32_16x16x64_i8 v[236:239], v[120:123], a[172:175], v[236:239]\n" \
  "v_mfma_i32_16x16x64_i8 v[236:239], v[124:127], a[188:191], v[236:239]\n" \
  "buffer_load_dwordx4 a[244:247], v91, s[28:31], s95 offen offset:1024 sc0 sc1\n" \
  "s_waitcnt lgkmcnt(8)\n" \
  "v_mfma_i32_16x16x64_i8 v[144:147], v[96:99], a[160:163], v[144:147]\n" \
  "ds_read_b64_tr_b8 v[120:121], v87 offset:16384\n" \
  "v_mfma_i32_16x16x64_i8 v[144:147], v[100:103], a[176:179], v[144:147]\n" \
  "ds_read_b64_tr_b8 v[122:123], v87 offset:17408\n" \
  "v_mfma_i32_16x16x64_i8 v[176:179], v[96:99], a[164:167], v[176:179]\n" \
  "v_mfma_i32_16x16x64_i8 v[176:179], v[100:103], a[180:183], v[176:179]\n" \
  "buffer_load_dwordx4 a[248:251], v91, s[28:31], s95 offen offset:2048 sc0 sc1\n" \
  "v_mfma_i32_16x16x64_i8 v[208:211], v[96:99], a[168:171], v[208:211]\n" \
  "ds_read_b64_tr_b8 v[124:125], v87 offset:24576\n" \
  "v_mfma_i32_16x16x64_i8 v[208:211], v[100:103], a[184:187], v[208:211]\n" \
  "ds_read_b64_tr_b8 v[126:127], v87 offset:25600\n" \
  "buffer_load_dwordx4 a[252:255], v91, s[28:31], s95 offen offset:3072 sc0 sc1\n" \
  "v_mfma_i32_16x16x64_i8 v[240:243], v[96:99], a[172:175], v[240:243]\n" \
  "v_mfma_i32_16x16x64_i8 v[240:243], v[100:103], a[188:191], v[240:243]\n" \
  "s_waitcnt lgkmcnt(8)\n" \
  "v_mfma_i32_16x16x64_i8 v[148:151], v[104:107], a[160:163], v[148:151]\n" \
  "ds_read_b64_tr_b8 v[96:97], v80 offset:32768\n" \
  "v_mfma_i32_16x16x64_i8 v[148:151], v[108:111], a[176:179], v[148:151]\n" \
  "ds_read_b64_tr_b8 v[98:99], v80 offset:33792\n" \
  "buffer_load_dwordx4 a[0:3], v44, s[28:31], s88 offen offset:0 sc0 sc1\n" \
  "v_mfma_i32_16x16x64_i8 v[180:183], v[104:107], a[164:167], v[180:183]\n" \
  "v_mfma_i32_16x16x64_i8 v[180:183], v[108:111], a[180:183], v[180:183]\n" \
  "buffer_load_dwordx4 a[4:7], v44, s[28:31], s88 offen offset:1024 sc0 sc1\n" \
  "v_mfma_i32_16x16x64_i8 v[212:215], v[104:107], a[168:171], v[212:215]\n" \
  "ds_read_b64_tr_b8 v[100:101], v80 offset:40960\n" \
  "v_mfma_i32_16x16x64_i8 v[212:215], v[108:111], a[184:187], v[212:215]\n" \
  "ds_read_b64_tr_b8 v[102:103], v80 offset:41984\n" \
  "v_mfma_i32_16x16x64_i8 v[244:247], v[104:107], a[172:175], v[244:247]\n" \
  "v_mfma_i32_16x16x64_i8 v[244:247], v[108:111], a[188:191], v[244:247]\n" \
  "buffer_load_dwordx4 a[8:11], v44, s[28:31], s88 offen offset:2048 sc0 sc1\n" \
  "s_waitcnt lgkmcnt(8)\n" \
  "v_mfma_i32_16x16x64_i8 v[152:155], v[112:115], a[160:163], v[152:155]\n" \
  "ds_read_b64_tr_b8 v[104:105], v81 offset:32768\n" \
  "v_mfma_i32_16x16x64_i8 v[152:155], v[116:119], a[176:179], v[152:155]\n" \
  "ds_read_b64_tr_b8 v[106:107], v81 offset:33792\n" \
  "buffer_load_dwordx4 a[12:15], v44, s[28:31], s88 offen offset:3072 sc0 sc1\n" \
  "v_mfma_i32_16x16x64_i8 v[184:187], v[112:115], a[164:167], v[184:187]\n" \
  "v_mfma_i32_16x16x64_i8 v[184:187], v[116:119], a[180:183], v[184:187]\n" \
  "v_mfma_i32_16x16x64_i8 v[216:219], v[112:115], a[168:171], v[216:219]\n" \
  "ds_read_b64_tr_b8 v[108:109], v81 offset:40960\n" \
  "v_mfma_i32_16x16x64_i8 v[216:219], v[116:119], a[184:187], v[216:219]\n" \
  "ds_read_b64_tr_b8 v[110:111], v81 offset:41984\n" \
  "buffer_load_dwordx4 a[16:19], v91, s[28:31], s88 offen offset:0 sc0 sc1\n" \
  "v_mfma_i32_16x16x64_i8 v[248:251], v[112:115], a[172:175], v[248:251]\n" \
  "v_mfma_i32_16x16x64_i8 v[248:251], v[116:119], a[188:191], v[248:251]\n" \
  "buffer_load_dwordx4 a[20:23], v91, s[28:31], s88 offen offset:1024 sc0 sc1\n" \
  "s_waitcnt lgkmcnt(8)\n" \
  "v_mfma_i32_16x16x64_i8 v[156:159], v[120:123], a[160:163], v[156:159]\n" \
  "ds_read_b64_tr_b8 v[112:113], v82 offset:32768\n" \
  "v_mfma_i32_16x16x64_i8 v[156:159], v[124:127], a[176:179], v[156:159]\n" \
  "ds_read_b64_tr_b8 v[114:115], v82 offset:33792\n" \
  "v_mfma_i32_16x16x64_i8 v[188:191], v[120:123], a[164:167], v[188:191]\n" \
  "v_mfma_i32_16x16x64_i8 v[188:191], v[124:127], a[180:183], v[188:191]\n" \
  "buffer_load_dwordx4 a[24:27], v91, s[28:31], s88 offen offset:2048 sc0 sc1\n" \
  "v_mfma_i32_16x16x64_i8 v[220:223], v[120:123], a[168:171], v[220:223]\n" \
  "ds_read_b64_tr_b8 v[116:117], v82 offset:40960\n" \
  "v_mfma_i32_16x16x64_i8 v[220:223], v[124:127], a[184:187], v[220:223]\n" \
  "ds_read_b64_tr_b8 v[118:119], v82 offset:41984\n" \
  "buffer_load_dwordx4 a[28:31], v91, s[28:31], s88 offen offset:3072 sc0 sc1\n" \
  "v_mfma_i32_16x16x64_i8 v[252:255], v[120:123], a[172:175], v[252:255]\n" \
  "v_mfma_i32_16x16x64_i8 v[252:255], v[124:127], a[188:191], v[252:255]\n" \
  "; ---- step 6\n" \
  "s_cmp_lg_u32 s11, 0\n" \
  "s_cbranch_scc1 Lskip16A_%=\n" \
  "global_load_dword v72, v50, s[38:39] sc0 sc1\n" \
  "Lskip16A_%=:\n" \
  "s_cmp_lg_u32 s46, 2\n" \
  "s_cbranch_scc1 Lskip15A_%=\n" \
  "s_waitcnt vmcnt(63)\n" \
  "global_store_dwordx4 v49, v[64:67], s[34:35] sc0 sc1\n" \
  "s_nop 1\n" \
  "global_store_dwordx4 v79, v[68:71], s[34:35] sc0 sc1\n" \
  "s_nop 1\n" \
  "Lskip15A_%=:\n" \
  "s_waitcnt vmcnt(24)\n" \
  "s_waitcnt lgkmcnt(8)\n" \
  "v_mfma_i32_16x16x64_i8 v[128:131], v[96:99], a[192:195], v[128:131]\n" \
  "ds_read_b64_tr_b8 v[120:121], v83 offset:32768\n" \
  "v_mfma_i32_16x16x64_i8 v[128:131], v[100:103], a[208:211], v[128:131]\n" \
  "ds_read_b64_tr_b8 v[122:123], v83 offset:33792\n" \
  "v_mfma_i32_16x16x64_i8 v[160:163], v[96:99], a[196:199], v[160:163]\n" \
  "v_mfma_i32_16x16x64_i8 v[160:163], v[100:103], a[212:215], v[160:163]\n" \
  "v_mfma_i32_16x16x64_i8 v[192:195], v[96:99], a[200:203], v[192:195]\n" \
  "ds_read_b64_tr_b8 v[124:125], v83 offset:40960\n" \
  "v_mfma_i32_16x16x64_i8 v[192:195], v[100:103], a[216:219], v[192:195]\n" \
  "ds_read_b64_tr_b8 v[126:127], v83 offset:41984\n" \
  "v_mfma_i32_16x16x64_i8 v[224:227], v[96:99], a[204:207], v[224:227]\n" \
  "v_mfma_i32_16x16x64_i8 v[224:227], v[100:103], a[220:223], v[224:227]\n" \
  "s_waitcnt lgkmcnt(8)\n" \
  "v_mfma_i32_16x16x64_i8 v[132:135], v[104:107], a[192:195], v[132:135]\n" \
  "ds_read_b64_tr_b8 v[96:97], v84 offset:32768\n" \
  "v_mfma_i32_16x16x64_i8 v[132:135], v[108:111], a[208:211], v[132:135]\n" \
  "ds_read_b64_tr_b8 v[98:99], v84 offset:33792\n" \
  "v_mfma_i32_16x16x64_i8 v[164:167], v[104:107], a[196:199], v[164:167]\n" \
  "v_mfma_i32_16x16x64_i8 v[164:167], v[108:111], a[212:215], v[164:167]\n" \
  "v_mfma_i32_16x16x64_i8 v[196:199], v[104:107], a[200:203], v[196:199]\n" \
  "ds_read_b64_tr_b8 v[100:101], v84 offset:40960\n" \
  "v_mfma_i32_16x16x64_i8 v[196:199], v[108:111], a[216:219], v[196:199]\n" \
  "ds_read_b64_tr_b8 v[102:103], v84 offset:41984\n" \
  "v_mfma_i32_16x16x64_i8 v[228:231], v[104:107], a[204:207], v[228:231]\n" \
  "v_mfma_i32_16x16x64_i8 v[228:231], v[108:111], a[220:223], v[228:231]\n" \
  "s_waitcnt lgkmcnt(8)\n" \
  "v_mfma_i32_16x16x64_i8 v[136:139], v[112:115], a[192:195], v[136:139]\n" \
  "ds_read_b64_tr_b8 v[104:105], v85 offset:32768\n" \
  "v_mfma_i32_16x16x64_i8 v[136:139], v[116:119], a[208:211], v[136:139]\n" \
  "ds_read_b64_tr_b8 v[106:107], v85 offset:33792\n" \
  "v_mfma_i32_16x16x64_i8 v[168:171], v[112:115], a[196:199], v[168:171]\n" \
  "v_mfma_i32_16x16x64_i8 v[168:171], v[116:119], a[212:215], v[168:171]\n" \
  "v_mfma_i32_16x16x64_i8 v[200:203], v[112:115], a[200:203], v[200:203]\n" \
  "ds_read_b64_tr_b8 v[108:109], v85 offset:40960\n" \
  "v_mfma_i32_16x16x64_i8 v[200:203], v[116:119], a[216:219], v[200:203]\n" \
  "ds_read_b64_tr_b8 v[110:111], v85 offset:41984\n" \
  "v_mfma_i32_16x16x64_i8 v[232:235], v[112:115], a[204:207], v[232:235]\n" \
  "v_mfma_i32_16x16x64_i8 v[232:235], v[116:119], a[220:223], v[232:235]\n" \
  "s_waitcnt lgkmcnt(8)\n" \
  "v_mfma_i32_16x16x64_i8 v[140:143], v[120:123], a[192:195], v[140:143]\n" \
  "ds_read_b64_tr_b8 v[112:113], v86 offset:32768\n" \
  "v_mfma_i32_16x16x64_i8 v[140:143], v[124:127], a[208:211], v[140:143]\n" \
  "ds_read_b64_tr_b8 v[114:115], v86 offset:33792\n" \
  "v_mfma_i32_16x16x64_i8 v[172:175], v[120:123], a[196:199], v[172:175]\n" \
  "v_mfma_i32_16x16x64_i8 v[172:175], v[124:127], a[212:215], v[172:175]\n" \
  "s_waitcnt vmcnt(63)\n" \
  "s_barrier\n" \
  "v_mfma_i32_16x16x64_i8 v[204:207], v[120:123], a[200:203], v[204:207]\n" \
  "ds_read_b64_tr_b8 v[116:117], v86 offset:40960\n" \
  "v_mfma_i32_16x16x64_i8 v[204:207], v[124:127], a[216:219], v[204:207]\n" \
  "ds_read_b64_tr_b8 v[118:119], v86 offset:41984\n" \
  "v_mfma_i32_16x16x64_i8 v[236:239], v[120:123], a[204:207], v[236:239]\n" \
  "v_mfma_i32_16x16x64_i8 v[236:239], v[124:127], a[220:223], v[236:239]\n" \
  "s_waitcnt lgkmcnt(8)\n" \
  "v_mfma_i32_16x16x64_i8 v[144:147], v[96:99], a[192:195], v[144:147]\n" \
  "ds_read_b64_tr_b8 v[120:121], v87 offset:32768\n" \
  "v_mfma_i32_16x16x64_i8 v[144:147], v[100:103], a[208:211], v[144:147]\n" \
  "ds_read_b64_tr_b8 v[122:123], v87 offset:33792\n" \
  "v_mfma_i32_16x16x64_i8 v[176:179], v[96:99], a[196:199], v[176:179]\n" \
  "v_mfma_i32_16x16x64_i8 v[176:179], v[100:103], a[212:215], v[176:179]\n" \
  "v_mfma_i32_16x16x64_i8 v[208:211], v[96:99], a[200:203], v[208:211]\n" \
  "ds_read_b64_tr_b8 v[124:125], v87 offset:40960\n" \
  "v_mfma_i32_16x16x64_i8 v[208:211], v[100:103], a[216:219], v[208:211]\n" \
  "ds_read_b64_tr_b8 v[126:127], v87 offset:41984\n" \
  "v_mfma_i32_16x16x64_i8 v[240:243], v[96:99], a[204:207], v[240:243]\n" \
  "v_mfma_i32_16x16x64_i8 v[240:243], v[100:103], a[220:223], v[240:243]\n" \
  "s_waitcnt lgkmcnt(8)\n" \
  "v_mfma_i32_16x16x64_i8 v[148:151], v[104:107], a[192:195], v[148:151]\n" \
  "ds_read_b64_tr_b8 v[96:97], v80 offset:49152\n" \
  "v_mfma_i32_16x16x64_i8 v[148:151], v[108:111], a[208:211], v[148:151]\n" \
  "ds_read_b64_tr_b8 v[98:99], v80 offset:50176\n" \
  "v_mfma_i32_16x16x64_i8 v[180:183], v[104:107], a[196:199], v[180:183]\n" \
  "v_mfma_i32_16x16x64_i8 v[180:183], v[108:111], a[212:215], v[180:183]\n" \
  "v_mfma_i32_16x16x64_i8 v[212:215], v[104:107], a[200:203], v[212:215]\n" \
  "ds_read_b64_tr_b8 v[100:101], v80 offset:57344\n" \
  "v_mfma_i32_16x16x64_i8 v[212:215], v[108:111], a[216:219], v[212:215]\n" \
  "ds_read_b64_tr_b8 v[102:103], v80 offset:58368\n" \
  "v_mfma_i32_16x16x64_i8 v[244:247], v[104:107], a[204:207], v[244:247]\n" \
  "v_mfma_i32_16x16x64_i8 v[244:247], v[108:111], a[220:223], v[244:247]\n" \
  "s_waitcnt lgkmcnt(8)\n" \
  "v_mfma_i32_16x16x64_i8 v[152:155], v[112:115], a[192:195], v[152:155]\n" \
  "ds_read_b64_tr_b8 v[104:105], v81 offset:49152\n" \
  "v_mfma_i32_16x16x64_i8 v[152:155], v[116:119], a[208:211], v[152:155]\n" \
  "ds_read_b64_tr_b8 v[106:107], v81 offset:50176\n" \
  "v_mfma_i32_16x16x64_i8 v[184:187], v[112:115], a[196:199], v[184:187]\n" \
  "v_mfma_i32_16x16x64_i8 v[184:187], v[116:119], a[212:215], v[184:187]\n" \
  "v_mfma_i32_16x16x64_i8 v[216:219], v[112:115], a[200:203], v[216:219]\n" \
  "ds_read_b64_tr_b8 v[108:109], v81 offset:57344\n" \
  "v_mfma_i32_16x16x64_i8 v[216:219], v[116:119], a[216:219], v[216:219]\n" \
  "ds_read_b64_tr_b8 v[110:111], v81 offset:58368\n" \
  "v_mfma_i32_16x16x64_i8 v[248:251], v[112:115], a[204:207], v[248:251]\n" \
  "v_mfma_i32_16x16x64_i8 v[248:251], v[116:119], a[220:223], v[248:251]\n" \
  "s_waitcnt lgkmcnt(8)\n" \
  "v_mfma_i32_16x16x64_i8 v[156:159], v[120:123], a[192:195], v[156:159]\n" \
  "ds_read_b64_tr_b8 v[112:113], v82 offset:49152\n" \
  "v_mfma_i32_16x16x64_i8 v[156:159], v[124:127], a[208:211], v[156:159]\n" \
  "ds_read_b64_tr_b8 v[114:115], v82 offset:50176\n" \
  "v_mfma_i32_16x16x64_i8 v[188:191], v[120:123], a[196:199], v[188:191]\n" \
  "v_mfma_i32_16x16x64_i8 v[188:191], v[124:127], a[212:215], v[188:191]\n" \
  "v_mfma_i32_16x16x64_i8 v[220:223], v[120:123], a[200:203], v[220:223]\n" \
  "ds_read_b64_tr_b8 v[116:117], v82 offset:57344\n" \
  "v_mfma_i32_16x16x64_i8 v[220:223], v[124:127], a[216:219], v[220:223]\n" \
  "ds_read_b64_tr_b8 v[118:119], v82 offset:58368\n" \
  "v_mfma_i32_16x16x64_i8 v[252:255], v[120:123], a[204:207], v[252:255]\n" \
  "v_mfma_i32_16x16x64_i8 v[252:255], v[124:127], a[220:223], v[252:255]\n" \
  "; ---- step 7\n" \
  "s_waitcnt vmcnt(8)\n" \
  "s_waitcnt lgkmcnt(8)\n" \
  "v_mfma_i32_16x16x64_i8 v[128:131], v[96:99], a[224:227], v[128:131]\n" \
  "ds_read_b64_tr_b8 v[120:121], v83 offset:49152\n" \
  "v_mfma_i32_16x16x64_i8 v[128:131], v[100:103], a[240:243], v[128:131]\n" \
  "ds_read_b64_tr_b8 v[122:123], v83 offset:50176\n" \
  "v_mfma_i32_16x16x64_i8 v[160:163], v[96:99], a[228:231], v[160:163]\n" \
  "v_mfma_i32_16x16x64_i8 v[160:163], v[100:103], a[244:247], v[160:163]\n" \
  "v_mfma_i32_16x16x64_i8 v[192:195], v[96:99], a[232:235], v[192:195]\n" \
  "ds_read_b64_tr_b8 v[124:125], v83 offset:57344\n" \
  "v_mfma_i32_16x16x64_i8 v[192:195], v[100:103], a[248:251], v[192:195]\n" \
  "ds_read_b64_tr_b8 v[126:127], v83 offset:58368\n" \
  "v_mfma_i32_16x16x64_i8 v[224:227], v[96:99], a[236:239], v[224:227]\n" \
  "v_mfma_i32_16x16x64_i8 v[224:227], v[100:103], a[252:255], v[224:227]\n" \
  "s_waitcnt lgkmcnt(8)\n" \
  "v_mfma_i32_16x16x64_i8 v[132:135], v[104:107], a[224:227], v[132:135]\n" \
  "ds_read_b64_tr_b8 v[96:97], v84 offset:49152\n" \
  "v_mfma_i32_16x16x64_i8 v[132:135], v[108:111], a[240:243], v[132:135]\n" \
  "ds_read_b64_tr_b8 v[98:99], v84 offset:50176\n" \
  "v_mfma_i32_16x16x64_i8 v[164:167], v[104:107], a[228:231], v[164:167]\n" \
  "v_mfma_i32_16x16x64_i8 v[164:167], v[108:111], a[244:247], v[164:167]\n" \
  "v_mfma_i32_16x16x64_i8 v[196:199], v[104:107], a[232:235], v[196:199]\n" \
  "ds_read_b64_tr_b8 v[100:101], v84 offset:57344\n" \
  "v_mfma_i32_16x16x64_i8 v[196:199], v[108:111], a[248:251], v[196:199]\n" \
  "ds_read_b64_tr_b8 v[102:103], v84 offset:58368\n" \
  "v_mfma_i32_16x16x64_i8 v[228:231], v[104:107], a[236:239], v[228:231]\n" \
  "v_readlane_b32 s50, v46, s49\n" \
  "v_mfma_i32_16x16x64_i8 v[228:231], v[108:111], a[252:255], v[228:231]\n" \
  "s_waitcnt lgkmcnt(8)\n" \
  "v_mfma_i32_16x16x64_i8 v[136:139], v[112:115], a[224:227], v[136:139]\n" \
  "ds_read_b64_tr_b8 v[104:105], v85 offset:49152\n" \
  "v_mfma_i32_16x16x64_i8 v[136:139], v[116:119], a[240:243], v[136:139]\n" \
  "ds_read_b64_tr_b8 v[106:107], v85 offset:50176\n" \
  "v_mfma_i32_16x16x64_i8 v[168:171], v[112:115], a[228:231], v[168:171]\n" \
  "v_mfma_i32_16x16x64_i8 v[168:171], v[116:119], a[244:247], v[168:171]\n" \
  "v_mfma_i32_16x16x64_i8 v[200:203], v[112:115], a[232:235], v[200:203]\n" \
  "ds_read_b64_tr_b8 v[108:109], v85 offset:57344\n" \
  "v_mfma_i32_16x16x64_i8 v[200:203], v[116:119], a[248:251], v[200:203]\n" \
  "ds_read_b64_tr_b8 v[110:111], v85 offset:58368\n" \
  "buffer_load_dwordx4 a[32:35], v44, s[28:31], s89 offen offset:0 sc0 sc1\n" \
  "v_mfma_i32_16x16x64_i8 v[232:235], v[112:115], a[236:239], v[232:235]\n" \
  "s_lshl_b32 s83, s50, 21\n" \
  "s_lshl_b32 s91, s50, 16\n" \
  "s_add_u32 s49, s49, 1\n" \
  "v_mfma_i32_16x16x64_i8 v[232:235], v[116:119], a[252:255], v[232:235]\n" \
  "buffer_load_dwordx4 a[36:39], v44, s[28:31], s89 offen offset:1024 sc0 sc1\n" \
  "s_waitcnt lgkmcnt(8)\n" \
  "v_mfma_i32_16x16x64_i8 v[140:143], v[120:123], a[224:227], v[140:143]\n" \
  "ds_read_b64_tr_b8 v[112:113], v86 offset:49152\n" \
  "v_mfma_i32_16x16x64_i8 v[140:143], v[124:127], a[240:243], v[140:143]\n" \
  "ds_read_b64_tr_b8 v[114:115], v86 offset:50176\n" \
  "buffer_load_dwordx4 a[40:43], v44, s[28:31], s89 offen offset:2048 sc0 sc1\n" \
  "v_mfma_i32_16x16x64_i8 v[172:175], v[120:123], a[228:231], v[172:175]\n" \
  "v_readlane_b32 s50, v46, s49\n" \
  "v_mfma_i32_16x16x64_i8 v[172:175], v[124:127], a[244:247], v[172:175]\n" \
  "s_cmp_lg_u32 s46, 2\n" \
  "s_cbranch_scc1 Lskip17A_%=\n" \
  "s_waitcnt vmcnt(3)\n" \
  "Lskip17A_%=:\n" \
  "s_waitcnt vmcnt(63)\n" \
  "s_barrier\n" \
  "s_cmp_lg_u32 s46, 2\n" \
  "s_cbranch_scc1 Lskip19A_%=\n" \
  "s_cmp_lg_u32 s11, 0\n" \
  "s_cbranch_scc1 Lskip18A_%=\n" \
  "s_mov_b64 exec, 1\n" \
  "global_store_dword v53, v76, s[36:37] sc0 sc1\n" \
  "s_mov_b64 exec, -1\n" \
  "Lskip18A_%=:\n" \
  "Lskip19A_%=:\n" \
  "v_mfma_i32_16x16x64_i8 v[204:207], v[120:123], a[232:235], v[204:207]\n" \
  "ds_read_b64_tr_b8 v[116:117], v86 offset:57344\n" \
  "v_mfma_i32_16x16x64_i8 v[204:207], v[124:127], a[248:251], v[204:207]\n" \
  "ds_read_b64_tr_b8 v[118:119], v86 offset:58368\n" \
  "buffer_load_dwordx4 a[44:47], v44, s[28:31], s89 offen offset:3072 sc0 sc1\n" \
  "v_mfma_i32_16x16x64_i8 v[236:239], v[120:123], a[236:239], v[236:239]\n" \
  "v_mfma_i32_16x16x64_i8 v[236:239], v[124:127], a[252:255], v[236:239]\n" \
  "buffer_load_dwordx4 a[48:51], v91, s[28:31], s89 offen offset:0 sc0 sc1\n" \
  "s_waitcnt lgkmcnt(8)\n" \
  "v_mfma_i32_16x16x64_i8 v[144:147], v[96:99], a[224:227], v[144:147]\n" \
  "ds_read_b64_tr_b8 v[120:121], v87 offset:49152\n" \
  "v_mfma_i32_16x16x64_i8 v[144:147], v[100:103], a[240:243], v[144:147]\n" \
  "ds_read_b64_tr_b8 v[122:123], v87 offset:50176\n" \
  "s_add_u32 m0, s47, 49152\n" \
  "buffer_load_dwordx4 a[52:55], v91, s[28:31], s89 offen offset:1024 sc0 sc1\n" \
  "v_mfma_i32_16x16x64_i8 v[176:179], v[96:99], a[228:231], v[176:179]\n" \
  "buffer_load_dwordx4 v40, s[16:19], s83 offen nt lds\n" \
  "s_lshl_b32 s84, s50, 21\n" \
  "s_lshl_b32 s92, s50, 16\n" \
  "s_add_u32 s49, s49, 1\n" \
  "v_mfma_i32_16x16x64_i8 v[176:179], v[100:103], a[244:247], v[176:179]\n" \
  "v_mfma_i32_16x16x64_i8 v[208:211], v[96:99], a[232:235], v[208:211]\n" \
  "ds_read_b64_tr_b8 v[124:125], v87 offset:57344\n" \
  "v_mfma_i32_16x16x64_i8 v[208:211], v[100:103], a[248:251], v[208:211]\n" \
  "ds_read_b64_tr_b8 v[126:127], v87 offset:58368\n" \
  "s_add_u32 m0, s47, 50176\n" \
  "buffer_load_dwordx4 a[56:59], v91, s[28:31], s89 offen offset:2048 sc0 sc1\n" \
  "v_mfma_i32_16x16x64_i8 v[240:243], v[96:99], a[236:239], v[240:243]\n" \
  "buffer_load_dwordx4 v41, s[16:19], s83 offen nt lds\n" \
  "v_readlane_b32 s50, v46, s49\n" \
  "v_mfma_i32_16x16x64_i8 v[240:243], v[100:103], a[252:255], v[240:243]\n" \
  "buffer_load_dwordx4 a[60:63], v91, s[28:31], s89 offen offset:3072 sc0 sc1\n" \
  "s_waitcnt lgkmcnt(8)\n" \
  "v_mfma_i32_16x16x64_i8 v[148:151], v[104:107], a[224:227], v[148:151]\n" \
  "ds_read_b64_tr_b8 v[96:97], v32 offset:0\n" \
  "v_mfma_i32_16x16x64_i8 v[148:151], v[108:111], a[240:243], v[148:151]\n" \
  "ds_read_b64_tr_b8 v[98:99], v32 offset:1024\n" \
  "s_add_u32 m0, s47, 51200\n" \
  "v_mfma_i32_16x16x64_i8 v[180:183], v[104:107], a[228:231], v[180:183]\n" \
  "buffer_load_dwordx4 v42, s[16:19], s83 offen nt lds\n" \
  "v_mfma_i32_16x16x64_i8 v[180:183], v[108:111], a[244:247], v[180:183]\n" \
  "buffer_load_dwordx4 a[64:67], v44, s[28:31], s90 offen offset:0 sc0 sc1\n" \
  "v_mfma_i32_16x16x64_i8 v[212:215], v[104:107], a[232:235], v[212:215]\n" \
  "ds_read_b64_tr_b8 v[100:101], v32 offset:8192\n" \
  "v_mfma_i32_16x16x64_i8 v[212:215], v[108:111], a[248:251], v[212:215]\n" \
  "ds_read_b64_tr_b8 v[102:103], v32 offset:9216\n" \
  "s_add_u32 m0, s47, 52224\n" \
  "buffer_load_dwordx4 a[68:71], v44, s[28:31], s90 offen offset:1024 sc0 sc1\n" \
  "v_mfma_i32_16x16x64_i8 v[244:247], v[104:107], a[236:239], v[244:247]\n" \
  "buffer_load_dwordx4 v43, s[16:19], s83 offen nt lds\n" \
  "s_lshl_b32 s85, s50, 21\n" \
  "s_lshl_b32 s93, s50, 16\n" \
  "s_add_u32 s49, s49, 1\n" \
  "v_mfma_i32_16x16x64_i8 v[244:247], v[108:111], a[252:255], v[244:247]\n" \
  "buffer_load_dwordx4 a[72:75], v44, s[28:31], s90 offen offset:2048 sc0 sc1\n" \
  "s_waitcnt lgkmcnt(8)\n" \
  "v_mfma_i32_16x16x64_i8 v[152:155], v[112:115], a[224:227], v[152:155]\n" \
  "ds_read_b64_tr_b8 v[104:105], v33 offset:0\n" \
  "v_mfma_i32_16x16x64_i8 v[152:155], v[116:119], a[240:243], v[152:155]\n" \
  "ds_read_b64_tr_b8 v[106:107], v33 offset:1024\n" \
  "s_add_u32 m0, s47, 65536\n" \
  "v_mfma_i32_16x16x64_i8 v[184:187], v[112:115], a[228:231], v[184:187]\n" \
  "buffer_load_dwordx4 v40, s[16:19], s84 offen nt lds\n" \
  "v_readlane_b32 s50, v46, s49\n" \
  "v_mfma_i32_16x16x64_i8 v[184:187], v[116:119], a[244:247], v[184:187]\n" \
  "buffer_load_dwordx4 a[76:79], v44, s[28:31], s90 offen offset:3072 sc0 sc1\n" \
  "s_cmp_lg_u32 s11, 0\n" \
  "s_cbranch_scc1 Lskip22A_%=\n" \
  "s_waitcnt vmcnt(17)\n" \
  "v_sub_u32 v72, v72, v76\n" \
  "v_cmp_gt_i32 vcc, 0, v72\n" \
  "s_nop 1\n" \
  "s_andn2_b64 vcc, vcc, s[14:15]\n" \
  "s_cbranch_scc1 Lslow20A_%=\n" \
  "Lback21A_%=:\n" \
  "v_add_u32 v50, 1024, v50\n" \
  "Lskip22A_%=:\n" \
  "v_mfma_i32_16x16x64_i8 v[216:219], v[112:115], a[232:235], v[216:219]\n" \
  "ds_read_b64_tr_b8 v[108:109], v33 offset:8192\n" \
  "v_mfma_i32_16x16x64_i8 v[216:219], v[116:119], a[248:251], v[216:219]\n" \
  "ds_read_b64_tr_b8 v[110:111], v33 offset:9216\n" \
  "s_add_u32 m0, s47, 66560\n" \
  "buffer_load_dwordx4 a[80:83], v91, s[28:31], s90 offen offset:0 sc0 sc1\n" \
  "v_mfma_i32_16x16x64_i8 v[248:251], v[112:115], a[236:239], v[248:251]\n" \
  "buffer_load_dwordx4 v41, s[16:19], s84 offen nt lds\n" \
  "v_mfma_i32_16x16x64_i8 v[248:251], v[116:119], a[252:255], v[248:251]\n" \
  "buffer_load_dwordx4 a[84:87], v91, s[28:31], s90 offen offset:1024 sc0 sc1\n" \
  "s_waitcnt lgkmcnt(8)\n" \
  "v_mfma_i32_16x16x64_i8 v[156:159], v[120:123], a[224:227], v[156:159]\n" \
  "ds_read_b64_tr_b8 v[112:113], v34 offset:0\n" \
  "v_mfma_i32_16x16x64_i8 v[156:159], v[124:127], a[240:243], v[156:159]\n" \
  "ds_read_b64_tr_b8 v[114:115], v34 offset:1024\n" \
  "s_add_u32 m0, s47, 67584\n" \
  "v_mfma_i32_16x16x64_i8 v[188:191], v[120:123], a[228:231], v[188:191]\n" \
  "buffer_load_dwordx4 v42, s[16:19], s84 offen nt lds\n" \
  "s_lshl_b32 s86, s50, 21\n" \
  "s_lshl_b32 s94, s50, 16\n" \
  "s_add_u32 s49, s49, 1\n" \
  "v_mfma_i32_16x16x64_i8 v[188:191], v[124:127], a[244:247], v[188:191]\n" \
  "buffer_load_dwordx4 a[88:91], v91, s[28:31], s90 offen offset:2048 sc0 sc1\n" \
  "v_mfma_i32_16x16x64_i8 v[220:223], v[120:123], a[232:235], v[220:223]\n" \
  "ds_read_b64_tr_b8 v[116:117], v34 offset:8192\n" \
  "v_mfma_i32_16x16x64_i8 v[220:223], v[124:127], a[248:251], v[220:223]\n" \
  "ds_read_b64_tr_b8 v[118:119], v34 offset:9216\n" \
  "s_add_u32 m0, s47, 68608\n" \
  "buffer_load_dwordx4 a[92:95], v91, s[28:31], s90 offen offset:3072 sc0 sc1\n" \
  "v_mfma_i32_16x16x64_i8 v[252:255], v[120:123], a[236:239], v[252:255]\n" \
  "buffer_load_dwordx4 v43, s[16:19], s84 offen nt lds\n" \
  "v_mfma_i32_16x16x64_i8 v[252:255], v[124:127], a[252:255], v[252:255]\n" \
  "s_barrier\n" \
  "buffer_load_dword v92, v52, s[28:31], 0 offen sc0 sc1\n" \
  "v_add_u32 v52, 0x10000, v52\n" \
  "s_add_u32 s53, s53, 1\n" \
  "s_add_u32 s54, s54, 1\n" \
  "s_mov_b32 s51, 1\n" \
  "L_loopA_%=:\n" \
  "; ---- step 0\n" \
  "s_waitcnt vmcnt(25)\n" \
  "s_waitcnt lgkmcnt(8)\n" \
  "v_mfma_i32_16x16x64_i8 v[128:131], v[96:99], a[0:3], v[128:131]\n" \
  "ds_read_b64_tr_b8 v[120:121], v35 offset:0\n" \
  "s_add_u32 m0, s47, 81920\n" \
  "v_mfma_i32_16x16x64_i8 v[128:131], v[100:103], a[16:19], v[128:131]\n" \
  "ds_read_b64_tr_b8 v[122:123], v35 offset:1024\n" \
  "buffer_load_dwordx4 v40, s[16:19], s85 offen lds\n" \
  "s_add_u32 m0, s47, 82944\n" \
  "v_mfma_i32_16x16x64_i8 v[160:163], v[96:99], a[4:7], v[160:163]\n" \
  "buffer_load_dwordx4 v41, s[16:19], s85 offen lds\n" \
  "s_add_u32 m0, s47, 83968\n" \
  "v_mfma_i32_16x16x64_i8 v[160:163], v[100:103], a[20:23], v[160:163]\n" \
  "buffer_load_dwordx4 v42, s[16:19], s85 offen lds\n" \
  "s_add_u32 m0, s47, 84992\n" \
  "v_mfma_i32_16x16x64_i8 v[192:195], v[96:99], a[8:11], v[192:195]\n" \
  "ds_read_b64_tr_b8 v[124:125], v35 offset:8192\n" \
  "buffer_load_dwordx4 v43, s[16:19], s85 offen lds\n" \
  "s_add_u32 m0, s47, 98304\n" \
  "v_mfma_i32_16x16x64_i8 v[192:195], v[100:103], a[24:27], v[192:195]\n" \
  "ds_read_b64_tr_b8 v[126:127], v35 offset:9216\n" \
  "buffer_load_dwordx4 v40, s[16:19], s86 offen lds\n" \
  "s_add_u32 m0, s47, 99328\n" \
  "v_mfma_i32_16x16x64_i8 v[224:227], v[96:99], a[12:15], v[224:227]\n" \
  "buffer_load_dwordx4 v41, s[16:19], s86 offen lds\n" \
  "s_add_u32 m0, s47, 100352\n" \
  "v_mfma_i32_16x16x64_i8 v[224:227], v[100:103], a[28:31], v[224:227]\n" \
  "buffer_load_dwordx4 v42, s[16:19], s86 offen lds\n" \
  "s_add_u32 m0, s47, 101376\n" \
  "s_waitcnt lgkmcnt(8)\n" \
  "v_mfma_i32_16x16x64_i8 v[132:135], v[104:107], a[0:3], v[132:135]\n" \
  "ds_read_b64_tr_b8 v[96:97], v36 offset:0\n" \
  "buffer_load_dwordx4 v43, s[16:19], s86 offen lds\n" \
  "v_mfma_i32_16x16x64_i8 v[132:135], v[108:111], a[16:19], v[132:135]\n" \
  "ds_read_b64_tr_b8 v[98:99], v36 offset:1024\n" \
  "v_mfma_i32_16x16x64_i8 v[164:167], v[104:107], a[4:7], v[164:167]\n" \
  "v_mfma_i32_16x16x64_i8 v[164:167], v[108:111], a[20:23], v[164:167]\n" \
  "v_mfma_i32_16x16x64_i8 v[196:199], v[104:107], a[8:11], v[196:199]\n" \
  "ds_read_b64_tr_b8 v[100:101], v36 offset:8192\n" \
  "v_mfma_i32_16x16x64_i8 v[196:199], v[108:111], a[24:27], v[196:199]\n" \
  "ds_read_b64_tr_b8 v[102:103], v36 offset:9216\n" \
  "v_mfma_i32_16x16x64_i8 v[228:231], v[104:107], a[12:15], v[228:231]\n" \
  "v_mfma_i32_16x16x64_i8 v[228:231], v[108:111], a[28:31], v[228:231]\n" \
  "s_waitcnt lgkmcnt(8)\n" \
  "v_mfma_i32_16x16x64_i8 v[136:139], v[112:115], a[0:3], v[136:139]\n" \
  "ds_read_b64_tr_b8 v[104:105], v37 offset:0\n" \
  "v_mfma_i32_16x16x64_i8 v[136:139], v[116:119], a[16:19], v[136:139]\n" \
  "ds_read_b64_tr_b8 v[106:107], v37 offset:1024\n" \
  "v_mfma_i32_16x16x64_i8 v[168:171], v[112:115], a[4:7], v[168:171]\n" \
  "v_mfma_i32_16x16x64_i8 v[168:171], v[116:119], a[20:23], v[168:171]\n" \
  "v_mfma_i32_16x16x64_i8 v[200:203], v[112:115], a[8:11], v[200:203]\n" \
  "ds_read_b64_tr_b8 v[108:109], v37 offset:8192\n" \
  "v_mfma_i32_16x16x64_i8 v[200:203], v[116:119], a[24:27], v[200:203]\n" \
  "ds_read_b64_tr_b8 v[110:111], v37 offset:9216\n" \
  "v_mfma_i32_16x16x64_i8 v[232:235], v[112:115], a[12:15], v[232:235]\n" \
  "v_mfma_i32_16x16x64_i8 v[232:235], v[116:119], a[28:31], v[232:235]\n" \
  "s_waitcnt lgkmcnt(8)\n" \
  "v_mfma_i32_16x16x64_i8 v[140:143], v[120:123], a[0:3], v[140:143]\n" \
  "ds_read_b64_tr_b8 v[112:113], v38 offset:0\n" \
  "v_mfma_i32_16x16x64_i8 v[140:143], v[124:127], a[16:19], v[140:143]\n" \
  "ds_read_b64_tr_b8 v[114:115], v38 offset:1024\n" \
  "v_mfma_i32_16x16x64_i8 v[172:175], v[120:123], a[4:7], v[172:175]\n" \
  "v_mfma_i32_16x16x64_i8 v[172:175], v[124:127], a[20:23], v[172:175]\n" \
  "s_waitcnt vmcnt(53)\n" \
  "s_barrier\n" \
  "v_mfma_i32_16x16x64_i8 v[204:207], v[120:123], a[8:11], v[204:207]\n" \
  "ds_read_b64_tr_b8 v[116:117], v38 offset:8192\n" \
  "v_mfma_i32_16x16x64_i8 v[204:207], v[124:127], a[24:27], v[204:207]\n" \
  "ds_read_b64_tr_b8 v[118:119], v38 offset:9216\n" \
  "v_mfma_i32_16x16x64_i8 v[236:239], v[120:123], a[12:15], v[236:239]\n" \
  "v_mfma_i32_16x16x64_i8 v[236:239], v[124:127], a[28:31], v[236:239]\n" \
  "s_waitcnt lgkmcnt(8)\n" \
  "v_mfma_i32_16x16x64_i8 v[144:147], v[96:99], a[0:3], v[144:147]\n" \
  "ds_read_b64_tr_b8 v[120:121], v39 offset:0\n" \
  "v_mfma_i32_16x16x64_i8 v[144:147], v[100:103], a[16:19], v[144:147]\n" \
  "ds_read_b64_tr_b8 v[122:123], v39 offset:1024\n" \
  "v_mfma_i32_16x16x64_i8 v[176:179], v[96:99], a[4:7], v[176:179]\n" \
  "v_mfma_i32_16x16x64_i8 v[176:179], v[100:103], a[20:23], v[176:179]\n" \
  "v_mfma_i32_16x16x64_i8 v[208:211], v[96:99], a[8:11], v[208:211]\n" \
  "ds_read_b64_tr_b8 v[124:125], v39 offset:8192\n" \
  "v_mfma_i32_16x16x64_i8 v[208:211], v[100:103], a[24:27], v[208:211]\n" \
  "ds_read_b64_tr_b8 v[126:127], v39 offset:9216\n" \
  "v_mfma_i32_16x16x64_i8 v[240:243], v[96:99], a[12:15], v[240:243]\n" \
  "v_mfma_i32_16x16x64_i8 v[240:243], v[100:103], a[28:31], v[240:243]\n" \
  "s_waitcnt lgkmcnt(8)\n" \
  "v_mfma_i32_16x16x64_i8 v[148:151], v[104:107], a[0:3], v[148:151]\n" \
  "ds_read_b64_tr_b8 v[96:97], v32 offset:16384\n" \
  "v_mfma_i32_16x16x64_i8 v[148:151], v[108:111], a[16:19], v[148:151]\n" \
  "ds_read_b64_tr_b8 v[98:99], v32 offset:17408\n" \
  "v_mfma_i32_16x16x64_i8 v[180:183], v[104:107], a[4:7], v[180:183]\n" \
  "v_mfma_i32_16x16x64_i8 v[180:183], v[108:111], a[20:23], v[180:183]\n" \
  "v_mfma_i32_16x16x64_i8 v[212:215], v[104:107], a[8:11], v[212:215]\n" \
  "ds_read_b64_tr_b8 v[100:101], v32 offset:24576\n" \
  "v_mfma_i32_16x16x64_i8 v[212:215], v[108:111], a[24:27], v[212:215]\n" \
  "ds_read_b64_tr_b8 v[102:103], v32 offset:25600\n" \
  "v_mfma_i32_16x16x64_i8 v[244:247], v[104:107], a[12:15], v[244:247]\n" \
  "v_mfma_i32_16x16x64_i8 v[244:247], v[108:111], a[28:31], v[244:247]\n" \
  "s_waitcnt lgkmcnt(8)\n" \
  "v_mfma_i32_16x16x64_i8 v[152:155], v[112:115], a[0:3], v[152:155]\n" \
  "ds_read_b64_tr_b8 v[104:105], v33 offset:16384\n" \
  "v_mfma_i32_16x16x64_i8 v[152:155], v[116:119], a[16:19], v[152:155]\n" \
  "ds_read_b64_tr_b8 v[106:107], v33 offset:17408\n" \
  "v_mfma_i32_16x16x64_i8 v[184:187], v[112:115], a[4:7], v[184:187]\n" \
  "v_mfma_i32_16x16x64_i8 v[184:187], v[116:119], a[20:23], v[184:187]\n" \
  "v_mfma_i32_16x16x64_i8 v[216:219], v[112:115], a[8:11], v[216:219]\n" \
  "ds_read_b64_tr_b8 v[108:109], v33 offset:24576\n" \
  "v_mfma_i32_16x16x64_i8 v[216:219], v[116:119], a[24:27], v[216:219]\n" \
  "ds_read_b64_tr_b8 v[110:111], v33 offset:25600\n" \
  "v_mfma_i32_16x16x64_i8 v[248:251], v[112:115], a[12:15], v[248:251]\n" \
  "v_mfma_i32_16x16x64_i8 v[248:251], v[116:119], a[28:31], v[248:251]\n" \
  "s_waitcnt lgkmcnt(8)\n" \
  "v_mfma_i32_16x16x64_i8 v[156:159], v[120:123], a[0:3], v[156:159]\n" \
  "ds_read_b64_tr_b8 v[112:113], v34 offset:16384\n" \
  "v_mfma_i32_16x16x64_i8 v[156:159], v[124:127], a[16:19], v[156:159]\n" \
  "ds_read_b64_tr_b8 v[114:115], v34 offset:17408\n" \
  "v_mfma_i32_16x16x64_i8 v[188:191], v[120:123], a[4:7], v[188:191]\n" \
  "v_mfma_i32_16x16x64_i8 v[188:191], v[124:127], a[20:23], v[188:191]\n" \
  "v_mfma_i32_16x16x64_i8 v[220:223], v[120:123], a[8:11], v[220:223]\n" \
  "ds_read_b64_tr_b8 v[116:117], v34 offset:24576\n" \
  "v_mfma_i32_16x16x64_i8 v[220:223], v[124:127], a[24:27], v[220:223]\n" \
  "ds_read_b64_tr_b8 v[118:119], v34 offset:25600\n" \
  "v_mfma_i32_16x16x64_i8 v[252:255], v[120:123], a[12:15], v[252:255]\n" \
  "v_mfma_i32_16x16x64_i8 v[252:255], v[124:127], a[28:31], v[252:255]\n" \
  "; ---- step 1\n" \
  "s_waitcnt vmcnt(23)\n" \
  "s_waitcnt lgkmcnt(8)\n" \
  "v_mfma_i32_16x16x64_i8 v[128:131], v[96:99], a[32:35], v[128:131]\n" \
  "ds_read_b64_tr_b8 v[120:121], v35 offset:16384\n" \
  "v_mfma_i32_16x16x64_i8 v[128:131], v[100:103], a[48:51], v[128:131]\n" \
  "ds_read_b64_tr_b8 v[122:123], v35 offset:17408\n" \
  "v_mfma_i32_16x16x64_i8 v[160:163], v[96:99], a[36:39], v[160:163]\n" \
  "v_mfma_i32_16x16x64_i8 v[160:163], v[100:103], a[52:55], v[160:163]\n" \
  "v_mfma_i32_16x16x64_i8 v[192:195], v[96:99], a[40:43], v[192:195]\n" \
  "ds_read_b64_tr_b8 v[124:125], v35 offset:24576\n" \
  "v_mfma_i32_16x16x64_i8 v[192:195], v[100:103], a[56:59], v[192:195]\n" \
  "ds_read_b64_tr_b8 v[126:127], v35 offset:25600\n" \
  "v_mfma_i32_16x16x64_i8 v[224:227], v[96:99], a[44:47], v[224:227]\n" \
  "v_mfma_i32_16x16x64_i8 v[224:227], v[100:103], a[60:63], v[224:227]\n" \
  "s_waitcnt lgkmcnt(8)\n" \
  "v_mfma_i32_16x16x64_i8 v[132:135], v[104:107], a[32:35], v[132:135]\n" \
  "ds_read_b64_tr_b8 v[96:97], v36 offset:16384\n" \
  "v_mfma_i32_16x16x64_i8 v[132:135], v[108:111], a[48:51], v[132:135]\n" \
  "ds_read_b64_tr_b8 v[98:99], v36 offset:17408\n" \
  "v_mfma_i32_16x16x64_i8 v[164:167], v[104:107], a[36:39], v[164:167]\n" \
  "v_mfma_i32_16x16x64_i8 v[164:167], v[108:111], a[52:55], v[164:167]\n" \
  "v_mfma_i32_16x16x64_i8 v[196:199], v[104:107], a[40:43], v[196:199]\n" \
  "ds_read_b64_tr_b8 v[100:101], v36 offset:24576\n" \
  "v_mfma_i32_16x16x64_i8 v[196:199], v[108:111], a[56:59], v[196:199]\n" \
  "ds_read_b64_tr_b8 v[102:103], v36 offset:25600\n" \
  "v_mfma_i32_16x16x64_i8 v[228:231], v[104:107], a[44:47], v[228:231]\n" \
  "v_mfma_i32_16x16x64_i8 v[228:231], v[108:111], a[60:63], v[228:231]\n" \
  "s_waitcnt lgkmcnt(8)\n" \
  "v_mfma_i32_16x16x64_i8 v[136:139], v[112:115], a[32:35], v[136:139]\n" \
  "ds_read_b64_tr_b8 v[104:105], v37 offset:16384\n" \
  "v_mfma_i32_16x16x64_i8 v[136:139], v[116:119], a[48:51], v[136:139]\n" \
  "ds_read_b64_tr_b8 v[106:107], v37 offset:17408\n" \
  "v_mfma_i32_16x16x64_i8 v[168:171], v[112:115], a[36:39], v[168:171]\n" \
  "v_mfma_i32_16x16x64_i8 v[168:171], v[116:119], a[52:55], v[168:171]\n" \
  "v_mfma_i32_16x16x64_i8 v[200:203], v[112:115], a[40:43], v[200:203]\n" \
  "ds_read_b64_tr_b8 v[108:109], v37 offset:24576\n" \
  "v_mfma_i32_16x16x64_i8 v[200:203], v[116:119], a[56:59], v[200:203]\n" \
  "ds_read_b64_tr_b8 v[110:111], v37 offset:25600\n" \
  "buffer_load_dwordx4 a[96:99], v44, s[28:31], s91 offen offset:0 sc0 sc1\n" \
  "v_mfma_i32_16x16x64_i8 v[232:235], v[112:115], a[44:47], v[232:235]\n" \
  "v_mfma_i32_16x16x64_i8 v[232:235], v[116:119], a[60:63], v[232:235]\n" \
  "buffer_load_dwordx4 a[100:103], v44, s[28:31], s91 offen offset:1024 sc0 sc1\n" \
  "s_waitcnt lgkmcnt(8)\n" \
  "v_mfma_i32_16x16x64_i8 v[140:143], v[120:123], a[32:35], v[140:143]\n" \
  "ds_read_b64_tr_b8 v[112:113], v38 offset:16384\n" \
  "v_mfma_i32_16x16x64_i8 v[140:143], v[124:127], a[48:51], v[140:143]\n" \
  "ds_read_b64_tr_b8 v[114:115], v38 offset:17408\n" \
  "v_mfma_i32_16x16x64_i8 v[172:175], v[120:123], a[36:39], v[172:175]\n" \
  "v_mfma_i32_16x16x64_i8 v[172:175], v[124:127], a[52:55], v[172:175]\n" \
  "buffer_load_dwordx4 a[104:107], v44, s[28:31], s91 offen offset:2048 sc0 sc1\n" \
  "s_waitcnt vmcnt(52)\n" \
  "s_barrier\n" \
  "v_mfma_i32_16x16x64_i8 v[204:207], v[120:123], a[40:43], v[204:207]\n" \
  "ds_read_b64_tr_b8 v[116:117], v38 offset:24576\n" \
  "v_mfma_i32_16x16x64_i8 v[204:207], v[124:127], a[56:59], v[204:207]\n" \
  "ds_read_b64_tr_b8 v[118:119], v38 offset:25600\n" \
  "v_mfma_i32_16x16x64_i8 v[236:239], v[120:123], a[44:47], v[236:239]\n" \
  "v_mfma_i32_16x16x64_i8 v[236:239], v[124:127], a[60:63], v[236:239]\n" \
  "buffer_load_dwordx4 a[108:111], v44, s[28:31], s91 offen offset:3072 sc0 sc1\n" \
  "s_waitcnt lgkmcnt(8)\n" \
  "v_mfma_i32_16x16x64_i8 v[144:147], v[96:99], a[32:35], v[144:147]\n" \
  "ds_read_b64_tr_b8 v[120:121], v39 offset:16384\n" \
  "v_mfma_i32_16x16x64_i8 v[144:147], v[100:103], a[48:51], v[144:147]\n" \
  "ds_read_b64_tr_b8 v[122:123], v39 offset:17408\n" \
  "v_mfma_i32_16x16x64_i8 v[176:179], v[96:99], a[36:39], v[176:179]\n" \
  "v_mfma_i32_16x16x64_i8 v[176:179], v[100:103], a[52:55], v[176:179]\n" \
  "buffer_load_dwordx4 a[112:115], v91, s[28:31], s91 offen offset:0 sc0 sc1\n" \
  "v_mfma_i32_16x16x64_i8 v[208:211], v[96:99], a[40:43], v[208:211]\n" \
  "ds_read_b64_tr_b8 v[124:125], v39 offset:24576\n" \
  "v_mfma_i32_16x16x64_i8 v[208:211], v[100:103], a[56:59], v[208:211]\n" \
  "ds_read_b64_tr_b8 v[126:127], v39 offset:25600\n" \
  "v_mfma_i32_16x16x64_i8 v[240:243], v[96:99], a[44:47], v[240:243]\n" \
  "v_mfma_i32_16x16x64_i8 v[240:243], v[100:103], a[60:63], v[240:243]\n" \
  "buffer_load_dwordx4 a[116:119], v91, s[28:31], s91 offen offset:1024 sc0 sc1\n" \
  "s_waitcnt lgkmcnt(8)\n" \
  "v_mfma_i32_16x16x64_i8 v[148:151], v[104:107], a[32:35], v[148:151]\n" \
  "ds_read_b64_tr_b8 v[96:97], v32 offset:32768\n" \
  "v_mfma_i32_16x16x64_i8 v[148:151], v[108:111], a[48:51], v[148:151]\n" \
  "ds_read_b64_tr_b8 v[98:99], v32 offset:33792\n" \
  "v_mfma_i32_16x16x64_i8 v[180:183], v[104:107], a[36:39], v[180:183]\n" \
  "v_mfma_i32_16x16x64_i8 v[180:183], v[108:111], a[52:55], v[180:183]\n" \
  "buffer_load_dwordx4 a[120:123], v91, s[28:31], s91 offen offset:2048 sc0 sc1\n" \
  "v_mfma_i32_16x16x64_i8 v[212:215], v[104:107], a[40:43], v[212:215]\n" \
  "ds_read_b64_tr_b8 v[100:101], v32 offset:40960\n" \
  "v_mfma_i32_16x16x64_i8 v[212:215], v[108:111], a[56:59], v[212:215]\n" \
  "ds_read_b64_tr_b8 v[102:103], v32 offset:41984\n" \
  "buffer_load_dwordx4 a[124:127], v91, s[28:31], s91 offen offset:3072 sc0 sc1\n" \
  "v_mfma_i32_16x16x64_i8 v[244:247], v[104:107], a[44:47], v[244:247]\n" \
  "v_mfma_i32_16x16x64_i8 v[244:247], v[108:111], a[60:63], v[244:247]\n" \
  "s_waitcnt lgkmcnt(8)\n" \
  "v_mfma_i32_16x16x64_i8 v[152:155], v[112:115], a[32:35], v[152:155]\n" \
  "ds_read_b64_tr_b8 v[104:105], v33 offset:32768\n" \
  "v_mfma_i32_16x16x64_i8 v[152:155], v[116:119], a[48:51], v[152:155]\n" \
  "ds_read_b64_tr_b8 v[106:107], v33 offset:33792\n" \
  "buffer_load_dwordx4 a[128:131], v44, s[28:31], s92 offen offset:0 sc0 sc1\n" \
  "v_mfma_i32_16x16x64_i8 v[184:187], v[112:115], a[36:39], v[184:187]\n" \
  "v_mfma_i32_16x16x64_i8 v[184:187], v[116:119], a[52:55], v[184:187]\n" \
  "v_mfma_i32_16x16x64_i8 v[216:219], v[112:115], a[40:43], v[216:219]\n" \
  "ds_read_b64_tr_b8 v[108:109], v33 offset:40960\n" \
  "v_mfma_i32_16x16x64_i8 v[216:219], v[116:119], a[56:59], v[216:219]\n" \
  "ds_read_b64_tr_b8 v[110:111], v33 offset:41984\n" \
  "buffer_load_dwordx4 a[132:135], v44, s[28:31], s92 offen offset:1024 sc0 sc1\n" \
  "v_mfma_i32_16x16x64_i8 v[248:251], v[112:115], a[44:47], v[248:251]\n" \
  "v_mfma_i32_16x16x64_i8 v[248:251], v[116:119], a[60:63], v[248:251]\n" \
  "s_waitcnt lgkmcnt(8)\n" \
  "v_mfma_i32_16x16x64_i8 v[156:159], v[120:123], a[32:35], v[156:159]\n" \
  "ds_read_b64_tr_b8 v[112:113], v34 offset:32768\n" \
  "v_mfma_i32_16x16x64_i8 v[156:159], v[124:127], a[48:51], v[156:159]\n" \
  "ds_read_b64_tr_b8 v[114:115], v34 offset:33792\n" \
  "buffer_load_dwordx4 a[136:139], v44, s[28:31], s92 offen offset:2048 sc0 sc1\n" \
  "v_mfma_i32_16x16x64_i8 v[188:191], v[120:123], a[36:39], v[188:191]\n" \
  "v_mfma_i32_16x16x64_i8 v[188:191], v[124:127], a[52:55], v[188:191]\n" \
  "v_mfma_i32_16x16x64_i8 v[220:223], v[120:123], a[40:43], v[220:223]\n" \
  "ds_read_b64_tr_b8 v[116:117], v34 offset:40960\n" \
  "v_mfma_i32_16x16x64_i8 v[220:223], v[124:127], a[56:59], v[220:223]\n" \
  "ds_read_b64_tr_b8 v[118:119], v34 offset:41984\n" \
  "buffer_load_dwordx4 a[140:143], v44, s[28:31], s92 offen offset:3072 sc0 sc1\n" \
  "v_mfma_i32_16x16x64_i8 v[252:255], v[120:123], a[44:47], v[252:255]\n" \
  "v_mfma_i32_16x16x64_i8 v[252:255], v[124:127], a[60:63], v[252:255]\n" \
  "; ---- step 2\n" \
  "s_cmp_lg_u32 s11, 0\n" \
  "s_cbranch_scc1 Lskip24A_%=\n" \
  "global_load_dword v72, v50, s[38:39] sc0 sc1\n" \
  "Lskip24A_%=:\n" \
  "s_cmp_lg_u32 s46, s54\n" \
  "s_cbranch_scc1 Lskip23A_%=\n" \
  "s_waitcnt vmcnt(63)\n" \
  "global_store_dwordx4 v49, v[64:67], s[34:35] sc0 sc1\n" \
  "s_nop 1\n" \
  "global_store_dwordx4 v79, v[68:71], s[34:35] sc0 sc1\n" \
  "s_nop 1\n" \
  "Lskip23A_%=:\n" \
  "s_waitcnt vmcnt(22)\n" \
  "s_waitcnt lgkmcnt(8)\n" \
  "v_mfma_i32_16x16x64_i8 v[128:131], v[96:99], a[64:67], v[128:131]\n" \
  "ds_read_b64_tr_b8 v[120:121], v35 offset:32768\n" \
  "v_mfma_i32_16x16x64_i8 v[128:131], v[100:103], a[80:83], v[128:131]\n" \
  "ds_read_b64_tr_b8 v[122:123], v35 offset:33792\n" \
  "v_mfma_i32_16x16x64_i8 v[160:163], v[96:99], a[68:71], v[160:163]\n" \
  "v_mfma_i32_16x16x64_i8 v[160:163], v[100:103], a[84:87], v[160:163]\n" \
  "buffer_load_dwordx4 a[144:147], v91, s[28:31], s92 offen offset:0 sc0 sc1\n" \
  "v_mfma_i32_16x16x64_i8 v[192:195], v[96:99], a[72:75], v[192:195]\n" \
  "ds_read_b64_tr_b8 v[124:125], v35 offset:40960\n" \
  "v_mfma_i32_16x16x64_i8 v[192:195], v[100:103], a[88:91], v[192:195]\n" \
  "ds_read_b64_tr_b8 v[126:127], v35 offset:41984\n" \
  "v_mfma_i32_16x16x64_i8 v[224:227], v[96:99], a[76:79], v[224:227]\n" \
  "v_mfma_i32_16x16x64_i8 v[224:227], v[100:103], a[92:95], v[224:227]\n" \
  "buffer_load_dwordx4 a[148:151], v91, s[28:31], s92 offen offset:1024 sc0 sc1\n" \
  "s_waitcnt lgkmcnt(8)\n" \
  "v_mfma_i32_16x16x64_i8 v[132:135], v[104:107], a[64:67], v[132:135]\n" \
  "ds_read_b64_tr_b8 v[96:97], v36 offset:32768\n" \
  "v_mfma_i32_16x16x64_i8 v[132:135], v[108:111], a[80:83], v[132:135]\n" \
  "ds_read_b64_tr_b8 v[98:99], v36 offset:33792\n" \
  "v_mfma_i32_16x16x64_i8 v[164:167], v[104:107], a[68:71], v[164:167]\n" \
  "v_mfma_i32_16x16x64_i8 v[164:167], v[108:111], a[84:87], v[164:167]\n" \
  "buffer_load_dwordx4 a[152:155], v91, s[28:31], s92 offen offset:2048 sc0 sc1\n" \
  "v_mfma_i32_16x16x64_i8 v[196:199], v[104:107], a[72:75], v[196:199]\n" \
  "ds_read_b64_tr_b8 v[100:101], v36 offset:40960\n" \
  "v_mfma_i32_16x16x64_i8 v[196:199], v[108:111], a[88:91], v[196:199]\n" \
  "ds_read_b64_tr_b8 v[102:103], v36 offset:41984\n" \
  "v_mfma_i32_16x16x64_i8 v[228:231], v[104:107], a[76:79], v[228:231]\n" \
  "v_mfma_i32_16x16x64_i8 v[228:231], v[108:111], a[92:95], v[228:231]\n" \
  "s_waitcnt lgkmcnt(8)\n" \
  "v_mfma_i32_16x16x64_i8 v[136:139], v[112:115], a[64:67], v[136:139]\n" \
  "ds_read_b64_tr_b8 v[104:105], v37 offset:32768\n" \
  "v_mfma_i32_16x16x64_i8 v[136:139], v[116:119], a[80:83], v[136:139]\n" \
  "ds_read_b64_tr_b8 v[106:107], v37 offset:33792\n" \
  "v_mfma_i32_16x16x64_i8 v[168:171], v[112:115], a[68:71], v[168:171]\n" \
  "v_mfma_i32_16x16x64_i8 v[168:171], v[116:119], a[84:87], v[168:171]\n" \
  "buffer_load_dwordx4 a[156:159], v91, s[28:31], s92 offen offset:3072 sc0 sc1\n" \
  "v_mfma_i32_16x16x64_i8 v[200:203], v[112:115], a[72:75], v[200:203]\n" \
  "ds_read_b64_tr_b8 v[108:109], v37 offset:40960\n" \
  "v_mfma_i32_16x16x64_i8 v[200:203], v[116:119], a[88:91], v[200:203]\n" \
  "ds_read_b64_tr_b8 v[110:111], v37 offset:41984\n" \
  "v_mfma_i32_16x16x64_i8 v[232:235], v[112:115], a[76:79], v[232:235]\n" \
  "v_mfma_i32_16x16x64_i8 v[232:235], v[116:119], a[92:95], v[232:235]\n" \
  "buffer_load_dwordx4 a[160:163], v44, s[28:31], s93 offen offset:0 sc0 sc1\n" \
  "s_waitcnt lgkmcnt(8)\n" \
  "v_mfma_i32_16x16x64_i8 v[140:143], v[120:123], a[64:67], v[140:143]\n" \
  "ds_read_b64_tr_b8 v[112:113], v38 offset:32768\n" \
  "v_mfma_i32_16x16x64_i8 v[140:143], v[124:127], a[80:83], v[140:143]\n" \
  "ds_read_b64_tr_b8 v[114:115], v38 offset:33792\n" \
  "v_mfma_i32_16x16x64_i8 v[172:175], v[120:123], a[68:71], v[172:175]\n" \
  "v_mfma_i32_16x16x64_i8 v[172:175], v[124:127], a[84:87], v[172:175]\n" \
  "buffer_load_dwordx4 a[164:167], v44, s[28:31], s93 offen offset:1024 sc0 sc1\n" \
  "s_waitcnt vmcnt(37)\n" \
  "s_barrier\n" \
  "v_mfma_i32_16x16x64_i8 v[204:207], v[120:123], a[72:75], v[204:207]\n" \
  "ds_read_b64_tr_b8 v[116:117], v38 offset:40960\n" \
  "v_mfma_i32_16x16x64_i8 v[204:207], v[124:127], a[88:91], v[204:207]\n" \
  "ds_read_b64_tr_b8 v[118:119], v38 offset:41984\n" \
  "v_mfma_i32_16x16x64_i8 v[236:239], v[120:123], a[76:79], v[236:239]\n" \
  "v_mfma_i32_16x16x64_i8 v[236:239], v[124:127], a[92:95], v[236:239]\n" \
  "s_waitcnt lgkmcnt(8)\n" \
  "v_mfma_i32_16x16x64_i8 v[144:147], v[96:99], a[64:67], v[144:147]\n" \
  "ds_read_b64_tr_b8 v[120:121], v39 offset:32768\n" \
  "v_mfma_i32_16x16x64_i8 v[144:147], v[100:103], a[80:83], v[144:147]\n" \
  "ds_read_b64_tr_b8 v[122:123], v39 offset:33792\n" \
  "v_mfma_i32_16x16x64_i8 v[176:179], v[96:99], a[68:71], v[176:179]\n" \
  "v_mfma_i32_16x16x64_i8 v[176:179], v[100:103], a[84:87], v[176:179]\n" \
  "buffer_load_dwordx4 a[168:171], v44, s[28:31], s93 offen offset:2048 sc0 sc1\n" \
  "v_mfma_i32_16x16x64_i8 v[208:211], v[96:99], a[72:75], v[208:211]\n" \
  "ds_read_b64_tr_b8 v[124:125], v39 offset:40960\n" \
  "v_mfma_i32_16x16x64_i8 v[208:211], v[100:103], a[88:91], v[208:211]\n" \
  "ds_read_b64_tr_b8 v[126:127], v39 offset:41984\n" \
  "v_mfma_i32_16x16x64_i8 v[240:243], v[96:99], a[76:79], v[240:243]\n" \
  "v_mfma_i32_16x16x64_i8 v[240:243], v[100:103], a[92:95], v[240:243]\n" \
  "buffer_load_dwordx4 a[172:175], v44, s[28:31], s93 offen offset:3072 sc0 sc1\n" \
  "s_waitcnt lgkmcnt(8)\n" \
  "v_mfma_i32_16x16x64_i8 v[148:151], v[104:107], a[64:67], v[148:151]\n" \
  "ds_read_b64_tr_b8 v[96:97], v32 offset:49152\n" \
  "v_mfma_i32_16x16x64_i8 v[148:151], v[108:111], a[80:83], v[148:151]\n" \
  "ds_read_b64_tr_b8 v[98:99], v32 offset:50176\n" \
  "v_mfma_i32_16x16x64_i8 v[180:183], v[104:107], a[68:71], v[180:183]\n" \
  "v_mfma_i32_16x16x64_i8 v[180:183], v[108:111], a[84:87], v[180:183]\n" \
  "buffer_load_dwordx4 a[176:179], v91, s[28:31], s93 offen offset:0 sc0 sc1\n" \
  "v_mfma_i32_16x16x64_i8 v[212:215], v[104:107], a[72:75], v[212:215]\n" \
  "ds_read_b64_tr_b8 v[100:101], v32 offset:57344\n" \
  "v_mfma_i32_16x16x64_i8 v[212:215], v[108:111], a[88:91], v[212:215]\n" \
  "ds_read_b64_tr_b8 v[102:103], v32 offset:58368\n" \
  "v_mfma_i32_16x16x64_i8 v[244:247], v[104:107], a[76:79], v[244:247]\n" \
  "v_mfma_i32_16x16x64_i8 v[244:247], v[108:111], a[92:95], v[244:247]\n" \
  "s_waitcnt lgkmcnt(8)\n" \
  "v_mfma_i32_16x16x64_i8 v[152:155], v[112:115], a[64:67], v[152:155]\n" \
  "ds_read_b64_tr_b8 v[104:105], v33 offset:49152\n" \
  "v_mfma_i32_16x16x64_i8 v[152:155], v[116:119], a[80:83], v[152:155]\n" \
  "ds_read_b64_tr_b8 v[106:107], v33 offset:50176\n" \
  "v_mfma_i32_16x16x64_i8 v[184:187], v[112:115], a[68:71], v[184:187]\n" \
  "v_mfma_i32_16x16x64_i8 v[184:187], v[116:119], a[84:87], v[184:187]\n" \
  "buffer_load_dwordx4 a[180:183], v91, s[28:31], s93 offen offset:1024 sc0 sc1\n" \
  "v_mfma_i32_16x16x64_i8 v[216:219], v[112:115], a[72:75], v[216:219]\n" \
  "ds_read_b64_tr_b8 v[108:109], v33 offset:57344\n" \
  "v_mfma_i32_16x16x64_i8 v[216:219], v[116:119], a[88:91], v[216:219]\n" \
  "ds_read_b64_tr_b8 v[110:111], v33 offset:58368\n" \
  "v_mfma_i32_16x16x64_i8 v[248:251], v[112:115], a[76:79], v[248:251]\n" \
  "v_mfma_i32_16x16x64_i8 v[248:251], v[116:119], a[92:95], v[248:251]\n" \
  "buffer_load_dwordx4 a[184:187], v91, s[28:31], s93 offen offset:2048 sc0 sc1\n" \
  "s_waitcnt lgkmcnt(8)\n" \
  "v_mfma_i32_16x16x64_i8 v[156:159], v[120:123], a[64:67], v[156:159]\n" \
  "ds_read_b64_tr_b8 v[112:113], v34 offset:49152\n" \
  "v_mfma_i32_16x16x64_i8 v[156:159], v[124:127], a[80:83], v[156:159]\n" \
  "ds_read_b64_tr_b8 v[114:115], v34 offset:50176\n" \
  "v_mfma_i32_16x16x64_i8 v[188:191], v[120:123], a[68:71], v[188:191]\n" \
  "v_mfma_i32_16x16x64_i8 v[188:191], v[124:127], a[84:87], v[188:191]\n" \
  "buffer_load_dwordx4 a[188:191], v91, s[28:31], s93 offen offset:3072 sc0 sc1\n" \
  "v_mfma_i32_16x16x64_i8 v[220:223], v[120:123], a[72:75], v[220:223]\n" \
  "ds_read_b64_tr_b8 v[116:117], v34 offset:57344\n" \
  "v_mfma_i32_16x16x64_i8 v[220:223], v[124:127], a[88:91], v[220:223]\n" \
  "ds_read_b64_tr_b8 v[118:119], v34 offset:58368\n" \
  "v_mfma_i32_16x16x64_i8 v[252:255], v[120:123], a[76:79], v[252:255]\n" \
  "v_mfma_i32_16x16x64_i8 v[252:255], v[124:127], a[92:95], v[252:255]\n" \
  "; ---- step 3\n" \
  "s_waitcnt vmcnt(16)\n" \
  "s_waitcnt lgkmcnt(8)\n" \
  "v_mfma_i32_16x16x64_i8 v[128:131], v[96:99], a[96:99], v[128:131]\n" \
  "ds_read_b64_tr_b8 v[120:121], v35 offset:49152\n" \
  "v_mfma_i32_16x16x64_i8 v[128:131], v[100:103], a[112:115], v[128:131]\n" \
  "ds_read_b64_tr_b8 v[122:123], v35 offset:50176\n" \
  "v_mfma_i32_16x16x64_i8 v[160:163], v[96:99], a[100:103], v[160:163]\n" \
  "v_mfma_i32_16x16x64_i8 v[160:163], v[100:103], a[116:119], v[160:163]\n" \
  "buffer_load_dwordx4 a[192:195], v44, s[28:31], s94 offen offset:0 sc0 sc1\n" \
  "v_mfma_i32_16x16x64_i8 v[192:195], v[96:99], a[104:107], v[192:195]\n" \
  "ds_read_b64_tr_b8 v[124:125], v35 offset:57344\n" \
  "v_mfma_i32_16x16x64_i8 v[192:195], v[100:103], a[120:123], v[192:195]\n" \
  "ds_read_b64_tr_b8 v[126:127], v35 offset:58368\n" \
  "v_mfma_i32_16x16x64_i8 v[224:227], v[96:99], a[108:111], v[224:227]\n" \
  "v_mfma_i32_16x16x64_i8 v[224:227], v[100:103], a[124:127], v[224:227]\n" \
  "buffer_load_dwordx4 a[196:199], v44, s[28:31], s94 offen offset:1024 sc0 sc1\n" \
  "s_waitcnt lgkmcnt(8)\n" \
  "v_mfma_i32_16x16x64_i8 v[132:135], v[104:107], a[96:99], v[132:135]\n" \
  "ds_read_b64_tr_b8 v[96:97], v36 offset:49152\n" \
  "v_mfma_i32_16x16x64_i8 v[132:135], v[108:111], a[112:115], v[132:135]\n" \
  "ds_read_b64_tr_b8 v[98:99], v36 offset:50176\n" \
  "v_mfma_i32_16x16x64_i8 v[164:167], v[104:107], a[100:103], v[164:167]\n" \
  "v_mfma_i32_16x16x64_i8 v[164:167], v[108:111], a[116:119], v[164:167]\n" \
  "buffer_load_dwordx4 a[200:203], v44, s[28:31], s94 offen offset:2048 sc0 sc1\n" \
  "v_mfma_i32_16x16x64_i8 v[196:199], v[104:107], a[104:107], v[196:199]\n" \
  "ds_read_b64_tr_b8 v[100:101], v36 offset:57344\n" \
  "v_mfma_i32_16x16x64_i8 v[196:199], v[108:111], a[120:123], v[196:199]\n" \
  "ds_read_b64_tr_b8 v[102:103], v36 offset:58368\n" \
  "v_mfma_i32_16x16x64_i8 v[228:231], v[104:107], a[108:111], v[228:231]\n" \
  "v_readlane_b32 s50, v46, s49\n" \
  "v_mfma_i32_16x16x64_i8 v[228:231], v[108:111], a[124:127], v[228:231]\n" \
  "buffer_load_dwordx4 a[204:207], v44, s[28:31], s94 offen offset:3072 sc0 sc1\n" \
  "s_waitcnt lgkmcnt(8)\n" \
  "v_mfma_i32_16x16x64_i8 v[136:139], v[112:115], a[96:99], v[136:139]\n" \
  "ds_read_b64_tr_b8 v[104:105], v37 offset:49152\n" \
  "v_mfma_i32_16x16x64_i8 v[136:139], v[116:119], a[112:115], v[136:139]\n" \
  "ds_read_b64_tr_b8 v[106:107], v37 offset:50176\n" \
  "v_mfma_i32_16x16x64_i8 v[168:171], v[112:115], a[100:103], v[168:171]\n" \
  "v_mfma_i32_16x16x64_i8 v[168:171], v[116:119], a[116:119], v[168:171]\n" \
  "buffer_load_dwordx4 a[208:211], v91, s[28:31], s94 offen offset:0 sc0 sc1\n" \
  "v_mfma_i32_16x16x64_i8 v[200:203], v[112:115], a[104:107], v[200:203]\n" \
  "ds_read_b64_tr_b8 v[108:109], v37 offset:57344\n" \
  "v_mfma_i32_16x16x64_i8 v[200:203], v[116:119], a[120:123], v[200:203]\n" \
  "ds_read_b64_tr_b8 v[110:111], v37 offset:58368\n" \
  "v_mfma_i32_16x16x64_i8 v[232:235], v[112:115], a[108:111], v[232:235]\n" \
  "s_lshl_b32 s87, s50, 21\n" \
  "s_lshl_b32 s95, s50, 16\n" \
  "s_add_u32 s49, s49, 1\n" \
  "v_mfma_i32_16x16x64_i8 v[232:235], v[116:119], a[124:127], v[232:235]\n" \
  "buffer_load_dwordx4 a[212:215], v91, s[28:31], s94 offen offset:1024 sc0 sc1\n" \
  "s_waitcnt lgkmcnt(8)\n" \
  "v_mfma_i32_16x16x64_i8 v[140:143], v[120:123], a[96:99], v[140:143]\n" \
  "ds_read_b64_tr_b8 v[112:113], v38 offset:49152\n" \
  "v_mfma_i32_16x16x64_i8 v[140:143], v[124:127], a[112:115], v[140:143]\n" \
  "ds_read_b64_tr_b8 v[114:115], v38 offset:50176\n" \
  "v_mfma_i32_16x16x64_i8 v[172:175], v[120:123], a[100:103], v[172:175]\n" \
  "v_readlane_b32 s50, v46, s49\n" \
  "v_mfma_i32_16x16x64_i8 v[172:175], v[124:127], a[116:119], v[172:175]\n" \
  "buffer_load_dwordx4 a[216:219], v91, s[28:31], s94 offen offset:2048 sc0 sc1\n" \
  "s_cmp_lg_u32 s46, s54\n" \
  "s_cbranch_scc1 Lskip25A_%=\n" \
  "s_waitcnt vmcnt(19)\n" \
  "Lskip25A_%=:\n" \
  "s_waitcnt vmcnt(40)\n" \
  "s_barrier\n" \
  "s_cmp_lg_u32 s46, s54\n" \
  "s_cbranch_scc1 Lskip27A_%=\n" \
  "s_cmp_lg_u32 s11, 0\n" \
  "s_cbranch_scc1 Lskip26A_%=\n" \
  "s_mov_b64 exec, 1\n" \
  "global_store_dword v53, v76, s[36:37] sc0 sc1\n" \
  "s_mov_b64 exec, -1\n" \
  "Lskip26A_%=:\n" \
  "Lskip27A_%=:\n" \
  "v_mfma_i32_16x16x64_i8 v[204:207], v[120:123], a[104:107], v[204:207]\n" \
  "ds_read_b64_tr_b8 v[116:117], v38 offset:57344\n" \
  "v_mfma_i32_16x16x64_i8 v[204:207], v[124:127], a[120:123], v[204:207]\n" \
  "ds_read_b64_tr_b8 v[118:119], v38 offset:58368\n" \
  "v_mfma_i32_16x16x64_i8 v[236:239], v[120:123], a[108:111], v[236:239]\n" \
  "v_mfma_i32_16x16x64_i8 v[236:239], v[124:127], a[124:127], v[236:239]\n" \
  "buffer_load_dwordx4 a[220:223], v91, s[28:31], s94 offen offset:3072 sc0 sc1\n" \
  "s_waitcnt lgkmcnt(8)\n" \
  "v_mfma_i32_16x16x64_i8 v[144:147], v[96:99], a[96:99], v[144:147]\n" \
  "ds_read_b64_tr_b8 v[120:121], v39 offset:49152\n" \
  "v_mfma_i32_16x16x64_i8 v[144:147], v[100:103], a[112:115], v[144:147]\n" \
  "ds_read_b64_tr_b8 v[122:123], v39 offset:50176\n" \
  "s_add_u32 m0, s47, 114688\n" \
  "v_mfma_i32_16x16x64_i8 v[176:179], v[96:99], a[100:103], v[176:179]\n" \
  "buffer_load_dwordx4 v40, s[16:19], s87 offen nt lds\n" \
  "s_lshl_b32 s80, s50, 21\n" \
  "s_lshl_b32 s88, s50, 16\n" \
  "s_add_u32 s49, s49, 1\n" \
  "v_mfma_i32_16x16x64_i8 v[176:179], v[100:103], a[116:119], v[176:179]\n" \
  "v_mfma_i32_16x16x64_i8 v[208:211], v[96:99], a[104:107], v[208:211]\n" \
  "ds_read_b64_tr_b8 v[124:125], v39 offset:57344\n" \
  "v_mfma_i32_16x16x64_i8 v[208:211], v[100:103], a[120:123], v[208:211]\n" \
  "ds_read_b64_tr_b8 v[126:127], v39 offset:58368\n" \
  "s_add_u32 m0, s47, 115712\n" \
  "v_mfma_i32_16x16x64_i8 v[240:243], v[96:99], a[108:111], v[240:243]\n" \
  "buffer_load_dwordx4 v41, s[16:19], s87 offen nt lds\n" \
  "v_readlane_b32 s50, v46, s49\n" \
  "v_mfma_i32_16x16x64_i8 v[240:243], v[100:103], a[124:127], v[240:243]\n" \
  "s_waitcnt lgkmcnt(8)\n" \
  "v_mfma_i32_16x16x64_i8 v[148:151], v[104:107], a[96:99], v[148:151]\n" \
  "ds_read_b64_tr_b8 v[96:97], v80 offset:0\n" \
  "v_mfma_i32_16x16x64_i8 v[148:151], v[108:111], a[112:115], v[148:151]\n" \
  "ds_read_b64_tr_b8 v[98:99], v80 offset:1024\n" \
  "s_add_u32 m0, s47, 116736\n" \
  "v_mfma_i32_16x16x64_i8 v[180:183], v[104:107], a[100:103], v[180:183]\n" \
  "buffer_load_dwordx4 v42, s[16:19], s87 offen nt lds\n" \
  "v_mfma_i32_16x16x64_i8 v[180:183], v[108:111], a[116:119], v[180:183]\n" \
  "v_mfma_i32_16x16x64_i8 v[212:215], v[104:107], a[104:107], v[212:215]\n" \
  "ds_read_b64_tr_b8 v[100:101], v80 offset:8192\n" \
  "v_mfma_i32_16x16x64_i8 v[212:215], v[108:111], a[120:123], v[212:215]\n" \
  "ds_read_b64_tr_b8 v[102:103], v80 offset:9216\n" \
  "s_add_u32 m0, s47, 117760\n" \
  "v_mfma_i32_16x16x64_i8 v[244:247], v[104:107], a[108:111], v[244:247]\n" \
  "buffer_load_dwordx4 v43, s[16:19], s87 offen nt lds\n" \
  "s_lshl_b32 s81, s50, 21\n" \
  "s_lshl_b32 s89, s50, 16\n" \
  "s_add_u32 s49, s49, 1\n" \
  "v_mfma_i32_16x16x64_i8 v[244:247], v[108:111], a[124:127], v[244:247]\n" \
  "s_waitcnt lgkmcnt(8)\n" \
  "v_mfma_i32_16x16x64_i8 v[152:155], v[112:115], a[96:99], v[152:155]\n" \
  "ds_read_b64_tr_b8 v[104:105], v81 offset:0\n" \
  "v_mfma_i32_16x16x64_i8 v[152:155], v[116:119], a[112:115], v[152:155]\n" \
  "ds_read_b64_tr_b8 v[106:107], v81 offset:1024\n" \
  "s_add_u32 m0, s47, 0\n" \
  "v_mfma_i32_16x16x64_i8 v[184:187], v[112:115], a[100:103], v[184:187]\n" \
  "buffer_load_dwordx4 v40, s[16:19], s80 offen nt lds\n" \
  "v_readlane_b32 s50, v46, s49\n" \
  "v_mfma_i32_16x16x64_i8 v[184:187], v[116:119], a[116:119], v[184:187]\n" \
  "s_cmp_lg_u32 s11, 0\n" \
  "s_cbranch_scc1 Lskip30A_%=\n" \
  "s_waitcnt vmcnt(25)\n" \
  "v_sub_u32 v72, v72, v76\n" \
  "v_cmp_gt_i32 vcc, 0, v72\n" \
  "s_nop 1\n" \
  "s_andn2_b64 vcc, vcc, s[14:15]\n" \
  "s_cbranch_scc1 Lslow28A_%=\n" \
  "Lback29A_%=:\n" \
  "v_add_u32 v50, 1024, v50\n" \
  "Lskip30A_%=:\n" \
  "v_mfma_i32_16x16x64_i8 v[216:219], v[112:115], a[104:107], v[216:219]\n" \
  "ds_read_b64_tr_b8 v[108:109], v81 offset:8192\n" \
  "v_mfma_i32_16x16x64_i8 v[216:219], v[116:119], a[120:123], v[216:219]\n" \
  "ds_read_b64_tr_b8 v[110:111], v81 offset:9216\n" \
  "s_add_u32 m0, s47, 1024\n" \
  "v_mfma_i32_16x16x64_i8 v[248:251], v[112:115], a[108:111], v[248:251]\n" \
  "buffer_load_dwordx4 v41, s[16:19], s80 offen nt lds\n" \
  "v_mfma_i32_16x16x64_i8 v[248:251], v[116:119], a[124:127], v[248:251]\n" \
  "s_waitcnt lgkmcnt(8)\n" \
  "v_mfma_i32_16x16x64_i8 v[156:159], v[120:123], a[96:99], v[156:159]\n" \
  "ds_read_b64_tr_b8 v[112:113], v82 offset:0\n" \
  "v_mfma_i32_16x16x64_i8 v[156:159], v[124:127], a[112:115], v[156:159]\n" \
  "ds_read_b64_tr_b8 v[114:115], v82 offset:1024\n" \
  "s_add_u32 m0, s47, 2048\n" \
  "v_mfma_i32_16x16x64_i8 v[188:191], v[120:123], a[100:103], v[188:191]\n" \
  "buffer_load_dwordx4 v42, s[16:19], s80 offen nt lds\n" \
  "s_lshl_b32 s82, s50, 21\n" \
  "s_lshl_b32 s90, s50, 16\n" \
  "s_add_u32 s49, s49, 1\n" \
  "v_mfma_i32_16x16x64_i8 v[188:191], v[124:127], a[116:119], v[188:191]\n" \
  "v_mfma_i32_16x16x64_i8 v[220:223], v[120:123], a[104:107], v[220:223]\n" \
  "ds_read_b64_tr_b8 v[116:117], v82 offset:8192\n" \
  "v_mfma_i32_16x16x64_i8 v[220:223], v[124:127], a[120:123], v[220:223]\n" \
  "ds_read_b64_tr_b8 v[118:119], v82 offset:9216\n" \
  "s_add_u32 m0, s47, 3072\n" \
  "v_mfma_i32_16x16x64_i8 v[252:255], v[120:123], a[108:111], v[252:255]\n" \
  "buffer_load_dwordx4 v43, s[16:19], s80 offen nt lds\n" \
  "v_mfma_i32_16x16x64_i8 v[252:255], v[124:127], a[124:127], v[252:255]\n" \
  "s_barrier\n" \
  "buffer_load_dword v92, v52, s[28:31], 0 offen sc0 sc1\n" \
  "v_add_u32 v52, 0x10000, v52\n" \
  "; ---- step 4\n" \
  "s_waitcnt vmcnt(25)\n" \
  "s_waitcnt lgkmcnt(8)\n" \
  "v_mfma_i32_16x16x64_i8 v[128:131], v[96:99], a[128:131], v[128:131]\n" \
  "ds_read_b64_tr_b8 v[120:121], v83 offset:0\n" \
  "s_add_u32 m0, s47, 16384\n" \
  "v_mfma_i32_16x16x64_i8 v[128:131], v[100:103], a[144:147], v[128:131]\n" \
  "ds_read_b64_tr_b8 v[122:123], v83 offset:1024\n" \
  "buffer_load_dwordx4 v40, s[16:19], s81 offen lds\n" \
  "s_add_u32 m0, s47, 17408\n" \
  "v_mfma_i32_16x16x64_i8 v[160:163], v[96:99], a[132:135], v[160:163]\n" \
  "buffer_load_dwordx4 v41, s[16:19], s81 offen lds\n" \
  "s_add_u32 m0, s47, 18432\n" \
  "v_mfma_i32_16x16x64_i8 v[160:163], v[100:103], a[148:151], v[160:163]\n" \
  "buffer_load_dwordx4 v42, s[16:19], s81 offen lds\n" \
  "s_add_u32 m0, s47, 19456\n" \
  "v_mfma_i32_16x16x64_i8 v[192:195], v[96:99], a[136:139], v[192:195]\n" \
  "ds_read_b64_tr_b8 v[124:125], v83 offset:8192\n" \
  "buffer_load_dwordx4 v43, s[16:19], s81 offen lds\n" \
  "s_add_u32 m0, s47, 32768\n" \
  "v_mfma_i32_16x16x64_i8 v[192:195], v[100:103], a[152:155], v[192:195]\n" \
  "ds_read_b64_tr_b8 v[126:127], v83 offset:9216\n" \
  "buffer_load_dwordx4 v40, s[16:19], s82 offen lds\n" \
  "s_add_u32 m0, s47, 33792\n" \
  "v_mfma_i32_16x16x64_i8 v[224:227], v[96:99], a[140:143], v[224:227]\n" \
  "buffer_load_dwordx4 v41, s[16:19], s82 offen lds\n" \
  "s_add_u32 m0, s47, 34816\n" \
  "v_mfma_i32_16x16x64_i8 v[224:227], v[100:103], a[156:159], v[224:227]\n" \
  "buffer_load_dwordx4 v42, s[16:19], s82 offen lds\n" \
  "s_add_u32 m0, s47, 35840\n" \
  "s_waitcnt lgkmcnt(8)\n" \
  "v_mfma_i32_16x16x64_i8 v[132:135], v[104:107], a[128:131], v[132:135]\n" \
  "ds_read_b64_tr_b8 v[96:97], v84 offset:0\n" \
  "buffer_load_dwordx4 v43, s[16:19], s82 offen lds\n" \
  "v_mfma_i32_16x16x64_i8 v[132:135], v[108:111], a[144:147], v[132:135]\n" \
  "ds_read_b64_tr_b8 v[98:99], v84 offset:1024\n" \
  "v_mfma_i32_16x16x64_i8 v[164:167], v[104:107], a[132:135], v[164:167]\n" \
  "v_mfma_i32_16x16x64_i8 v[164:167], v[108:111], a[148:151], v[164:167]\n" \
  "v_mfma_i32_16x16x64_i8 v[196:199], v[104:107], a[136:139], v[196:199]\n" \
  "ds_read_b64_tr_b8 v[100:101], v84 offset:8192\n" \
  "v_mfma_i32_16x16x64_i8 v[196:199], v[108:111], a[152:155], v[196:199]\n" \
  "ds_read_b64_tr_b8 v[102:103], v84 offset:9216\n" \
  "v_mfma_i32_16x16x64_i8 v[228:231], v[104:107], a[140:143], v[228:231]\n" \
  "v_mfma_i32_16x16x64_i8 v[228:231], v[108:111], a[156:159], v[228:231]\n" \
  "s_waitcnt lgkmcnt(8)\n" \
  "v_mfma_i32_16x16x64_i8 v[136:139], v[112:115], a[128:131], v[136:139]\n" \
  "ds_read_b64_tr_b8 v[104:105], v85 offset:0\n" \
  "v_mfma_i32_16x16x64_i8 v[136:139], v[116:119], a[144:147], v[136:139]\n" \
  "ds_read_b64_tr_b8 v[106:107], v85 offset:1024\n" \
  "v_mfma_i32_16x16x64_i8 v[168:171], v[112:115], a[132:135], v[168:171]\n" \
  "v_mfma_i32_16x16x64_i8 v[168:171], v[116:119], a[148:151], v[168:171]\n" \
  "v_mfma_i32_16x16x64_i8 v[200:203], v[112:115], a[136:139], v[200:203]\n" \
  "ds_read_b64_tr_b8 v[108:109], v85 offset:8192\n" \
  "v_mfma_i32_16x16x64_i8 v[200:203], v[116:119], a[152:155], v[200:203]\n" \
  "ds_read_b64_tr_b8 v[110:111], v85 offset:9216\n" \
  "v_mfma_i32_16x16x64_i8 v[232:235], v[112:115], a[140:143], v[232:235]\n" \
  "v_mfma_i32_16x16x64_i8 v[232:235], v[116:119], a[156:159], v[232:235]\n" \
  "s_waitcnt lgkmcnt(8)\n" \
  "v_mfma_i32_16x16x64_i8 v[140:143], v[120:123], a[128:131], v[140:143]\n" \
  "ds_read_b64_tr_b8 v[112:113], v86 offset:0\n" \
  "v_mfma_i32_16x16x64_i8 v[140:143], v[124:127], a[144:147], v[140:143]\n" \
  "ds_read_b64_tr_b8 v[114:115], v86 offset:1024\n" \
  "v_mfma_i32_16x16x64_i8 v[172:175], v[120:123], a[132:135], v[172:175]\n" \
  "v_mfma_i32_16x16x64_i8 v[172:175], v[124:127], a[148:151], v[172:175]\n" \
  "s_waitcnt vmcnt(53)\n" \
  "s_barrier\n" \
  "v_mfma_i32_16x16x64_i8 v[204:207], v[120:123], a[136:139], v[204:207]\n" \
  "ds_read_b64_tr_b8 v[116:117], v86 offset:8192\n" \
  "v_mfma_i32_16x16x64_i8 v[204:207], v[124:127], a[152:155], v[204:207]\n" \
  "ds_read_b64_tr_b8 v[118:119], v86 offset:9216\n" \
  "v_mfma_i32_16x16x64_i8 v[236:239], v[120:123], a[140:143], v[236:239]\n" \
  "v_mfma_i32_16x16x64_i8 v[236:239], v[124:127], a[156:159], v[236:239]\n" \
  "s_waitcnt lgkmcnt(8)\n" \
  "v_mfma_i32_16x16x64_i8 v[144:147], v[96:99], a[128:131], v[144:147]\n" \
  "ds_read_b64_tr_b8 v[120:121], v87 offset:0\n" \
  "v_mfma_i32_16x16x64_i8 v[144:147], v[100:103], a[144:147], v[144:147]\n" \
  "ds_read_b64_tr_b8 v[122:123], v87 offset:1024\n" \
  "v_mfma_i32_16x16x64_i8 v[176:179], v[96:99], a[132:135], v[176:179]\n" \
  "v_mfma_i32_16x16x64_i8 v[176:179], v[100:103], a[148:151], v[176:179]\n" \
  "v_mfma_i32_16x16x64_i8 v[208:211], v[96:99], a[136:139], v[208:211]\n" \
  "ds_read_b64_tr_b8 v[124:125], v87 offset:8192\n" \
  "v_mfma_i32_16x16x64_i8 v[208:211], v[100:103], a[152:155], v[208:211]\n" \
  "ds_read_b64_tr_b8 v[126:127], v87 offset:9216\n" \
  "v_mfma_i32_16x16x64_i8 v[240:243], v[96:99], a[140:143], v[240:243]\n" \
  "v_mfma_i32_16x16x64_i8 v[240:243], v[100:103], a[156:159], v[240:243]\n" \
  "s_waitcnt lgkmcnt(8)\n" \
  "v_mfma_i32_16x16x64_i8 v[148:151], v[104:107], a[128:131], v[148:151]\n" \
  "ds_read_b64_tr_b8 v[96:97], v80 offset:16384\n" \
  "v_mfma_i32_16x16x64_i8 v[148:151], v[108:111], a[144:147], v[148:151]\n" \
  "ds_read_b64_tr_b8 v[98:99], v80 offset:17408\n" \
  "v_mfma_i32_16x16x64_i8 v[180:183], v[104:107], a[132:135], v[180:183]\n" \
  "v_mfma_i32_16x16x64_i8 v[180:183], v[108:111], a[148:151], v[180:183]\n" \
  "v_mfma_i32_16x16x64_i8 v[212:215], v[104:107], a[136:139], v[212:215]\n" \
  "ds_read_b64_tr_b8 v[100:101], v80 offset:24576\n" \
  "v_mfma_i32_16x16x64_i8 v[212:215], v[108:111], a[152:155], v[212:215]\n" \
  "ds_read_b64_tr_b8 v[102:103], v80 offset:25600\n" \
  "v_mfma_i32_16x16x64_i8 v[244:247], v[104:107], a[140:143], v[244:247]\n" \
  "v_mfma_i32_16x16x64_i8 v[244:247], v[108:111], a[156:159], v[244:247]\n" \
  "s_waitcnt lgkmcnt(8)\n" \
  "v_mfma_i32_16x16x64_i8 v[152:155], v[112:115], a[128:131], v[152:155]\n" \
  "ds_read_b64_tr_b8 v[104:105], v81 offset:16384\n" \
  "v_mfma_i32_16x16x64_i8 v[152:155], v[116:119], a[144:147], v[152:155]\n" \
  "ds_read_b64_tr_b8 v[106:107], v81 offset:17408\n" \
  "v_mfma_i32_16x16x64_i8 v[184:187], v[112:115], a[132:135], v[184:187]\n" \
  "v_mfma_i32_16x16x64_i8 v[184:187], v[116:119], a[148:151], v[184:187]\n" \
  "v_mfma_i32_16x16x64_i8 v[216:219], v[112:115], a[136:139], v[216:219]\n" \
  "ds_read_b64_tr_b8 v[108:109], v81 offset:24576\n" \
  "v_mfma_i32_16x16x64_i8 v[216:219], v[116:119], a[152:155], v[216:219]\n" \
  "ds_read_b64_tr_b8 v[110:111], v81 offset:25600\n" \
  "v_mfma_i32_16x16x64_i8 v[248:251], v[112:115], a[140:143], v[248:251]\n" \
  "v_mfma_i32_16x16x64_i8 v[248:251], v[116:119], a[156:159], v[248:251]\n" \
  "s_waitcnt lgkmcnt(8)\n" \
  "v_mfma_i32_16x16x64_i8 v[156:159], v[120:123], a[128:131], v[156:159]\n" \
  "ds_read_b64_tr_b8 v[112:113], v82 offset:16384\n" \
  "v_mfma_i32_16x16x64_i8 v[156:159], v[124:127], a[144:147], v[156:159]\n" \
  "ds_read_b64_tr_b8 v[114:115], v82 offset:17408\n" \
  "v_mfma_i32_16x16x64_i8 v[188:191], v[120:123], a[132:135], v[188:191]\n" \
  "v_mfma_i32_16x16x64_i8 v[188:191], v[124:127], a[148:151], v[188:191]\n" \
  "v_mfma_i32_16x16x64_i8 v[220:223], v[120:123], a[136:139], v[220:223]\n" \
  "ds_read_b64_tr_b8 v[116:117], v82 offset:24576\n" \
  "v_mfma_i32_16x16x64_i8 v[220:223], v[124:127], a[152:155], v[220:223]\n" \
  "ds_read_b64_tr_b8 v[118:119], v82 offset:25600\n" \
  "v_mfma_i32_16x16x64_i8 v[252:255], v[120:123], a[140:143], v[252:255]\n" \
  "v_mfma_i32_16x16x64_i8 v[252:255], v[124:127], a[156:159], v[252:255]\n" \
  "; ---- step 5\n" \
  "s_waitcnt vmcnt(25)\n" \
  "s_waitcnt lgkmcnt(8)\n" \
  "v_mfma_i32_16x16x64_i8 v[128:131], v[96:99], a[160:163], v[128:131]\n" \
  "ds_read_b64_tr_b8 v[120:121], v83 offset:16384\n" \
  "v_mfma_i32_16x16x64_i8 v[128:131], v[100:103], a[176:179], v[128:131]\n" \
  "ds_read_b64_tr_b8 v[122:123], v83 offset:17408\n" \
  "v_mfma_i32_16x16x64_i8 v[160:163], v[96:99], a[164:167], v[160:163]\n" \
  "v_mfma_i32_16x16x64_i8 v[160:163], v[100:103], a[180:183], v[160:163]\n" \
  "v_mfma_i32_16x16x64_i8 v[192:195], v[96:99], a[168:171], v[192:195]\n" \
  "ds_read_b64_tr_b8 v[124:125], v83 offset:24576\n" \
  "v_mfma_i32_16x16x64_i8 v[192:195], v[100:103], a[184:187], v[192:195]\n" \
  "ds_read_b64_tr_b8 v[126:127], v83 offset:25600\n" \
  "v_mfma_i32_16x16x64_i8 v[224:227], v[96:99], a[172:175], v[224:227]\n" \
  "v_mfma_i32_16x16x64_i8 v[224:227], v[100:103], a[188:191], v[224:227]\n" \
  "s_waitcnt lgkmcnt(8)\n" \
  "v_mfma_i32_16x16x64_i8 v[132:135], v[104:107], a[160:163], v[132:135]\n" \
  "ds_read_b64_tr_b8 v[96:97], v84 offset:16384\n" \
  "v_mfma_i32_16x16x64_i8 v[132:135], v[108:111], a[176:179], v[132:135]\n" \
  "ds_read_b64_tr_b8 v[98:99], v84 offset:17408\n" \
  "v_mfma_i32_16x16x64_i8 v[164:167], v[104:107], a[164:167], v[164:167]\n" \
  "v_mfma_i32_16x16x64_i8 v[164:167], v[108:111], a[180:183], v[164:167]\n" \
  "v_mfma_i32_16x16x64_i8 v[196:199], v[104:107], a[168:171], v[196:199]\n" \
  "ds_read_b64_tr_b8 v[100:101], v84 offset:24576\n" \
  "v_mfma_i32_16x16x64_i8 v[196:199], v[108:111], a[184:187], v[196:199]\n" \
  "ds_read_b64_tr_b8 v[102:103], v84 offset:25600\n" \
  "v_mfma_i32_16x16x64_i8 v[228:231], v[104:107], a[172:175], v[228:231]\n" \
  "v_mfma_i32_16x16x64_i8 v[228:231], v[108:111], a[188:191], v[228:231]\n" \
  "s_waitcnt lgkmcnt(8)\n" \
  "v_mfma_i32_16x16x64_i8 v[136:139], v[112:115], a[160:163], v[136:139]\n" \
  "ds_read_b64_tr_b8 v[104:105], v85 offset:16384\n" \
  "v_mfma_i32_16x16x64_i8 v[136:139], v[116:119], a[176:179], v[136:139]\n" \
  "ds_read_b64_tr_b8 v[106:107], v85 offset:17408\n" \
  "v_mfma_i32_16x16x64_i8 v[168:171], v[112:115], a[164:167], v[168:171]\n" \
  "v_mfma_i32_16x16x64_i8 v[168:171], v[116:119], a[180:183], v[168:171]\n" \
  "v_mfma_i32_16x16x64_i8 v[200:203], v[112:115], a[168:171], v[200:203]\n" \
  "ds_read_b64_tr_b8 v[108:109], v85 offset:24576\n" \
  "v_mfma_i32_16x16x64_i8 v[200:203], v[116:119], a[184:187], v[200:203]\n" \
  "ds_read_b64_tr_b8 v[110:111], v85 offset:25600\n" \
  "buffer_load_dwordx4 a[224:227], v44, s[28:31], s95 offen offset:0 sc0 sc1\n" \
  "v_mfma_i32_16x16x64_i8 v[232:235], v[112:115], a[172:175], v[232:235]\n" \
  "v_mfma_i32_16x16x64_i8 v[232:235], v[116:119], a[188:191], v[232:235]\n" \
  "buffer_load_dwordx4 a[228:231], v44, s[28:31], s95 offen offset:1024 sc0 sc1\n" \
  "s_waitcnt lgkmcnt(8)\n" \
  "v_mfma_i32_16x16x64_i8 v[140:143], v[120:123], a[160:163], v[140:143]\n" \
  "ds_read_b64_tr_b8 v[112:113], v86 offset:16384\n" \
  "v_mfma_i32_16x16x64_i8 v[140:143], v[124:127], a[176:179], v[140:143]\n" \
  "ds_read_b64_tr_b8 v[114:115], v86 offset:17408\n" \
  "v_mfma_i32_16x16x64_i8 v[172:175], v[120:123], a[164:167], v[172:175]\n" \
  "v_mfma_i32_16x16x64_i8 v[172:175], v[124:127], a[180:183], v[172:175]\n" \
  "buffer_load_dwordx4 a[232:235], v44, s[28:31], s95 offen offset:2048 sc0 sc1\n" \
  "s_waitcnt vmcnt(52)\n" \
  "s_barrier\n" \
  "v_mfma_i32_16x16x64_i8 v[204:207], v[120:123], a[168:171], v[204:207]\n" \
  "ds_read_b64_tr_b8 v[116:117], v86 offset:24576\n" \
  "v_mfma_i32_16x16x64_i8 v[204:207], v[124:127], a[184:187], v[204:207]\n" \
  "ds_read_b64_tr_b8 v[118:119], v86 offset:25600\n" \
  "v_mfma_i32_16x16x64_i8 v[236:239], v[120:123], a[172:175], v[236:239]\n" \
  "v_mfma_i32_16x16x64_i8 v[236:239], v[124:127], a[188:191], v[236:239]\n" \
  "buffer_load_dwordx4 a[236:239], v44, s[28:31], s95 offen offset:3072 sc0 sc1\n" \
  "s_waitcnt lgkmcnt(8)\n" \
  "v_mfma_i32_16x16x64_i8 v[144:147], v[96:99], a[160:163], v[144:147]\n" \
  "ds_read_b64_tr_b8 v[120:121], v87 offset:16384\n" \
  "v_mfma_i32_16x16x64_i8 v[144:147], v[100:103], a[176:179], v[144:147]\n" \
  "ds_read_b64_tr_b8 v[122:123], v87 offset:17408\n" \
  "v_mfma_i32_16x16x64_i8 v[176:179], v[96:99], a[164:167], v[176:179]\n" \
  "v_mfma_i32_16x16x64_i8 v[176:179], v[100:103], a[180:183], v[176:179]\n" \
  "buffer_load_dwordx4 a[240:243], v91, s[28:31], s95 offen offset:0 sc0 sc1\n" \
  "v_mfma_i32_16x16x64_i8 v[208:211], v[96:99], a[168:171], v[208:211]\n" \
  "ds_read_b64_tr_b8 v[124:125], v87 offset:24576\n" \
  "v_mfma_i32_16x16x64_i8 v[208:211], v[100:103], a[184:187], v[208:211]\n" \
  "ds_read_b64_tr_b8 v[126:127], v87 offset:25600\n" \
  "v_mfma_i32_16x16x64_i8 v[240:243], v[96:99], a[172:175], v[240:243]\n" \
  "v_mfma_i32_16x16x64_i8 v[240:243], v[100:103], a[188:191], v[240:243]\n" \
  "buffer_load_dwordx4 a[244:247], v91, s[28:31], s95 offen offset:1024 sc0 sc1\n" \
  "s_waitcnt lgkmcnt(8)\n" \
  "v_mfma_i32_16x16x64_i8 v[148:151], v[104:107], a[160:163], v[148:151]\n" \
  "ds_read_b64_tr_b8 v[96:97], v80 offset:32768\n" \
  "v_mfma_i32_16x16x64_i8 v[148:151], v[108:111], a[176:179], v[148:151]\n" \
  "ds_read_b64_tr_b8 v[98:99], v80 offset:33792\n" \
  "v_mfma_i32_16x16x64_i8 v[180:183], v[104:107], a[164:167], v[180:183]\n" \
  "v_mfma_i32_16x16x64_i8 v[180:183], v[108:111], a[180:183], v[180:183]\n" \
  "buffer_load_dwordx4 a[248:251], v91, s[28:31], s95 offen offset:2048 sc0 sc1\n" \
  "v_mfma_i32_16x16x64_i8 v[212:215], v[104:107], a[168:171], v[212:215]\n" \
  "ds_read_b64_tr_b8 v[100:101], v80 offset:40960\n" \
  "v_mfma_i32_16x16x64_i8 v[212:215], v[108:111], a[184:187], v[212:215]\n" \
  "ds_read_b64_tr_b8 v[102:103], v80 offset:41984\n" \
  "buffer_load_dwordx4 a[252:255], v91, s[28:31], s95 offen offset:3072 sc0 sc1\n" \
  "v_mfma_i32_16x16x64_i8 v[244:247], v[104:107], a[172:175], v[244:247]\n" \
  "v_mfma_i32_16x16x64_i8 v[244:247], v[108:111], a[188:191], v[244:247]\n" \
  "s_waitcnt lgkmcnt(8)\n" \
  "v_mfma_i32_16x16x64_i8 v[152:155], v[112:115], a[160:163], v[152:155]\n" \
  "ds_read_b64_tr_b8 v[104:105], v81 offset:32768\n" \
  "v_mfma_i32_16x16x64_i8 v[152:155], v[116:119], a[176:179], v[152:155]\n" \
  "ds_read_b64_tr_b8 v[106:107], v81 offset:33792\n" \
  "buffer_load_dwordx4 a[0:3], v44, s[28:31], s88 offen offset:0 sc0 sc1\n" \
  "v_mfma_i32_16x16x64_i8 v[184:187], v[112:115], a[164:167], v[184:187]\n" \
  "v_mfma_i32_16x16x64_i8 v[184:187], v[116:119], a[180:183], v[184:187]\n" \
  "v_mfma_i32_16x16x64_i8 v[216:219], v[112:115], a[168:171], v[216:219]\n" \
  "ds_read_b64_tr_b8 v[108:109], v81 offset:40960\n" \
  "v_mfma_i32_16x16x64_i8 v[216:219], v[116:119], a[184:187], v[216:219]\n" \
  "ds_read_b64_tr_b8 v[110:111], v81 offset:41984\n" \
  "buffer_load_dwordx4 a[4:7], v44, s[28:31], s88 offen offset:1024 sc0 sc1\n" \
  "v_mfma_i32_16x16x64_i8 v[248:251], v[112:115], a[172:175], v[248:251]\n" \
  "v_mfma_i32_16x16x64_i8 v[248:251], v[116:119], a[188:191], v[248:251]\n" \
  "s_waitcnt lgkmcnt(8)\n" \
  "v_mfma_i32_16x16x64_i8 v[156:159], v[120:123], a[160:163], v[156:159]\n" \
  "ds_read_b64_tr_b8 v[112:113], v82 offset:32768\n" \
  "v_mfma_i32_16x16x64_i8 v[156:159], v[124:127], a[176:179], v[156:159]\n" \
  "ds_read_b64_tr_b8 v[114:115], v82 offset:33792\n" \
  "buffer_load_dwordx4 a[8:11], v44, s[28:31], s88 offen offset:2048 sc0 sc1\n" \
  "v_mfma_i32_16x16x64_i8 v[188:191], v[120:123], a[164:167], v[188:191]\n" \
  "v_mfma_i32_16x16x64_i8 v[188:191], v[124:127], a[180:183], v[188:191]\n" \
  "v_mfma_i32_16x16x64_i8 v[220:223], v[120:123], a[168:171], v[220:223]\n" \
  "ds_read_b64_tr_b8 v[116:117], v82 offset:40960\n" \
  "v_mfma_i32_16x16x64_i8 v[220:223], v[124:127], a[184:187], v[220:223]\n" \
  "ds_read_b64_tr_b8 v[118:119], v82 offset:41984\n" \
  "buffer_load_dwordx4 a[12:15], v44, s[28:31], s88 offen offset:3072 sc0 sc1\n" \
  "v_mfma_i32_16x16x64_i8 v[252:255], v[120:123], a[172:175], v[252:255]\n" \
  "v_mfma_i32_16x16x64_i8 v[252:255], v[124:127], a[188:191], v[252:255]\n" \
  "; ---- step 6\n" \
  "s_waitcnt vmcnt(29)\n" \
  "s_waitcnt lgkmcnt(8)\n" \
  "v_mfma_i32_16x16x64_i8 v[128:131], v[96:99], a[192:195], v[128:131]\n" \
  "ds_read_b64_tr_b8 v[120:121], v83 offset:32768\n" \
  "v_mfma_i32_16x16x64_i8 v[128:131], v[100:103], a[208:211], v[128:131]\n" \
  "ds_read_b64_tr_b8 v[122:123], v83 offset:33792\n" \
  "v_mfma_i32_16x16x64_i8 v[160:163], v[96:99], a[196:199], v[160:163]\n" \
  "v_mfma_i32_16x16x64_i8 v[160:163], v[100:103], a[212:215], v[160:163]\n" \
  "buffer_load_dwordx4 a[16:19], v91, s[28:31], s88 offen offset:0 sc0 sc1\n" \
  "v_mfma_i32_16x16x64_i8 v[192:195], v[96:99], a[200:203], v[192:195]\n" \
  "ds_read_b64_tr_b8 v[124:125], v83 offset:40960\n" \
  "v_mfma_i32_16x16x64_i8 v[192:195], v[100:103], a[216:219], v[192:195]\n" \
  "ds_read_b64_tr_b8 v[126:127], v83 offset:41984\n" \
  "v_mfma_i32_16x16x64_i8 v[224:227], v[96:99], a[204:207], v[224:227]\n" \
  "v_mfma_i32_16x16x64_i8 v[224:227], v[100:103], a[220:223], v[224:227]\n" \
  "buffer_load_dwordx4 a[20:23], v91, s[28:31], s88 offen offset:1024 sc0 sc1\n" \
  "s_waitcnt lgkmcnt(8)\n" \
  "v_mfma_i32_16x16x64_i8 v[132:135], v[104:107], a[192:195], v[132:135]\n" \
  "ds_read_b64_tr_b8 v[96:97], v84 offset:32768\n" \
  "v_mfma_i32_16x16x64_i8 v[132:135], v[108:111], a[208:211], v[132:135]\n" \
  "ds_read_b64_tr_b8 v[98:99], v84 offset:33792\n" \
  "v_mfma_i32_16x16x64_i8 v[164:167], v[104:107], a[196:199], v[164:167]\n" \
  "v_mfma_i32_16x16x64_i8 v[164:167], v[108:111], a[212:215], v[164:167]\n" \
  "buffer_load_dwordx4 a[24:27], v91, s[28:31], s88 offen offset:2048 sc0 sc1\n" \
  "v_mfma_i32_16x16x64_i8 v[196:199], v[104:107], a[200:203], v[196:199]\n" \
  "ds_read_b64_tr_b8 v[100:101], v84 offset:40960\n" \
  "v_mfma_i32_16x16x64_i8 v[196:199], v[108:111], a[216:219], v[196:199]\n" \
  "ds_read_b64_tr_b8 v[102:103], v84 offset:41984\n" \
  "v_mfma_i32_16x16x64_i8 v[228:231], v[104:107], a[204:207], v[228:231]\n" \
  "v_mfma_i32_16x16x64_i8 v[228:231], v[108:111], a[220:223], v[228:231]\n" \
  "s_waitcnt lgkmcnt(8)\n" \
  "v_mfma_i32_16x16x64_i8 v[136:139], v[112:115], a[192:195], v[136:139]\n" \
  "ds_read_b64_tr_b8 v[104:105], v85 offset:32768\n" \
  "v_mfma_i32_16x16x64_i8 v[136:139], v[116:119], a[208:211], v[136:139]\n" \
  "ds_read_b64_tr_b8 v[106:107], v85 offset:33792\n" \
  "v_mfma_i32_16x16x64_i8 v[168:171], v[112:115], a[196:199], v[168:171]\n" \
  "v_mfma_i32_16x16x64_i8 v[168:171], v[116:119], a[212:215], v[168:171]\n" \
  "buffer_load_dwordx4 a[28:31], v91, s[28:31], s88 offen offset:3072 sc0 sc1\n" \
  "v_mfma_i32_16x16x64_i8 v[200:203], v[112:115], a[200:203], v[200:203]\n" \
  "ds_read_b64_tr_b8 v[108:109], v85 offset:40960\n" \
  "v_mfma_i32_16x16x64_i8 v[200:203], v[116:119], a[216:219], v[200:203]\n" \
  "ds_read_b64_tr_b8 v[110:111], v85 offset:41984\n" \
  "v_mfma_i32_16x16x64_i8 v[232:235], v[112:115], a[204:207], v[232:235]\n" \
  "v_mfma_i32_16x16x64_i8 v[232:235], v[116:119], a[220:223], v[232:235]\n" \
  "buffer_load_dwordx4 a[32:35], v44, s[28:31], s89 offen offset:0 sc0 sc1\n" \
  "s_waitcnt lgkmcnt(8)\n" \
  "v_mfma_i32_16x16x64_i8 v[140:143], v[120:123], a[192:195], v[140:143]\n" \
  "ds_read_b64_tr_b8 v[112:113], v86 offset:32768\n" \
  "v_mfma_i32_16x16x64_i8 v[140:143], v[124:127], a[208:211], v[140:143]\n" \
  "ds_read_b64_tr_b8 v[114:115], v86 offset:33792\n" \
  "v_mfma_i32_16x16x64_i8 v[172:175], v[120:123], a[196:199], v[172:175]\n" \
  "v_mfma_i32_16x16x64_i8 v[172:175], v[124:127], a[212:215], v[172:175]\n" \
  "buffer_load_dwordx4 a[36:39], v44, s[28:31], s89 offen offset:1024 sc0 sc1\n" \
  "s_waitcnt vmcnt(31)\n" \
  "s_barrier\n" \
  "v_mfma_i32_16x16x64_i8 v[204:207], v[120:123], a[200:203], v[204:207]\n" \
  "ds_read_b64_tr_b8 v[116:117], v86 offset:40960\n" \
  "v_mfma_i32_16x16x64_i8 v[204:207], v[124:127], a[216:219], v[204:207]\n" \
  "ds_read_b64_tr_b8 v[118:119], v86 offset:41984\n" \
  "v_mfma_i32_16x16x64_i8 v[236:239], v[120:123], a[204:207], v[236:239]\n" \
  "v_mfma_i32_16x16x64_i8 v[236:239], v[124:127], a[220:223], v[236:239]\n" \
  "s_waitcnt lgkmcnt(8)\n" \
  "v_mfma_i32_16x16x64_i8 v[144:147], v[96:99], a[192:195], v[144:147]\n" \
  "ds_read_b64_tr_b8 v[120:121], v87 offset:32768\n" \
  "v_mfma_i32_16x16x64_i8 v[144:147], v[100:103], a[208:211], v[144:147]\n" \
  "ds_read_b64_tr_b8 v[122:123], v87 offset:33792\n" \
  "v_mfma_i32_16x16x64_i8 v[176:179], v[96:99], a[196:199], v[176:179]\n" \
  "v_mfma_i32_16x16x64_i8 v[176:179], v[100:103], a[212:215], v[176:179]\n" \
  "buffer_load_dwordx4 a[40:43], v44, s[28:31], s89 offen offset:2048 sc0 sc1\n" \
  "v_mfma_i32_16x16x64_i8 v[208:211], v[96:99], a[200:203], v[208:211]\n" \
  "ds_read_b64_tr_b8 v[124:125], v87 offset:40960\n" \
  "v_mfma_i32_16x16x64_i8 v[208:211], v[100:103], a[216:219], v[208:211]\n" \
  "ds_read_b64_tr_b8 v[126:127], v87 offset:41984\n" \
  "v_mfma_i32_16x16x64_i8 v[240:243], v[96:99], a[204:207], v[240:243]\n" \
  "v_mfma_i32_16x16x64_i8 v[240:243], v[100:103], a[220:223], v[240:243]\n" \
  "buffer_load_dwordx4 a[44:47], v44, s[28:31], s89 offen offset:3072 sc0 sc1\n" \
  "s_waitcnt lgkmcnt(8)\n" \
  "v_mfma_i32_16x16x64_i8 v[148:151], v[104:107], a[192:195], v[148:151]\n" \
  "ds_read_b64_tr_b8 v[96:97], v80 offset:49152\n" \
  "v_mfma_i32_16x16x64_i8 v[148:151], v[108:111], a[208:211], v[148:151]\n" \
  "ds_read_b64_tr_b8 v[98:99], v80 offset:50176\n" \
  "v_mfma_i32_16x16x64_i8 v[180:183], v[104:107], a[196:199], v[180:183]\n" \
  "v_mfma_i32_16x16x64_i8 v[180:183], v[108:111], a[212:215], v[180:183]\n" \
  "buffer_load_dwordx4 a[48:51], v91, s[28:31], s89 offen offset:0 sc0 sc1\n" \
  "v_mfma_i32_16x16x64_i8 v[212:215], v[104:107], a[200:203], v[212:215]\n" \
  "ds_read_b64_tr_b8 v[100:101], v80 offset:57344\n" \
  "v_mfma_i32_16x16x64_i8 v[212:215], v[108:111], a[216:219], v[212:215]\n" \
  "ds_read_b64_tr_b8 v[102:103], v80 offset:58368\n" \
  "v_mfma_i32_16x16x64_i8 v[244:247], v[104:107], a[204:207], v[244:247]\n" \
  "v_mfma_i32_16x16x64_i8 v[244:247], v[108:111], a[220:223], v[244:247]\n" \
  "s_waitcnt lgkmcnt(8)\n" \
  "v_mfma_i32_16x16x64_i8 v[152:155], v[112:115], a[192:195], v[152:155]\n" \
  "ds_read_b64_tr_b8 v[104:105], v81 offset:49152\n" \
  "v_mfma_i32_16x16x64_i8 v[152:155], v[116:119], a[208:211], v[152:155]\n" \
  "ds_read_b64_tr_b8 v[106:107], v81 offset:50176\n" \
  "v_mfma_i32_16x16x64_i8 v[184:187], v[112:115], a[196:199], v[184:187]\n" \
  "v_mfma_i32_16x16x64_i8 v[184:187], v[116:119], a[212:215], v[184:187]\n" \
  "buffer_load_dwordx4 a[52:55], v91, s[28:31], s89 offen offset:1024 sc0 sc1\n" \
  "v_mfma_i32_16x16x64_i8 v[216:219], v[112:115], a[200:203], v[216:219]\n" \
  "ds_read_b64_tr_b8 v[108:109], v81 offset:57344\n" \
  "v_mfma_i32_16x16x64_i8 v[216:219], v[116:119], a[216:219], v[216:219]\n" \
  "ds_read_b64_tr_b8 v[110:111], v81 offset:58368\n" \
  "v_mfma_i32_16x16x64_i8 v[248:251], v[112:115], a[204:207], v[248:251]\n" \
  "v_mfma_i32_16x16x64_i8 v[248:251], v[116:119], a[220:223], v[248:251]\n" \
  "buffer_load_dwordx4 a[56:59], v91, s[28:31], s89 offen offset:2048 sc0 sc1\n" \
  "s_waitcnt lgkmcnt(8)\n" \
  "v_mfma_i32_16x16x64_i8 v[156:159], v[120:123], a[192:195], v[156:159]\n" \
  "ds_read_b64_tr_b8 v[112:113], v82 offset:49152\n" \
  "v_mfma_i32_16x16x64_i8 v[156:159], v[124:127], a[208:211], v[156:159]\n" \
  "ds_read_b64_tr_b8 v[114:115], v82 offset:50176\n" \
  "v_mfma_i32_16x16x64_i8 v[188:191], v[120:123], a[196:199], v[188:191]\n" \
  "v_mfma_i32_16x16x64_i8 v[188:191], v[124:127], a[212:215], v[188:191]\n" \
  "buffer_load_dwordx4 a[60:63], v91, s[28:31], s89 offen offset:3072 sc0 sc1\n" \
  "v_mfma_i32_16x16x64_i8 v[220:223], v[120:123], a[200:203], v[220:223]\n" \
  "ds_read_b64_tr_b8 v[116:117], v82 offset:57344\n" \
  "v_mfma_i32_16x16x64_i8 v[220:223], v[124:127], a[216:219], v[220:223]\n" \
  "ds_read_b64_tr_b8 v[118:119], v82 offset:58368\n" \
  "v_mfma_i32_16x16x64_i8 v[252:255], v[120:123], a[204:207], v[252:255]\n" \
  "v_mfma_i32_16x16x64_i8 v[252:255], v[124:127], a[220:223], v[252:255]\n" \
  "; ---- step 7\n" \
  "s_waitcnt vmcnt(16)\n" \
  "s_waitcnt lgkmcnt(8)\n" \
  "v_mfma_i32_16x16x64_i8 v[128:131], v[96:99], a[224:227], v[128:131]\n" \
  "ds_read_b64_tr_b8 v[120:121], v83 offset:49152\n" \
  "v_mfma_i32_16x16x64_i8 v[128:131], v[100:103], a[240:243], v[128:131]\n" \
  "ds_read_b64_tr_b8 v[122:123], v83 offset:50176\n" \
  "v_mfma_i32_16x16x64_i8 v[160:163], v[96:99], a[228:231], v[160:163]\n" \
  "v_mfma_i32_16x16x64_i8 v[160:163], v[100:103], a[244:247], v[160:163]\n" \
  "buffer_load_dwordx4 a[64:67], v44, s[28:31], s90 offen offset:0 sc0 sc1\n" \
  "v_mfma_i32_16x16x64_i8 v[192:195], v[96:99], a[232:235], v[192:195]\n" \
  "ds_read_b64_tr_b8 v[124:125], v83 offset:57344\n" \
  "v_mfma_i32_16x16x64_i8 v[192:195], v[100:103], a[248:251], v[192:195]\n" \
  "ds_read_b64_tr_b8 v[126:127], v83 offset:58368\n" \
  "v_mfma_i32_16x16x64_i8 v[224:227], v[96:99], a[236:239], v[224:227]\n" \
  "v_mfma_i32_16x16x64_i8 v[224:227], v[100:103], a[252:255], v[224:227]\n" \
  "buffer_load_dwordx4 a[68:71], v44, s[28:31], s90 offen offset:1024 sc0 sc1\n" \
  "s_waitcnt lgkmcnt(8)\n" \
  "v_mfma_i32_16x16x64_i8 v[132:135], v[104:107], a[224:227], v[132:135]\n" \
  "ds_read_b64_tr_b8 v[96:97], v84 offset:49152\n" \
  "v_mfma_i32_16x16x64_i8 v[132:135], v[108:111], a[240:243], v[132:135]\n" \
  "ds_read_b64_tr_b8 v[98:99], v84 offset:50176\n" \
  "v_mfma_i32_16x16x64_i8 v[164:167], v[104:107], a[228:231], v[164:167]\n" \
  "v_mfma_i32_16x16x64_i8 v[164:167], v[108:111], a[244:247], v[164:167]\n" \
  "buffer_load_dwordx4 a[72:75], v44, s[28:31], s90 offen offset:2048 sc0 sc1\n" \
  "v_mfma_i32_16x16x64_i8 v[196:199], v[104:107], a[232:235], v[196:199]\n" \
  "ds_read_b64_tr_b8 v[100:101], v84 offset:57344\n" \
  "v_mfma_i32_16x16x64_i8 v[196:199], v[108:111], a[248:251], v[196:199]\n" \
  "ds_read_b64_tr_b8 v[102:103], v84 offset:58368\n" \
  "v_mfma_i32_16x16x64_i8 v[228:231], v[104:107], a[236:239], v[228:231]\n" \
  "v_readlane_b32 s50, v46, s49\n" \
  "v_mfma_i32_16x16x64_i8 v[228:231], v[108:111], a[252:255], v[228:231]\n" \
  "buffer_load_dwordx4 a[76:79], v44, s[28:31], s90 offen offset:3072 sc0 sc1\n" \
  "s_waitcnt lgkmcnt(8)\n" \
  "v_mfma_i32_16x16x64_i8 v[136:139], v[112:115], a[224:227], v[136:139]\n" \
  "ds_read_b64_tr_b8 v[104:105], v85 offset:49152\n" \
  "v_mfma_i32_16x16x64_i8 v[136:139], v[116:119], a[240:243], v[136:139]\n" \
  "ds_read_b64_tr_b8 v[106:107], v85 offset:50176\n" \
  "v_mfma_i32_16x16x64_i8 v[168:171], v[112:115], a[228:231], v[168:171]\n" \
  "v_mfma_i32_16x16x64_i8 v[168:171], v[116:119], a[244:247], v[168:171]\n" \
  "buffer_load_dwordx4 a[80:83], v91, s[28:31], s90 offen offset:0 sc0 sc1\n" \
  "v_mfma_i32_16x16x64_i8 v[200:203], v[112:115], a[232:235], v[200:203]\n" \
  "ds_read_b64_tr_b8 v[108:109], v85 offset:57344\n" \
  "v_mfma_i32_16x16x64_i8 v[200:203], v[116:119], a[248:251], v[200:203]\n" \
  "ds_read_b64_tr_b8 v[110:111], v85 offset:58368\n" \
  "v_mfma_i32_16x16x64_i8 v[232:235], v[112:115], a[236:239], v[232:235]\n" \
  "s_lshl_b32 s83, s50, 21\n" \
  "s_lshl_b32 s91, s50, 16\n" \
  "s_add_u32 s49, s49, 1\n" \
  "v_mfma_i32_16x16x64_i8 v[232:235], v[116:119], a[252:255], v[232:235]\n" \
  "buffer_load_dwordx4 a[84:87], v91, s[28:31], s90 offen offset:1024 sc0 sc1\n" \
  "s_waitcnt lgkmcnt(8)\n" \
  "v_mfma_i32_16x16x64_i8 v[140:143], v[120:123], a[224:227], v[140:143]\n" \
  "ds_read_b64_tr_b8 v[112:113], v86 offset:49152\n" \
  "v_mfma_i32_16x16x64_i8 v[140:143], v[124:127], a[240:243], v[140:143]\n" \
  "ds_read_b64_tr_b8 v[114:115], v86 offset:50176\n" \
  "v_mfma_i32_16x16x64_i8 v[172:175], v[120:123], a[228:231], v[172:175]\n" \
  "v_readlane_b32 s50, v46, s49\n" \
  "v_mfma_i32_16x16x64_i8 v[172:175], v[124:127], a[244:247], v[172:175]\n" \
  "buffer_load_dwordx4 a[88:91], v91, s[28:31], s90 offen offset:2048 sc0 sc1\n" \
  "s_waitcnt vmcnt(40)\n" \
  "s_barrier\n" \
  "v_mfma_i32_16x16x64_i8 v[204:207], v[120:123], a[232:235], v[204:207]\n" \
  "ds_read_b64_tr_b8 v[116:117], v86 offset:57344\n" \
  "v_mfma_i32_16x16x64_i8 v[204:207], v[124:127], a[248:251], v[204:207]\n" \
  "ds_read_b64_tr_b8 v[118:119], v86 offset:58368\n" \
  "v_mfma_i32_16x16x64_i8 v[236:239], v[120:123], a[236:239], v[236:239]\n" \
  "v_mfma_i32_16x16x64_i8 v[236:239], v[124:127], a[252:255], v[236:239]\n" \
  "buffer_load_dwordx4 a[92:95], v91, s[28:31], s90 offen offset:3072 sc0 sc1\n" \
  "s_waitcnt lgkmcnt(8)\n" \
  "v_mfma_i32_16x16x64_i8 v[144:147], v[96:99], a[224:227], v[144:147]\n" \
  "ds_read_b64_tr_b8 v[120:121], v87 offset:49152\n" \
  "v_mfma_i32_16x16x64_i8 v[144:147], v[100:103], a[240:243], v[144:147]\n" \
  "ds_read_b64_tr_b8 v[122:123], v87 offset:50176\n" \
  "s_add_u32 m0, s47, 49152\n" \
  "v_mfma_i32_16x16x64_i8 v[176:179], v[96:99], a[228:231], v[176:179]\n" \
  "buffer_load_dwordx4 v40, s[16:19], s83 offen nt lds\n" \
  "s_lshl_b32 s84, s50, 21\n" \
  "s_lshl_b32 s92, s50, 16\n" \
  "s_add_u32 s49, s49, 1\n" \
  "v_mfma_i32_16x16x64_i8 v[176:179], v[100:103], a[244:247], v[176:179]\n" \
  "v_mfma_i32_16x16x64_i8 v[208:211], v[96:99], a[232:235], v[208:211]\n" \
  "ds_read_b64_tr_b8 v[124:125], v87 offset:57344\n" \
  "v_mfma_i32_16x16x64_i8 v[208:211], v[100:103], a[248:251], v[208:211]\n" \
  "ds_read_b64_tr_b8 v[126:127], v87 offset:58368\n" \
  "s_add_u32 m0, s47, 50176\n" \
  "v_mfma_i32_16x16x64_i8 v[240:243], v[96:99], a[236:239], v[240:243]\n" \
  "buffer_load_dwordx4 v41, s[16:19], s83 offen nt lds\n" \
  "v_readlane_b32 s50, v46, s49\n" \
  "v_mfma_i32_16x16x64_i8 v[240:243], v[100:103], a[252:255], v[240:243]\n" \
  "s_waitcnt lgkmcnt(8)\n" \
  "v_mfma_i32_16x16x64_i8 v[148:151], v[104:107], a[224:227], v[148:151]\n" \
  "ds_read_b64_tr_b8 v[96:97], v32 offset:0\n" \
  "v_mfma_i32_16x16x64_i8 v[148:151], v[108:111], a[240:243], v[148:151]\n" \
  "ds_read_b64_tr_b8 v[98:99], v32 offset:1024\n" \
  "s_add_u32 m0, s47, 51200\n" \
  "v_mfma_i32_16x16x64_i8 v[180:183], v[104:107], a[228:231], v[180:183]\n" \
  "buffer_load_dwordx4 v42, s[16:19], s83 offen nt lds\n" \
  "v_mfma_i32_16x16x64_i8 v[180:183], v[108:111], a[244:247], v[180:183]\n" \
  "v_mfma_i32_16x16x64_i8 v[212:215], v[104:107], a[232:235], v[212:215]\n" \
  "ds_read_b64_tr_b8 v[100:101], v32 offset:8192\n" \
  "v_mfma_i32_16x16x64_i8 v[212:215], v[108:111], a[248:251], v[212:215]\n" \
  "ds_read_b64_tr_b8 v[102:103], v32 offset:9216\n" \
  "s_add_u32 m0, s47, 52224\n" \
  "v_mfma_i32_16x16x64_i8 v[244:247], v[104:107], a[236:239], v[244:247]\n" \
  "buffer_load_dwordx4 v43, s[16:19], s83 offen nt lds\n" \
  "s_lshl_b32 s85, s50, 21\n" \
  "s_lshl_b32 s93, s50, 16\n" \
  "s_add_u32 s49, s49, 1\n" \
  "v_mfma_i32_16x16x64_i8 v[244:247], v[108:111], a[252:255], v[244:247]\n" \
  "s_waitcnt lgkmcnt(8)\n" \
  "v_mfma_i32_16x16x64_i8 v[152:155], v[112:115], a[224:227], v[152:155]\n" \
  "ds_read_b64_tr_b8 v[104:105], v33 offset:0\n" \
  "v_mfma_i32_16x16x64_i8 v[152:155], v[116:119], a[240:243], v[152:155]\n" \
  "ds_read_b64_tr_b8 v[106:107], v33 offset:1024\n" \
  "s_add_u32 m0, s47, 65536\n" \
  "v_mfma_i32_16x16x64_i8 v[184:187], v[112:115], a[228:231], v[184:187]\n" \
  "buffer_load_dwordx4 v40, s[16:19], s84 offen nt lds\n" \
  "v_readlane_b32 s50, v46, s49\n" \
  "v_mfma_i32_16x16x64_i8 v[184:187], v[116:119], a[244:247], v[184:187]\n" \
  "v_mfma_i32_16x16x64_i8 v[216:219], v[112:115], a[232:235], v[216:219]\n" \
  "ds_read_b64_tr_b8 v[108:109], v33 offset:8192\n" \
  "v_mfma_i32_16x16x64_i8 v[216:219], v[116:119], a[248:251], v[216:219]\n" \
  "ds_read_b64_tr_b8 v[110:111], v33 offset:9216\n" \
  "s_add_u32 m0, s47, 66560\n" \
  "v_mfma_i32_16x16x64_i8 v[248:251], v[112:115], a[236:239], v[248:251]\n" \
  "buffer_load_dwordx4 v41, s[16:19], s84 offen nt lds\n" \
  "v_mfma_i32_16x16x64_i8 v[248:251], v[116:119], a[252:255], v[248:251]\n" \
  "s_waitcnt lgkmcnt(8)\n" \
  "v_mfma_i32_16x16x64_i8 v[156:159], v[120:123], a[224:227], v[156:159]\n" \
  "ds_read_b64_tr_b8 v[112:113], v34 offset:0\n" \
  "v_mfma_i32_16x16x64_i8 v[156:159], v[124:127], a[240:243], v[156:159]\n" \
  "ds_read_b64_tr_b8 v[114:115], v34 offset:1024\n" \
  "s_add_u32 m0, s47, 67584\n" \
  "v_mfma_i32_16x16x64_i8 v[188:191], v[120:123], a[228:231], v[188:191]\n" \
  "buffer_load_dwordx4 v42, s[16:19], s84 offen nt lds\n" \
  "s_lshl_b32 s86, s50, 21\n" \
  "s_lshl_b32 s94, s50, 16\n" \
  "s_add_u32 s49, s49, 1\n" \
  "v_mfma_i32_16x16x64_i8 v[188:191], v[124:127], a[244:247], v[188:191]\n" \
  "v_mfma_i32_16x16x64_i8 v[220:223], v[120:123], a[232:235], v[220:223]\n" \
  "ds_read_b64_tr_b8 v[116:117], v34 offset:8192\n" \
  "v_mfma_i32_16x16x64_i8 v[220:223], v[124:127], a[248:251], v[220:223]\n" \
  "ds_read_b64_tr_b8 v[118:119], v34 offset:9216\n" \
  "s_add_u32 m0, s47, 68608\n" \
  "v_mfma_i32_16x16x64_i8 v[252:255], v[120:123], a[236:239], v[252:255]\n" \
  "buffer_load_dwordx4 v43, s[16:19], s84 offen nt lds\n" \
  "v_mfma_i32_16x16x64_i8 v[252:255], v[124:127], a[252:255], v[252:255]\n" \
  "s_add_u32 s53, s53, 1\n" \
  "s_add_u32 s54, s54, 1\n" \
  "s_sub_u32 s51, s51, 1\n" \
  "s_cmp_lg_u32 s51, 0\n" \
  "s_cbranch_scc1 L_loopA_%=\n" \
  "; ---- step 0\n" \
  "s_waitcnt vmcnt(24)\n" \
  "s_waitcnt lgkmcnt(8)\n" \
  "v_mfma_i32_16x16x64_i8 v[128:131], v[96:99], a[0:3], v[128:131]\n" \
  "ds_read_b64_tr_b8 v[120:121], v35 offset:0\n" \
  "s_add_u32 m0, s47, 81920\n" \
  "v_mfma_i32_16x16x64_i8 v[128:131], v[100:103], a[16:19], v[128:131]\n" \
  "ds_read_b64_tr_b8 v[122:123], v35 offset:1024\n" \
  "buffer_load_dwordx4 v40, s[16:19], s85 offen lds\n" \
  "s_add_u32 m0, s47, 82944\n" \
  "v_mfma_i32_16x16x64_i8 v[160:163], v[96:99], a[4:7], v[160:163]\n" \
  "buffer_load_dwordx4 v41, s[16:19], s85 offen lds\n" \
  "s_add_u32 m0, s47, 83968\n" \
  "v_mfma_i32_16x16x64_i8 v[160:163], v[100:103], a[20:23], v[160:163]\n" \
  "buffer_load_dwordx4 v42, s[16:19], s85 offen lds\n" \
  "s_add_u32 m0, s47, 84992\n" \
  "v_mfma_i32_16x16x64_i8 v[192:195], v[96:99], a[8:11], v[192:195]\n" \
  "ds_read_b64_tr_b8 v[124:125], v35 offset:8192\n" \
  "buffer_load_dwordx4 v43, s[16:19], s85 offen lds\n" \
  "s_add_u32 m0, s47, 98304\n" \
  "v_mfma_i32_16x16x64_i8 v[192:195], v[100:103], a[24:27], v[192:195]\n" \
  "ds_read_b64_tr_b8 v[126:127], v35 offset:9216\n" \
  "buffer_load_dwordx4 v40, s[16:19], s86 offen lds\n" \
  "s_add_u32 m0, s47, 99328\n" \
  "v_mfma_i32_16x16x64_i8 v[224:227], v[96:99], a[12:15], v[224:227]\n" \
  "buffer_load_dwordx4 v41, s[16:19], s86 offen lds\n" \
  "s_add_u32 m0, s47, 100352\n" \
  "v_mfma_i32_16x16x64_i8 v[224:227], v[100:103], a[28:31], v[224:227]\n" \
  "buffer_load_dwordx4 v42, s[16:19], s86 offen lds\n" \
  "s_add_u32 m0, s47, 101376\n" \
  "s_waitcnt lgkmcnt(8)\n" \
  "v_mfma_i32_16x16x64_i8 v[132:135], v[104:107], a[0:3], v[132:135]\n" \
  "ds_read_b64_tr_b8 v[96:97], v36 offset:0\n" \
  "buffer_load_dwordx4 v43, s[16:19], s86 offen lds\n" \
  "v_mfma_i32_16x16x64_i8 v[132:135], v[108:111], a[16:19], v[132:135]\n" \
  "ds_read_b64_tr_b8 v[98:99], v36 offset:1024\n" \
  "v_mfma_i32_16x16x64_i8 v[164:167], v[104:107], a[4:7], v[164:167]\n" \
  "v_mfma_i32_16x16x64_i8 v[164:167], v[108:111], a[20:23], v[164:167]\n" \
  "v_mfma_i32_16x16x64_i8 v[196:199], v[104:107], a[8:11], v[196:199]\n" \
  "ds_read_b64_tr_b8 v[100:101], v36 offset:8192\n" \
  "v_mfma_i32_16x16x64_i8 v[196:199], v[108:111], a[24:27], v[196:199]\n" \
  "ds_read_b64_tr_b8 v[102:103], v36 offset:9216\n" \
  "v_mfma_i32_16x16x64_i8 v[228:231], v[104:107], a[12:15], v[228:231]\n" \
  "v_mfma_i32_16x16x64_i8 v[228:231], v[108:111], a[28:31], v[228:231]\n" \
  "s_waitcnt lgkmcnt(8)\n" \
  "v_mfma_i32_16x16x64_i8 v[136:139], v[112:115], a[0:3], v[136:139]\n" \
  "ds_read_b64_tr_b8 v[104:105], v37 offset:0\n" \
  "v_mfma_i32_16x16x64_i8 v[136:139], v[116:119], a[16:19], v[136:139]\n" \
  "ds_read_b64_tr_b8 v[106:107], v37 offset:1024\n" \
  "v_mfma_i32_16x16x64_i8 v[168:171], v[112:115], a[4:7], v[168:171]\n" \
  "v_mfma_i32_16x16x64_i8 v[168:171], v[116:119], a[20:23], v[168:171]\n" \
  "v_mfma_i32_16x16x64_i8 v[200:203], v[112:115], a[8:11], v[200:203]\n" \
  "ds_read_b64_tr_b8 v[108:109], v37 offset:8192\n" \
  "v_mfma_i32_16x16x64_i8 v[200:203], v[116:119], a[24:27], v[200:203]\n" \
  "ds_read_b64_tr_b8 v[110:111], v37 offset:9216\n" \
  "v_mfma_i32_16x16x64_i8 v[232:235], v[112:115], a[12:15], v[232:235]\n" \
  "v_mfma_i32_16x16x64_i8 v[232:235], v[116:119], a[28:31], v[232:235]\n" \
  "s_waitcnt lgkmcnt(8)\n" \
  "v_mfma_i32_16x16x64_i8 v[140:143], v[120:123], a[0:3], v[140:143]\n" \
  "ds_read_b64_tr_b8 v[112:113], v38 offset:0\n" \
  "v_mfma_i32_16x16x64_i8 v[140:143], v[124:127], a[16:19], v[140:143]\n" \
  "ds_read_b64_tr_b8 v[114:115], v38 offset:1024\n" \
  "v_mfma_i32_16x16x64_i8 v[172:175], v[120:123], a[4:7], v[172:175]\n" \
  "v_mfma_i32_16x16x64_i8 v[172:175], v[124:127], a[20:23], v[172:175]\n" \
  "s_waitcnt vmcnt(52)\n" \
  "s_barrier\n" \
  "v_mfma_i32_16x16x64_i8 v[204:207], v[120:123], a[8:11], v[204:207]\n" \
  "ds_read_b64_tr_b8 v[116:117], v38 offset:8192\n" \
  "v_mfma_i32_16x16x64_i8 v[204:207], v[124:127], a[24:27], v[204:207]\n" \
  "ds_read_b64_tr_b8 v[118:119], v38 offset:9216\n" \
  "v_mfma_i32_16x16x64_i8 v[236:239], v[120:123], a[12:15], v[236:239]\n" \
  "v_mfma_i32_16x16x64_i8 v[236:239], v[124:127], a[28:31], v[236:239]\n" \
  "s_waitcnt lgkmcnt(8)\n" \
  "v_mfma_i32_16x16x64_i8 v[144:147], v[96:99], a[0:3], v[144:147]\n" \
  "ds_read_b64_tr_b8 v[120:121], v39 offset:0\n" \
  "v_mfma_i32_16x16x64_i8 v[144:147], v[100:103], a[16:19], v[144:147]\n" \
  "ds_read_b64_tr_b8 v[122:123], v39 offset:1024\n" \
  "v_mfma_i32_16x16x64_i8 v[176:179], v[96:99], a[4:7], v[176:179]\n" \
  "v_mfma_i32_16x16x64_i8 v[176:179], v[100:103], a[20:23], v[176:179]\n" \
  "v_mfma_i32_16x16x64_i8 v[208:211], v[96:99], a[8:11], v[208:211]\n" \
  "ds_read_b64_tr_b8 v[124:125], v39 offset:8192\n" \
  "v_mfma_i32_16x16x64_i8 v[208:211], v[100:103], a[24:27], v[208:211]\n" \
  "ds_read_b64_tr_b8 v[126:127], v39 offset:9216\n" \
  "v_mfma_i32_16x16x64_i8 v[240:243], v[96:99], a[12:15], v[240:243]\n" \
  "v_mfma_i32_16x16x64_i8 v[240:243], v[100:103], a[28:31], v[240:243]\n" \
  "s_waitcnt lgkmcnt(8)\n" \
  "v_mfma_i32_16x16x64_i8 v[148:151], v[104:107], a[0:3], v[148:151]\n" \
  "ds_read_b64_tr_b8 v[96:97], v32 offset:16384\n" \
  "v_mfma_i32_16x16x64_i8 v[148:151], v[108:111], a[16:19], v[148:151]\n" \
  "ds_read_b64_tr_b8 v[98:99], v32 offset:17408\n" \
  "v_mfma_i32_16x16x64_i8 v[180:183], v[104:107], a[4:7], v[180:183]\n" \
  "v_mfma_i32_16x16x64_i8 v[180:183], v[108:111], a[20:23], v[180:183]\n" \
  "v_mfma_i32_16x16x64_i8 v[212:215], v[104:107], a[8:11], v[212:215]\n" \
  "ds_read_b64_tr_b8 v[100:101], v32 offset:24576\n" \
  "v_mfma_i32_16x16x64_i8 v[212:215], v[108:111], a[24:27], v[212:215]\n" \
  "ds_read_b64_tr_b8 v[102:103], v32 offset:25600\n" \
  "v_mfma_i32_16x16x64_i8 v[244:247], v[104:107], a[12:15], v[244:247]\n" \
  "v_mfma_i32_16x16x64_i8 v[244:247], v[108:111], a[28:31], v[244:247]\n" \
  "s_waitcnt lgkmcnt(8)\n" \
  "v_mfma_i32_16x16x64_i8 v[152:155], v[112:115], a[0:3], v[152:155]\n" \
  "ds_read_b64_tr_b8 v[104:105], v33 offset:16384\n" \
  "v_mfma_i32_16x16x64_i8 v[152:155], v[116:119], a[16:19], v[152:155]\n" \
  "ds_read_b64_tr_b8 v[106:107], v33 offset:17408\n" \
  "v_mfma_i32_16x16x64_i8 v[184:187], v[112:115], a[4:7], v[184:187]\n" \
  "v_mfma_i32_16x16x64_i8 v[184:187], v[116:119], a[20:23], v[184:187]\n" \
  "v_mfma_i32_16x16x64_i8 v[216:219], v[112:115], a[8:11], v[216:219]\n" \
  "ds_read_b64_tr_b8 v[108:109], v33 offset:24576\n" \
  "v_mfma_i32_16x16x64_i8 v[216:219], v[116:119], a[24:27], v[216:219]\n" \
  "ds_read_b64_tr_b8 v[110:111], v33 offset:25600\n" \
  "v_mfma_i32_16x16x64_i8 v[248:251], v[112:115], a[12:15], v[248:251]\n" \
  "v_mfma_i32_16x16x64_i8 v[248:251], v[116:119], a[28:31], v[248:251]\n" \
  "s_waitcnt lgkmcnt(8)\n" \
  "v_mfma_i32_16x16x64_i8 v[156:159], v[120:123], a[0:3], v[156:159]\n" \
  "ds_read_b64_tr_b8 v[112:113], v34 offset:16384\n" \
  "v_mfma_i32_16x16x64_i8 v[156:159], v[124:127], a[16:19], v[156:159]\n" \
  "ds_read_b64_tr_b8 v[114:115], v34 offset:17408\n" \
  "v_mfma_i32_16x16x64_i8 v[188:191], v[120:123], a[4:7], v[188:191]\n" \
  "v_mfma_i32_16x16x64_i8 v[188:191], v[124:127], a[20:23], v[188:191]\n" \
  "v_mfma_i32_16x16x64_i8 v[220:223], v[120:123], a[8:11], v[220:223]\n" \
  "ds_read_b64_tr_b8 v[116:117], v34 offset:24576\n" \
  "v_mfma_i32_16x16x64_i8 v[220:223], v[124:127], a[24:27], v[220:223]\n" \
  "ds_read_b64_tr_b8 v[118:119], v34 offset:25600\n" \
  "v_mfma_i32_16x16x64_i8 v[252:255], v[120:123], a[12:15], v[252:255]\n" \
  "v_mfma_i32_16x16x64_i8 v[252:255], v[124:127], a[28:31], v[252:255]\n" \
  "; ---- step 1\n" \
  "s_waitcnt vmcnt(24)\n" \
  "s_waitcnt lgkmcnt(8)\n" \
  "v_mfma_i32_16x16x64_i8 v[128:131], v[96:99], a[32:35], v[128:131]\n" \
  "ds_read_b64_tr_b8 v[120:121], v35 offset:16384\n" \
  "v_mfma_i32_16x16x64_i8 v[128:131], v[100:103], a[48:51], v[128:131]\n" \
  "ds_read_b64_tr_b8 v[122:123], v35 offset:17408\n" \
  "v_mfma_i32_16x16x64_i8 v[160:163], v[96:99], a[36:39], v[160:163]\n" \
  "v_mfma_i32_16x16x64_i8 v[160:163], v[100:103], a[52:55], v[160:163]\n" \
  "v_mfma_i32_16x16x64_i8 v[192:195], v[96:99], a[40:43], v[192:195]\n" \
  "ds_read_b64_tr_b8 v[124:125], v35 offset:24576\n" \
  "v_mfma_i32_16x16x64_i8 v[192:195], v[100:103], a[56:59], v[192:195]\n" \
  "ds_read_b64_tr_b8 v[126:127], v35 offset:25600\n" \
  "v_mfma_i32_16x16x64_i8 v[224:227], v[96:99], a[44:47], v[224:227]\n" \
  "v_mfma_i32_16x16x64_i8 v[224:227], v[100:103], a[60:63], v[224:227]\n" \
  "s_waitcnt lgkmcnt(8)\n" \
  "v_mfma_i32_16x16x64_i8 v[132:135], v[104:107], a[32:35], v[132:135]\n" \
  "ds_read_b64_tr_b8 v[96:97], v36 offset:16384\n" \
  "v_mfma_i32_16x16x64_i8 v[132:135], v[108:111], a[48:51], v[132:135]\n" \
  "ds_read_b64_tr_b8 v[98:99], v36 offset:17408\n" \
  "v_mfma_i32_16x16x64_i8 v[164:167], v[104:107], a[36:39], v[164:167]\n" \
  "v_mfma_i32_16x16x64_i8 v[164:167], v[108:111], a[52:55], v[164:167]\n" \
  "v_mfma_i32_16x16x64_i8 v[196:199], v[104:107], a[40:43], v[196:199]\n" \
  "ds_read_b64_tr_b8 v[100:101], v36 offset:24576\n" \
  "v_mfma_i32_16x16x64_i8 v[196:199], v[108:111], a[56:59], v[196:199]\n" \
  "ds_read_b64_tr_b8 v[102:103], v36 offset:25600\n" \
  "v_mfma_i32_16x16x64_i8 v[228:231], v[104:107], a[44:47], v[228:231]\n" \
  "v_mfma_i32_16x16x64_i8 v[228:231], v[108:111], a[60:63], v[228:231]\n" \
  "s_waitcnt lgkmcnt(8)\n" \
  "v_mfma_i32_16x16x64_i8 v[136:139], v[112:115], a[32:35], v[136:139]\n" \
  "ds_read_b64_tr_b8 v[104:105], v37 offset:16384\n" \
  "v_mfma_i32_16x16x64_i8 v[136:139], v[116:119], a[48:51], v[136:139]\n" \
  "ds_read_b64_tr_b8 v[106:107], v37 offset:17408\n" \
  "v_mfma_i32_16x16x64_i8 v[168:171], v[112:115], a[36:39], v[168:171]\n" \
  "v_mfma_i32_16x16x64_i8 v[168:171], v[116:119], a[52:55], v[168:171]\n" \
  "v_mfma_i32_16x16x64_i8 v[200:203], v[112:115], a[40:43], v[200:203]\n" \
  "ds_read_b64_tr_b8 v[108:109], v37 offset:24576\n" \
  "v_mfma_i32_16x16x64_i8 v[200:203], v[116:119], a[56:59], v[200:203]\n" \
  "ds_read_b64_tr_b8 v[110:111], v37 offset:25600\n" \
  "buffer_load_dwordx4 a[96:99], v44, s[28:31], s91 offen offset:0 sc0 sc1\n" \
  "v_mfma_i32_16x16x64_i8 v[232:235], v[112:115], a[44:47], v[232:235]\n" \
  "v_mfma_i32_16x16x64_i8 v[232:235], v[116:119], a[60:63], v[232:235]\n" \
  "buffer_load_dwordx4 a[100:103], v44, s[28:31], s91 offen offset:1024 sc0 sc1\n" \
  "s_waitcnt lgkmcnt(8)\n" \
  "v_mfma_i32_16x16x64_i8 v[140:143], v[120:123], a[32:35], v[140:143]\n" \
  "ds_read_b64_tr_b8 v[112:113], v38 offset:16384\n" \
  "v_mfma_i32_16x16x64_i8 v[140:143], v[124:127], a[48:51], v[140:143]\n" \
  "ds_read_b64_tr_b8 v[114:115], v38 offset:17408\n" \
  "v_mfma_i32_16x16x64_i8 v[172:175], v[120:123], a[36:39], v[172:175]\n" \
  "v_mfma_i32_16x16x64_i8 v[172:175], v[124:127], a[52:55], v[172:175]\n" \
  "buffer_load_dwordx4 a[104:107], v44, s[28:31], s91 offen offset:2048 sc0 sc1\n" \
  "s_waitcnt vmcnt(51)\n" \
  "s_barrier\n" \
  "v_mfma_i32_16x16x64_i8 v[204:207], v[120:123], a[40:43], v[204:207]\n" \
  "ds_read_b64_tr_b8 v[116:117], v38 offset:24576\n" \
  "v_mfma_i32_16x16x64_i8 v[204:207], v[124:127], a[56:59], v[204:207]\n" \
  "ds_read_b64_tr_b8 v[118:119], v38 offset:25600\n" \
  "v_mfma_i32_16x16x64_i8 v[236:239], v[120:123], a[44:47], v[236:239]\n" \
  "v_mfma_i32_16x16x64_i8 v[236:239], v[124:127], a[60:63], v[236:239]\n" \
  "buffer_load_dwordx4 a[108:111], v44, s[28:31], s91 offen offset:3072 sc0 sc1\n" \
  "s_waitcnt lgkmcnt(8)\n" \
  "v_mfma_i32_16x16x64_i8 v[144:147], v[96:99], a[32:35], v[144:147]\n" \
  "ds_read_b64_tr_b8 v[120:121], v39 offset:16384\n" \
  "v_mfma_i32_16x16x64_i8 v[144:147], v[100:103], a[48:51], v[144:147]\n" \
  "ds_read_b64_tr_b8 v[122:123], v39 offset:17408\n" \
  "v_mfma_i32_16x16x64_i8 v[176:179], v[96:99], a[36:39], v[176:179]\n" \
  "v_mfma_i32_16x16x64_i8 v[176:179], v[100:103], a[52:55], v[176:179]\n" \
  "buffer_load_dwordx4 a[112:115], v91, s[28:31], s91 offen offset:0 sc0 sc1\n" \
  "v_mfma_i32_16x16x64_i8 v[208:211], v[96:99], a[40:43], v[208:211]\n" \
  "ds_read_b64_tr_b8 v[124:125], v39 offset:24576\n" \
  "v_mfma_i32_16x16x64_i8 v[208:211], v[100:103], a[56:59], v[208:211]\n" \
  "ds_read_b64_tr_b8 v[126:127], v39 offset:25600\n" \
  "v_mfma_i32_16x16x64_i8 v[240:243], v[96:99], a[44:47], v[240:243]\n" \
  "v_mfma_i32_16x16x64_i8 v[240:243], v[100:103], a[60:63], v[240:243]\n" \
  "buffer_load_dwordx4 a[116:119], v91, s[28:31], s91 offen offset:1024 sc0 sc1\n" \
  "s_waitcnt lgkmcnt(8)\n" \
  "v_mfma_i32_16x16x64_i8 v[148:151], v[104:107], a[32:35], v[148:151]\n" \
  "ds_read_b64_tr_b8 v[96:97], v32 offset:32768\n" \
  "v_mfma_i32_16x16x64_i8 v[148:151], v[108:111], a[48:51], v[148:151]\n" \
  "ds_read_b64_tr_b8 v[98:99], v32 offset:33792\n" \
  "v_mfma_i32_16x16x64_i8 v[180:183], v[104:107], a[36:39], v[180:183]\n" \
  "v_mfma_i32_16x16x64_i8 v[180:183], v[108:111], a[52:55], v[180:183]\n" \
  "buffer_load_dwordx4 a[120:123], v91, s[28:31], s91 offen offset:2048 sc0 sc1\n" \
  "v_mfma_i32_16x16x64_i8 v[212:215], v[104:107], a[40:43], v[212:215]\n" \
  "ds_read_b64_tr_b8 v[100:101], v32 offset:40960\n" \
  "v_mfma_i32_16x16x64_i8 v[212:215], v[108:111], a[56:59], v[212:215]\n" \
  "ds_read_b64_tr_b8 v[102:103], v32 offset:41984\n" \
  "buffer_load_dwordx4 a[124:127], v91, s[28:31], s91 offen offset:3072 sc0 sc1\n" \
  "v_mfma_i32_16x16x64_i8 v[244:247], v[104:107], a[44:47], v[244:247]\n" \
  "v_mfma_i32_16x16x64_i8 v[244:247], v[108:111], a[60:63], v[244:247]\n" \
  "s_waitcnt lgkmcnt(8)\n" \
  "v_mfma_i32_16x16x64_i8 v[152:155], v[112:115], a[32:35], v[152:155]\n" \
  "ds_read_b64_tr_b8 v[104:105], v33 offset:32768\n" \
  "v_mfma_i32_16x16x64_i8 v[152:155], v[116:119], a[48:51], v[152:155]\n" \
  "ds_read_b64_tr_b8 v[106:107], v33 offset:33792\n" \
  "buffer_load_dwordx4 a[128:131], v44, s[28:31], s92 offen offset:0 sc0 sc1\n" \
  "v_mfma_i32_16x16x64_i8 v[184:187], v[112:115], a[36:39], v[184:187]\n" \
  "v_mfma_i32_16x16x64_i8 v[184:187], v[116:119], a[52:55], v[184:187]\n" \
  "v_mfma_i32_16x16x64_i8 v[216:219], v[112:115], a[40:43], v[216:219]\n" \
  "ds_read_b64_tr_b8 v[108:109], v33 offset:40960\n" \
  "v_mfma_i32_16x16x64_i8 v[216:219], v[116:119], a[56:59], v[216:219]\n" \
  "ds_read_b64_tr_b8 v[110:111], v33 offset:41984\n" \
  "buffer_load_dwordx4 a[132:135], v44, s[28:31], s92 offen offset:1024 sc0 sc1\n" \
  "v_mfma_i32_16x16x64_i8 v[248:251], v[112:115], a[44:47], v[248:251]\n" \
  "v_mfma_i32_16x16x64_i8 v[248:251], v[116:119], a[60:63], v[248:251]\n" \
  "s_waitcnt lgkmcnt(8)\n" \
  "v_mfma_i32_16x16x64_i8 v[156:159], v[120:123], a[32:35], v[156:159]\n" \
  "ds_read_b64_tr_b8 v[112:113], v34 offset:32768\n" \
  "v_mfma_i32_16x16x64_i8 v[156:159], v[124:127], a[48:51], v[156:159]\n" \
  "ds_read_b64_tr_b8 v[114:115], v34 offset:33792\n" \
  "buffer_load_dwordx4 a[136:139], v44, s[28:31], s92 offen offset:2048 sc0 sc1\n" \
  "v_mfma_i32_16x16x64_i8 v[188:191], v[120:123], a[36:39], v[188:191]\n" \
  "v_mfma_i32_16x16x64_i8 v[188:191], v[124:127], a[52:55], v[188:191]\n" \
  "v_mfma_i32_16x16x64_i8 v[220:223], v[120:123], a[40:43], v[220:223]\n" \
  "ds_read_b64_tr_b8 v[116:117], v34 offset:40960\n" \
  "v_mfma_i32_16x16x64_i8 v[220:223], v[124:127], a[56:59], v[220:223]\n" \
  "ds_read_b64_tr_b8 v[118:119], v34 offset:41984\n" \
  "buffer_load_dwordx4 a[140:143], v44, s[28:31], s92 offen offset:3072 sc0 sc1\n" \
  "v_mfma_i32_16x16x64_i8 v[252:255], v[120:123], a[44:47], v[252:255]\n" \
  "v_mfma_i32_16x16x64_i8 v[252:255], v[124:127], a[60:63], v[252:255]\n" \
  "; ---- step 2\n" \
  "s_cmp_lg_u32 s11, 0\n" \
  "s_cbranch_scc1 Lskip32A_%=\n" \
  "global_load_dword v72, v50, s[38:39] sc0 sc1\n" \
  "Lskip32A_%=:\n" \
  "s_cmp_lg_u32 s46, s54\n" \
  "s_cbranch_scc1 Lskip31A_%=\n" \
  "s_waitcnt vmcnt(63)\n" \
  "global_store_dwordx4 v49, v[64:67], s[34:35] sc0 sc1\n" \
  "s_nop 1\n" \
  "global_store_dwordx4 v79, v[68:71], s[34:35] sc0 sc1\n" \
  "s_nop 1\n" \
  "Lskip31A_%=:\n" \
  "s_waitcnt vmcnt(28)\n" \
  "s_waitcnt lgkmcnt(8)\n" \
  "v_mfma_i32_16x16x64_i8 v[128:131], v[96:99], a[64:67], v[128:131]\n" \
  "ds_read_b64_tr_b8 v[120:121], v35 offset:32768\n" \
  "v_mfma_i32_16x16x64_i8 v[128:131], v[100:103], a[80:83], v[128:131]\n" \
  "ds_read_b64_tr_b8 v[122:123], v35 offset:33792\n" \
  "v_mfma_i32_16x16x64_i8 v[160:163], v[96:99], a[68:71], v[160:163]\n" \
  "v_mfma_i32_16x16x64_i8 v[160:163], v[100:103], a[84:87], v[160:163]\n" \
  "buffer_load_dwordx4 a[144:147], v91, s[28:31], s92 offen offset:0 sc0 sc1\n" \
  "v_mfma_i32_16x16x64_i8 v[192:195], v[96:99], a[72:75], v[192:195]\n" \
  "ds_read_b64_tr_b8 v[124:125], v35 offset:40960\n" \
  "v_mfma_i32_16x16x64_i8 v[192:195], v[100:103], a[88:91], v[192:195]\n" \
  "ds_read_b64_tr_b8 v[126:127], v35 offset:41984\n" \
  "v_mfma_i32_16x16x64_i8 v[224:227], v[96:99], a[76:79], v[224:227]\n" \
  "v_mfma_i32_16x16x64_i8 v[224:227], v[100:103], a[92:95], v[224:227]\n" \
  "buffer_load_dwordx4 a[148:151], v91, s[28:31], s92 offen offset:1024 sc0 sc1\n" \
  "s_waitcnt lgkmcnt(8)\n" \
  "v_mfma_i32_16x16x64_i8 v[132:135], v[104:107], a[64:67], v[132:135]\n" \
  "ds_read_b64_tr_b8 v[96:97], v36 offset:32768\n" \
  "v_mfma_i32_16x16x64_i8 v[132:135], v[108:111], a[80:83], v[132:135]\n" \
  "ds_read_b64_tr_b8 v[98:99], v36 offset:33792\n" \
  "v_mfma_i32_16x16x64_i8 v[164:167], v[104:107], a[68:71], v[164:167]\n" \
  "v_mfma_i32_16x16x64_i8 v[164:167], v[108:111], a[84:87], v[164:167]\n" \
  "buffer_load_dwordx4 a[152:155], v91, s[28:31], s92 offen offset:2048 sc0 sc1\n" \
  "v_mfma_i32_16x16x64_i8 v[196:199], v[104:107], a[72:75], v[196:199]\n" \
  "ds_read_b64_tr_b8 v[100:101], v36 offset:40960\n" \
  "v_mfma_i32_16x16x64_i8 v[196:199], v[108:111], a[88:91], v[196:199]\n" \
  "ds_read_b64_tr_b8 v[102:103], v36 offset:41984\n" \
  "v_mfma_i32_16x16x64_i8 v[228:231], v[104:107], a[76:79], v[228:231]\n" \
  "v_mfma_i32_16x16x64_i8 v[228:231], v[108:111], a[92:95], v[228:231]\n" \
  "s_waitcnt lgkmcnt(8)\n" \
  "v_mfma_i32_16x16x64_i8 v[136:139], v[112:115], a[64:67], v[136:139]\n" \
  "ds_read_b64_tr_b8 v[104:105], v37 offset:32768\n" \
  "v_mfma_i32_16x16x64_i8 v[136:139], v[116:119], a[80:83], v[136:139]\n" \
  "ds_read_b64_tr_b8 v[106:107], v37 offset:33792\n" \
  "v_mfma_i32_16x16x64_i8 v[168:171], v[112:115], a[68:71], v[168:171]\n" \
  "v_mfma_i32_16x16x64_i8 v[168:171], v[116:119], a[84:87], v[168:171]\n" \
  "buffer_load_dwordx4 a[156:159], v91, s[28:31], s92 offen offset:3072 sc0 sc1\n" \
  "v_mfma_i32_16x16x64_i8 v[200:203], v[112:115], a[72:75], v[200:203]\n" \
  "ds_read_b64_tr_b8 v[108:109], v37 offset:40960\n" \
  "v_mfma_i32_16x16x64_i8 v[200:203], v[116:119], a[88:91], v[200:203]\n" \
  "ds_read_b64_tr_b8 v[110:111], v37 offset:41984\n" \
  "v_mfma_i32_16x16x64_i8 v[232:235], v[112:115], a[76:79], v[232:235]\n" \
  "v_mfma_i32_16x16x64_i8 v[232:235], v[116:119], a[92:95], v[232:235]\n" \
  "buffer_load_dwordx4 a[160:163], v44, s[28:31], s93 offen offset:0 sc0 sc1\n" \
  "s_waitcnt lgkmcnt(8)\n" \
  "v_mfma_i32_16x16x64_i8 v[140:143], v[120:123], a[64:67], v[140:143]\n" \
  "ds_read_b64_tr_b8 v[112:113], v38 offset:32768\n" \
  "v_mfma_i32_16x16x64_i8 v[140:143], v[124:127], a[80:83], v[140:143]\n" \
  "ds_read_b64_tr_b8 v[114:115], v38 offset:33792\n" \
  "v_mfma_i32_16x16x64_i8 v[172:175], v[120:123], a[68:71], v[172:175]\n" \
  "v_mfma_i32_16x16x64_i8 v[172:175], v[124:127], a[84:87], v[172:175]\n" \
  "buffer_load_dwordx4 a[164:167], v44, s[28:31], s93 offen offset:1024 sc0 sc1\n" \
  "s_waitcnt vmcnt(30)\n" \
  "s_barrier\n" \
  "v_mfma_i32_16x16x64_i8 v[204:207], v[120:123], a[72:75], v[204:207]\n" \
  "ds_read_b64_tr_b8 v[116:117], v38 offset:40960\n" \
  "v_mfma_i32_16x16x64_i8 v[204:207], v[124:127], a[88:91], v[204:207]\n" \
  "ds_read_b64_tr_b8 v[118:119], v38 offset:41984\n" \
  "v_mfma_i32_16x16x64_i8 v[236:239], v[120:123], a[76:79], v[236:239]\n" \
  "v_mfma_i32_16x16x64_i8 v[236:239], v[124:127], a[92:95], v[236:239]\n" \
  "s_waitcnt lgkmcnt(8)\n" \
  "v_mfma_i32_16x16x64_i8 v[144:147], v[96:99], a[64:67], v[144:147]\n" \
  "ds_read_b64_tr_b8 v[120:121], v39 offset:32768\n" \
  "v_mfma_i32_16x16x64_i8 v[144:147], v[100:103], a[80:83], v[144:147]\n" \
  "ds_read_b64_tr_b8 v[122:123], v39 offset:33792\n" \
  "v_mfma_i32_16x16x64_i8 v[176:179], v[96:99], a[68:71], v[176:179]\n" \
  "v_mfma_i32_16x16x64_i8 v[176:179], v[100:103], a[84:87], v[176:179]\n" \
  "buffer_load_dwordx4 a[168:171], v44, s[28:31], s93 offen offset:2048 sc0 sc1\n" \
  "v_mfma_i32_16x16x64_i8 v[208:211], v[96:99], a[72:75], v[208:211]\n" \
  "ds_read_b64_tr_b8 v[124:125], v39 offset:40960\n" \
  "v_mfma_i32_16x16x64_i8 v[208:211], v[100:103], a[88:91], v[208:211]\n" \
  "ds_read_b64_tr_b8 v[126:127], v39 offset:41984\n" \
  "v_mfma_i32_16x16x64_i8 v[240:243], v[96:99], a[76:79], v[240:243]\n" \
  "v_mfma_i32_16x16x64_i8 v[240:243], v[100:103], a[92:95], v[240:243]\n" \
  "buffer_load_dwordx4 a[172:175], v44, s[28:31], s93 offen offset:3072 sc0 sc1\n" \
  "s_waitcnt lgkmcnt(8)\n" \
  "v_mfma_i32_16x16x64_i8 v[148:151], v[104:107], a[64:67], v[148:151]\n" \
  "ds_read_b64_tr_b8 v[96:97], v32 offset:49152\n" \
  "v_mfma_i32_16x16x64_i8 v[148:151], v[108:111], a[80:83], v[148:151]\n" \
  "ds_read_b64_tr_b8 v[98:99], v32 offset:50176\n" \
  "v_mfma_i32_16x16x64_i8 v[180:183], v[104:107], a[68:71], v[180:183]\n" \
  "v_mfma_i32_16x16x64_i8 v[180:183], v[108:111], a[84:87], v[180:183]\n" \
  "buffer_load_dwordx4 a[176:179], v91, s[28:31], s93 offen offset:0 sc0 sc1\n" \
  "v_mfma_i32_16x16x64_i8 v[212:215], v[104:107], a[72:75], v[212:215]\n" \
  "ds_read_b64_tr_b8 v[100:101], v32 offset:57344\n" \
  "v_mfma_i32_16x16x64_i8 v[212:215], v[108:111], a[88:91], v[212:215]\n" \
  "ds_read_b64_tr_b8 v[102:103], v32 offset:58368\n" \
  "v_mfma_i32_16x16x64_i8 v[244:247], v[104:107], a[76:79], v[244:247]\n" \
  "v_mfma_i32_16x16x64_i8 v[244:247], v[108:111], a[92:95], v[244:247]\n" \
  "s_waitcnt lgkmcnt(8)\n" \
  "v_mfma_i32_16x16x64_i8 v[152:155], v[112:115], a[64:67], v[152:155]\n" \
  "ds_read_b64_tr_b8 v[104:105], v33 offset:49152\n" \
  "v_mfma_i32_16x16x64_i8 v[152:155], v[116:119], a[80:83], v[152:155]\n" \
  "ds_read_b64_tr_b8 v[106:107], v33 offset:50176\n" \
  "v_mfma_i32_16x16x64_i8 v[184:187], v[112:115], a[68:71], v[184:187]\n" \
  "v_mfma_i32_16x16x64_i8 v[184:187], v[116:119], a[84:87], v[184:187]\n" \
  "buffer_load_dwordx4 a[180:183], v91, s[28:31], s93 offen offset:1024 sc0 sc1\n" \
  "v_mfma_i32_16x16x64_i8 v[216:219], v[112:115], a[72:75], v[216:219]\n" \
  "ds_read_b64_tr_b8 v[108:109], v33 offset:57344\n" \
  "v_mfma_i32_16x16x64_i8 v[216:219], v[116:119], a[88:91], v[216:219]\n" \
  "ds_read_b64_tr_b8 v[110:111], v33 offset:58368\n" \
  "v_mfma_i32_16x16x64_i8 v[248:251], v[112:115], a[76:79], v[248:251]\n" \
  "v_mfma_i32_16x16x64_i8 v[248:251], v[116:119], a[92:95], v[248:251]\n" \
  "buffer_load_dwordx4 a[184:187], v91, s[28:31], s93 offen offset:2048 sc0 sc1\n" \
  "s_waitcnt lgkmcnt(8)\n" \
  "v_mfma_i32_16x16x64_i8 v[156:159], v[120:123], a[64:67], v[156:159]\n" \
  "ds_read_b64_tr_b8 v[112:113], v34 offset:49152\n" \
  "v_mfma_i32_16x16x64_i8 v[156:159], v[124:127], a[80:83], v[156:159]\n" \
  "ds_read_b64_tr_b8 v[114:115], v34 offset:50176\n" \
  "v_mfma_i32_16x16x64_i8 v[188:191], v[120:123], a[68:71], v[188:191]\n" \
  "v_mfma_i32_16x16x64_i8 v[188:191], v[124:127], a[84:87], v[188:191]\n" \
  "buffer_load_dwordx4 a[188:191], v91, s[28:31], s93 offen offset:3072 sc0 sc1\n" \
  "v_mfma_i32_16x16x64_i8 v[220:223], v[120:123], a[72:75], v[220:223]\n" \
  "ds_read_b64_tr_b8 v[116:117], v34 offset:57344\n" \
  "v_mfma_i32_16x16x64_i8 v[220:223], v[124:127], a[88:91], v[220:223]\n" \
  "ds_read_b64_tr_b8 v[118:119], v34 offset:58368\n" \
  "v_mfma_i32_16x16x64_i8 v[252:255], v[120:123], a[76:79], v[252:255]\n" \
  "v_mfma_i32_16x16x64_i8 v[252:255], v[124:127], a[92:95], v[252:255]\n" \
  "; ---- step 3\n" \
  "s_waitcnt vmcnt(16)\n" \
  "s_waitcnt lgkmcnt(8)\n" \
  "v_mfma_i32_16x16x64_i8 v[128:131], v[96:99], a[96:99], v[128:131]\n" \
  "ds_read_b64_tr_b8 v[120:121], v35 offset:49152\n" \
  "v_mfma_i32_16x16x64_i8 v[128:131], v[100:103], a[112:115], v[128:131]\n" \
  "ds_read_b64_tr_b8 v[122:123], v35 offset:50176\n" \
  "v_mfma_i32_16x16x64_i8 v[160:163], v[96:99], a[100:103], v[160:163]\n" \
  "v_mfma_i32_16x16x64_i8 v[160:163], v[100:103], a[116:119], v[160:163]\n" \
  "buffer_load_dwordx4 a[192:195], v44, s[28:31], s94 offen offset:0 sc0 sc1\n" \
  "v_mfma_i32_16x16x64_i8 v[192:195], v[96:99], a[104:107], v[192:195]\n" \
  "ds_read_b64_tr_b8 v[124:125], v35 offset:57344\n" \
  "v_mfma_i32_16x16x64_i8 v[192:195], v[100:103], a[120:123], v[192:195]\n" \
  "ds_read_b64_tr_b8 v[126:127], v35 offset:58368\n" \
  "v_mfma_i32_16x16x64_i8 v[224:227], v[96:99], a[108:111], v[224:227]\n" \
  "v_mfma_i32_16x16x64_i8 v[224:227], v[100:103], a[124:127], v[224:227]\n" \
  "buffer_load_dwordx4 a[196:199], v44, s[28:31], s94 offen offset:1024 sc0 sc1\n" \
  "s_waitcnt lgkmcnt(8)\n" \
  "v_mfma_i32_16x16x64_i8 v[132:135], v[104:107], a[96:99], v[132:135]\n" \
  "ds_read_b64_tr_b8 v[96:97], v36 offset:49152\n" \
  "v_mfma_i32_16x16x64_i8 v[132:135], v[108:111], a[112:115], v[132:135]\n" \
  "ds_read_b64_tr_b8 v[98:99], v36 offset:50176\n" \
  "v_mfma_i32_16x16x64_i8 v[164:167], v[104:107], a[100:103], v[164:167]\n" \
  "v_mfma_i32_16x16x64_i8 v[164:167], v[108:111], a[116:119], v[164:167]\n" \
  "buffer_load_dwordx4 a[200:203], v44, s[28:31], s94 offen offset:2048 sc0 sc1\n" \
  "v_mfma_i32_16x16x64_i8 v[196:199], v[104:107], a[104:107], v[196:199]\n" \
  "ds_read_b64_tr_b8 v[100:101], v36 offset:57344\n" \
  "v_mfma_i32_16x16x64_i8 v[196:199], v[108:111], a[120:123], v[196:199]\n" \
  "ds_read_b64_tr_b8 v[102:103], v36 offset:58368\n" \
  "v_mfma_i32_16x16x64_i8 v[228:231], v[104:107], a[108:111], v[228:231]\n" \
  "v_readlane_b32 s50, v46, s49\n" \
  "v_mfma_i32_16x16x64_i8 v[228:231], v[108:111], a[124:127], v[228:231]\n" \
  "buffer_load_dwordx4 a[204:207], v44, s[28:31], s94 offen offset:3072 sc0 sc1\n" \
  "s_waitcnt lgkmcnt(8)\n" \
  "v_mfma_i32_16x16x64_i8 v[136:139], v[112:115], a[96:99], v[136:139]\n" \
  "ds_read_b64_tr_b8 v[104:105], v37 offset:49152\n" \
  "v_mfma_i32_16x16x64_i8 v[136:139], v[116:119], a[112:115], v[136:139]\n" \
  "ds_read_b64_tr_b8 v[106:107], v37 offset:50176\n" \
  "v_mfma_i32_16x16x64_i8 v[168:171], v[112:115], a[100:103], v[168:171]\n" \
  "v_mfma_i32_16x16x64_i8 v[168:171], v[116:119], a[116:119], v[168:171]\n" \
  "buffer_load_dwordx4 a[208:211], v91, s[28:31], s94 offen offset:0 sc0 sc1\n" \
  "v_mfma_i32_16x16x64_i8 v[200:203], v[112:115], a[104:107], v[200:203]\n" \
  "ds_read_b64_tr_b8 v[108:109], v37 offset:57344\n" \
  "v_mfma_i32_16x16x64_i8 v[200:203], v[116:119], a[120:123], v[200:203]\n" \
  "ds_read_b64_tr_b8 v[110:111], v37 offset:58368\n" \
  "v_mfma_i32_16x16x64_i8 v[232:235], v[112:115], a[108:111], v[232:235]\n" \
  "s_lshl_b32 s87, s50, 21\n" \
  "s_lshl_b32 s95, s50, 16\n" \
  "s_add_u32 s49, s49, 1\n" \
  "v_mfma_i32_16x16x64_i8 v[232:235], v[116:119], a[124:127], v[232:235]\n" \
  "buffer_load_dwordx4 a[212:215], v91, s[28:31], s94 offen offset:1024 sc0 sc1\n" \
  "s_waitcnt lgkmcnt(8)\n" \
  "v_mfma_i32_16x16x64_i8 v[140:143], v[120:123], a[96:99], v[140:143]\n" \
  "ds_read_b64_tr_b8 v[112:113], v38 offset:49152\n" \
  "v_mfma_i32_16x16x64_i8 v[140:143], v[124:127], a[112:115], v[140:143]\n" \
  "ds_read_b64_tr_b8 v[114:115], v38 offset:50176\n" \
  "v_mfma_i32_16x16x64_i8 v[172:175], v[120:123], a[100:103], v[172:175]\n" \
  "v_readlane_b32 s50, v46, s49\n" \
  "v_mfma_i32_16x16x64_i8 v[172:175], v[124:127], a[116:119], v[172:175]\n" \
  "buffer_load_dwordx4 a[216:219], v91, s[28:31], s94 offen offset:2048 sc0 sc1\n" \
  "s_cmp_lg_u32 s46, s54\n" \
  "s_cbranch_scc1 Lskip33A_%=\n" \
  "s_waitcnt vmcnt(19)\n" \
  "Lskip33A_%=:\n" \
  "s_waitcnt vmcnt(39)\n" \
  "s_barrier\n" \
  "s_cmp_lg_u32 s46, s54\n" \
  "s_cbranch_scc1 Lskip35A_%=\n" \
  "s_cmp_lg_u32 s11, 0\n" \
  "s_cbranch_scc1 Lskip34A_%=\n" \
  "s_mov_b64 exec, 1\n" \
  "global_store_dword v53, v76, s[36:37] sc0 sc1\n" \
  "s_mov_b64 exec, -1\n" \
  "Lskip34A_%=:\n" \
  "Lskip35A_%=:\n" \
  "v_mfma_i32_16x16x64_i8 v[204:207], v[120:123], a[104:107], v[204:207]\n" \
  "ds_read_b64_tr_b8 v[116:117], v38 offset:57344\n" \
  "v_mfma_i32_16x16x64_i8 v[204:207], v[124:127], a[120:123], v[204:207]\n" \
  "ds_read_b64_tr_b8 v[118:119], v38 offset:58368\n" \
  "v_mfma_i32_16x16x64_i8 v[236:239], v[120:123], a[108:111], v[236:239]\n" \
  "v_mfma_i32_16x16x64_i8 v[236:239], v[124:127], a[124:127], v[236:239]\n" \
  "buffer_load_dwordx4 a[220:223], v91, s[28:31], s94 offen offset:3072 sc0 sc1\n" \
  "s_waitcnt lgkmcnt(8)\n" \
  "v_mfma_i32_16x16x64_i8 v[144:147], v[96:99], a[96:99], v[144:147]\n" \
  "ds_read_b64_tr_b8 v[120:121], v39 offset:49152\n" \
  "v_mfma_i32_16x16x64_i8 v[144:147], v[100:103], a[112:115], v[144:147]\n" \
  "ds_read_b64_tr_b8 v[122:123], v39 offset:50176\n" \
  "s_add_u32 m0, s47, 114688\n" \
  "v_mfma_i32_16x16x64_i8 v[176:179], v[96:99], a[100:103], v[176:179]\n" \
  "buffer_load_dwordx4 v40, s[16:19], s87 offen nt lds\n" \
  "s_lshl_b32 s80, s50, 21\n" \
  "s_lshl_b32 s88, s50, 16\n" \
  "s_add_u32 s49, s49, 1\n" \
  "v_mfma_i32_16x16x64_i8 v[176:179], v[100:103], a[116:119], v[176:179]\n" \
  "v_mfma_i32_16x16x64_i8 v[208:211], v[96:99], a[104:107], v[208:211]\n" \
  "ds_read_b64_tr_b8 v[124:125], v39 offset:57344\n" \
  "v_mfma_i32_16x16x64_i8 v[208:211], v[100:103], a[120:123], v[208:211]\n" \
  "ds_read_b64_tr_b8 v[126:127], v39 offset:58368\n" \
  "s_add_u32 m0, s47, 115712\n" \
  "v_mfma_i32_16x16x64_i8 v[240:243], v[96:99], a[108:111], v[240:243]\n" \
  "buffer_load_dwordx4 v41, s[16:19], s87 offen nt lds\n" \
  "v_readlane_b32 s50, v46, s49\n" \
  "v_mfma_i32_16x16x64_i8 v[240:243], v[100:103], a[124:127], v[240:243]\n" \
  "s_waitcnt lgkmcnt(8)\n" \
  "v_mfma_i32_16x16x64_i8 v[148:151], v[104:107], a[96:99], v[148:151]\n" \
  "ds_read_b64_tr_b8 v[96:97], v80 offset:0\n" \
  "v_mfma_i32_16x16x64_i8 v[148:151], v[108:111], a[112:115], v[148:151]\n" \
  "ds_read_b64_tr_b8 v[98:99], v80 offset:1024\n" \
  "s_add_u32 m0, s47, 116736\n" \
  "v_mfma_i32_16x16x64_i8 v[180:183], v[104:107], a[100:103], v[180:183]\n" \
  "buffer_load_dwordx4 v42, s[16:19], s87 offen nt lds\n" \
  "v_mfma_i32_16x16x64_i8 v[180:183], v[108:111], a[116:119], v[180:183]\n" \
  "v_mfma_i32_16x16x64_i8 v[212:215], v[104:107], a[104:107], v[212:215]\n" \
  "ds_read_b64_tr_b8 v[100:101], v80 offset:8192\n" \
  "v_mfma_i32_16x16x64_i8 v[212:215], v[108:111], a[120:123], v[212:215]\n" \
  "ds_read_b64_tr_b8 v[102:103], v80 offset:9216\n" \
  "s_add_u32 m0, s47, 117760\n" \
  "v_mfma_i32_16x16x64_i8 v[244:247], v[104:107], a[108:111], v[244:247]\n" \
  "buffer_load_dwordx4 v43, s[16:19], s87 offen nt lds\n" \
  "s_lshl_b32 s81, s50, 21\n" \
  "s_lshl_b32 s89, s50, 16\n" \
  "s_add_u32 s49, s49, 1\n" \
  "v_mfma_i32_16x16x64_i8 v[244:247], v[108:111], a[124:127], v[244:247]\n" \
  "s_waitcnt lgkmcnt(8)\n" \
  "v_mfma_i32_16x16x64_i8 v[152:155], v[112:115], a[96:99], v[152:155]\n" \
  "ds_read_b64_tr_b8 v[104:105], v81 offset:0\n" \
  "v_mfma_i32_16x16x64_i8 v[152:155], v[116:119], a[112:115], v[152:155]\n" \
  "ds_read_b64_tr_b8 v[106:107], v81 offset:1024\n" \
  "s_add_u32 m0, s47, 0\n" \
  "v_mfma_i32_16x16x64_i8 v[184:187], v[112:115], a[100:103], v[184:187]\n" \
  "buffer_load_dwordx4 v40, s[16:19], s80 offen nt lds\n" \
  "v_readlane_b32 s50, v46, s49\n" \
  "v_mfma_i32_16x16x64_i8 v[184:187], v[116:119], a[116:119], v[184:187]\n" \
  "s_cmp_lg_u32 s11, 0\n" \
  "s_cbranch_scc1 Lskip38A_%=\n" \
  "s_waitcnt vmcnt(25)\n" \
  "v_sub_u32 v72, v72, v76\n" \
  "v_cmp_gt_i32 vcc, 0, v72\n" \
  "s_nop 1\n" \
  "s_andn2_b64 vcc, vcc, s[14:15]\n" \
  "s_cbranch_scc1 Lslow36A_%=\n" \
  "Lback37A_%=:\n" \
  "v_add_u32 v50, 1024, v50\n" \
  "Lskip38A_%=:\n" \
  "v_mfma_i32_16x16x64_i8 v[216:219], v[112:115], a[104:107], v[216:219]\n" \
  "ds_read_b64_tr_b8 v[108:109], v81 offset:8192\n" \
  "v_mfma_i32_16x16x64_i8 v[216:219], v[116:119], a[120:123], v[216:219]\n" \
  "ds_read_b64_tr_b8 v[110:111], v81 offset:9216\n" \
  "s_add_u32 m0, s47, 1024\n" \
  "v_mfma_i32_16x16x64_i8 v[248:251], v[112:115], a[108:111], v[248:251]\n" \
  "buffer_load_dwordx4 v41, s[16:19], s80 offen nt lds\n" \
  "v_mfma_i32_16x16x64_i8 v[248:251], v[116:119], a[124:127], v[248:251]\n" \
  "s_waitcnt lgkmcnt(8)\n" \
  "v_mfma_i32_16x16x64_i8 v[156:159], v[120:123], a[96:99], v[156:159]\n" \
  "ds_read_b64_tr_b8 v[112:113], v82 offset:0\n" \
  "v_mfma_i32_16x16x64_i8 v[156:159], v[124:127], a[112:115], v[156:159]\n" \
  "ds_read_b64_tr_b8 v[114:115], v82 offset:1024\n" \
  "s_add_u32 m0, s47, 2048\n" \
  "v_mfma_i32_16x16x64_i8 v[188:191], v[120:123], a[100:103], v[188:191]\n" \
  "buffer_load_dwordx4 v42, s[16:19], s80 offen nt lds\n" \
  "s_lshl_b32 s82, s50, 21\n" \
  "s_lshl_b32 s90, s50, 16\n" \
  "s_add_u32 s49, s49, 1\n" \
  "v_mfma_i32_16x16x64_i8 v[188:191], v[124:127], a[116:119], v[188:191]\n" \
  "v_mfma_i32_16x16x64_i8 v[220:223], v[120:123], a[104:107], v[220:223]\n" \
  "ds_read_b64_tr_b8 v[116:117], v82 offset:8192\n" \
  "v_mfma_i32_16x16x64_i8 v[220:223], v[124:127], a[120:123], v[220:223]\n" \
  "ds_read_b64_tr_b8 v[118:119], v82 offset:9216\n" \
  "s_add_u32 m0, s47, 3072\n" \
  "v_mfma_i32_16x16x64_i8 v[252:255], v[120:123], a[108:111], v[252:255]\n" \
  "buffer_load_dwordx4 v43, s[16:19], s80 offen nt lds\n" \
  "v_mfma_i32_16x16x64_i8 v[252:255], v[124:127], a[124:127], v[252:255]\n" \
  "s_barrier\n" \
  "buffer_load_dword v92, v52, s[28:31], 0 offen sc0 sc1\n" \
  "v_add_u32 v52, 0x10000, v52\n" \
  "; ---- step 4\n" \
  "s_waitcnt vmcnt(25)\n" \
  "s_waitcnt lgkmcnt(8)\n" \
  "v_mfma_i32_16x16x64_i8 v[128:131], v[96:99], a[128:131], v[128:131]\n" \
  "ds_read_b64_tr_b8 v[120:121], v83 offset:0\n" \
  "s_add_u32 m0, s47, 16384\n" \
  "v_mfma_i32_16x16x64_i8 v[128:131], v[100:103], a[144:147], v[128:131]\n" \
  "ds_read_b64_tr_b8 v[122:123], v83 offset:1024\n" \
  "buffer_load_dwordx4 v40, s[16:19], s81 offen lds\n" \
  "s_add_u32 m0, s47, 17408\n" \
  "v_mfma_i32_16x16x64_i8 v[160:163], v[96:99], a[132:135], v[160:163]\n" \
  "buffer_load_dwordx4 v41, s[16:19], s81 offen lds\n" \
  "s_add_u32 m0, s47, 18432\n" \
  "v_mfma_i32_16x16x64_i8 v[160:163], v[100:103], a[148:151], v[160:163]\n" \
  "buffer_load_dwordx4 v42, s[16:19], s81 offen lds\n" \
  "s_add_u32 m0, s47, 19456\n" \
  "v_mfma_i32_16x16x64_i8 v[192:195], v[96:99], a[136:139], v[192:195]\n" \
  "ds_read_b64_tr_b8 v[124:125], v83 offset:8192\n" \
  "buffer_load_dwordx4 v43, s[16:19], s81 offen lds\n" \
  "s_add_u32 m0, s47, 32768\n" \
  "v_mfma_i32_16x16x64_i8 v[192:195], v[100:103], a[152:155], v[192:195]\n" \
  "ds_read_b64_tr_b8 v[126:127], v83 offset:9216\n" \
  "buffer_load_dwordx4 v40, s[16:19], s82 offen lds\n" \
  "s_add_u32 m0, s47, 33792\n" \
  "v_mfma_i32_16x16x64_i8 v[224:227], v[96:99], a[140:143], v[224:227]\n" \
  "buffer_load_dwordx4 v41, s[16:19], s82 offen lds\n" \
  "s_add_u32 m0, s47, 34816\n" \
  "v_mfma_i32_16x16x64_i8 v[224:227], v[100:103], a[156:159], v[224:227]\n" \
  "buffer_load_dwordx4 v42, s[16:19], s82 offen lds\n" \
  "s_add_u32 m0, s47, 35840\n" \
  "s_waitcnt lgkmcnt(8)\n" \
  "v_mfma_i32_16x16x64_i8 v[132:135], v[104:107], a[128:131], v[132:135]\n" \
  "ds_read_b64_tr_b8 v[96:97], v84 offset:0\n" \
  "buffer_load_dwordx4 v43, s[16:19], s82 offen lds\n" \
  "v_mfma_i32_16x16x64_i8 v[132:135], v[108:111], a[144:147], v[132:135]\n" \
  "ds_read_b64_tr_b8 v[98:99], v84 offset:1024\n" \
  "v_mfma_i32_16x16x64_i8 v[164:167], v[104:107], a[132:135], v[164:167]\n" \
  "v_mfma_i32_16x16x64_i8 v[164:167], v[108:111], a[148:151], v[164:167]\n" \
  "v_mfma_i32_16x16x64_i8 v[196:199], v[104:107], a[136:139], v[196:199]\n" \
  "ds_read_b64_tr_b8 v[100:101], v84 offset:8192\n" \
  "v_mfma_i32_16x16x64_i8 v[196:199], v[108:111], a[152:155], v[196:199]\n" \
  "ds_read_b64_tr_b8 v[102:103], v84 offset:9216\n" \
  "v_mfma_i32_16x16x64_i8 v[228:231], v[104:107], a[140:143], v[228:231]\n" \
  "v_mfma_i32_16x16x64_i8 v[228:231], v[108:111], a[156:159], v[228:231]\n" \
  "s_waitcnt lgkmcnt(8)\n" \
  "v_mfma_i32_16x16x64_i8 v[136:139], v[112:115], a[128:131], v[136:139]\n" \
  "ds_read_b64_tr_b8 v[104:105], v85 offset:0\n" \
  "v_mfma_i32_16x16x64_i8 v[136:139], v[116:119], a[144:147], v[136:139]\n" \
  "ds_read_b64_tr_b8 v[106:107], v85 offset:1024\n" \
  "v_mfma_i32_16x16x64_i8 v[168:171], v[112:115], a[132:135], v[168:171]\n" \
  "v_mfma_i32_16x16x64_i8 v[168:171], v[116:119], a[148:151], v[168:171]\n" \
  "v_mfma_i32_16x16x64_i8 v[200:203], v[112:115], a[136:139], v[200:203]\n" \
  "ds_read_b64_tr_b8 v[108:109], v85 offset:8192\n" \
  "v_mfma_i32_16x16x64_i8 v[200:203], v[116:119], a[152:155], v[200:203]\n" \
  "ds_read_b64_tr_b8 v[110:111], v85 offset:9216\n" \
  "v_mfma_i32_16x16x64_i8 v[232:235], v[112:115], a[140:143], v[232:235]\n" \
  "v_mfma_i32_16x16x64_i8 v[232:235], v[116:119], a[156:159], v[232:235]\n" \
  "s_waitcnt lgkmcnt(8)\n" \
  "v_mfma_i32_16x16x64_i8 v[140:143], v[120:123], a[128:131], v[140:143]\n" \
  "ds_read_b64_tr_b8 v[112:113], v86 offset:0\n" \
  "v_mfma_i32_16x16x64_i8 v[140:143], v[124:127], a[144:147], v[140:143]\n" \
  "ds_read_b64_tr_b8 v[114:115], v86 offset:1024\n" \
  "v_mfma_i32_16x16x64_i8 v[172:175], v[120:123], a[132:135], v[172:175]\n" \
  "v_mfma_i32_16x16x64_i8 v[172:175], v[124:127], a[148:151], v[172:175]\n" \
  "s_waitcnt vmcnt(53)\n" \
  "s_barrier\n" \
  "v_mfma_i32_16x16x64_i8 v[204:207], v[120:123], a[136:139], v[204:207]\n" \
  "ds_read_b64_tr_b8 v[116:117], v86 offset:8192\n" \
  "v_mfma_i32_16x16x64_i8 v[204:207], v[124:127], a[152:155], v[204:207]\n" \
  "ds_read_b64_tr_b8 v[118:119], v86 offset:9216\n" \
  "v_mfma_i32_16x16x64_i8 v[236:239], v[120:123], a[140:143], v[236:239]\n" \
  "v_mfma_i32_16x16x64_i8 v[236:239], v[124:127], a[156:159], v[236:239]\n" \
  "s_waitcnt lgkmcnt(8)\n" \
  "v_mfma_i32_16x16x64_i8 v[144:147], v[96:99], a[128:131], v[144:147]\n" \
  "ds_read_b64_tr_b8 v[120:121], v87 offset:0\n" \
  "v_mfma_i32_16x16x64_i8 v[144:147], v[100:103], a[144:147], v[144:147]\n" \
  "ds_read_b64_tr_b8 v[122:123], v87 offset:1024\n" \
  "v_mfma_i32_16x16x64_i8 v[176:179], v[96:99], a[132:135], v[176:179]\n" \
  "v_mfma_i32_16x16x64_i8 v[176:179], v[100:103], a[148:151], v[176:179]\n" \
  "v_mfma_i32_16x16x64_i8 v[208:211], v[96:99], a[136:139], v[208:211]\n" \
  "ds_read_b64_tr_b8 v[124:125], v87 offset:8192\n" \
  "v_mfma_i32_16x16x64_i8 v[208:211], v[100:103], a[152:155], v[208:211]\n" \
  "ds_read_b64_tr_b8 v[126:127], v87 offset:9216\n" \
  "v_mfma_i32_16x16x64_i8 v[240:243], v[96:99], a[140:143], v[240:243]\n" \
  "v_mfma_i32_16x16x64_i8 v[240:243], v[100:103], a[156:159], v[240:243]\n" \
  "s_waitcnt lgkmcnt(8)\n" \
  "v_mfma_i32_16x16x64_i8 v[148:151], v[104:107], a[128:131], v[148:151]\n" \
  "ds_read_b64_tr_b8 v[96:97], v80 offset:16384\n" \
  "v_mfma_i32_16x16x64_i8 v[148:151], v[108:111], a[144:147], v[148:151]\n" \
  "ds_read_b64_tr_b8 v[98:99], v80 offset:17408\n" \
  "v_mfma_i32_16x16x64_i8 v[180:183], v[104:107], a[132:135], v[180:183]\n" \
  "v_mfma_i32_16x16x64_i8 v[180:183], v[108:111], a[148:151], v[180:183]\n" \
  "v_mfma_i32_16x16x64_i8 v[212:215], v[104:107], a[136:139], v[212:215]\n" \
  "ds_read_b64_tr_b8 v[100:101], v80 offset:24576\n" \
  "v_mfma_i32_16x16x64_i8 v[212:215], v[108:111], a[152:155], v[212:215]\n" \
  "ds_read_b64_tr_b8 v[102:103], v80 offset:25600\n" \
  "v_mfma_i32_16x16x64_i8 v[244:247], v[104:107], a[140:143], v[244:247]\n" \
  "v_mfma_i32_16x16x64_i8 v[244:247], v[108:111], a[156:159], v[244:247]\n" \
  "s_waitcnt lgkmcnt(8)\n" \
  "v_mfma_i32_16x16x64_i8 v[152:155], v[112:115], a[128:131], v[152:155]\n" \
  "ds_read_b64_tr_b8 v[104:105], v81 offset:16384\n" \
  "v_mfma_i32_16x16x64_i8 v[152:155], v[116:119], a[144:147], v[152:155]\n" \
  "ds_read_b64_tr_b8 v[106:107], v81 offset:17408\n" \
  "v_mfma_i32_16x16x64_i8 v[184:187], v[112:115], a[132:135], v[184:187]\n" \
  "v_mfma_i32_16x16x64_i8 v[184:187], v[116:119], a[148:151], v[184:187]\n" \
  "v_mfma_i32_16x16x64_i8 v[216:219], v[112:115], a[136:139], v[216:219]\n" \
  "ds_read_b64_tr_b8 v[108:109], v81 offset:24576\n" \
  "v_mfma_i32_16x16x64_i8 v[216:219], v[116:119], a[152:155], v[216:219]\n" \
  "ds_read_b64_tr_b8 v[110:111], v81 offset:25600\n" \
  "v_mfma_i32_16x16x64_i8 v[248:251], v[112:115], a[140:143], v[248:251]\n" \
  "v_mfma_i32_16x16x64_i8 v[248:251], v[116:119], a[156:159], v[248:251]\n" \
  "s_waitcnt lgkmcnt(8)\n" \
  "v_mfma_i32_16x16x64_i8 v[156:159], v[120:123], a[128:131], v[156:159]\n" \
  "ds_read_b64_tr_b8 v[112:113], v82 offset:16384\n" \
  "v_mfma_i32_16x16x64_i8 v[156:159], v[124:127], a[144:147], v[156:159]\n" \
  "ds_read_b64_tr_b8 v[114:115], v82 offset:17408\n" \
  "v_mfma_i32_16x16x64_i8 v[188:191], v[120:123], a[132:135], v[188:191]\n" \
  "v_mfma_i32_16x16x64_i8 v[188:191], v[124:127], a[148:151], v[188:191]\n" \
  "v_mfma_i32_16x16x64_i8 v[220:223], v[120:123], a[136:139], v[220:223]\n" \
  "ds_read_b64_tr_b8 v[116:117], v82 offset:24576\n" \
  "v_mfma_i32_16x16x64_i8 v[220:223], v[124:127], a[152:155], v[220:223]\n" \
  "ds_read_b64_tr_b8 v[118:119], v82 offset:25600\n" \
  "v_mfma_i32_16x16x64_i8 v[252:255], v[120:123], a[140:143], v[252:255]\n" \
  "v_mfma_i32_16x16x64_i8 v[252:255], v[124:127], a[156:159], v[252:255]\n" \
  "; ---- step 5\n" \
  "s_waitcnt vmcnt(25)\n" \
  "s_waitcnt lgkmcnt(8)\n" \
  "v_mfma_i32_16x16x64_i8 v[128:131], v[96:99], a[160:163], v[128:131]\n" \
  "ds_read_b64_tr_b8 v[120:121], v83 offset:16384\n" \
  "v_mfma_i32_16x16x64_i8 v[128:131], v[100:103], a[176:179], v[128:131]\n" \
  "ds_read_b64_tr_b8 v[122:123], v83 offset:17408\n" \
  "v_mfma_i32_16x16x64_i8 v[160:163], v[96:99], a[164:167], v[160:163]\n" \
  "v_mfma_i32_16x16x64_i8 v[160:163], v[100:103], a[180:183], v[160:163]\n" \
  "v_mfma_i32_16x16x64_i8 v[192:195], v[96:99], a[168:171], v[192:195]\n" \
  "ds_read_b64_tr_b8 v[124:125], v83 offset:24576\n" \
  "v_mfma_i32_16x16x64_i8 v[192:195], v[100:103], a[184:187], v[192:195]\n" \
  "ds_read_b64_tr_b8 v[126:127], v83 offset:25600\n" \
  "v_mfma_i32_16x16x64_i8 v[224:227], v[96:99], a[172:175], v[224:227]\n" \
  "v_mfma_i32_16x16x64_i8 v[224:227], v[100:103], a[188:191], v[224:227]\n" \
  "s_waitcnt lgkmcnt(8)\n" \
  "v_mfma_i32_16x16x64_i8 v[132:135], v[104:107], a[160:163], v[132:135]\n" \
  "ds_read_b64_tr_b8 v[96:97], v84 offset:16384\n" \
  "v_mfma_i32_16x16x64_i8 v[132:135], v[108:111], a[176:179], v[132:135]\n" \
  "ds_read_b64_tr_b8 v[98:99], v84 offset:17408\n" \
  "v_mfma_i32_16x16x64_i8 v[164:167], v[104:107], a[164:167], v[164:167]\n" \
  "v_mfma_i32_16x16x64_i8 v[164:167], v[108:111], a[180:183], v[164:167]\n" \
  "v_mfma_i32_16x16x64_i8 v[196:199], v[104:107], a[168:171], v[196:199]\n" \
  "ds_read_b64_tr_b8 v[100:101], v84 offset:24576\n" \
  "v_mfma_i32_16x16x64_i8 v[196:199], v[108:111], a[184:187], v[196:199]\n" \
  "ds_read_b64_tr_b8 v[102:103], v84 offset:25600\n" \
  "v_mfma_i32_16x16x64_i8 v[228:231], v[104:107], a[172:175], v[228:231]\n" \
  "v_mfma_i32_16x16x64_i8 v[228:231], v[108:111], a[188:191], v[228:231]\n" \
  "s_waitcnt lgkmcnt(8)\n" \
  "v_mfma_i32_16x16x64_i8 v[136:139], v[112:115], a[160:163], v[136:139]\n" \
  "ds_read_b64_tr_b8 v[104:105], v85 offset:16384\n" \
  "v_mfma_i32_16x16x64_i8 v[136:139], v[116:119], a[176:179], v[136:139]\n" \
  "ds_read_b64_tr_b8 v[106:107], v85 offset:17408\n" \
  "v_mfma_i32_16x16x64_i8 v[168:171], v[112:115], a[164:167], v[168:171]\n" \
  "v_mfma_i32_16x16x64_i8 v[168:171], v[116:119], a[180:183], v[168:171]\n" \
  "v_mfma_i32_16x16x64_i8 v[200:203], v[112:115], a[168:171], v[200:203]\n" \
  "ds_read_b64_tr_b8 v[108:109], v85 offset:24576\n" \
  "v_mfma_i32_16x16x64_i8 v[200:203], v[116:119], a[184:187], v[200:203]\n" \
  "ds_read_b64_tr_b8 v[110:111], v85 offset:25600\n" \
  "buffer_load_dwordx4 a[224:227], v44, s[28:31], s95 offen offset:0 sc0 sc1\n" \
  "v_mfma_i32_16x16x64_i8 v[232:235], v[112:115], a[172:175], v[232:235]\n" \
  "v_mfma_i32_16x16x64_i8 v[232:235], v[116:119], a[188:191], v[232:235]\n" \
  "buffer_load_dwordx4 a[228:231], v44, s[28:31], s95 offen offset:1024 sc0 sc1\n" \
  "s_waitcnt lgkmcnt(8)\n" \
  "v_mfma_i32_16x16x64_i8 v[140:143], v[120:123], a[160:163], v[140:143]\n" \
  "ds_read_b64_tr_b8 v[112:113], v86 offset:16384\n" \
  "v_mfma_i32_16x16x64_i8 v[140:143], v[124:127], a[176:179], v[140:143]\n" \
  "ds_read_b64_tr_b8 v[114:115], v86 offset:17408\n" \
  "v_mfma_i32_16x16x64_i8 v[172:175], v[120:123], a[164:167], v[172:175]\n" \
  "v_mfma_i32_16x16x64_i8 v[172:175], v[124:127], a[180:183], v[172:175]\n" \
  "buffer_load_dwordx4 a[232:235], v44, s[28:31], s95 offen offset:2048 sc0 sc1\n" \
  "s_waitcnt vmcnt(52)\n" \
  "s_barrier\n" \
  "v_mfma_i32_16x16x64_i8 v[204:207], v[120:123], a[168:171], v[204:207]\n" \
  "ds_read_b64_tr_b8 v[116:117], v86 offset:24576\n" \
  "v_mfma_i32_16x16x64_i8 v[204:207], v[124:127], a[184:187], v[204:207]\n" \
  "ds_read_b64_tr_b8 v[118:119], v86 offset:25600\n" \
  "v_mfma_i32_16x16x64_i8 v[236:239], v[120:123], a[172:175], v[236:239]\n" \
  "v_mfma_i32_16x16x64_i8 v[236:239], v[124:127], a[188:191], v[236:239]\n" \
  "buffer_load_dwordx4 a[236:239], v44, s[28:31], s95 offen offset:3072 sc0 sc1\n" \
  "s_waitcnt lgkmcnt(8)\n" \
  "v_mfma_i32_16x16x64_i8 v[144:147], v[96:99], a[160:163], v[144:147]\n" \
  "ds_read_b64_tr_b8 v[120:121], v87 offset:16384\n" \
  "v_mfma_i32_16x16x64_i8 v[144:147], v[100:103], a[176:179], v[144:147]\n" \
  "ds_read_b64_tr_b8 v[122:123], v87 offset:17408\n" \
  "v_mfma_i32_16x16x64_i8 v[176:179], v[96:99], a[164:167], v[176:179]\n" \
  "v_mfma_i32_16x16x64_i8 v[176:179], v[100:103], a[180:183], v[176:179]\n" \
  "buffer_load_dwordx4 a[240:243], v91, s[28:31], s95 offen offset:0 sc0 sc1\n" \
  "v_mfma_i32_16x16x64_i8 v[208:211], v[96:99], a[168:171], v[208:211]\n" \
  "ds_read_b64_tr_b8 v[124:125], v87 offset:24576\n" \
  "v_mfma_i32_16x16x64_i8 v[208:211], v[100:103], a[184:187], v[208:211]\n" \
  "ds_read_b64_tr_b8 v[126:127], v87 offset:25600\n" \
  "v_mfma_i32_16x16x64_i8 v[240:243], v[96:99], a[172:175], v[240:243]\n" \
  "v_mfma_i32_16x16x64_i8 v[240:243], v[100:103], a[188:191], v[240:243]\n" \
  "buffer_load_dwordx4 a[244:247], v91, s[28:31], s95 offen offset:1024 sc0 sc1\n" \
  "s_waitcnt lgkmcnt(8)\n" \
  "v_mfma_i32_16x16x64_i8 v[148:151], v[104:107], a[160:163], v[148:151]\n" \
  "ds_read_b64_tr_b8 v[96:97], v80 offset:32768\n" \
  "v_mfma_i32_16x16x64_i8 v[148:151], v[108:111], a[176:179], v[148:151]\n" \
  "ds_read_b64_tr_b8 v[98:99], v80 offset:33792\n" \
  "v_mfma_i32_16x16x64_i8 v[180:183], v[104:107], a[164:167], v[180:183]\n" \
  "v_mfma_i32_16x16x64_i8 v[180:183], v[108:111], a[180:183], v[180:183]\n" \
  "buffer_load_dwordx4 a[248:251], v91, s[28:31], s95 offen offset:2048 sc0 sc1\n" \
  "v_mfma_i32_16x16x64_i8 v[212:215], v[104:107], a[168:171], v[212:215]\n" \
  "ds_read_b64_tr_b8 v[100:101], v80 offset:40960\n" \
  "v_mfma_i32_16x16x64_i8 v[212:215], v[108:111], a[184:187], v[212:215]\n" \
  "ds_read_b64_tr_b8 v[102:103], v80 offset:41984\n" \
  "buffer_load_dwordx4 a[252:255], v91, s[28:31], s95 offen offset:3072 sc0 sc1\n" \
  "v_mfma_i32_16x16x64_i8 v[244:247], v[104:107], a[172:175], v[244:247]\n" \
  "v_mfma_i32_16x16x64_i8 v[244:247], v[108:111], a[188:191], v[244:247]\n" \
  "s_waitcnt lgkmcnt(8)\n" \
  "v_mfma_i32_16x16x64_i8 v[152:155], v[112:115], a[160:163], v[152:155]\n" \
  "ds_read_b64_tr_b8 v[104:105], v81 offset:32768\n" \
  "v_mfma_i32_16x16x64_i8 v[152:155], v[116:119], a[176:179], v[152:155]\n" \
  "ds_read_b64_tr_b8 v[106:107], v81 offset:33792\n" \
  "buffer_load_dwordx4 a[0:3], v44, s[28:31], s88 offen offset:0 sc0 sc1\n" \
  "v_mfma_i32_16x16x64_i8 v[184:187], v[112:115], a[164:167], v[184:187]\n" \
  "v_mfma_i32_16x16x64_i8 v[184:187], v[116:119], a[180:183], v[184:187]\n" \
  "v_mfma_i32_16x16x64_i8 v[216:219], v[112:115], a[168:171], v[216:219]\n" \
  "ds_read_b64_tr_b8 v[108:109], v81 offset:40960\n" \
  "v_mfma_i32_16x16x64_i8 v[216:219], v[116:119], a[184:187], v[216:219]\n" \
  "ds_read_b64_tr_b8 v[110:111], v81 offset:41984\n" \
  "buffer_load_dwordx4 a[4:7], v44, s[28:31], s88 offen offset:1024 sc0 sc1\n" \
  "v_mfma_i32_16x16x64_i8 v[248:251], v[112:115], a[172:175], v[248:251]\n" \
  "v_mfma_i32_16x16x64_i8 v[248:251], v[116:119], a[188:191], v[248:251]\n" \
  "s_waitcnt lgkmcnt(8)\n" \
  "v_mfma_i32_16x16x64_i8 v[156:159], v[120:123], a[160:163], v[156:159]\n" \
  "ds_read_b64_tr_b8 v[112:113], v82 offset:32768\n" \
  "v_mfma_i32_16x16x64_i8 v[156:159], v[124:127], a[176:179], v[156:159]\n" \
  "ds_read_b64_tr_b8 v[114:115], v82 offset:33792\n" \
  "buffer_load_dwordx4 a[8:11], v44, s[28:31], s88 offen offset:2048 sc0 sc1\n" \
  "v_mfma_i32_16x16x64_i8 v[188:191], v[120:123], a[164:167], v[188:191]\n" \
  "v_mfma_i32_16x16x64_i8 v[188:191], v[124:127], a[180:183], v[188:191]\n" \
  "v_mfma_i32_16x16x64_i8 v[220:223], v[120:123], a[168:171], v[220:223]\n" \
  "ds_read_b64_tr_b8 v[116:117], v82 offset:40960\n" \
  "v_mfma_i32_16x16x64_i8 v[220:223], v[124:127], a[184:187], v[220:223]\n" \
  "ds_read_b64_tr_b8 v[118:119], v82 offset:41984\n" \
  "buffer_load_dwordx4 a[12:15], v44, s[28:31], s88 offen offset:3072 sc0 sc1\n" \
  "v_mfma_i32_16x16x64_i8 v[252:255], v[120:123], a[172:175], v[252:255]\n" \
  "v_mfma_i32_16x16x64_i8 v[252:255], v[124:127], a[188:191], v[252:255]\n" \
  "; ---- step 6\n" \
  "s_waitcnt vmcnt(29)\n" \
  "s_waitcnt lgkmcnt(8)\n" \
  "v_mfma_i32_16x16x64_i8 v[128:131], v[96:99], a[192:195], v[128:131]\n" \
  "ds_read_b64_tr_b8 v[120:121], v83 offset:32768\n" \
  "v_mfma_i32_16x16x64_i8 v[128:131], v[100:103], a[208:211], v[128:131]\n" \
  "ds_read_b64_tr_b8 v[122:123], v83 offset:33792\n" \
  "v_mfma_i32_16x16x64_i8 v[160:163], v[96:99], a[196:199], v[160:163]\n" \
  "v_mfma_i32_16x16x64_i8 v[160:163], v[100:103], a[212:215], v[160:163]\n" \
  "buffer_load_dwordx4 a[16:19], v91, s[28:31], s88 offen offset:0 sc0 sc1\n" \
  "v_mfma_i32_16x16x64_i8 v[192:195], v[96:99], a[200:203], v[192:195]\n" \
  "ds_read_b64_tr_b8 v[124:125], v83 offset:40960\n" \
  "v_mfma_i32_16x16x64_i8 v[192:195], v[100:103], a[216:219], v[192:195]\n" \
  "ds_read_b64_tr_b8 v[126:127], v83 offset:41984\n" \
  "v_mfma_i32_16x16x64_i8 v[224:227], v[96:99], a[204:207], v[224:227]\n" \
  "v_mfma_i32_16x16x64_i8 v[224:227], v[100:103], a[220:223], v[224:227]\n" \
  "buffer_load_dwordx4 a[20:23], v91, s[28:31], s88 offen offset:1024 sc0 sc1\n" \
  "s_waitcnt lgkmcnt(8)\n" \
  "v_mfma_i32_16x16x64_i8 v[132:135], v[104:107], a[192:195], v[132:135]\n" \
  "ds_read_b64_tr_b8 v[96:97], v84 offset:32768\n" \
  "v_mfma_i32_16x16x64_i8 v[132:135], v[108:111], a[208:211], v[132:135]\n" \
  "ds_read_b64_tr_b8 v[98:99], v84 offset:33792\n" \
  "v_mfma_i32_16x16x64_i8 v[164:167], v[104:107], a[196:199], v[164:167]\n" \
  "v_mfma_i32_16x16x64_i8 v[164:167], v[108:111], a[212:215], v[164:167]\n" \
  "buffer_load_dwordx4 a[24:27], v91, s[28:31], s88 offen offset:2048 sc0 sc1\n" \
  "v_mfma_i32_16x16x64_i8 v[196:199], v[104:107], a[200:203], v[196:199]\n" \
  "ds_read_b64_tr_b8 v[100:101], v84 offset:40960\n" \
  "v_mfma_i32_16x16x64_i8 v[196:199], v[108:111], a[216:219], v[196:199]\n" \
  "ds_read_b64_tr_b8 v[102:103], v84 offset:41984\n" \
  "v_mfma_i32_16x16x64_i8 v[228:231], v[104:107], a[204:207], v[228:231]\n" \
  "v_mfma_i32_16x16x64_i8 v[228:231], v[108:111], a[220:223], v[228:231]\n" \
  "s_waitcnt lgkmcnt(8)\n" \
  "v_mfma_i32_16x16x64_i8 v[136:139], v[112:115], a[192:195], v[136:139]\n" \
  "ds_read_b64_tr_b8 v[104:105], v85 offset:32768\n" \
  "v_mfma_i32_16x16x64_i8 v[136:139], v[116:119], a[208:211], v[136:139]\n" \
  "ds_read_b64_tr_b8 v[106:107], v85 offset:33792\n" \
  "v_mfma_i32_16x16x64_i8 v[168:171], v[112:115], a[196:199], v[168:171]\n" \
  "v_mfma_i32_16x16x64_i8 v[168:171], v[116:119], a[212:215], v[168:171]\n" \
  "buffer_load_dwordx4 a[28:31], v91, s[28:31], s88 offen offset:3072 sc0 sc1\n" \
  "v_mfma_i32_16x16x64_i8 v[200:203], v[112:115], a[200:203], v[200:203]\n" \
  "ds_read_b64_tr_b8 v[108:109], v85 offset:40960\n" \
  "v_mfma_i32_16x16x64_i8 v[200:203], v[116:119], a[216:219], v[200:203]\n" \
  "ds_read_b64_tr_b8 v[110:111], v85 offset:41984\n" \
  "v_mfma_i32_16x16x64_i8 v[232:235], v[112:115], a[204:207], v[232:235]\n" \
  "v_mfma_i32_16x16x64_i8 v[232:235], v[116:119], a[220:223], v[232:235]\n" \
  "buffer_load_dwordx4 a[32:35], v44, s[28:31], s89 offen offset:0 sc0 sc1\n" \
  "s_waitcnt lgkmcnt(8)\n" \
  "v_mfma_i32_16x16x64_i8 v[140:143], v[120:123], a[192:195], v[140:143]\n" \
  "ds_read_b64_tr_b8 v[112:113], v86 offset:32768\n" \
  "v_mfma_i32_16x16x64_i8 v[140:143], v[124:127], a[208:211], v[140:143]\n" \
  "ds_read_b64_tr_b8 v[114:115], v86 offset:33792\n" \
  "v_mfma_i32_16x16x64_i8 v[172:175], v[120:123], a[196:199], v[172:175]\n" \
  "v_mfma_i32_16x16x64_i8 v[172:175], v[124:127], a[212:215], v[172:175]\n" \
  "buffer_load_dwordx4 a[36:39], v44, s[28:31], s89 offen offset:1024 sc0 sc1\n" \
  "s_waitcnt vmcnt(31)\n" \
  "s_barrier\n" \
  "v_mfma_i32_16x16x64_i8 v[204:207], v[120:123], a[200:203], v[204:207]\n" \
  "ds_read_b64_tr_b8 v[116:117], v86 offset:40960\n" \
  "v_mfma_i32_16x16x64_i8 v[204:207], v[124:127], a[216:219], v[204:207]\n" \
  "ds_read_b64_tr_b8 v[118:119], v86 offset:41984\n" \
  "v_mfma_i32_16x16x64_i8 v[236:239], v[120:123], a[204:207], v[236:239]\n" \
  "v_mfma_i32_16x16x64_i8 v[236:239], v[124:127], a[220:223], v[236:239]\n" \
  "s_waitcnt lgkmcnt(8)\n" \
  "v_mfma_i32_16x16x64_i8 v[144:147], v[96:99], a[192:195], v[144:147]\n" \
  "ds_read_b64_tr_b8 v[120:121], v87 offset:32768\n" \
  "v_mfma_i32_16x16x64_i8 v[144:147], v[100:103], a[208:211], v[144:147]\n" \
  "ds_read_b64_tr_b8 v[122:123], v87 offset:33792\n" \
  "v_mfma_i32_16x16x64_i8 v[176:179], v[96:99], a[196:199], v[176:179]\n" \
  "v_mfma_i32_16x16x64_i8 v[176:179], v[100:103], a[212:215], v[176:179]\n" \
  "buffer_load_dwordx4 a[40:43], v44, s[28:31], s89 offen offset:2048 sc0 sc1\n" \
  "v_mfma_i32_16x16x64_i8 v[208:211], v[96:99], a[200:203], v[208:211]\n" \
  "ds_read_b64_tr_b8 v[124:125], v87 offset:40960\n" \
  "v_mfma_i32_16x16x64_i8 v[208:211], v[100:103], a[216:219], v[208:211]\n" \
  "ds_read_b64_tr_b8 v[126:127], v87 offset:41984\n" \
  "v_mfma_i32_16x16x64_i8 v[240:243], v[96:99], a[204:207], v[240:243]\n" \
  "v_mfma_i32_16x16x64_i8 v[240:243], v[100:103], a[220:223], v[240:243]\n" \
  "buffer_load_dwordx4 a[44:47], v44, s[28:31], s89 offen offset:3072 sc0 sc1\n" \
  "s_waitcnt lgkmcnt(8)\n" \
  "v_mfma_i32_16x16x64_i8 v[148:151], v[104:107], a[192:195], v[148:151]\n" \
  "ds_read_b64_tr_b8 v[96:97], v80 offset:49152\n" \
  "v_mfma_i32_16x16x64_i8 v[148:151], v[108:111], a[208:211], v[148:151]\n" \
  "ds_read_b64_tr_b8 v[98:99], v80 offset:50176\n" \
  "v_mfma_i32_16x16x64_i8 v[180:183], v[104:107], a[196:199], v[180:183]\n" \
  "v_mfma_i32_16x16x64_i8 v[180:183], v[108:111], a[212:215], v[180:183]\n" \
  "buffer_load_dwordx4 a[48:51], v91, s[28:31], s89 offen offset:0 sc0 sc1\n" \
  "v_mfma_i32_16x16x64_i8 v[212:215], v[104:107], a[200:203], v[212:215]\n" \
  "ds_read_b64_tr_b8 v[100:101], v80 offset:57344\n" \
  "v_mfma_i32_16x16x64_i8 v[212:215], v[108:111], a[216:219], v[212:215]\n" \
  "ds_read_b64_tr_b8 v[102:103], v80 offset:58368\n" \
  "v_mfma_i32_16x16x64_i8 v[244:247], v[104:107], a[204:207], v[244:247]\n" \
  "v_mfma_i32_16x16x64_i8 v[244:247], v[108:111], a[220:223], v[244:247]\n" \
  "s_waitcnt lgkmcnt(8)\n" \
  "v_mfma_i32_16x16x64_i8 v[152:155], v[112:115], a[192:195], v[152:155]\n" \
  "ds_read_b64_tr_b8 v[104:105], v81 offset:49152\n" \
  "v_mfma_i32_16x16x64_i8 v[152:155], v[116:119], a[208:211], v[152:155]\n" \
  "ds_read_b64_tr_b8 v[106:107], v81 offset:50176\n" \
  "v_mfma_i32_16x16x64_i8 v[184:187], v[112:115], a[196:199], v[184:187]\n" \
  "v_mfma_i32_16x16x64_i8 v[184:187], v[116:119], a[212:215], v[184:187]\n" \
  "buffer_load_dwordx4 a[52:55], v91, s[28:31], s89 offen offset:1024 sc0 sc1\n" \
  "v_mfma_i32_16x16x64_i8 v[216:219], v[112:115], a[200:203], v[216:219]\n" \
  "ds_read_b64_tr_b8 v[108:109], v81 offset:57344\n" \
  "v_mfma_i32_16x16x64_i8 v[216:219], v[116:119], a[216:219], v[216:219]\n" \
  "ds_read_b64_tr_b8 v[110:111], v81 offset:58368\n" \
  "v_mfma_i32_16x16x64_i8 v[248:251], v[112:115], a[204:207], v[248:251]\n" \
  "v_mfma_i32_16x16x64_i8 v[248:251], v[116:119], a[220:223], v[248:251]\n" \
  "buffer_load_dwordx4 a[56:59], v91, s[28:31], s89 offen offset:2048 sc0 sc1\n" \
  "s_waitcnt lgkmcnt(8)\n" \
  "v_mfma_i32_16x16x64_i8 v[156:159], v[120:123], a[192:195], v[156:159]\n" \
  "ds_read_b64_tr_b8 v[112:113], v82 offset:49152\n" \
  "v_mfma_i32_16x16x64_i8 v[156:159], v[124:127], a[208:211], v[156:159]\n" \
  "ds_read_b64_tr_b8 v[114:115], v82 offset:50176\n" \
  "v_mfma_i32_16x16x64_i8 v[188:191], v[120:123], a[196:199], v[188:191]\n" \
  "v_mfma_i32_16x16x64_i8 v[188:191], v[124:127], a[212:215], v[188:191]\n" \
  "buffer_load_dwordx4 a[60:63], v91, s[28:31], s89 offen offset:3072 sc0 sc1\n" \
  "v_mfma_i32_16x16x64_i8 v[220:223], v[120:123], a[200:203], v[220:223]\n" \
  "ds_read_b64_tr_b8 v[116:117], v82 offset:57344\n" \
  "v_mfma_i32_16x16x64_i8 v[220:223], v[124:127], a[216:219], v[220:223]\n" \
  "ds_read_b64_tr_b8 v[118:119], v82 offset:58368\n" \
  "v_mfma_i32_16x16x64_i8 v[252:255], v[120:123], a[204:207], v[252:255]\n" \
  "v_mfma_i32_16x16x64_i8 v[252:255], v[124:127], a[220:223], v[252:255]\n" \
  "; ---- step 7\n" \
  "s_waitcnt vmcnt(16)\n" \
  "s_waitcnt lgkmcnt(8)\n" \
  "v_mfma_i32_16x16x64_i8 v[128:131], v[96:99], a[224:227], v[128:131]\n" \
  "ds_read_b64_tr_b8 v[120:121], v83 offset:49152\n" \
  "v_mfma_i32_16x16x64_i8 v[128:131], v[100:103], a[240:243], v[128:131]\n" \
  "ds_read_b64_tr_b8 v[122:123], v83 offset:50176\n" \
  "v_mfma_i32_16x16x64_i8 v[160:163], v[96:99], a[228:231], v[160:163]\n" \
  "v_mfma_i32_16x16x64_i8 v[160:163], v[100:103], a[244:247], v[160:163]\n" \
  "buffer_load_dwordx4 a[64:67], v44, s[28:31], s90 offen offset:0 sc0 sc1\n" \
  "v_mfma_i32_16x16x64_i8 v[192:195], v[96:99], a[232:235], v[192:195]\n" \
  "ds_read_b64_tr_b8 v[124:125], v83 offset:57344\n" \
  "v_mfma_i32_16x16x64_i8 v[192:195], v[100:103], a[248:251], v[192:195]\n" \
  "ds_read_b64_tr_b8 v[126:127], v83 offset:58368\n" \
  "v_mfma_i32_16x16x64_i8 v[224:227], v[96:99], a[236:239], v[224:227]\n" \
  "v_mfma_i32_16x16x64_i8 v[224:227], v[100:103], a[252:255], v[224:227]\n" \
  "buffer_load_dwordx4 a[68:71], v44, s[28:31], s90 offen offset:1024 sc0 sc1\n" \
  "s_waitcnt lgkmcnt(8)\n" \
  "v_mfma_i32_16x16x64_i8 v[132:135], v[104:107], a[224:227], v[132:135]\n" \
  "ds_read_b64_tr_b8 v[96:97], v84 offset:49152\n" \
  "v_mfma_i32_16x16x64_i8 v[132:135], v[108:111], a[240:243], v[132:135]\n" \
  "ds_read_b64_tr_b8 v[98:99], v84 offset:50176\n" \
  "v_mfma_i32_16x16x64_i8 v[164:167], v[104:107], a[228:231], v[164:167]\n" \
  "v_mfma_i32_16x16x64_i8 v[164:167], v[108:111], a[244:247], v[164:167]\n" \
  "buffer_load_dwordx4 a[72:75], v44, s[28:31], s90 offen offset:2048 sc0 sc1\n" \
  "v_mfma_i32_16x16x64_i8 v[196:199], v[104:107], a[232:235], v[196:199]\n" \
  "ds_read_b64_tr_b8 v[100:101], v84 offset:57344\n" \
  "v_mfma_i32_16x16x64_i8 v[196:199], v[108:111], a[248:251], v[196:199]\n" \
  "ds_read_b64_tr_b8 v[102:103], v84 offset:58368\n" \
  "v_mfma_i32_16x16x64_i8 v[228:231], v[104:107], a[236:239], v[228:231]\n" \
  "v_mfma_i32_16x16x64_i8 v[228:231], v[108:111], a[252:255], v[228:231]\n" \
  "buffer_load_dwordx4 a[76:79], v44, s[28:31], s90 offen offset:3072 sc0 sc1\n" \
  "s_waitcnt lgkmcnt(8)\n" \
  "v_mfma_i32_16x16x64_i8 v[136:139], v[112:115], a[224:227], v[136:139]\n" \
  "ds_read_b64_tr_b8 v[104:105], v85 offset:49152\n" \
  "v_mfma_i32_16x16x64_i8 v[136:139], v[116:119], a[240:243], v[136:139]\n" \
  "ds_read_b64_tr_b8 v[106:107], v85 offset:50176\n" \
  "v_mfma_i32_16x16x64_i8 v[168:171], v[112:115], a[228:231], v[168:171]\n" \
  "v_readlane_b32 s50, v46, s49\n" \
  "v_mfma_i32_16x16x64_i8 v[168:171], v[116:119], a[244:247], v[168:171]\n" \
  "buffer_load_dwordx4 a[80:83], v91, s[28:31], s90 offen offset:0 sc0 sc1\n" \
  "v_mfma_i32_16x16x64_i8 v[200:203], v[112:115], a[232:235], v[200:203]\n" \
  "ds_read_b64_tr_b8 v[108:109], v85 offset:57344\n" \
  "v_mfma_i32_16x16x64_i8 v[200:203], v[116:119], a[248:251], v[200:203]\n" \
  "ds_read_b64_tr_b8 v[110:111], v85 offset:58368\n" \
  "v_mfma_i32_16x16x64_i8 v[232:235], v[112:115], a[236:239], v[232:235]\n" \
  "v_mfma_i32_16x16x64_i8 v[232:235], v[116:119], a[252:255], v[232:235]\n" \
  "buffer_load_dwordx4 a[84:87], v91, s[28:31], s90 offen offset:1024 sc0 sc1\n" \
  "s_waitcnt lgkmcnt(8)\n" \
  "v_mfma_i32_16x16x64_i8 v[140:143], v[120:123], a[224:227], v[140:143]\n" \
  "ds_read_b64_tr_b8 v[112:113], v86 offset:49152\n" \
  "v_mfma_i32_16x16x64_i8 v[140:143], v[124:127], a[240:243], v[140:143]\n" \
  "ds_read_b64_tr_b8 v[114:115], v86 offset:50176\n" \
  "v_mfma_i32_16x16x64_i8 v[172:175], v[120:123], a[228:231], v[172:175]\n" \
  "s_lshl_b32 s83, s50, 21\n" \
  "s_lshl_b32 s91, s50, 16\n" \
  "s_add_u32 s49, s49, 1\n" \
  "v_mfma_i32_16x16x64_i8 v[172:175], v[124:127], a[244:247], v[172:175]\n" \
  "buffer_load_dwordx4 a[88:91], v91, s[28:31], s90 offen offset:2048 sc0 sc1\n" \
  "s_waitcnt vmcnt(40)\n" \
  "s_barrier\n" \
  "v_mfma_i32_16x16x64_i8 v[204:207], v[120:123], a[232:235], v[204:207]\n" \
  "ds_read_b64_tr_b8 v[116:117], v86 offset:57344\n" \
  "v_mfma_i32_16x16x64_i8 v[204:207], v[124:127], a[248:251], v[204:207]\n" \
  "ds_read_b64_tr_b8 v[118:119], v86 offset:58368\n" \
  "v_mfma_i32_16x16x64_i8 v[236:239], v[120:123], a[236:239], v[236:239]\n" \
  "v_mfma_i32_16x16x64_i8 v[236:239], v[124:127], a[252:255], v[236:239]\n" \
  "buffer_load_dwordx4 a[92:95], v91, s[28:31], s90 offen offset:3072 sc0 sc1\n" \
  "s_waitcnt lgkmcnt(8)\n" \
  "v_mfma_i32_16x16x64_i8 v[144:147], v[96:99], a[224:227], v[144:147]\n" \
  "ds_read_b64_tr_b8 v[120:121], v87 offset:49152\n" \
  "v_mfma_i32_16x16x64_i8 v[144:147], v[100:103], a[240:243], v[144:147]\n" \
  "ds_read_b64_tr_b8 v[122:123], v87 offset:50176\n" \
  "s_add_u32 m0, s47, 49152\n" \
  "v_mfma_i32_16x16x64_i8 v[176:179], v[96:99], a[228:231], v[176:179]\n" \
  "buffer_load_dwordx4 v40, s[16:19], s83 offen nt lds\n" \
  "v_readlane_b32 s50, v46, s49\n" \
  "v_mfma_i32_16x16x64_i8 v[176:179], v[100:103], a[244:247], v[176:179]\n" \
  "v_mfma_i32_16x16x64_i8 v[208:211], v[96:99], a[232:235], v[208:211]\n" \
  "ds_read_b64_tr_b8 v[124:125], v87 offset:57344\n" \
  "v_mfma_i32_16x16x64_i8 v[208:211], v[100:103], a[248:251], v[208:211]\n" \
  "ds_read_b64_tr_b8 v[126:127], v87 offset:58368\n" \
  "s_add_u32 m0, s47, 50176\n" \
  "v_mfma_i32_16x16x64_i8 v[240:243], v[96:99], a[236:239], v[240:243]\n" \
  "buffer_load_dwordx4 v41, s[16:19], s83 offen nt lds\n" \
  "v_mfma_i32_16x16x64_i8 v[240:243], v[100:103], a[252:255], v[240:243]\n" \
  "s_waitcnt lgkmcnt(8)\n" \
  "v_mfma_i32_16x16x64_i8 v[148:151], v[104:107], a[224:227], v[148:151]\n" \
  "ds_read_b64_tr_b8 v[96:97], v32 offset:0\n" \
  "v_mfma_i32_16x16x64_i8 v[148:151], v[108:111], a[240:243], v[148:151]\n" \
  "ds_read_b64_tr_b8 v[98:99], v32 offset:1024\n" \
  "s_add_u32 m0, s47, 51200\n" \
  "v_mfma_i32_16x16x64_i8 v[180:183], v[104:107], a[228:231], v[180:183]\n" \
  "buffer_load_dwordx4 v42, s[16:19], s83 offen nt lds\n" \
  "s_lshl_b32 s84, s50, 21\n" \
  "s_lshl_b32 s92, s50, 16\n" \
  "s_add_u32 s49, s49, 1\n" \
  "v_mfma_i32_16x16x64_i8 v[180:183], v[108:111], a[244:247], v[180:183]\n" \
  "v_mfma_i32_16x16x64_i8 v[212:215], v[104:107], a[232:235], v[212:215]\n" \
  "ds_read_b64_tr_b8 v[100:101], v32 offset:8192\n" \
  "v_mfma_i32_16x16x64_i8 v[212:215], v[108:111], a[248:251], v[212:215]\n" \
  "ds_read_b64_tr_b8 v[102:103], v32 offset:9216\n" \
  "s_add_u32 m0, s47, 52224\n" \
  "v_mfma_i32_16x16x64_i8 v[244:247], v[104:107], a[236:239], v[244:247]\n" \
  "buffer_load_dwordx4 v43, s[16:19], s83 offen nt lds\n" \
  "v_mfma_i32_16x16x64_i8 v[244:247], v[108:111], a[252:255], v[244:247]\n" \
  "s_waitcnt lgkmcnt(8)\n" \
  "v_mfma_i32_16x16x64_i8 v[152:155], v[112:115], a[224:227], v[152:155]\n" \
  "ds_read_b64_tr_b8 v[104:105], v33 offset:0\n" \
  "v_mfma_i32_16x16x64_i8 v[152:155], v[116:119], a[240:243], v[152:155]\n" \
  "ds_read_b64_tr_b8 v[106:107], v33 offset:1024\n" \
  "s_add_u32 m0, s47, 65536\n" \
  "v_mfma_i32_16x16x64_i8 v[184:187], v[112:115], a[228:231], v[184:187]\n" \
  "buffer_load_dwordx4 v40, s[16:19], s84 offen nt lds\n" \
  "v_readlane_b32 s50, v46, s49\n" \
  "v_mfma_i32_16x16x64_i8 v[184:187], v[116:119], a[244:247], v[184:187]\n" \
  "v_mfma_i32_16x16x64_i8 v[216:219], v[112:115], a[232:235], v[216:219]\n" \
  "ds_read_b64_tr_b8 v[108:109], v33 offset:8192\n" \
  "v_mfma_i32_16x16x64_i8 v[216:219], v[116:119], a[248:251], v[216:219]\n" \
  "ds_read_b64_tr_b8 v[110:111], v33 offset:9216\n" \
  "s_add_u32 m0, s47, 66560\n" \
  "v_mfma_i32_16x16x64_i8 v[248:251], v[112:115], a[236:239], v[248:251]\n" \
  "buffer_load_dwordx4 v41, s[16:19], s84 offen nt lds\n" \
  "v_mfma_i32_16x16x64_i8 v[248:251], v[116:119], a[252:255], v[248:251]\n" \
  "s_waitcnt lgkmcnt(8)\n" \
  "v_mfma_i32_16x16x64_i8 v[156:159], v[120:123], a[224:227], v[156:159]\n" \
  "ds_read_b64_tr_b8 v[112:113], v34 offset:0\n" \
  "v_mfma_i32_16x16x64_i8 v[156:159], v[124:127], a[240:243], v[156:159]\n" \
  "ds_read_b64_tr_b8 v[114:115], v34 offset:1024\n" \
  "s_add_u32 m0, s47, 67584\n" \
  "v_mfma_i32_16x16x64_i8 v[188:191], v[120:123], a[228:231], v[188:191]\n" \
  "buffer_load_dwordx4 v42, s[16:19], s84 offen nt lds\n" \
  "s_lshl_b32 s85, s50, 21\n" \
  "s_lshl_b32 s93, s50, 16\n" \
  "s_add_u32 s49, s49, 1\n" \
  "v_mfma_i32_16x16x64_i8 v[188:191], v[124:127], a[244:247], v[188:191]\n" \
  "v_mfma_i32_16x16x64_i8 v[220:223], v[120:123], a[232:235], v[220:223]\n" \
  "ds_read_b64_tr_b8 v[116:117], v34 offset:8192\n" \
  "v_mfma_i32_16x16x64_i8 v[220:223], v[124:127], a[248:251], v[220:223]\n" \
  "ds_read_b64_tr_b8 v[118:119], v34 offset:9216\n" \
  "s_add_u32 m0, s47, 68608\n" \
  "v_mfma_i32_16x16x64_i8 v[252:255], v[120:123], a[236:239], v[252:255]\n" \
  "buffer_load_dwordx4 v43, s[16:19], s84 offen nt lds\n" \
  "v_mfma_i32_16x16x64_i8 v[252:255], v[124:127], a[252:255], v[252:255]\n" \
  "s_add_u32 s53, s53, 1\n" \
  "s_add_u32 s54, s54, 1\n" \
  "; ---- step 0\n" \
  "global_load_dword v90, v51, s[12:13] sc1\n" \
  "s_waitcnt vmcnt(25)\n" \
  "s_waitcnt lgkmcnt(8)\n" \
  "v_mfma_i32_16x16x64_i8 v[128:131], v[96:99], a[0:3], v[128:131]\n" \
  "ds_read_b64_tr_b8 v[120:121], v35 offset:0\n" \
  "v_mfma_i32_16x16x64_i8 v[128:131], v[100:103], a[16:19], v[128:131]\n" \
  "ds_read_b64_tr_b8 v[122:123], v35 offset:1024\n" \
  "s_add_u32 m0, s47, 81920\n" \
  "v_mfma_i32_16x16x64_i8 v[160:163], v[96:99], a[4:7], v[160:163]\n" \
  "buffer_load_dwordx4 v40, s[16:19], s85 offen nt lds\n" \
  "v_mfma_i32_16x16x64_i8 v[160:163], v[100:103], a[20:23], v[160:163]\n" \
  "v_mfma_i32_16x16x64_i8 v[192:195], v[96:99], a[8:11], v[192:195]\n" \
  "ds_read_b64_tr_b8 v[124:125], v35 offset:8192\n" \
  "v_mfma_i32_16x16x64_i8 v[192:195], v[100:103], a[24:27], v[192:195]\n" \
  "ds_read_b64_tr_b8 v[126:127], v35 offset:9216\n" \
  "s_add_u32 m0, s47, 82944\n" \
  "v_mfma_i32_16x16x64_i8 v[224:227], v[96:99], a[12:15], v[224:227]\n" \
  "buffer_load_dwordx4 v41, s[16:19], s85 offen nt lds\n" \
  "v_mfma_i32_16x16x64_i8 v[224:227], v[100:103], a[28:31], v[224:227]\n" \
  "s_waitcnt lgkmcnt(8)\n" \
  "v_mfma_i32_16x16x64_i8 v[132:135], v[104:107], a[0:3], v[132:135]\n" \
  "ds_read_b64_tr_b8 v[96:97], v36 offset:0\n" \
  "v_mfma_i32_16x16x64_i8 v[132:135], v[108:111], a[16:19], v[132:135]\n" \
  "ds_read_b64_tr_b8 v[98:99], v36 offset:1024\n" \
  "s_add_u32 m0, s47, 83968\n" \
  "v_mfma_i32_16x16x64_i8 v[164:167], v[104:107], a[4:7], v[164:167]\n" \
  "buffer_load_dwordx4 v42, s[16:19], s85 offen nt lds\n" \
  "v_mfma_i32_16x16x64_i8 v[164:167], v[108:111], a[20:23], v[164:167]\n" \
  "v_mfma_i32_16x16x64_i8 v[196:199], v[104:107], a[8:11], v[196:199]\n" \
  "ds_read_b64_tr_b8 v[100:101], v36 offset:8192\n" \
  "v_mfma_i32_16x16x64_i8 v[196:199], v[108:111], a[24:27], v[196:199]\n" \
  "ds_read_b64_tr_b8 v[102:103], v36 offset:9216\n" \
  "s_add_u32 m0, s47, 84992\n" \
  "v_mfma_i32_16x16x64_i8 v[228:231], v[104:107], a[12:15], v[228:231]\n" \
  "buffer_load_dwordx4 v43, s[16:19], s85 offen nt lds\n" \
  "v_mfma_i32_16x16x64_i8 v[228:231], v[108:111], a[28:31], v[228:231]\n" \
  "s_waitcnt lgkmcnt(8)\n" \
  "v_mfma_i32_16x16x64_i8 v[136:139], v[112:115], a[0:3], v[136:139]\n" \
  "ds_read_b64_tr_b8 v[104:105], v37 offset:0\n" \
  "v_mfma_i32_16x16x64_i8 v[136:139], v[116:119], a[16:19], v[136:139]\n" \
  "ds_read_b64_tr_b8 v[106:107], v37 offset:1024\n" \
  "v_mfma_i32_16x16x64_i8 v[168:171], v[112:115], a[4:7], v[168:171]\n" \
  "v_mfma_i32_16x16x64_i8 v[168:171], v[116:119], a[20:23], v[168:171]\n" \
  "v_mfma_i32_16x16x64_i8 v[200:203], v[112:115], a[8:11], v[200:203]\n" \
  "ds_read_b64_tr_b8 v[108:109], v37 offset:8192\n" \
  "v_mfma_i32_16x16x64_i8 v[200:203], v[116:119], a[24:27], v[200:203]\n" \
  "ds_read_b64_tr_b8 v[110:111], v37 offset:9216\n" \
  "v_mfma_i32_16x16x64_i8 v[232:235], v[112:115], a[12:15], v[232:235]\n" \
  "v_mfma_i32_16x16x64_i8 v[232:235], v[116:119], a[28:31], v[232:235]\n" \
  "s_waitcnt lgkmcnt(8)\n" \
  "v_mfma_i32_16x16x64_i8 v[140:143], v[120:123], a[0:3], v[140:143]\n" \
  "ds_read_b64_tr_b8 v[112:113], v38 offset:0\n" \
  "v_mfma_i32_16x16x64_i8 v[140:143], v[124:127], a[16:19], v[140:143]\n" \
  "ds_read_b64_tr_b8 v[114:115], v38 offset:1024\n" \
  "v_mfma_i32_16x16x64_i8 v[172:175], v[120:123], a[4:7], v[172:175]\n" \
  "v_mfma_i32_16x16x64_i8 v[172:175], v[124:127], a[20:23], v[172:175]\n" \
  "s_waitcnt vmcnt(49)\n" \
  "s_barrier\n" \
  "v_mfma_i32_16x16x64_i8 v[204:207], v[120:123], a[8:11], v[204:207]\n" \
  "ds_read_b64_tr_b8 v[116:117], v38 offset:8192\n" \
  "v_mfma_i32_16x16x64_i8 v[204:207], v[124:127], a[24:27], v[204:207]\n" \
  "ds_read_b64_tr_b8 v[118:119], v38 offset:9216\n" \
  "v_mfma_i32_16x16x64_i8 v[236:239], v[120:123], a[12:15], v[236:239]\n" \
  "v_mfma_i32_16x16x64_i8 v[236:239], v[124:127], a[28:31], v[236:239]\n" \
  "s_waitcnt lgkmcnt(8)\n" \
  "v_mfma_i32_16x16x64_i8 v[144:147], v[96:99], a[0:3], v[144:147]\n" \
  "ds_read_b64_tr_b8 v[120:121], v39 offset:0\n" \
  "v_mfma_i32_16x16x64_i8 v[144:147], v[100:103], a[16:19], v[144:147]\n" \
  "ds_read_b64_tr_b8 v[122:123], v39 offset:1024\n" \
  "v_mfma_i32_16x16x64_i8 v[176:179], v[96:99], a[4:7], v[176:179]\n" \
  "v_mfma_i32_16x16x64_i8 v[176:179], v[100:103], a[20:23], v[176:179]\n" \
  "v_mfma_i32_16x16x64_i8 v[208:211], v[96:99], a[8:11], v[208:211]\n" \
  "ds_read_b64_tr_b8 v[124:125], v39 offset:8192\n" \
  "v_mfma_i32_16x16x64_i8 v[208:211], v[100:103], a[24:27], v[208:211]\n" \
  "ds_read_b64_tr_b8 v[126:127], v39 offset:9216\n" \
  "v_mfma_i32_16x16x64_i8 v[240:243], v[96:99], a[12:15], v[240:243]\n" \
  "v_mfma_i32_16x16x64_i8 v[240:243], v[100:103], a[28:31], v[240:243]\n" \
  "s_waitcnt lgkmcnt(8)\n" \
  "v_mfma_i32_16x16x64_i8 v[148:151], v[104:107], a[0:3], v[148:151]\n" \
  "ds_read_b64_tr_b8 v[96:97], v32 offset:16384\n" \
  "v_mfma_i32_16x16x64_i8 v[148:151], v[108:111], a[16:19], v[148:151]\n" \
  "ds_read_b64_tr_b8 v[98:99], v32 offset:17408\n" \
  "v_mfma_i32_16x16x64_i8 v[180:183], v[104:107], a[4:7], v[180:183]\n" \
  "v_mfma_i32_16x16x64_i8 v[180:183], v[108:111], a[20:23], v[180:183]\n" \
  "v_mfma_i32_16x16x64_i8 v[212:215], v[104:107], a[8:11], v[212:215]\n" \
  "ds_read_b64_tr_b8 v[100:101], v32 offset:24576\n" \
  "v_mfma_i32_16x16x64_i8 v[212:215], v[108:111], a[24:27], v[212:215]\n" \
  "ds_read_b64_tr_b8 v[102:103], v32 offset:25600\n" \
  "v_mfma_i32_16x16x64_i8 v[244:247], v[104:107], a[12:15], v[244:247]\n" \
  "v_mfma_i32_16x16x64_i8 v[244:247], v[108:111], a[28:31], v[244:247]\n" \
  "s_waitcnt lgkmcnt(8)\n" \
  "v_mfma_i32_16x16x64_i8 v[152:155], v[112:115], a[0:3], v[152:155]\n" \
  "ds_read_b64_tr_b8 v[104:105], v33 offset:16384\n" \
  "v_mfma_i32_16x16x64_i8 v[152:155], v[116:119], a[16:19], v[152:155]\n" \
  "ds_read_b64_tr_b8 v[106:107], v33 offset:17408\n" \
  "v_mfma_i32_16x16x64_i8 v[184:187], v[112:115], a[4:7], v[184:187]\n" \
  "v_mfma_i32_16x16x64_i8 v[184:187], v[116:119], a[20:23], v[184:187]\n" \
  "v_mfma_i32_16x16x64_i8 v[216:219], v[112:115], a[8:11], v[216:219]\n" \
  "ds_read_b64_tr_b8 v[108:109], v33 offset:24576\n" \
  "v_mfma_i32_16x16x64_i8 v[216:219], v[116:119], a[24:27], v[216:219]\n" \
  "ds_read_b64_tr_b8 v[110:111], v33 offset:25600\n" \
  "v_mfma_i32_16x16x64_i8 v[248:251], v[112:115], a[12:15], v[248:251]\n" \
  "v_mfma_i32_16x16x64_i8 v[248:251], v[116:119], a[28:31], v[248:251]\n" \
  "s_waitcnt lgkmcnt(8)\n" \
  "v_mfma_i32_16x16x64_i8 v[156:159], v[120:123], a[0:3], v[156:159]\n" \
  "ds_read_b64_tr_b8 v[112:113], v34 offset:16384\n" \
  "v_mfma_i32_16x16x64_i8 v[156:159], v[124:127], a[16:19], v[156:159]\n" \
  "ds_read_b64_tr_b8 v[114:115], v34 offset:17408\n" \
  "v_mfma_i32_16x16x64_i8 v[188:191], v[120:123], a[4:7], v[188:191]\n" \
  "v_mfma_i32_16x16x64_i8 v[188:191], v[124:127], a[20:23], v[188:191]\n" \
  "v_mfma_i32_16x16x64_i8 v[220:223], v[120:123], a[8:11], v[220:223]\n" \
  "ds_read_b64_tr_b8 v[116:117], v34 offset:24576\n" \
  "v_mfma_i32_16x16x64_i8 v[220:223], v[124:127], a[24:27], v[220:223]\n" \
  "ds_read_b64_tr_b8 v[118:119], v34 offset:25600\n" \
  "v_mfma_i32_16x16x64_i8 v[252:255], v[120:123], a[12:15], v[252:255]\n" \
  "v_mfma_i32_16x16x64_i8 v[252:255], v[124:127], a[28:31], v[252:255]\n" \
  "; ---- step 1\n" \
  "s_waitcnt vmcnt(21)\n" \
  "s_waitcnt lgkmcnt(8)\n" \
  "v_mfma_i32_16x16x64_i8 v[128:131], v[96:99], a[32:35], v[128:131]\n" \
  "ds_read_b64_tr_b8 v[120:121], v35 offset:16384\n" \
  "v_mfma_i32_16x16x64_i8 v[128:131], v[100:103], a[48:51], v[128:131]\n" \
  "ds_read_b64_tr_b8 v[122:123], v35 offset:17408\n" \
  "v_mfma_i32_16x16x64_i8 v[160:163], v[96:99], a[36:39], v[160:163]\n" \
  "v_mfma_i32_16x16x64_i8 v[160:163], v[100:103], a[52:55], v[160:163]\n" \
  "v_mfma_i32_16x16x64_i8 v[192:195], v[96:99], a[40:43], v[192:195]\n" \
  "ds_read_b64_tr_b8 v[124:125], v35 offset:24576\n" \
  "v_mfma_i32_16x16x64_i8 v[192:195], v[100:103], a[56:59], v[192:195]\n" \
  "ds_read_b64_tr_b8 v[126:127], v35 offset:25600\n" \
  "v_mfma_i32_16x16x64_i8 v[224:227], v[96:99], a[44:47], v[224:227]\n" \
  "v_mfma_i32_16x16x64_i8 v[224:227], v[100:103], a[60:63], v[224:227]\n" \
  "s_waitcnt lgkmcnt(8)\n" \
  "v_mfma_i32_16x16x64_i8 v[132:135], v[104:107], a[32:35], v[132:135]\n" \
  "ds_read_b64_tr_b8 v[96:97], v36 offset:16384\n" \
  "v_mfma_i32_16x16x64_i8 v[132:135], v[108:111], a[48:51], v[132:135]\n" \
  "ds_read_b64_tr_b8 v[98:99], v36 offset:17408\n" \
  "v_mfma_i32_16x16x64_i8 v[164:167], v[104:107], a[36:39], v[164:167]\n" \
  "v_mfma_i32_16x16x64_i8 v[164:167], v[108:111], a[52:55], v[164:167]\n" \
  "v_mfma_i32_16x16x64_i8 v[196:199], v[104:107], a[40:43], v[196:199]\n" \
  "ds_read_b64_tr_b8 v[100:101], v36 offset:24576\n" \
  "v_mfma_i32_16x16x64_i8 v[196:199], v[108:111], a[56:59], v[196:199]\n" \
  "ds_read_b64_tr_b8 v[102:103], v36 offset:25600\n" \
  "v_mfma_i32_16x16x64_i8 v[228:231], v[104:107], a[44:47], v[228:231]\n" \
  "v_mfma_i32_16x16x64_i8 v[228:231], v[108:111], a[60:63], v[228:231]\n" \
  "s_waitcnt lgkmcnt(8)\n" \
  "v_mfma_i32_16x16x64_i8 v[136:139], v[112:115], a[32:35], v[136:139]\n" \
  "ds_read_b64_tr_b8 v[104:105], v37 offset:16384\n" \
  "v_mfma_i32_16x16x64_i8 v[136:139], v[116:119], a[48:51], v[136:139]\n" \
  "ds_read_b64_tr_b8 v[106:107], v37 offset:17408\n" \
  "v_mfma_i32_16x16x64_i8 v[168:171], v[112:115], a[36:39], v[168:171]\n" \
  "v_mfma_i32_16x16x64_i8 v[168:171], v[116:119], a[52:55], v[168:171]\n" \
  "v_mfma_i32_16x16x64_i8 v[200:203], v[112:115], a[40:43], v[200:203]\n" \
  "ds_read_b64_tr_b8 v[108:109], v37 offset:24576\n" \
  "v_mfma_i32_16x16x64_i8 v[200:203], v[116:119], a[56:59], v[200:203]\n" \
  "ds_read_b64_tr_b8 v[110:111], v37 offset:25600\n" \
  "buffer_load_dwordx4 a[96:99], v44, s[28:31], s91 offen offset:0 sc0 sc1\n" \
  "v_mfma_i32_16x16x64_i8 v[232:235], v[112:115], a[44:47], v[232:235]\n" \
  "v_mfma_i32_16x16x64_i8 v[232:235], v[116:119], a[60:63], v[232:235]\n" \
  "buffer_load_dwordx4 a[100:103], v44, s[28:31], s91 offen offset:1024 sc0 sc1\n" \
  "s_waitcnt lgkmcnt(8)\n" \
  "v_mfma_i32_16x16x64_i8 v[140:143], v[120:123], a[32:35], v[140:143]\n" \
  "ds_read_b64_tr_b8 v[112:113], v38 offset:16384\n" \
  "v_mfma_i32_16x16x64_i8 v[140:143], v[124:127], a[48:51], v[140:143]\n" \
  "ds_read_b64_tr_b8 v[114:115], v38 offset:17408\n" \
  "v_mfma_i32_16x16x64_i8 v[172:175], v[120:123], a[36:39], v[172:175]\n" \
  "v_mfma_i32_16x16x64_i8 v[172:175], v[124:127], a[52:55], v[172:175]\n" \
  "buffer_load_dwordx4 a[104:107], v44, s[28:31], s91 offen offset:2048 sc0 sc1\n" \
  "s_waitcnt vmcnt(48)\n" \
  "s_barrier\n" \
  "v_mfma_i32_16x16x64_i8 v[204:207], v[120:123], a[40:43], v[204:207]\n" \
  "ds_read_b64_tr_b8 v[116:117], v38 offset:24576\n" \
  "v_mfma_i32_16x16x64_i8 v[204:207], v[124:127], a[56:59], v[204:207]\n" \
  "ds_read_b64_tr_b8 v[118:119], v38 offset:25600\n" \
  "v_mfma_i32_16x16x64_i8 v[236:239], v[120:123], a[44:47], v[236:239]\n" \
  "v_mfma_i32_16x16x64_i8 v[236:239], v[124:127], a[60:63], v[236:239]\n" \
  "buffer_load_dwordx4 a[108:111], v44, s[28:31], s91 offen offset:3072 sc0 sc1\n" \
  "s_waitcnt lgkmcnt(8)\n" \
  "v_mfma_i32_16x16x64_i8 v[144:147], v[96:99], a[32:35], v[144:147]\n" \
  "ds_read_b64_tr_b8 v[120:121], v39 offset:16384\n" \
  "v_mfma_i32_16x16x64_i8 v[144:147], v[100:103], a[48:51], v[144:147]\n" \
  "ds_read_b64_tr_b8 v[122:123], v39 offset:17408\n" \
  "v_mfma_i32_16x16x64_i8 v[176:179], v[96:99], a[36:39], v[176:179]\n" \
  "v_mfma_i32_16x16x64_i8 v[176:179], v[100:103], a[52:55], v[176:179]\n" \
  "buffer_load_dwordx4 a[112:115], v91, s[28:31], s91 offen offset:0 sc0 sc1\n" \
  "v_mfma_i32_16x16x64_i8 v[208:211], v[96:99], a[40:43], v[208:211]\n" \
  "ds_read_b64_tr_b8 v[124:125], v39 offset:24576\n" \
  "v_mfma_i32_16x16x64_i8 v[208:211], v[100:103], a[56:59], v[208:211]\n" \
  "ds_read_b64_tr_b8 v[126:127], v39 offset:25600\n" \
  "v_mfma_i32_16x16x64_i8 v[240:243], v[96:99], a[44:47], v[240:243]\n" \
  "v_mfma_i32_16x16x64_i8 v[240:243], v[100:103], a[60:63], v[240:243]\n" \
  "buffer_load_dwordx4 a[116:119], v91, s[28:31], s91 offen offset:1024 sc0 sc1\n" \
  "s_waitcnt lgkmcnt(8)\n" \
  "v_mfma_i32_16x16x64_i8 v[148:151], v[104:107], a[32:35], v[148:151]\n" \
  "ds_read_b64_tr_b8 v[96:97], v32 offset:32768\n" \
  "v_mfma_i32_16x16x64_i8 v[148:151], v[108:111], a[48:51], v[148:151]\n" \
  "ds_read_b64_tr_b8 v[98:99], v32 offset:33792\n" \
  "v_mfma_i32_16x16x64_i8 v[180:183], v[104:107], a[36:39], v[180:183]\n" \
  "v_mfma_i32_16x16x64_i8 v[180:183], v[108:111], a[52:55], v[180:183]\n" \
  "buffer_load_dwordx4 a[120:123], v91, s[28:31], s91 offen offset:2048 sc0 sc1\n" \
  "v_mfma_i32_16x16x64_i8 v[212:215], v[104:107], a[40:43], v[212:215]\n" \
  "ds_read_b64_tr_b8 v[100:101], v32 offset:40960\n" \
  "v_mfma_i32_16x16x64_i8 v[212:215], v[108:111], a[56:59], v[212:215]\n" \
  "ds_read_b64_tr_b8 v[102:103], v32 offset:41984\n" \
  "buffer_load_dwordx4 a[124:127], v91, s[28:31], s91 offen offset:3072 sc0 sc1\n" \
  "v_mfma_i32_16x16x64_i8 v[244:247], v[104:107], a[44:47], v[244:247]\n" \
  "v_mfma_i32_16x16x64_i8 v[244:247], v[108:111], a[60:63], v[244:247]\n" \
  "s_waitcnt lgkmcnt(8)\n" \
  "v_mfma_i32_16x16x64_i8 v[152:155], v[112:115], a[32:35], v[152:155]\n" \
  "ds_read_b64_tr_b8 v[104:105], v33 offset:32768\n" \
  "v_mfma_i32_16x16x64_i8 v[152:155], v[116:119], a[48:51], v[152:155]\n" \
  "ds_read_b64_tr_b8 v[106:107], v33 offset:33792\n" \
  "buffer_load_dwordx4 a[128:131], v44, s[28:31], s92 offen offset:0 sc0 sc1\n" \
  "v_mfma_i32_16x16x64_i8 v[184:187], v[112:115], a[36:39], v[184:187]\n" \
  "v_mfma_i32_16x16x64_i8 v[184:187], v[116:119], a[52:55], v[184:187]\n" \
  "v_mfma_i32_16x16x64_i8 v[216:219], v[112:115], a[40:43], v[216:219]\n" \
  "ds_read_b64_tr_b8 v[108:109], v33 offset:40960\n" \
  "v_mfma_i32_16x16x64_i8 v[216:219], v[116:119], a[56:59], v[216:219]\n" \
  "ds_read_b64_tr_b8 v[110:111], v33 offset:41984\n" \
  "buffer_load_dwordx4 a[132:135], v44, s[28:31], s92 offen offset:1024 sc0 sc1\n" \
  "v_mfma_i32_16x16x64_i8 v[248:251], v[112:115], a[44:47], v[248:251]\n" \
  "v_mfma_i32_16x16x64_i8 v[248:251], v[116:119], a[60:63], v[248:251]\n" \
  "s_waitcnt lgkmcnt(8)\n" \
  "v_mfma_i32_16x16x64_i8 v[156:159], v[120:123], a[32:35], v[156:159]\n" \
  "ds_read_b64_tr_b8 v[112:113], v34 offset:32768\n" \
  "v_mfma_i32_16x16x64_i8 v[156:159], v[124:127], a[48:51], v[156:159]\n" \
  "ds_read_b64_tr_b8 v[114:115], v34 offset:33792\n" \
  "buffer_load_dwordx4 a[136:139], v44, s[28:31], s92 offen offset:2048 sc0 sc1\n" \
  "v_mfma_i32_16x16x64_i8 v[188:191], v[120:123], a[36:39], v[188:191]\n" \
  "v_mfma_i32_16x16x64_i8 v[188:191], v[124:127], a[52:55], v[188:191]\n" \
  "v_mfma_i32_16x16x64_i8 v[220:223], v[120:123], a[40:43], v[220:223]\n" \
  "ds_read_b64_tr_b8 v[116:117], v34 offset:40960\n" \
  "v_mfma_i32_16x16x64_i8 v[220:223], v[124:127], a[56:59], v[220:223]\n" \
  "ds_read_b64_tr_b8 v[118:119], v34 offset:41984\n" \
  "buffer_load_dwordx4 a[140:143], v44, s[28:31], s92 offen offset:3072 sc0 sc1\n" \
  "v_mfma_i32_16x16x64_i8 v[252:255], v[120:123], a[44:47], v[252:255]\n" \
  "v_mfma_i32_16x16x64_i8 v[252:255], v[124:127], a[60:63], v[252:255]\n" \
  "; ---- step 2\n" \
  "s_waitcnt vmcnt(16)\n" \
  "v_mul_f32 v88, v77, v78\n" \
  "v_cmp_eq_u32 vcc, v90, v76\n" \
  "s_nop 1\n" \
  "s_cmp_lg_u32 s55, 0\n" \
  "s_cselect_b64 vcc, -1, vcc\n" \
  "v_mov_b32 v89, 0x7fc00000\n" \
  "s_nop 1\n" \
  "v_cndmask_b32 v88, v88, v89, vcc\n" \
  "v_mov_b32 v89, v88\n" \
  "s_waitcnt vmcnt(25)\n" \
  "s_waitcnt lgkmcnt(8)\n" \
  "v_mfma_i32_16x16x64_i8 v[128:131], v[96:99], a[64:67], v[128:131]\n" \
  "ds_read_b64_tr_b8 v[120:121], v35 offset:32768\n" \
  "v_mfma_i32_16x16x64_i8 v[128:131], v[100:103], a[80:83], v[128:131]\n" \
  "ds_read_b64_tr_b8 v[122:123], v35 offset:33792\n" \
  "v_mfma_i32_16x16x64_i8 v[160:163], v[96:99], a[68:71], v[160:163]\n" \
  "v_mfma_i32_16x16x64_i8 v[160:163], v[100:103], a[84:87], v[160:163]\n" \
  "buffer_load_dwordx4 a[144:147], v91, s[28:31], s92 offen offset:0 sc0 sc1\n" \
  "v_mfma_i32_16x16x64_i8 v[192:195], v[96:99], a[72:75], v[192:195]\n" \
  "ds_read_b64_tr_b8 v[124:125], v35 offset:40960\n" \
  "v_mfma_i32_16x16x64_i8 v[192:195], v[100:103], a[88:91], v[192:195]\n" \
  "ds_read_b64_tr_b8 v[126:127], v35 offset:41984\n" \
  "v_mfma_i32_16x16x64_i8 v[224:227], v[96:99], a[76:79], v[224:227]\n" \
  "v_mfma_i32_16x16x64_i8 v[224:227], v[100:103], a[92:95], v[224:227]\n" \
  "buffer_load_dwordx4 a[148:151], v91, s[28:31], s92 offen offset:1024 sc0 sc1\n" \
  "s_waitcnt lgkmcnt(8)\n" \
  "v_mfma_i32_16x16x64_i8 v[132:135], v[104:107], a[64:67], v[132:135]\n" \
  "ds_read_b64_tr_b8 v[96:97], v36 offset:32768\n" \
  "v_mfma_i32_16x16x64_i8 v[132:135], v[108:111], a[80:83], v[132:135]\n" \
  "ds_read_b64_tr_b8 v[98:99], v36 offset:33792\n" \
  "v_mfma_i32_16x16x64_i8 v[164:167], v[104:107], a[68:71], v[164:167]\n" \
  "v_mfma_i32_16x16x64_i8 v[164:167], v[108:111], a[84:87], v[164:167]\n" \
  "buffer_load_dwordx4 a[152:155], v91, s[28:31], s92 offen offset:2048 sc0 sc1\n" \
  "v_mfma_i32_16x16x64_i8 v[196:199], v[104:107], a[72:75], v[196:199]\n" \
  "ds_read_b64_tr_b8 v[100:101], v36 offset:40960\n" \
  "v_mfma_i32_16x16x64_i8 v[196:199], v[108:111], a[88:91], v[196:199]\n" \
  "ds_read_b64_tr_b8 v[102:103], v36 offset:41984\n" \
  "v_mfma_i32_16x16x64_i8 v[228:231], v[104:107], a[76:79], v[228:231]\n" \
  "v_mfma_i32_16x16x64_i8 v[228:231], v[108:111], a[92:95], v[228:231]\n" \
  "s_waitcnt lgkmcnt(8)\n" \
  "v_mfma_i32_16x16x64_i8 v[136:139], v[112:115], a[64:67], v[136:139]\n" \
  "ds_read_b64_tr_b8 v[104:105], v37 offset:32768\n" \
  "v_mfma_i32_16x16x64_i8 v[136:139], v[116:119], a[80:83], v[136:139]\n" \
  "ds_read_b64_tr_b8 v[106:107], v37 offset:33792\n" \
  "v_mfma_i32_16x16x64_i8 v[168:171], v[112:115], a[68:71], v[168:171]\n" \
  "v_mfma_i32_16x16x64_i8 v[168:171], v[116:119], a[84:87], v[168:171]\n" \
  "buffer_load_dwordx4 a[156:159], v91, s[28:31], s92 offen offset:3072 sc0 sc1\n" \
  "v_mfma_i32_16x16x64_i8 v[200:203], v[112:115], a[72:75], v[200:203]\n" \
  "ds_read_b64_tr_b8 v[108:109], v37 offset:40960\n" \
  "v_mfma_i32_16x16x64_i8 v[200:203], v[116:119], a[88:91], v[200:203]\n" \
  "ds_read_b64_tr_b8 v[110:111], v37 offset:41984\n" \
  "v_mfma_i32_16x16x64_i8 v[232:235], v[112:115], a[76:79], v[232:235]\n" \
  "v_mfma_i32_16x16x64_i8 v[232:235], v[116:119], a[92:95], v[232:235]\n" \
  "buffer_load_dwordx4 a[160:163], v44, s[28:31], s93 offen offset:0 sc0 sc1\n" \
  "s_waitcnt lgkmcnt(8)\n" \
  "v_mfma_i32_16x16x64_i8 v[140:143], v[120:123], a[64:67], v[140:143]\n" \
  "ds_read_b64_tr_b8 v[112:113], v38 offset:32768\n" \
  "v_mfma_i32_16x16x64_i8 v[140:143], v[124:127], a[80:83], v[140:143]\n" \
  "ds_read_b64_tr_b8 v[114:115], v38 offset:33792\n" \
  "v_mfma_i32_16x16x64_i8 v[172:175], v[120:123], a[68:71], v[172:175]\n" \
  "v_mfma_i32_16x16x64_i8 v[172:175], v[124:127], a[84:87], v[172:175]\n" \
  "buffer_load_dwordx4 a[164:167], v44, s[28:31], s93 offen offset:1024 sc0 sc1\n" \
  "s_waitcnt vmcnt(18)\n" \
  "s_barrier\n" \
  "v_mfma_i32_16x16x64_i8 v[204:207], v[120:123], a[72:75], v[204:207]\n" \
  "ds_read_b64_tr_b8 v[116:117], v38 offset:40960\n" \
  "v_mfma_i32_16x16x64_i8 v[204:207], v[124:127], a[88:91], v[204:207]\n" \
  "ds_read_b64_tr_b8 v[118:119], v38 offset:41984\n" \
  "v_mfma_i32_16x16x64_i8 v[236:239], v[120:123], a[76:79], v[236:239]\n" \
  "v_mfma_i32_16x16x64_i8 v[236:239], v[124:127], a[92:95], v[236:239]\n" \
  "s_waitcnt lgkmcnt(8)\n" \
  "v_mfma_i32_16x16x64_i8 v[144:147], v[96:99], a[64:67], v[144:147]\n" \
  "ds_read_b64_tr_b8 v[120:121], v39 offset:32768\n" \
  "v_mfma_i32_16x16x64_i8 v[144:147], v[100:103], a[80:83], v[144:147]\n" \
  "ds_read_b64_tr_b8 v[122:123], v39 offset:33792\n" \
  "v_mfma_i32_16x16x64_i8 v[176:179], v[96:99], a[68:71], v[176:179]\n" \
  "v_mfma_i32_16x16x64_i8 v[176:179], v[100:103], a[84:87], v[176:179]\n" \
  "buffer_load_dwordx4 a[168:171], v44, s[28:31], s93 offen offset:2048 sc0 sc1\n" \
  "v_mfma_i32_16x16x64_i8 v[208:211], v[96:99], a[72:75], v[208:211]\n" \
  "ds_read_b64_tr_b8 v[124:125], v39 offset:40960\n" \
  "v_mfma_i32_16x16x64_i8 v[208:211], v[100:103], a[88:91], v[208:211]\n" \
  "ds_read_b64_tr_b8 v[126:127], v39 offset:41984\n" \
  "v_mfma_i32_16x16x64_i8 v[240:243], v[96:99], a[76:79], v[240:243]\n" \
  "v_readlane_b32 s50, v46, s49\n" \
  "v_mfma_i32_16x16x64_i8 v[240:243], v[100:103], a[92:95], v[240:243]\n" \
  "buffer_load_dwordx4 a[172:175], v44, s[28:31], s93 offen offset:3072 sc0 sc1\n" \
  "s_waitcnt lgkmcnt(8)\n" \
  "v_mfma_i32_16x16x64_i8 v[148:151], v[104:107], a[64:67], v[148:151]\n" \
  "ds_read_b64_tr_b8 v[96:97], v32 offset:49152\n" \
  "v_mfma_i32_16x16x64_i8 v[148:151], v[108:111], a[80:83], v[148:151]\n" \
  "ds_read_b64_tr_b8 v[98:99], v32 offset:50176\n" \
  "v_mfma_i32_16x16x64_i8 v[180:183], v[104:107], a[68:71], v[180:183]\n" \
  "v_mfma_i32_16x16x64_i8 v[180:183], v[108:111], a[84:87], v[180:183]\n" \
  "buffer_load_dwordx4 a[176:179], v91, s[28:31], s93 offen offset:0 sc0 sc1\n" \
  "v_mfma_i32_16x16x64_i8 v[212:215], v[104:107], a[72:75], v[212:215]\n" \
  "ds_read_b64_tr_b8 v[100:101], v32 offset:57344\n" \
  "v_mfma_i32_16x16x64_i8 v[212:215], v[108:111], a[88:91], v[212:215]\n" \
  "ds_read_b64_tr_b8 v[102:103], v32 offset:58368\n" \
  "v_mfma_i32_16x16x64_i8 v[244:247], v[104:107], a[76:79], v[244:247]\n" \
  "s_lshl_b32 s86, s50, 21\n" \
  "s_lshl_b32 s94, s50, 16\n" \
  "s_add_u32 s49, s49, 1\n" \
  "v_mfma_i32_16x16x64_i8 v[244:247], v[108:111], a[92:95], v[244:247]\n" \
  "s_waitcnt lgkmcnt(8)\n" \
  "v_mfma_i32_16x16x64_i8 v[152:155], v[112:115], a[64:67], v[152:155]\n" \
  "ds_read_b64_tr_b8 v[104:105], v33 offset:49152\n" \
  "v_mfma_i32_16x16x64_i8 v[152:155], v[116:119], a[80:83], v[152:155]\n" \
  "ds_read_b64_tr_b8 v[106:107], v33 offset:50176\n" \
  "v_mfma_i32_16x16x64_i8 v[184:187], v[112:115], a[68:71], v[184:187]\n" \
  "v_readlane_b32 s50, v46, s49\n" \
  "v_mfma_i32_16x16x64_i8 v[184:187], v[116:119], a[84:87], v[184:187]\n" \
  "buffer_load_dwordx4 a[180:183], v91, s[28:31], s93 offen offset:1024 sc0 sc1\n" \
  "v_mfma_i32_16x16x64_i8 v[216:219], v[112:115], a[72:75], v[216:219]\n" \
  "ds_read_b64_tr_b8 v[108:109], v33 offset:57344\n" \
  "v_mfma_i32_16x16x64_i8 v[216:219], v[116:119], a[88:91], v[216:219]\n" \
  "ds_read_b64_tr_b8 v[110:111], v33 offset:58368\n" \
  "v_mfma_i32_16x16x64_i8 v[248:251], v[112:115], a[76:79], v[248:251]\n" \
  "v_mfma_i32_16x16x64_i8 v[248:251], v[116:119], a[92:95], v[248:251]\n" \
  "buffer_load_dwordx4 a[184:187], v91, s[28:31], s93 offen offset:2048 sc0 sc1\n" \
  "s_waitcnt lgkmcnt(8)\n" \
  "v_mfma_i32_16x16x64_i8 v[156:159], v[120:123], a[64:67], v[156:159]\n" \
  "ds_read_b64_tr_b8 v[112:113], v80 offset:0\n" \
  "v_mfma_i32_16x16x64_i8 v[156:159], v[124:127], a[80:83], v[156:159]\n" \
  "ds_read_b64_tr_b8 v[114:115], v80 offset:1024\n" \
  "v_mfma_i32_16x16x64_i8 v[188:191], v[120:123], a[68:71], v[188:191]\n" \
  "s_lshl_b32 s87, s50, 21\n" \
  "s_lshl_b32 s95, s50, 16\n" \
  "s_add_u32 s49, s49, 1\n" \
  "v_mfma_i32_16x16x64_i8 v[188:191], v[124:127], a[84:87], v[188:191]\n" \
  "buffer_load_dwordx4 a[188:191], v91, s[28:31], s93 offen offset:3072 sc0 sc1\n" \
  "v_mfma_i32_16x16x64_i8 v[220:223], v[120:123], a[72:75], v[220:223]\n" \
  "ds_read_b64_tr_b8 v[116:117], v80 offset:8192\n" \
  "v_mfma_i32_16x16x64_i8 v[220:223], v[124:127], a[88:91], v[220:223]\n" \
  "ds_read_b64_tr_b8 v[118:119], v80 offset:9216\n" \
  "v_mfma_i32_16x16x64_i8 v[252:255], v[120:123], a[76:79], v[252:255]\n" \
  "v_mfma_i32_16x16x64_i8 v[252:255], v[124:127], a[92:95], v[252:255]\n" \
  "; ---- step 3\n" \
  "s_waitcnt vmcnt(16)\n" \
  "s_waitcnt lgkmcnt(8)\n" \
  "v_mfma_i32_16x16x64_i8 v[128:131], v[96:99], a[96:99], v[128:131]\n" \
  "ds_read_b64_tr_b8 v[120:121], v81 offset:0\n" \
  "v_mfma_i32_16x16x64_i8 v[128:131], v[100:103], a[112:115], v[128:131]\n" \
  "ds_read_b64_tr_b8 v[122:123], v81 offset:1024\n" \
  "buffer_load_dwordx4 a[192:195], v44, s[28:31], s94 offen offset:0 sc0 sc1\n" \
  "v_mfma_i32_16x16x64_i8 v[160:163], v[96:99], a[100:103], v[160:163]\n" \
  "v_mfma_i32_16x16x64_i8 v[160:163], v[100:103], a[116:119], v[160:163]\n" \
  "buffer_load_dwordx4 a[196:199], v44, s[28:31], s94 offen offset:1024 sc0 sc1\n" \
  "v_mfma_i32_16x16x64_i8 v[192:195], v[96:99], a[104:107], v[192:195]\n" \
  "ds_read_b64_tr_b8 v[124:125], v81 offset:8192\n" \
  "v_mfma_i32_16x16x64_i8 v[192:195], v[100:103], a[120:123], v[192:195]\n" \
  "ds_read_b64_tr_b8 v[126:127], v81 offset:9216\n" \
  "buffer_load_dwordx4 a[200:203], v44, s[28:31], s94 offen offset:2048 sc0 sc1\n" \
  "v_mfma_i32_16x16x64_i8 v[224:227], v[96:99], a[108:111], v[224:227]\n" \
  "v_mfma_i32_16x16x64_i8 v[224:227], v[100:103], a[124:127], v[224:227]\n" \
  "buffer_load_dwordx4 a[204:207], v44, s[28:31], s94 offen offset:3072 sc0 sc1\n" \
  "s_waitcnt lgkmcnt(8)\n" \
  "v_mfma_i32_16x16x64_i8 v[132:135], v[104:107], a[96:99], v[132:135]\n" \
  "ds_read_b64_tr_b8 v[96:97], v80 offset:16384\n" \
  "v_mfma_i32_16x16x64_i8 v[132:135], v[108:111], a[112:115], v[132:135]\n" \
  "ds_read_b64_tr_b8 v[98:99], v80 offset:17408\n" \
  "buffer_load_dwordx4 a[208:211], v91, s[28:31], s94 offen offset:0 sc0 sc1\n" \
  "v_mfma_i32_16x16x64_i8 v[164:167], v[104:107], a[100:103], v[164:167]\n" \
  "v_mfma_i32_16x16x64_i8 v[164:167], v[108:111], a[116:119], v[164:167]\n" \
  "buffer_load_dwordx4 a[212:215], v91, s[28:31], s94 offen offset:1024 sc0 sc1\n" \
  "v_mfma_i32_16x16x64_i8 v[196:199], v[104:107], a[104:107], v[196:199]\n" \
  "ds_read_b64_tr_b8 v[100:101], v80 offset:24576\n" \
  "v_mfma_i32_16x16x64_i8 v[196:199], v[108:111], a[120:123], v[196:199]\n" \
  "ds_read_b64_tr_b8 v[102:103], v80 offset:25600\n" \
  "buffer_load_dwordx4 a[216:219], v91, s[28:31], s94 offen offset:2048 sc0 sc1\n" \
  "v_mfma_i32_16x16x64_i8 v[228:231], v[104:107], a[108:111], v[228:231]\n" \
  "v_mfma_i32_16x16x64_i8 v[228:231], v[108:111], a[124:127], v[228:231]\n" \
  "buffer_load_dwordx4 a[220:223], v91, s[28:31], s94 offen offset:3072 sc0 sc1\n" \
  "s_waitcnt vmcnt(16)\n" \
  "s_waitcnt lgkmcnt(8)\n" \
  "v_mfma_i32_16x16x64_i8 v[128:131], v[112:115], a[128:131], v[128:131]\n" \
  "ds_read_b64_tr_b8 v[104:105], v81 offset:16384\n" \
  "v_mfma_i32_16x16x64_i8 v[128:131], v[116:119], a[144:147], v[128:131]\n" \
  "ds_read_b64_tr_b8 v[106:107], v81 offset:17408\n" \
  "buffer_load_dwordx4 a[224:227], v44, s[28:31], s95 offen offset:0 sc0 sc1\n" \
  "v_mfma_i32_16x16x64_i8 v[160:163], v[112:115], a[132:135], v[160:163]\n" \
  "v_mfma_i32_16x16x64_i8 v[160:163], v[116:119], a[148:151], v[160:163]\n" \
  "buffer_load_dwordx4 a[228:231], v44, s[28:31], s95 offen offset:1024 sc0 sc1\n" \
  "v_mfma_i32_16x16x64_i8 v[192:195], v[112:115], a[136:139], v[192:195]\n" \
  "ds_read_b64_tr_b8 v[108:109], v81 offset:24576\n" \
  "v_mfma_i32_16x16x64_i8 v[192:195], v[116:119], a[152:155], v[192:195]\n" \
  "ds_read_b64_tr_b8 v[110:111], v81 offset:25600\n" \
  "buffer_load_dwordx4 a[232:235], v44, s[28:31], s95 offen offset:2048 sc0 sc1\n" \
  "v_mfma_i32_16x16x64_i8 v[224:227], v[112:115], a[140:143], v[224:227]\n" \
  "v_mfma_i32_16x16x64_i8 v[224:227], v[116:119], a[156:159], v[224:227]\n" \
  "buffer_load_dwordx4 a[236:239], v44, s[28:31], s95 offen offset:3072 sc0 sc1\n" \
  "s_waitcnt lgkmcnt(8)\n" \
  "v_mfma_i32_16x16x64_i8 v[132:135], v[120:123], a[128:131], v[132:135]\n" \
  "ds_read_b64_tr_b8 v[112:113], v24 offset:0\n" \
  "v_mfma_i32_16x16x64_i8 v[132:135], v[124:127], a[144:147], v[132:135]\n" \
  "ds_read_b64_tr_b8 v[114:115], v24 offset:1024\n" \
  "buffer_load_dwordx4 a[240:243], v91, s[28:31], s95 offen offset:0 sc0 sc1\n" \
  "v_mfma_i32_16x16x64_i8 v[164:167], v[120:123], a[132:135], v[164:167]\n" \
  "v_mfma_i32_16x16x64_i8 v[164:167], v[124:127], a[148:151], v[164:167]\n" \
  "buffer_load_dwordx4 a[244:247], v91, s[28:31], s95 offen offset:1024 sc0 sc1\n" \
  "v_mfma_i32_16x16x64_i8 v[196:199], v[120:123], a[136:139], v[196:199]\n" \
  "ds_read_b64_tr_b8 v[116:117], v24 offset:8192\n" \
  "v_mfma_i32_16x16x64_i8 v[196:199], v[124:127], a[152:155], v[196:199]\n" \
  "ds_read_b64_tr_b8 v[118:119], v24 offset:9216\n" \
  "buffer_load_dwordx4 a[248:251], v91, s[28:31], s95 offen offset:2048 sc0 sc1\n" \
  "v_mfma_i32_16x16x64_i8 v[228:231], v[120:123], a[140:143], v[228:231]\n" \
  "v_mfma_i32_16x16x64_i8 v[228:231], v[124:127], a[156:159], v[228:231]\n" \
  "buffer_load_dwordx4 a[252:255], v91, s[28:31], s95 offen offset:3072 sc0 sc1\n" \
  "s_waitcnt vmcnt(16)\n" \
  "s_waitcnt lgkmcnt(8)\n" \
  "v_mfma_i32_16x16x64_i8 v[128:131], v[96:99], a[160:163], v[128:131]\n" \
  "ds_read_b64_tr_b8 v[120:121], v25 offset:0\n" \
  "v_mfma_i32_16x16x64_i8 v[128:131], v[100:103], a[176:179], v[128:131]\n" \
  "ds_read_b64_tr_b8 v[122:123], v25 offset:1024\n" \
  "v_mfma_i32_16x16x64_i8 v[160:163], v[96:99], a[164:167], v[160:163]\n" \
  "v_mfma_i32_16x16x64_i8 v[160:163], v[100:103], a[180:183], v[160:163]\n" \
  "v_mfma_i32_16x16x64_i8 v[192:195], v[96:99], a[168:171], v[192:195]\n" \
  "ds_read_b64_tr_b8 v[124:125], v25 offset:8192\n" \
  "v_mfma_i32_16x16x64_i8 v[192:195], v[100:103], a[184:187], v[192:195]\n" \
  "ds_read_b64_tr_b8 v[126:127], v25 offset:9216\n" \
  "v_mfma_i32_16x16x64_i8 v[224:227], v[96:99], a[172:175], v[224:227]\n" \
  "v_mfma_i32_16x16x64_i8 v[224:227], v[100:103], a[188:191], v[224:227]\n" \
  "s_waitcnt lgkmcnt(8)\n" \
  "v_mfma_i32_16x16x64_i8 v[132:135], v[104:107], a[160:163], v[132:135]\n" \
  "ds_read_b64_tr_b8 v[96:97], v24 offset:16384\n" \
  "v_mfma_i32_16x16x64_i8 v[132:135], v[108:111], a[176:179], v[132:135]\n" \
  "ds_read_b64_tr_b8 v[98:99], v24 offset:17408\n" \
  "v_mfma_i32_16x16x64_i8 v[164:167], v[104:107], a[164:167], v[164:167]\n" \
  "v_mfma_i32_16x16x64_i8 v[164:167], v[108:111], a[180:183], v[164:167]\n" \
  "v_mfma_i32_16x16x64_i8 v[196:199], v[104:107], a[168:171], v[196:199]\n" \
  "ds_read_b64_tr_b8 v[100:101], v24 offset:24576\n" \
  "v_mfma_i32_16x16x64_i8 v[196:199], v[108:111], a[184:187], v[196:199]\n" \
  "ds_read_b64_tr_b8 v[102:103], v24 offset:25600\n" \
  "v_mfma_i32_16x16x64_i8 v[228:231], v[104:107], a[172:175], v[228:231]\n" \
  "v_mfma_i32_16x16x64_i8 v[228:231], v[108:111], a[188:191], v[228:231]\n" \
  "s_waitcnt vmcnt(8)\n" \
  "s_waitcnt lgkmcnt(8)\n" \
  "v_mfma_i32_16x16x64_i8 v[128:131], v[112:115], a[192:195], v[128:131]\n" \
  "ds_read_b64_tr_b8 v[104:105], v25 offset:16384\n" \
  "v_mfma_i32_16x16x64_i8 v[128:131], v[116:119], a[208:211], v[128:131]\n" \
  "ds_read_b64_tr_b8 v[106:107], v25 offset:17408\n" \
  "v_mfma_i32_16x16x64_i8 v[160:163], v[112:115], a[196:199], v[160:163]\n" \
  "v_mfma_i32_16x16x64_i8 v[160:163], v[116:119], a[212:215], v[160:163]\n" \
  "v_mfma_i32_16x16x64_i8 v[192:195], v[112:115], a[200:203], v[192:195]\n" \
  "ds_read_b64_tr_b8 v[108:109], v25 offset:24576\n" \
  "v_mfma_i32_16x16x64_i8 v[192:195], v[116:119], a[216:219], v[192:195]\n" \
  "ds_read_b64_tr_b8 v[110:111], v25 offset:25600\n" \
  "v_mfma_i32_16x16x64_i8 v[224:227], v[112:115], a[204:207], v[224:227]\n" \
  "v_mfma_i32_16x16x64_i8 v[224:227], v[116:119], a[220:223], v[224:227]\n" \
  "s_waitcnt lgkmcnt(8)\n" \
  "v_mfma_i32_16x16x64_i8 v[132:135], v[120:123], a[192:195], v[132:135]\n" \
  "ds_read_b64_tr_b8 v[112:113], v34 offset:49152\n" \
  "v_mfma_i32_16x16x64_i8 v[132:135], v[124:127], a[208:211], v[132:135]\n" \
  "ds_read_b64_tr_b8 v[114:115], v34 offset:50176\n" \
  "v_mfma_i32_16x16x64_i8 v[164:167], v[120:123], a[196:199], v[164:167]\n" \
  "v_mfma_i32_16x16x64_i8 v[164:167], v[124:127], a[212:215], v[164:167]\n" \
  "v_mfma_i32_16x16x64_i8 v[196:199], v[120:123], a[200:203], v[196:199]\n" \
  "ds_read_b64_tr_b8 v[116:117], v34 offset:57344\n" \
  "v_mfma_i32_16x16x64_i8 v[196:199], v[124:127], a[216:219], v[196:199]\n" \
  "ds_read_b64_tr_b8 v[118:119], v34 offset:58368\n" \
  "v_mfma_i32_16x16x64_i8 v[228:231], v[120:123], a[204:207], v[228:231]\n" \
  "v_mfma_i32_16x16x64_i8 v[228:231], v[124:127], a[220:223], v[228:231]\n" \
  "; ---- step 4\n" \
  "s_waitcnt vmcnt(0)\n" \
  "s_waitcnt lgkmcnt(8)\n" \
  "v_mfma_i32_16x16x64_i8 v[128:131], v[96:99], a[224:227], v[128:131]\n" \
  "ds_read_b64_tr_b8 v[120:121], v35 offset:49152\n" \
  "v_mfma_i32_16x16x64_i8 v[128:131], v[100:103], a[240:243], v[128:131]\n" \
  "ds_read_b64_tr_b8 v[122:123], v35 offset:50176\n" \
  "v_mfma_i32_16x16x64_i8 v[160:163], v[96:99], a[228:231], v[160:163]\n" \
  "v_mfma_i32_16x16x64_i8 v[160:163], v[100:103], a[244:247], v[160:163]\n" \
  "v_mfma_i32_16x16x64_i8 v[192:195], v[96:99], a[232:235], v[192:195]\n" \
  "ds_read_b64_tr_b8 v[124:125], v35 offset:57344\n" \
  "v_mfma_i32_16x16x64_i8 v[192:195], v[100:103], a[248:251], v[192:195]\n" \
  "ds_read_b64_tr_b8 v[126:127], v35 offset:58368\n" \
  "v_mfma_i32_16x16x64_i8 v[224:227], v[96:99], a[236:239], v[224:227]\n" \
  "v_mfma_i32_16x16x64_i8 v[224:227], v[100:103], a[252:255], v[224:227]\n" \
  "s_waitcnt lgkmcnt(8)\n" \
  "v_mfma_i32_16x16x64_i8 v[132:135], v[104:107], a[224:227], v[132:135]\n" \
  "ds_read_b64_tr_b8 v[96:97], v82 offset:0\n" \
  "v_mfma_i32_16x16x64_i8 v[132:135], v[108:111], a[240:243], v[132:135]\n" \
  "ds_read_b64_tr_b8 v[98:99], v82 offset:1024\n" \
  "v_mfma_i32_16x16x64_i8 v[164:167], v[104:107], a[228:231], v[164:167]\n" \
  "v_mfma_i32_16x16x64_i8 v[164:167], v[108:111], a[244:247], v[164:167]\n" \
  "v_mfma_i32_16x16x64_i8 v[196:199], v[104:107], a[232:235], v[196:199]\n" \
  "ds_read_b64_tr_b8 v[100:101], v82 offset:8192\n" \
  "v_mfma_i32_16x16x64_i8 v[196:199], v[108:111], a[248:251], v[196:199]\n" \
  "ds_read_b64_tr_b8 v[102:103], v82 offset:9216\n" \
  "v_mfma_i32_16x16x64_i8 v[228:231], v[104:107], a[236:239], v[228:231]\n" \
  "v_mfma_i32_16x16x64_i8 v[228:231], v[108:111], a[252:255], v[228:231]\n" \
  "s_waitcnt lgkmcnt(8)\n" \
  "v_mfma_i32_16x16x64_i8 v[136:139], v[112:115], a[96:99], v[136:139]\n" \
  "ds_read_b64_tr_b8 v[104:105], v83 offset:0\n" \
  "v_mfma_i32_16x16x64_i8 v[136:139], v[116:119], a[112:115], v[136:139]\n" \
  "ds_read_b64_tr_b8 v[106:107], v83 offset:1024\n" \
  "v_mfma_i32_16x16x64_i8 v[168:171], v[112:115], a[100:103], v[168:171]\n" \
  "v_mfma_i32_16x16x64_i8 v[168:171], v[116:119], a[116:119], v[168:171]\n" \
  "v_mfma_i32_16x16x64_i8 v[200:203], v[112:115], a[104:107], v[200:203]\n" \
  "ds_read_b64_tr_b8 v[108:109], v83 offset:8192\n" \
  "v_mfma_i32_16x16x64_i8 v[200:203], v[116:119], a[120:123], v[200:203]\n" \
  "ds_read_b64_tr_b8 v[110:111], v83 offset:9216\n" \
  "v_mfma_i32_16x16x64_i8 v[232:235], v[112:115], a[108:111], v[232:235]\n" \
  "v_mfma_i32_16x16x64_i8 v[232:235], v[116:119], a[124:127], v[232:235]\n" \
  "s_waitcnt lgkmcnt(8)\n" \
  "v_mfma_i32_16x16x64_i8 v[140:143], v[120:123], a[96:99], v[140:143]\n" \
  "ds_read_b64_tr_b8 v[112:113], v82 offset:16384\n" \
  "v_cvt_f32_i32 v128, v128\n" \
  "v_mfma_i32_16x16x64_i8 v[140:143], v[124:127], a[112:115], v[140:143]\n" \
  "ds_read_b64_tr_b8 v[114:115], v82 offset:17408\n" \
  "v_cvt_f32_i32 v129, v129\n" \
  "v_mfma_i32_16x16x64_i8 v[172:175], v[120:123], a[100:103], v[172:175]\n" \
  "v_cvt_f32_i32 v130, v130\n" \
  "v_mfma_i32_16x16x64_i8 v[172:175], v[124:127], a[116:119], v[172:175]\n" \
  "v_cvt_f32_i32 v131, v131\n" \
  "v_mfma_i32_16x16x64_i8 v[204:207], v[120:123], a[104:107], v[204:207]\n" \
  "ds_read_b64_tr_b8 v[116:117], v82 offset:24576\n" \
  "v_mfma_i32_16x16x64_i8 v[204:207], v[124:127], a[120:123], v[204:207]\n" \
  "ds_read_b64_tr_b8 v[118:119], v82 offset:25600\n" \
  "v_pk_mul_f32 v[128:129], v[128:129], v[88:89]\n" \
  "v_mfma_i32_16x16x64_i8 v[236:239], v[120:123], a[108:111], v[236:239]\n" \
  "v_pk_mul_f32 v[130:131], v[130:131], v[88:89]\n" \
  "v_mfma_i32_16x16x64_i8 v[236:239], v[124:127], a[124:127], v[236:239]\n" \
  "buffer_store_dwordx4 v[128:131], v47, s[20:23], s76 offen offset:0 sc0 sc1\n" \
  "s_waitcnt lgkmcnt(8)\n" \
  "v_mfma_i32_16x16x64_i8 v[136:139], v[96:99], a[128:131], v[136:139]\n" \
  "ds_read_b64_tr_b8 v[120:121], v83 offset:16384\n" \
  "v_mfma_i32_16x16x64_i8 v[136:139], v[100:103], a[144:147], v[136:139]\n" \
  "ds_read_b64_tr_b8 v[122:123], v83 offset:17408\n" \
  "v_cvt_f32_i32 v132, v132\n" \
  "v_mfma_i32_16x16x64_i8 v[168:171], v[96:99], a[132:135], v[168:171]\n" \
  "v_cvt_f32_i32 v133, v133\n" \
  "v_mfma_i32_16x16x64_i8 v[168:171], v[100:103], a[148:151], v[168:171]\n" \
  "v_cvt_f32_i32 v134, v134\n" \
  "v_mfma_i32_16x16x64_i8 v[200:203], v[96:99], a[136:139], v[200:203]\n" \
  "ds_read_b64_tr_b8 v[124:125], v83 offset:24576\n" \
  "v_cvt_f32_i32 v135, v135\n" \
  "v_mfma_i32_16x16x64_i8 v[200:203], v[100:103], a[152:155], v[200:203]\n" \
  "ds_read_b64_tr_b8 v[126:127], v83 offset:25600\n" \
  "v_mfma_i32_16x16x64_i8 v[232:235], v[96:99], a[140:143], v[232:235]\n" \
  "v_pk_mul_f32 v[132:133], v[132:133], v[88:89]\n" \
  "v_mfma_i32_16x16x64_i8 v[232:235], v[100:103], a[156:159], v[232:235]\n" \
  "v_pk_mul_f32 v[134:135], v[134:135], v[88:89]\n" \
  "s_waitcnt lgkmcnt(8)\n" \
  "v_mfma_i32_16x16x64_i8 v[140:143], v[104:107], a[128:131], v[140:143]\n" \
  "ds_read_b64_tr_b8 v[96:97], v26 offset:0\n" \
  "buffer_store_dwordx4 v[132:135], v47, s[20:23], s76 offen offset:64 sc0 sc1\n" \
  "v_mfma_i32_16x16x64_i8 v[140:143], v[108:111], a[144:147], v[140:143]\n" \
  "ds_read_b64_tr_b8 v[98:99], v26 offset:1024\n" \
  "v_mfma_i32_16x16x64_i8 v[172:175], v[104:107], a[132:135], v[172:175]\n" \
  "v_cvt_f32_i32 v160, v160\n" \
  "v_mfma_i32_16x16x64_i8 v[172:175], v[108:111], a[148:151], v[172:175]\n" \
  "v_cvt_f32_i32 v161, v161\n" \
  "v_mfma_i32_16x16x64_i8 v[204:207], v[104:107], a[136:139], v[204:207]\n" \
  "ds_read_b64_tr_b8 v[100:101], v26 offset:8192\n" \
  "v_cvt_f32_i32 v162, v162\n" \
  "v_mfma_i32_16x16x64_i8 v[204:207], v[108:111], a[152:155], v[204:207]\n" \
  "ds_read_b64_tr_b8 v[102:103], v26 offset:9216\n" \
  "v_cvt_f32_i32 v163, v163\n" \
  "v_mfma_i32_16x16x64_i8 v[236:239], v[104:107], a[140:143], v[236:239]\n" \
  "v_mfma_i32_16x16x64_i8 v[236:239], v[108:111], a[156:159], v[236:239]\n" \
  "v_pk_mul_f32 v[160:161], v[160:161], v[88:89]\n" \
  "s_waitcnt lgkmcnt(8)\n" \
  "v_mfma_i32_16x16x64_i8 v[136:139], v[112:115], a[160:163], v[136:139]\n" \
  "ds_read_b64_tr_b8 v[104:105], v27 offset:0\n" \
  "v_pk_mul_f32 v[162:163], v[162:163], v[88:89]\n" \
  "v_mfma_i32_16x16x64_i8 v[136:139], v[116:119], a[176:179], v[136:139]\n" \
  "ds_read_b64_tr_b8 v[106:107], v27 offset:1024\n" \
  "buffer_store_dwordx4 v[160:163], v47, s[20:23], s77 offen offset:0 sc0 sc1\n" \
  "v_mfma_i32_16x16x64_i8 v[168:171], v[112:115], a[164:167], v[168:171]\n" \
  "v_mfma_i32_16x16x64_i8 v[168:171], v[116:119], a[180:183], v[168:171]\n" \
  "v_cvt_f32_i32 v164, v164\n" \
  "v_mfma_i32_16x16x64_i8 v[200:203], v[112:115], a[168:171], v[200:203]\n" \
  "ds_read_b64_tr_b8 v[108:109], v27 offset:8192\n" \
  "v_cvt_f32_i32 v165, v165\n" \
  "v_mfma_i32_16x16x64_i8 v[200:203], v[116:119], a[184:187], v[200:203]\n" \
  "ds_read_b64_tr_b8 v[110:111], v27 offset:9216\n" \
  "v_cvt_f32_i32 v166, v166\n" \
  "v_mfma_i32_16x16x64_i8 v[232:235], v[112:115], a[172:175], v[232:235]\n" \
  "v_cvt_f32_i32 v167, v167\n" \
  "v_mfma_i32_16x16x64_i8 v[232:235], v[116:119], a[188:191], v[232:235]\n" \
  "s_waitcnt lgkmcnt(8)\n" \
  "v_mfma_i32_16x16x64_i8 v[140:143], v[120:123], a[160:163], v[140:143]\n" \
  "ds_read_b64_tr_b8 v[112:113], v26 offset:16384\n" \
  "v_pk_mul_f32 v[164:165], v[164:165], v[88:89]\n" \
  "v_mfma_i32_16x16x64_i8 v[140:143], v[124:127], a[176:179], v[140:143]\n" \
  "ds_read_b64_tr_b8 v[114:115], v26 offset:17408\n" \
  "v_pk_mul_f32 v[166:167], v[166:167], v[88:89]\n" \
  "v_mfma_i32_16x16x64_i8 v[172:175], v[120:123], a[164:167], v[172:175]\n" \
  "buffer_store_dwordx4 v[164:167], v47, s[20:23], s77 offen offset:64 sc0 sc1\n" \
  "v_mfma_i32_16x16x64_i8 v[172:175], v[124:127], a[180:183], v[172:175]\n" \
  "v_mfma_i32_16x16x64_i8 v[204:207], v[120:123], a[168:171], v[204:207]\n" \
  "ds_read_b64_tr_b8 v[116:117], v26 offset:24576\n" \
  "v_cvt_f32_i32 v192, v192\n" \
  "v_mfma_i32_16x16x64_i8 v[204:207], v[124:127], a[184:187], v[204:207]\n" \
  "ds_read_b64_tr_b8 v[118:119], v26 offset:25600\n" \
  "v_cvt_f32_i32 v193, v193\n" \
  "v_mfma_i32_16x16x64_i8 v[236:239], v[120:123], a[172:175], v[236:239]\n" \
  "v_cvt_f32_i32 v194, v194\n" \
  "v_mfma_i32_16x16x64_i8 v[236:239], v[124:127], a[188:191], v[236:239]\n" \
  "v_cvt_f32_i32 v195, v195\n" \
  "; ---- step 5\n" \
  "s_waitcnt lgkmcnt(8)\n" \
  "v_mfma_i32_16x16x64_i8 v[136:139], v[96:99], a[192:195], v[136:139]\n" \
  "ds_read_b64_tr_b8 v[120:121], v27 offset:16384\n" \
  "v_mfma_i32_16x16x64_i8 v[136:139], v[100:103], a[208:211], v[136:139]\n" \
  "ds_read_b64_tr_b8 v[122:123], v27 offset:17408\n" \
  "v_pk_mul_f32 v[192:193], v[192:193], v[88:89]\n" \
  "v_mfma_i32_16x16x64_i8 v[168:171], v[96:99], a[196:199], v[168:171]\n" \
  "v_pk_mul_f32 v[194:195], v[194:195], v[88:89]\n" \
  "v_mfma_i32_16x16x64_i8 v[168:171], v[100:103], a[212:215], v[168:171]\n" \
  "buffer_store_dwordx4 v[192:195], v47, s[20:23], s78 offen offset:0 sc0 sc1\n" \
  "v_mfma_i32_16x16x64_i8 v[200:203], v[96:99], a[200:203], v[200:203]\n" \
  "ds_read_b64_tr_b8 v[124:125], v27 offset:24576\n" \
  "v_mfma_i32_16x16x64_i8 v[200:203], v[100:103], a[216:219], v[200:203]\n" \
  "ds_read_b64_tr_b8 v[126:127], v27 offset:25600\n" \
  "v_cvt_f32_i32 v196, v196\n" \
  "v_mfma_i32_16x16x64_i8 v[232:235], v[96:99], a[204:207], v[232:235]\n" \
  "v_cvt_f32_i32 v197, v197\n" \
  "v_mfma_i32_16x16x64_i8 v[232:235], v[100:103], a[220:223], v[232:235]\n" \
  "v_cvt_f32_i32 v198, v198\n" \
  "s_waitcnt lgkmcnt(8)\n" \
  "v_mfma_i32_16x16x64_i8 v[140:143], v[104:107], a[192:195], v[140:143]\n" \
  "ds_read_b64_tr_b8 v[96:97], v36 offset:49152\n" \
  "v_cvt_f32_i32 v199, v199\n" \
  "v_mfma_i32_16x16x64_i8 v[140:143], v[108:111], a[208:211], v[140:143]\n" \
  "ds_read_b64_tr_b8 v[98:99], v36 offset:50176\n" \
  "v_mfma_i32_16x16x64_i8 v[172:175], v[104:107], a[196:199], v[172:175]\n" \
  "v_pk_mul_f32 v[196:197], v[196:197], v[88:89]\n" \
  "v_mfma_i32_16x16x64_i8 v[172:175], v[108:111], a[212:215], v[172:175]\n" \
  "v_pk_mul_f32 v[198:199], v[198:199], v[88:89]\n" \
  "v_mfma_i32_16x16x64_i8 v[204:207], v[104:107], a[200:203], v[204:207]\n" \
  "ds_read_b64_tr_b8 v[100:101], v36 offset:57344\n" \
  "buffer_store_dwordx4 v[196:199], v47, s[20:23], s78 offen offset:64 sc0 sc1\n" \
  "v_mfma_i32_16x16x64_i8 v[204:207], v[108:111], a[216:219], v[204:207]\n" \
  "ds_read_b64_tr_b8 v[102:103], v36 offset:58368\n" \
  "v_mfma_i32_16x16x64_i8 v[236:239], v[104:107], a[204:207], v[236:239]\n" \
  "v_cvt_f32_i32 v224, v224\n" \
  "v_mfma_i32_16x16x64_i8 v[236:239], v[108:111], a[220:223], v[236:239]\n" \
  "v_cvt_f32_i32 v225, v225\n" \
  "s_waitcnt lgkmcnt(8)\n" \
  "v_mfma_i32_16x16x64_i8 v[136:139], v[112:115], a[224:227], v[136:139]\n" \
  "ds_read_b64_tr_b8 v[104:105], v37 offset:49152\n" \
  "v_cvt_f32_i32 v226, v226\n" \
  "v_mfma_i32_16x16x64_i8 v[136:139], v[116:119], a[240:243], v[136:139]\n" \
  "ds_read_b64_tr_b8 v[106:107], v37 offset:50176\n" \
  "v_cvt_f32_i32 v227, v227\n" \
  "v_mfma_i32_16x16x64_i8 v[168:171], v[112:115], a[228:231], v[168:171]\n" \
  "v_mfma_i32_16x16x64_i8 v[168:171], v[116:119], a[244:247], v[168:171]\n" \
  "v_pk_mul_f32 v[224:225], v[224:225], v[88:89]\n" \
  "v_mfma_i32_16x16x64_i8 v[200:203], v[112:115], a[232:235], v[200:203]\n" \
  "ds_read_b64_tr_b8 v[108:109], v37 offset:57344\n" \
  "v_pk_mul_f32 v[226:227], v[226:227], v[88:89]\n" \
  "v_mfma_i32_16x16x64_i8 v[200:203], v[116:119], a[248:251], v[200:203]\n" \
  "ds_read_b64_tr_b8 v[110:111], v37 offset:58368\n" \
  "buffer_store_dwordx4 v[224:227], v47, s[20:23], s79 offen offset:0 sc0 sc1\n" \
  "v_mfma_i32_16x16x64_i8 v[232:235], v[112:115], a[236:239], v[232:235]\n" \
  "v_mfma_i32_16x16x64_i8 v[232:235], v[116:119], a[252:255], v[232:235]\n" \
  "v_cvt_f32_i32 v228, v228\n" \
  "s_waitcnt lgkmcnt(8)\n" \
  "v_mfma_i32_16x16x64_i8 v[140:143], v[120:123], a[224:227], v[140:143]\n" \
  "ds_read_b64_tr_b8 v[112:113], v84 offset:0\n" \
  "v_cvt_f32_i32 v229, v229\n" \
  "v_mfma_i32_16x16x64_i8 v[140:143], v[124:127], a[240:243], v[140:143]\n" \
  "ds_read_b64_tr_b8 v[114:115], v84 offset:1024\n" \
  "v_cvt_f32_i32 v230, v230\n" \
  "v_mfma_i32_16x16x64_i8 v[172:175], v[120:123], a[228:231], v[172:175]\n" \
  "v_cvt_f32_i32 v231, v231\n" \
  "v_mfma_i32_16x16x64_i8 v[172:175], v[124:127], a[244:247], v[172:175]\n" \
  "v_mfma_i32_16x16x64_i8 v[204:207], v[120:123], a[232:235], v[204:207]\n" \
  "ds_read_b64_tr_b8 v[116:117], v84 offset:8192\n" \
  "v_pk_mul_f32 v[228:229], v[228:229], v[88:89]\n" \
  "v_mfma_i32_16x16x64_i8 v[204:207], v[124:127], a[248:251], v[204:207]\n" \
  "ds_read_b64_tr_b8 v[118:119], v84 offset:9216\n" \
  "v_pk_mul_f32 v[230:231], v[230:231], v[88:89]\n" \
  "v_mfma_i32_16x16x64_i8 v[236:239], v[120:123], a[236:239], v[236:239]\n" \
  "buffer_store_dwordx4 v[228:231], v47, s[20:23], s79 offen offset:64 sc0 sc1\n" \
  "v_mfma_i32_16x16x64_i8 v[236:239], v[124:127], a[252:255], v[236:239]\n" \
  "s_waitcnt lgkmcnt(8)\n" \
  "v_mfma_i32_16x16x64_i8 v[144:147], v[96:99], a[96:99], v[144:147]\n" \
  "ds_read_b64_tr_b8 v[120:121], v85 offset:0\n" \
  "v_mfma_i32_16x16x64_i8 v[144:147], v[100:103], a[112:115], v[144:147]\n" \
  "ds_read_b64_tr_b8 v[122:123], v85 offset:1024\n" \
  "v_mfma_i32_16x16x64_i8 v[176:179], v[96:99], a[100:103], v[176:179]\n" \
  "v_mfma_i32_16x16x64_i8 v[176:179], v[100:103], a[116:119], v[176:179]\n" \
  "v_mfma_i32_16x16x64_i8 v[208:211], v[96:99], a[104:107], v[208:211]\n" \
  "ds_read_b64_tr_b8 v[124:125], v85 offset:8192\n" \
  "v_mfma_i32_16x16x64_i8 v[208:211], v[100:103], a[120:123], v[208:211]\n" \
  "ds_read_b64_tr_b8 v[126:127], v85 offset:9216\n" \
  "v_mfma_i32_16x16x64_i8 v[240:243], v[96:99], a[108:111], v[240:243]\n" \
  "v_mfma_i32_16x16x64_i8 v[240:243], v[100:103], a[124:127], v[240:243]\n" \
  "s_waitcnt lgkmcnt(8)\n" \
  "v_mfma_i32_16x16x64_i8 v[148:151], v[104:107], a[96:99], v[148:151]\n" \
  "ds_read_b64_tr_b8 v[96:97], v84 offset:16384\n" \
  "v_cvt_f32_i32 v136, v136\n" \
  "v_mfma_i32_16x16x64_i8 v[148:151], v[108:111], a[112:115], v[148:151]\n" \
  "ds_read_b64_tr_b8 v[98:99], v84 offset:17408\n" \
  "v_cvt_f32_i32 v137, v137\n" \
  "v_mfma_i32_16x16x64_i8 v[180:183], v[104:107], a[100:103], v[180:183]\n" \
  "v_cvt_f32_i32 v138, v138\n" \
  "v_mfma_i32_16x16x64_i8 v[180:183], v[108:111], a[116:119], v[180:183]\n" \
  "v_cvt_f32_i32 v139, v139\n" \
  "v_mfma_i32_16x16x64_i8 v[212:215], v[104:107], a[104:107], v[212:215]\n" \
  "ds_read_b64_tr_b8 v[100:101], v84 offset:24576\n" \
  "v_mfma_i32_16x16x64_i8 v[212:215], v[108:111], a[120:123], v[212:215]\n" \
  "ds_read_b64_tr_b8 v[102:103], v84 offset:25600\n" \
  "v_pk_mul_f32 v[136:137], v[136:137], v[88:89]\n" \
  "v_mfma_i32_16x16x64_i8 v[244:247], v[104:107], a[108:111], v[244:247]\n" \
  "v_pk_mul_f32 v[138:139], v[138:139], v[88:89]\n" \
  "v_mfma_i32_16x16x64_i8 v[244:247], v[108:111], a[124:127], v[244:247]\n" \
  "buffer_store_dwordx4 v[136:139], v47, s[20:23], s76 offen offset:128 sc0 sc1\n" \
  "s_waitcnt lgkmcnt(8)\n" \
  "v_mfma_i32_16x16x64_i8 v[144:147], v[112:115], a[128:131], v[144:147]\n" \
  "ds_read_b64_tr_b8 v[104:105], v85 offset:16384\n" \
  "v_mfma_i32_16x16x64_i8 v[144:147], v[116:119], a[144:147], v[144:147]\n" \
  "ds_read_b64_tr_b8 v[106:107], v85 offset:17408\n" \
  "v_cvt_f32_i32 v140, v140\n" \
  "v_mfma_i32_16x16x64_i8 v[176:179], v[112:115], a[132:135], v[176:179]\n" \
  "v_cvt_f32_i32 v141, v141\n" \
  "v_mfma_i32_16x16x64_i8 v[176:179], v[116:119], a[148:151], v[176:179]\n" \
  "v_cvt_f32_i32 v142, v142\n" \
  "v_mfma_i32_16x16x64_i8 v[208:211], v[112:115], a[136:139], v[208:211]\n" \
  "ds_read_b64_tr_b8 v[108:109], v85 offset:24576\n" \
  "v_cvt_f32_i32 v143, v143\n" \
  "v_mfma_i32_16x16x64_i8 v[208:211], v[116:119], a[152:155], v[208:211]\n" \
  "ds_read_b64_tr_b8 v[110:111], v85 offset:25600\n" \
  "v_mfma_i32_16x16x64_i8 v[240:243], v[112:115], a[140:143], v[240:243]\n" \
  "v_pk_mul_f32 v[140:141], v[140:141], v[88:89]\n" \
  "v_mfma_i32_16x16x64_i8 v[240:243], v[116:119], a[156:159], v[240:243]\n" \
  "v_pk_mul_f32 v[142:143], v[142:143], v[88:89]\n" \
  "s_waitcnt lgkmcnt(8)\n" \
  "v_mfma_i32_16x16x64_i8 v[148:151], v[120:123], a[128:131], v[148:151]\n" \
  "ds_read_b64_tr_b8 v[112:113], v28 offset:0\n" \
  "buffer_store_dwordx4 v[140:143], v47, s[20:23], s76 offen offset:192 sc0 sc1\n" \
  "v_mfma_i32_16x16x64_i8 v[148:151], v[124:127], a[144:147], v[148:151]\n" \
  "ds_read_b64_tr_b8 v[114:115], v28 offset:1024\n" \
  "v_mfma_i32_16x16x64_i8 v[180:183], v[120:123], a[132:135], v[180:183]\n" \
  "v_cvt_f32_i32 v168, v168\n" \
  "v_mfma_i32_16x16x64_i8 v[180:183], v[124:127], a[148:151], v[180:183]\n" \
  "v_cvt_f32_i32 v169, v169\n" \
  "v_mfma_i32_16x16x64_i8 v[212:215], v[120:123], a[136:139], v[212:215]\n" \
  "ds_read_b64_tr_b8 v[116:117], v28 offset:8192\n" \
  "v_cvt_f32_i32 v170, v170\n" \
  "v_mfma_i32_16x16x64_i8 v[212:215], v[124:127], a[152:155], v[212:215]\n" \
  "ds_read_b64_tr_b8 v[118:119], v28 offset:9216\n" \
  "v_cvt_f32_i32 v171, v171\n" \
  "v_mfma_i32_16x16x64_i8 v[244:247], v[120:123], a[140:143], v[244:247]\n" \
  "v_mfma_i32_16x16x64_i8 v[244:247], v[124:127], a[156:159], v[244:247]\n" \
  "v_pk_mul_f32 v[168:169], v[168:169], v[88:89]\n" \
  "; ---- step 6\n" \
  "s_waitcnt lgkmcnt(8)\n" \
  "v_mfma_i32_16x16x64_i8 v[144:147], v[96:99], a[160:163], v[144:147]\n" \
  "ds_read_b64_tr_b8 v[120:121], v29 offset:0\n" \
  "v_pk_mul_f32 v[170:171], v[170:171], v[88:89]\n" \
  "v_mfma_i32_16x16x64_i8 v[144:147], v[100:103], a[176:179], v[144:147]\n" \
  "ds_read_b64_tr_b8 v[122:123], v29 offset:1024\n" \
  "buffer_store_dwordx4 v[168:171], v47, s[20:23], s77 offen offset:128 sc0 sc1\n" \
  "v_mfma_i32_16x16x64_i8 v[176:179], v[96:99], a[164:167], v[176:179]\n" \
  "v_mfma_i32_16x16x64_i8 v[176:179], v[100:103], a[180:183], v[176:179]\n" \
  "v_cvt_f32_i32 v172, v172\n" \
  "v_mfma_i32_16x16x64_i8 v[208:211], v[96:99], a[168:171], v[208:211]\n" \
  "ds_read_b64_tr_b8 v[124:125], v29 offset:8192\n" \
  "v_cvt_f32_i32 v173, v173\n" \
  "v_mfma_i32_16x16x64_i8 v[208:211], v[100:103], a[184:187], v[208:211]\n" \
  "ds_read_b64_tr_b8 v[126:127], v29 offset:9216\n" \
  "v_cvt_f32_i32 v174, v174\n" \
  "v_mfma_i32_16x16x64_i8 v[240:243], v[96:99], a[172:175], v[240:243]\n" \
  "v_cvt_f32_i32 v175, v175\n" \
  "v_mfma_i32_16x16x64_i8 v[240:243], v[100:103], a[188:191], v[240:243]\n" \
  "s_waitcnt lgkmcnt(8)\n" \
  "v_mfma_i32_16x16x64_i8 v[148:151], v[104:107], a[160:163], v[148:151]\n" \
  "ds_read_b64_tr_b8 v[96:97], v28 offset:16384\n" \
  "v_pk_mul_f32 v[172:173], v[172:173], v[88:89]\n" \
  "v_mfma_i32_16x16x64_i8 v[148:151], v[108:111], a[176:179], v[148:151]\n" \
  "ds_read_b64_tr_b8 v[98:99], v28 offset:17408\n" \
  "v_pk_mul_f32 v[174:175], v[174:175], v[88:89]\n" \
  "v_mfma_i32_16x16x64_i8 v[180:183], v[104:107], a[164:167], v[180:183]\n" \
  "buffer_store_dwordx4 v[172:175], v47, s[20:23], s77 offen offset:192 sc0 sc1\n" \
  "v_mfma_i32_16x16x64_i8 v[180:183], v[108:111], a[180:183], v[180:183]\n" \
  "v_mfma_i32_16x16x64_i8 v[212:215], v[104:107], a[168:171], v[212:215]\n" \
  "ds_read_b64_tr_b8 v[100:101], v28 offset:24576\n" \
  "v_cvt_f32_i32 v200, v200\n" \
  "v_mfma_i32_16x16x64_i8 v[212:215], v[108:111], a[184:187], v[212:215]\n" \
  "ds_read_b64_tr_b8 v[102:103], v28 offset:25600\n" \
  "v_cvt_f32_i32 v201, v201\n" \
  "v_mfma_i32_16x16x64_i8 v[244:247], v[104:107], a[172:175], v[244:247]\n" \
  "v_cvt_f32_i32 v202, v202\n" \
  "v_mfma_i32_16x16x64_i8 v[244:247], v[108:111], a[188:191], v[244:247]\n" \
  "v_cvt_f32_i32 v203, v203\n" \
  "s_waitcnt lgkmcnt(8)\n" \
  "v_mfma_i32_16x16x64_i8 v[144:147], v[112:115], a[192:195], v[144:147]\n" \
  "ds_read_b64_tr_b8 v[104:105], v29 offset:16384\n" \
  "v_mfma_i32_16x16x64_i8 v[144:147], v[116:119], a[208:211], v[144:147]\n" \
  "ds_read_b64_tr_b8 v[106:107], v29 offset:17408\n" \
  "v_pk_mul_f32 v[200:201], v[200:201], v[88:89]\n" \
  "v_mfma_i32_16x16x64_i8 v[176:179], v[112:115], a[196:199], v[176:179]\n" \
  "v_pk_mul_f32 v[202:203], v[202:203], v[88:89]\n" \
  "v_mfma_i32_16x16x64_i8 v[176:179], v[116:119], a[212:215], v[176:179]\n" \
  "buffer_store_dwordx4 v[200:203], v47, s[20:23], s78 offen offset:128 sc0 sc1\n" \
  "v_mfma_i32_16x16x64_i8 v[208:211], v[112:115], a[200:203], v[208:211]\n" \
  "ds_read_b64_tr_b8 v[108:109], v29 offset:24576\n" \
  "v_mfma_i32_16x16x64_i8 v[208:211], v[116:119], a[216:219], v[208:211]\n" \
  "ds_read_b64_tr_b8 v[110:111], v29 offset:25600\n" \
  "v_cvt_f32_i32 v204, v204\n" \
  "v_mfma_i32_16x16x64_i8 v[240:243], v[112:115], a[204:207], v[240:243]\n" \
  "v_cvt_f32_i32 v205, v205\n" \
  "v_mfma_i32_16x16x64_i8 v[240:243], v[116:119], a[220:223], v[240:243]\n" \
  "v_cvt_f32_i32 v206, v206\n" \
  "s_waitcnt lgkmcnt(8)\n" \
  "v_mfma_i32_16x16x64_i8 v[148:151], v[120:123], a[192:195], v[148:151]\n" \
  "ds_read_b64_tr_b8 v[112:113], v38 offset:49152\n" \
  "v_cvt_f32_i32 v207, v207\n" \
  "v_mfma_i32_16x16x64_i8 v[148:151], v[124:127], a[208:211], v[148:151]\n" \
  "ds_read_b64_tr_b8 v[114:115], v38 offset:50176\n" \
  "v_mfma_i32_16x16x64_i8 v[180:183], v[120:123], a[196:199], v[180:183]\n" \
  "v_pk_mul_f32 v[204:205], v[204:205], v[88:89]\n" \
  "v_mfma_i32_16x16x64_i8 v[180:183], v[124:127], a[212:215], v[180:183]\n" \
  "v_pk_mul_f32 v[206:207], v[206:207], v[88:89]\n" \
  "v_mfma_i32_16x16x64_i8 v[212:215], v[120:123], a[200:203], v[212:215]\n" \
  "ds_read_b64_tr_b8 v[116:117], v38 offset:57344\n" \
  "buffer_store_dwordx4 v[204:207], v47, s[20:23], s78 offen offset:192 sc0 sc1\n" \
  "v_mfma_i32_16x16x64_i8 v[212:215], v[124:127], a[216:219], v[212:215]\n" \
  "ds_read_b64_tr_b8 v[118:119], v38 offset:58368\n" \
  "v_mfma_i32_16x16x64_i8 v[244:247], v[120:123], a[204:207], v[244:247]\n" \
  "v_cvt_f32_i32 v232, v232\n" \
  "v_mfma_i32_16x16x64_i8 v[244:247], v[124:127], a[220:223], v[244:247]\n" \
  "v_cvt_f32_i32 v233, v233\n" \
  "s_waitcnt lgkmcnt(8)\n" \
  "v_mfma_i32_16x16x64_i8 v[144:147], v[96:99], a[224:227], v[144:147]\n" \
  "ds_read_b64_tr_b8 v[120:121], v39 offset:49152\n" \
  "v_cvt_f32_i32 v234, v234\n" \
  "v_mfma_i32_16x16x64_i8 v[144:147], v[100:103], a[240:243], v[144:147]\n" \
  "ds_read_b64_tr_b8 v[122:123], v39 offset:50176\n" \
  "v_cvt_f32_i32 v235, v235\n" \
  "v_mfma_i32_16x16x64_i8 v[176:179], v[96:99], a[228:231], v[176:179]\n" \
  "v_mfma_i32_16x16x64_i8 v[176:179], v[100:103], a[244:247], v[176:179]\n" \
  "v_pk_mul_f32 v[232:233], v[232:233], v[88:89]\n" \
  "v_mfma_i32_16x16x64_i8 v[208:211], v[96:99], a[232:235], v[208:211]\n" \
  "ds_read_b64_tr_b8 v[124:125], v39 offset:57344\n" \
  "v_pk_mul_f32 v[234:235], v[234:235], v[88:89]\n" \
  "v_mfma_i32_16x16x64_i8 v[208:211], v[100:103], a[248:251], v[208:211]\n" \
  "ds_read_b64_tr_b8 v[126:127], v39 offset:58368\n" \
  "buffer_store_dwordx4 v[232:235], v47, s[20:23], s79 offen offset:128 sc0 sc1\n" \
  "v_mfma_i32_16x16x64_i8 v[240:243], v[96:99], a[236:239], v[240:243]\n" \
  "v_mfma_i32_16x16x64_i8 v[240:243], v[100:103], a[252:255], v[240:243]\n" \
  "v_cvt_f32_i32 v236, v236\n" \
  "s_waitcnt lgkmcnt(8)\n" \
  "v_mfma_i32_16x16x64_i8 v[148:151], v[104:107], a[224:227], v[148:151]\n" \
  "ds_read_b64_tr_b8 v[96:97], v86 offset:0\n" \
  "v_cvt_f32_i32 v237, v237\n" \
  "v_mfma_i32_16x16x64_i8 v[148:151], v[108:111], a[240:243], v[148:151]\n" \
  "ds_read_b64_tr_b8 v[98:99], v86 offset:1024\n" \
  "v_cvt_f32_i32 v238, v238\n" \
  "v_mfma_i32_16x16x64_i8 v[180:183], v[104:107], a[228:231], v[180:183]\n" \
  "v_cvt_f32_i32 v239, v239\n" \
  "v_mfma_i32_16x16x64_i8 v[180:183], v[108:111], a[244:247], v[180:183]\n" \
  "v_mfma_i32_16x16x64_i8 v[212:215], v[104:107], a[232:235], v[212:215]\n" \
  "ds_read_b64_tr_b8 v[100:101], v86 offset:8192\n" \
  "v_pk_mul_f32 v[236:237], v[236:237], v[88:89]\n" \
  "v_mfma_i32_16x16x64_i8 v[212:215], v[108:111], a[248:251], v[212:215]\n" \
  "ds_read_b64_tr_b8 v[102:103], v86 offset:9216\n" \
  "v_pk_mul_f32 v[238:239], v[238:239], v[88:89]\n" \
  "v_mfma_i32_16x16x64_i8 v[244:247], v[104:107], a[236:239], v[244:247]\n" \
  "buffer_store_dwordx4 v[236:239], v47, s[20:23], s79 offen offset:192 sc0 sc1\n" \
  "v_mfma_i32_16x16x64_i8 v[244:247], v[108:111], a[252:255], v[244:247]\n" \
  "s_waitcnt lgkmcnt(8)\n" \
  "v_mfma_i32_16x16x64_i8 v[152:155], v[112:115], a[96:99], v[152:155]\n" \
  "ds_read_b64_tr_b8 v[104:105], v87 offset:0\n" \
  "v_mfma_i32_16x16x64_i8 v[152:155], v[116:119], a[112:115], v[152:155]\n" \
  "ds_read_b64_tr_b8 v[106:107], v87 offset:1024\n" \
  "v_mfma_i32_16x16x64_i8 v[184:187], v[112:115], a[100:103], v[184:187]\n" \
  "v_mfma_i32_16x16x64_i8 v[184:187], v[116:119], a[116:119], v[184:187]\n" \
  "v_mfma_i32_16x16x64_i8 v[216:219], v[112:115], a[104:107], v[216:219]\n" \
  "ds_read_b64_tr_b8 v[108:109], v87 offset:8192\n" \
  "v_mfma_i32_16x16x64_i8 v[216:219], v[116:119], a[120:123], v[216:219]\n" \
  "ds_read_b64_tr_b8 v[110:111], v87 offset:9216\n" \
  "v_mfma_i32_16x16x64_i8 v[248:251], v[112:115], a[108:111], v[248:251]\n" \
  "v_mfma_i32_16x16x64_i8 v[248:251], v[116:119], a[124:127], v[248:251]\n" \
  "s_waitcnt lgkmcnt(8)\n" \
  "v_mfma_i32_16x16x64_i8 v[156:159], v[120:123], a[96:99], v[156:159]\n" \
  "ds_read_b64_tr_b8 v[112:113], v86 offset:16384\n" \
  "v_cvt_f32_i32 v144, v144\n" \
  "v_mfma_i32_16x16x64_i8 v[156:159], v[124:127], a[112:115], v[156:159]\n" \
  "ds_read_b64_tr_b8 v[114:115], v86 offset:17408\n" \
  "v_cvt_f32_i32 v145, v145\n" \
  "v_mfma_i32_16x16x64_i8 v[188:191], v[120:123], a[100:103], v[188:191]\n" \
  "v_cvt_f32_i32 v146, v146\n" \
  "v_mfma_i32_16x16x64_i8 v[188:191], v[124:127], a[116:119], v[188:191]\n" \
  "v_cvt_f32_i32 v147, v147\n" \
  "v_mfma_i32_16x16x64_i8 v[220:223], v[120:123], a[104:107], v[220:223]\n" \
  "ds_read_b64_tr_b8 v[116:117], v86 offset:24576\n" \
  "v_mfma_i32_16x16x64_i8 v[220:223], v[124:127], a[120:123], v[220:223]\n" \
  "ds_read_b64_tr_b8 v[118:119], v86 offset:25600\n" \
  "v_pk_mul_f32 v[144:145], v[144:145], v[88:89]\n" \
  "v_mfma_i32_16x16x64_i8 v[252:255], v[120:123], a[108:111], v[252:255]\n" \
  "v_pk_mul_f32 v[146:147], v[146:147], v[88:89]\n" \
  "v_mfma_i32_16x16x64_i8 v[252:255], v[124:127], a[124:127], v[252:255]\n" \
  "buffer_store_dwordx4 v[144:147], v47, s[20:23], s76 offen offset:256 sc0 sc1\n" \
  "; ---- step 7\n" \
  "s_waitcnt lgkmcnt(8)\n" \
  "v_mfma_i32_16x16x64_i8 v[152:155], v[96:99], a[128:131], v[152:155]\n" \
  "ds_read_b64_tr_b8 v[120:121], v87 offset:16384\n" \
  "v_mfma_i32_16x16x64_i8 v[152:155], v[100:103], a[144:147], v[152:155]\n" \
  "ds_read_b64_tr_b8 v[122:123], v87 offset:17408\n" \
  "v_cvt_f32_i32 v148, v148\n" \
  "v_mfma_i32_16x16x64_i8 v[184:187], v[96:99], a[132:135], v[184:187]\n" \
  "v_cvt_f32_i32 v149, v149\n" \
  "v_mfma_i32_16x16x64_i8 v[184:187], v[100:103], a[148:151], v[184:187]\n" \
  "v_cvt_f32_i32 v150, v150\n" \
  "v_mfma_i32_16x16x64_i8 v[216:219], v[96:99], a[136:139], v[216:219]\n" \
  "ds_read_b64_tr_b8 v[124:125], v87 offset:24576\n" \
  "v_cvt_f32_i32 v151, v151\n" \
  "v_mfma_i32_16x16x64_i8 v[216:219], v[100:103], a[152:155], v[216:219]\n" \
  "ds_read_b64_tr_b8 v[126:127], v87 offset:25600\n" \
  "v_mfma_i32_16x16x64_i8 v[248:251], v[96:99], a[140:143], v[248:251]\n" \
  "v_pk_mul_f32 v[148:149], v[148:149], v[88:89]\n" \
  "v_mfma_i32_16x16x64_i8 v[248:251], v[100:103], a[156:159], v[248:251]\n" \
  "v_pk_mul_f32 v[150:151], v[150:151], v[88:89]\n" \
  "s_waitcnt lgkmcnt(8)\n" \
  "v_mfma_i32_16x16x64_i8 v[156:159], v[104:107], a[128:131], v[156:159]\n" \
  "ds_read_b64_tr_b8 v[96:97], v30 offset:0\n" \
  "buffer_store_dwordx4 v[148:151], v47, s[20:23], s76 offen offset:320 sc0 sc1\n" \
  "v_mfma_i32_16x16x64_i8 v[156:159], v[108:111], a[144:147], v[156:159]\n" \
  "ds_read_b64_tr_b8 v[98:99], v30 offset:1024\n" \
  "v_mfma_i32_16x16x64_i8 v[188:191], v[104:107], a[132:135], v[188:191]\n" \
  "v_cvt_f32_i32 v176, v176\n" \
  "v_mfma_i32_16x16x64_i8 v[188:191], v[108:111], a[148:151], v[188:191]\n" \
  "v_cvt_f32_i32 v177, v177\n" \
  "v_mfma_i32_16x16x64_i8 v[220:223], v[104:107], a[136:139], v[220:223]\n" \
  "ds_read_b64_tr_b8 v[100:101], v30 offset:8192\n" \
  "v_cvt_f32_i32 v178, v178\n" \
  "v_mfma_i32_16x16x64_i8 v[220:223], v[108:111], a[152:155], v[220:223]\n" \
  "ds_read_b64_tr_b8 v[102:103], v30 offset:9216\n" \
  "v_cvt_f32_i32 v179, v179\n" \
  "v_mfma_i32_16x16x64_i8 v[252:255], v[104:107], a[140:143], v[252:255]\n" \
  "v_mfma_i32_16x16x64_i8 v[252:255], v[108:111], a[156:159], v[252:255]\n" \
  "v_pk_mul_f32 v[176:177], v[176:177], v[88:89]\n" \
  "s_waitcnt lgkmcnt(8)\n" \
  "v_mfma_i32_16x16x64_i8 v[152:155], v[112:115], a[160:163], v[152:155]\n" \
  "ds_read_b64_tr_b8 v[104:105], v31 offset:0\n" \
  "v_pk_mul_f32 v[178:179], v[178:179], v[88:89]\n" \
  "v_mfma_i32_16x16x64_i8 v[152:155], v[116:119], a[176:179], v[152:155]\n" \
  "ds_read_b64_tr_b8 v[106:107], v31 offset:1024\n" \
  "buffer_store_dwordx4 v[176:179], v47, s[20:23], s77 offen offset:256 sc0 sc1\n" \
  "v_mfma_i32_16x16x64_i8 v[184:187], v[112:115], a[164:167], v[184:187]\n" \
  "v_mfma_i32_16x16x64_i8 v[184:187], v[116:119], a[180:183], v[184:187]\n" \
  "v_cvt_f32_i32 v180, v180\n" \
  "v_mfma_i32_16x16x64_i8 v[216:219], v[112:115], a[168:171], v[216:219]\n" \
  "ds_read_b64_tr_b8 v[108:109], v31 offset:8192\n" \
  "v_cvt_f32_i32 v181, v181\n" \
  "v_mfma_i32_16x16x64_i8 v[216:219], v[116:119], a[184:187], v[216:219]\n" \
  "ds_read_b64_tr_b8 v[110:111], v31 offset:9216\n" \
  "v_cvt_f32_i32 v182, v182\n" \
  "v_mfma_i32_16x16x64_i8 v[248:251], v[112:115], a[172:175], v[248:251]\n" \
  "v_cvt_f32_i32 v183, v183\n" \
  "v_mfma_i32_16x16x64_i8 v[248:251], v[116:119], a[188:191], v[248:251]\n" \
  "s_waitcnt lgkmcnt(8)\n" \
  "v_mfma_i32_16x16x64_i8 v[156:159], v[120:123], a[160:163], v[156:159]\n" \
  "ds_read_b64_tr_b8 v[112:113], v30 offset:16384\n" \
  "v_pk_mul_f32 v[180:181], v[180:181], v[88:89]\n" \
  "v_mfma_i32_16x16x64_i8 v[156:159], v[124:127], a[176:179], v[156:159]\n" \
  "ds_read_b64_tr_b8 v[114:115], v30 offset:17408\n" \
  "v_pk_mul_f32 v[182:183], v[182:183], v[88:89]\n" \
  "v_mfma_i32_16x16x64_i8 v[188:191], v[120:123], a[164:167], v[188:191]\n" \
  "buffer_store_dwordx4 v[180:183], v47, s[20:23], s77 offen offset:320 sc0 sc1\n" \
  "v_mfma_i32_16x16x64_i8 v[188:191], v[124:127], a[180:183], v[188:191]\n" \
  "v_mfma_i32_16x16x64_i8 v[220:223], v[120:123], a[168:171], v[220:223]\n" \
  "ds_read_b64_tr_b8 v[116:117], v30 offset:24576\n" \
  "v_cvt_f32_i32 v208, v208\n" \
  "v_mfma_i32_16x16x64_i8 v[220:223], v[124:127], a[184:187], v[220:223]\n" \
  "ds_read_b64_tr_b8 v[118:119], v30 offset:25600\n" \
  "v_cvt_f32_i32 v209, v209\n" \
  "v_mfma_i32_16x16x64_i8 v[252:255], v[120:123], a[172:175], v[252:255]\n" \
  "v_cvt_f32_i32 v210, v210\n" \
  "v_mfma_i32_16x16x64_i8 v[252:255], v[124:127], a[188:191], v[252:255]\n" \
  "v_cvt_f32_i32 v211, v211\n" \
  "s_waitcnt lgkmcnt(8)\n" \
  "v_mfma_i32_16x16x64_i8 v[152:155], v[96:99], a[192:195], v[152:155]\n" \
  "ds_read_b64_tr_b8 v[120:121], v31 offset:16384\n" \
  "v_mfma_i32_16x16x64_i8 v[152:155], v[100:103], a[208:211], v[152:155]\n" \
  "ds_read_b64_tr_b8 v[122:123], v31 offset:17408\n" \
  "v_pk_mul_f32 v[208:209], v[208:209], v[88:89]\n" \
  "v_mfma_i32_16x16x64_i8 v[184:187], v[96:99], a[196:199], v[184:187]\n" \
  "v_pk_mul_f32 v[210:211], v[210:211], v[88:89]\n" \
  "v_mfma_i32_16x16x64_i8 v[184:187], v[100:103], a[212:215], v[184:187]\n" \
  "buffer_store_dwordx4 v[208:211], v47, s[20:23], s78 offen offset:256 sc0 sc1\n" \
  "v_mfma_i32_16x16x64_i8 v[216:219], v[96:99], a[200:203], v[216:219]\n" \
  "ds_read_b64_tr_b8 v[124:125], v31 offset:24576\n" \
  "v_mfma_i32_16x16x64_i8 v[216:219], v[100:103], a[216:219], v[216:219]\n" \
  "ds_read_b64_tr_b8 v[126:127], v31 offset:25600\n" \
  "v_cvt_f32_i32 v212, v212\n" \
  "v_mfma_i32_16x16x64_i8 v[248:251], v[96:99], a[204:207], v[248:251]\n" \
  "v_cvt_f32_i32 v213, v213\n" \
  "v_mfma_i32_16x16x64_i8 v[248:251], v[100:103], a[220:223], v[248:251]\n" \
  "v_cvt_f32_i32 v214, v214\n" \
  "s_waitcnt lgkmcnt(8)\n" \
  "v_mfma_i32_16x16x64_i8 v[156:159], v[104:107], a[192:195], v[156:159]\n" \
  "v_cvt_f32_i32 v215, v215\n" \
  "v_mfma_i32_16x16x64_i8 v[156:159], v[108:111], a[208:211], v[156:159]\n" \
  "v_mfma_i32_16x16x64_i8 v[188:191], v[104:107], a[196:199], v[188:191]\n" \
  "v_pk_mul_f32 v[212:213], v[212:213], v[88:89]\n" \
  "v_mfma_i32_16x16x64_i8 v[188:191], v[108:111], a[212:215], v[188:191]\n" \
  "v_pk_mul_f32 v[214:215], v[214:215], v[88:89]\n" \
  "v_mfma_i32_16x16x64_i8 v[220:223], v[104:107], a[200:203], v[220:223]\n" \
  "buffer_store_dwordx4 v[212:215], v47, s[20:23], s78 offen offset:320 sc0 sc1\n" \
  "v_mfma_i32_16x16x64_i8 v[220:223], v[108:111], a[216:219], v[220:223]\n" \
  "v_mfma_i32_16x16x64_i8 v[252:255], v[104:107], a[204:207], v[252:255]\n" \
  "v_cvt_f32_i32 v240, v240\n" \
  "v_mfma_i32_16x16x64_i8 v[252:255], v[108:111], a[220:223], v[252:255]\n" \
  "v_cvt_f32_i32 v241, v241\n" \
  "s_waitcnt lgkmcnt(4)\n" \
  "v_mfma_i32_16x16x64_i8 v[152:155], v[112:115], a[224:227], v[152:155]\n" \
  "v_cvt_f32_i32 v242, v242\n" \
  "v_mfma_i32_16x16x64_i8 v[152:155], v[116:119], a[240:243], v[152:155]\n" \
  "v_cvt_f32_i32 v243, v243\n" \
  "v_mfma_i32_16x16x64_i8 v[184:187], v[112:115], a[228:231], v[184:187]\n" \
  "v_mfma_i32_16x16x64_i8 v[184:187], v[116:119], a[244:247], v[184:187]\n" \
  "v_pk_mul_f32 v[240:241], v[240:241], v[88:89]\n" \
  "v_mfma_i32_16x16x64_i8 v[216:219], v[112:115], a[232:235], v[216:219]\n" \
  "v_pk_mul_f32 v[242:243], v[242:243], v[88:89]\n" \
  "v_mfma_i32_16x16x64_i8 v[216:219], v[116:119], a[248:251], v[216:219]\n" \
  "buffer_store_dwordx4 v[240:243], v47, s[20:23], s79 offen offset:256 sc0 sc1\n" \
  "v_mfma_i32_16x16x64_i8 v[248:251], v[112:115], a[236:239], v[248:251]\n" \
  "v_mfma_i32_16x16x64_i8 v[248:251], v[116:119], a[252:255], v[248:251]\n" \
  "v_cvt_f32_i32 v244, v244\n" \
  "s_waitcnt lgkmcnt(0)\n" \
  "v_mfma_i32_16x16x64_i8 v[156:159], v[120:123], a[224:227], v[156:159]\n" \
  "v_cvt_f32_i32 v245, v245\n" \
  "v_mfma_i32_16x16x64_i8 v[156:159], v[124:127], a[240:243], v[156:159]\n" \
  "v_cvt_f32_i32 v246, v246\n" \
  "v_mfma_i32_16x16x64_i8 v[188:191], v[120:123], a[228:231], v[188:191]\n" \
  "v_cvt_f32_i32 v247, v247\n" \
  "v_mfma_i32_16x16x64_i8 v[188:191], v[124:127], a[244:247], v[188:191]\n" \
  "v_mfma_i32_16x16x64_i8 v[220:223], v[120:123], a[232:235], v[220:223]\n" \
  "v_pk_mul_f32 v[244:245], v[244:245], v[88:89]\n" \
  "v_mfma_i32_16x16x64_i8 v[220:223], v[124:127], a[248:251], v[220:223]\n" \
  "v_pk_mul_f32 v[246:247], v[246:247], v[88:89]\n" \
  "v_mfma_i32_16x16x64_i8 v[252:255], v[120:123], a[236:239], v[252:255]\n" \
  "buffer_store_dwordx4 v[244:247], v47, s[20:23], s79 offen offset:320 sc0 sc1\n" \
  "v_mfma_i32_16x16x64_i8 v[252:255], v[124:127], a[252:255], v[252:255]\n" \
  "s_nop 15\n" \
  "v_cvt_f32_i32 v152, v152\n" \
  "v_cvt_f32_i32 v153, v153\n" \
  "v_cvt_f32_i32 v154, v154\n" \
  "v_cvt_f32_i32 v155, v155\n" \
  "v_pk_mul_f32 v[152:153], v[152:153], v[88:89]\n" \
  "v_pk_mul_f32 v[154:155], v[154:155], v[88:89]\n" \
  "buffer_store_dwordx4 v[152:155], v47, s[20:23], s76 offen offset:384 sc0 sc1\n" \
  "v_cvt_f32_i32 v156, v156\n" \
  "v_cvt_f32_i32 v157, v157\n" \
  "v_cvt_f32_i32 v158, v158\n" \
  "v_cvt_f32_i32 v159, v159\n" \
  "v_pk_mul_f32 v[156:157], v[156:157], v[88:89]\n" \
  "v_pk_mul_f32 v[158:159], v[158:159], v[88:89]\n" \
  "buffer_store_dwordx4 v[156:159], v47, s[20:23], s76 offen offset:448 sc0 sc1\n" \
  "v_cvt_f32_i32 v184, v184\n" \
  "v_cvt_f32_i32 v185, v185\n" \
  "v_cvt_f32_i32 v186, v186\n" \
  "v_cvt_f32_i32 v187, v187\n" \
  "v_pk_mul_f32 v[184:185], v[184:185], v[88:89]\n" \
  "v_pk_mul_f32 v[186:187], v[186:187], v[88:89]\n" \
  "buffer_store_dwordx4 v[184:187], v47, s[20:23], s77 offen offset:384 sc0 sc1\n" \
  "v_cvt_f32_i32 v188, v188\n" \
  "v_cvt_f32_i32 v189, v189\n" \
  "v_cvt_f32_i32 v190, v190\n" \
  "v_cvt_f32_i32 v191, v191\n" \
  "v_pk_mul_f32 v[188:189], v[188:189], v[88:89]\n" \
  "v_pk_mul_f32 v[190:191], v[190:191], v[88:89]\n" \
  "buffer_store_dwordx4 v[188:191], v47, s[20:23], s77 offen offset:448 sc0 sc1\n" \
  "v_cvt_f32_i32 v216, v216\n" \
  "v_cvt_f32_i32 v217, v217\n" \
  "v_cvt_f32_i32 v218, v218\n" \
  "v_cvt_f32_i32 v219, v219\n" \
  "v_pk_mul_f32 v[216:217], v[216:217], v[88:89]\n" \
  "v_pk_mul_f32 v[218:219], v[218:219], v[88:89]\n" \
  "buffer_store_dwordx4 v[216:219], v47, s[20:23], s78 offen offset:384 sc0 sc1\n" \
  "v_cvt_f32_i32 v220, v220\n" \
  "v_cvt_f32_i32 v221, v221\n" \
  "v_cvt_f32_i32 v222, v222\n" \
  "v_cvt_f32_i32 v223, v223\n" \
  "v_pk_mul_f32 v[220:221], v[220:221], v[88:89]\n" \
  "v_pk_mul_f32 v[222:223], v[222:223], v[88:89]\n" \
  "buffer_store_dwordx4 v[220:223], v47, s[20:23], s78 offen offset:448 sc0 sc1\n" \
  "v_cvt_f32_i32 v248, v248\n" \
  "v_cvt_f32_i32 v249, v249\n" \
  "v_cvt_f32_i32 v250, v250\n" \
  "v_cvt_f32_i32 v251, v251\n" \
  "v_pk_mul_f32 v[248:249], v[248:249], v[88:89]\n" \
  "v_pk_mul_f32 v[250:251], v[250:251], v[88:89]\n" \
  "buffer_store_dwordx4 v[248:251], v47, s[20:23], s79 offen offset:384 sc0 sc1\n" \
  "v_cvt_f32_i32 v252, v252\n" \
  "v_cvt_f32_i32 v253, v253\n" \
  "v_cvt_f32_i32 v254, v254\n" \
  "v_cvt_f32_i32 v255, v255\n" \
  "v_pk_mul_f32 v[252:253], v[252:253], v[88:89]\n" \
  "v_pk_mul_f32 v[254:255], v[254:255], v[88:89]\n" \
  "buffer_store_dwordx4 v[252:255], v47, s[20:23], s79 offen offset:448 sc0 sc1\n" \
  "s_mov_b64 exec, 1\n" \
  "global_store_dword v51, v76, s[32:33] sc0 sc1\n" \
  "s_mov_b64 exec, -1\n" \
  "s_waitcnt vmcnt(0)\n" \
  "s_branch L_endA_%=\n" \
  "Lslow12A_%=:\n" \
  "s_memrealtime s[96:97]\n" \
  "s_waitcnt lgkmcnt(0)\n" \
  "Lspin12A_%=:\n" \
  "s_sleep 4\n" \
  "global_load_dword v72, v50, s[38:39] sc0 sc1\n" \
  "global_load_dword v90, v51, s[12:13] sc1\n" \
  "s_waitcnt vmcnt(0)\n" \
  "v_sub_u32 v72, v72, v76\n" \
  "v_cmp_gt_i32 vcc, 0, v72\n" \
  "s_nop 1\n" \
  "s_andn2_b64 vcc, vcc, s[14:15]\n" \
  "s_cbranch_scc0 Lback13A_%=\n" \
  "v_cmp_eq_u32 vcc, v90, v76\n" \
  "s_nop 1\n" \
  "s_cmp_lg_u64 vcc, 0\n" \
  "s_cbranch_scc1 Ltmo12A_%=\n" \
  "s_memrealtime s[98:99]\n" \
  "s_waitcnt lgkmcnt(0)\n" \
  "s_sub_u32 s98, s98, s96\n" \
  "s_cmp_lt_u32 s98, 50000000\n" \
  "s_cbranch_scc1 Lspin12A_%=\n" \
  "Ltmo12A_%=:\n" \
  "global_store_dword v51, v76, s[12:13] sc1\n" \
  "s_mov_b32 s55, 1\n" \
  "s_branch Lback13A_%=\n" \
  "Lslow20A_%=:\n" \
  "s_memrealtime s[96:97]\n" \
  "s_waitcnt lgkmcnt(0)\n" \
  "Lspin20A_%=:\n" \
  "s_sleep 4\n" \
  "global_load_dword v72, v50, s[38:39] sc0 sc1\n" \
  "global_load_dword v90, v51, s[12:13] sc1\n" \
  "s_waitcnt vmcnt(0)\n" \
  "v_sub_u32 v72, v72, v76\n" \
  "v_cmp_gt_i32 vcc, 0, v72\n" \
  "s_nop 1\n" \
  "s_andn2_b64 vcc, vcc, s[14:15]\n" \
  "s_cbranch_scc0 Lback21A_%=\n" \
  "v_cmp_eq_u32 vcc, v90, v76\n" \
  "s_nop 1\n" \
  "s_cmp_lg_u64 vcc, 0\n" \
  "s_cbranch_scc1 Ltmo20A_%=\n" \
  "s_memrealtime s[98:99]\n" \
  "s_waitcnt lgkmcnt(0)\n" \
  "s_sub_u32 s98, s98, s96\n" \
  "s_cmp_lt_u32 s98, 50000000\n" \
  "s_cbranch_scc1 Lspin20A_%=\n" \
  "Ltmo20A_%=:\n" \
  "global_store_dword v51, v76, s[12:13] sc1\n" \
  "s_mov_b32 s55, 1\n" \
  "s_branch Lback21A_%=\n" \
  "Lslow28A_%=:\n" \
  "s_memrealtime s[96:97]\n" \
  "s_waitcnt lgkmcnt(0)\n" \
  "Lspin28A_%=:\n" \
  "s_sleep 4\n" \
  "global_load_dword v72, v50, s[38:39] sc0 sc1\n" \
  "global_load_dword v90, v51, s[12:13] sc1\n" \
  "s_waitcnt vmcnt(0)\n" \
  "v_sub_u32 v72, v72, v76\n" \
  "v_cmp_gt_i32 vcc, 0, v72\n" \
  "s_nop 1\n" \
  "s_andn2_b64 vcc, vcc, s[14:15]\n" \
  "s_cbranch_scc0 Lback29A_%=\n" \
  "v_cmp_eq_u32 vcc, v90, v76\n" \
  "s_nop 1\n" \
  "s_cmp_lg_u64 vcc, 0\n" \
  "s_cbranch_scc1 Ltmo28A_%=\n" \
  "s_memrealtime s[98:99]\n" \
  "s_waitcnt lgkmcnt(0)\n" \
  "s_sub_u32 s98, s98, s96\n" \
  "s_cmp_lt_u32 s98, 50000000\n" \
  "s_cbranch_scc1 Lspin28A_%=\n" \
  "Ltmo28A_%=:\n" \
  "global_store_dword v51, v76, s[12:13] sc1\n" \
  "s_mov_b32 s55, 1\n" \
  "s_branch Lback29A_%=\n" \
  "Lslow36A_%=:\n" \
  "s_memrealtime s[96:97]\n" \
  "s_waitcnt lgkmcnt(0)\n" \
  "Lspin36A_%=:\n" \
  "s_sleep 4\n" \
  "global_load_dword v72, v50, s[38:39] sc0 sc1\n" \
  "global_load_dword v90, v51, s[12:13] sc1\n" \
  "s_waitcnt vmcnt(0)\n" \
  "v_sub_u32 v72, v72, v76\n" \
  "v_cmp_gt_i32 vcc, 0, v72\n" \
  "s_nop 1\n" \
  "s_andn2_b64 vcc, vcc, s[14:15]\n" \
  "s_cbranch_scc0 Lback37A_%=\n" \
  "v_cmp_eq_u32 vcc, v90, v76\n" \
  "s_nop 1\n" \
  "s_cmp_lg_u64 vcc, 0\n" \
  "s_cbranch_scc1 Ltmo36A_%=\n" \
  "s_memrealtime s[98:99]\n" \
  "s_waitcnt lgkmcnt(0)\n" \
  "s_sub_u32 s98, s98, s96\n" \
  "s_cmp_lt_u32 s98, 50000000\n" \
  "s_cbranch_scc1 Lspin36A_%=\n" \
  "Ltmo36A_%=:\n" \
  "global_store_dword v51, v76, s[12:13] sc1\n" \
  "s_mov_b32 s55, 1\n" \
  "s_branch Lback37A_%=\n" \
  "L_endA_%=:\n" \
  "s_branch Lfin_%=\n" \
  "LvarB_%=:\n" \
  "s_mov_b32 s16, s66\n" \
  "s_and_b32 s17, s67, 0xffff\n" \
  "s_mov_b32 s18, 0x4000000\n" \
  "s_mov_b32 s19, 0x20000\n" \
  "s_mov_b32 s24, s64\n" \
  "s_and_b32 s25, s65, 0xffff\n" \
  "s_mov_b32 s26, 0x200000\n" \
  "s_mov_b32 s27, 0x20000\n" \
  "s_lshl_b32 s81, s2, 6\n" \
  "s_add_u32 s32, s74, s81\n" \
  "s_addc_u32 s33, s75, 0\n" \
  "s_mov_b64 s[44:45], s[64:65]\n" \
  "s_lshl_b32 s96, s50, 9\n" \
  "s_lshl_b32 s97, s53, 8\n" \
  "s_add_u32 s96, s96, s97\n" \
  "s_lshl_b32 s97, s11, 6\n" \
  "s_add_u32 s97, s96, s97\n" \
  "v_add_u32 v10, s97, v3\n" \
  "v_lshl_add_u32 v45, v10, 9, v9\n" \
  "s_add_u32 s98, s50, 1\n" \
  "s_add_u32 s99, s98, s87\n" \
  "s_and_b32 s99, s99, 7\n" \
  "s_lshl_b32 s99, s99, 9\n" \
  "s_lshl_b32 s81, s52, 6\n" \
  "s_add_u32 s99, s99, s81\n" \
  "s_lshl_b32 s81, s11, 4\n" \
  "s_add_u32 s99, s99, s81\n" \
  "v_add_u32 v10, s99, v6\n" \
  "v_lshlrev_b32 v10, 9, v10\n" \
  "v_lshl_add_u32 v10, v5, 4, v10\n" \
  "s_lshl_b32 s81, s51, 7\n" \
  "v_add_u32 v48, s81, v10\n" \
  "s_lshl_b32 s81, s54, 4\n" \
  "s_add_u32 s81, s96, s81\n" \
  "v_add_u32 v10, s81, v2\n" \
  "v_lshlrev_b32 v10, 9, v10\n" \
  "s_lshl_b32 s81, s11, 7\n" \
  "v_add_u32 v10, s81, v10\n" \
  "v_mov_b32 v11, 0x80000000\n" \
  "v_cmp_gt_u32 vcc, 16, v2\n" \
  "s_nop 1\n" \
  "v_cndmask_b32 v56, v11, v10, vcc\n" \
  "global_load_dword v76, v51, s[32:33] sc0 sc1\n" \
  "s_cmp_gt_u32 s46, 3\n" \
  "s_cbranch_scc1 Lnox1B_%=\n" \
  "global_load_dwordx4 v[64:67], v48, s[44:45]\n" \
  "global_load_dwordx4 v[68:71], v48, s[44:45] offset:64\n" \
  "Lnox1B_%=:\n" \
  "buffer_load_dword v92, v56, s[24:27], 0 offen\n" \
  "s_mov_b32 s49, 0\n" \
  "s_lshl_b32 s80, s50, 23\n" \
  "s_lshl_b32 s88, s50, 18\n" \
  "s_mov_b32 s49, 1\n" \
  "s_add_u32 m0, s47, 0\n" \
  "s_nop 0\n" \
  "buffer_load_dwordx4 v40, s[16:19], s80 offen nt lds\n" \
  "s_add_u32 m0, s47, 1024\n" \
  "s_nop 0\n" \
  "buffer_load_dwordx4 v41, s[16:19], s80 offen nt lds\n" \
  "s_add_u32 m0, s47, 2048\n" \
  "s_nop 0\n" \
  "buffer_load_dwordx4 v42, s[16:19], s80 offen nt lds\n" \
  "s_add_u32 m0, s47, 3072\n" \
  "s_nop 0\n" \
  "buffer_load_dwordx4 v43, s[16:19], s80 offen nt lds\n" \
  "s_lshl1_add_u32 m0, s47, 131072\n" \
  "s_mov_b32 s98, 0\n" \
  "buffer_load_dwordx4 v45, s[24:27], s98 offen lds\n" \
  "s_lshl1_add_u32 m0, s47, 132096\n" \
  "s_mov_b32 s98, 4096\n" \
  "buffer_load_dwordx4 v45, s[24:27], s98 offen lds\n" \
  "s_lshl1_add_u32 m0, s47, 133120\n" \
  "s_mov_b32 s98, 8192\n" \
  "buffer_load_dwordx4 v45, s[24:27], s98 offen lds\n" \
  "s_lshl1_add_u32 m0, s47, 134144\n" \
  "s_mov_b32 s98, 12288\n" \
  "buffer_load_dwordx4 v45, s[24:27], s98 offen lds\n" \
  "s_lshl1_add_u32 m0, s47, 135168\n" \
  "s_mov_b32 s98, 16384\n" \
  "buffer_load_dwordx4 v45, s[24:27], s98 offen lds\n" \
  "s_lshl1_add_u32 m0, s47, 136192\n" \
  "s_mov_b32 s98, 20480\n" \
  "buffer_load_dwordx4 v45, s[24:27], s98 offen lds\n" \
  "s_lshl1_add_u32 m0, s47, 137216\n" \
  "s_mov_b32 s98, 24576\n" \
  "buffer_load_dwordx4 v45, s[24:27], s98 offen lds\n" \
  "s_lshl1_add_u32 m0, s47, 138240\n" \
  "s_mov_b32 s98, 28672\n" \
  "buffer_load_dwordx4 v45, s[24:27], s98 offen lds\n" \
  "s_mov_b32 s20, s72\n" \
  "s_and_b32 s21, s73, 0xffff\n" \
  "s_mov_b32 s22, 0x2000000\n" \
  "s_mov_b32 s23, 0x20000\n" \
  "s_add_u32 s28, s74, 1048576\n" \
  "s_addc_u32 s29, s75, 0\n" \
  "s_and_b32 s29, s29, 0xffff\n" \
  "s_mov_b32 s30, 0x200000\n" \
  "s_mov_b32 s31, 0x20000\n" \
  "s_mov_b64 s[40:41], s[68:69]\n" \
  "s_mov_b64 s[42:43], s[70:71]\n" \
  "s_add_u32 s12, s74, 16384\n" \
  "s_addc_u32 s13, s75, 0\n" \
  "s_lshl_b32 s81, s50, 3\n" \
  "s_mov_b64 s[14:15], 0xff\n" \
  "s_lshl_b64 s[14:15], s[14:15], s81\n" \
  "s_add_u32 s36, s34, 32768\n" \
  "s_addc_u32 s37, s35, 0\n" \
  "s_add_u32 s38, s74, 32768\n" \
  "s_addc_u32 s39, s75, 0\n" \
  "v_lshrrev_b32 v7, 1, v6\n" \
  "v_lshl_add_u32 v8, v5, 4, v7\n" \
  "v_lshlrev_b32 v8, 7, v8\n" \
  "v_and_b32 v10, 1, v2\n" \
  "v_lshl_add_u32 v8, v10, 3, v8\n" \
  "v_lshrrev_b32 v10, 1, v7\n" \
  "v_and_b32 v10, 3, v10\n" \
  "v_and_b32 v11, 1, v5\n" \
  "v_lshl_or_b32 v10, v11, 2, v10\n" \
  "v_lshl_or_b32 v32, v10, 4, v8\n" \
  "v_xor_b32 v33, 16, v32\n" \
  "v_xor_b32 v34, 32, v32\n" \
  "v_xor_b32 v35, 48, v32\n" \
  "v_xor_b32 v36, 64, v32\n" \
  "v_xor_b32 v37, 80, v32\n" \
  "v_xor_b32 v38, 96, v32\n" \
  "v_xor_b32 v39, 112, v32\n" \
  "s_lshl_b32 s81, s53, 2\n" \
  "s_add_u32 s81, s81, s11\n" \
  "s_lshl_b32 s81, s81, 13\n" \
  "v_lshl_add_u32 v44, v2, 4, s81\n" \
  "v_add_u32 v10, -4, v2\n" \
  "v_mul_u32_u24 v11, 37, v10\n" \
  "v_lshrrev_b32 v11, 8, v11\n" \
  "v_mul_u32_u24 v12, 7, v11\n" \
  "v_sub_u32 v12, v10, v12\n" \
  "s_add_u32 s81, s50, 1\n" \
  "v_add_u32 v12, s81, v12\n" \
  "v_and_b32 v12, 7, v12\n" \
  "v_lshl_add_u32 v12, v12, 2, v11\n" \
  "s_lshl_b32 s81, s50, 2\n" \
  "v_add_u32 v13, s81, v2\n" \
  "v_cmp_gt_u32 vcc, 4, v2\n" \
  "s_nop 1\n" \
  "v_cndmask_b32 v46, v12, v13, vcc\n" \
  "v_mov_b32 v13, 32\n" \
  "v_cmp_gt_u32 vcc, 32, v2\n" \
  "s_nop 1\n" \
  "v_cndmask_b32 v46, v13, v46, vcc\n" \
  "s_lshl_b32 s81, s53, 8\n" \
  "s_lshl_b32 s89, s11, 6\n" \
  "s_add_u32 s81, s81, s89\n" \
  "v_add_u32 v10, s81, v6\n" \
  "v_lshlrev_b32 v10, 16, v10\n" \
  "s_lshl_b32 s81, s55, 9\n" \
  "v_lshl_add_u32 v10, v5, 4, v10\n" \
  "v_add_u32 v47, s81, v10\n" \
  "s_lshl_b32 s81, s50, 2\n" \
  "s_add_u32 s81, s81, s51\n" \
  "s_lshl_b32 s81, s81, 16\n" \
  "s_add_u32 s81, s81, 1048576\n" \
  "s_lshl_b32 s89, s52, 13\n" \
  "s_add_u32 s81, s81, s89\n" \
  "s_lshl_b32 s89, s11, 10\n" \
  "s_add_u32 s81, s81, s89\n" \
  "v_lshl_add_u32 v49, v2, 4, s81\n" \
  "v_lshlrev_b32 v10, 2, v4\n" \
  "v_lshl_add_u32 v50, v3, 7, v10\n" \
  "s_lshl_b32 s81, s51, 10\n" \
  "s_lshl_b32 s89, s50, 7\n" \
  "s_add_u32 s81, s81, s89\n" \
  "s_lshl_b32 s89, s52, 2\n" \
  "s_add_u32 s81, s81, s89\n" \
  "v_mov_b32 v53, s81\n" \
  "s_mul_i32 s81, s11, 28\n" \
  "v_add_u32 v10, s81, v2\n" \
  "v_lshrrev_b32 v11, 4, v10\n" \
  "v_and_b32 v10, 15, v10\n" \
  "s_add_u32 s81, s50, 1\n" \
  "v_add_u32 v11, s81, v11\n" \
  "v_and_b32 v11, 7, v11\n" \
  "v_lshlrev_b32 v11, 18, v11\n" \
  "s_lshl_b32 s81, s54, 4\n" \
  "v_add_u32 v10, s81, v10\n" \
  "v_lshl_add_u32 v11, v10, 7, v11\n" \
  "s_lshl_b32 s81, s53, 15\n" \
  "v_add_u32 v11, s81, v11\n" \
  "v_mov_b32 v12, 0x80000000\n" \
  "v_cmp_gt_u32 vcc, 28, v2\n" \
  "s_nop 1\n" \
  "v_cndmask_b32 v52, v12, v11, vcc\n" \
  "v_xor_b32 v10, v5, v4\n" \
  "v_lshlrev_b32 v10, 4, v10\n" \
  "v_lshl_add_u32 v57, v6, 7, v10\n" \
  "s_lshr_b32 s4, s2, 3\n" \
  "s_lshl_b32 s4, s4, 2\n" \
  "s_add_u32 s4, s4, s11\n" \
  "s_lshl_b32 s5, s4, 21\n" \
  "s_cmp_lt_u32 s4, 32\n" \
  "s_cselect_b32 s5, s5, 0x80000000\n" \
  "s_buffer_load_dword s75, s[16:19], s5\n" \
  "s_sub_u32 s5, s4, 32\n" \
  "s_lshl_b32 s6, s5, 21\n" \
  "s_cmp_lt_u32 s5, 16\n" \
  "s_cselect_b32 s6, s6, 0x80000000\n" \
  "s_buffer_load_dword s75, s[20:23], s6\n" \
  "s_mov_b32 s9, 0\n" \
  "s_mov_b32 s10, 0\n" \
  "global_load_dword v77, v51, s[40:41]\n" \
  "global_load_dword v78, v51, s[42:43]\n" \
  "v_readlane_b32 s50, v46, s49\n" \
  "s_nop 3\n" \
  "s_lshl_b32 s81, s50, 21\n" \
  "s_lshl_b32 s89, s50, 16\n" \
  "s_add_u32 s49, s49, 1\n" \
  "v_readlane_b32 s50, v46, s49\n" \
  "s_nop 3\n" \
  "s_lshl_b32 s82, s50, 21\n" \
  "s_lshl_b32 s90, s50, 16\n" \
  "s_add_u32 s49, s49, 1\n" \
  "s_mov_b32 s76, 0\n" \
  "s_mov_b32 s77, 1048576\n" \
  "s_mov_b32 s78, 2097152\n" \
  "s_mov_b32 s79, 3145728\n" \
  "v_add_u32 v80, 0x10000, v32\n" \
  "v_add_u32 v81, 0x10000, v33\n" \
  "v_add_u32 v82, 0x10000, v34\n" \
  "v_add_u32 v83, 0x10000, v35\n" \
  "v_add_u32 v84, 0x10000, v36\n" \
  "v_add_u32 v85, 0x10000, v37\n" \
  "v_add_u32 v86, 0x10000, v38\n" \
  "v_add_u32 v87, 0x10000, v39\n" \
  "v_add_u32 v91, 4096, v44\n" \
  "v_add_u32 v24, 0x20000, v32\n" \
  "v_add_u32 v25, 0x20000, v33\n" \
  "v_add_u32 v26, 0x20000, v34\n" \
  "v_add_u32 v27, 0x20000, v35\n" \
  "v_add_u32 v28, 0x20000, v36\n" \
  "v_add_u32 v29, 0x20000, v37\n" \
  "v_add_u32 v30, 0x20000, v38\n" \
  "v_add_u32 v31, 0x20000, v39\n" \
  "v_add_u32 v79, 4096, v49\n" \
  "s_lshl1_add_u32 s96, s47, 131072\n" \
  "v_add_u32 v58, s96, v57\n" \
  "v_xor_b32 v59, 64, v58\n" \
  "s_add_u32 s96, s47, 98304\n" \
  "v_add_u32 v60, s96, v57\n" \
  "v_xor_b32 v61, 64, v60\n" \
  "s_waitcnt vmcnt(15)\n" \
  "v_add_u32 v76, 1, v76\n" \
  "s_nop 0\n" \
  "v_readfirstlane_b32 s52, v76\n" \
  "s_nop 3\n" \
  "s_and_b32 s48, s52, 1\n" \
  "s_lshl_b32 s48, s48, 21\n" \
  "s_add_u32 s34, s34, s48\n" \
  "s_addc_u32 s35, s35, 0\n" \
  "s_add_u32 s28, s28, s48\n" \
  "s_addc_u32 s29, s29, 0\n" \
  "s_mov_b32 s55, 0\n" \
  "s_mov_b32 s53, 1\n" \
  "s_mov_b32 s54, 2\n" \
  "s_cmp_lg_u32 s46, 0\n" \
  "s_cbranch_scc1 Lskip2B_%=\n" \
  "s_waitcnt vmcnt(15)\n" \
  "global_store_dwordx4 v49, v[64:67], s[34:35] sc0 sc1\n" \
  "s_nop 1\n" \
  "global_store_dwordx4 v79, v[68:71], s[34:35] sc0 sc1\n" \
  "s_nop 1\n" \
  "Lskip2B_%=:\n" \
  "s_add_u32 m0, s47, 16384\n" \
  "s_nop 0\n" \
  "buffer_load_dwordx4 v40, s[16:19], s81 offen nt lds\n" \
  "s_add_u32 m0, s47, 17408\n" \
  "s_nop 0\n" \
  "buffer_load_dwordx4 v41, s[16:19], s81 offen nt lds\n" \
  "s_add_u32 m0, s47, 18432\n" \
  "s_nop 0\n" \
  "buffer_load_dwordx4 v42, s[16:19], s81 offen nt lds\n" \
  "s_add_u32 m0, s47, 19456\n" \
  "s_nop 0\n" \
  "buffer_load_dwordx4 v43, s[16:19], s81 offen nt lds\n" \
  "s_add_u32 m0, s47, 98304\n" \
  "s_mov_b32 s98, 128\n" \
  "buffer_load_dwordx4 v45, s[24:27], s98 offen lds\n" \
  "s_add_u32 m0, s47, 99328\n" \
  "s_mov_b32 s98, 4224\n" \
  "buffer_load_dwordx4 v45, s[24:27], s98 offen lds\n" \
  "s_add_u32 m0, s47, 100352\n" \
  "s_mov_b32 s98, 8320\n" \
  "buffer_load_dwordx4 v45, s[24:27], s98 offen lds\n" \
  "s_add_u32 m0, s47, 101376\n" \
  "s_mov_b32 s98, 12416\n" \
  "buffer_load_dwordx4 v45, s[24:27], s98 offen lds\n" \
  "s_add_u32 m0, s47, 114688\n" \
  "s_mov_b32 s98, 16512\n" \
  "buffer_load_dwordx4 v45, s[24:27], s98 offen lds\n" \
  "s_add_u32 m0, s47, 115712\n" \
  "s_mov_b32 s98, 20608\n" \
  "buffer_load_dwordx4 v45, s[24:27], s98 offen lds\n" \
  "s_add_u32 m0, s47, 116736\n" \
  "s_mov_b32 s98, 24704\n" \
  "buffer_load_dwordx4 v45, s[24:27], s98 offen lds\n" \
  "s_add_u32 m0, s47, 117760\n" \
  "s_mov_b32 s98, 28800\n" \
  "buffer_load_dwordx4 v45, s[24:27], s98 offen lds\n" \
  "s_add_u32 m0, s47, 32768\n" \
  "s_nop 0\n" \
  "buffer_load_dwordx4 v40, s[16:19], s82 offen nt lds\n" \
  "s_add_u32 m0, s47, 33792\n" \
  "s_nop 0\n" \
  "buffer_load_dwordx4 v41, s[16:19], s82 offen nt lds\n" \
  "s_add_u32 m0, s47, 34816\n" \
  "s_nop 0\n" \
  "buffer_load_dwordx4 v42, s[16:19], s82 offen nt lds\n" \
  "s_add_u32 m0, s47, 35840\n" \
  "s_nop 0\n" \
  "buffer_load_dwordx4 v43, s[16:19], s82 offen nt lds\n" \
  "v_mov_b32 v128, 0\n" \
  "v_mov_b32 v129, 0\n" \
  "v_mov_b32 v130, 0\n" \
  "v_mov_b32 v131, 0\n" \
  "v_mov_b32 v132, 0\n" \
  "v_mov_b32 v133, 0\n" \
  "v_mov_b32 v134, 0\n" \
  "v_mov_b32 v135, 0\n" \
  "v_mov_b32 v136, 0\n" \
  "v_mov_b32 v137, 0\n" \
  "v_mov_b32 v138, 0\n" \
  "v_mov_b32 v139, 0\n" \
  "v_mov_b32 v140, 0\n" \
  "v_mov_b32 v141, 0\n" \
  "v_mov_b32 v142, 0\n" \
  "v_mov_b32 v143, 0\n" \
  "v_mov_b32 v144, 0\n" \
  "v_mov_b32 v145, 0\n" \
  "v_mov_b32 v146, 0\n" \
  "v_mov_b32 v147, 0\n" \
  "v_mov_b32 v148, 0\n" \
  "v_mov_b32 v149, 0\n" \
  "v_mov_b32 v150, 0\n" \
  "v_mov_b32 v151, 0\n" \
  "v_mov_b32 v152, 0\n" \
  "v_mov_b32 v153, 0\n" \
  "v_mov_b32 v154, 0\n" \
  "v_mov_b32 v155, 0\n" \
  "v_mov_b32 v156, 0\n" \
  "v_mov_b32 v157, 0\n" \
  "v_mov_b32 v158, 0\n" \
  "v_mov_b32 v159, 0\n" \
  "v_mov_b32 v160, 0\n" \
  "v_mov_b32 v161, 0\n" \
  "v_mov_b32 v162, 0\n" \
  "v_mov_b32 v163, 0\n" \
  "v_mov_b32 v164, 0\n" \
  "v_mov_b32 v165, 0\n" \
  "v_mov_b32 v166, 0\n" \
  "v_mov_b32 v167, 0\n" \
  "v_mov_b32 v168, 0\n" \
  "v_mov_b32 v169, 0\n" \
  "v_mov_b32 v170, 0\n" \
  "v_mov_b32 v171, 0\n" \
  "v_mov_b32 v172, 0\n" \
  "v_mov_b32 v173, 0\n" \
  "v_mov_b32 v174, 0\n" \
  "v_mov_b32 v175, 0\n" \
  "v_mov_b32 v176, 0\n" \
  "v_mov_b32 v177, 0\n" \
  "v_mov_b32 v178, 0\n" \
  "v_mov_b32 v179, 0\n" \
  "v_mov_b32 v180, 0\n" \
  "v_mov_b32 v181, 0\n" \
  "v_mov_b32 v182, 0\n" \
  "v_mov_b32 v183, 0\n" \
  "v_mov_b32 v184, 0\n" \
  "v_mov_b32 v185, 0\n" \
  "v_mov_b32 v186, 0\n" \
  "v_mov_b32 v187, 0\n" \
  "v_mov_b32 v188, 0\n" \
  "v_mov_b32 v189, 0\n" \
  "v_mov_b32 v190, 0\n" \
  "v_mov_b32 v191, 0\n" \
  "v_mov_b32 v192, 0\n" \
  "v_mov_b32 v193, 0\n" \
  "v_mov_b32 v194, 0\n" \
  "v_mov_b32 v195, 0\n" \
  "v_mov_b32 v196, 0\n" \
  "v_mov_b32 v197, 0\n" \
  "v_mov_b32 v198, 0\n" \
  "v_mov_b32 v199, 0\n" \
  "v_mov_b32 v200, 0\n" \
  "v_mov_b32 v201, 0\n" \
  "v_mov_b32 v202, 0\n" \
  "v_mov_b32 v203, 0\n" \
  "v_mov_b32 v204, 0\n" \
  "v_mov_b32 v205, 0\n" \
  "v_mov_b32 v206, 0\n" \
  "v_mov_b32 v207, 0\n" \
  "v_mov_b32 v208, 0\n" \
  "v_mov_b32 v209, 0\n" \
  "v_mov_b32 v210, 0\n" \
  "v_mov_b32 v211, 0\n" \
  "v_mov_b32 v212, 0\n" \
  "v_mov_b32 v213, 0\n" \
  "v_mov_b32 v214, 0\n" \
  "v_mov_b32 v215, 0\n" \
  "v_mov_b32 v216, 0\n" \
  "v_mov_b32 v217, 0\n" \
  "v_mov_b32 v218, 0\n" \
  "v_mov_b32 v219, 0\n" \
  "v_mov_b32 v220, 0\n" \
  "v_mov_b32 v221, 0\n" \
  "v_mov_b32 v222, 0\n" \
  "v_mov_b32 v223, 0\n" \
  "v_mov_b32 v224, 0\n" \
  "v_mov_b32 v225, 0\n" \
  "v_mov_b32 v226, 0\n" \
  "v_mov_b32 v227, 0\n" \
  "v_mov_b32 v228, 0\n" \
  "v_mov_b32 v229, 0\n" \
  "v_mov_b32 v230, 0\n" \
  "v_mov_b32 v231, 0\n" \
  "v_mov_b32 v232, 0\n" \
  "v_mov_b32 v233, 0\n" \
  "v_mov_b32 v234, 0\n" \
  "v_mov_b32 v235, 0\n" \
  "v_mov_b32 v236, 0\n" \
  "v_mov_b32 v237, 0\n" \
  "v_mov_b32 v238, 0\n" \
  "v_mov_b32 v239, 0\n" \
  "v_mov_b32 v240, 0\n" \
  "v_mov_b32 v241, 0\n" \
  "v_mov_b32 v242, 0\n" \
  "v_mov_b32 v243, 0\n" \
  "v_mov_b32 v244, 0\n" \
  "v_mov_b32 v245, 0\n" \
  "v_mov_b32 v246, 0\n" \
  "v_mov_b32 v247, 0\n" \
  "v_mov_b32 v248, 0\n" \
  "v_mov_b32 v249, 0\n" \
  "v_mov_b32 v250, 0\n" \
  "v_mov_b32 v251, 0\n" \
  "v_mov_b32 v252, 0\n" \
  "v_mov_b32 v253, 0\n" \
  "v_mov_b32 v254, 0\n" \
  "v_mov_b32 v255, 0\n" \
  "s_cmp_eq_u32 s46, 0\n" \
  "s_cbranch_scc1 Lskip3B_%=\n" \
  "s_waitcnt vmcnt(18)\n" \
  "s_branch Lskip4B_%=\n" \
  "Lskip3B_%=:\n" \
  "s_waitcnt vmcnt(20)\n" \
  "Lskip4B_%=:\n" \
  "ds_read_b128 a[0:3], v58 offset:0\n" \
  "ds_read_b128 a[4:7], v58 offset:2048\n" \
  "ds_read_b128 a[8:11], v58 offset:4096\n" \
  "ds_read_b128 a[12:15], v58 offset:6144\n" \
  "ds_read_b128 a[16:19], v59 offset:0\n" \
  "ds_read_b128 a[20:23], v59 offset:2048\n" \
  "ds_read_b128 a[24:27], v59 offset:4096\n" \
  "ds_read_b128 a[28:31], v59 offset:6144\n" \
  "s_waitcnt vmcnt(26)\n" \
  "s_barrier\n" \
  "ds_read_b64_tr_b8 v[96:97], v32 offset:0\n" \
  "ds_read_b64_tr_b8 v[98:99], v32 offset:1024\n" \
  "ds_read_b64_tr_b8 v[100:101], v32 offset:8192\n" \
  "ds_read_b64_tr_b8 v[102:103], v32 offset:9216\n" \
  "ds_read_b64_tr_b8 v[104:105], v33 offset:0\n" \
  "ds_read_b64_tr_b8 v[106:107], v33 offset:1024\n" \
  "ds_read_b64_tr_b8 v[108:109], v33 offset:8192\n" \
  "ds_read_b64_tr_b8 v[110:111], v33 offset:9216\n" \
  "ds_read_b64_tr_b8 v[112:113], v34 offset:0\n" \
  "ds_read_b64_tr_b8 v[114:115], v34 offset:1024\n" \
  "ds_read_b64_tr_b8 v[116:117], v34 offset:8192\n" \
  "ds_read_b64_tr_b8 v[118:119], v34 offset:9216\n" \
  "; ---- step 0\n" \
  "s_waitcnt lgkmcnt(8)\n" \
  "v_mfma_i32_16x16x64_i8 v[128:131], v[96:99], a[0:3], v[128:131]\n" \
  "ds_read_b64_tr_b8 v[120:121], v35 offset:0\n" \
  "v_mfma_i32_16x16x64_i8 v[128:131], v[100:103], a[16:19], v[128:131]\n" \
  "ds_read_b64_tr_b8 v[122:123], v35 offset:1024\n" \
  "v_mfma_i32_16x16x64_i8 v[160:163], v[96:99], a[4:7], v[160:163]\n" \
  "s_lshl1_add_u32 m0, s47, 131072\n" \
  "s_mov_b32 s98, 256\n" \
  "v_mfma_i32_16x16x64_i8 v[160:163], v[100:103], a[20:23], v[160:163]\n" \
  "buffer_load_dwordx4 v45, s[24:27], s98 offen lds\n" \
  "v_mfma_i32_16x16x64_i8 v[192:195], v[96:99], a[8:11], v[192:195]\n" \
  "ds_read_b64_tr_b8 v[124:125], v35 offset:8192\n" \
  "v_mfma_i32_16x16x64_i8 v[192:195], v[100:103], a[24:27], v[192:195]\n" \
  "ds_read_b64_tr_b8 v[126:127], v35 offset:9216\n" \
  "v_mfma_i32_16x16x64_i8 v[224:227], v[96:99], a[12:15], v[224:227]\n" \
  "s_lshl1_add_u32 m0, s47, 132096\n" \
  "s_mov_b32 s98, 4352\n" \
  "v_mfma_i32_16x16x64_i8 v[224:227], v[100:103], a[28:31], v[224:227]\n" \
  "buffer_load_dwordx4 v45, s[24:27], s98 offen lds\n" \
  "s_waitcnt lgkmcnt(8)\n" \
  "v_mfma_i32_16x16x64_i8 v[132:135], v[104:107], a[0:3], v[132:135]\n" \
  "ds_read_b64_tr_b8 v[96:97], v36 offset:0\n" \
  "v_mfma_i32_16x16x64_i8 v[132:135], v[108:111], a[16:19], v[132:135]\n" \
  "ds_read_b64_tr_b8 v[98:99], v36 offset:1024\n" \
  "v_mfma_i32_16x16x64_i8 v[164:167], v[104:107], a[4:7], v[164:167]\n" \
  "s_lshl1_add_u32 m0, s47, 133120\n" \
  "s_mov_b32 s98, 8448\n" \
  "v_mfma_i32_16x16x64_i8 v[164:167], v[108:111], a[20:23], v[164:167]\n" \
  "buffer_load_dwordx4 v45, s[24:27], s98 offen lds\n" \
  "v_mfma_i32_16x16x64_i8 v[196:199], v[104:107], a[8:11], v[196:199]\n" \
  "ds_read_b64_tr_b8 v[100:101], v36 offset:8192\n" \
  "v_mfma_i32_16x16x64_i8 v[196:199], v[108:111], a[24:27], v[196:199]\n" \
  "ds_read_b64_tr_b8 v[102:103], v36 offset:9216\n" \
  "v_mfma_i32_16x16x64_i8 v[228:231], v[104:107], a[12:15], v[228:231]\n" \
  "s_lshl1_add_u32 m0, s47, 134144\n" \
  "s_mov_b32 s98, 12544\n" \
  "v_mfma_i32_16x16x64_i8 v[228:231], v[108:111], a[28:31], v[228:231]\n" \
  "buffer_load_dwordx4 v45, s[24:27], s98 offen lds\n" \
  "s_waitcnt lgkmcnt(8)\n" \
  "v_mfma_i32_16x16x64_i8 v[136:139], v[112:115], a[0:3], v[136:139]\n" \
  "ds_read_b64_tr_b8 v[104:105], v37 offset:0\n" \
  "v_mfma_i32_16x16x64_i8 v[136:139], v[116:119], a[16:19], v[136:139]\n" \
  "ds_read_b64_tr_b8 v[106:107], v37 offset:1024\n" \
  "v_mfma_i32_16x16x64_i8 v[168:171], v[112:115], a[4:7], v[168:171]\n" \
  "s_lshl1_add_u32 m0, s47, 135168\n" \
  "s_mov_b32 s98, 16640\n" \
  "v_mfma_i32_16x16x64_i8 v[168:171], v[116:119], a[20:23], v[168:171]\n" \
  "buffer_load_dwordx4 v45, s[24:27], s98 offen lds\n" \
  "v_mfma_i32_16x16x64_i8 v[200:203], v[112:115], a[8:11], v[200:203]\n" \
  "ds_read_b64_tr_b8 v[108:109], v37 offset:8192\n" \
  "v_mfma_i32_16x16x64_i8 v[200:203], v[116:119], a[24:27], v[200:203]\n" \
  "ds_read_b64_tr_b8 v[110:111], v37 offset:9216\n" \
  "v_mfma_i32_16x16x64_i8 v[232:235], v[112:115], a[12:15], v[232:235]\n" \
  "s_lshl1_add_u32 m0, s47, 136192\n" \
  "s_mov_b32 s98, 20736\n" \
  "v_mfma_i32_16x16x64_i8 v[232:235], v[116:119], a[28:31], v[232:235]\n" \
  "buffer_load_dwordx4 v45, s[24:27], s98 offen lds\n" \
  "s_waitcnt lgkmcnt(8)\n" \
  "v_mfma_i32_16x16x64_i8 v[140:143], v[120:123], a[0:3], v[140:143]\n" \
  "ds_read_b64_tr_b8 v[112:113], v38 offset:0\n" \
  "v_mfma_i32_16x16x64_i8 v[140:143], v[124:127], a[16:19], v[140:143]\n" \
  "ds_read_b64_tr_b8 v[114:115], v38 offset:1024\n" \
  "v_mfma_i32_16x16x64_i8 v[172:175], v[120:123], a[4:7], v[172:175]\n" \
  "s_lshl1_add_u32 m0, s47, 137216\n" \
  "s_mov_b32 s98, 24832\n" \
  "v_mfma_i32_16x16x64_i8 v[172:175], v[124:127], a[20:23], v[172:175]\n" \
  "buffer_load_dwordx4 v45, s[24:27], s98 offen lds\n" \
  "s_cmp_lg_u32 s46, 0\n" \
  "s_cbranch_scc1 Lskip5B_%=\n" \
  "s_waitcnt vmcnt(23)\n" \
  "Lskip5B_%=:\n" \
  "s_waitcnt vmcnt(19)\n" \
  "s_barrier\n" \
  "s_cmp_lg_u32 s46, 0\n" \
  "s_cbranch_scc1 Lskip7B_%=\n" \
  "s_cmp_lg_u32 s11, 0\n" \
  "s_cbranch_scc1 Lskip6B_%=\n" \
  "s_mov_b64 exec, 1\n" \
  "global_store_dword v53, v76, s[36:37] sc0 sc1\n" \
  "s_mov_b64 exec, -1\n" \
  "Lskip6B_%=:\n" \
  "Lskip7B_%=:\n" \
  "v_mfma_i32_16x16x64_i8 v[204:207], v[120:123], a[8:11], v[204:207]\n" \
  "ds_read_b64_tr_b8 v[116:117], v38 offset:8192\n" \
  "v_mfma_i32_16x16x64_i8 v[204:207], v[124:127], a[24:27], v[204:207]\n" \
  "ds_read_b64_tr_b8 v[118:119], v38 offset:9216\n" \
  "v_mfma_i32_16x16x64_i8 v[236:239], v[120:123], a[12:15], v[236:239]\n" \
  "s_lshl1_add_u32 m0, s47, 138240\n" \
  "s_mov_b32 s98, 28928\n" \
  "v_mfma_i32_16x16x64_i8 v[236:239], v[124:127], a[28:31], v[236:239]\n" \
  "buffer_load_dwordx4 v45, s[24:27], s98 offen lds\n" \
  "s_waitcnt lgkmcnt(8)\n" \
  "v_mfma_i32_16x16x64_i8 v[144:147], v[96:99], a[0:3], v[144:147]\n" \
  "ds_read_b64_tr_b8 v[120:121], v39 offset:0\n" \
  "v_mfma_i32_16x16x64_i8 v[144:147], v[100:103], a[16:19], v[144:147]\n" \
  "ds_read_b64_tr_b8 v[122:123], v39 offset:1024\n" \
  "v_mfma_i32_16x16x64_i8 v[176:179], v[96:99], a[4:7], v[176:179]\n" \
  "v_mfma_i32_16x16x64_i8 v[176:179], v[100:103], a[20:23], v[176:179]\n" \
  "v_mfma_i32_16x16x64_i8 v[208:211], v[96:99], a[8:11], v[208:211]\n" \
  "ds_read_b64_tr_b8 v[124:125], v39 offset:8192\n" \
  "v_mfma_i32_16x16x64_i8 v[208:211], v[100:103], a[24:27], v[208:211]\n" \
  "ds_read_b64_tr_b8 v[126:127], v39 offset:9216\n" \
  "v_mfma_i32_16x16x64_i8 v[240:243], v[96:99], a[12:15], v[240:243]\n" \
  "v_mfma_i32_16x16x64_i8 v[240:243], v[100:103], a[28:31], v[240:243]\n" \
  "s_waitcnt lgkmcnt(8)\n" \
  "v_mfma_i32_16x16x64_i8 v[148:151], v[104:107], a[0:3], v[148:151]\n" \
  "ds_read_b64_tr_b8 v[96:97], v32 offset:16384\n" \
  "v_mfma_i32_16x16x64_i8 v[148:151], v[108:111], a[16:19], v[148:151]\n" \
  "ds_read_b64_tr_b8 v[98:99], v32 offset:17408\n" \
  "v_mfma_i32_16x16x64_i8 v[180:183], v[104:107], a[4:7], v[180:183]\n" \
  "v_mfma_i32_16x16x64_i8 v[180:183], v[108:111], a[20:23], v[180:183]\n" \
  "s_waitcnt vmcnt(12)\n" \
  "v_mfma_i32_16x16x64_i8 v[212:215], v[104:107], a[8:11], v[212:215]\n" \
  "ds_read_b64_tr_b8 v[100:101], v32 offset:24576\n" \
  "v_mfma_i32_16x16x64_i8 v[212:215], v[108:111], a[24:27], v[212:215]\n" \
  "ds_read_b64_tr_b8 v[102:103], v32 offset:25600\n" \
  "v_mfma_i32_16x16x64_i8 v[244:247], v[104:107], a[12:15], v[244:247]\n" \
  "ds_read_b128 a[32:35], v60 offset:0\n" \
  "v_mfma_i32_16x16x64_i8 v[244:247], v[108:111], a[28:31], v[244:247]\n" \
  "s_waitcnt lgkmcnt(9)\n" \
  "v_mfma_i32_16x16x64_i8 v[152:155], v[112:115], a[0:3], v[152:155]\n" \
  "ds_read_b64_tr_b8 v[104:105], v33 offset:16384\n" \
  "ds_read_b128 a[36:39], v60 offset:2048\n" \
  "v_mfma_i32_16x16x64_i8 v[152:155], v[116:119], a[16:19], v[152:155]\n" \
  "ds_read_b64_tr_b8 v[106:107], v33 offset:17408\n" \
  "v_mfma_i32_16x16x64_i8 v[184:187], v[112:115], a[4:7], v[184:187]\n" \
  "ds_read_b128 a[40:43], v60 offset:16384\n" \
  "v_mfma_i32_16x16x64_i8 v[184:187], v[116:119], a[20:23], v[184:187]\n" \
  "v_mfma_i32_16x16x64_i8 v[216:219], v[112:115], a[8:11], v[216:219]\n" \
  "ds_read_b64_tr_b8 v[108:109], v33 offset:24576\n" \
  "ds_read_b128 a[44:47], v60 offset:18432\n" \
  "v_mfma_i32_16x16x64_i8 v[216:219], v[116:119], a[24:27], v[216:219]\n" \
  "ds_read_b64_tr_b8 v[110:111], v33 offset:25600\n" \
  "v_mfma_i32_16x16x64_i8 v[248:251], v[112:115], a[12:15], v[248:251]\n" \
  "ds_read_b128 a[48:51], v61 offset:0\n" \
  "v_mfma_i32_16x16x64_i8 v[248:251], v[116:119], a[28:31], v[248:251]\n" \
  "s_waitcnt lgkmcnt(13)\n" \
  "v_mfma_i32_16x16x64_i8 v[156:159], v[120:123], a[0:3], v[156:159]\n" \
  "ds_read_b64_tr_b8 v[112:113], v34 offset:16384\n" \
  "ds_read_b128 a[52:55], v61 offset:2048\n" \
  "v_mfma_i32_16x16x64_i8 v[156:159], v[124:127], a[16:19], v[156:159]\n" \
  "ds_read_b64_tr_b8 v[114:115], v34 offset:17408\n" \
  "v_mfma_i32_16x16x64_i8 v[188:191], v[120:123], a[4:7], v[188:191]\n" \
  "ds_read_b128 a[56:59], v61 offset:16384\n" \
  "v_mfma_i32_16x16x64_i8 v[188:191], v[124:127], a[20:23], v[188:191]\n" \
  "v_mfma_i32_16x16x64_i8 v[220:223], v[120:123], a[8:11], v[220:223]\n" \
  "ds_read_b64_tr_b8 v[116:117], v34 offset:24576\n" \
  "ds_read_b128 a[60:63], v61 offset:18432\n" \
  "v_mfma_i32_16x16x64_i8 v[220:223], v[124:127], a[24:27], v[220:223]\n" \
  "ds_read_b64_tr_b8 v[118:119], v34 offset:25600\n" \
  "v_mfma_i32_16x16x64_i8 v[252:255], v[120:123], a[12:15], v[252:255]\n" \
  "v_mfma_i32_16x16x64_i8 v[252:255], v[124:127], a[28:31], v[252:255]\n" \
  "; ---- step 1\n" \
  "s_waitcnt lgkmcnt(1)\n" \
  "v_mfma_i32_16x16x64_i8 v[128:131], v[96:99], a[32:35], v[128:131]\n" \
  "ds_read_b64_tr_b8 v[120:121], v35 offset:16384\n" \
  "v_mfma_i32_16x16x64_i8 v[128:131], v[100:103], a[48:51], v[128:131]\n" \
  "ds_read_b64_tr_b8 v[122:123], v35 offset:17408\n" \
  "v_mfma_i32_16x16x64_i8 v[160:163], v[96:99], a[36:39], v[160:163]\n" \
  "s_add_u32 m0, s47, 98304\n" \
  "s_mov_b32 s98, 384\n" \
  "v_mfma_i32_16x16x64_i8 v[160:163], v[100:103], a[52:55], v[160:163]\n" \
  "buffer_load_dwordx4 v45, s[24:27], s98 offen lds\n" \
  "v_mfma_i32_16x16x64_i8 v[192:195], v[96:99], a[40:43], v[192:195]\n" \
  "ds_read_b64_tr_b8 v[124:125], v35 offset:24576\n" \
  "v_mfma_i32_16x16x64_i8 v[192:195], v[100:103], a[56:59], v[192:195]\n" \
  "ds_read_b64_tr_b8 v[126:127], v35 offset:25600\n" \
  "v_mfma_i32_16x16x64_i8 v[224:227], v[96:99], a[44:47], v[224:227]\n" \
  "s_add_u32 m0, s47, 99328\n" \
  "s_mov_b32 s98, 4480\n" \
  "v_mfma_i32_16x16x64_i8 v[224:227], v[100:103], a[60:63], v[224:227]\n" \
  "buffer_load_dwordx4 v45, s[24:27], s98 offen lds\n" \
  "s_waitcnt lgkmcnt(12)\n" \
  "v_mfma_i32_16x16x64_i8 v[132:135], v[104:107], a[32:35], v[132:135]\n" \
  "ds_read_b64_tr_b8 v[96:97], v36 offset:16384\n" \
  "v_mfma_i32_16x16x64_i8 v[132:135], v[108:111], a[48:51], v[132:135]\n" \
  "ds_read_b64_tr_b8 v[98:99], v36 offset:17408\n" \
  "v_mfma_i32_16x16x64_i8 v[164:167], v[104:107], a[36:39], v[164:167]\n" \
  "s_add_u32 m0, s47, 100352\n" \
  "s_mov_b32 s98, 8576\n" \
  "v_mfma_i32_16x16x64_i8 v[164:167], v[108:111], a[52:55], v[164:167]\n" \
  "buffer_load_dwordx4 v45, s[24:27], s98 offen lds\n" \
  "v_mfma_i32_16x16x64_i8 v[196:199], v[104:107], a[40:43], v[196:199]\n" \
  "ds_read_b64_tr_b8 v[100:101], v36 offset:24576\n" \
  "v_mfma_i32_16x16x64_i8 v[196:199], v[108:111], a[56:59], v[196:199]\n" \
  "ds_read_b64_tr_b8 v[102:103], v36 offset:25600\n" \
  "v_mfma_i32_16x16x64_i8 v[228:231], v[104:107], a[44:47], v[228:231]\n" \
  "v_readlane_b32 s50, v46, s49\n" \
  "s_add_u32 m0, s47, 101376\n" \
  "s_mov_b32 s98, 12672\n" \
  "v_mfma_i32_16x16x64_i8 v[228:231], v[108:111], a[60:63], v[228:231]\n" \
  "buffer_load_dwordx4 v45, s[24:27], s98 offen lds\n" \
  "s_waitcnt lgkmcnt(8)\n" \
  "v_mfma_i32_16x16x64_i8 v[136:139], v[112:115], a[32:35], v[136:139]\n" \
  "ds_read_b64_tr_b8 v[104:105], v37 offset:16384\n" \
  "v_mfma_i32_16x16x64_i8 v[136:139], v[116:119], a[48:51], v[136:139]\n" \
  "ds_read_b64_tr_b8 v[106:107], v37 offset:17408\n" \
  "v_mfma_i32_16x16x64_i8 v[168:171], v[112:115], a[36:39], v[168:171]\n" \
  "s_add_u32 m0, s47, 114688\n" \
  "s_mov_b32 s98, 16768\n" \
  "v_mfma_i32_16x16x64_i8 v[168:171], v[116:119], a[52:55], v[168:171]\n" \
  "buffer_load_dwordx4 v45, s[24:27], s98 offen lds\n" \
  "v_mfma_i32_16x16x64_i8 v[200:203], v[112:115], a[40:43], v[200:203]\n" \
  "ds_read_b64_tr_b8 v[108:109], v37 offset:24576\n" \
  "v_mfma_i32_16x16x64_i8 v[200:203], v[116:119], a[56:59], v[200:203]\n" \
  "ds_read_b64_tr_b8 v[110:111], v37 offset:25600\n" \
  "v_mfma_i32_16x16x64_i8 v[232:235], v[112:115], a[44:47], v[232:235]\n" \
  "s_lshl_b32 s83, s50, 21\n" \
  "s_lshl_b32 s91, s50, 16\n" \
  "s_add_u32 s49, s49, 1\n" \
  "s_add_u32 m0, s47, 115712\n" \
  "s_mov_b32 s98, 20864\n" \
  "v_mfma_i32_16x16x64_i8 v[232:235], v[116:119], a[60:63], v[232:235]\n" \
  "buffer_load_dwordx4 v45, s[24:27], s98 offen lds\n" \
  "s_waitcnt lgkmcnt(8)\n" \
  "v_mfma_i32_16x16x64_i8 v[140:143], v[120:123], a[32:35], v[140:143]\n" \
  "ds_read_b64_tr_b8 v[112:113], v38 offset:16384\n" \
  "v_mfma_i32_16x16x64_i8 v[140:143], v[124:127], a[48:51], v[140:143]\n" \
  "ds_read_b64_tr_b8 v[114:115], v38 offset:17408\n" \
  "v_mfma_i32_16x16x64_i8 v[172:175], v[120:123], a[36:39], v[172:175]\n" \
  "v_readlane_b32 s50, v46, s49\n" \
  "s_add_u32 m0, s47, 116736\n" \
  "s_mov_b32 s98, 24960\n" \
  "v_mfma_i32_16x16x64_i8 v[172:175], v[124:127], a[52:55], v[172:175]\n" \
  "buffer_load_dwordx4 v45, s[24:27], s98 offen lds\n" \
  "s_waitcnt vmcnt(15)\n" \
  "s_barrier\n" \
  "v_mfma_i32_16x16x64_i8 v[204:207], v[120:123], a[40:43], v[204:207]\n" \
  "ds_read_b64_tr_b8 v[116:117], v38 offset:24576\n" \
  "v_mfma_i32_16x16x64_i8 v[204:207], v[124:127], a[56:59], v[204:207]\n" \
  "ds_read_b64_tr_b8 v[118:119], v38 offset:25600\n" \
  "v_mfma_i32_16x16x64_i8 v[236:239], v[120:123], a[44:47], v[236:239]\n" \
  "s_add_u32 m0, s47, 117760\n" \
  "s_mov_b32 s98, 29056\n" \
  "v_mfma_i32_16x16x64_i8 v[236:239], v[124:127], a[60:63], v[236:239]\n" \
  "buffer_load_dwordx4 v45, s[24:27], s98 offen lds\n" \
  "s_waitcnt vmcnt(8)\n" \
  "s_waitcnt lgkmcnt(8)\n" \
  "v_mfma_i32_16x16x64_i8 v[144:147], v[96:99], a[32:35], v[144:147]\n" \
  "ds_read_b64_tr_b8 v[120:121], v39 offset:16384\n" \
  "v_mfma_i32_16x16x64_i8 v[144:147], v[100:103], a[48:51], v[144:147]\n" \
  "ds_read_b64_tr_b8 v[122:123], v39 offset:17408\n" \
  "s_add_u32 m0, s47, 49152\n" \
  "v_mfma_i32_16x16x64_i8 v[176:179], v[96:99], a[36:39], v[176:179]\n" \
  "buffer_load_dwordx4 v40, s[16:19], s83 offen nt lds\n" \
  "s_lshl_b32 s84, s50, 21\n" \
  "s_lshl_b32 s92, s50, 16\n" \
  "s_add_u32 s49, s49, 1\n" \
  "ds_read_b128 a[64:67], v58 offset:0\n" \
  "v_mfma_i32_16x16x64_i8 v[176:179], v[100:103], a[52:55], v[176:179]\n" \
  "v_mfma_i32_16x16x64_i8 v[208:211], v[96:99], a[40:43], v[208:211]\n" \
  "ds_read_b64_tr_b8 v[124:125], v39 offset:24576\n" \
  "ds_read_b128 a[68:71], v58 offset:2048\n" \
  "v_mfma_i32_16x16x64_i8 v[208:211], v[100:103], a[56:59], v[208:211]\n" \
  "ds_read_b64_tr_b8 v[126:127], v39 offset:25600\n" \
  "s_add_u32 m0, s47, 50176\n" \
  "v_mfma_i32_16x16x64_i8 v[240:243], v[96:99], a[44:47], v[240:243]\n" \
  "buffer_load_dwordx4 v41, s[16:19], s83 offen nt lds\n" \
  "v_readlane_b32 s50, v46, s49\n" \
  "ds_read_b128 a[72:75], v58 offset:4096\n" \
  "v_mfma_i32_16x16x64_i8 v[240:243], v[100:103], a[60:63], v[240:243]\n" \
  "s_waitcnt lgkmcnt(11)\n" \
  "v_mfma_i32_16x16x64_i8 v[148:151], v[104:107], a[32:35], v[148:151]\n" \
  "ds_read_b64_tr_b8 v[96:97], v32 offset:32768\n" \
  "ds_read_b128 a[76:79], v58 offset:6144\n" \
  "v_mfma_i32_16x16x64_i8 v[148:151], v[108:111], a[48:51], v[148:151]\n" \
  "ds_read_b64_tr_b8 v[98:99], v32 offset:33792\n" \
  "s_add_u32 m0, s47, 51200\n" \
  "v_mfma_i32_16x16x64_i8 v[180:183], v[104:107], a[36:39], v[180:183]\n" \
  "buffer_load_dwordx4 v42, s[16:19], s83 offen nt lds\n" \
  "ds_read_b128 a[80:83], v59 offset:0\n" \
  "v_mfma_i32_16x16x64_i8 v[180:183], v[108:111], a[52:55], v[180:183]\n" \
  "v_mfma_i32_16x16x64_i8 v[212:215], v[104:107], a[40:43], v[212:215]\n" \
  "ds_read_b64_tr_b8 v[100:101], v32 offset:40960\n" \
  "ds_read_b128 a[84:87], v59 offset:2048\n" \
  "v_mfma_i32_16x16x64_i8 v[212:215], v[108:111], a[56:59], v[212:215]\n" \
  "ds_read_b64_tr_b8 v[102:103], v32 offset:41984\n" \
  "s_add_u32 m0, s47, 52224\n" \
  "v_mfma_i32_16x16x64_i8 v[244:247], v[104:107], a[44:47], v[244:247]\n" \
  "buffer_load_dwordx4 v43, s[16:19], s83 offen nt lds\n" \
  "s_lshl_b32 s85, s50, 21\n" \
  "s_lshl_b32 s93, s50, 16\n" \
  "s_add_u32 s49, s49, 1\n" \
  "ds_read_b128 a[88:91], v59 offset:4096\n" \
  "v_mfma_i32_16x16x64_i8 v[244:247], v[108:111], a[60:63], v[244:247]\n" \
  "s_waitcnt vmcnt(4)\n" \
  "s_waitcnt lgkmcnt(15)\n" \
  "v_mfma_i32_16x16x64_i8 v[152:155], v[112:115], a[32:35], v[152:155]\n" \
  "ds_read_b64_tr_b8 v[104:105], v33 offset:32768\n" \
  "ds_read_b128 a[92:95], v59 offset:6144\n" \
  "v_mfma_i32_16x16x64_i8 v[152:155], v[116:119], a[48:51], v[152:155]\n" \
  "ds_read_b64_tr_b8 v[106:107], v33 offset:33792\n" \
  "s_add_u32 m0, s47, 65536\n" \
  "v_mfma_i32_16x16x64_i8 v[184:187], v[112:115], a[36:39], v[184:187]\n" \
  "buffer_load_dwordx4 v40, s[16:19], s84 offen nt lds\n" \
  "v_readlane_b32 s50, v46, s49\n" \
  "ds_read_b128 a[96:99], v60 offset:0\n" \
  "v_mfma_i32_16x16x64_i8 v[184:187], v[116:119], a[52:55], v[184:187]\n" \
  "v_mfma_i32_16x16x64_i8 v[216:219], v[112:115], a[40:43], v[216:219]\n" \
  "ds_read_b64_tr_b8 v[108:109], v33 offset:40960\n" \
  "ds_read_b128 a[100:103], v60 offset:2048\n" \
  "v_mfma_i32_16x16x64_i8 v[216:219], v[116:119], a[56:59], v[216:219]\n" \
  "ds_read_b64_tr_b8 v[110:111], v33 offset:41984\n" \
  "s_add_u32 m0, s47, 66560\n" \
  "v_mfma_i32_16x16x64_i8 v[248:251], v[112:115], a[44:47], v[248:251]\n" \
  "buffer_load_dwordx4 v41, s[16:19], s84 offen nt lds\n" \
  "ds_read_b128 a[104:107], v60 offset:16384\n" \
  "v_mfma_i32_16x16x64_i8 v[248:251], v[116:119], a[60:63], v[248:251]\n" \
  "s_waitcnt lgkmcnt(15)\n" \
  "v_mfma_i32_16x16x64_i8 v[156:159], v[120:123], a[32:35], v[156:159]\n" \
  "ds_read_b64_tr_b8 v[112:113], v34 offset:32768\n" \
  "ds_read_b128 a[108:111], v60 offset:18432\n" \
  "v_mfma_i32_16x16x64_i8 v[156:159], v[124:127], a[48:51], v[156:159]\n" \
  "ds_read_b64_tr_b8 v[114:115], v34 offset:33792\n" \
  "s_add_u32 m0, s47, 67584\n" \
  "v_mfma_i32_16x16x64_i8 v[188:191], v[120:123], a[36:39], v[188:191]\n" \
  "buffer_load_dwordx4 v42, s[16:19], s84 offen nt lds\n" \
  "s_lshl_b32 s86, s50, 21\n" \
  "s_lshl_b32 s94, s50, 16\n" \
  "s_add_u32 s49, s49, 1\n" \
  "ds_read_b128 a[112:115], v61 offset:0\n" \
  "v_mfma_i32_16x16x64_i8 v[188:191], v[124:127], a[52:55], v[188:191]\n" \
  "v_mfma_i32_16x16x64_i8 v[220:223], v[120:123], a[40:43], v[220:223]\n" \
  "ds_read_b64_tr_b8 v[116:117], v34 offset:40960\n" \
  "ds_read_b128 a[116:119], v61 offset:2048\n" \
  "v_mfma_i32_16x16x64_i8 v[220:223], v[124:127], a[56:59], v[220:223]\n" \
  "ds_read_b64_tr_b8 v[118:119], v34 offset:41984\n" \
  "s_add_u32 m0, s47, 68608\n" \
  "v_mfma_i32_16x16x64_i8 v[252:255], v[120:123], a[44:47], v[252:255]\n" \
  "buffer_load_dwordx4 v43, s[16:19], s84 offen nt lds\n" \
  "ds_read_b128 a[120:123], v61 offset:16384\n" \
  "v_mfma_i32_16x16x64_i8 v[252:255], v[124:127], a[60:63], v[252:255]\n" \
  "ds_read_b128 a[124:127], v61 offset:18432\n" \
  "; ---- step 2\n" \
  "s_cmp_lg_u32 s46, 1\n" \
  "s_cbranch_scc1 Lskip8B_%=\n" \
  "s_waitcnt vmcnt(55)\n" \
  "global_store_dwordx4 v49, v[64:67], s[34:35] sc0 sc1\n" \
  "s_nop 1\n" \
  "global_store_dwordx4 v79, v[68:71], s[34:35] sc0 sc1\n" \
  "s_nop 1\n" \
  "Lskip8B_%=:\n" \
  "s_waitcnt lgkmcnt(0)\n" \
  "v_mfma_i32_16x16x64_i8 v[128:131], v[96:99], a[64:67], v[128:131]\n" \
  "ds_read_b64_tr_b8 v[120:121], v35 offset:32768\n" \
  "s_add_u32 m0, s47, 81920\n" \
  "v_mfma_i32_16x16x64_i8 v[128:131], v[100:103], a[80:83], v[128:131]\n" \
  "ds_read_b64_tr_b8 v[122:123], v35 offset:33792\n" \
  "buffer_load_dwordx4 v40, s[16:19], s85 offen lds\n" \
  "s_add_u32 m0, s47, 82944\n" \
  "v_mfma_i32_16x16x64_i8 v[160:163], v[96:99], a[68:71], v[160:163]\n" \
  "buffer_load_dwordx4 v41, s[16:19], s85 offen lds\n" \
  "s_add_u32 m0, s47, 83968\n" \
  "v_mfma_i32_16x16x64_i8 v[160:163], v[100:103], a[84:87], v[160:163]\n" \
  "buffer_load_dwordx4 v42, s[16:19], s85 offen lds\n" \
  "s_add_u32 m0, s47, 84992\n" \
  "v_mfma_i32_16x16x64_i8 v[192:195], v[96:99], a[72:75], v[192:195]\n" \
  "ds_read_b64_tr_b8 v[124:125], v35 offset:40960\n" \
  "buffer_load_dwordx4 v43, s[16:19], s85 offen lds\n" \
  "s_add_u32 m0, s47, 98304\n" \
  "v_mfma_i32_16x16x64_i8 v[192:195], v[100:103], a[88:91], v[192:195]\n" \
  "ds_read_b64_tr_b8 v[126:127], v35 offset:41984\n" \
  "buffer_load_dwordx4 v40, s[16:19], s86 offen lds\n" \
  "s_add_u32 m0, s47, 99328\n" \
  "v_mfma_i32_16x16x64_i8 v[224:227], v[96:99], a[76:79], v[224:227]\n" \
  "buffer_load_dwordx4 v41, s[16:19], s86 offen lds\n" \
  "s_add_u32 m0, s47, 100352\n" \
  "v_mfma_i32_16x16x64_i8 v[224:227], v[100:103], a[92:95], v[224:227]\n" \
  "buffer_load_dwordx4 v42, s[16:19], s86 offen lds\n" \
  "s_add_u32 m0, s47, 101376\n" \
  "s_waitcnt lgkmcnt(14)\n" \
  "v_mfma_i32_16x16x64_i8 v[132:135], v[104:107], a[64:67], v[132:135]\n" \
  "ds_read_b64_tr_b8 v[96:97], v36 offset:32768\n" \
  "buffer_load_dwordx4 v43, s[16:19], s86 offen lds\n" \
  "v_mfma_i32_16x16x64_i8 v[132:135], v[108:111], a[80:83], v[132:135]\n" \
  "ds_read_b64_tr_b8 v[98:99], v36 offset:33792\n" \
  "v_mfma_i32_16x16x64_i8 v[164:167], v[104:107], a[68:71], v[164:167]\n" \
  "v_mfma_i32_16x16x64_i8 v[164:167], v[108:111], a[84:87], v[164:167]\n" \
  "v_mfma_i32_16x16x64_i8 v[196:199], v[104:107], a[72:75], v[196:199]\n" \
  "ds_read_b64_tr_b8 v[100:101], v36 offset:40960\n" \
  "v_mfma_i32_16x16x64_i8 v[196:199], v[108:111], a[88:91], v[196:199]\n" \
  "ds_read_b64_tr_b8 v[102:103], v36 offset:41984\n" \
  "v_mfma_i32_16x16x64_i8 v[228:231], v[104:107], a[76:79], v[228:231]\n" \
  "v_mfma_i32_16x16x64_i8 v[228:231], v[108:111], a[92:95], v[228:231]\n" \
  "s_waitcnt lgkmcnt(10)\n" \
  "v_mfma_i32_16x16x64_i8 v[136:139], v[112:115], a[64:67], v[136:139]\n" \
  "ds_read_b64_tr_b8 v[104:105], v37 offset:32768\n" \
  "v_readlane_b32 s98, v46, 30\n" \
  "v_readlane_b32 s99, v46, 31\n" \
  "v_mfma_i32_16x16x64_i8 v[136:139], v[116:119], a[80:83], v[136:139]\n" \
  "ds_read_b64_tr_b8 v[106:107], v37 offset:33792\n" \
  "v_mfma_i32_16x16x64_i8 v[168:171], v[112:115], a[68:71], v[168:171]\n" \
  "v_mfma_i32_16x16x64_i8 v[168:171], v[116:119], a[84:87], v[168:171]\n" \
  "v_mfma_i32_16x16x64_i8 v[200:203], v[112:115], a[72:75], v[200:203]\n" \
  "ds_read_b64_tr_b8 v[108:109], v37 offset:40960\n" \
  "v_mfma_i32_16x16x64_i8 v[200:203], v[116:119], a[88:91], v[200:203]\n" \
  "ds_read_b64_tr_b8 v[110:111], v37 offset:41984\n" \
  "v_mfma_i32_16x16x64_i8 v[232:235], v[112:115], a[76:79], v[232:235]\n" \
  "v_mfma_i32_16x16x64_i8 v[232:235], v[116:119], a[92:95], v[232:235]\n" \
  "s_waitcnt lgkmcnt(8)\n" \
  "v_mfma_i32_16x16x64_i8 v[140:143], v[120:123], a[64:67], v[140:143]\n" \
  "ds_read_b64_tr_b8 v[112:113], v38 offset:32768\n" \
  "v_mfma_i32_16x16x64_i8 v[140:143], v[124:127], a[80:83], v[140:143]\n" \
  "ds_read_b64_tr_b8 v[114:115], v38 offset:33792\n" \
  "v_mfma_i32_16x16x64_i8 v[172:175], v[120:123], a[68:71], v[172:175]\n" \
  "s_lshl_b32 s98, s98, 21\n" \
  "s_lshl_b32 s99, s99, 21\n" \
  "v_mfma_i32_16x16x64_i8 v[172:175], v[124:127], a[84:87], v[172:175]\n" \
  "s_waitcnt vmcnt(12)\n" \
  "s_barrier\n" \
  "v_mfma_i32_16x16x64_i8 v[204:207], v[120:123], a[72:75], v[204:207]\n" \
  "ds_read_b64_tr_b8 v[116:117], v38 offset:40960\n" \
  "v_mfma_i32_16x16x64_i8 v[204:207], v[124:127], a[88:91], v[204:207]\n" \
  "ds_read_b64_tr_b8 v[118:119], v38 offset:41984\n" \
  "v_mfma_i32_16x16x64_i8 v[236:239], v[120:123], a[76:79], v[236:239]\n" \
  "v_mfma_i32_16x16x64_i8 v[236:239], v[124:127], a[92:95], v[236:239]\n" \
  "s_waitcnt lgkmcnt(8)\n" \
  "v_mfma_i32_16x16x64_i8 v[144:147], v[96:99], a[64:67], v[144:147]\n" \
  "ds_read_b64_tr_b8 v[120:121], v39 offset:32768\n" \
  "v_mfma_i32_16x16x64_i8 v[144:147], v[100:103], a[80:83], v[144:147]\n" \
  "ds_read_b64_tr_b8 v[122:123], v39 offset:33792\n" \
  "v_mfma_i32_16x16x64_i8 v[176:179], v[96:99], a[68:71], v[176:179]\n" \
  "v_mfma_i32_16x16x64_i8 v[176:179], v[100:103], a[84:87], v[176:179]\n" \
  "v_mfma_i32_16x16x64_i8 v[208:211], v[96:99], a[72:75], v[208:211]\n" \
  "ds_read_b64_tr_b8 v[124:125], v39 offset:40960\n" \
  "v_mfma_i32_16x16x64_i8 v[208:211], v[100:103], a[88:91], v[208:211]\n" \
  "ds_read_b64_tr_b8 v[126:127], v39 offset:41984\n" \
  "v_mfma_i32_16x16x64_i8 v[240:243], v[96:99], a[76:79], v[240:243]\n" \
  "v_mfma_i32_16x16x64_i8 v[240:243], v[100:103], a[92:95], v[240:243]\n" \
  "s_waitcnt lgkmcnt(8)\n" \
  "v_mfma_i32_16x16x64_i8 v[148:151], v[104:107], a[64:67], v[148:151]\n" \
  "ds_read_b64_tr_b8 v[96:97], v32 offset:49152\n" \
  "v_mfma_i32_16x16x64_i8 v[148:151], v[108:111], a[80:83], v[148:151]\n" \
  "ds_read_b64_tr_b8 v[98:99], v32 offset:50176\n" \
  "v_mfma_i32_16x16x64_i8 v[180:183], v[104:107], a[68:71], v[180:183]\n" \
  "v_mfma_i32_16x16x64_i8 v[180:183], v[108:111], a[84:87], v[180:183]\n" \
  "v_mfma_i32_16x16x64_i8 v[212:215], v[104:107], a[72:75], v[212:215]\n" \
  "ds_read_b64_tr_b8 v[100:101], v32 offset:57344\n" \
  "v_mfma_i32_16x16x64_i8 v[212:215], v[108:111], a[88:91], v[212:215]\n" \
  "ds_read_b64_tr_b8 v[102:103], v32 offset:58368\n" \
  "v_mfma_i32_16x16x64_i8 v[244:247], v[104:107], a[76:79], v[244:247]\n" \
  "v_mfma_i32_16x16x64_i8 v[244:247], v[108:111], a[92:95], v[244:247]\n" \
  "s_waitcnt lgkmcnt(8)\n" \
  "v_mfma_i32_16x16x64_i8 v[152:155], v[112:115], a[64:67], v[152:155]\n" \
  "ds_read_b64_tr_b8 v[104:105], v33 offset:49152\n" \
  "v_mfma_i32_16x16x64_i8 v[152:155], v[116:119], a[80:83], v[152:155]\n" \
  "ds_read_b64_tr_b8 v[106:107], v33 offset:50176\n" \
  "v_mfma_i32_16x16x64_i8 v[184:187], v[112:115], a[68:71], v[184:187]\n" \
  "v_mfma_i32_16x16x64_i8 v[184:187], v[116:119], a[84:87], v[184:187]\n" \
  "v_mfma_i32_16x16x64_i8 v[216:219], v[112:115], a[72:75], v[216:219]\n" \
  "ds_read_b64_tr_b8 v[108:109], v33 offset:57344\n" \
  "v_mfma_i32_16x16x64_i8 v[216:219], v[116:119], a[88:91], v[216:219]\n" \
  "ds_read_b64_tr_b8 v[110:111], v33 offset:58368\n" \
  "v_mfma_i32_16x16x64_i8 v[248:251], v[112:115], a[76:79], v[248:251]\n" \
  "v_mfma_i32_16x16x64_i8 v[248:251], v[116:119], a[92:95], v[248:251]\n" \
  "s_waitcnt lgkmcnt(8)\n" \
  "v_mfma_i32_16x16x64_i8 v[156:159], v[120:123], a[64:67], v[156:159]\n" \
  "ds_read_b64_tr_b8 v[112:113], v34 offset:49152\n" \
  "v_mfma_i32_16x16x64_i8 v[156:159], v[124:127], a[80:83], v[156:159]\n" \
  "ds_read_b64_tr_b8 v[114:115], v34 offset:50176\n" \
  "v_mfma_i32_16x16x64_i8 v[188:191], v[120:123], a[68:71], v[188:191]\n" \
  "v_mfma_i32_16x16x64_i8 v[188:191], v[124:127], a[84:87], v[188:191]\n" \
  "v_mfma_i32_16x16x64_i8 v[220:223], v[120:123], a[72:75], v[220:223]\n" \
  "ds_read_b64_tr_b8 v[116:117], v34 offset:57344\n" \
  "v_mfma_i32_16x16x64_i8 v[220:223], v[124:127], a[88:91], v[220:223]\n" \
  "ds_read_b64_tr_b8 v[118:119], v34 offset:58368\n" \
  "v_mfma_i32_16x16x64_i8 v[252:255], v[120:123], a[76:79], v[252:255]\n" \
  "v_mfma_i32_16x16x64_i8 v[252:255], v[124:127], a[92:95], v[252:255]\n" \
  "; ---- step 3\n" \
  "s_waitcnt lgkmcnt(8)\n" \
  "v_mfma_i32_16x16x64_i8 v[128:131], v[96:99], a[96:99], v[128:131]\n" \
  "ds_read_b64_tr_b8 v[120:121], v35 offset:49152\n" \
  "v_mfma_i32_16x16x64_i8 v[128:131], v[100:103], a[112:115], v[128:131]\n" \
  "ds_read_b64_tr_b8 v[122:123], v35 offset:50176\n" \
  "v_mfma_i32_16x16x64_i8 v[160:163], v[96:99], a[100:103], v[160:163]\n" \
  "v_mfma_i32_16x16x64_i8 v[160:163], v[100:103], a[116:119], v[160:163]\n" \
  "v_mfma_i32_16x16x64_i8 v[192:195], v[96:99], a[104:107], v[192:195]\n" \
  "ds_read_b64_tr_b8 v[124:125], v35 offset:57344\n" \
  "s_add_u32 m0, s47, 131072\n" \
  "v_mfma_i32_16x16x64_i8 v[192:195], v[100:103], a[120:123], v[192:195]\n" \
  "ds_read_b64_tr_b8 v[126:127], v35 offset:58368\n" \
  "buffer_load_dwordx4 v40, s[16:19], s98 offen lds\n" \
  "s_add_u32 m0, s47, 132096\n" \
  "v_mfma_i32_16x16x64_i8 v[224:227], v[96:99], a[108:111], v[224:227]\n" \
  "buffer_load_dwordx4 v41, s[16:19], s98 offen lds\n" \
  "s_add_u32 m0, s47, 133120\n" \
  "v_mfma_i32_16x16x64_i8 v[224:227], v[100:103], a[124:127], v[224:227]\n" \
  "buffer_load_dwordx4 v42, s[16:19], s98 offen lds\n" \
  "s_add_u32 m0, s47, 134144\n" \
  "s_waitcnt lgkmcnt(8)\n" \
  "v_mfma_i32_16x16x64_i8 v[132:135], v[104:107], a[96:99], v[132:135]\n" \
  "ds_read_b64_tr_b8 v[96:97], v36 offset:49152\n" \
  "buffer_load_dwordx4 v43, s[16:19], s98 offen lds\n" \
  "s_add_u32 m0, s47, 147456\n" \
  "v_mfma_i32_16x16x64_i8 v[132:135], v[108:111], a[112:115], v[132:135]\n" \
  "ds_read_b64_tr_b8 v[98:99], v36 offset:50176\n" \
  "buffer_load_dwordx4 v40, s[16:19], s99 offen lds\n" \
  "s_add_u32 m0, s47, 148480\n" \
  "v_mfma_i32_16x16x64_i8 v[164:167], v[104:107], a[100:103], v[164:167]\n" \
  "buffer_load_dwordx4 v41, s[16:19], s99 offen lds\n" \
  "s_add_u32 m0, s47, 149504\n" \
  "v_mfma_i32_16x16x64_i8 v[164:167], v[108:111], a[116:119], v[164:167]\n" \
  "buffer_load_dwordx4 v42, s[16:19], s99 offen lds\n" \
  "s_add_u32 m0, s47, 150528\n" \
  "v_mfma_i32_16x16x64_i8 v[196:199], v[104:107], a[104:107], v[196:199]\n" \
  "ds_read_b64_tr_b8 v[100:101], v36 offset:57344\n" \
  "buffer_load_dwordx4 v43, s[16:19], s99 offen lds\n" \
  "v_mfma_i32_16x16x64_i8 v[196:199], v[108:111], a[120:123], v[196:199]\n" \
  "ds_read_b64_tr_b8 v[102:103], v36 offset:58368\n" \
  "v_mfma_i32_16x16x64_i8 v[228:231], v[104:107], a[108:111], v[228:231]\n" \
  "v_mfma_i32_16x16x64_i8 v[228:231], v[108:111], a[124:127], v[228:231]\n" \
  "s_waitcnt lgkmcnt(8)\n" \
  "v_mfma_i32_16x16x64_i8 v[136:139], v[112:115], a[96:99], v[136:139]\n" \
  "ds_read_b64_tr_b8 v[104:105], v37 offset:49152\n" \
  "v_mfma_i32_16x16x64_i8 v[136:139], v[116:119], a[112:115], v[136:139]\n" \
  "ds_read_b64_tr_b8 v[106:107], v37 offset:50176\n" \
  "v_mfma_i32_16x16x64_i8 v[168:171], v[112:115], a[100:103], v[168:171]\n" \
  "v_readlane_b32 s50, v46, s49\n" \
  "v_mfma_i32_16x16x64_i8 v[168:171], v[116:119], a[116:119], v[168:171]\n" \
  "v_mfma_i32_16x16x64_i8 v[200:203], v[112:115], a[104:107], v[200:203]\n" \
  "ds_read_b64_tr_b8 v[108:109], v37 offset:57344\n" \
  "v_mfma_i32_16x16x64_i8 v[200:203], v[116:119], a[120:123], v[200:203]\n" \
  "ds_read_b64_tr_b8 v[110:111], v37 offset:58368\n" \
  "v_mfma_i32_16x16x64_i8 v[232:235], v[112:115], a[108:111], v[232:235]\n" \
  "v_mfma_i32_16x16x64_i8 v[232:235], v[116:119], a[124:127], v[232:235]\n" \
  "s_waitcnt lgkmcnt(8)\n" \
  "v_mfma_i32_16x16x64_i8 v[140:143], v[120:123], a[96:99], v[140:143]\n" \
  "ds_read_b64_tr_b8 v[112:113], v38 offset:49152\n" \
  "v_mfma_i32_16x16x64_i8 v[140:143], v[124:127], a[112:115], v[140:143]\n" \
  "ds_read_b64_tr_b8 v[114:115], v38 offset:50176\n" \
  "v_mfma_i32_16x16x64_i8 v[172:175], v[120:123], a[100:103], v[172:175]\n" \
  "s_lshl_b32 s87, s50, 21\n" \
  "s_lshl_b32 s95, s50, 16\n" \
  "s_add_u32 s49, s49, 1\n" \
  "v_mfma_i32_16x16x64_i8 v[172:175], v[124:127], a[116:119], v[172:175]\n" \
  "s_cmp_lg_u32 s46, 1\n" \
  "s_cbranch_scc1 Lskip9B_%=\n" \
  "s_waitcnt vmcnt(16)\n" \
  "Lskip9B_%=:\n" \
  "s_waitcnt vmcnt(16)\n" \
  "s_barrier\n" \
  "s_cmp_lg_u32 s46, 1\n" \
  "s_cbranch_scc1 Lskip11B_%=\n" \
  "s_cmp_lg_u32 s11, 0\n" \
  "s_cbranch_scc1 Lskip10B_%=\n" \
  "s_mov_b64 exec, 1\n" \
  "global_store_dword v53, v76, s[36:37] sc0 sc1\n" \
  "s_mov_b64 exec, -1\n" \
  "Lskip10B_%=:\n" \
  "Lskip11B_%=:\n" \
  "v_mfma_i32_16x16x64_i8 v[204:207], v[120:123], a[104:107], v[204:207]\n" \
  "ds_read_b64_tr_b8 v[116:117], v38 offset:57344\n" \
  "v_mfma_i32_16x16x64_i8 v[204:207], v[124:127], a[120:123], v[204:207]\n" \
  "ds_read_b64_tr_b8 v[118:119], v38 offset:58368\n" \
  "v_mfma_i32_16x16x64_i8 v[236:239], v[120:123], a[108:111], v[236:239]\n" \
  "v_mfma_i32_16x16x64_i8 v[236:239], v[124:127], a[124:127], v[236:239]\n" \
  "s_waitcnt lgkmcnt(8)\n" \
  "v_mfma_i32_16x16x64_i8 v[144:147], v[96:99], a[96:99], v[144:147]\n" \
  "ds_read_b64_tr_b8 v[120:121], v39 offset:49152\n" \
  "v_mfma_i32_16x16x64_i8 v[144:147], v[100:103], a[112:115], v[144:147]\n" \
  "ds_read_b64_tr_b8 v[122:123], v39 offset:50176\n" \
  "s_add_u32 m0, s47, 114688\n" \
  "v_mfma_i32_16x16x64_i8 v[176:179], v[96:99], a[100:103], v[176:179]\n" \
  "buffer_load_dwordx4 v40, s[16:19], s87 offen nt lds\n" \
  "v_readlane_b32 s50, v46, s49\n" \
  "v_mfma_i32_16x16x64_i8 v[176:179], v[100:103], a[116:119], v[176:179]\n" \
  "v_mfma_i32_16x16x64_i8 v[208:211], v[96:99], a[104:107], v[208:211]\n" \
  "ds_read_b64_tr_b8 v[124:125], v39 offset:57344\n" \
  "v_mfma_i32_16x16x64_i8 v[208:211], v[100:103], a[120:123], v[208:211]\n" \
  "ds_read_b64_tr_b8 v[126:127], v39 offset:58368\n" \
  "s_add_u32 m0, s47, 115712\n" \
  "v_mfma_i32_16x16x64_i8 v[240:243], v[96:99], a[108:111], v[240:243]\n" \
  "buffer_load_dwordx4 v41, s[16:19], s87 offen nt lds\n" \
  "v_mfma_i32_16x16x64_i8 v[240:243], v[100:103], a[124:127], v[240:243]\n" \
  "s_waitcnt lgkmcnt(8)\n" \
  "v_mfma_i32_16x16x64_i8 v[148:151], v[104:107], a[96:99], v[148:151]\n" \
  "ds_read_b64_tr_b8 v[96:97], v80 offset:0\n" \
  "v_mfma_i32_16x16x64_i8 v[148:151], v[108:111], a[112:115], v[148:151]\n" \
  "ds_read_b64_tr_b8 v[98:99], v80 offset:1024\n" \
  "s_add_u32 m0, s47, 116736\n" \
  "v_mfma_i32_16x16x64_i8 v[180:183], v[104:107], a[100:103], v[180:183]\n" \
  "buffer_load_dwordx4 v42, s[16:19], s87 offen nt lds\n" \
  "s_lshl_b32 s80, s50, 21\n" \
  "s_lshl_b32 s88, s50, 16\n" \
  "s_add_u32 s49, s49, 1\n" \
  "v_mfma_i32_16x16x64_i8 v[180:183], v[108:111], a[116:119], v[180:183]\n" \
  "v_mfma_i32_16x16x64_i8 v[212:215], v[104:107], a[104:107], v[212:215]\n" \
  "ds_read_b64_tr_b8 v[100:101], v80 offset:8192\n" \
  "v_mfma_i32_16x16x64_i8 v[212:215], v[108:111], a[120:123], v[212:215]\n" \
  "ds_read_b64_tr_b8 v[102:103], v80 offset:9216\n" \
  "s_add_u32 m0, s47, 117760\n" \
  "v_mfma_i32_16x16x64_i8 v[244:247], v[104:107], a[108:111], v[244:247]\n" \
  "buffer_load_dwordx4 v43, s[16:19], s87 offen nt lds\n" \
  "v_mfma_i32_16x16x64_i8 v[244:247], v[108:111], a[124:127], v[244:247]\n" \
  "s_waitcnt lgkmcnt(8)\n" \
  "v_mfma_i32_16x16x64_i8 v[152:155], v[112:115], a[96:99], v[152:155]\n" \
  "ds_read_b64_tr_b8 v[104:105], v81 offset:0\n" \
  "v_mfma_i32_16x16x64_i8 v[152:155], v[116:119], a[112:115], v[152:155]\n" \
  "ds_read_b64_tr_b8 v[106:107], v81 offset:1024\n" \
  "s_add_u32 m0, s47, 0\n" \
  "v_mfma_i32_16x16x64_i8 v[184:187], v[112:115], a[100:103], v[184:187]\n" \
  "buffer_load_dwordx4 v40, s[16:19], s80 offen nt lds\n" \
  "v_mfma_i32_16x16x64_i8 v[184:187], v[116:119], a[116:119], v[184:187]\n" \
  "v_mfma_i32_16x16x64_i8 v[216:219], v[112:115], a[104:107], v[216:219]\n" \
  "ds_read_b64_tr_b8 v[108:109], v81 offset:8192\n" \
  "v_mfma_i32_16x16x64_i8 v[216:219], v[116:119], a[120:123], v[216:219]\n" \
  "ds_read_b64_tr_b8 v[110:111], v81 offset:9216\n" \
  "s_add_u32 m0, s47, 1024\n" \
  "v_mfma_i32_16x16x64_i8 v[248:251], v[112:115], a[108:111], v[248:251]\n" \
  "buffer_load_dwordx4 v41, s[16:19], s80 offen nt lds\n" \
  "v_mfma_i32_16x16x64_i8 v[248:251], v[116:119], a[124:127], v[248:251]\n" \
  "s_waitcnt lgkmcnt(8)\n" \
  "v_mfma_i32_16x16x64_i8 v[156:159], v[120:123], a[96:99], v[156:159]\n" \
  "ds_read_b64_tr_b8 v[112:113], v82 offset:0\n" \
  "v_mfma_i32_16x16x64_i8 v[156:159], v[124:127], a[112:115], v[156:159]\n" \
  "ds_read_b64_tr_b8 v[114:115], v82 offset:1024\n" \
  "s_add_u32 m0, s47, 2048\n" \
  "v_mfma_i32_16x16x64_i8 v[188:191], v[120:123], a[100:103], v[188:191]\n" \
  "buffer_load_dwordx4 v42, s[16:19], s80 offen nt lds\n" \
  "v_mfma_i32_16x16x64_i8 v[188:191], v[124:127], a[116:119], v[188:191]\n" \
  "v_mfma_i32_16x16x64_i8 v[220:223], v[120:123], a[104:107], v[220:223]\n" \
  "ds_read_b64_tr_b8 v[116:117], v82 offset:8192\n" \
  "v_mfma_i32_16x16x64_i8 v[220:223], v[124:127], a[120:123], v[220:223]\n" \
  "ds_read_b64_tr_b8 v[118:119], v82 offset:9216\n" \
  "s_add_u32 m0, s47, 3072\n" \
  "v_mfma_i32_16x16x64_i8 v[252:255], v[120:123], a[108:111], v[252:255]\n" \
  "buffer_load_dwordx4 v43, s[16:19], s80 offen nt lds\n" \
  "v_mfma_i32_16x16x64_i8 v[252:255], v[124:127], a[124:127], v[252:255]\n" \
  "s_cmp_lg_u32 s11, 0\n" \
  "s_cbranch_scc1 Lskip14B_%=\n" \
  "global_load_dword v72, v50, s[38:39] sc0 sc1\n" \
  "s_waitcnt vmcnt(0)\n" \
  "v_sub_u32 v72, v72, v76\n" \
  "v_cmp_gt_i32 vcc, 0, v72\n" \
  "s_nop 1\n" \
  "s_andn2_b64 vcc, vcc, s[14:15]\n" \
  "s_cbranch_scc1 Lslow12B_%=\n" \
  "Lback13B_%=:\n" \
  "v_add_u32 v50, 1024, v50\n" \
  "Lskip14B_%=:\n" \
  "s_barrier\n" \
  "buffer_load_dwordx4 a[128:131], v44, s[28:31], s92 offen offset:0 sc0 sc1\n" \
  "buffer_load_dwordx4 a[132:135], v44, s[28:31], s92 offen offset:1024 sc0 sc1\n" \
  "buffer_load_dwordx4 a[136:139], v44, s[28:31], s92 offen offset:2048 sc0 sc1\n" \
  "buffer_load_dwordx4 a[140:143], v44, s[28:31], s92 offen offset:3072 sc0 sc1\n" \
  "buffer_load_dword v92, v52, s[28:31], 0 offen sc0 sc1\n" \
  "v_add_u32 v52, 0x10000, v52\n" \
  "buffer_load_dwordx4 a[144:147], v91, s[28:31], s92 offen offset:0 sc0 sc1\n" \
  "buffer_load_dwordx4 a[148:151], v91, s[28:31], s92 offen offset:1024 sc0 sc1\n" \
  "buffer_load_dwordx4 a[152:155], v91, s[28:31], s92 offen offset:2048 sc0 sc1\n" \
  "buffer_load_dwordx4 a[156:159], v91, s[28:31], s92 offen offset:3072 sc0 sc1\n" \
  "; ---- step 4\n" \
  "s_waitcnt vmcnt(0)\n" \
  "s_waitcnt lgkmcnt(8)\n" \
  "v_mfma_i32_16x16x64_i8 v[128:131], v[96:99], a[128:131], v[128:131]\n" \
  "ds_read_b64_tr_b8 v[120:121], v83 offset:0\n" \
  "v_mfma_i32_16x16x64_i8 v[128:131], v[100:103], a[144:147], v[128:131]\n" \
  "ds_read_b64_tr_b8 v[122:123], v83 offset:1024\n" \
  "buffer_load_dwordx4 a[160:163], v44, s[28:31], s93 offen offset:0 sc0 sc1\n" \
  "v_mfma_i32_16x16x64_i8 v[160:163], v[96:99], a[132:135], v[160:163]\n" \
  "v_mfma_i32_16x16x64_i8 v[160:163], v[100:103], a[148:151], v[160:163]\n" \
  "buffer_load_dwordx4 a[164:167], v44, s[28:31], s93 offen offset:1024 sc0 sc1\n" \
  "v_mfma_i32_16x16x64_i8 v[192:195], v[96:99], a[136:139], v[192:195]\n" \
  "ds_read_b64_tr_b8 v[124:125], v83 offset:8192\n" \
  "v_mfma_i32_16x16x64_i8 v[192:195], v[100:103], a[152:155], v[192:195]\n" \
  "ds_read_b64_tr_b8 v[126:127], v83 offset:9216\n" \
  "buffer_load_dwordx4 a[168:171], v44, s[28:31], s93 offen offset:2048 sc0 sc1\n" \
  "v_mfma_i32_16x16x64_i8 v[224:227], v[96:99], a[140:143], v[224:227]\n" \
  "v_mfma_i32_16x16x64_i8 v[224:227], v[100:103], a[156:159], v[224:227]\n" \
  "s_waitcnt lgkmcnt(8)\n" \
  "v_mfma_i32_16x16x64_i8 v[132:135], v[104:107], a[128:131], v[132:135]\n" \
  "ds_read_b64_tr_b8 v[96:97], v84 offset:0\n" \
  "v_mfma_i32_16x16x64_i8 v[132:135], v[108:111], a[144:147], v[132:135]\n" \
  "ds_read_b64_tr_b8 v[98:99], v84 offset:1024\n" \
  "buffer_load_dwordx4 a[172:175], v44, s[28:31], s93 offen offset:3072 sc0 sc1\n" \
  "v_mfma_i32_16x16x64_i8 v[164:167], v[104:107], a[132:135], v[164:167]\n" \
  "v_mfma_i32_16x16x64_i8 v[164:167], v[108:111], a[148:151], v[164:167]\n" \
  "buffer_load_dwordx4 a[176:179], v91, s[28:31], s93 offen offset:0 sc0 sc1\n" \
  "v_mfma_i32_16x16x64_i8 v[196:199], v[104:107], a[136:139], v[196:199]\n" \
  "ds_read_b64_tr_b8 v[100:101], v84 offset:8192\n" \
  "v_mfma_i32_16x16x64_i8 v[196:199], v[108:111], a[152:155], v[196:199]\n" \
  "ds_read_b64_tr_b8 v[102:103], v84 offset:9216\n" \
  "buffer_load_dwordx4 a[180:183], v91, s[28:31], s93 offen offset:1024 sc0 sc1\n" \
  "v_mfma_i32_16x16x64_i8 v[228:231], v[104:107], a[140:143], v[228:231]\n" \
  "v_mfma_i32_16x16x64_i8 v[228:231], v[108:111], a[156:159], v[228:231]\n" \
  "s_waitcnt lgkmcnt(8)\n" \
  "v_mfma_i32_16x16x64_i8 v[136:139], v[112:115], a[128:131], v[136:139]\n" \
  "ds_read_b64_tr_b8 v[104:105], v85 offset:0\n" \
  "v_mfma_i32_16x16x64_i8 v[136:139], v[116:119], a[144:147], v[136:139]\n" \
  "ds_read_b64_tr_b8 v[106:107], v85 offset:1024\n" \
  "buffer_load_dwordx4 a[184:187], v91, s[28:31], s93 offen offset:2048 sc0 sc1\n" \
  "v_mfma_i32_16x16x64_i8 v[168:171], v[112:115], a[132:135], v[168:171]\n" \
  "v_mfma_i32_16x16x64_i8 v[168:171], v[116:119], a[148:151], v[168:171]\n" \
  "buffer_load_dwordx4 a[188:191], v91, s[28:31], s93 offen offset:3072 sc0 sc1\n" \
  "v_mfma_i32_16x16x64_i8 v[200:203], v[112:115], a[136:139], v[200:203]\n" \
  "ds_read_b64_tr_b8 v[108:109], v85 offset:8192\n" \
  "v_mfma_i32_16x16x64_i8 v[200:203], v[116:119], a[152:155], v[200:203]\n" \
  "ds_read_b64_tr_b8 v[110:111], v85 offset:9216\n" \
  "v_mfma_i32_16x16x64_i8 v[232:235], v[112:115], a[140:143], v[232:235]\n" \
  "v_mfma_i32_16x16x64_i8 v[232:235], v[116:119], a[156:159], v[232:235]\n" \
  "buffer_load_dwordx4 a[192:195], v44, s[28:31], s94 offen offset:0 sc0 sc1\n" \
  "s_waitcnt lgkmcnt(8)\n" \
  "v_mfma_i32_16x16x64_i8 v[140:143], v[120:123], a[128:131], v[140:143]\n" \
  "ds_read_b64_tr_b8 v[112:113], v86 offset:0\n" \
  "v_mfma_i32_16x16x64_i8 v[140:143], v[124:127], a[144:147], v[140:143]\n" \
  "ds_read_b64_tr_b8 v[114:115], v86 offset:1024\n" \
  "buffer_load_dwordx4 a[196:199], v44, s[28:31], s94 offen offset:1024 sc0 sc1\n" \
  "v_mfma_i32_16x16x64_i8 v[172:175], v[120:123], a[132:135], v[172:175]\n" \
  "v_mfma_i32_16x16x64_i8 v[172:175], v[124:127], a[148:151], v[172:175]\n" \
  "buffer_load_dwordx4 a[200:203], v44, s[28:31], s94 offen offset:2048 sc0 sc1\n" \
  "s_waitcnt vmcnt(40)\n" \
  "s_barrier\n" \
  "v_mfma_i32_16x16x64_i8 v[204:207], v[120:123], a[136:139], v[204:207]\n" \
  "ds_read_b64_tr_b8 v[116:117], v86 offset:8192\n" \
  "v_mfma_i32_16x16x64_i8 v[204:207], v[124:127], a[152:155], v[204:207]\n" \
  "ds_read_b64_tr_b8 v[118:119], v86 offset:9216\n" \
  "v_mfma_i32_16x16x64_i8 v[236:239], v[120:123], a[140:143], v[236:239]\n" \
  "v_mfma_i32_16x16x64_i8 v[236:239], v[124:127], a[156:159], v[236:239]\n" \
  "buffer_load_dwordx4 a[204:207], v44, s[28:31], s94 offen offset:3072 sc0 sc1\n" \
  "s_waitcnt lgkmcnt(8)\n" \
  "v_mfma_i32_16x16x64_i8 v[144:147], v[96:99], a[128:131], v[144:147]\n" \
  "ds_read_b64_tr_b8 v[120:121], v87 offset:0\n" \
  "v_mfma_i32_16x16x64_i8 v[144:147], v[100:103], a[144:147], v[144:147]\n" \
  "ds_read_b64_tr_b8 v[122:123], v87 offset:1024\n" \
  "buffer_load_dwordx4 a[208:211], v91, s[28:31], s94 offen offset:0 sc0 sc1\n" \
  "v_mfma_i32_16x16x64_i8 v[176:179], v[96:99], a[132:135], v[176:179]\n" \
  "v_mfma_i32_16x16x64_i8 v[176:179], v[100:103], a[148:151], v[176:179]\n" \
  "buffer_load_dwordx4 a[212:215], v91, s[28:31], s94 offen offset:1024 sc0 sc1\n" \
  "v_mfma_i32_16x16x64_i8 v[208:211], v[96:99], a[136:139], v[208:211]\n" \
  "ds_read_b64_tr_b8 v[124:125], v87 offset:8192\n" \
  "v_mfma_i32_16x16x64_i8 v[208:211], v[100:103], a[152:155], v[208:211]\n" \
  "ds_read_b64_tr_b8 v[126:127], v87 offset:9216\n" \
  "v_mfma_i32_16x16x64_i8 v[240:243], v[96:99], a[140:143], v[240:243]\n" \
  "v_mfma_i32_16x16x64_i8 v[240:243], v[100:103], a[156:159], v[240:243]\n" \
  "buffer_load_dwordx4 a[216:219], v91, s[28:31], s94 offen offset:2048 sc0 sc1\n" \
  "s_waitcnt lgkmcnt(8)\n" \
  "v_mfma_i32_16x16x64_i8 v[148:151], v[104:107], a[128:131], v[148:151]\n" \
  "ds_read_b64_tr_b8 v[96:97], v80 offset:16384\n" \
  "v_mfma_i32_16x16x64_i8 v[148:151], v[108:111], a[144:147], v[148:151]\n" \
  "ds_read_b64_tr_b8 v[98:99], v80 offset:17408\n" \
  "buffer_load_dwordx4 a[220:223], v91, s[28:31], s94 offen offset:3072 sc0 sc1\n" \
  "v_mfma_i32_16x16x64_i8 v[180:183], v[104:107], a[132:135], v[180:183]\n" \
  "v_mfma_i32_16x16x64_i8 v[180:183], v[108:111], a[148:151], v[180:183]\n" \
  "v_mfma_i32_16x16x64_i8 v[212:215], v[104:107], a[136:139], v[212:215]\n" \
  "ds_read_b64_tr_b8 v[100:101], v80 offset:24576\n" \
  "v_mfma_i32_16x16x64_i8 v[212:215], v[108:111], a[152:155], v[212:215]\n" \
  "ds_read_b64_tr_b8 v[102:103], v80 offset:25600\n" \
  "v_mfma_i32_16x16x64_i8 v[244:247], v[104:107], a[140:143], v[244:247]\n" \
  "v_mfma_i32_16x16x64_i8 v[244:247], v[108:111], a[156:159], v[244:247]\n" \
  "s_waitcnt lgkmcnt(8)\n" \
  "v_mfma_i32_16x16x64_i8 v[152:155], v[112:115], a[128:131], v[152:155]\n" \
  "ds_read_b64_tr_b8 v[104:105], v81 offset:16384\n" \
  "v_mfma_i32_16x16x64_i8 v[152:155], v[116:119], a[144:147], v[152:155]\n" \
  "ds_read_b64_tr_b8 v[106:107], v81 offset:17408\n" \
  "v_mfma_i32_16x16x64_i8 v[184:187], v[112:115], a[132:135], v[184:187]\n" \
  "v_mfma_i32_16x16x64_i8 v[184:187], v[116:119], a[148:151], v[184:187]\n" \
  "v_mfma_i32_16x16x64_i8 v[216:219], v[112:115], a[136:139], v[216:219]\n" \
  "ds_read_b64_tr_b8 v[108:109], v81 offset:24576\n" \
  "v_mfma_i32_16x16x64_i8 v[216:219], v[116:119], a[152:155], v[216:219]\n" \
  "ds_read_b64_tr_b8 v[110:111], v81 offset:25600\n" \
  "v_mfma_i32_16x16x64_i8 v[248:251], v[112:115], a[140:143], v[248:251]\n" \
  "v_mfma_i32_16x16x64_i8 v[248:251], v[116:119], a[156:159], v[248:251]\n" \
  "s_waitcnt lgkmcnt(8)\n" \
  "v_mfma_i32_16x16x64_i8 v[156:159], v[120:123], a[128:131], v[156:159]\n" \
  "ds_read_b64_tr_b8 v[112:113], v82 offset:16384\n" \
  "v_mfma_i32_16x16x64_i8 v[156:159], v[124:127], a[144:147], v[156:159]\n" \
  "ds_read_b64_tr_b8 v[114:115], v82 offset:17408\n" \
  "v_mfma_i32_16x16x64_i8 v[188:191], v[120:123], a[132:135], v[188:191]\n" \
  "v_mfma_i32_16x16x64_i8 v[188:191], v[124:127], a[148:151], v[188:191]\n" \
  "v_mfma_i32_16x16x64_i8 v[220:223], v[120:123], a[136:139], v[220:223]\n" \
  "ds_read_b64_tr_b8 v[116:117], v82 offset:24576\n" \
  "v_mfma_i32_16x16x64_i8 v[220:223], v[124:127], a[152:155], v[220:223]\n" \
  "ds_read_b64_tr_b8 v[118:119], v82 offset:25600\n" \
  "v_mfma_i32_16x16x64_i8 v[252:255], v[120:123], a[140:143], v[252:255]\n" \
  "v_mfma_i32_16x16x64_i8 v[252:255], v[124:127], a[156:159], v[252:255]\n" \
  "; ---- step 5\n" \
  "s_waitcnt vmcnt(8)\n" \
  "s_waitcnt lgkmcnt(8)\n" \
  "v_mfma_i32_16x16x64_i8 v[128:131], v[96:99], a[160:163], v[128:131]\n" \
  "ds_read_b64_tr_b8 v[120:121], v83 offset:16384\n" \
  "v_mfma_i32_16x16x64_i8 v[128:131], v[100:103], a[176:179], v[128:131]\n" \
  "ds_read_b64_tr_b8 v[122:123], v83 offset:17408\n" \
  "v_mfma_i32_16x16x64_i8 v[160:163], v[96:99], a[164:167], v[160:163]\n" \
  "v_mfma_i32_16x16x64_i8 v[160:163], v[100:103], a[180:183], v[160:163]\n" \
  "v_mfma_i32_16x16x64_i8 v[192:195], v[96:99], a[168:171], v[192:195]\n" \
  "ds_read_b64_tr_b8 v[124:125], v83 offset:24576\n" \
  "v_mfma_i32_16x16x64_i8 v[192:195], v[100:103], a[184:187], v[192:195]\n" \
  "ds_read_b64_tr_b8 v[126:127], v83 offset:25600\n" \
  "v_mfma_i32_16x16x64_i8 v[224:227], v[96:99], a[172:175], v[224:227]\n" \
  "v_mfma_i32_16x16x64_i8 v[224:227], v[100:103], a[188:191], v[224:227]\n" \
  "s_waitcnt lgkmcnt(8)\n" \
  "v_mfma_i32_16x16x64_i8 v[132:135], v[104:107], a[160:163], v[132:135]\n" \
  "ds_read_b64_tr_b8 v[96:97], v84 offset:16384\n" \
  "v_mfma_i32_16x16x64_i8 v[132:135], v[108:111], a[176:179], v[132:135]\n" \
  "ds_read_b64_tr_b8 v[98:99], v84 offset:17408\n" \
  "v_mfma_i32_16x16x64_i8 v[164:167], v[104:107], a[164:167], v[164:167]\n" \
  "v_mfma_i32_16x16x64_i8 v[164:167], v[108:111], a[180:183], v[164:167]\n" \
  "v_mfma_i32_16x16x64_i8 v[196:199], v[104:107], a[168:171], v[196:199]\n" \
  "ds_read_b64_tr_b8 v[100:101], v84 offset:24576\n" \
  "v_mfma_i32_16x16x64_i8 v[196:199], v[108:111], a[184:187], v[196:199]\n" \
  "ds_read_b64_tr_b8 v[102:103], v84 offset:25600\n" \
  "v_mfma_i32_16x16x64_i8 v[228:231], v[104:107], a[172:175], v[228:231]\n" \
  "v_readlane_b32 s50, v46, s49\n" \
  "v_mfma_i32_16x16x64_i8 v[228:231], v[108:111], a[188:191], v[228:231]\n" \
  "s_waitcnt lgkmcnt(8)\n" \
  "v_mfma_i32_16x16x64_i8 v[136:139], v[112:115], a[160:163], v[136:139]\n" \
  "ds_read_b64_tr_b8 v[104:105], v85 offset:16384\n" \
  "v_mfma_i32_16x16x64_i8 v[136:139], v[116:119], a[176:179], v[136:139]\n" \
  "ds_read_b64_tr_b8 v[106:107], v85 offset:17408\n" \
  "buffer_load_dwordx4 a[224:227], v44, s[28:31], s95 offen offset:0 sc0 sc1\n" \
  "v_mfma_i32_16x16x64_i8 v[168:171], v[112:115], a[164:167], v[168:171]\n" \
  "v_mfma_i32_16x16x64_i8 v[168:171], v[116:119], a[180:183], v[168:171]\n" \
  "buffer_load_dwordx4 a[228:231], v44, s[28:31], s95 offen offset:1024 sc0 sc1\n" \
  "v_mfma_i32_16x16x64_i8 v[200:203], v[112:115], a[168:171], v[200:203]\n" \
  "ds_read_b64_tr_b8 v[108:109], v85 offset:24576\n" \
  "v_mfma_i32_16x16x64_i8 v[200:203], v[116:119], a[184:187], v[200:203]\n" \
  "ds_read_b64_tr_b8 v[110:111], v85 offset:25600\n" \
  "v_mfma_i32_16x16x64_i8 v[232:235], v[112:115], a[172:175], v[232:235]\n" \
  "s_lshl_b32 s81, s50, 21\n" \
  "s_lshl_b32 s89, s50, 16\n" \
  "s_add_u32 s49, s49, 1\n" \
  "v_mfma_i32_16x16x64_i8 v[232:235], v[116:119], a[188:191], v[232:235]\n" \
  "buffer_load_dwordx4 a[232:235], v44, s[28:31], s95 offen offset:2048 sc0 sc1\n" \
  "s_waitcnt lgkmcnt(8)\n" \
  "v_mfma_i32_16x16x64_i8 v[140:143], v[120:123], a[160:163], v[140:143]\n" \
  "ds_read_b64_tr_b8 v[112:113], v86 offset:16384\n" \
  "v_mfma_i32_16x16x64_i8 v[140:143], v[124:127], a[176:179], v[140:143]\n" \
  "ds_read_b64_tr_b8 v[114:115], v86 offset:17408\n" \
  "buffer_load_dwordx4 a[236:239], v44, s[28:31], s95 offen offset:3072 sc0 sc1\n" \
  "v_mfma_i32_16x16x64_i8 v[172:175], v[120:123], a[164:167], v[172:175]\n" \
  "v_readlane_b32 s50, v46, s49\n" \
  "v_mfma_i32_16x16x64_i8 v[172:175], v[124:127], a[180:183], v[172:175]\n" \
  "s_waitcnt vmcnt(45)\n" \
  "s_barrier\n" \
  "v_mfma_i32_16x16x64_i8 v[204:207], v[120:123], a[168:171], v[204:207]\n" \
  "ds_read_b64_tr_b8 v[116:117], v86 offset:24576\n" \
  "v_mfma_i32_16x16x64_i8 v[204:207], v[124:127], a[184:187], v[204:207]\n" \
  "ds_read_b64_tr_b8 v[118:119], v86 offset:25600\n" \
  "buffer_load_dwordx4 a[240:243], v91, s[28:31], s95 offen offset:0 sc0 sc1\n" \
  "v_mfma_i32_16x16x64_i8 v[236:239], v[120:123], a[172:175], v[236:239]\n" \
  "v_mfma_i32_16x16x64_i8 v[236:239], v[124:127], a[188:191], v[236:239]\n" \
  "buffer_load_dwordx4 a[244:247], v91, s[28:31], s95 offen offset:1024 sc0 sc1\n" \
  "s_waitcnt lgkmcnt(8)\n" \
  "v_mfma_i32_16x16x64_i8 v[144:147], v[96:99], a[160:163], v[144:147]\n" \
  "ds_read_b64_tr_b8 v[120:121], v87 offset:16384\n" \
  "v_mfma_i32_16x16x64_i8 v[144:147], v[100:103], a[176:179], v[144:147]\n" \
  "ds_read_b64_tr_b8 v[122:123], v87 offset:17408\n" \
  "s_add_u32 m0, s47, 16384\n" \
  "v_mfma_i32_16x16x64_i8 v[176:179], v[96:99], a[164:167], v[176:179]\n" \
  "buffer_load_dwordx4 v40, s[16:19], s81 offen nt lds\n" \
  "s_lshl_b32 s82, s50, 21\n" \
  "s_lshl_b32 s90, s50, 16\n" \
  "s_add_u32 s49, s49, 1\n" \
  "v_mfma_i32_16x16x64_i8 v[176:179], v[100:103], a[180:183], v[176:179]\n" \
  "buffer_load_dwordx4 a[248:251], v91, s[28:31], s95 offen offset:2048 sc0 sc1\n" \
  "v_mfma_i32_16x16x64_i8 v[208:211], v[96:99], a[168:171], v[208:211]\n" \
  "ds_read_b64_tr_b8 v[124:125], v87 offset:24576\n" \
  "v_mfma_i32_16x16x64_i8 v[208:211], v[100:103], a[184:187], v[208:211]\n" \
  "ds_read_b64_tr_b8 v[126:127], v87 offset:25600\n" \
  "s_add_u32 m0, s47, 17408\n" \
  "buffer_load_dwordx4 a[252:255], v91, s[28:31], s95 offen offset:3072 sc0 sc1\n" \
  "v_mfma_i32_16x16x64_i8 v[240:243], v[96:99], a[172:175], v[240:243]\n" \
  "buffer_load_dwordx4 v41, s[16:19], s81 offen nt lds\n" \
  "v_readlane_b32 s50, v46, s49\n" \
  "v_mfma_i32_16x16x64_i8 v[240:243], v[100:103], a[188:191], v[240:243]\n" \
  "s_waitcnt lgkmcnt(8)\n" \
  "v_mfma_i32_16x16x64_i8 v[148:151], v[104:107], a[160:163], v[148:151]\n" \
  "ds_read_b64_tr_b8 v[96:97], v80 offset:32768\n" \
  "v_mfma_i32_16x16x64_i8 v[148:151], v[108:111], a[176:179], v[148:151]\n" \
  "ds_read_b64_tr_b8 v[98:99], v80 offset:33792\n" \
  "s_add_u32 m0, s47, 18432\n" \
  "buffer_load_dwordx4 a[0:3], v44, s[28:31], s88 offen offset:0 sc0 sc1\n" \
  "v_mfma_i32_16x16x64_i8 v[180:183], v[104:107], a[164:167], v[180:183]\n" \
  "buffer_load_dwordx4 v42, s[16:19], s81 offen nt lds\n" \
  "v_mfma_i32_16x16x64_i8 v[180:183], v[108:111], a[180:183], v[180:183]\n" \
  "buffer_load_dwordx4 a[4:7], v44, s[28:31], s88 offen offset:1024 sc0 sc1\n" \
  "v_mfma_i32_16x16x64_i8 v[212:215], v[104:107], a[168:171], v[212:215]\n" \
  "ds_read_b64_tr_b8 v[100:101], v80 offset:40960\n" \
  "v_mfma_i32_16x16x64_i8 v[212:215], v[108:111], a[184:187], v[212:215]\n" \
  "ds_read_b64_tr_b8 v[102:103], v80 offset:41984\n" \
  "s_add_u32 m0, s47, 19456\n" \
  "v_mfma_i32_16x16x64_i8 v[244:247], v[104:107], a[172:175], v[244:247]\n" \
  "buffer_load_dwordx4 v43, s[16:19], s81 offen nt lds\n" \
  "s_lshl_b32 s83, s50, 21\n" \
  "s_lshl_b32 s91, s50, 16\n" \
  "s_add_u32 s49, s49, 1\n" \
  "v_mfma_i32_16x16x64_i8 v[244:247], v[108:111], a[188:191], v[244:247]\n" \
  "buffer_load_dwordx4 a[8:11], v44, s[28:31], s88 offen offset:2048 sc0 sc1\n" \
  "s_waitcnt lgkmcnt(8)\n" \
  "v_mfma_i32_16x16x64_i8 v[152:155], v[112:115], a[160:163], v[152:155]\n" \
  "ds_read_b64_tr_b8 v[104:105], v81 offset:32768\n" \
  "v_mfma_i32_16x16x64_i8 v[152:155], v[116:119], a[176:179], v[152:155]\n" \
  "ds_read_b64_tr_b8 v[106:107], v81 offset:33792\n" \
  "s_add_u32 m0, s47, 32768\n" \
  "buffer_load_dwordx4 a[12:15], v44, s[28:31], s88 offen offset:3072 sc0 sc1\n" \
  "v_mfma_i32_16x16x64_i8 v[184:187], v[112:115], a[164:167], v[184:187]\n" \
  "buffer_load_dwordx4 v40, s[16:19], s82 offen nt lds\n" \
  "v_readlane_b32 s50, v46, s49\n" \
  "v_mfma_i32_16x16x64_i8 v[184:187], v[116:119], a[180:183], v[184:187]\n" \
  "v_mfma_i32_16x16x64_i8 v[216:219], v[112:115], a[168:171], v[216:219]\n" \
  "ds_read_b64_tr_b8 v[108:109], v81 offset:40960\n" \
  "v_mfma_i32_16x16x64_i8 v[216:219], v[116:119], a[184:187], v[216:219]\n" \
  "ds_read_b64_tr_b8 v[110:111], v81 offset:41984\n" \
  "s_add_u32 m0, s47, 33792\n" \
  "buffer_load_dwordx4 a[16:19], v91, s[28:31], s88 offen offset:0 sc0 sc1\n" \
  "v_mfma_i32_16x16x64_i8 v[248:251], v[112:115], a[172:175], v[248:251]\n" \
  "buffer_load_dwordx4 v41, s[16:19], s82 offen nt lds\n" \
  "v_mfma_i32_16x16x64_i8 v[248:251], v[116:119], a[188:191], v[248:251]\n" \
  "buffer_load_dwordx4 a[20:23], v91, s[28:31], s88 offen offset:1024 sc0 sc1\n" \
  "s_waitcnt lgkmcnt(8)\n" \
  "v_mfma_i32_16x16x64_i8 v[156:159], v[120:123], a[160:163], v[156:159]\n" \
  "ds_read_b64_tr_b8 v[112:113], v82 offset:32768\n" \
  "v_mfma_i32_16x16x64_i8 v[156:159], v[124:127], a[176:179], v[156:159]\n" \
  "ds_read_b64_tr_b8 v[114:115], v82 offset:33792\n" \
  "s_add_u32 m0, s47, 34816\n" \
  "v_mfma_i32_16x16x64_i8 v[188:191], v[120:123], a[164:167], v[188:191]\n" \
  "buffer_load_dwordx4 v42, s[16:19], s82 offen nt lds\n" \
  "s_lshl_b32 s84, s50, 21\n" \
  "s_lshl_b32 s92, s50, 16\n" \
  "s_add_u32 s49, s49, 1\n" \
  "v_mfma_i32_16x16x64_i8 v[188:191], v[124:127], a[180:183], v[188:191]\n" \
  "buffer_load_dwordx4 a[24:27], v91, s[28:31], s88 offen offset:2048 sc0 sc1\n" \
  "v_mfma_i32_16x16x64_i8 v[220:223], v[120:123], a[168:171], v[220:223]\n" \
  "ds_read_b64_tr_b8 v[116:117], v82 offset:40960\n" \
  "v_mfma_i32_16x16x64_i8 v[220:223], v[124:127], a[184:187], v[220:223]\n" \
  "ds_read_b64_tr_b8 v[118:119], v82 offset:41984\n" \
  "s_add_u32 m0, s47, 35840\n" \
  "buffer_load_dwordx4 a[28:31], v91, s[28:31], s88 offen offset:3072 sc0 sc1\n" \
  "v_mfma_i32_16x16x64_i8 v[252:255], v[120:123], a[172:175], v[252:255]\n" \
  "buffer_load_dwordx4 v43, s[16:19], s82 offen nt lds\n" \
  "v_mfma_i32_16x16x64_i8 v[252:255], v[124:127], a[188:191], v[252:255]\n" \
  "; ---- step 6\n" \
  "s_cmp_lg_u32 s11, 0\n" \
  "s_cbranch_scc1 Lskip16B_%=\n" \
  "global_load_dword v72, v50, s[38:39] sc0 sc1\n" \
  "Lskip16B_%=:\n" \
  "s_cmp_lg_u32 s46, 2\n" \
  "s_cbranch_scc1 Lskip15B_%=\n" \
  "s_waitcnt vmcnt(63)\n" \
  "global_store_dwordx4 v49, v[64:67], s[34:35] sc0 sc1\n" \
  "s_nop 1\n" \
  "global_store_dwordx4 v79, v[68:71], s[34:35] sc0 sc1\n" \
  "s_nop 1\n" \
  "Lskip15B_%=:\n" \
  "s_waitcnt vmcnt(24)\n" \
  "s_waitcnt lgkmcnt(8)\n" \
  "v_mfma_i32_16x16x64_i8 v[128:131], v[96:99], a[192:195], v[128:131]\n" \
  "ds_read_b64_tr_b8 v[120:121], v83 offset:32768\n" \
  "s_add_u32 m0, s47, 49152\n" \
  "v_mfma_i32_16x16x64_i8 v[128:131], v[100:103], a[208:211], v[128:131]\n" \
  "ds_read_b64_tr_b8 v[122:123], v83 offset:33792\n" \
  "buffer_load_dwordx4 v40, s[16:19], s83 offen lds\n" \
  "s_add_u32 m0, s47, 50176\n" \
  "v_mfma_i32_16x16x64_i8 v[160:163], v[96:99], a[196:199], v[160:163]\n" \
  "buffer_load_dwordx4 v41, s[16:19], s83 offen lds\n" \
  "s_add_u32 m0, s47, 51200\n" \
  "v_mfma_i32_16x16x64_i8 v[160:163], v[100:103], a[212:215], v[160:163]\n" \
  "buffer_load_dwordx4 v42, s[16:19], s83 offen lds\n" \
  "s_add_u32 m0, s47, 52224\n" \
  "v_mfma_i32_16x16x64_i8 v[192:195], v[96:99], a[200:203], v[192:195]\n" \
  "ds_read_b64_tr_b8 v[124:125], v83 offset:40960\n" \
  "buffer_load_dwordx4 v43, s[16:19], s83 offen lds\n" \
  "s_add_u32 m0, s47, 65536\n" \
  "v_mfma_i32_16x16x64_i8 v[192:195], v[100:103], a[216:219], v[192:195]\n" \
  "ds_read_b64_tr_b8 v[126:127], v83 offset:41984\n" \
  "buffer_load_dwordx4 v40, s[16:19], s84 offen lds\n" \
  "s_add_u32 m0, s47, 66560\n" \
  "v_mfma_i32_16x16x64_i8 v[224:227], v[96:99], a[204:207], v[224:227]\n" \
  "buffer_load_dwordx4 v41, s[16:19], s84 offen lds\n" \
  "s_add_u32 m0, s47, 67584\n" \
  "v_mfma_i32_16x16x64_i8 v[224:227], v[100:103], a[220:223], v[224:227]\n" \
  "buffer_load_dwordx4 v42, s[16:19], s84 offen lds\n" \
  "s_add_u32 m0, s47, 68608\n" \
  "s_waitcnt lgkmcnt(8)\n" \
  "v_mfma_i32_16x16x64_i8 v[132:135], v[104:107], a[192:195], v[132:135]\n" \
  "ds_read_b64_tr_b8 v[96:97], v84 offset:32768\n" \
  "buffer_load_dwordx4 v43, s[16:19], s84 offen lds\n" \
  "v_mfma_i32_16x16x64_i8 v[132:135], v[108:111], a[208:211], v[132:135]\n" \
  "ds_read_b64_tr_b8 v[98:99], v84 offset:33792\n" \
  "v_mfma_i32_16x16x64_i8 v[164:167], v[104:107], a[196:199], v[164:167]\n" \
  "v_mfma_i32_16x16x64_i8 v[164:167], v[108:111], a[212:215], v[164:167]\n" \
  "v_mfma_i32_16x16x64_i8 v[196:199], v[104:107], a[200:203], v[196:199]\n" \
  "ds_read_b64_tr_b8 v[100:101], v84 offset:40960\n" \
  "v_mfma_i32_16x16x64_i8 v[196:199], v[108:111], a[216:219], v[196:199]\n" \
  "ds_read_b64_tr_b8 v[102:103], v84 offset:41984\n" \
  "v_mfma_i32_16x16x64_i8 v[228:231], v[104:107], a[204:207], v[228:231]\n" \
  "v_mfma_i32_16x16x64_i8 v[228:231], v[108:111], a[220:223], v[228:231]\n" \
  "s_waitcnt lgkmcnt(8)\n" \
  "v_mfma_i32_16x16x64_i8 v[136:139], v[112:115], a[192:195], v[136:139]\n" \
  "ds_read_b64_tr_b8 v[104:105], v85 offset:32768\n" \
  "v_mfma_i32_16x16x64_i8 v[136:139], v[116:119], a[208:211], v[136:139]\n" \
  "ds_read_b64_tr_b8 v[106:107], v85 offset:33792\n" \
  "v_mfma_i32_16x16x64_i8 v[168:171], v[112:115], a[196:199], v[168:171]\n" \
  "v_mfma_i32_16x16x64_i8 v[168:171], v[116:119], a[212:215], v[168:171]\n" \
  "v_mfma_i32_16x16x64_i8 v[200:203], v[112:115], a[200:203], v[200:203]\n" \
  "ds_read_b64_tr_b8 v[108:109], v85 offset:40960\n" \
  "v_mfma_i32_16x16x64_i8 v[200:203], v[116:119], a[216:219], v[200:203]\n" \
  "ds_read_b64_tr_b8 v[110:111], v85 offset:41984\n" \
  "v_mfma_i32_16x16x64_i8 v[232:235], v[112:115], a[204:207], v[232:235]\n" \
  "v_mfma_i32_16x16x64_i8 v[232:235], v[116:119], a[220:223], v[232:235]\n" \
  "s_waitcnt lgkmcnt(8)\n" \
  "v_mfma_i32_16x16x64_i8 v[140:143], v[120:123], a[192:195], v[140:143]\n" \
  "ds_read_b64_tr_b8 v[112:113], v86 offset:32768\n" \
  "v_mfma_i32_16x16x64_i8 v[140:143], v[124:127], a[208:211], v[140:143]\n" \
  "ds_read_b64_tr_b8 v[114:115], v86 offset:33792\n" \
  "v_mfma_i32_16x16x64_i8 v[172:175], v[120:123], a[196:199], v[172:175]\n" \
  "v_mfma_i32_16x16x64_i8 v[172:175], v[124:127], a[212:215], v[172:175]\n" \
  "s_waitcnt vmcnt(61)\n" \
  "s_barrier\n" \
  "v_mfma_i32_16x16x64_i8 v[204:207], v[120:123], a[200:203], v[204:207]\n" \
  "ds_read_b64_tr_b8 v[116:117], v86 offset:40960\n" \
  "v_mfma_i32_16x16x64_i8 v[204:207], v[124:127], a[216:219], v[204:207]\n" \
  "ds_read_b64_tr_b8 v[118:119], v86 offset:41984\n" \
  "v_mfma_i32_16x16x64_i8 v[236:239], v[120:123], a[204:207], v[236:239]\n" \
  "v_mfma_i32_16x16x64_i8 v[236:239], v[124:127], a[220:223], v[236:239]\n" \
  "s_waitcnt lgkmcnt(8)\n" \
  "v_mfma_i32_16x16x64_i8 v[144:147], v[96:99], a[192:195], v[144:147]\n" \
  "ds_read_b64_tr_b8 v[120:121], v87 offset:32768\n" \
  "v_mfma_i32_16x16x64_i8 v[144:147], v[100:103], a[208:211], v[144:147]\n" \
  "ds_read_b64_tr_b8 v[122:123], v87 offset:33792\n" \
  "v_mfma_i32_16x16x64_i8 v[176:179], v[96:99], a[196:199], v[176:179]\n" \
  "v_mfma_i32_16x16x64_i8 v[176:179], v[100:103], a[212:215], v[176:179]\n" \
  "v_mfma_i32_16x16x64_i8 v[208:211], v[96:99], a[200:203], v[208:211]\n" \
  "ds_read_b64_tr_b8 v[124:125], v87 offset:40960\n" \
  "v_mfma_i32_16x16x64_i8 v[208:211], v[100:103], a[216:219], v[208:211]\n" \
  "ds_read_b64_tr_b8 v[126:127], v87 offset:41984\n" \
  "v_mfma_i32_16x16x64_i8 v[240:243], v[96:99], a[204:207], v[240:243]\n" \
  "v_mfma_i32_16x16x64_i8 v[240:243], v[100:103], a[220:223], v[240:243]\n" \
  "s_waitcnt lgkmcnt(8)\n" \
  "v_mfma_i32_16x16x64_i8 v[148:151], v[104:107], a[192:195], v[148:151]\n" \
  "ds_read_b64_tr_b8 v[96:97], v80 offset:49152\n" \
  "v_mfma_i32_16x16x64_i8 v[148:151], v[108:111], a[208:211], v[148:151]\n" \
  "ds_read_b64_tr_b8 v[98:99], v80 offset:50176\n" \
  "v_mfma_i32_16x16x64_i8 v[180:183], v[104:107], a[196:199], v[180:183]\n" \
  "v_mfma_i32_16x16x64_i8 v[180:183], v[108:111], a[212:215], v[180:183]\n" \
  "v_mfma_i32_16x16x64_i8 v[212:215], v[104:107], a[200:203], v[212:215]\n" \
  "ds_read_b64_tr_b8 v[100:101], v80 offset:57344\n" \
  "v_mfma_i32_16x16x64_i8 v[212:215], v[108:111], a[216:219], v[212:215]\n" \
  "ds_read_b64_tr_b8 v[102:103], v80 offset:58368\n" \
  "v_mfma_i32_16x16x64_i8 v[244:247], v[104:107], a[204:207], v[244:247]\n" \
  "v_mfma_i32_16x16x64_i8 v[244:247], v[108:111], a[220:223], v[244:247]\n" \
  "s_waitcnt lgkmcnt(8)\n" \
  "v_mfma_i32_16x16x64_i8 v[152:155], v[112:115], a[192:195], v[152:155]\n" \
  "ds_read_b64_tr_b8 v[104:105], v81 offset:49152\n" \
  "v_mfma_i32_16x16x64_i8 v[152:155], v[116:119], a[208:211], v[152:155]\n" \
  "ds_read_b64_tr_b8 v[106:107], v81 offset:50176\n" \
  "v_mfma_i32_16x16x64_i8 v[184:187], v[112:115], a[196:199], v[184:187]\n" \
  "v_mfma_i32_16x16x64_i8 v[184:187], v[116:119], a[212:215], v[184:187]\n" \
  "v_mfma_i32_16x16x64_i8 v[216:219], v[112:115], a[200:203], v[216:219]\n" \
  "ds_read_b64_tr_b8 v[108:109], v81 offset:57344\n" \
  "v_mfma_i32_16x16x64_i8 v[216:219], v[116:119], a[216:219], v[216:219]\n" \
  "ds_read_b64_tr_b8 v[110:111], v81 offset:58368\n" \
  "v_mfma_i32_16x16x64_i8 v[248:251], v[112:115], a[204:207], v[248:251]\n" \
  "v_mfma_i32_16x16x64_i8 v[248:251], v[116:119], a[220:223], v[248:251]\n" \
  "s_waitcnt lgkmcnt(8)\n" \
  "v_mfma_i32_16x16x64_i8 v[156:159], v[120:123], a[192:195], v[156:159]\n" \
  "ds_read_b64_tr_b8 v[112:113], v82 offset:49152\n" \
  "v_mfma_i32_16x16x64_i8 v[156:159], v[124:127], a[208:211], v[156:159]\n" \
  "ds_read_b64_tr_b8 v[114:115], v82 offset:50176\n" \
  "v_mfma_i32_16x16x64_i8 v[188:191], v[120:123], a[196:199], v[188:191]\n" \
  "v_mfma_i32_16x16x64_i8 v[188:191], v[124:127], a[212:215], v[188:191]\n" \
  "v_mfma_i32_16x16x64_i8 v[220:223], v[120:123], a[200:203], v[220:223]\n" \
  "ds_read_b64_tr_b8 v[116:117], v82 offset:57344\n" \
  "v_mfma_i32_16x16x64_i8 v[220:223], v[124:127], a[216:219], v[220:223]\n" \
  "ds_read_b64_tr_b8 v[118:119], v82 offset:58368\n" \
  "v_mfma_i32_16x16x64_i8 v[252:255], v[120:123], a[204:207], v[252:255]\n" \
  "v_mfma_i32_16x16x64_i8 v[252:255], v[124:127], a[220:223], v[252:255]\n" \
  "; ---- step 7\n" \
  "s_waitcnt vmcnt(23)\n" \
  "s_waitcnt lgkmcnt(8)\n" \
  "v_mfma_i32_16x16x64_i8 v[128:131], v[96:99], a[224:227], v[128:131]\n" \
  "ds_read_b64_tr_b8 v[120:121], v83 offset:49152\n" \
  "v_mfma_i32_16x16x64_i8 v[128:131], v[100:103], a[240:243], v[128:131]\n" \
  "ds_read_b64_tr_b8 v[122:123], v83 offset:50176\n" \
  "v_mfma_i32_16x16x64_i8 v[160:163], v[96:99], a[228:231], v[160:163]\n" \
  "v_mfma_i32_16x16x64_i8 v[160:163], v[100:103], a[244:247], v[160:163]\n" \
  "v_mfma_i32_16x16x64_i8 v[192:195], v[96:99], a[232:235], v[192:195]\n" \
  "ds_read_b64_tr_b8 v[124:125], v83 offset:57344\n" \
  "v_mfma_i32_16x16x64_i8 v[192:195], v[100:103], a[248:251], v[192:195]\n" \
  "ds_read_b64_tr_b8 v[126:127], v83 offset:58368\n" \
  "v_mfma_i32_16x16x64_i8 v[224:227], v[96:99], a[236:239], v[224:227]\n" \
  "v_mfma_i32_16x16x64_i8 v[224:227], v[100:103], a[252:255], v[224:227]\n" \
  "s_waitcnt lgkmcnt(8)\n" \
  "v_mfma_i32_16x16x64_i8 v[132:135], v[104:107], a[224:227], v[132:135]\n" \
  "ds_read_b64_tr_b8 v[96:97], v84 offset:49152\n" \
  "v_mfma_i32_16x16x64_i8 v[132:135], v[108:111], a[240:243], v[132:135]\n" \
  "ds_read_b64_tr_b8 v[98:99], v84 offset:50176\n" \
  "v_mfma_i32_16x16x64_i8 v[164:167], v[104:107], a[228:231], v[164:167]\n" \
  "v_mfma_i32_16x16x64_i8 v[164:167], v[108:111], a[244:247], v[164:167]\n" \
  "v_mfma_i32_16x16x64_i8 v[196:199], v[104:107], a[232:235], v[196:199]\n" \
  "ds_read_b64_tr_b8 v[100:101], v84 offset:57344\n" \
  "v_mfma_i32_16x16x64_i8 v[196:199], v[108:111], a[248:251], v[196:199]\n" \
  "ds_read_b64_tr_b8 v[102:103], v84 offset:58368\n" \
  "v_mfma_i32_16x16x64_i8 v[228:231], v[104:107], a[236:239], v[228:231]\n" \
  "v_mfma_i32_16x16x64_i8 v[228:231], v[108:111], a[252:255], v[228:231]\n" \
  "s_waitcnt lgkmcnt(8)\n" \
  "v_mfma_i32_16x16x64_i8 v[136:139], v[112:115], a[224:227], v[136:139]\n" \
  "ds_read_b64_tr_b8 v[104:105], v85 offset:49152\n" \
  "v_mfma_i32_16x16x64_i8 v[136:139], v[116:119], a[240:243], v[136:139]\n" \
  "ds_read_b64_tr_b8 v[106:107], v85 offset:50176\n" \
  "v_mfma_i32_16x16x64_i8 v[168:171], v[112:115], a[228:231], v[168:171]\n" \
  "v_mfma_i32_16x16x64_i8 v[168:171], v[116:119], a[244:247], v[168:171]\n" \
  "v_mfma_i32_16x16x64_i8 v[200:203], v[112:115], a[232:235], v[200:203]\n" \
  "ds_read_b64_tr_b8 v[108:109], v85 offset:57344\n" \
  "v_mfma_i32_16x16x64_i8 v[200:203], v[116:119], a[248:251], v[200:203]\n" \
  "ds_read_b64_tr_b8 v[110:111], v85 offset:58368\n" \
  "buffer_load_dwordx4 a[32:35], v44, s[28:31], s89 offen offset:0 sc0 sc1\n" \
  "v_mfma_i32_16x16x64_i8 v[232:235], v[112:115], a[236:239], v[232:235]\n" \
  "v_mfma_i32_16x16x64_i8 v[232:235], v[116:119], a[252:255], v[232:235]\n" \
  "buffer_load_dwordx4 a[36:39], v44, s[28:31], s89 offen offset:1024 sc0 sc1\n" \
  "s_waitcnt lgkmcnt(8)\n" \
  "v_mfma_i32_16x16x64_i8 v[140:143], v[120:123], a[224:227], v[140:143]\n" \
  "ds_read_b64_tr_b8 v[112:113], v86 offset:49152\n" \
  "v_mfma_i32_16x16x64_i8 v[140:143], v[124:127], a[240:243], v[140:143]\n" \
  "ds_read_b64_tr_b8 v[114:115], v86 offset:50176\n" \
  "buffer_load_dwordx4 a[40:43], v44, s[28:31], s89 offen offset:2048 sc0 sc1\n" \
  "v_mfma_i32_16x16x64_i8 v[172:175], v[120:123], a[228:231], v[172:175]\n" \
  "v_mfma_i32_16x16x64_i8 v[172:175], v[124:127], a[244:247], v[172:175]\n" \
  "s_cmp_lg_u32 s46, 2\n" \
  "s_cbranch_scc1 Lskip17B_%=\n" \
  "s_waitcnt vmcnt(11)\n" \
  "Lskip17B_%=:\n" \
  "s_waitcnt vmcnt(60)\n" \
  "s_barrier\n" \
  "s_cmp_lg_u32 s46, 2\n" \
  "s_cbranch_scc1 Lskip19B_%=\n" \
  "s_cmp_lg_u32 s11, 0\n" \
  "s_cbranch_scc1 Lskip18B_%=\n" \
  "s_mov_b64 exec, 1\n" \
  "global_store_dword v53, v76, s[36:37] sc0 sc1\n" \
  "s_mov_b64 exec, -1\n" \
  "Lskip18B_%=:\n" \
  "Lskip19B_%=:\n" \
  "v_mfma_i32_16x16x64_i8 v[204:207], v[120:123], a[232:235], v[204:207]\n" \
  "ds_read_b64_tr_b8 v[116:117], v86 offset:57344\n" \
  "v_mfma_i32_16x16x64_i8 v[204:207], v[124:127], a[248:251], v[204:207]\n" \
  "ds_read_b64_tr_b8 v[118:119], v86 offset:58368\n" \
  "buffer_load_dwordx4 a[44:47], v44, s[28:31], s89 offen offset:3072 sc0 sc1\n" \
  "v_mfma_i32_16x16x64_i8 v[236:239], v[120:123], a[236:239], v[236:239]\n" \
  "v_mfma_i32_16x16x64_i8 v[236:239], v[124:127], a[252:255], v[236:239]\n" \
  "buffer_load_dwordx4 a[48:51], v91, s[28:31], s89 offen offset:0 sc0 sc1\n" \
  "s_waitcnt lgkmcnt(8)\n" \
  "v_mfma_i32_16x16x64_i8 v[144:147], v[96:99], a[224:227], v[144:147]\n" \
  "ds_read_b64_tr_b8 v[120:121], v87 offset:49152\n" \
  "v_mfma_i32_16x16x64_i8 v[144:147], v[100:103], a[240:243], v[144:147]\n" \
  "ds_read_b64_tr_b8 v[122:123], v87 offset:50176\n" \
  "buffer_load_dwordx4 a[52:55], v91, s[28:31], s89 offen offset:1024 sc0 sc1\n" \
  "v_mfma_i32_16x16x64_i8 v[176:179], v[96:99], a[228:231], v[176:179]\n" \
  "v_mfma_i32_16x16x64_i8 v[176:179], v[100:103], a[244:247], v[176:179]\n" \
  "v_mfma_i32_16x16x64_i8 v[208:211], v[96:99], a[232:235], v[208:211]\n" \
  "ds_read_b64_tr_b8 v[124:125], v87 offset:57344\n" \
  "v_mfma_i32_16x16x64_i8 v[208:211], v[100:103], a[248:251], v[208:211]\n" \
  "ds_read_b64_tr_b8 v[126:127], v87 offset:58368\n" \
  "buffer_load_dwordx4 a[56:59], v91, s[28:31], s89 offen offset:2048 sc0 sc1\n" \
  "v_mfma_i32_16x16x64_i8 v[240:243], v[96:99], a[236:239], v[240:243]\n" \
  "v_mfma_i32_16x16x64_i8 v[240:243], v[100:103], a[252:255], v[240:243]\n" \
  "buffer_load_dwordx4 a[60:63], v91, s[28:31], s89 offen offset:3072 sc0 sc1\n" \
  "s_waitcnt lgkmcnt(8)\n" \
  "v_mfma_i32_16x16x64_i8 v[148:151], v[104:107], a[224:227], v[148:151]\n" \
  "ds_read_b64_tr_b8 v[96:97], v32 offset:0\n" \
  "v_mfma_i32_16x16x64_i8 v[148:151], v[108:111], a[240:243], v[148:151]\n" \
  "ds_read_b64_tr_b8 v[98:99], v32 offset:1024\n" \
  "v_mfma_i32_16x16x64_i8 v[180:183], v[104:107], a[228:231], v[180:183]\n" \
  "v_mfma_i32_16x16x64_i8 v[180:183], v[108:111], a[244:247], v[180:183]\n" \
  "buffer_load_dwordx4 a[64:67], v44, s[28:31], s90 offen offset:0 sc0 sc1\n" \
  "v_mfma_i32_16x16x64_i8 v[212:215], v[104:107], a[232:235], v[212:215]\n" \
  "ds_read_b64_tr_b8 v[100:101], v32 offset:8192\n" \
  "v_mfma_i32_16x16x64_i8 v[212:215], v[108:111], a[248:251], v[212:215]\n" \
  "ds_read_b64_tr_b8 v[102:103], v32 offset:9216\n" \
  "buffer_load_dwordx4 a[68:71], v44, s[28:31], s90 offen offset:1024 sc0 sc1\n" \
  "v_mfma_i32_16x16x64_i8 v[244:247], v[104:107], a[236:239], v[244:247]\n" \
  "v_mfma_i32_16x16x64_i8 v[244:247], v[108:111], a[252:255], v[244:247]\n" \
  "buffer_load_dwordx4 a[72:75], v44, s[28:31], s90 offen offset:2048 sc0 sc1\n" \
  "s_waitcnt lgkmcnt(8)\n" \
  "v_mfma_i32_16x16x64_i8 v[152:155], v[112:115], a[224:227], v[152:155]\n" \
  "ds_read_b64_tr_b8 v[104:105], v33 offset:0\n" \
  "v_mfma_i32_16x16x64_i8 v[152:155], v[116:119], a[240:243], v[152:155]\n" \
  "ds_read_b64_tr_b8 v[106:107], v33 offset:1024\n" \
  "v_mfma_i32_16x16x64_i8 v[184:187], v[112:115], a[228:231], v[184:187]\n" \
  "v_mfma_i32_16x16x64_i8 v[184:187], v[116:119], a[244:247], v[184:187]\n" \
  "buffer_load_dwordx4 a[76:79], v44, s[28:31], s90 offen offset:3072 sc0 sc1\n" \
  "s_cmp_lg_u32 s11, 0\n" \
  "s_cbranch_scc1 Lskip22B_%=\n" \
  "s_waitcnt vmcnt(20)\n" \
  "v_sub_u32 v72, v72, v76\n" \
  "v_cmp_gt_i32 vcc, 0, v72\n" \
  "s_nop 1\n" \
  "s_andn2_b64 vcc, vcc, s[14:15]\n" \
  "s_cbranch_scc1 Lslow20B_%=\n" \
  "Lback21B_%=:\n" \
  "v_add_u32 v50, 1024, v50\n" \
  "Lskip22B_%=:\n" \
  "v_mfma_i32_16x16x64_i8 v[216:219], v[112:115], a[232:235], v[216:219]\n" \
  "ds_read_b64_tr_b8 v[108:109], v33 offset:8192\n" \
  "v_mfma_i32_16x16x64_i8 v[216:219], v[116:119], a[248:251], v[216:219]\n" \
  "ds_read_b64_tr_b8 v[110:111], v33 offset:9216\n" \
  "buffer_load_dwordx4 a[80:83], v91, s[28:31], s90 offen offset:0 sc0 sc1\n" \
  "v_mfma_i32_16x16x64_i8 v[248:251], v[112:115], a[236:239], v[248:251]\n" \
  "v_mfma_i32_16x16x64_i8 v[248:251], v[116:119], a[252:255], v[248:251]\n" \
  "buffer_load_dwordx4 a[84:87], v91, s[28:31], s90 offen offset:1024 sc0 sc1\n" \
  "s_waitcnt lgkmcnt(8)\n" \
  "v_mfma_i32_16x16x64_i8 v[156:159], v[120:123], a[224:227], v[156:159]\n" \
  "ds_read_b64_tr_b8 v[112:113], v34 offset:0\n" \
  "v_mfma_i32_16x16x64_i8 v[156:159], v[124:127], a[240:243], v[156:159]\n" \
  "ds_read_b64_tr_b8 v[114:115], v34 offset:1024\n" \
  "v_mfma_i32_16x16x64_i8 v[188:191], v[120:123], a[228:231], v[188:191]\n" \
  "v_mfma_i32_16x16x64_i8 v[188:191], v[124:127], a[244:247], v[188:191]\n" \
  "buffer_load_dwordx4 a[88:91], v91, s[28:31], s90 offen offset:2048 sc0 sc1\n" \
  "v_mfma_i32_16x16x64_i8 v[220:223], v[120:123], a[232:235], v[220:223]\n" \
  "ds_read_b64_tr_b8 v[116:117], v34 offset:8192\n" \
  "v_mfma_i32_16x16x64_i8 v[220:223], v[124:127], a[248:251], v[220:223]\n" \
  "ds_read_b64_tr_b8 v[118:119], v34 offset:9216\n" \
  "buffer_load_dwordx4 a[92:95], v91, s[28:31], s90 offen offset:3072 sc0 sc1\n" \
  "v_mfma_i32_16x16x64_i8 v[252:255], v[120:123], a[236:239], v[252:255]\n" \
  "v_mfma_i32_16x16x64_i8 v[252:255], v[124:127], a[252:255], v[252:255]\n" \
  "s_barrier\n" \
  "buffer_load_dword v92, v52, s[28:31], 0 offen sc0 sc1\n" \
  "v_add_u32 v52, 0x10000, v52\n" \
  "s_add_u32 s53, s53, 1\n" \
  "s_add_u32 s54, s54, 1\n" \
  "s_mov_b32 s51, 1\n" \
  "L_loopB_%=:\n" \
  "; ---- step 0\n" \
  "s_waitcnt vmcnt(26)\n" \
  "s_waitcnt lgkmcnt(8)\n" \
  "v_mfma_i32_16x16x64_i8 v[128:131], v[96:99], a[0:3], v[128:131]\n" \
  "ds_read_b64_tr_b8 v[120:121], v35 offset:0\n" \
  "v_mfma_i32_16x16x64_i8 v[128:131], v[100:103], a[16:19], v[128:131]\n" \
  "ds_read_b64_tr_b8 v[122:123], v35 offset:1024\n" \
  "v_mfma_i32_16x16x64_i8 v[160:163], v[96:99], a[4:7], v[160:163]\n" \
  "v_mfma_i32_16x16x64_i8 v[160:163], v[100:103], a[20:23], v[160:163]\n" \
  "buffer_load_dwordx4 a[96:99], v44, s[28:31], s91 offen offset:0 sc0 sc1\n" \
  "v_mfma_i32_16x16x64_i8 v[192:195], v[96:99], a[8:11], v[192:195]\n" \
  "ds_read_b64_tr_b8 v[124:125], v35 offset:8192\n" \
  "v_mfma_i32_16x16x64_i8 v[192:195], v[100:103], a[24:27], v[192:195]\n" \
  "ds_read_b64_tr_b8 v[126:127], v35 offset:9216\n" \
  "v_mfma_i32_16x16x64_i8 v[224:227], v[96:99], a[12:15], v[224:227]\n" \
  "v_mfma_i32_16x16x64_i8 v[224:227], v[100:103], a[28:31], v[224:227]\n" \
  "buffer_load_dwordx4 a[100:103], v44, s[28:31], s91 offen offset:1024 sc0 sc1\n" \
  "s_waitcnt lgkmcnt(8)\n" \
  "v_mfma_i32_16x16x64_i8 v[132:135], v[104:107], a[0:3], v[132:135]\n" \
  "ds_read_b64_tr_b8 v[96:97], v36 offset:0\n" \
  "v_mfma_i32_16x16x64_i8 v[132:135], v[108:111], a[16:19], v[132:135]\n" \
  "ds_read_b64_tr_b8 v[98:99], v36 offset:1024\n" \
  "v_mfma_i32_16x16x64_i8 v[164:167], v[104:107], a[4:7], v[164:167]\n" \
  "v_mfma_i32_16x16x64_i8 v[164:167], v[108:111], a[20:23], v[164:167]\n" \
  "buffer_load_dwordx4 a[104:107], v44, s[28:31], s91 offen offset:2048 sc0 sc1\n" \
  "v_mfma_i32_16x16x64_i8 v[196:199], v[104:107], a[8:11], v[196:199]\n" \
  "ds_read_b64_tr_b8 v[100:101], v36 offset:8192\n" \
  "v_mfma_i32_16x16x64_i8 v[196:199], v[108:111], a[24:27], v[196:199]\n" \
  "ds_read_b64_tr_b8 v[102:103], v36 offset:9216\n" \
  "v_mfma_i32_16x16x64_i8 v[228:231], v[104:107], a[12:15], v[228:231]\n" \
  "v_mfma_i32_16x16x64_i8 v[228:231], v[108:111], a[28:31], v[228:231]\n" \
  "s_waitcnt lgkmcnt(8)\n" \
  "v_mfma_i32_16x16x64_i8 v[136:139], v[112:115], a[0:3], v[136:139]\n" \
  "ds_read_b64_tr_b8 v[104:105], v37 offset:0\n" \
  "v_mfma_i32_16x16x64_i8 v[136:139], v[116:119], a[16:19], v[136:139]\n" \
  "ds_read_b64_tr_b8 v[106:107], v37 offset:1024\n" \
  "v_mfma_i32_16x16x64_i8 v[168:171], v[112:115], a[4:7], v[168:171]\n" \
  "v_mfma_i32_16x16x64_i8 v[168:171], v[116:119], a[20:23], v[168:171]\n" \
  "buffer_load_dwordx4 a[108:111], v44, s[28:31], s91 offen offset:3072 sc0 sc1\n" \
  "v_mfma_i32_16x16x64_i8 v[200:203], v[112:115], a[8:11], v[200:203]\n" \
  "ds_read_b64_tr_b8 v[108:109], v37 offset:8192\n" \
  "v_mfma_i32_16x16x64_i8 v[200:203], v[116:119], a[24:27], v[200:203]\n" \
  "ds_read_b64_tr_b8 v[110:111], v37 offset:9216\n" \
  "v_mfma_i32_16x16x64_i8 v[232:235], v[112:115], a[12:15], v[232:235]\n" \
  "v_mfma_i32_16x16x64_i8 v[232:235], v[116:119], a[28:31], v[232:235]\n" \
  "buffer_load_dwordx4 a[112:115], v91, s[28:31], s91 offen offset:0 sc0 sc1\n" \
  "s_waitcnt lgkmcnt(8)\n" \
  "v_mfma_i32_16x16x64_i8 v[140:143], v[120:123], a[0:3], v[140:143]\n" \
  "ds_read_b64_tr_b8 v[112:113], v38 offset:0\n" \
  "v_mfma_i32_16x16x64_i8 v[140:143], v[124:127], a[16:19], v[140:143]\n" \
  "ds_read_b64_tr_b8 v[114:115], v38 offset:1024\n" \
  "v_mfma_i32_16x16x64_i8 v[172:175], v[120:123], a[4:7], v[172:175]\n" \
  "v_mfma_i32_16x16x64_i8 v[172:175], v[124:127], a[20:23], v[172:175]\n" \
  "buffer_load_dwordx4 a[116:119], v91, s[28:31], s91 offen offset:1024 sc0 sc1\n" \
  "s_waitcnt vmcnt(41)\n" \
  "s_barrier\n" \
  "v_mfma_i32_16x16x64_i8 v[204:207], v[120:123], a[8:11], v[204:207]\n" \
  "ds_read_b64_tr_b8 v[116:117], v38 offset:8192\n" \
  "v_mfma_i32_16x16x64_i8 v[204:207], v[124:127], a[24:27], v[204:207]\n" \
  "ds_read_b64_tr_b8 v[118:119], v38 offset:9216\n" \
  "v_mfma_i32_16x16x64_i8 v[236:239], v[120:123], a[12:15], v[236:239]\n" \
  "v_mfma_i32_16x16x64_i8 v[236:239], v[124:127], a[28:31], v[236:239]\n" \
  "s_waitcnt lgkmcnt(8)\n" \
  "v_mfma_i32_16x16x64_i8 v[144:147], v[96:99], a[0:3], v[144:147]\n" \
  "ds_read_b64_tr_b8 v[120:121], v39 offset:0\n" \
  "v_mfma_i32_16x16x64_i8 v[144:147], v[100:103], a[16:19], v[144:147]\n" \
  "ds_read_b64_tr_b8 v[122:123], v39 offset:1024\n" \
  "v_mfma_i32_16x16x64_i8 v[176:179], v[96:99], a[4:7], v[176:179]\n" \
  "v_mfma_i32_16x16x64_i8 v[176:179], v[100:103], a[20:23], v[176:179]\n" \
  "buffer_load_dwordx4 a[120:123], v91, s[28:31], s91 offen offset:2048 sc0 sc1\n" \
  "v_mfma_i32_16x16x64_i8 v[208:211], v[96:99], a[8:11], v[208:211]\n" \
  "ds_read_b64_tr_b8 v[124:125], v39 offset:8192\n" \
  "v_mfma_i32_16x16x64_i8 v[208:211], v[100:103], a[24:27], v[208:211]\n" \
  "ds_read_b64_tr_b8 v[126:127], v39 offset:9216\n" \
  "v_mfma_i32_16x16x64_i8 v[240:243], v[96:99], a[12:15], v[240:243]\n" \
  "v_mfma_i32_16x16x64_i8 v[240:243], v[100:103], a[28:31], v[240:243]\n" \
  "buffer_load_dwordx4 a[124:127], v91, s[28:31], s91 offen offset:3072 sc0 sc1\n" \
  "s_waitcnt lgkmcnt(8)\n" \
  "v_mfma_i32_16x16x64_i8 v[148:151], v[104:107], a[0:3], v[148:151]\n" \
  "ds_read_b64_tr_b8 v[96:97], v32 offset:16384\n" \
  "v_mfma_i32_16x16x64_i8 v[148:151], v[108:111], a[16:19], v[148:151]\n" \
  "ds_read_b64_tr_b8 v[98:99], v32 offset:17408\n" \
  "v_mfma_i32_16x16x64_i8 v[180:183], v[104:107], a[4:7], v[180:183]\n" \
  "v_mfma_i32_16x16x64_i8 v[180:183], v[108:111], a[20:23], v[180:183]\n" \
  "buffer_load_dwordx4 a[128:131], v44, s[28:31], s92 offen offset:0 sc0 sc1\n" \
  "v_mfma_i32_16x16x64_i8 v[212:215], v[104:107], a[8:11], v[212:215]\n" \
  "ds_read_b64_tr_b8 v[100:101], v32 offset:24576\n" \
  "v_mfma_i32_16x16x64_i8 v[212:215], v[108:111], a[24:27], v[212:215]\n" \
  "ds_read_b64_tr_b8 v[102:103], v32 offset:25600\n" \
  "v_mfma_i32_16x16x64_i8 v[244:247], v[104:107], a[12:15], v[244:247]\n" \
  "v_mfma_i32_16x16x64_i8 v[244:247], v[108:111], a[28:31], v[244:247]\n" \
  "s_waitcnt lgkmcnt(8)\n" \
  "v_mfma_i32_16x16x64_i8 v[152:155], v[112:115], a[0:3], v[152:155]\n" \
  "ds_read_b64_tr_b8 v[104:105], v33 offset:16384\n" \
  "v_mfma_i32_16x16x64_i8 v[152:155], v[116:119], a[16:19], v[152:155]\n" \
  "ds_read_b64_tr_b8 v[106:107], v33 offset:17408\n" \
  "v_mfma_i32_16x16x64_i8 v[184:187], v[112:115], a[4:7], v[184:187]\n" \
  "v_readlane_b32 s50, v46, s49\n" \
  "v_mfma_i32_16x16x64_i8 v[184:187], v[116:119], a[20:23], v[184:187]\n" \
  "buffer_load_dwordx4 a[132:135], v44, s[28:31], s92 offen offset:1024 sc0 sc1\n" \
  "v_mfma_i32_16x16x64_i8 v[216:219], v[112:115], a[8:11], v[216:219]\n" \
  "ds_read_b64_tr_b8 v[108:109], v33 offset:24576\n" \
  "v_mfma_i32_16x16x64_i8 v[216:219], v[116:119], a[24:27], v[216:219]\n" \
  "ds_read_b64_tr_b8 v[110:111], v33 offset:25600\n" \
  "v_mfma_i32_16x16x64_i8 v[248:251], v[112:115], a[12:15], v[248:251]\n" \
  "v_mfma_i32_16x16x64_i8 v[248:251], v[116:119], a[28:31], v[248:251]\n" \
  "buffer_load_dwordx4 a[136:139], v44, s[28:31], s92 offen offset:2048 sc0 sc1\n" \
  "s_waitcnt lgkmcnt(8)\n" \
  "v_mfma_i32_16x16x64_i8 v[156:159], v[120:123], a[0:3], v[156:159]\n" \
  "ds_read_b64_tr_b8 v[112:113], v34 offset:16384\n" \
  "v_mfma_i32_16x16x64_i8 v[156:159], v[124:127], a[16:19], v[156:159]\n" \
  "ds_read_b64_tr_b8 v[114:115], v34 offset:17408\n" \
  "v_mfma_i32_16x16x64_i8 v[188:191], v[120:123], a[4:7], v[188:191]\n" \
  "s_lshl_b32 s85, s50, 21\n" \
  "s_lshl_b32 s93, s50, 16\n" \
  "s_add_u32 s49, s49, 1\n" \
  "v_mfma_i32_16x16x64_i8 v[188:191], v[124:127], a[20:23], v[188:191]\n" \
  "buffer_load_dwordx4 a[140:143], v44, s[28:31], s92 offen offset:3072 sc0 sc1\n" \
  "v_mfma_i32_16x16x64_i8 v[220:223], v[120:123], a[8:11], v[220:223]\n" \
  "ds_read_b64_tr_b8 v[116:117], v34 offset:24576\n" \
  "v_mfma_i32_16x16x64_i8 v[220:223], v[124:127], a[24:27], v[220:223]\n" \
  "ds_read_b64_tr_b8 v[118:119], v34 offset:25600\n" \
  "v_mfma_i32_16x16x64_i8 v[252:255], v[120:123], a[12:15], v[252:255]\n" \
  "v_mfma_i32_16x16x64_i8 v[252:255], v[124:127], a[28:31], v[252:255]\n" \
  "; ---- step 1\n" \
  "s_waitcnt vmcnt(21)\n" \
  "s_waitcnt lgkmcnt(8)\n" \
  "v_mfma_i32_16x16x64_i8 v[128:131], v[96:99], a[32:35], v[128:131]\n" \
  "ds_read_b64_tr_b8 v[120:121], v35 offset:16384\n" \
  "v_mfma_i32_16x16x64_i8 v[128:131], v[100:103], a[48:51], v[128:131]\n" \
  "ds_read_b64_tr_b8 v[122:123], v35 offset:17408\n" \
  "v_mfma_i32_16x16x64_i8 v[160:163], v[96:99], a[36:39], v[160:163]\n" \
  "v_mfma_i32_16x16x64_i8 v[160:163], v[100:103], a[52:55], v[160:163]\n" \
  "buffer_load_dwordx4 a[144:147], v91, s[28:31], s92 offen offset:0 sc0 sc1\n" \
  "v_mfma_i32_16x16x64_i8 v[192:195], v[96:99], a[40:43], v[192:195]\n" \
  "ds_read_b64_tr_b8 v[124:125], v35 offset:24576\n" \
  "v_mfma_i32_16x16x64_i8 v[192:195], v[100:103], a[56:59], v[192:195]\n" \
  "ds_read_b64_tr_b8 v[126:127], v35 offset:25600\n" \
  "v_mfma_i32_16x16x64_i8 v[224:227], v[96:99], a[44:47], v[224:227]\n" \
  "v_mfma_i32_16x16x64_i8 v[224:227], v[100:103], a[60:63], v[224:227]\n" \
  "buffer_load_dwordx4 a[148:151], v91, s[28:31], s92 offen offset:1024 sc0 sc1\n" \
  "s_waitcnt lgkmcnt(8)\n" \
  "v_mfma_i32_16x16x64_i8 v[132:135], v[104:107], a[32:35], v[132:135]\n" \
  "ds_read_b64_tr_b8 v[96:97], v36 offset:16384\n" \
  "v_mfma_i32_16x16x64_i8 v[132:135], v[108:111], a[48:51], v[132:135]\n" \
  "ds_read_b64_tr_b8 v[98:99], v36 offset:17408\n" \
  "v_mfma_i32_16x16x64_i8 v[164:167], v[104:107], a[36:39], v[164:167]\n" \
  "v_mfma_i32_16x16x64_i8 v[164:167], v[108:111], a[52:55], v[164:167]\n" \
  "buffer_load_dwordx4 a[152:155], v91, s[28:31], s92 offen offset:2048 sc0 sc1\n" \
  "v_mfma_i32_16x16x64_i8 v[196:199], v[104:107], a[40:43], v[196:199]\n" \
  "ds_read_b64_tr_b8 v[100:101], v36 offset:24576\n" \
  "v_mfma_i32_16x16x64_i8 v[196:199], v[108:111], a[56:59], v[196:199]\n" \
  "ds_read_b64_tr_b8 v[102:103], v36 offset:25600\n" \
  "v_mfma_i32_16x16x64_i8 v[228:231], v[104:107], a[44:47], v[228:231]\n" \
  "v_mfma_i32_16x16x64_i8 v[228:231], v[108:111], a[60:63], v[228:231]\n" \
  "buffer_load_dwordx4 a[156:159], v91, s[28:31], s92 offen offset:3072 sc0 sc1\n" \
  "s_waitcnt lgkmcnt(8)\n" \
  "v_mfma_i32_16x16x64_i8 v[136:139], v[112:115], a[32:35], v[136:139]\n" \
  "ds_read_b64_tr_b8 v[104:105], v37 offset:16384\n" \
  "v_mfma_i32_16x16x64_i8 v[136:139], v[116:119], a[48:51], v[136:139]\n" \
  "ds_read_b64_tr_b8 v[106:107], v37 offset:17408\n" \
  "v_mfma_i32_16x16x64_i8 v[168:171], v[112:115], a[36:39], v[168:171]\n" \
  "v_mfma_i32_16x16x64_i8 v[168:171], v[116:119], a[52:55], v[168:171]\n" \
  "buffer_load_dwordx4 a[160:163], v44, s[28:31], s93 offen offset:0 sc0 sc1\n" \
  "v_mfma_i32_16x16x64_i8 v[200:203], v[112:115], a[40:43], v[200:203]\n" \
  "ds_read_b64_tr_b8 v[108:109], v37 offset:24576\n" \
  "v_mfma_i32_16x16x64_i8 v[200:203], v[116:119], a[56:59], v[200:203]\n" \
  "ds_read_b64_tr_b8 v[110:111], v37 offset:25600\n" \
  "v_mfma_i32_16x16x64_i8 v[232:235], v[112:115], a[44:47], v[232:235]\n" \
  "v_mfma_i32_16x16x64_i8 v[232:235], v[116:119], a[60:63], v[232:235]\n" \
  "buffer_load_dwordx4 a[164:167], v44, s[28:31], s93 offen offset:1024 sc0 sc1\n" \
  "s_waitcnt lgkmcnt(8)\n" \
  "v_mfma_i32_16x16x64_i8 v[140:143], v[120:123], a[32:35], v[140:143]\n" \
  "ds_read_b64_tr_b8 v[112:113], v38 offset:16384\n" \
  "v_mfma_i32_16x16x64_i8 v[140:143], v[124:127], a[48:51], v[140:143]\n" \
  "ds_read_b64_tr_b8 v[114:115], v38 offset:17408\n" \
  "v_mfma_i32_16x16x64_i8 v[172:175], v[120:123], a[36:39], v[172:175]\n" \
  "v_readlane_b32 s50, v46, s49\n" \
  "v_mfma_i32_16x16x64_i8 v[172:175], v[124:127], a[52:55], v[172:175]\n" \
  "buffer_load_dwordx4 a[168:171], v44, s[28:31], s93 offen offset:2048 sc0 sc1\n" \
  "s_waitcnt vmcnt(44)\n" \
  "s_barrier\n" \
  "v_mfma_i32_16x16x64_i8 v[204:207], v[120:123], a[40:43], v[204:207]\n" \
  "ds_read_b64_tr_b8 v[116:117], v38 offset:24576\n" \
  "v_mfma_i32_16x16x64_i8 v[204:207], v[124:127], a[56:59], v[204:207]\n" \
  "ds_read_b64_tr_b8 v[118:119], v38 offset:25600\n" \
  "v_mfma_i32_16x16x64_i8 v[236:239], v[120:123], a[44:47], v[236:239]\n" \
  "v_mfma_i32_16x16x64_i8 v[236:239], v[124:127], a[60:63], v[236:239]\n" \
  "buffer_load_dwordx4 a[172:175], v44, s[28:31], s93 offen offset:3072 sc0 sc1\n" \
  "s_waitcnt lgkmcnt(8)\n" \
  "v_mfma_i32_16x16x64_i8 v[144:147], v[96:99], a[32:35], v[144:147]\n" \
  "ds_read_b64_tr_b8 v[120:121], v39 offset:16384\n" \
  "v_mfma_i32_16x16x64_i8 v[144:147], v[100:103], a[48:51], v[144:147]\n" \
  "ds_read_b64_tr_b8 v[122:123], v39 offset:17408\n" \
  "s_add_u32 m0, s47, 81920\n" \
  "v_mfma_i32_16x16x64_i8 v[176:179], v[96:99], a[36:39], v[176:179]\n" \
  "buffer_load_dwordx4 v40, s[16:19], s85 offen nt lds\n" \
  "s_lshl_b32 s86, s50, 21\n" \
  "s_lshl_b32 s94, s50, 16\n" \
  "s_add_u32 s49, s49, 1\n" \
  "v_mfma_i32_16x16x64_i8 v[176:179], v[100:103], a[52:55], v[176:179]\n" \
  "v_mfma_i32_16x16x64_i8 v[208:211], v[96:99], a[40:43], v[208:211]\n" \
  "ds_read_b64_tr_b8 v[124:125], v39 offset:24576\n" \
  "v_mfma_i32_16x16x64_i8 v[208:211], v[100:103], a[56:59], v[208:211]\n" \
  "ds_read_b64_tr_b8 v[126:127], v39 offset:25600\n" \
  "s_add_u32 m0, s47, 82944\n" \
  "v_mfma_i32_16x16x64_i8 v[240:243], v[96:99], a[44:47], v[240:243]\n" \
  "buffer_load_dwordx4 v41, s[16:19], s85 offen nt lds\n" \
  "v_readlane_b32 s50, v46, s49\n" \
  "v_mfma_i32_16x16x64_i8 v[240:243], v[100:103], a[60:63], v[240:243]\n" \
  "s_waitcnt lgkmcnt(8)\n" \
  "v_mfma_i32_16x16x64_i8 v[148:151], v[104:107], a[32:35], v[148:151]\n" \
  "ds_read_b64_tr_b8 v[96:97], v32 offset:32768\n" \
  "v_mfma_i32_16x16x64_i8 v[148:151], v[108:111], a[48:51], v[148:151]\n" \
  "ds_read_b64_tr_b8 v[98:99], v32 offset:33792\n" \
  "s_add_u32 m0, s47, 83968\n" \
  "v_mfma_i32_16x16x64_i8 v[180:183], v[104:107], a[36:39], v[180:183]\n" \
  "buffer_load_dwordx4 v42, s[16:19], s85 offen nt lds\n" \
  "v_mfma_i32_16x16x64_i8 v[180:183], v[108:111], a[52:55], v[180:183]\n" \
  "v_mfma_i32_16x16x64_i8 v[212:215], v[104:107], a[40:43], v[212:215]\n" \
  "ds_read_b64_tr_b8 v[100:101], v32 offset:40960\n" \
  "v_mfma_i32_16x16x64_i8 v[212:215], v[108:111], a[56:59], v[212:215]\n" \
  "ds_read_b64_tr_b8 v[102:103], v32 offset:41984\n" \
  "s_add_u32 m0, s47, 84992\n" \
  "v_mfma_i32_16x16x64_i8 v[244:247], v[104:107], a[44:47], v[244:247]\n" \
  "buffer_load_dwordx4 v43, s[16:19], s85 offen nt lds\n" \
  "s_lshl_b32 s87, s50, 21\n" \
  "s_lshl_b32 s95, s50, 16\n" \
  "s_add_u32 s49, s49, 1\n" \
  "v_mfma_i32_16x16x64_i8 v[244:247], v[108:111], a[60:63], v[244:247]\n" \
  "s_waitcnt lgkmcnt(8)\n" \
  "v_mfma_i32_16x16x64_i8 v[152:155], v[112:115], a[32:35], v[152:155]\n" \
  "ds_read_b64_tr_b8 v[104:105], v33 offset:32768\n" \
  "v_mfma_i32_16x16x64_i8 v[152:155], v[116:119], a[48:51], v[152:155]\n" \
  "ds_read_b64_tr_b8 v[106:107], v33 offset:33792\n" \
  "s_add_u32 m0, s47, 98304\n" \
  "v_mfma_i32_16x16x64_i8 v[184:187], v[112:115], a[36:39], v[184:187]\n" \
  "buffer_load_dwordx4 v40, s[16:19], s86 offen nt lds\n" \
  "v_readlane_b32 s50, v46, s49\n" \
  "v_mfma_i32_16x16x64_i8 v[184:187], v[116:119], a[52:55], v[184:187]\n" \
  "v_mfma_i32_16x16x64_i8 v[216:219], v[112:115], a[40:43], v[216:219]\n" \
  "ds_read_b64_tr_b8 v[108:109], v33 offset:40960\n" \
  "v_mfma_i32_16x16x64_i8 v[216:219], v[116:119], a[56:59], v[216:219]\n" \
  "ds_read_b64_tr_b8 v[110:111], v33 offset:41984\n" \
  "s_add_u32 m0, s47, 99328\n" \
  "v_mfma_i32_16x16x64_i8 v[248:251], v[112:115], a[44:47], v[248:251]\n" \
  "buffer_load_dwordx4 v41, s[16:19], s86 offen nt lds\n" \
  "v_mfma_i32_16x16x64_i8 v[248:251], v[116:119], a[60:63], v[248:251]\n" \
  "s_waitcnt lgkmcnt(8)\n" \
  "v_mfma_i32_16x16x64_i8 v[156:159], v[120:123], a[32:35], v[156:159]\n" \
  "ds_read_b64_tr_b8 v[112:113], v34 offset:32768\n" \
  "v_mfma_i32_16x16x64_i8 v[156:159], v[124:127], a[48:51], v[156:159]\n" \
  "ds_read_b64_tr_b8 v[114:115], v34 offset:33792\n" \
  "s_add_u32 m0, s47, 100352\n" \
  "v_mfma_i32_16x16x64_i8 v[188:191], v[120:123], a[36:39], v[188:191]\n" \
  "buffer_load_dwordx4 v42, s[16:19], s86 offen nt lds\n" \
  "s_lshl_b32 s80, s50, 21\n" \
  "s_lshl_b32 s88, s50, 16\n" \
  "s_add_u32 s49, s49, 1\n" \
  "v_mfma_i32_16x16x64_i8 v[188:191], v[124:127], a[52:55], v[188:191]\n" \
  "v_mfma_i32_16x16x64_i8 v[220:223], v[120:123], a[40:43], v[220:223]\n" \
  "ds_read_b64_tr_b8 v[116:117], v34 offset:40960\n" \
  "v_mfma_i32_16x16x64_i8 v[220:223], v[124:127], a[56:59], v[220:223]\n" \
  "ds_read_b64_tr_b8 v[118:119], v34 offset:41984\n" \
  "s_add_u32 m0, s47, 101376\n" \
  "v_mfma_i32_16x16x64_i8 v[252:255], v[120:123], a[44:47], v[252:255]\n" \
  "buffer_load_dwordx4 v43, s[16:19], s86 offen nt lds\n" \
  "v_mfma_i32_16x16x64_i8 v[252:255], v[124:127], a[60:63], v[252:255]\n" \
  "; ---- step 2\n" \
  "s_cmp_lg_u32 s11, 0\n" \
  "s_cbranch_scc1 Lskip24B_%=\n" \
  "global_load_dword v72, v50, s[38:39] sc0 sc1\n" \
  "Lskip24B_%=:\n" \
  "s_cmp_lg_u32 s46, s54\n" \
  "s_cbranch_scc1 Lskip23B_%=\n" \
  "s_waitcnt vmcnt(63)\n" \
  "global_store_dwordx4 v49, v[64:67], s[34:35] sc0 sc1\n" \
  "s_nop 1\n" \
  "global_store_dwordx4 v79, v[68:71], s[34:35] sc0 sc1\n" \
  "s_nop 1\n" \
  "Lskip23B_%=:\n" \
  "s_waitcnt vmcnt(29)\n" \
  "s_waitcnt lgkmcnt(8)\n" \
  "v_mfma_i32_16x16x64_i8 v[128:131], v[96:99], a[64:67], v[128:131]\n" \
  "ds_read_b64_tr_b8 v[120:121], v35 offset:32768\n" \
  "s_add_u32 m0, s47, 114688\n" \
  "v_mfma_i32_16x16x64_i8 v[128:131], v[100:103], a[80:83], v[128:131]\n" \
  "ds_read_b64_tr_b8 v[122:123], v35 offset:33792\n" \
  "buffer_load_dwordx4 v40, s[16:19], s87 offen lds\n" \
  "s_add_u32 m0, s47, 115712\n" \
  "v_mfma_i32_16x16x64_i8 v[160:163], v[96:99], a[68:71], v[160:163]\n" \
  "buffer_load_dwordx4 v41, s[16:19], s87 offen lds\n" \
  "s_add_u32 m0, s47, 116736\n" \
  "v_mfma_i32_16x16x64_i8 v[160:163], v[100:103], a[84:87], v[160:163]\n" \
  "buffer_load_dwordx4 v42, s[16:19], s87 offen lds\n" \
  "s_add_u32 m0, s47, 117760\n" \
  "v_mfma_i32_16x16x64_i8 v[192:195], v[96:99], a[72:75], v[192:195]\n" \
  "ds_read_b64_tr_b8 v[124:125], v35 offset:40960\n" \
  "buffer_load_dwordx4 v43, s[16:19], s87 offen lds\n" \
  "s_add_u32 m0, s47, 0\n" \
  "v_mfma_i32_16x16x64_i8 v[192:195], v[100:103], a[88:91], v[192:195]\n" \
  "ds_read_b64_tr_b8 v[126:127], v35 offset:41984\n" \
  "buffer_load_dwordx4 v40, s[16:19], s80 offen lds\n" \
  "s_add_u32 m0, s47, 1024\n" \
  "v_mfma_i32_16x16x64_i8 v[224:227], v[96:99], a[76:79], v[224:227]\n" \
  "buffer_load_dwordx4 v41, s[16:19], s80 offen lds\n" \
  "v_mfma_i32_16x16x64_i8 v[224:227], v[100:103], a[92:95], v[224:227]\n" \
  "s_waitcnt lgkmcnt(8)\n" \
  "v_mfma_i32_16x16x64_i8 v[132:135], v[104:107], a[64:67], v[132:135]\n" \
  "ds_read_b64_tr_b8 v[96:97], v36 offset:32768\n" \
  "v_mfma_i32_16x16x64_i8 v[132:135], v[108:111], a[80:83], v[132:135]\n" \
  "ds_read_b64_tr_b8 v[98:99], v36 offset:33792\n" \
  "v_mfma_i32_16x16x64_i8 v[164:167], v[104:107], a[68:71], v[164:167]\n" \
  "v_mfma_i32_16x16x64_i8 v[164:167], v[108:111], a[84:87], v[164:167]\n" \
  "v_mfma_i32_16x16x64_i8 v[196:199], v[104:107], a[72:75], v[196:199]\n" \
  "ds_read_b64_tr_b8 v[100:101], v36 offset:40960\n" \
  "v_mfma_i32_16x16x64_i8 v[196:199], v[108:111], a[88:91], v[196:199]\n" \
  "ds_read_b64_tr_b8 v[102:103], v36 offset:41984\n" \
  "v_mfma_i32_16x16x64_i8 v[228:231], v[104:107], a[76:79], v[228:231]\n" \
  "v_mfma_i32_16x16x64_i8 v[228:231], v[108:111], a[92:95], v[228:231]\n" \
  "s_waitcnt lgkmcnt(8)\n" \
  "v_mfma_i32_16x16x64_i8 v[136:139], v[112:115], a[64:67], v[136:139]\n" \
  "ds_read_b64_tr_b8 v[104:105], v37 offset:32768\n" \
  "v_mfma_i32_16x16x64_i8 v[136:139], v[116:119], a[80:83], v[136:139]\n" \
  "ds_read_b64_tr_b8 v[106:107], v37 offset:33792\n" \
  "v_mfma_i32_16x16x64_i8 v[168:171], v[112:115], a[68:71], v[168:171]\n" \
  "v_mfma_i32_16x16x64_i8 v[168:171], v[116:119], a[84:87], v[168:171]\n" \
  "v_mfma_i32_16x16x64_i8 v[200:203], v[112:115], a[72:75], v[200:203]\n" \
  "ds_read_b64_tr_b8 v[108:109], v37 offset:40960\n" \
  "v_mfma_i32_16x16x64_i8 v[200:203], v[116:119], a[88:91], v[200:203]\n" \
  "ds_read_b64_tr_b8 v[110:111], v37 offset:41984\n" \
  "v_mfma_i32_16x16x64_i8 v[232:235], v[112:115], a[76:79], v[232:235]\n" \
  "v_mfma_i32_16x16x64_i8 v[232:235], v[116:119], a[92:95], v[232:235]\n" \
  "s_waitcnt lgkmcnt(8)\n" \
  "v_mfma_i32_16x16x64_i8 v[140:143], v[120:123], a[64:67], v[140:143]\n" \
  "ds_read_b64_tr_b8 v[112:113], v38 offset:32768\n" \
  "v_mfma_i32_16x16x64_i8 v[140:143], v[124:127], a[80:83], v[140:143]\n" \
  "ds_read_b64_tr_b8 v[114:115], v38 offset:33792\n" \
  "v_mfma_i32_16x16x64_i8 v[172:175], v[120:123], a[68:71], v[172:175]\n" \
  "v_mfma_i32_16x16x64_i8 v[172:175], v[124:127], a[84:87], v[172:175]\n" \
  "s_waitcnt vmcnt(55)\n" \
  "s_barrier\n" \
  "v_mfma_i32_16x16x64_i8 v[204:207], v[120:123], a[72:75], v[204:207]\n" \
  "ds_read_b64_tr_b8 v[116:117], v38 offset:40960\n" \
  "v_mfma_i32_16x16x64_i8 v[204:207], v[124:127], a[88:91], v[204:207]\n" \
  "ds_read_b64_tr_b8 v[118:119], v38 offset:41984\n" \
  "v_mfma_i32_16x16x64_i8 v[236:239], v[120:123], a[76:79], v[236:239]\n" \
  "v_mfma_i32_16x16x64_i8 v[236:239], v[124:127], a[92:95], v[236:239]\n" \
  "s_waitcnt lgkmcnt(8)\n" \
  "v_mfma_i32_16x16x64_i8 v[144:147], v[96:99], a[64:67], v[144:147]\n" \
  "ds_read_b64_tr_b8 v[120:121], v39 offset:32768\n" \
  "v_mfma_i32_16x16x64_i8 v[144:147], v[100:103], a[80:83], v[144:147]\n" \
  "ds_read_b64_tr_b8 v[122:123], v39 offset:33792\n" \
  "v_mfma_i32_16x16x64_i8 v[176:179], v[96:99], a[68:71], v[176:179]\n" \
  "v_mfma_i32_16x16x64_i8 v[176:179], v[100:103], a[84:87], v[176:179]\n" \
  "v_mfma_i32_16x16x64_i8 v[208:211], v[96:99], a[72:75], v[208:211]\n" \
  "ds_read_b64_tr_b8 v[124:125], v39 offset:40960\n" \
  "v_mfma_i32_16x16x64_i8 v[208:211], v[100:103], a[88:91], v[208:211]\n" \
  "ds_read_b64_tr_b8 v[126:127], v39 offset:41984\n" \
  "v_mfma_i32_16x16x64_i8 v[240:243], v[96:99], a[76:79], v[240:243]\n" \
  "v_mfma_i32_16x16x64_i8 v[240:243], v[100:103], a[92:95], v[240:243]\n" \
  "s_waitcnt lgkmcnt(8)\n" \
  "v_mfma_i32_16x16x64_i8 v[148:151], v[104:107], a[64:67], v[148:151]\n" \
  "ds_read_b64_tr_b8 v[96:97], v32 offset:49152\n" \
  "s_add_u32 m0, s47, 2048\n" \
  "v_mfma_i32_16x16x64_i8 v[148:151], v[108:111], a[80:83], v[148:151]\n" \
  "ds_read_b64_tr_b8 v[98:99], v32 offset:50176\n" \
  "buffer_load_dwordx4 v42, s[16:19], s80 offen lds\n" \
  "s_add_u32 m0, s47, 3072\n" \
  "v_mfma_i32_16x16x64_i8 v[180:183], v[104:107], a[68:71], v[180:183]\n" \
  "buffer_load_dwordx4 v43, s[16:19], s80 offen lds\n" \
  "v_mfma_i32_16x16x64_i8 v[180:183], v[108:111], a[84:87], v[180:183]\n" \
  "v_mfma_i32_16x16x64_i8 v[212:215], v[104:107], a[72:75], v[212:215]\n" \
  "ds_read_b64_tr_b8 v[100:101], v32 offset:57344\n" \
  "v_mfma_i32_16x16x64_i8 v[212:215], v[108:111], a[88:91], v[212:215]\n" \
  "ds_read_b64_tr_b8 v[102:103], v32 offset:58368\n" \
  "v_mfma_i32_16x16x64_i8 v[244:247], v[104:107], a[76:79], v[244:247]\n" \
  "v_mfma_i32_16x16x64_i8 v[244:247], v[108:111], a[92:95], v[244:247]\n" \
  "s_waitcnt lgkmcnt(8)\n" \
  "v_mfma_i32_16x16x64_i8 v[152:155], v[112:115], a[64:67], v[152:155]\n" \
  "ds_read_b64_tr_b8 v[104:105], v33 offset:49152\n" \
  "v_mfma_i32_16x16x64_i8 v[152:155], v[116:119], a[80:83], v[152:155]\n" \
  "ds_read_b64_tr_b8 v[106:107], v33 offset:50176\n" \
  "v_mfma_i32_16x16x64_i8 v[184:187], v[112:115], a[68:71], v[184:187]\n" \
  "v_mfma_i32_16x16x64_i8 v[184:187], v[116:119], a[84:87], v[184:187]\n" \
  "v_mfma_i32_16x16x64_i8 v[216:219], v[112:115], a[72:75], v[216:219]\n" \
  "ds_read_b64_tr_b8 v[108:109], v33 offset:57344\n" \
  "v_mfma_i32_16x16x64_i8 v[216:219], v[116:119], a[88:91], v[216:219]\n" \
  "ds_read_b64_tr_b8 v[110:111], v33 offset:58368\n" \
  "v_mfma_i32_16x16x64_i8 v[248:251], v[112:115], a[76:79], v[248:251]\n" \
  "v_mfma_i32_16x16x64_i8 v[248:251], v[116:119], a[92:95], v[248:251]\n" \
  "s_waitcnt lgkmcnt(8)\n" \
  "v_mfma_i32_16x16x64_i8 v[156:159], v[120:123], a[64:67], v[156:159]\n" \
  "ds_read_b64_tr_b8 v[112:113], v34 offset:49152\n" \
  "v_mfma_i32_16x16x64_i8 v[156:159], v[124:127], a[80:83], v[156:159]\n" \
  "ds_read_b64_tr_b8 v[114:115], v34 offset:50176\n" \
  "v_mfma_i32_16x16x64_i8 v[188:191], v[120:123], a[68:71], v[188:191]\n" \
  "v_mfma_i32_16x16x64_i8 v[188:191], v[124:127], a[84:87], v[188:191]\n" \
  "v_mfma_i32_16x16x64_i8 v[220:223], v[120:123], a[72:75], v[220:223]\n" \
  "ds_read_b64_tr_b8 v[116:117], v34 offset:57344\n" \
  "v_mfma_i32_16x16x64_i8 v[220:223], v[124:127], a[88:91], v[220:223]\n" \
  "ds_read_b64_tr_b8 v[118:119], v34 offset:58368\n" \
  "v_mfma_i32_16x16x64_i8 v[252:255], v[120:123], a[76:79], v[252:255]\n" \
  "v_mfma_i32_16x16x64_i8 v[252:255], v[124:127], a[92:95], v[252:255]\n" \
  "; ---- step 3\n" \
  "s_waitcnt vmcnt(28)\n" \
  "s_waitcnt lgkmcnt(8)\n" \
  "v_mfma_i32_16x16x64_i8 v[128:131], v[96:99], a[96:99], v[128:131]\n" \
  "ds_read_b64_tr_b8 v[120:121], v35 offset:49152\n" \
  "v_mfma_i32_16x16x64_i8 v[128:131], v[100:103], a[112:115], v[128:131]\n" \
  "ds_read_b64_tr_b8 v[122:123], v35 offset:50176\n" \
  "v_mfma_i32_16x16x64_i8 v[160:163], v[96:99], a[100:103], v[160:163]\n" \
  "v_mfma_i32_16x16x64_i8 v[160:163], v[100:103], a[116:119], v[160:163]\n" \
  "v_mfma_i32_16x16x64_i8 v[192:195], v[96:99], a[104:107], v[192:195]\n" \
  "ds_read_b64_tr_b8 v[124:125], v35 offset:57344\n" \
  "v_mfma_i32_16x16x64_i8 v[192:195], v[100:103], a[120:123], v[192:195]\n" \
  "ds_read_b64_tr_b8 v[126:127], v35 offset:58368\n" \
  "v_mfma_i32_16x16x64_i8 v[224:227], v[96:99], a[108:111], v[224:227]\n" \
  "v_mfma_i32_16x16x64_i8 v[224:227], v[100:103], a[124:127], v[224:227]\n" \
  "s_waitcnt lgkmcnt(8)\n" \
  "v_mfma_i32_16x16x64_i8 v[132:135], v[104:107], a[96:99], v[132:135]\n" \
  "ds_read_b64_tr_b8 v[96:97], v36 offset:49152\n" \
  "v_mfma_i32_16x16x64_i8 v[132:135], v[108:111], a[112:115], v[132:135]\n" \
  "ds_read_b64_tr_b8 v[98:99], v36 offset:50176\n" \
  "v_mfma_i32_16x16x64_i8 v[164:167], v[104:107], a[100:103], v[164:167]\n" \
  "v_mfma_i32_16x16x64_i8 v[164:167], v[108:111], a[116:119], v[164:167]\n" \
  "v_mfma_i32_16x16x64_i8 v[196:199], v[104:107], a[104:107], v[196:199]\n" \
  "ds_read_b64_tr_b8 v[100:101], v36 offset:57344\n" \
  "v_mfma_i32_16x16x64_i8 v[196:199], v[108:111], a[120:123], v[196:199]\n" \
  "ds_read_b64_tr_b8 v[102:103], v36 offset:58368\n" \
  "v_mfma_i32_16x16x64_i8 v[228:231], v[104:107], a[108:111], v[228:231]\n" \
  "v_mfma_i32_16x16x64_i8 v[228:231], v[108:111], a[124:127], v[228:231]\n" \
  "s_waitcnt lgkmcnt(8)\n" \
  "v_mfma_i32_16x16x64_i8 v[136:139], v[112:115], a[96:99], v[136:139]\n" \
  "ds_read_b64_tr_b8 v[104:105], v37 offset:49152\n" \
  "v_mfma_i32_16x16x64_i8 v[136:139], v[116:119], a[112:115], v[136:139]\n" \
  "ds_read_b64_tr_b8 v[106:107], v37 offset:50176\n" \
  "v_mfma_i32_16x16x64_i8 v[168:171], v[112:115], a[100:103], v[168:171]\n" \
  "v_mfma_i32_16x16x64_i8 v[168:171], v[116:119], a[116:119], v[168:171]\n" \
  "v_mfma_i32_16x16x64_i8 v[200:203], v[112:115], a[104:107], v[200:203]\n" \
  "ds_read_b64_tr_b8 v[108:109], v37 offset:57344\n" \
  "v_mfma_i32_16x16x64_i8 v[200:203], v[116:119], a[120:123], v[200:203]\n" \
  "ds_read_b64_tr_b8 v[110:111], v37 offset:58368\n" \
  "buffer_load_dwordx4 a[176:179], v91, s[28:31], s93 offen offset:0 sc0 sc1\n" \
  "v_mfma_i32_16x16x64_i8 v[232:235], v[112:115], a[108:111], v[232:235]\n" \
  "v_mfma_i32_16x16x64_i8 v[232:235], v[116:119], a[124:127], v[232:235]\n" \
  "buffer_load_dwordx4 a[180:183], v91, s[28:31], s93 offen offset:1024 sc0 sc1\n" \
  "s_waitcnt lgkmcnt(8)\n" \
  "v_mfma_i32_16x16x64_i8 v[140:143], v[120:123], a[96:99], v[140:143]\n" \
  "ds_read_b64_tr_b8 v[112:113], v38 offset:49152\n" \
  "v_mfma_i32_16x16x64_i8 v[140:143], v[124:127], a[112:115], v[140:143]\n" \
  "ds_read_b64_tr_b8 v[114:115], v38 offset:50176\n" \
  "v_mfma_i32_16x16x64_i8 v[172:175], v[120:123], a[100:103], v[172:175]\n" \
  "v_mfma_i32_16x16x64_i8 v[172:175], v[124:127], a[116:119], v[172:175]\n" \
  "buffer_load_dwordx4 a[184:187], v91, s[28:31], s93 offen offset:2048 sc0 sc1\n" \
  "s_cmp_lg_u32 s46, s54\n" \
  "s_cbranch_scc1 Lskip25B_%=\n" \
  "s_waitcnt vmcnt(11)\n" \
  "Lskip25B_%=:\n" \
  "s_waitcnt vmcnt(56)\n" \
  "s_barrier\n" \
  "s_cmp_lg_u32 s46, s54\n" \
  "s_cbranch_scc1 Lskip27B_%=\n" \
  "s_cmp_lg_u32 s11, 0\n" \
  "s_cbranch_scc1 Lskip26B_%=\n" \
  "s_mov_b64 exec, 1\n" \
  "global_store_dword v53, v76, s[36:37] sc0 sc1\n" \
  "s_mov_b64 exec, -1\n" \
  "Lskip26B_%=:\n" \
  "Lskip27B_%=:\n" \
  "v_mfma_i32_16x16x64_i8 v[204:207], v[120:123], a[104:107], v[204:207]\n" \
  "ds_read_b64_tr_b8 v[116:117], v38 offset:57344\n" \
  "v_mfma_i32_16x16x64_i8 v[204:207], v[124:127], a[120:123], v[204:207]\n" \
  "ds_read_b64_tr_b8 v[118:119], v38 offset:58368\n" \
  "v_mfma_i32_16x16x64_i8 v[236:239], v[120:123], a[108:111], v[236:239]\n" \
  "v_mfma_i32_16x16x64_i8 v[236:239], v[124:127], a[124:127], v[236:239]\n" \
  "buffer_load_dwordx4 a[188:191], v91, s[28:31], s93 offen offset:3072 sc0 sc1\n" \
  "s_waitcnt lgkmcnt(8)\n" \
  "v_mfma_i32_16x16x64_i8 v[144:147], v[96:99], a[96:99], v[144:147]\n" \
  "ds_read_b64_tr_b8 v[120:121], v39 offset:49152\n" \
  "v_mfma_i32_16x16x64_i8 v[144:147], v[100:103], a[112:115], v[144:147]\n" \
  "ds_read_b64_tr_b8 v[122:123], v39 offset:50176\n" \
  "v_mfma_i32_16x16x64_i8 v[176:179], v[96:99], a[100:103], v[176:179]\n" \
  "v_mfma_i32_16x16x64_i8 v[176:179], v[100:103], a[116:119], v[176:179]\n" \
  "buffer_load_dwordx4 a[192:195], v44, s[28:31], s94 offen offset:0 sc0 sc1\n" \
  "v_mfma_i32_16x16x64_i8 v[208:211], v[96:99], a[104:107], v[208:211]\n" \
  "ds_read_b64_tr_b8 v[124:125], v39 offset:57344\n" \
  "v_mfma_i32_16x16x64_i8 v[208:211], v[100:103], a[120:123], v[208:211]\n" \
  "ds_read_b64_tr_b8 v[126:127], v39 offset:58368\n" \
  "v_mfma_i32_16x16x64_i8 v[240:243], v[96:99], a[108:111], v[240:243]\n" \
  "v_mfma_i32_16x16x64_i8 v[240:243], v[100:103], a[124:127], v[240:243]\n" \
  "buffer_load_dwordx4 a[196:199], v44, s[28:31], s94 offen offset:1024 sc0 sc1\n" \
  "s_waitcnt lgkmcnt(8)\n" \
  "v_mfma_i32_16x16x64_i8 v[148:151], v[104:107], a[96:99], v[148:151]\n" \
  "ds_read_b64_tr_b8 v[96:97], v80 offset:0\n" \
  "v_mfma_i32_16x16x64_i8 v[148:151], v[108:111], a[112:115], v[148:151]\n" \
  "ds_read_b64_tr_b8 v[98:99], v80 offset:1024\n" \
  "v_mfma_i32_16x16x64_i8 v[180:183], v[104:107], a[100:103], v[180:183]\n" \
  "v_mfma_i32_16x16x64_i8 v[180:183], v[108:111], a[116:119], v[180:183]\n" \
  "buffer_load_dwordx4 a[200:203], v44, s[28:31], s94 offen offset:2048 sc0 sc1\n" \
  "v_mfma_i32_16x16x64_i8 v[212:215], v[104:107], a[104:107], v[212:215]\n" \
  "ds_read_b64_tr_b8 v[100:101], v80 offset:8192\n" \
  "v_mfma_i32_16x16x64_i8 v[212:215], v[108:111], a[120:123], v[212:215]\n" \
  "ds_read_b64_tr_b8 v[102:103], v80 offset:9216\n" \
  "buffer_load_dwordx4 a[204:207], v44, s[28:31], s94 offen offset:3072 sc0 sc1\n" \
  "v_mfma_i32_16x16x64_i8 v[244:247], v[104:107], a[108:111], v[244:247]\n" \
  "v_mfma_i32_16x16x64_i8 v[244:247], v[108:111], a[124:127], v[244:247]\n" \
  "s_waitcnt lgkmcnt(8)\n" \
  "v_mfma_i32_16x16x64_i8 v[152:155], v[112:115], a[96:99], v[152:155]\n" \
  "ds_read_b64_tr_b8 v[104:105], v81 offset:0\n" \
  "v_mfma_i32_16x16x64_i8 v[152:155], v[116:119], a[112:115], v[152:155]\n" \
  "ds_read_b64_tr_b8 v[106:107], v81 offset:1024\n" \
  "buffer_load_dwordx4 a[208:211], v91, s[28:31], s94 offen offset:0 sc0 sc1\n" \
  "v_mfma_i32_16x16x64_i8 v[184:187], v[112:115], a[100:103], v[184:187]\n" \
  "v_mfma_i32_16x16x64_i8 v[184:187], v[116:119], a[116:119], v[184:187]\n" \
  "s_cmp_lg_u32 s11, 0\n" \
  "s_cbranch_scc1 Lskip30B_%=\n" \
  "s_waitcnt vmcnt(17)\n" \
  "v_sub_u32 v72, v72, v76\n" \
  "v_cmp_gt_i32 vcc, 0, v72\n" \
  "s_nop 1\n" \
  "s_andn2_b64 vcc, vcc, s[14:15]\n" \
  "s_cbranch_scc1 Lslow28B_%=\n" \
  "Lback29B_%=:\n" \
  "v_add_u32 v50, 1024, v50\n" \
  "Lskip30B_%=:\n" \
  "v_mfma_i32_16x16x64_i8 v[216:219], v[112:115], a[104:107], v[216:219]\n" \
  "ds_read_b64_tr_b8 v[108:109], v81 offset:8192\n" \
  "v_mfma_i32_16x16x64_i8 v[216:219], v[116:119], a[120:123], v[216:219]\n" \
  "ds_read_b64_tr_b8 v[110:111], v81 offset:9216\n" \
  "buffer_load_dwordx4 a[212:215], v91, s[28:31], s94 offen offset:1024 sc0 sc1\n" \
  "v_mfma_i32_16x16x64_i8 v[248:251], v[112:115], a[108:111], v[248:251]\n" \
  "v_mfma_i32_16x16x64_i8 v[248:251], v[116:119], a[124:127], v[248:251]\n" \
  "s_waitcnt lgkmcnt(8)\n" \
  "v_mfma_i32_16x16x64_i8 v[156:159], v[120:123], a[96:99], v[156:159]\n" \
  "ds_read_b64_tr_b8 v[112:113], v82 offset:0\n" \
  "v_mfma_i32_16x16x64_i8 v[156:159], v[124:127], a[112:115], v[156:159]\n" \
  "ds_read_b64_tr_b8 v[114:115], v82 offset:1024\n" \
  "buffer_load_dwordx4 a[216:219], v91, s[28:31], s94 offen offset:2048 sc0 sc1\n" \
  "v_mfma_i32_16x16x64_i8 v[188:191], v[120:123], a[100:103], v[188:191]\n" \
  "v_mfma_i32_16x16x64_i8 v[188:191], v[124:127], a[116:119], v[188:191]\n" \
  "v_mfma_i32_16x16x64_i8 v[220:223], v[120:123], a[104:107], v[220:223]\n" \
  "ds_read_b64_tr_b8 v[116:117], v82 offset:8192\n" \
  "v_mfma_i32_16x16x64_i8 v[220:223], v[124:127], a[120:123], v[220:223]\n" \
  "ds_read_b64_tr_b8 v[118:119], v82 offset:9216\n" \
  "buffer_load_dwordx4 a[220:223], v91, s[28:31], s94 offen offset:3072 sc0 sc1\n" \
  "v_mfma_i32_16x16x64_i8 v[252:255], v[120:123], a[108:111], v[252:255]\n" \
  "v_mfma_i32_16x16x64_i8 v[252:255], v[124:127], a[124:127], v[252:255]\n" \
  "s_barrier\n" \
  "buffer_load_dword v92, v52, s[28:31], 0 offen sc0 sc1\n" \
  "v_add_u32 v52, 0x10000, v52\n" \
  "; ---- step 4\n" \
  "s_waitcnt vmcnt(33)\n" \
  "s_waitcnt lgkmcnt(8)\n" \
  "v_mfma_i32_16x16x64_i8 v[128:131], v[96:99], a[128:131], v[128:131]\n" \
  "ds_read_b64_tr_b8 v[120:121], v83 offset:0\n" \
  "v_mfma_i32_16x16x64_i8 v[128:131], v[100:103], a[144:147], v[128:131]\n" \
  "ds_read_b64_tr_b8 v[122:123], v83 offset:1024\n" \
  "v_mfma_i32_16x16x64_i8 v[160:163], v[96:99], a[132:135], v[160:163]\n" \
  "v_mfma_i32_16x16x64_i8 v[160:163], v[100:103], a[148:151], v[160:163]\n" \
  "buffer_load_dwordx4 a[224:227], v44, s[28:31], s95 offen offset:0 sc0 sc1\n" \
  "v_mfma_i32_16x16x64_i8 v[192:195], v[96:99], a[136:139], v[192:195]\n" \
  "ds_read_b64_tr_b8 v[124:125], v83 offset:8192\n" \
  "v_mfma_i32_16x16x64_i8 v[192:195], v[100:103], a[152:155], v[192:195]\n" \
  "ds_read_b64_tr_b8 v[126:127], v83 offset:9216\n" \
  "v_mfma_i32_16x16x64_i8 v[224:227], v[96:99], a[140:143], v[224:227]\n" \
  "v_mfma_i32_16x16x64_i8 v[224:227], v[100:103], a[156:159], v[224:227]\n" \
  "buffer_load_dwordx4 a[228:231], v44, s[28:31], s95 offen offset:1024 sc0 sc1\n" \
  "s_waitcnt lgkmcnt(8)\n" \
  "v_mfma_i32_16x16x64_i8 v[132:135], v[104:107], a[128:131], v[132:135]\n" \
  "ds_read_b64_tr_b8 v[96:97], v84 offset:0\n" \
  "v_mfma_i32_16x16x64_i8 v[132:135], v[108:111], a[144:147], v[132:135]\n" \
  "ds_read_b64_tr_b8 v[98:99], v84 offset:1024\n" \
  "v_mfma_i32_16x16x64_i8 v[164:167], v[104:107], a[132:135], v[164:167]\n" \
  "v_mfma_i32_16x16x64_i8 v[164:167], v[108:111], a[148:151], v[164:167]\n" \
  "buffer_load_dwordx4 a[232:235], v44, s[28:31], s95 offen offset:2048 sc0 sc1\n" \
  "v_mfma_i32_16x16x64_i8 v[196:199], v[104:107], a[136:139], v[196:199]\n" \
  "ds_read_b64_tr_b8 v[100:101], v84 offset:8192\n" \
  "v_mfma_i32_16x16x64_i8 v[196:199], v[108:111], a[152:155], v[196:199]\n" \
  "ds_read_b64_tr_b8 v[102:103], v84 offset:9216\n" \
  "v_mfma_i32_16x16x64_i8 v[228:231], v[104:107], a[140:143], v[228:231]\n" \
  "v_mfma_i32_16x16x64_i8 v[228:231], v[108:111], a[156:159], v[228:231]\n" \
  "s_waitcnt lgkmcnt(8)\n" \
  "v_mfma_i32_16x16x64_i8 v[136:139], v[112:115], a[128:131], v[136:139]\n" \
  "ds_read_b64_tr_b8 v[104:105], v85 offset:0\n" \
  "v_mfma_i32_16x16x64_i8 v[136:139], v[116:119], a[144:147], v[136:139]\n" \
  "ds_read_b64_tr_b8 v[106:107], v85 offset:1024\n" \
  "v_mfma_i32_16x16x64_i8 v[168:171], v[112:115], a[132:135], v[168:171]\n" \
  "v_mfma_i32_16x16x64_i8 v[168:171], v[116:119], a[148:151], v[168:171]\n" \
  "buffer_load_dwordx4 a[236:239], v44, s[28:31], s95 offen offset:3072 sc0 sc1\n" \
  "v_mfma_i32_16x16x64_i8 v[200:203], v[112:115], a[136:139], v[200:203]\n" \
  "ds_read_b64_tr_b8 v[108:109], v85 offset:8192\n" \
  "v_mfma_i32_16x16x64_i8 v[200:203], v[116:119], a[152:155], v[200:203]\n" \
  "ds_read_b64_tr_b8 v[110:111], v85 offset:9216\n" \
  "v_mfma_i32_16x16x64_i8 v[232:235], v[112:115], a[140:143], v[232:235]\n" \
  "v_mfma_i32_16x16x64_i8 v[232:235], v[116:119], a[156:159], v[232:235]\n" \
  "buffer_load_dwordx4 a[240:243], v91, s[28:31], s95 offen offset:0 sc0 sc1\n" \
  "s_waitcnt lgkmcnt(8)\n" \
  "v_mfma_i32_16x16x64_i8 v[140:143], v[120:123], a[128:131], v[140:143]\n" \
  "ds_read_b64_tr_b8 v[112:113], v86 offset:0\n" \
  "v_mfma_i32_16x16x64_i8 v[140:143], v[124:127], a[144:147], v[140:143]\n" \
  "ds_read_b64_tr_b8 v[114:115], v86 offset:1024\n" \
  "v_mfma_i32_16x16x64_i8 v[172:175], v[120:123], a[132:135], v[172:175]\n" \
  "v_mfma_i32_16x16x64_i8 v[172:175], v[124:127], a[148:151], v[172:175]\n" \
  "buffer_load_dwordx4 a[244:247], v91, s[28:31], s95 offen offset:1024 sc0 sc1\n" \
  "s_waitcnt vmcnt(31)\n" \
  "s_barrier\n" \
  "v_mfma_i32_16x16x64_i8 v[204:207], v[120:123], a[136:139], v[204:207]\n" \
  "ds_read_b64_tr_b8 v[116:117], v86 offset:8192\n" \
  "v_mfma_i32_16x16x64_i8 v[204:207], v[124:127], a[152:155], v[204:207]\n" \
  "ds_read_b64_tr_b8 v[118:119], v86 offset:9216\n" \
  "v_mfma_i32_16x16x64_i8 v[236:239], v[120:123], a[140:143], v[236:239]\n" \
  "v_mfma_i32_16x16x64_i8 v[236:239], v[124:127], a[156:159], v[236:239]\n" \
  "s_waitcnt lgkmcnt(8)\n" \
  "v_mfma_i32_16x16x64_i8 v[144:147], v[96:99], a[128:131], v[144:147]\n" \
  "ds_read_b64_tr_b8 v[120:121], v87 offset:0\n" \
  "v_mfma_i32_16x16x64_i8 v[144:147], v[100:103], a[144:147], v[144:147]\n" \
  "ds_read_b64_tr_b8 v[122:123], v87 offset:1024\n" \
  "v_mfma_i32_16x16x64_i8 v[176:179], v[96:99], a[132:135], v[176:179]\n" \
  "v_mfma_i32_16x16x64_i8 v[176:179], v[100:103], a[148:151], v[176:179]\n" \
  "buffer_load_dwordx4 a[248:251], v91, s[28:31], s95 offen offset:2048 sc0 sc1\n" \
  "v_mfma_i32_16x16x64_i8 v[208:211], v[96:99], a[136:139], v[208:211]\n" \
  "ds_read_b64_tr_b8 v[124:125], v87 offset:8192\n" \
  "v_mfma_i32_16x16x64_i8 v[208:211], v[100:103], a[152:155], v[208:211]\n" \
  "ds_read_b64_tr_b8 v[126:127], v87 offset:9216\n" \
  "v_mfma_i32_16x16x64_i8 v[240:243], v[96:99], a[140:143], v[240:243]\n" \
  "v_mfma_i32_16x16x64_i8 v[240:243], v[100:103], a[156:159], v[240:243]\n" \
  "buffer_load_dwordx4 a[252:255], v91, s[28:31], s95 offen offset:3072 sc0 sc1\n" \
  "s_waitcnt lgkmcnt(8)\n" \
  "v_mfma_i32_16x16x64_i8 v[148:151], v[104:107], a[128:131], v[148:151]\n" \
  "ds_read_b64_tr_b8 v[96:97], v80 offset:16384\n" \
  "v_mfma_i32_16x16x64_i8 v[148:151], v[108:111], a[144:147], v[148:151]\n" \
  "ds_read_b64_tr_b8 v[98:99], v80 offset:17408\n" \
  "v_mfma_i32_16x16x64_i8 v[180:183], v[104:107], a[132:135], v[180:183]\n" \
  "v_mfma_i32_16x16x64_i8 v[180:183], v[108:111], a[148:151], v[180:183]\n" \
  "buffer_load_dwordx4 a[0:3], v44, s[28:31], s88 offen offset:0 sc0 sc1\n" \
  "v_mfma_i32_16x16x64_i8 v[212:215], v[104:107], a[136:139], v[212:215]\n" \
  "ds_read_b64_tr_b8 v[100:101], v80 offset:24576\n" \
  "v_mfma_i32_16x16x64_i8 v[212:215], v[108:111], a[152:155], v[212:215]\n" \
  "ds_read_b64_tr_b8 v[102:103], v80 offset:25600\n" \
  "v_mfma_i32_16x16x64_i8 v[244:247], v[104:107], a[140:143], v[244:247]\n" \
  "v_mfma_i32_16x16x64_i8 v[244:247], v[108:111], a[156:159], v[244:247]\n" \
  "s_waitcnt lgkmcnt(8)\n" \
  "v_mfma_i32_16x16x64_i8 v[152:155], v[112:115], a[128:131], v[152:155]\n" \
  "ds_read_b64_tr_b8 v[104:105], v81 offset:16384\n" \
  "v_mfma_i32_16x16x64_i8 v[152:155], v[116:119], a[144:147], v[152:155]\n" \
  "ds_read_b64_tr_b8 v[106:107], v81 offset:17408\n" \
  "v_mfma_i32_16x16x64_i8 v[184:187], v[112:115], a[132:135], v[184:187]\n" \
  "v_readlane_b32 s50, v46, s49\n" \
  "v_mfma_i32_16x16x64_i8 v[184:187], v[116:119], a[148:151], v[184:187]\n" \
  "buffer_load_dwordx4 a[4:7], v44, s[28:31], s88 offen offset:1024 sc0 sc1\n" \
  "v_mfma_i32_16x16x64_i8 v[216:219], v[112:115], a[136:139], v[216:219]\n" \
  "ds_read_b64_tr_b8 v[108:109], v81 offset:24576\n" \
  "v_mfma_i32_16x16x64_i8 v[216:219], v[116:119], a[152:155], v[216:219]\n" \
  "ds_read_b64_tr_b8 v[110:111], v81 offset:25600\n" \
  "v_mfma_i32_16x16x64_i8 v[248:251], v[112:115], a[140:143], v[248:251]\n" \
  "v_mfma_i32_16x16x64_i8 v[248:251], v[116:119], a[156:159], v[248:251]\n" \
  "buffer_load_dwordx4 a[8:11], v44, s[28:31], s88 offen offset:2048 sc0 sc1\n" \
  "s_waitcnt lgkmcnt(8)\n" \
  "v_mfma_i32_16x16x64_i8 v[156:159], v[120:123], a[128:131], v[156:159]\n" \
  "ds_read_b64_tr_b8 v[112:113], v82 offset:16384\n" \
  "v_mfma_i32_16x16x64_i8 v[156:159], v[124:127], a[144:147], v[156:159]\n" \
  "ds_read_b64_tr_b8 v[114:115], v82 offset:17408\n" \
  "v_mfma_i32_16x16x64_i8 v[188:191], v[120:123], a[132:135], v[188:191]\n" \
  "s_lshl_b32 s81, s50, 21\n" \
  "s_lshl_b32 s89, s50, 16\n" \
  "s_add_u32 s49, s49, 1\n" \
  "v_mfma_i32_16x16x64_i8 v[188:191], v[124:127], a[148:151], v[188:191]\n" \
  "buffer_load_dwordx4 a[12:15], v44, s[28:31], s88 offen offset:3072 sc0 sc1\n" \
  "v_mfma_i32_16x16x64_i8 v[220:223], v[120:123], a[136:139], v[220:223]\n" \
  "ds_read_b64_tr_b8 v[116:117], v82 offset:24576\n" \
  "v_mfma_i32_16x16x64_i8 v[220:223], v[124:127], a[152:155], v[220:223]\n" \
  "ds_read_b64_tr_b8 v[118:119], v82 offset:25600\n" \
  "v_mfma_i32_16x16x64_i8 v[252:255], v[120:123], a[140:143], v[252:255]\n" \
  "v_mfma_i32_16x16x64_i8 v[252:255], v[124:127], a[156:159], v[252:255]\n" \
  "; ---- step 5\n" \
  "s_waitcnt vmcnt(21)\n" \
  "s_waitcnt lgkmcnt(8)\n" \
  "v_mfma_i32_16x16x64_i8 v[128:131], v[96:99], a[160:163], v[128:131]\n" \
  "ds_read_b64_tr_b8 v[120:121], v83 offset:16384\n" \
  "v_mfma_i32_16x16x64_i8 v[128:131], v[100:103], a[176:179], v[128:131]\n" \
  "ds_read_b64_tr_b8 v[122:123], v83 offset:17408\n" \
  "v_mfma_i32_16x16x64_i8 v[160:163], v[96:99], a[164:167], v[160:163]\n" \
  "v_mfma_i32_16x16x64_i8 v[160:163], v[100:103], a[180:183], v[160:163]\n" \
  "buffer_load_dwordx4 a[16:19], v91, s[28:31], s88 offen offset:0 sc0 sc1\n" \
  "v_mfma_i32_16x16x64_i8 v[192:195], v[96:99], a[168:171], v[192:195]\n" \
  "ds_read_b64_tr_b8 v[124:125], v83 offset:24576\n" \
  "v_mfma_i32_16x16x64_i8 v[192:195], v[100:103], a[184:187], v[192:195]\n" \
  "ds_read_b64_tr_b8 v[126:127], v83 offset:25600\n" \
  "v_mfma_i32_16x16x64_i8 v[224:227], v[96:99], a[172:175], v[224:227]\n" \
  "v_mfma_i32_16x16x64_i8 v[224:227], v[100:103], a[188:191], v[224:227]\n" \
  "buffer_load_dwordx4 a[20:23], v91, s[28:31], s88 offen offset:1024 sc0 sc1\n" \
  "s_waitcnt lgkmcnt(8)\n" \
  "v_mfma_i32_16x16x64_i8 v[132:135], v[104:107], a[160:163], v[132:135]\n" \
  "ds_read_b64_tr_b8 v[96:97], v84 offset:16384\n" \
  "v_mfma_i32_16x16x64_i8 v[132:135], v[108:111], a[176:179], v[132:135]\n" \
  "ds_read_b64_tr_b8 v[98:99], v84 offset:17408\n" \
  "v_mfma_i32_16x16x64_i8 v[164:167], v[104:107], a[164:167], v[164:167]\n" \
  "v_mfma_i32_16x16x64_i8 v[164:167], v[108:111], a[180:183], v[164:167]\n" \
  "buffer_load_dwordx4 a[24:27], v91, s[28:31], s88 offen offset:2048 sc0 sc1\n" \
  "v_mfma_i32_16x16x64_i8 v[196:199], v[104:107], a[168:171], v[196:199]\n" \
  "ds_read_b64_tr_b8 v[100:101], v84 offset:24576\n" \
  "v_mfma_i32_16x16x64_i8 v[196:199], v[108:111], a[184:187], v[196:199]\n" \
  "ds_read_b64_tr_b8 v[102:103], v84 offset:25600\n" \
  "v_mfma_i32_16x16x64_i8 v[228:231], v[104:107], a[172:175], v[228:231]\n" \
  "v_mfma_i32_16x16x64_i8 v[228:231], v[108:111], a[188:191], v[228:231]\n" \
  "buffer_load_dwordx4 a[28:31], v91, s[28:31], s88 offen offset:3072 sc0 sc1\n" \
  "s_waitcnt lgkmcnt(8)\n" \
  "v_mfma_i32_16x16x64_i8 v[136:139], v[112:115], a[160:163], v[136:139]\n" \
  "ds_read_b64_tr_b8 v[104:105], v85 offset:16384\n" \
  "v_mfma_i32_16x16x64_i8 v[136:139], v[116:119], a[176:179], v[136:139]\n" \
  "ds_read_b64_tr_b8 v[106:107], v85 offset:17408\n" \
  "v_mfma_i32_16x16x64_i8 v[168:171], v[112:115], a[164:167], v[168:171]\n" \
  "v_mfma_i32_16x16x64_i8 v[168:171], v[116:119], a[180:183], v[168:171]\n" \
  "buffer_load_dwordx4 a[32:35], v44, s[28:31], s89 offen offset:0 sc0 sc1\n" \
  "v_mfma_i32_16x16x64_i8 v[200:203], v[112:115], a[168:171], v[200:203]\n" \
  "ds_read_b64_tr_b8 v[108:109], v85 offset:24576\n" \
  "v_mfma_i32_16x16x64_i8 v[200:203], v[116:119], a[184:187], v[200:203]\n" \
  "ds_read_b64_tr_b8 v[110:111], v85 offset:25600\n" \
  "v_mfma_i32_16x16x64_i8 v[232:235], v[112:115], a[172:175], v[232:235]\n" \
  "v_mfma_i32_16x16x64_i8 v[232:235], v[116:119], a[188:191], v[232:235]\n" \
  "buffer_load_dwordx4 a[36:39], v44, s[28:31], s89 offen offset:1024 sc0 sc1\n" \
  "s_waitcnt lgkmcnt(8)\n" \
  "v_mfma_i32_16x16x64_i8 v[140:143], v[120:123], a[160:163], v[140:143]\n" \
  "ds_read_b64_tr_b8 v[112:113], v86 offset:16384\n" \
  "v_mfma_i32_16x16x64_i8 v[140:143], v[124:127], a[176:179], v[140:143]\n" \
  "ds_read_b64_tr_b8 v[114:115], v86 offset:17408\n" \
  "v_mfma_i32_16x16x64_i8 v[172:175], v[120:123], a[164:167], v[172:175]\n" \
  "v_readlane_b32 s50, v46, s49\n" \
  "v_mfma_i32_16x16x64_i8 v[172:175], v[124:127], a[180:183], v[172:175]\n" \
  "buffer_load_dwordx4 a[40:43], v44, s[28:31], s89 offen offset:2048 sc0 sc1\n" \
  "s_waitcnt vmcnt(40)\n" \
  "s_barrier\n" \
  "v_mfma_i32_16x16x64_i8 v[204:207], v[120:123], a[168:171], v[204:207]\n" \
  "ds_read_b64_tr_b8 v[116:117], v86 offset:24576\n" \
  "v_mfma_i32_16x16x64_i8 v[204:207], v[124:127], a[184:187], v[204:207]\n" \
  "ds_read_b64_tr_b8 v[118:119], v86 offset:25600\n" \
  "v_mfma_i32_16x16x64_i8 v[236:239], v[120:123], a[172:175], v[236:239]\n" \
  "v_mfma_i32_16x16x64_i8 v[236:239], v[124:127], a[188:191], v[236:239]\n" \
  "buffer_load_dwordx4 a[44:47], v44, s[28:31], s89 offen offset:3072 sc0 sc1\n" \
  "s_waitcnt lgkmcnt(8)\n" \
  "v_mfma_i32_16x16x64_i8 v[144:147], v[96:99], a[160:163], v[144:147]\n" \
  "ds_read_b64_tr_b8 v[120:121], v87 offset:16384\n" \
  "v_mfma_i32_16x16x64_i8 v[144:147], v[100:103], a[176:179], v[144:147]\n" \
  "ds_read_b64_tr_b8 v[122:123], v87 offset:17408\n" \
  "s_add_u32 m0, s47, 16384\n" \
  "v_mfma_i32_16x16x64_i8 v[176:179], v[96:99], a[164:167], v[176:179]\n" \
  "buffer_load_dwordx4 v40, s[16:19], s81 offen nt lds\n" \
  "s_lshl_b32 s82, s50, 21\n" \
  "s_lshl_b32 s90, s50, 16\n" \
  "s_add_u32 s49, s49, 1\n" \
  "v_mfma_i32_16x16x64_i8 v[176:179], v[100:103], a[180:183], v[176:179]\n" \
  "v_mfma_i32_16x16x64_i8 v[208:211], v[96:99], a[168:171], v[208:211]\n" \
  "ds_read_b64_tr_b8 v[124:125], v87 offset:24576\n" \
  "v_mfma_i32_16x16x64_i8 v[208:211], v[100:103], a[184:187], v[208:211]\n" \
  "ds_read_b64_tr_b8 v[126:127], v87 offset:25600\n" \
  "s_add_u32 m0, s47, 17408\n" \
  "v_mfma_i32_16x16x64_i8 v[240:243], v[96:99], a[172:175], v[240:243]\n" \
  "buffer_load_dwordx4 v41, s[16:19], s81 offen nt lds\n" \
  "v_readlane_b32 s50, v46, s49\n" \
  "v_mfma_i32_16x16x64_i8 v[240:243], v[100:103], a[188:191], v[240:243]\n" \
  "s_waitcnt lgkmcnt(8)\n" \
  "v_mfma_i32_16x16x64_i8 v[148:151], v[104:107], a[160:163], v[148:151]\n" \
  "ds_read_b64_tr_b8 v[96:97], v80 offset:32768\n" \
  "v_mfma_i32_16x16x64_i8 v[148:151], v[108:111], a[176:179], v[148:151]\n" \
  "ds_read_b64_tr_b8 v[98:99], v80 offset:33792\n" \
  "s_add_u32 m0, s47, 18432\n" \
  "v_mfma_i32_16x16x64_i8 v[180:183], v[104:107], a[164:167], v[180:183]\n" \
  "buffer_load_dwordx4 v42, s[16:19], s81 offen nt lds\n" \
  "v_mfma_i32_16x16x64_i8 v[180:183], v[108:111], a[180:183], v[180:183]\n" \
  "v_mfma_i32_16x16x64_i8 v[212:215], v[104:107], a[168:171], v[212:215]\n" \
  "ds_read_b64_tr_b8 v[100:101], v80 offset:40960\n" \
  "v_mfma_i32_16x16x64_i8 v[212:215], v[108:111], a[184:187], v[212:215]\n" \
  "ds_read_b64_tr_b8 v[102:103], v80 offset:41984\n" \
  "s_add_u32 m0, s47, 19456\n" \
  "v_mfma_i32_16x16x64_i8 v[244:247], v[104:107], a[172:175], v[244:247]\n" \
  "buffer_load_dwordx4 v43, s[16:19], s81 offen nt lds\n" \
  "s_lshl_b32 s83, s50, 21\n" \
  "s_lshl_b32 s91, s50, 16\n" \
  "s_add_u32 s49, s49, 1\n" \
  "v_mfma_i32_16x16x64_i8 v[244:247], v[108:111], a[188:191], v[244:247]\n" \
  "s_waitcnt lgkmcnt(8)\n" \
  "v_mfma_i32_16x16x64_i8 v[152:155], v[112:115], a[160:163], v[152:155]\n" \
  "ds_read_b64_tr_b8 v[104:105], v81 offset:32768\n" \
  "v_mfma_i32_16x16x64_i8 v[152:155], v[116:119], a[176:179], v[152:155]\n" \
  "ds_read_b64_tr_b8 v[106:107], v81 offset:33792\n" \
  "s_add_u32 m0, s47, 32768\n" \
  "v_mfma_i32_16x16x64_i8 v[184:187], v[112:115], a[164:167], v[184:187]\n" \
  "buffer_load_dwordx4 v40, s[16:19], s82 offen nt lds\n" \
  "v_readlane_b32 s50, v46, s49\n" \
  "v_mfma_i32_16x16x64_i8 v[184:187], v[116:119], a[180:183], v[184:187]\n" \
  "v_mfma_i32_16x16x64_i8 v[216:219], v[112:115], a[168:171], v[216:219]\n" \
  "ds_read_b64_tr_b8 v[108:109], v81 offset:40960\n" \
  "v_mfma_i32_16x16x64_i8 v[216:219], v[116:119], a[184:187], v[216:219]\n" \
  "ds_read_b64_tr_b8 v[110:111], v81 offset:41984\n" \
  "s_add_u32 m0, s47, 33792\n" \
  "v_mfma_i32_16x16x64_i8 v[248:251], v[112:115], a[172:175], v[248:251]\n" \
  "buffer_load_dwordx4 v41, s[16:19], s82 offen nt lds\n" \
  "v_mfma_i32_16x16x64_i8 v[248:251], v[116:119], a[188:191], v[248:251]\n" \
  "s_waitcnt lgkmcnt(8)\n" \
  "v_mfma_i32_16x16x64_i8 v[156:159], v[120:123], a[160:163], v[156:159]\n" \
  "ds_read_b64_tr_b8 v[112:113], v82 offset:32768\n" \
  "v_mfma_i32_16x16x64_i8 v[156:159], v[124:127], a[176:179], v[156:159]\n" \
  "ds_read_b64_tr_b8 v[114:115], v82 offset:33792\n" \
  "s_add_u32 m0, s47, 34816\n" \
  "v_mfma_i32_16x16x64_i8 v[188:191], v[120:123], a[164:167], v[188:191]\n" \
  "buffer_load_dwordx4 v42, s[16:19], s82 offen nt lds\n" \
  "s_lshl_b32 s84, s50, 21\n" \
  "s_lshl_b32 s92, s50, 16\n" \
  "s_add_u32 s49, s49, 1\n" \
  "v_mfma_i32_16x16x64_i8 v[188:191], v[124:127], a[180:183], v[188:191]\n" \
  "v_mfma_i32_16x16x64_i8 v[220:223], v[120:123], a[168:171], v[220:223]\n" \
  "ds_read_b64_tr_b8 v[116:117], v82 offset:40960\n" \
  "v_mfma_i32_16x16x64_i8 v[220:223], v[124:127], a[184:187], v[220:223]\n" \
  "ds_read_b64_tr_b8 v[118:119], v82 offset:41984\n" \
  "s_add_u32 m0, s47, 35840\n" \
  "v_mfma_i32_16x16x64_i8 v[252:255], v[120:123], a[172:175], v[252:255]\n" \
  "buffer_load_dwordx4 v43, s[16:19], s82 offen nt lds\n" \
  "v_mfma_i32_16x16x64_i8 v[252:255], v[124:127], a[188:191], v[252:255]\n" \
  "; ---- step 6\n" \
  "s_waitcnt vmcnt(29)\n" \
  "s_waitcnt lgkmcnt(8)\n" \
  "v_mfma_i32_16x16x64_i8 v[128:131], v[96:99], a[192:195], v[128:131]\n" \
  "ds_read_b64_tr_b8 v[120:121], v83 offset:32768\n" \
  "s_add_u32 m0, s47, 49152\n" \
  "v_mfma_i32_16x16x64_i8 v[128:131], v[100:103], a[208:211], v[128:131]\n" \
  "ds_read_b64_tr_b8 v[122:123], v83 offset:33792\n" \
  "buffer_load_dwordx4 v40, s[16:19], s83 offen lds\n" \
  "s_add_u32 m0, s47, 50176\n" \
  "v_mfma_i32_16x16x64_i8 v[160:163], v[96:99], a[196:199], v[160:163]\n" \
  "buffer_load_dwordx4 v41, s[16:19], s83 offen lds\n" \
  "s_add_u32 m0, s47, 51200\n" \
  "v_mfma_i32_16x16x64_i8 v[160:163], v[100:103], a[212:215], v[160:163]\n" \
  "buffer_load_dwordx4 v42, s[16:19], s83 offen lds\n" \
  "s_add_u32 m0, s47, 52224\n" \
  "v_mfma_i32_16x16x64_i8 v[192:195], v[96:99], a[200:203], v[192:195]\n" \
  "ds_read_b64_tr_b8 v[124:125], v83 offset:40960\n" \
  "buffer_load_dwordx4 v43, s[16:19], s83 offen lds\n" \
  "s_add_u32 m0, s47, 65536\n" \
  "v_mfma_i32_16x16x64_i8 v[192:195], v[100:103], a[216:219], v[192:195]\n" \
  "ds_read_b64_tr_b8 v[126:127], v83 offset:41984\n" \
  "buffer_load_dwordx4 v40, s[16:19], s84 offen lds\n" \
  "s_add_u32 m0, s47, 66560\n" \
  "v_mfma_i32_16x16x64_i8 v[224:227], v[96:99], a[204:207], v[224:227]\n" \
  "buffer_load_dwordx4 v41, s[16:19], s84 offen lds\n" \
  "s_add_u32 m0, s47, 67584\n" \
  "v_mfma_i32_16x16x64_i8 v[224:227], v[100:103], a[220:223], v[224:227]\n" \
  "buffer_load_dwordx4 v42, s[16:19], s84 offen lds\n" \
  "s_add_u32 m0, s47, 68608\n" \
  "s_waitcnt lgkmcnt(8)\n" \
  "v_mfma_i32_16x16x64_i8 v[132:135], v[104:107], a[192:195], v[132:135]\n" \
  "ds_read_b64_tr_b8 v[96:97], v84 offset:32768\n" \
  "buffer_load_dwordx4 v43, s[16:19], s84 offen lds\n" \
  "v_mfma_i32_16x16x64_i8 v[132:135], v[108:111], a[208:211], v[132:135]\n" \
  "ds_read_b64_tr_b8 v[98:99], v84 offset:33792\n" \
  "v_mfma_i32_16x16x64_i8 v[164:167], v[104:107], a[196:199], v[164:167]\n" \
  "v_mfma_i32_16x16x64_i8 v[164:167], v[108:111], a[212:215], v[164:167]\n" \
  "v_mfma_i32_16x16x64_i8 v[196:199], v[104:107], a[200:203], v[196:199]\n" \
  "ds_read_b64_tr_b8 v[100:101], v84 offset:40960\n" \
  "v_mfma_i32_16x16x64_i8 v[196:199], v[108:111], a[216:219], v[196:199]\n" \
  "ds_read_b64_tr_b8 v[102:103], v84 offset:41984\n" \
  "v_mfma_i32_16x16x64_i8 v[228:231], v[104:107], a[204:207], v[228:231]\n" \
  "v_mfma_i32_16x16x64_i8 v[228:231], v[108:111], a[220:223], v[228:231]\n" \
  "s_waitcnt lgkmcnt(8)\n" \
  "v_mfma_i32_16x16x64_i8 v[136:139], v[112:115], a[192:195], v[136:139]\n" \
  "ds_read_b64_tr_b8 v[104:105], v85 offset:32768\n" \
  "v_mfma_i32_16x16x64_i8 v[136:139], v[116:119], a[208:211], v[136:139]\n" \
  "ds_read_b64_tr_b8 v[106:107], v85 offset:33792\n" \
  "v_mfma_i32_16x16x64_i8 v[168:171], v[112:115], a[196:199], v[168:171]\n" \
  "v_mfma_i32_16x16x64_i8 v[168:171], v[116:119], a[212:215], v[168:171]\n" \
  "v_mfma_i32_16x16x64_i8 v[200:203], v[112:115], a[200:203], v[200:203]\n" \
  "ds_read_b64_tr_b8 v[108:109], v85 offset:40960\n" \
  "v_mfma_i32_16x16x64_i8 v[200:203], v[116:119], a[216:219], v[200:203]\n" \
  "ds_read_b64_tr_b8 v[110:111], v85 offset:41984\n" \
  "v_mfma_i32_16x16x64_i8 v[232:235], v[112:115], a[204:207], v[232:235]\n" \
  "v_mfma_i32_16x16x64_i8 v[232:235], v[116:119], a[220:223], v[232:235]\n" \
  "s_waitcnt lgkmcnt(8)\n" \
  "v_mfma_i32_16x16x64_i8 v[140:143], v[120:123], a[192:195], v[140:143]\n" \
  "ds_read_b64_tr_b8 v[112:113], v86 offset:32768\n" \
  "v_mfma_i32_16x16x64_i8 v[140:143], v[124:127], a[208:211], v[140:143]\n" \
  "ds_read_b64_tr_b8 v[114:115], v86 offset:33792\n" \
  "v_mfma_i32_16x16x64_i8 v[172:175], v[120:123], a[196:199], v[172:175]\n" \
  "v_mfma_i32_16x16x64_i8 v[172:175], v[124:127], a[212:215], v[172:175]\n" \
  "s_waitcnt vmcnt(53)\n" \
  "s_barrier\n" \
  "v_mfma_i32_16x16x64_i8 v[204:207], v[120:123], a[200:203], v[204:207]\n" \
  "ds_read_b64_tr_b8 v[116:117], v86 offset:40960\n" \
  "v_mfma_i32_16x16x64_i8 v[204:207], v[124:127], a[216:219], v[204:207]\n" \
  "ds_read_b64_tr_b8 v[118:119], v86 offset:41984\n" \
  "v_mfma_i32_16x16x64_i8 v[236:239], v[120:123], a[204:207], v[236:239]\n" \
  "v_mfma_i32_16x16x64_i8 v[236:239], v[124:127], a[220:223], v[236:239]\n" \
  "s_waitcnt lgkmcnt(8)\n" \
  "v_mfma_i32_16x16x64_i8 v[144:147], v[96:99], a[192:195], v[144:147]\n" \
  "ds_read_b64_tr_b8 v[120:121], v87 offset:32768\n" \
  "v_mfma_i32_16x16x64_i8 v[144:147], v[100:103], a[208:211], v[144:147]\n" \
  "ds_read_b64_tr_b8 v[122:123], v87 offset:33792\n" \
  "v_mfma_i32_16x16x64_i8 v[176:179], v[96:99], a[196:199], v[176:179]\n" \
  "v_mfma_i32_16x16x64_i8 v[176:179], v[100:103], a[212:215], v[176:179]\n" \
  "v_mfma_i32_16x16x64_i8 v[208:211], v[96:99], a[200:203], v[208:211]\n" \
  "ds_read_b64_tr_b8 v[124:125], v87 offset:40960\n" \
  "v_mfma_i32_16x16x64_i8 v[208:211], v[100:103], a[216:219], v[208:211]\n" \
  "ds_read_b64_tr_b8 v[126:127], v87 offset:41984\n" \
  "v_mfma_i32_16x16x64_i8 v[240:243], v[96:99], a[204:207], v[240:243]\n" \
  "v_mfma_i32_16x16x64_i8 v[240:243], v[100:103], a[220:223], v[240:243]\n" \
  "s_waitcnt lgkmcnt(8)\n" \
  "v_mfma_i32_16x16x64_i8 v[148:151], v[104:107], a[192:195], v[148:151]\n" \
  "ds_read_b64_tr_b8 v[96:97], v80 offset:49152\n" \
  "v_mfma_i32_16x16x64_i8 v[148:151], v[108:111], a[208:211], v[148:151]\n" \
  "ds_read_b64_tr_b8 v[98:99], v80 offset:50176\n" \
  "v_mfma_i32_16x16x64_i8 v[180:183], v[104:107], a[196:199], v[180:183]\n" \
  "v_mfma_i32_16x16x64_i8 v[180:183], v[108:111], a[212:215], v[180:183]\n" \
  "v_mfma_i32_16x16x64_i8 v[212:215], v[104:107], a[200:203], v[212:215]\n" \
  "ds_read_b64_tr_b8 v[100:101], v80 offset:57344\n" \
  "v_mfma_i32_16x16x64_i8 v[212:215], v[108:111], a[216:219], v[212:215]\n" \
  "ds_read_b64_tr_b8 v[102:103], v80 offset:58368\n" \
  "v_mfma_i32_16x16x64_i8 v[244:247], v[104:107], a[204:207], v[244:247]\n" \
  "v_mfma_i32_16x16x64_i8 v[244:247], v[108:111], a[220:223], v[244:247]\n" \
  "s_waitcnt lgkmcnt(8)\n" \
  "v_mfma_i32_16x16x64_i8 v[152:155], v[112:115], a[192:195], v[152:155]\n" \
  "ds_read_b64_tr_b8 v[104:105], v81 offset:49152\n" \
  "v_mfma_i32_16x16x64_i8 v[152:155], v[116:119], a[208:211], v[152:155]\n" \
  "ds_read_b64_tr_b8 v[106:107], v81 offset:50176\n" \
  "v_mfma_i32_16x16x64_i8 v[184:187], v[112:115], a[196:199], v[184:187]\n" \
  "v_mfma_i32_16x16x64_i8 v[184:187], v[116:119], a[212:215], v[184:187]\n" \
  "v_mfma_i32_16x16x64_i8 v[216:219], v[112:115], a[200:203], v[216:219]\n" \
  "ds_read_b64_tr_b8 v[108:109], v81 offset:57344\n" \
  "v_mfma_i32_16x16x64_i8 v[216:219], v[116:119], a[216:219], v[216:219]\n" \
  "ds_read_b64_tr_b8 v[110:111], v81 offset:58368\n" \
  "v_mfma_i32_16x16x64_i8 v[248:251], v[112:115], a[204:207], v[248:251]\n" \
  "v_mfma_i32_16x16x64_i8 v[248:251], v[116:119], a[220:223], v[248:251]\n" \
  "s_waitcnt lgkmcnt(8)\n" \
  "v_mfma_i32_16x16x64_i8 v[156:159], v[120:123], a[192:195], v[156:159]\n" \
  "ds_read_b64_tr_b8 v[112:113], v82 offset:49152\n" \
  "v_mfma_i32_16x16x64_i8 v[156:159], v[124:127], a[208:211], v[156:159]\n" \
  "ds_read_b64_tr_b8 v[114:115], v82 offset:50176\n" \
  "v_mfma_i32_16x16x64_i8 v[188:191], v[120:123], a[196:199], v[188:191]\n" \
  "v_mfma_i32_16x16x64_i8 v[188:191], v[124:127], a[212:215], v[188:191]\n" \
  "v_mfma_i32_16x16x64_i8 v[220:223], v[120:123], a[200:203], v[220:223]\n" \
  "ds_read_b64_tr_b8 v[116:117], v82 offset:57344\n" \
  "v_mfma_i32_16x16x64_i8 v[220:223], v[124:127], a[216:219], v[220:223]\n" \
  "ds_read_b64_tr_b8 v[118:119], v82 offset:58368\n" \
  "v_mfma_i32_16x16x64_i8 v[252:255], v[120:123], a[204:207], v[252:255]\n" \
  "v_mfma_i32_16x16x64_i8 v[252:255], v[124:127], a[220:223], v[252:255]\n" \
  "; ---- step 7\n" \
  "s_waitcnt vmcnt(28)\n" \
  "s_waitcnt lgkmcnt(8)\n" \
  "v_mfma_i32_16x16x64_i8 v[128:131], v[96:99], a[224:227], v[128:131]\n" \
  "ds_read_b64_tr_b8 v[120:121], v83 offset:49152\n" \
  "v_mfma_i32_16x16x64_i8 v[128:131], v[100:103], a[240:243], v[128:131]\n" \
  "ds_read_b64_tr_b8 v[122:123], v83 offset:50176\n" \
  "v_mfma_i32_16x16x64_i8 v[160:163], v[96:99], a[228:231], v[160:163]\n" \
  "v_mfma_i32_16x16x64_i8 v[160:163], v[100:103], a[244:247], v[160:163]\n" \
  "v_mfma_i32_16x16x64_i8 v[192:195], v[96:99], a[232:235], v[192:195]\n" \
  "ds_read_b64_tr_b8 v[124:125], v83 offset:57344\n" \
  "v_mfma_i32_16x16x64_i8 v[192:195], v[100:103], a[248:251], v[192:195]\n" \
  "ds_read_b64_tr_b8 v[126:127], v83 offset:58368\n" \
  "v_mfma_i32_16x16x64_i8 v[224:227], v[96:99], a[236:239], v[224:227]\n" \
  "v_mfma_i32_16x16x64_i8 v[224:227], v[100:103], a[252:255], v[224:227]\n" \
  "s_waitcnt lgkmcnt(8)\n" \
  "v_mfma_i32_16x16x64_i8 v[132:135], v[104:107], a[224:227], v[132:135]\n" \
  "ds_read_b64_tr_b8 v[96:97], v84 offset:49152\n" \
  "v_mfma_i32_16x16x64_i8 v[132:135], v[108:111], a[240:243], v[132:135]\n" \
  "ds_read_b64_tr_b8 v[98:99], v84 offset:50176\n" \
  "v_mfma_i32_16x16x64_i8 v[164:167], v[104:107], a[228:231], v[164:167]\n" \
  "v_mfma_i32_16x16x64_i8 v[164:167], v[108:111], a[244:247], v[164:167]\n" \
  "v_mfma_i32_16x16x64_i8 v[196:199], v[104:107], a[232:235], v[196:199]\n" \
  "ds_read_b64_tr_b8 v[100:101], v84 offset:57344\n" \
  "v_mfma_i32_16x16x64_i8 v[196:199], v[108:111], a[248:251], v[196:199]\n" \
  "ds_read_b64_tr_b8 v[102:103], v84 offset:58368\n" \
  "v_mfma_i32_16x16x64_i8 v[228:231], v[104:107], a[236:239], v[228:231]\n" \
  "v_mfma_i32_16x16x64_i8 v[228:231], v[108:111], a[252:255], v[228:231]\n" \
  "s_waitcnt lgkmcnt(8)\n" \
  "v_mfma_i32_16x16x64_i8 v[136:139], v[112:115], a[224:227], v[136:139]\n" \
  "ds_read_b64_tr_b8 v[104:105], v85 offset:49152\n" \
  "v_mfma_i32_16x16x64_i8 v[136:139], v[116:119], a[240:243], v[136:139]\n" \
  "ds_read_b64_tr_b8 v[106:107], v85 offset:50176\n" \
  "v_mfma_i32_16x16x64_i8 v[168:171], v[112:115], a[228:231], v[168:171]\n" \
  "v_mfma_i32_16x16x64_i8 v[168:171], v[116:119], a[244:247], v[168:171]\n" \
  "v_mfma_i32_16x16x64_i8 v[200:203], v[112:115], a[232:235], v[200:203]\n" \
  "ds_read_b64_tr_b8 v[108:109], v85 offset:57344\n" \
  "v_mfma_i32_16x16x64_i8 v[200:203], v[116:119], a[248:251], v[200:203]\n" \
  "ds_read_b64_tr_b8 v[110:111], v85 offset:58368\n" \
  "buffer_load_dwordx4 a[48:51], v91, s[28:31], s89 offen offset:0 sc0 sc1\n" \
  "v_mfma_i32_16x16x64_i8 v[232:235], v[112:115], a[236:239], v[232:235]\n" \
  "v_mfma_i32_16x16x64_i8 v[232:235], v[116:119], a[252:255], v[232:235]\n" \
  "buffer_load_dwordx4 a[52:55], v91, s[28:31], s89 offen offset:1024 sc0 sc1\n" \
  "s_waitcnt lgkmcnt(8)\n" \
  "v_mfma_i32_16x16x64_i8 v[140:143], v[120:123], a[224:227], v[140:143]\n" \
  "ds_read_b64_tr_b8 v[112:113], v86 offset:49152\n" \
  "v_mfma_i32_16x16x64_i8 v[140:143], v[124:127], a[240:243], v[140:143]\n" \
  "ds_read_b64_tr_b8 v[114:115], v86 offset:50176\n" \
  "v_mfma_i32_16x16x64_i8 v[172:175], v[120:123], a[228:231], v[172:175]\n" \
  "v_mfma_i32_16x16x64_i8 v[172:175], v[124:127], a[244:247], v[172:175]\n" \
  "buffer_load_dwordx4 a[56:59], v91, s[28:31], s89 offen offset:2048 sc0 sc1\n" \
  "s_waitcnt vmcnt(52)\n" \
  "s_barrier\n" \
  "v_mfma_i32_16x16x64_i8 v[204:207], v[120:123], a[232:235], v[204:207]\n" \
  "ds_read_b64_tr_b8 v[116:117], v86 offset:57344\n" \
  "v_mfma_i32_16x16x64_i8 v[204:207], v[124:127], a[248:251], v[204:207]\n" \
  "ds_read_b64_tr_b8 v[118:119], v86 offset:58368\n" \
  "v_mfma_i32_16x16x64_i8 v[236:239], v[120:123], a[236:239], v[236:239]\n" \
  "v_mfma_i32_16x16x64_i8 v[236:239], v[124:127], a[252:255], v[236:239]\n" \
  "buffer_load_dwordx4 a[60:63], v91, s[28:31], s89 offen offset:3072 sc0 sc1\n" \
  "s_waitcnt lgkmcnt(8)\n" \
  "v_mfma_i32_16x16x64_i8 v[144:147], v[96:99], a[224:227], v[144:147]\n" \
  "ds_read_b64_tr_b8 v[120:121], v87 offset:49152\n" \
  "v_mfma_i32_16x16x64_i8 v[144:147], v[100:103], a[240:243], v[144:147]\n" \
  "ds_read_b64_tr_b8 v[122:123], v87 offset:50176\n" \
  "v_mfma_i32_16x16x64_i8 v[176:179], v[96:99], a[228:231], v[176:179]\n" \
  "v_mfma_i32_16x16x64_i8 v[176:179], v[100:103], a[244:247], v[176:179]\n" \
  "buffer_load_dwordx4 a[64:67], v44, s[28:31], s90 offen offset:0 sc0 sc1\n" \
  "v_mfma_i32_16x16x64_i8 v[208:211], v[96:99], a[232:235], v[208:211]\n" \
  "ds_read_b64_tr_b8 v[124:125], v87 offset:57344\n" \
  "v_mfma_i32_16x16x64_i8 v[208:211], v[100:103], a[248:251], v[208:211]\n" \
  "ds_read_b64_tr_b8 v[126:127], v87 offset:58368\n" \
  "v_mfma_i32_16x16x64_i8 v[240:243], v[96:99], a[236:239], v[240:243]\n" \
  "v_mfma_i32_16x16x64_i8 v[240:243], v[100:103], a[252:255], v[240:243]\n" \
  "buffer_load_dwordx4 a[68:71], v44, s[28:31], s90 offen offset:1024 sc0 sc1\n" \
  "s_waitcnt lgkmcnt(8)\n" \
  "v_mfma_i32_16x16x64_i8 v[148:151], v[104:107], a[224:227], v[148:151]\n" \
  "ds_read_b64_tr_b8 v[96:97], v32 offset:0\n" \
  "v_mfma_i32_16x16x64_i8 v[148:151], v[108:111], a[240:243], v[148:151]\n" \
  "ds_read_b64_tr_b8 v[98:99], v32 offset:1024\n" \
  "v_mfma_i32_16x16x64_i8 v[180:183], v[104:107], a[228:231], v[180:183]\n" \
  "v_mfma_i32_16x16x64_i8 v[180:183], v[108:111], a[244:247], v[180:183]\n" \
  "buffer_load_dwordx4 a[72:75], v44, s[28:31], s90 offen offset:2048 sc0 sc1\n" \
  "v_mfma_i32_16x16x64_i8 v[212:215], v[104:107], a[232:235], v[212:215]\n" \
  "ds_read_b64_tr_b8 v[100:101], v32 offset:8192\n" \
  "v_mfma_i32_16x16x64_i8 v[212:215], v[108:111], a[248:251], v[212:215]\n" \
  "ds_read_b64_tr_b8 v[102:103], v32 offset:9216\n" \
  "buffer_load_dwordx4 a[76:79], v44, s[28:31], s90 offen offset:3072 sc0 sc1\n" \
  "v_mfma_i32_16x16x64_i8 v[244:247], v[104:107], a[236:239], v[244:247]\n" \
  "v_mfma_i32_16x16x64_i8 v[244:247], v[108:111], a[252:255], v[244:247]\n" \
  "s_waitcnt lgkmcnt(8)\n" \
  "v_mfma_i32_16x16x64_i8 v[152:155], v[112:115], a[224:227], v[152:155]\n" \
  "ds_read_b64_tr_b8 v[104:105], v33 offset:0\n" \
  "v_mfma_i32_16x16x64_i8 v[152:155], v[116:119], a[240:243], v[152:155]\n" \
  "ds_read_b64_tr_b8 v[106:107], v33 offset:1024\n" \
  "buffer_load_dwordx4 a[80:83], v91, s[28:31], s90 offen offset:0 sc0 sc1\n" \
  "v_mfma_i32_16x16x64_i8 v[184:187], v[112:115], a[228:231], v[184:187]\n" \
  "v_mfma_i32_16x16x64_i8 v[184:187], v[116:119], a[244:247], v[184:187]\n" \
  "v_mfma_i32_16x16x64_i8 v[216:219], v[112:115], a[232:235], v[216:219]\n" \
  "ds_read_b64_tr_b8 v[108:109], v33 offset:8192\n" \
  "v_mfma_i32_16x16x64_i8 v[216:219], v[116:119], a[248:251], v[216:219]\n" \
  "ds_read_b64_tr_b8 v[110:111], v33 offset:9216\n" \
  "buffer_load_dwordx4 a[84:87], v91, s[28:31], s90 offen offset:1024 sc0 sc1\n" \
  "v_mfma_i32_16x16x64_i8 v[248:251], v[112:115], a[236:239], v[248:251]\n" \
  "v_mfma_i32_16x16x64_i8 v[248:251], v[116:119], a[252:255], v[248:251]\n" \
  "s_waitcnt lgkmcnt(8)\n" \
  "v_mfma_i32_16x16x64_i8 v[156:159], v[120:123], a[224:227], v[156:159]\n" \
  "ds_read_b64_tr_b8 v[112:113], v34 offset:0\n" \
  "v_mfma_i32_16x16x64_i8 v[156:159], v[124:127], a[240:243], v[156:159]\n" \
  "ds_read_b64_tr_b8 v[114:115], v34 offset:1024\n" \
  "buffer_load_dwordx4 a[88:91], v91, s[28:31], s90 offen offset:2048 sc0 sc1\n" \
  "v_mfma_i32_16x16x64_i8 v[188:191], v[120:123], a[228:231], v[188:191]\n" \
  "v_mfma_i32_16x16x64_i8 v[188:191], v[124:127], a[244:247], v[188:191]\n" \
  "v_mfma_i32_16x16x64_i8 v[220:223], v[120:123], a[232:235], v[220:223]\n" \
  "ds_read_b64_tr_b8 v[116:117], v34 offset:8192\n" \
  "v_mfma_i32_16x16x64_i8 v[220:223], v[124:127], a[248:251], v[220:223]\n" \
  "ds_read_b64_tr_b8 v[118:119], v34 offset:9216\n" \
  "buffer_load_dwordx4 a[92:95], v91, s[28:31], s90 offen offset:3072 sc0 sc1\n" \
  "v_mfma_i32_16x16x64_i8 v[252:255], v[120:123], a[236:239], v[252:255]\n" \
  "v_mfma_i32_16x16x64_i8 v[252:255], v[124:127], a[252:255], v[252:255]\n" \
  "s_add_u32 s53, s53, 1\n" \
  "s_add_u32 s54, s54, 1\n" \
  "s_sub_u32 s51, s51, 1\n" \
  "s_cmp_lg_u32 s51, 0\n" \
  "s_cbranch_scc1 L_loopB_%=\n" \
  "; ---- step 0\n" \
  "s_waitcnt vmcnt(32)\n" \
  "s_waitcnt lgkmcnt(8)\n" \
  "v_mfma_i32_16x16x64_i8 v[128:131], v[96:99], a[0:3], v[128:131]\n" \
  "ds_read_b64_tr_b8 v[120:121], v35 offset:0\n" \
  "v_mfma_i32_16x16x64_i8 v[128:131], v[100:103], a[16:19], v[128:131]\n" \
  "ds_read_b64_tr_b8 v[122:123], v35 offset:1024\n" \
  "v_mfma_i32_16x16x64_i8 v[160:163], v[96:99], a[4:7], v[160:163]\n" \
  "v_mfma_i32_16x16x64_i8 v[160:163], v[100:103], a[20:23], v[160:163]\n" \
  "buffer_load_dwordx4 a[96:99], v44, s[28:31], s91 offen offset:0 sc0 sc1\n" \
  "v_mfma_i32_16x16x64_i8 v[192:195], v[96:99], a[8:11], v[192:195]\n" \
  "ds_read_b64_tr_b8 v[124:125], v35 offset:8192\n" \
  "v_mfma_i32_16x16x64_i8 v[192:195], v[100:103], a[24:27], v[192:195]\n" \
  "ds_read_b64_tr_b8 v[126:127], v35 offset:9216\n" \
  "v_mfma_i32_16x16x64_i8 v[224:227], v[96:99], a[12:15], v[224:227]\n" \
  "v_mfma_i32_16x16x64_i8 v[224:227], v[100:103], a[28:31], v[224:227]\n" \
  "buffer_load_dwordx4 a[100:103], v44, s[28:31], s91 offen offset:1024 sc0 sc1\n" \
  "s_waitcnt lgkmcnt(8)\n" \
  "v_mfma_i32_16x16x64_i8 v[132:135], v[104:107], a[0:3], v[132:135]\n" \
  "ds_read_b64_tr_b8 v[96:97], v36 offset:0\n" \
  "v_mfma_i32_16x16x64_i8 v[132:135], v[108:111], a[16:19], v[132:135]\n" \
  "ds_read_b64_tr_b8 v[98:99], v36 offset:1024\n" \
  "v_mfma_i32_16x16x64_i8 v[164:167], v[104:107], a[4:7], v[164:167]\n" \
  "v_mfma_i32_16x16x64_i8 v[164:167], v[108:111], a[20:23], v[164:167]\n" \
  "buffer_load_dwordx4 a[104:107], v44, s[28:31], s91 offen offset:2048 sc0 sc1\n" \
  "v_mfma_i32_16x16x64_i8 v[196:199], v[104:107], a[8:11], v[196:199]\n" \
  "ds_read_b64_tr_b8 v[100:101], v36 offset:8192\n" \
  "v_mfma_i32_16x16x64_i8 v[196:199], v[108:111], a[24:27], v[196:199]\n" \
  "ds_read_b64_tr_b8 v[102:103], v36 offset:9216\n" \
  "v_mfma_i32_16x16x64_i8 v[228:231], v[104:107], a[12:15], v[228:231]\n" \
  "v_mfma_i32_16x16x64_i8 v[228:231], v[108:111], a[28:31], v[228:231]\n" \
  "s_waitcnt lgkmcnt(8)\n" \
  "v_mfma_i32_16x16x64_i8 v[136:139], v[112:115], a[0:3], v[136:139]\n" \
  "ds_read_b64_tr_b8 v[104:105], v37 offset:0\n" \
  "v_mfma_i32_16x16x64_i8 v[136:139], v[116:119], a[16:19], v[136:139]\n" \
  "ds_read_b64_tr_b8 v[106:107], v37 offset:1024\n" \
  "v_mfma_i32_16x16x64_i8 v[168:171], v[112:115], a[4:7], v[168:171]\n" \
  "v_mfma_i32_16x16x64_i8 v[168:171], v[116:119], a[20:23], v[168:171]\n" \
  "buffer_load_dwordx4 a[108:111], v44, s[28:31], s91 offen offset:3072 sc0 sc1\n" \
  "v_mfma_i32_16x16x64_i8 v[200:203], v[112:115], a[8:11], v[200:203]\n" \
  "ds_read_b64_tr_b8 v[108:109], v37 offset:8192\n" \
  "v_mfma_i32_16x16x64_i8 v[200:203], v[116:119], a[24:27], v[200:203]\n" \
  "ds_read_b64_tr_b8 v[110:111], v37 offset:9216\n" \
  "v_mfma_i32_16x16x64_i8 v[232:235], v[112:115], a[12:15], v[232:235]\n" \
  "v_mfma_i32_16x16x64_i8 v[232:235], v[116:119], a[28:31], v[232:235]\n" \
  "buffer_load_dwordx4 a[112:115], v91, s[28:31], s91 offen offset:0 sc0 sc1\n" \
  "s_waitcnt lgkmcnt(8)\n" \
  "v_mfma_i32_16x16x64_i8 v[140:143], v[120:123], a[0:3], v[140:143]\n" \
  "ds_read_b64_tr_b8 v[112:113], v38 offset:0\n" \
  "v_mfma_i32_16x16x64_i8 v[140:143], v[124:127], a[16:19], v[140:143]\n" \
  "ds_read_b64_tr_b8 v[114:115], v38 offset:1024\n" \
  "v_mfma_i32_16x16x64_i8 v[172:175], v[120:123], a[4:7], v[172:175]\n" \
  "v_mfma_i32_16x16x64_i8 v[172:175], v[124:127], a[20:23], v[172:175]\n" \
  "buffer_load_dwordx4 a[116:119], v91, s[28:31], s91 offen offset:1024 sc0 sc1\n" \
  "s_waitcnt vmcnt(30)\n" \
  "s_barrier\n" \
  "v_mfma_i32_16x16x64_i8 v[204:207], v[120:123], a[8:11], v[204:207]\n" \
  "ds_read_b64_tr_b8 v[116:117], v38 offset:8192\n" \
  "v_mfma_i32_16x16x64_i8 v[204:207], v[124:127], a[24:27], v[204:207]\n" \
  "ds_read_b64_tr_b8 v[118:119], v38 offset:9216\n" \
  "v_mfma_i32_16x16x64_i8 v[236:239], v[120:123], a[12:15], v[236:239]\n" \
  "v_mfma_i32_16x16x64_i8 v[236:239], v[124:127], a[28:31], v[236:239]\n" \
  "s_waitcnt lgkmcnt(8)\n" \
  "v_mfma_i32_16x16x64_i8 v[144:147], v[96:99], a[0:3], v[144:147]\n" \
  "ds_read_b64_tr_b8 v[120:121], v39 offset:0\n" \
  "v_mfma_i32_16x16x64_i8 v[144:147], v[100:103], a[16:19], v[144:147]\n" \
  "ds_read_b64_tr_b8 v[122:123], v39 offset:1024\n" \
  "v_mfma_i32_16x16x64_i8 v[176:179], v[96:99], a[4:7], v[176:179]\n" \
  "v_mfma_i32_16x16x64_i8 v[176:179], v[100:103], a[20:23], v[176:179]\n" \
  "buffer_load_dwordx4 a[120:123], v91, s[28:31], s91 offen offset:2048 sc0 sc1\n" \
  "v_mfma_i32_16x16x64_i8 v[208:211], v[96:99], a[8:11], v[208:211]\n" \
  "ds_read_b64_tr_b8 v[124:125], v39 offset:8192\n" \
  "v_mfma_i32_16x16x64_i8 v[208:211], v[100:103], a[24:27], v[208:211]\n" \
  "ds_read_b64_tr_b8 v[126:127], v39 offset:9216\n" \
  "v_mfma_i32_16x16x64_i8 v[240:243], v[96:99], a[12:15], v[240:243]\n" \
  "v_mfma_i32_16x16x64_i8 v[240:243], v[100:103], a[28:31], v[240:243]\n" \
  "buffer_load_dwordx4 a[124:127], v91, s[28:31], s91 offen offset:3072 sc0 sc1\n" \
  "s_waitcnt lgkmcnt(8)\n" \
  "v_mfma_i32_16x16x64_i8 v[148:151], v[104:107], a[0:3], v[148:151]\n" \
  "ds_read_b64_tr_b8 v[96:97], v32 offset:16384\n" \
  "v_mfma_i32_16x16x64_i8 v[148:151], v[108:111], a[16:19], v[148:151]\n" \
  "ds_read_b64_tr_b8 v[98:99], v32 offset:17408\n" \
  "v_mfma_i32_16x16x64_i8 v[180:183], v[104:107], a[4:7], v[180:183]\n" \
  "v_mfma_i32_16x16x64_i8 v[180:183], v[108:111], a[20:23], v[180:183]\n" \
  "buffer_load_dwordx4 a[128:131], v44, s[28:31], s92 offen offset:0 sc0 sc1\n" \
  "v_mfma_i32_16x16x64_i8 v[212:215], v[104:107], a[8:11], v[212:215]\n" \
  "ds_read_b64_tr_b8 v[100:101], v32 offset:24576\n" \
  "v_mfma_i32_16x16x64_i8 v[212:215], v[108:111], a[24:27], v[212:215]\n" \
  "ds_read_b64_tr_b8 v[102:103], v32 offset:25600\n" \
  "v_mfma_i32_16x16x64_i8 v[244:247], v[104:107], a[12:15], v[244:247]\n" \
  "v_mfma_i32_16x16x64_i8 v[244:247], v[108:111], a[28:31], v[244:247]\n" \
  "s_waitcnt lgkmcnt(8)\n" \
  "v_mfma_i32_16x16x64_i8 v[152:155], v[112:115], a[0:3], v[152:155]\n" \
  "ds_read_b64_tr_b8 v[104:105], v33 offset:16384\n" \
  "v_mfma_i32_16x16x64_i8 v[152:155], v[116:119], a[16:19], v[152:155]\n" \
  "ds_read_b64_tr_b8 v[106:107], v33 offset:17408\n" \
  "v_mfma_i32_16x16x64_i8 v[184:187], v[112:115], a[4:7], v[184:187]\n" \
  "v_readlane_b32 s50, v46, s49\n" \
  "v_mfma_i32_16x16x64_i8 v[184:187], v[116:119], a[20:23], v[184:187]\n" \
  "buffer_load_dwordx4 a[132:135], v44, s[28:31], s92 offen offset:1024 sc0 sc1\n" \
  "v_mfma_i32_16x16x64_i8 v[216:219], v[112:115], a[8:11], v[216:219]\n" \
  "ds_read_b64_tr_b8 v[108:109], v33 offset:24576\n" \
  "v_mfma_i32_16x16x64_i8 v[216:219], v[116:119], a[24:27], v[216:219]\n" \
  "ds_read_b64_tr_b8 v[110:111], v33 offset:25600\n" \
  "v_mfma_i32_16x16x64_i8 v[248:251], v[112:115], a[12:15], v[248:251]\n" \
  "v_mfma_i32_16x16x64_i8 v[248:251], v[116:119], a[28:31], v[248:251]\n" \
  "buffer_load_dwordx4 a[136:139], v44, s[28:31], s92 offen offset:2048 sc0 sc1\n" \
  "s_waitcnt lgkmcnt(8)\n" \
  "v_mfma_i32_16x16x64_i8 v[156:159], v[120:123], a[0:3], v[156:159]\n" \
  "ds_read_b64_tr_b8 v[112:113], v34 offset:16384\n" \
  "v_mfma_i32_16x16x64_i8 v[156:159], v[124:127], a[16:19], v[156:159]\n" \
  "ds_read_b64_tr_b8 v[114:115], v34 offset:17408\n" \
  "v_mfma_i32_16x16x64_i8 v[188:191], v[120:123], a[4:7], v[188:191]\n" \
  "s_lshl_b32 s85, s50, 21\n" \
  "s_lshl_b32 s93, s50, 16\n" \
  "s_add_u32 s49, s49, 1\n" \
  "v_mfma_i32_16x16x64_i8 v[188:191], v[124:127], a[20:23], v[188:191]\n" \
  "buffer_load_dwordx4 a[140:143], v44, s[28:31], s92 offen offset:3072 sc0 sc1\n" \
  "v_mfma_i32_16x16x64_i8 v[220:223], v[120:123], a[8:11], v[220:223]\n" \
  "ds_read_b64_tr_b8 v[116:117], v34 offset:24576\n" \
  "v_mfma_i32_16x16x64_i8 v[220:223], v[124:127], a[24:27], v[220:223]\n" \
  "ds_read_b64_tr_b8 v[118:119], v34 offset:25600\n" \
  "v_mfma_i32_16x16x64_i8 v[252:255], v[120:123], a[12:15], v[252:255]\n" \
  "v_mfma_i32_16x16x64_i8 v[252:255], v[124:127], a[28:31], v[252:255]\n" \
  "; ---- step 1\n" \
  "s_waitcnt vmcnt(20)\n" \
  "s_waitcnt lgkmcnt(8)\n" \
  "v_mfma_i32_16x16x64_i8 v[128:131], v[96:99], a[32:35], v[128:131]\n" \
  "ds_read_b64_tr_b8 v[120:121], v35 offset:16384\n" \
  "v_mfma_i32_16x16x64_i8 v[128:131], v[100:103], a[48:51], v[128:131]\n" \
  "ds_read_b64_tr_b8 v[122:123], v35 offset:17408\n" \
  "v_mfma_i32_16x16x64_i8 v[160:163], v[96:99], a[36:39], v[160:163]\n" \
  "v_mfma_i32_16x16x64_i8 v[160:163], v[100:103], a[52:55], v[160:163]\n" \
  "buffer_load_dwordx4 a[144:147], v91, s[28:31], s92 offen offset:0 sc0 sc1\n" \
  "v_mfma_i32_16x16x64_i8 v[192:195], v[96:99], a[40:43], v[192:195]\n" \
  "ds_read_b64_tr_b8 v[124:125], v35 offset:24576\n" \
  "v_mfma_i32_16x16x64_i8 v[192:195], v[100:103], a[56:59], v[192:195]\n" \
  "ds_read_b64_tr_b8 v[126:127], v35 offset:25600\n" \
  "v_mfma_i32_16x16x64_i8 v[224:227], v[96:99], a[44:47], v[224:227]\n" \
  "v_mfma_i32_16x16x64_i8 v[224:227], v[100:103], a[60:63], v[224:227]\n" \
  "buffer_load_dwordx4 a[148:151], v91, s[28:31], s92 offen offset:1024 sc0 sc1\n" \
  "s_waitcnt lgkmcnt(8)\n" \
  "v_mfma_i32_16x16x64_i8 v[132:135], v[104:107], a[32:35], v[132:135]\n" \
  "ds_read_b64_tr_b8 v[96:97], v36 offset:16384\n" \
  "v_mfma_i32_16x16x64_i8 v[132:135], v[108:111], a[48:51], v[132:135]\n" \
  "ds_read_b64_tr_b8 v[98:99], v36 offset:17408\n" \
  "v_mfma_i32_16x16x64_i8 v[164:167], v[104:107], a[36:39], v[164:167]\n" \
  "v_mfma_i32_16x16x64_i8 v[164:167], v[108:111], a[52:55], v[164:167]\n" \
  "buffer_load_dwordx4 a[152:155], v91, s[28:31], s92 offen offset:2048 sc0 sc1\n" \
  "v_mfma_i32_16x16x64_i8 v[196:199], v[104:107], a[40:43], v[196:199]\n" \
  "ds_read_b64_tr_b8 v[100:101], v36 offset:24576\n" \
  "v_mfma_i32_16x16x64_i8 v[196:199], v[108:111], a[56:59], v[196:199]\n" \
  "ds_read_b64_tr_b8 v[102:103], v36 offset:25600\n" \
  "v_mfma_i32_16x16x64_i8 v[228:231], v[104:107], a[44:47], v[228:231]\n" \
  "v_mfma_i32_16x16x64_i8 v[228:231], v[108:111], a[60:63], v[228:231]\n" \
  "buffer_load_dwordx4 a[156:159], v91, s[28:31], s92 offen offset:3072 sc0 sc1\n" \
  "s_waitcnt lgkmcnt(8)\n" \
  "v_mfma_i32_16x16x64_i8 v[136:139], v[112:115], a[32:35], v[136:139]\n" \
  "ds_read_b64_tr_b8 v[104:105], v37 offset:16384\n" \
  "v_mfma_i32_16x16x64_i8 v[136:139], v[116:119], a[48:51], v[136:139]\n" \
  "ds_read_b64_tr_b8 v[106:107], v37 offset:17408\n" \
  "v_mfma_i32_16x16x64_i8 v[168:171], v[112:115], a[36:39], v[168:171]\n" \
  "v_mfma_i32_16x16x64_i8 v[168:171], v[116:119], a[52:55], v[168:171]\n" \
  "buffer_load_dwordx4 a[160:163], v44, s[28:31], s93 offen offset:0 sc0 sc1\n" \
  "v_mfma_i32_16x16x64_i8 v[200:203], v[112:115], a[40:43], v[200:203]\n" \
  "ds_read_b64_tr_b8 v[108:109], v37 offset:24576\n" \
  "v_mfma_i32_16x16x64_i8 v[200:203], v[116:119], a[56:59], v[200:203]\n" \
  "ds_read_b64_tr_b8 v[110:111], v37 offset:25600\n" \
  "v_mfma_i32_16x16x64_i8 v[232:235], v[112:115], a[44:47], v[232:235]\n" \
  "v_mfma_i32_16x16x64_i8 v[232:235], v[116:119], a[60:63], v[232:235]\n" \
  "buffer_load_dwordx4 a[164:167], v44, s[28:31], s93 offen offset:1024 sc0 sc1\n" \
  "s_waitcnt lgkmcnt(8)\n" \
  "v_mfma_i32_16x16x64_i8 v[140:143], v[120:123], a[32:35], v[140:143]\n" \
  "ds_read_b64_tr_b8 v[112:113], v38 offset:16384\n" \
  "v_mfma_i32_16x16x64_i8 v[140:143], v[124:127], a[48:51], v[140:143]\n" \
  "ds_read_b64_tr_b8 v[114:115], v38 offset:17408\n" \
  "v_mfma_i32_16x16x64_i8 v[172:175], v[120:123], a[36:39], v[172:175]\n" \
  "v_readlane_b32 s50, v46, s49\n" \
  "v_mfma_i32_16x16x64_i8 v[172:175], v[124:127], a[52:55], v[172:175]\n" \
  "buffer_load_dwordx4 a[168:171], v44, s[28:31], s93 offen offset:2048 sc0 sc1\n" \
  "s_waitcnt vmcnt(39)\n" \
  "s_barrier\n" \
  "v_mfma_i32_16x16x64_i8 v[204:207], v[120:123], a[40:43], v[204:207]\n" \
  "ds_read_b64_tr_b8 v[116:117], v38 offset:24576\n" \
  "v_mfma_i32_16x16x64_i8 v[204:207], v[124:127], a[56:59], v[204:207]\n" \
  "ds_read_b64_tr_b8 v[118:119], v38 offset:25600\n" \
  "v_mfma_i32_16x16x64_i8 v[236:239], v[120:123], a[44:47], v[236:239]\n" \
  "v_mfma_i32_16x16x64_i8 v[236:239], v[124:127], a[60:63], v[236:239]\n" \
  "buffer_load_dwordx4 a[172:175], v44, s[28:31], s93 offen offset:3072 sc0 sc1\n" \
  "s_waitcnt lgkmcnt(8)\n" \
  "v_mfma_i32_16x16x64_i8 v[144:147], v[96:99], a[32:35], v[144:147]\n" \
  "ds_read_b64_tr_b8 v[120:121], v39 offset:16384\n" \
  "v_mfma_i32_16x16x64_i8 v[144:147], v[100:103], a[48:51], v[144:147]\n" \
  "ds_read_b64_tr_b8 v[122:123], v39 offset:17408\n" \
  "s_add_u32 m0, s47, 81920\n" \
  "v_mfma_i32_16x16x64_i8 v[176:179], v[96:99], a[36:39], v[176:179]\n" \
  "buffer_load_dwordx4 v40, s[16:19], s85 offen nt lds\n" \
  "s_lshl_b32 s86, s50, 21\n" \
  "s_lshl_b32 s94, s50, 16\n" \
  "s_add_u32 s49, s49, 1\n" \
  "v_mfma_i32_16x16x64_i8 v[176:179], v[100:103], a[52:55], v[176:179]\n" \
  "v_mfma_i32_16x16x64_i8 v[208:211], v[96:99], a[40:43], v[208:211]\n" \
  "ds_read_b64_tr_b8 v[124:125], v39 offset:24576\n" \
  "v_mfma_i32_16x16x64_i8 v[208:211], v[100:103], a[56:59], v[208:211]\n" \
  "ds_read_b64_tr_b8 v[126:127], v39 offset:25600\n" \
  "s_add_u32 m0, s47, 82944\n" \
  "v_mfma_i32_16x16x64_i8 v[240:243], v[96:99], a[44:47], v[240:243]\n" \
  "buffer_load_dwordx4 v41, s[16:19], s85 offen nt lds\n" \
  "v_readlane_b32 s50, v46, s49\n" \
  "v_mfma_i32_16x16x64_i8 v[240:243], v[100:103], a[60:63], v[240:243]\n" \
  "s_waitcnt lgkmcnt(8)\n" \
  "v_mfma_i32_16x16x64_i8 v[148:151], v[104:107], a[32:35], v[148:151]\n" \
  "ds_read_b64_tr_b8 v[96:97], v32 offset:32768\n" \
  "v_mfma_i32_16x16x64_i8 v[148:151], v[108:111], a[48:51], v[148:151]\n" \
  "ds_read_b64_tr_b8 v[98:99], v32 offset:33792\n" \
  "s_add_u32 m0, s47, 83968\n" \
  "v_mfma_i32_16x16x64_i8 v[180:183], v[104:107], a[36:39], v[180:183]\n" \
  "buffer_load_dwordx4 v42, s[16:19], s85 offen nt lds\n" \
  "v_mfma_i32_16x16x64_i8 v[180:183], v[108:111], a[52:55], v[180:183]\n" \
  "v_mfma_i32_16x16x64_i8 v[212:215], v[104:107], a[40:43], v[212:215]\n" \
  "ds_read_b64_tr_b8 v[100:101], v32 offset:40960\n" \
  "v_mfma_i32_16x16x64_i8 v[212:215], v[108:111], a[56:59], v[212:215]\n" \
  "ds_read_b64_tr_b8 v[102:103], v32 offset:41984\n" \
  "s_add_u32 m0, s47, 84992\n" \
  "v_mfma_i32_16x16x64_i8 v[244:247], v[104:107], a[44:47], v[244:247]\n" \
  "buffer_load_dwordx4 v43, s[16:19], s85 offen nt lds\n" \
  "s_lshl_b32 s87, s50, 21\n" \
  "s_lshl_b32 s95, s50, 16\n" \
  "s_add_u32 s49, s49, 1\n" \
  "v_mfma_i32_16x16x64_i8 v[244:247], v[108:111], a[60:63], v[244:247]\n" \
  "s_waitcnt lgkmcnt(8)\n" \
  "v_mfma_i32_16x16x64_i8 v[152:155], v[112:115], a[32:35], v[152:155]\n" \
  "ds_read_b64_tr_b8 v[104:105], v33 offset:32768\n" \
  "v_mfma_i32_16x16x64_i8 v[152:155], v[116:119], a[48:51], v[152:155]\n" \
  "ds_read_b64_tr_b8 v[106:107], v33 offset:33792\n" \
  "s_add_u32 m0, s47, 98304\n" \
  "v_mfma_i32_16x16x64_i8 v[184:187], v[112:115], a[36:39], v[184:187]\n" \
  "buffer_load_dwordx4 v40, s[16:19], s86 offen nt lds\n" \
  "v_readlane_b32 s50, v46, s49\n" \
  "v_mfma_i32_16x16x64_i8 v[184:187], v[116:119], a[52:55], v[184:187]\n" \
  "v_mfma_i32_16x16x64_i8 v[216:219], v[112:115], a[40:43], v[216:219]\n" \
  "ds_read_b64_tr_b8 v[108:109], v33 offset:40960\n" \
  "v_mfma_i32_16x16x64_i8 v[216:219], v[116:119], a[56:59], v[216:219]\n" \
  "ds_read_b64_tr_b8 v[110:111], v33 offset:41984\n" \
  "s_add_u32 m0, s47, 99328\n" \
  "v_mfma_i32_16x16x64_i8 v[248:251], v[112:115], a[44:47], v[248:251]\n" \
  "buffer_load_dwordx4 v41, s[16:19], s86 offen nt lds\n" \
  "v_mfma_i32_16x16x64_i8 v[248:251], v[116:119], a[60:63], v[248:251]\n" \
  "s_waitcnt lgkmcnt(8)\n" \
  "v_mfma_i32_16x16x64_i8 v[156:159], v[120:123], a[32:35], v[156:159]\n" \
  "ds_read_b64_tr_b8 v[112:113], v34 offset:32768\n" \
  "v_mfma_i32_16x16x64_i8 v[156:159], v[124:127], a[48:51], v[156:159]\n" \
  "ds_read_b64_tr_b8 v[114:115], v34 offset:33792\n" \
  "s_add_u32 m0, s47, 100352\n" \
  "v_mfma_i32_16x16x64_i8 v[188:191], v[120:123], a[36:39], v[188:191]\n" \
  "buffer_load_dwordx4 v42, s[16:19], s86 offen nt lds\n" \
  "s_lshl_b32 s80, s50, 21\n" \
  "s_lshl_b32 s88, s50, 16\n" \
  "s_add_u32 s49, s49, 1\n" \
  "v_mfma_i32_16x16x64_i8 v[188:191], v[124:127], a[52:55], v[188:191]\n" \
  "v_mfma_i32_16x16x64_i8 v[220:223], v[120:123], a[40:43], v[220:223]\n" \
  "ds_read_b64_tr_b8 v[116:117], v34 offset:40960\n" \
  "v_mfma_i32_16x16x64_i8 v[220:223], v[124:127], a[56:59], v[220:223]\n" \
  "ds_read_b64_tr_b8 v[118:119], v34 offset:41984\n" \
  "s_add_u32 m0, s47, 101376\n" \
  "v_mfma_i32_16x16x64_i8 v[252:255], v[120:123], a[44:47], v[252:255]\n" \
  "buffer_load_dwordx4 v43, s[16:19], s86 offen nt lds\n" \
  "v_mfma_i32_16x16x64_i8 v[252:255], v[124:127], a[60:63], v[252:255]\n" \
  "; ---- step 2\n" \
  "s_cmp_lg_u32 s11, 0\n" \
  "s_cbranch_scc1 Lskip32B_%=\n" \
  "global_load_dword v72, v50, s[38:39] sc0 sc1\n" \
  "Lskip32B_%=:\n" \
  "s_cmp_lg_u32 s46, s54\n" \
  "s_cbranch_scc1 Lskip31B_%=\n" \
  "s_waitcnt vmcnt(63)\n" \
  "global_store_dwordx4 v49, v[64:67], s[34:35] sc0 sc1\n" \
  "s_nop 1\n" \
  "global_store_dwordx4 v79, v[68:71], s[34:35] sc0 sc1\n" \
  "s_nop 1\n" \
  "Lskip31B_%=:\n" \
  "s_waitcnt vmcnt(28)\n" \
  "s_waitcnt lgkmcnt(8)\n" \
  "v_mfma_i32_16x16x64_i8 v[128:131], v[96:99], a[64:67], v[128:131]\n" \
  "ds_read_b64_tr_b8 v[120:121], v35 offset:32768\n" \
  "s_add_u32 m0, s47, 114688\n" \
  "v_mfma_i32_16x16x64_i8 v[128:131], v[100:103], a[80:83], v[128:131]\n" \
  "ds_read_b64_tr_b8 v[122:123], v35 offset:33792\n" \
  "buffer_load_dwordx4 v40, s[16:19], s87 offen lds\n" \
  "s_add_u32 m0, s47, 115712\n" \
  "v_mfma_i32_16x16x64_i8 v[160:163], v[96:99], a[68:71], v[160:163]\n" \
  "buffer_load_dwordx4 v41, s[16:19], s87 offen lds\n" \
  "s_add_u32 m0, s47, 116736\n" \
  "v_mfma_i32_16x16x64_i8 v[160:163], v[100:103], a[84:87], v[160:163]\n" \
  "buffer_load_dwordx4 v42, s[16:19], s87 offen lds\n" \
  "s_add_u32 m0, s47, 117760\n" \
  "v_mfma_i32_16x16x64_i8 v[192:195], v[96:99], a[72:75], v[192:195]\n" \
  "ds_read_b64_tr_b8 v[124:125], v35 offset:40960\n" \
  "buffer_load_dwordx4 v43, s[16:19], s87 offen lds\n" \
  "s_add_u32 m0, s47, 0\n" \
  "v_mfma_i32_16x16x64_i8 v[192:195], v[100:103], a[88:91], v[192:195]\n" \
  "ds_read_b64_tr_b8 v[126:127], v35 offset:41984\n" \
  "buffer_load_dwordx4 v40, s[16:19], s80 offen lds\n" \
  "s_add_u32 m0, s47, 1024\n" \
  "v_mfma_i32_16x16x64_i8 v[224:227], v[96:99], a[76:79], v[224:227]\n" \
  "buffer_load_dwordx4 v41, s[16:19], s80 offen lds\n" \
  "v_mfma_i32_16x16x64_i8 v[224:227], v[100:103], a[92:95], v[224:227]\n" \
  "s_waitcnt lgkmcnt(8)\n" \
  "v_mfma_i32_16x16x64_i8 v[132:135], v[104:107], a[64:67], v[132:135]\n" \
  "ds_read_b64_tr_b8 v[96:97], v36 offset:32768\n" \
  "v_mfma_i32_16x16x64_i8 v[132:135], v[108:111], a[80:83], v[132:135]\n" \
  "ds_read_b64_tr_b8 v[98:99], v36 offset:33792\n" \
  "v_mfma_i32_16x16x64_i8 v[164:167], v[104:107], a[68:71], v[164:167]\n" \
  "v_mfma_i32_16x16x64_i8 v[164:167], v[108:111], a[84:87], v[164:167]\n" \
  "v_mfma_i32_16x16x64_i8 v[196:199], v[104:107], a[72:75], v[196:199]\n" \
  "ds_read_b64_tr_b8 v[100:101], v36 offset:40960\n" \
  "v_mfma_i32_16x16x64_i8 v[196:199], v[108:111], a[88:91], v[196:199]\n" \
  "ds_read_b64_tr_b8 v[102:103], v36 offset:41984\n" \
  "v_mfma_i32_16x16x64_i8 v[228:231], v[104:107], a[76:79], v[228:231]\n" \
  "v_mfma_i32_16x16x64_i8 v[228:231], v[108:111], a[92:95], v[228:231]\n" \
  "s_waitcnt lgkmcnt(8)\n" \
  "v_mfma_i32_16x16x64_i8 v[136:139], v[112:115], a[64:67], v[136:139]\n" \
  "ds_read_b64_tr_b8 v[104:105], v37 offset:32768\n" \
  "v_mfma_i32_16x16x64_i8 v[136:139], v[116:119], a[80:83], v[136:139]\n" \
  "ds_read_b64_tr_b8 v[106:107], v37 offset:33792\n" \
  "v_mfma_i32_16x16x64_i8 v[168:171], v[112:115], a[68:71], v[168:171]\n" \
  "v_mfma_i32_16x16x64_i8 v[168:171], v[116:119], a[84:87], v[168:171]\n" \
  "v_mfma_i32_16x16x64_i8 v[200:203], v[112:115], a[72:75], v[200:203]\n" \
  "ds_read_b64_tr_b8 v[108:109], v37 offset:40960\n" \
  "v_mfma_i32_16x16x64_i8 v[200:203], v[116:119], a[88:91], v[200:203]\n" \
  "ds_read_b64_tr_b8 v[110:111], v37 offset:41984\n" \
  "v_mfma_i32_16x16x64_i8 v[232:235], v[112:115], a[76:79], v[232:235]\n" \
  "v_mfma_i32_16x16x64_i8 v[232:235], v[116:119], a[92:95], v[232:235]\n" \
  "s_waitcnt lgkmcnt(8)\n" \
  "v_mfma_i32_16x16x64_i8 v[140:143], v[120:123], a[64:67], v[140:143]\n" \
  "ds_read_b64_tr_b8 v[112:113], v38 offset:32768\n" \
  "v_mfma_i32_16x16x64_i8 v[140:143], v[124:127], a[80:83], v[140:143]\n" \
  "ds_read_b64_tr_b8 v[114:115], v38 offset:33792\n" \
  "v_mfma_i32_16x16x64_i8 v[172:175], v[120:123], a[68:71], v[172:175]\n" \
  "v_mfma_i32_16x16x64_i8 v[172:175], v[124:127], a[84:87], v[172:175]\n" \
  "s_waitcnt vmcnt(50)\n" \
  "s_barrier\n" \
  "v_mfma_i32_16x16x64_i8 v[204:207], v[120:123], a[72:75], v[204:207]\n" \
  "ds_read_b64_tr_b8 v[116:117], v38 offset:40960\n" \
  "v_mfma_i32_16x16x64_i8 v[204:207], v[124:127], a[88:91], v[204:207]\n" \
  "ds_read_b64_tr_b8 v[118:119], v38 offset:41984\n" \
  "v_mfma_i32_16x16x64_i8 v[236:239], v[120:123], a[76:79], v[236:239]\n" \
  "v_mfma_i32_16x16x64_i8 v[236:239], v[124:127], a[92:95], v[236:239]\n" \
  "s_waitcnt lgkmcnt(8)\n" \
  "v_mfma_i32_16x16x64_i8 v[144:147], v[96:99], a[64:67], v[144:147]\n" \
  "ds_read_b64_tr_b8 v[120:121], v39 offset:32768\n" \
  "v_mfma_i32_16x16x64_i8 v[144:147], v[100:103], a[80:83], v[144:147]\n" \
  "ds_read_b64_tr_b8 v[122:123], v39 offset:33792\n" \
  "v_mfma_i32_16x16x64_i8 v[176:179], v[96:99], a[68:71], v[176:179]\n" \
  "v_mfma_i32_16x16x64_i8 v[176:179], v[100:103], a[84:87], v[176:179]\n" \
  "v_mfma_i32_16x16x64_i8 v[208:211], v[96:99], a[72:75], v[208:211]\n" \
  "ds_read_b64_tr_b8 v[124:125], v39 offset:40960\n" \
  "v_mfma_i32_16x16x64_i8 v[208:211], v[100:103], a[88:91], v[208:211]\n" \
  "ds_read_b64_tr_b8 v[126:127], v39 offset:41984\n" \
  "v_mfma_i32_16x16x64_i8 v[240:243], v[96:99], a[76:79], v[240:243]\n" \
  "v_mfma_i32_16x16x64_i8 v[240:243], v[100:103], a[92:95], v[240:243]\n" \
  "s_waitcnt lgkmcnt(8)\n" \
  "v_mfma_i32_16x16x64_i8 v[148:151], v[104:107], a[64:67], v[148:151]\n" \
  "ds_read_b64_tr_b8 v[96:97], v32 offset:49152\n" \
  "s_add_u32 m0, s47, 2048\n" \
  "v_mfma_i32_16x16x64_i8 v[148:151], v[108:111], a[80:83], v[148:151]\n" \
  "ds_read_b64_tr_b8 v[98:99], v32 offset:50176\n" \
  "buffer_load_dwordx4 v42, s[16:19], s80 offen lds\n" \
  "s_add_u32 m0, s47, 3072\n" \
  "v_mfma_i32_16x16x64_i8 v[180:183], v[104:107], a[68:71], v[180:183]\n" \
  "buffer_load_dwordx4 v43, s[16:19], s80 offen lds\n" \
  "v_mfma_i32_16x16x64_i8 v[180:183], v[108:111], a[84:87], v[180:183]\n" \
  "v_mfma_i32_16x16x64_i8 v[212:215], v[104:107], a[72:75], v[212:215]\n" \
  "ds_read_b64_tr_b8 v[100:101], v32 offset:57344\n" \
  "v_mfma_i32_16x16x64_i8 v[212:215], v[108:111], a[88:91], v[212:215]\n" \
  "ds_read_b64_tr_b8 v[102:103], v32 offset:58368\n" \
  "v_mfma_i32_16x16x64_i8 v[244:247], v[104:107], a[76:79], v[244:247]\n" \
  "v_mfma_i32_16x16x64_i8 v[244:247], v[108:111], a[92:95], v[244:247]\n" \
  "s_waitcnt lgkmcnt(8)\n" \
  "v_mfma_i32_16x16x64_i8 v[152:155], v[112:115], a[64:67], v[152:155]\n" \
  "ds_read_b64_tr_b8 v[104:105], v33 offset:49152\n" \
  "v_mfma_i32_16x16x64_i8 v[152:155], v[116:119], a[80:83], v[152:155]\n" \
  "ds_read_b64_tr_b8 v[106:107], v33 offset:50176\n" \
  "v_mfma_i32_16x16x64_i8 v[184:187], v[112:115], a[68:71], v[184:187]\n" \
  "v_mfma_i32_16x16x64_i8 v[184:187], v[116:119], a[84:87], v[184:187]\n" \
  "v_mfma_i32_16x16x64_i8 v[216:219], v[112:115], a[72:75], v[216:219]\n" \
  "ds_read_b64_tr_b8 v[108:109], v33 offset:57344\n" \
  "v_mfma_i32_16x16x64_i8 v[216:219], v[116:119], a[88:91], v[216:219]\n" \
  "ds_read_b64_tr_b8 v[110:111], v33 offset:58368\n" \
  "v_mfma_i32_16x16x64_i8 v[248:251], v[112:115], a[76:79], v[248:251]\n" \
  "v_mfma_i32_16x16x64_i8 v[248:251], v[116:119], a[92:95], v[248:251]\n" \
  "s_waitcnt lgkmcnt(8)\n" \
  "v_mfma_i32_16x16x64_i8 v[156:159], v[120:123], a[64:67], v[156:159]\n" \
  "ds_read_b64_tr_b8 v[112:113], v34 offset:49152\n" \
  "v_mfma_i32_16x16x64_i8 v[156:159], v[124:127], a[80:83], v[156:159]\n" \
  "ds_read_b64_tr_b8 v[114:115], v34 offset:50176\n" \
  "v_mfma_i32_16x16x64_i8 v[188:191], v[120:123], a[68:71], v[188:191]\n" \
  "v_mfma_i32_16x16x64_i8 v[188:191], v[124:127], a[84:87], v[188:191]\n" \
  "v_mfma_i32_16x16x64_i8 v[220:223], v[120:123], a[72:75], v[220:223]\n" \
  "ds_read_b64_tr_b8 v[116:117], v34 offset:57344\n" \
  "v_mfma_i32_16x16x64_i8 v[220:223], v[124:127], a[88:91], v[220:223]\n" \
  "ds_read_b64_tr_b8 v[118:119], v34 offset:58368\n" \
  "v_mfma_i32_16x16x64_i8 v[252:255], v[120:123], a[76:79], v[252:255]\n" \
  "v_mfma_i32_16x16x64_i8 v[252:255], v[124:127], a[92:95], v[252:255]\n" \
  "; ---- step 3\n" \
  "s_waitcnt vmcnt(28)\n" \
  "s_waitcnt lgkmcnt(8)\n" \
  "v_mfma_i32_16x16x64_i8 v[128:131], v[96:99], a[96:99], v[128:131]\n" \
  "ds_read_b64_tr_b8 v[120:121], v35 offset:49152\n" \
  "v_mfma_i32_16x16x64_i8 v[128:131], v[100:103], a[112:115], v[128:131]\n" \
  "ds_read_b64_tr_b8 v[122:123], v35 offset:50176\n" \
  "v_mfma_i32_16x16x64_i8 v[160:163], v[96:99], a[100:103], v[160:163]\n" \
  "v_mfma_i32_16x16x64_i8 v[160:163], v[100:103], a[116:119], v[160:163]\n" \
  "v_mfma_i32_16x16x64_i8 v[192:195], v[96:99], a[104:107], v[192:195]\n" \
  "ds_read_b64_tr_b8 v[124:125], v35 offset:57344\n" \
  "v_mfma_i32_16x16x64_i8 v[192:195], v[100:103], a[120:123], v[192:195]\n" \
  "ds_read_b64_tr_b8 v[126:127], v35 offset:58368\n" \
  "v_mfma_i32_16x16x64_i8 v[224:227], v[96:99], a[108:111], v[224:227]\n" \
  "v_mfma_i32_16x16x64_i8 v[224:227], v[100:103], a[124:127], v[224:227]\n" \
  "s_waitcnt lgkmcnt(8)\n" \
  "v_mfma_i32_16x16x64_i8 v[132:135], v[104:107], a[96:99], v[132:135]\n" \
  "ds_read_b64_tr_b8 v[96:97], v36 offset:49152\n" \
  "v_mfma_i32_16x16x64_i8 v[132:135], v[108:111], a[112:115], v[132:135]\n" \
  "ds_read_b64_tr_b8 v[98:99], v36 offset:50176\n" \
  "v_mfma_i32_16x16x64_i8 v[164:167], v[104:107], a[100:103], v[164:167]\n" \
  "v_mfma_i32_16x16x64_i8 v[164:167], v[108:111], a[116:119], v[164:167]\n" \
  "v_mfma_i32_16x16x64_i8 v[196:199], v[104:107], a[104:107], v[196:199]\n" \
  "ds_read_b64_tr_b8 v[100:101], v36 offset:57344\n" \
  "v_mfma_i32_16x16x64_i8 v[196:199], v[108:111], a[120:123], v[196:199]\n" \
  "ds_read_b64_tr_b8 v[102:103], v36 offset:58368\n" \
  "v_mfma_i32_16x16x64_i8 v[228:231], v[104:107], a[108:111], v[228:231]\n" \
  "v_mfma_i32_16x16x64_i8 v[228:231], v[108:111], a[124:127], v[228:231]\n" \
  "s_waitcnt lgkmcnt(8)\n" \
  "v_mfma_i32_16x16x64_i8 v[136:139], v[112:115], a[96:99], v[136:139]\n" \
  "ds_read_b64_tr_b8 v[104:105], v37 offset:49152\n" \
  "v_mfma_i32_16x16x64_i8 v[136:139], v[116:119], a[112:115], v[136:139]\n" \
  "ds_read_b64_tr_b8 v[106:107], v37 offset:50176\n" \
  "v_mfma_i32_16x16x64_i8 v[168:171], v[112:115], a[100:103], v[168:171]\n" \
  "v_mfma_i32_16x16x64_i8 v[168:171], v[116:119], a[116:119], v[168:171]\n" \
  "v_mfma_i32_16x16x64_i8 v[200:203], v[112:115], a[104:107], v[200:203]\n" \
  "ds_read_b64_tr_b8 v[108:109], v37 offset:57344\n" \
  "v_mfma_i32_16x16x64_i8 v[200:203], v[116:119], a[120:123], v[200:203]\n" \
  "ds_read_b64_tr_b8 v[110:111], v37 offset:58368\n" \
  "buffer_load_dwordx4 a[176:179], v91, s[28:31], s93 offen offset:0 sc0 sc1\n" \
  "v_mfma_i32_16x16x64_i8 v[232:235], v[112:115], a[108:111], v[232:235]\n" \
  "v_mfma_i32_16x16x64_i8 v[232:235], v[116:119], a[124:127], v[232:235]\n" \
  "buffer_load_dwordx4 a[180:183], v91, s[28:31], s93 offen offset:1024 sc0 sc1\n" \
  "s_waitcnt lgkmcnt(8)\n" \
  "v_mfma_i32_16x16x64_i8 v[140:143], v[120:123], a[96:99], v[140:143]\n" \
  "ds_read_b64_tr_b8 v[112:113], v38 offset:49152\n" \
  "v_mfma_i32_16x16x64_i8 v[140:143], v[124:127], a[112:115], v[140:143]\n" \
  "ds_read_b64_tr_b8 v[114:115], v38 offset:50176\n" \
  "v_mfma_i32_16x16x64_i8 v[172:175], v[120:123], a[100:103], v[172:175]\n" \
  "v_mfma_i32_16x16x64_i8 v[172:175], v[124:127], a[116:119], v[172:175]\n" \
  "buffer_load_dwordx4 a[184:187], v91, s[28:31], s93 offen offset:2048 sc0 sc1\n" \
  "s_cmp_lg_u32 s46, s54\n" \
  "s_cbranch_scc1 Lskip33B_%=\n" \
  "s_waitcnt vmcnt(11)\n" \
  "Lskip33B_%=:\n" \
  "s_waitcnt vmcnt(51)\n" \
  "s_barrier\n" \
  "s_cmp_lg_u32 s46, s54\n" \
  "s_cbranch_scc1 Lskip35B_%=\n" \
  "s_cmp_lg_u32 s11, 0\n" \
  "s_cbranch_scc1 Lskip34B_%=\n" \
  "s_mov_b64 exec, 1\n" \
  "global_store_dword v53, v76, s[36:37] sc0 sc1\n" \
  "s_mov_b64 exec, -1\n" \
  "Lskip34B_%=:\n" \
  "Lskip35B_%=:\n" \
  "v_mfma_i32_16x16x64_i8 v[204:207], v[120:123], a[104:107], v[204:207]\n" \
  "ds_read_b64_tr_b8 v[116:117], v38 offset:57344\n" \
  "v_mfma_i32_16x16x64_i8 v[204:207], v[124:127], a[120:123], v[204:207]\n" \
  "ds_read_b64_tr_b8 v[118:119], v38 offset:58368\n" \
  "v_mfma_i32_16x16x64_i8 v[236:239], v[120:123], a[108:111], v[236:239]\n" \
  "v_mfma_i32_16x16x64_i8 v[236:239], v[124:127], a[124:127], v[236:239]\n" \
  "buffer_load_dwordx4 a[188:191], v91, s[28:31], s93 offen offset:3072 sc0 sc1\n" \
  "s_waitcnt lgkmcnt(8)\n" \
  "v_mfma_i32_16x16x64_i8 v[144:147], v[96:99], a[96:99], v[144:147]\n" \
  "ds_read_b64_tr_b8 v[120:121], v39 offset:49152\n" \
  "v_mfma_i32_16x16x64_i8 v[144:147], v[100:103], a[112:115], v[144:147]\n" \
  "ds_read_b64_tr_b8 v[122:123], v39 offset:50176\n" \
  "v_mfma_i32_16x16x64_i8 v[176:179], v[96:99], a[100:103], v[176:179]\n" \
  "v_mfma_i32_16x16x64_i8 v[176:179], v[100:103], a[116:119], v[176:179]\n" \
  "buffer_load_dwordx4 a[192:195], v44, s[28:31], s94 offen offset:0 sc0 sc1\n" \
  "v_mfma_i32_16x16x64_i8 v[208:211], v[96:99], a[104:107], v[208:211]\n" \
  "ds_read_b64_tr_b8 v[124:125], v39 offset:57344\n" \
  "v_mfma_i32_16x16x64_i8 v[208:211], v[100:103], a[120:123], v[208:211]\n" \
  "ds_read_b64_tr_b8 v[126:127], v39 offset:58368\n" \
  "v_mfma_i32_16x16x64_i8 v[240:243], v[96:99], a[108:111], v[240:243]\n" \
  "v_mfma_i32_16x16x64_i8 v[240:243], v[100:103], a[124:127], v[240:243]\n" \
  "buffer_load_dwordx4 a[196:199], v44, s[28:31], s94 offen offset:1024 sc0 sc1\n" \
  "s_waitcnt lgkmcnt(8)\n" \
  "v_mfma_i32_16x16x64_i8 v[148:151], v[104:107], a[96:99], v[148:151]\n" \
  "ds_read_b64_tr_b8 v[96:97], v80 offset:0\n" \
  "v_mfma_i32_16x16x64_i8 v[148:151], v[108:111], a[112:115], v[148:151]\n" \
  "ds_read_b64_tr_b8 v[98:99], v80 offset:1024\n" \
  "v_mfma_i32_16x16x64_i8 v[180:183], v[104:107], a[100:103], v[180:183]\n" \
  "v_mfma_i32_16x16x64_i8 v[180:183], v[108:111], a[116:119], v[180:183]\n" \
  "buffer_load_dwordx4 a[200:203], v44, s[28:31], s94 offen offset:2048 sc0 sc1\n" \
  "v_mfma_i32_16x16x64_i8 v[212:215], v[104:107], a[104:107], v[212:215]\n" \
  "ds_read_b64_tr_b8 v[100:101], v80 offset:8192\n" \
  "v_mfma_i32_16x16x64_i8 v[212:215], v[108:111], a[120:123], v[212:215]\n" \
  "ds_read_b64_tr_b8 v[102:103], v80 offset:9216\n" \
  "buffer_load_dwordx4 a[204:207], v44, s[28:31], s94 offen offset:3072 sc0 sc1\n" \
  "v_mfma_i32_16x16x64_i8 v[244:247], v[104:107], a[108:111], v[244:247]\n" \
  "v_mfma_i32_16x16x64_i8 v[244:247], v[108:111], a[124:127], v[244:247]\n" \
  "s_waitcnt lgkmcnt(8)\n" \
  "v_mfma_i32_16x16x64_i8 v[152:155], v[112:115], a[96:99], v[152:155]\n" \
  "ds_read_b64_tr_b8 v[104:105], v81 offset:0\n" \
  "v_mfma_i32_16x16x64_i8 v[152:155], v[116:119], a[112:115], v[152:155]\n" \
  "ds_read_b64_tr_b8 v[106:107], v81 offset:1024\n" \
  "buffer_load_dwordx4 a[208:211], v91, s[28:31], s94 offen offset:0 sc0 sc1\n" \
  "v_mfma_i32_16x16x64_i8 v[184:187], v[112:115], a[100:103], v[184:187]\n" \
  "v_mfma_i32_16x16x64_i8 v[184:187], v[116:119], a[116:119], v[184:187]\n" \
  "s_cmp_lg_u32 s11, 0\n" \
  "s_cbranch_scc1 Lskip38B_%=\n" \
  "s_waitcnt vmcnt(17)\n" \
  "v_sub_u32 v72, v72, v76\n" \
  "v_cmp_gt_i32 vcc, 0, v72\n" \
  "s_nop 1\n" \
  "s_andn2_b64 vcc, vcc, s[14:15]\n" \
  "s_cbranch_scc1 Lslow36B_%=\n" \
  "Lback37B_%=:\n" \
  "v_add_u32 v50, 1024, v50\n" \
  "Lskip38B_%=:\n" \
  "v_mfma_i32_16x16x64_i8 v[216:219], v[112:115], a[104:107], v[216:219]\n" \
  "ds_read_b64_tr_b8 v[108:109], v81 offset:8192\n" \
  "v_mfma_i32_16x16x64_i8 v[216:219], v[116:119], a[120:123], v[216:219]\n" \
  "ds_read_b64_tr_b8 v[110:111], v81 offset:9216\n" \
  "buffer_load_dwordx4 a[212:215], v91, s[28:31], s94 offen offset:1024 sc0 sc1\n" \
  "v_mfma_i32_16x16x64_i8 v[248:251], v[112:115], a[108:111], v[248:251]\n" \
  "v_mfma_i32_16x16x64_i8 v[248:251], v[116:119], a[124:127], v[248:251]\n" \
  "s_waitcnt lgkmcnt(8)\n" \
  "v_mfma_i32_16x16x64_i8 v[156:159], v[120:123], a[96:99], v[156:159]\n" \
  "ds_read_b64_tr_b8 v[112:113], v82 offset:0\n" \
  "v_mfma_i32_16x16x64_i8 v[156:159], v[124:127], a[112:115], v[156:159]\n" \
  "ds_read_b64_tr_b8 v[114:115], v82 offset:1024\n" \
  "buffer_load_dwordx4 a[216:219], v91, s[28:31], s94 offen offset:2048 sc0 sc1\n" \
  "v_mfma_i32_16x16x64_i8 v[188:191], v[120:123], a[100:103], v[188:191]\n" \
  "v_mfma_i32_16x16x64_i8 v[188:191], v[124:127], a[116:119], v[188:191]\n" \
  "v_mfma_i32_16x16x64_i8 v[220:223], v[120:123], a[104:107], v[220:223]\n" \
  "ds_read_b64_tr_b8 v[116:117], v82 offset:8192\n" \
  "v_mfma_i32_16x16x64_i8 v[220:223], v[124:127], a[120:123], v[220:223]\n" \
  "ds_read_b64_tr_b8 v[118:119], v82 offset:9216\n" \
  "buffer_load_dwordx4 a[220:223], v91, s[28:31], s94 offen offset:3072 sc0 sc1\n" \
  "v_mfma_i32_16x16x64_i8 v[252:255], v[120:123], a[108:111], v[252:255]\n" \
  "v_mfma_i32_16x16x64_i8 v[252:255], v[124:127], a[124:127], v[252:255]\n" \
  "s_barrier\n" \
  "buffer_load_dword v92, v52, s[28:31], 0 offen sc0 sc1\n" \
  "v_add_u32 v52, 0x10000, v52\n" \
  "; ---- step 4\n" \
  "s_waitcnt vmcnt(33)\n" \
  "s_waitcnt lgkmcnt(8)\n" \
  "v_mfma_i32_16x16x64_i8 v[128:131], v[96:99], a[128:131], v[128:131]\n" \
  "ds_read_b64_tr_b8 v[120:121], v83 offset:0\n" \
  "v_mfma_i32_16x16x64_i8 v[128:131], v[100:103], a[144:147], v[128:131]\n" \
  "ds_read_b64_tr_b8 v[122:123], v83 offset:1024\n" \
  "v_mfma_i32_16x16x64_i8 v[160:163], v[96:99], a[132:135], v[160:163]\n" \
  "v_mfma_i32_16x16x64_i8 v[160:163], v[100:103], a[148:151], v[160:163]\n" \
  "buffer_load_dwordx4 a[224:227], v44, s[28:31], s95 offen offset:0 sc0 sc1\n" \
  "v_mfma_i32_16x16x64_i8 v[192:195], v[96:99], a[136:139], v[192:195]\n" \
  "ds_read_b64_tr_b8 v[124:125], v83 offset:8192\n" \
  "v_mfma_i32_16x16x64_i8 v[192:195], v[100:103], a[152:155], v[192:195]\n" \
  "ds_read_b64_tr_b8 v[126:127], v83 offset:9216\n" \
  "v_mfma_i32_16x16x64_i8 v[224:227], v[96:99], a[140:143], v[224:227]\n" \
  "v_mfma_i32_16x16x64_i8 v[224:227], v[100:103], a[156:159], v[224:227]\n" \
  "buffer_load_dwordx4 a[228:231], v44, s[28:31], s95 offen offset:1024 sc0 sc1\n" \
  "s_waitcnt lgkmcnt(8)\n" \
  "v_mfma_i32_16x16x64_i8 v[132:135], v[104:107], a[128:131], v[132:135]\n" \
  "ds_read_b64_tr_b8 v[96:97], v84 offset:0\n" \
  "v_mfma_i32_16x16x64_i8 v[132:135], v[108:111], a[144:147], v[132:135]\n" \
  "ds_read_b64_tr_b8 v[98:99], v84 offset:1024\n" \
  "v_mfma_i32_16x16x64_i8 v[164:167], v[104:107], a[132:135], v[164:167]\n" \
  "v_mfma_i32_16x16x64_i8 v[164:167], v[108:111], a[148:151], v[164:167]\n" \
  "buffer_load_dwordx4 a[232:235], v44, s[28:31], s95 offen offset:2048 sc0 sc1\n" \
  "v_mfma_i32_16x16x64_i8 v[196:199], v[104:107], a[136:139], v[196:199]\n" \
  "ds_read_b64_tr_b8 v[100:101], v84 offset:8192\n" \
  "v_mfma_i32_16x16x64_i8 v[196:199], v[108:111], a[152:155], v[196:199]\n" \
  "ds_read_b64_tr_b8 v[102:103], v84 offset:9216\n" \
  "v_mfma_i32_16x16x64_i8 v[228:231], v[104:107], a[140:143], v[228:231]\n" \
  "v_mfma_i32_16x16x64_i8 v[228:231], v[108:111], a[156:159], v[228:231]\n" \
  "s_waitcnt lgkmcnt(8)\n" \
  "v_mfma_i32_16x16x64_i8 v[136:139], v[112:115], a[128:131], v[136:139]\n" \
  "ds_read_b64_tr_b8 v[104:105], v85 offset:0\n" \
  "v_mfma_i32_16x16x64_i8 v[136:139], v[116:119], a[144:147], v[136:139]\n" \
  "ds_read_b64_tr_b8 v[106:107], v85 offset:1024\n" \
  "v_mfma_i32_16x16x64_i8 v[168:171], v[112:115], a[132:135], v[168:171]\n" \
  "v_mfma_i32_16x16x64_i8 v[168:171], v[116:119], a[148:151], v[168:171]\n" \
  "buffer_load_dwordx4 a[236:239], v44, s[28:31], s95 offen offset:3072 sc0 sc1\n" \
  "v_mfma_i32_16x16x64_i8 v[200:203], v[112:115], a[136:139], v[200:203]\n" \
  "ds_read_b64_tr_b8 v[108:109], v85 offset:8192\n" \
  "v_mfma_i32_16x16x64_i8 v[200:203], v[116:119], a[152:155], v[200:203]\n" \
  "ds_read_b64_tr_b8 v[110:111], v85 offset:9216\n" \
  "v_mfma_i32_16x16x64_i8 v[232:235], v[112:115], a[140:143], v[232:235]\n" \
  "v_mfma_i32_16x16x64_i8 v[232:235], v[116:119], a[156:159], v[232:235]\n" \
  "buffer_load_dwordx4 a[240:243], v91, s[28:31], s95 offen offset:0 sc0 sc1\n" \
  "s_waitcnt lgkmcnt(8)\n" \
  "v_mfma_i32_16x16x64_i8 v[140:143], v[120:123], a[128:131], v[140:143]\n" \
  "ds_read_b64_tr_b8 v[112:113], v86 offset:0\n" \
  "v_mfma_i32_16x16x64_i8 v[140:143], v[124:127], a[144:147], v[140:143]\n" \
  "ds_read_b64_tr_b8 v[114:115], v86 offset:1024\n" \
  "v_mfma_i32_16x16x64_i8 v[172:175], v[120:123], a[132:135], v[172:175]\n" \
  "v_mfma_i32_16x16x64_i8 v[172:175], v[124:127], a[148:151], v[172:175]\n" \
  "buffer_load_dwordx4 a[244:247], v91, s[28:31], s95 offen offset:1024 sc0 sc1\n" \
  "s_waitcnt vmcnt(31)\n" \
  "s_barrier\n" \
  "v_mfma_i32_16x16x64_i8 v[204:207], v[120:123], a[136:139], v[204:207]\n" \
  "ds_read_b64_tr_b8 v[116:117], v86 offset:8192\n" \
  "v_mfma_i32_16x16x64_i8 v[204:207], v[124:127], a[152:155], v[204:207]\n" \
  "ds_read_b64_tr_b8 v[118:119], v86 offset:9216\n" \
  "v_mfma_i32_16x16x64_i8 v[236:239], v[120:123], a[140:143], v[236:239]\n" \
  "v_mfma_i32_16x16x64_i8 v[236:239], v[124:127], a[156:159], v[236:239]\n" \
  "s_waitcnt lgkmcnt(8)\n" \
  "v_mfma_i32_16x16x64_i8 v[144:147], v[96:99], a[128:131], v[144:147]\n" \
  "ds_read_b64_tr_b8 v[120:121], v87 offset:0\n" \
  "v_mfma_i32_16x16x64_i8 v[144:147], v[100:103], a[144:147], v[144:147]\n" \
  "ds_read_b64_tr_b8 v[122:123], v87 offset:1024\n" \
  "v_mfma_i32_16x16x64_i8 v[176:179], v[96:99], a[132:135], v[176:179]\n" \
  "v_mfma_i32_16x16x64_i8 v[176:179], v[100:103], a[148:151], v[176:179]\n" \
  "buffer_load_dwordx4 a[248:251], v91, s[28:31], s95 offen offset:2048 sc0 sc1\n" \
  "v_mfma_i32_16x16x64_i8 v[208:211], v[96:99], a[136:139], v[208:211]\n" \
  "ds_read_b64_tr_b8 v[124:125], v87 offset:8192\n" \
  "v_mfma_i32_16x16x64_i8 v[208:211], v[100:103], a[152:155], v[208:211]\n" \
  "ds_read_b64_tr_b8 v[126:127], v87 offset:9216\n" \
  "v_mfma_i32_16x16x64_i8 v[240:243], v[96:99], a[140:143], v[240:243]\n" \
  "v_mfma_i32_16x16x64_i8 v[240:243], v[100:103], a[156:159], v[240:243]\n" \
  "buffer_load_dwordx4 a[252:255], v91, s[28:31], s95 offen offset:3072 sc0 sc1\n" \
  "s_waitcnt lgkmcnt(8)\n" \
  "v_mfma_i32_16x16x64_i8 v[148:151], v[104:107], a[128:131], v[148:151]\n" \
  "ds_read_b64_tr_b8 v[96:97], v80 offset:16384\n" \
  "v_mfma_i32_16x16x64_i8 v[148:151], v[108:111], a[144:147], v[148:151]\n" \
  "ds_read_b64_tr_b8 v[98:99], v80 offset:17408\n" \
  "v_mfma_i32_16x16x64_i8 v[180:183], v[104:107], a[132:135], v[180:183]\n" \
  "v_mfma_i32_16x16x64_i8 v[180:183], v[108:111], a[148:151], v[180:183]\n" \
  "buffer_load_dwordx4 a[0:3], v44, s[28:31], s88 offen offset:0 sc0 sc1\n" \
  "v_mfma_i32_16x16x64_i8 v[212:215], v[104:107], a[136:139], v[212:215]\n" \
  "ds_read_b64_tr_b8 v[100:101], v80 offset:24576\n" \
  "v_mfma_i32_16x16x64_i8 v[212:215], v[108:111], a[152:155], v[212:215]\n" \
  "ds_read_b64_tr_b8 v[102:103], v80 offset:25600\n" \
  "v_mfma_i32_16x16x64_i8 v[244:247], v[104:107], a[140:143], v[244:247]\n" \
  "v_mfma_i32_16x16x64_i8 v[244:247], v[108:111], a[156:159], v[244:247]\n" \
  "s_waitcnt lgkmcnt(8)\n" \
  "v_mfma_i32_16x16x64_i8 v[152:155], v[112:115], a[128:131], v[152:155]\n" \
  "ds_read_b64_tr_b8 v[104:105], v81 offset:16384\n" \
  "v_mfma_i32_16x16x64_i8 v[152:155], v[116:119], a[144:147], v[152:155]\n" \
  "ds_read_b64_tr_b8 v[106:107], v81 offset:17408\n" \
  "v_mfma_i32_16x16x64_i8 v[184:187], v[112:115], a[132:135], v[184:187]\n" \
  "v_readlane_b32 s50, v46, s49\n" \
  "v_mfma_i32_16x16x64_i8 v[184:187], v[116:119], a[148:151], v[184:187]\n" \
  "buffer_load_dwordx4 a[4:7], v44, s[28:31], s88 offen offset:1024 sc0 sc1\n" \
  "v_mfma_i32_16x16x64_i8 v[216:219], v[112:115], a[136:139], v[216:219]\n" \
  "ds_read_b64_tr_b8 v[108:109], v81 offset:24576\n" \
  "v_mfma_i32_16x16x64_i8 v[216:219], v[116:119], a[152:155], v[216:219]\n" \
  "ds_read_b64_tr_b8 v[110:111], v81 offset:25600\n" \
  "v_mfma_i32_16x16x64_i8 v[248:251], v[112:115], a[140:143], v[248:251]\n" \
  "v_mfma_i32_16x16x64_i8 v[248:251], v[116:119], a[156:159], v[248:251]\n" \
  "buffer_load_dwordx4 a[8:11], v44, s[28:31], s88 offen offset:2048 sc0 sc1\n" \
  "s_waitcnt lgkmcnt(8)\n" \
  "v_mfma_i32_16x16x64_i8 v[156:159], v[120:123], a[128:131], v[156:159]\n" \
  "ds_read_b64_tr_b8 v[112:113], v82 offset:16384\n" \
  "v_mfma_i32_16x16x64_i8 v[156:159], v[124:127], a[144:147], v[156:159]\n" \
  "ds_read_b64_tr_b8 v[114:115], v82 offset:17408\n" \
  "v_mfma_i32_16x16x64_i8 v[188:191], v[120:123], a[132:135], v[188:191]\n" \
  "s_lshl_b32 s81, s50, 21\n" \
  "s_lshl_b32 s89, s50, 16\n" \
  "s_add_u32 s49, s49, 1\n" \
  "v_mfma_i32_16x16x64_i8 v[188:191], v[124:127], a[148:151], v[188:191]\n" \
  "buffer_load_dwordx4 a[12:15], v44, s[28:31], s88 offen offset:3072 sc0 sc1\n" \
  "v_mfma_i32_16x16x64_i8 v[220:223], v[120:123], a[136:139], v[220:223]\n" \
  "ds_read_b64_tr_b8 v[116:117], v82 offset:24576\n" \
  "v_mfma_i32_16x16x64_i8 v[220:223], v[124:127], a[152:155], v[220:223]\n" \
  "ds_read_b64_tr_b8 v[118:119], v82 offset:25600\n" \
  "v_mfma_i32_16x16x64_i8 v[252:255], v[120:123], a[140:143], v[252:255]\n" \
  "v_mfma_i32_16x16x64_i8 v[252:255], v[124:127], a[156:159], v[252:255]\n" \
  "; ---- step 5\n" \
  "s_waitcnt vmcnt(21)\n" \
  "s_waitcnt lgkmcnt(8)\n" \
  "v_mfma_i32_16x16x64_i8 v[128:131], v[96:99], a[160:163], v[128:131]\n" \
  "ds_read_b64_tr_b8 v[120:121], v83 offset:16384\n" \
  "v_mfma_i32_16x16x64_i8 v[128:131], v[100:103], a[176:179], v[128:131]\n" \
  "ds_read_b64_tr_b8 v[122:123], v83 offset:17408\n" \
  "v_mfma_i32_16x16x64_i8 v[160:163], v[96:99], a[164:167], v[160:163]\n" \
  "v_mfma_i32_16x16x64_i8 v[160:163], v[100:103], a[180:183], v[160:163]\n" \
  "buffer_load_dwordx4 a[16:19], v91, s[28:31], s88 offen offset:0 sc0 sc1\n" \
  "v_mfma_i32_16x16x64_i8 v[192:195], v[96:99], a[168:171], v[192:195]\n" \
  "ds_read_b64_tr_b8 v[124:125], v83 offset:24576\n" \
  "v_mfma_i32_16x16x64_i8 v[192:195], v[100:103], a[184:187], v[192:195]\n" \
  "ds_read_b64_tr_b8 v[126:127], v83 offset:25600\n" \
  "v_mfma_i32_16x16x64_i8 v[224:227], v[96:99], a[172:175], v[224:227]\n" \
  "v_mfma_i32_16x16x64_i8 v[224:227], v[100:103], a[188:191], v[224:227]\n" \
  "buffer_load_dwordx4 a[20:23], v91, s[28:31], s88 offen offset:1024 sc0 sc1\n" \
  "s_waitcnt lgkmcnt(8)\n" \
  "v_mfma_i32_16x16x64_i8 v[132:135], v[104:107], a[160:163], v[132:135]\n" \
  "ds_read_b64_tr_b8 v[96:97], v84 offset:16384\n" \
  "v_mfma_i32_16x16x64_i8 v[132:135], v[108:111], a[176:179], v[132:135]\n" \
  "ds_read_b64_tr_b8 v[98:99], v84 offset:17408\n" \
  "v_mfma_i32_16x16x64_i8 v[164:167], v[104:107], a[164:167], v[164:167]\n" \
  "v_mfma_i32_16x16x64_i8 v[164:167], v[108:111], a[180:183], v[164:167]\n" \
  "buffer_load_dwordx4 a[24:27], v91, s[28:31], s88 offen offset:2048 sc0 sc1\n" \
  "v_mfma_i32_16x16x64_i8 v[196:199], v[104:107], a[168:171], v[196:199]\n" \
  "ds_read_b64_tr_b8 v[100:101], v84 offset:24576\n" \
  "v_mfma_i32_16x16x64_i8 v[196:199], v[108:111], a[184:187], v[196:199]\n" \
  "ds_read_b64_tr_b8 v[102:103], v84 offset:25600\n" \
  "v_mfma_i32_16x16x64_i8 v[228:231], v[104:107], a[172:175], v[228:231]\n" \
  "v_mfma_i32_16x16x64_i8 v[228:231], v[108:111], a[188:191], v[228:231]\n" \
  "buffer_load_dwordx4 a[28:31], v91, s[28:31], s88 offen offset:3072 sc0 sc1\n" \
  "s_waitcnt lgkmcnt(8)\n" \
  "v_mfma_i32_16x16x64_i8 v[136:139], v[112:115], a[160:163], v[136:139]\n" \
  "ds_read_b64_tr_b8 v[104:105], v85 offset:16384\n" \
  "v_mfma_i32_16x16x64_i8 v[136:139], v[116:119], a[176:179], v[136:139]\n" \
  "ds_read_b64_tr_b8 v[106:107], v85 offset:17408\n" \
  "v_mfma_i32_16x16x64_i8 v[168:171], v[112:115], a[164:167], v[168:171]\n" \
  "v_mfma_i32_16x16x64_i8 v[168:171], v[116:119], a[180:183], v[168:171]\n" \
  "buffer_load_dwordx4 a[32:35], v44, s[28:31], s89 offen offset:0 sc0 sc1\n" \
  "v_mfma_i32_16x16x64_i8 v[200:203], v[112:115], a[168:171], v[200:203]\n" \
  "ds_read_b64_tr_b8 v[108:109], v85 offset:24576\n" \
  "v_mfma_i32_16x16x64_i8 v[200:203], v[116:119], a[184:187], v[200:203]\n" \
  "ds_read_b64_tr_b8 v[110:111], v85 offset:25600\n" \
  "v_mfma_i32_16x16x64_i8 v[232:235], v[112:115], a[172:175], v[232:235]\n" \
  "v_mfma_i32_16x16x64_i8 v[232:235], v[116:119], a[188:191], v[232:235]\n" \
  "buffer_load_dwordx4 a[36:39], v44, s[28:31], s89 offen offset:1024 sc0 sc1\n" \
  "s_waitcnt lgkmcnt(8)\n" \
  "v_mfma_i32_16x16x64_i8 v[140:143], v[120:123], a[160:163], v[140:143]\n" \
  "ds_read_b64_tr_b8 v[112:113], v86 offset:16384\n" \
  "v_mfma_i32_16x16x64_i8 v[140:143], v[124:127], a[176:179], v[140:143]\n" \
  "ds_read_b64_tr_b8 v[114:115], v86 offset:17408\n" \
  "v_mfma_i32_16x16x64_i8 v[172:175], v[120:123], a[164:167], v[172:175]\n" \
  "v_readlane_b32 s50, v46, s49\n" \
  "v_mfma_i32_16x16x64_i8 v[172:175], v[124:127], a[180:183], v[172:175]\n" \
  "buffer_load_dwordx4 a[40:43], v44, s[28:31], s89 offen offset:2048 sc0 sc1\n" \
  "s_waitcnt vmcnt(40)\n" \
  "s_barrier\n" \
  "v_mfma_i32_16x16x64_i8 v[204:207], v[120:123], a[168:171], v[204:207]\n" \
  "ds_read_b64_tr_b8 v[116:117], v86 offset:24576\n" \
  "v_mfma_i32_16x16x64_i8 v[204:207], v[124:127], a[184:187], v[204:207]\n" \
  "ds_read_b64_tr_b8 v[118:119], v86 offset:25600\n" \
  "v_mfma_i32_16x16x64_i8 v[236:239], v[120:123], a[172:175], v[236:239]\n" \
  "v_mfma_i32_16x16x64_i8 v[236:239], v[124:127], a[188:191], v[236:239]\n" \
  "buffer_load_dwordx4 a[44:47], v44, s[28:31], s89 offen offset:3072 sc0 sc1\n" \
  "s_waitcnt lgkmcnt(8)\n" \
  "v_mfma_i32_16x16x64_i8 v[144:147], v[96:99], a[160:163], v[144:147]\n" \
  "ds_read_b64_tr_b8 v[120:121], v87 offset:16384\n" \
  "v_mfma_i32_16x16x64_i8 v[144:147], v[100:103], a[176:179], v[144:147]\n" \
  "ds_read_b64_tr_b8 v[122:123], v87 offset:17408\n" \
  "s_add_u32 m0, s47, 16384\n" \
  "v_mfma_i32_16x16x64_i8 v[176:179], v[96:99], a[164:167], v[176:179]\n" \
  "buffer_load_dwordx4 v40, s[16:19], s81 offen nt lds\n" \
  "s_lshl_b32 s82, s50, 21\n" \
  "s_lshl_b32 s90, s50, 16\n" \
  "s_add_u32 s49, s49, 1\n" \
  "v_mfma_i32_16x16x64_i8 v[176:179], v[100:103], a[180:183], v[176:179]\n" \
  "v_mfma_i32_16x16x64_i8 v[208:211], v[96:99], a[168:171], v[208:211]\n" \
  "ds_read_b64_tr_b8 v[124:125], v87 offset:24576\n" \
  "v_mfma_i32_16x16x64_i8 v[208:211], v[100:103], a[184:187], v[208:211]\n" \
  "ds_read_b64_tr_b8 v[126:127], v87 offset:25600\n" \
  "s_add_u32 m0, s47, 17408\n" \
  "v_mfma_i32_16x16x64_i8 v[240:243], v[96:99], a[172:175], v[240:243]\n" \
  "buffer_load_dwordx4 v41, s[16:19], s81 offen nt lds\n" \
  "v_readlane_b32 s50, v46, s49\n" \
  "v_mfma_i32_16x16x64_i8 v[240:243], v[100:103], a[188:191], v[240:243]\n" \
  "s_waitcnt lgkmcnt(8)\n" \
  "v_mfma_i32_16x16x64_i8 v[148:151], v[104:107], a[160:163], v[148:151]\n" \
  "ds_read_b64_tr_b8 v[96:97], v80 offset:32768\n" \
  "v_mfma_i32_16x16x64_i8 v[148:151], v[108:111], a[176:179], v[148:151]\n" \
  "ds_read_b64_tr_b8 v[98:99], v80 offset:33792\n" \
  "s_add_u32 m0, s47, 18432\n" \
  "v_mfma_i32_16x16x64_i8 v[180:183], v[104:107], a[164:167], v[180:183]\n" \
  "buffer_load_dwordx4 v42, s[16:19], s81 offen nt lds\n" \
  "v_mfma_i32_16x16x64_i8 v[180:183], v[108:111], a[180:183], v[180:183]\n" \
  "v_mfma_i32_16x16x64_i8 v[212:215], v[104:107], a[168:171], v[212:215]\n" \
  "ds_read_b64_tr_b8 v[100:101], v80 offset:40960\n" \
  "v_mfma_i32_16x16x64_i8 v[212:215], v[108:111], a[184:187], v[212:215]\n" \
  "ds_read_b64_tr_b8 v[102:103], v80 offset:41984\n" \
  "s_add_u32 m0, s47, 19456\n" \
  "v_mfma_i32_16x16x64_i8 v[244:247], v[104:107], a[172:175], v[244:247]\n" \
  "buffer_load_dwordx4 v43, s[16:19], s81 offen nt lds\n" \
  "s_lshl_b32 s83, s50, 21\n" \
  "s_lshl_b32 s91, s50, 16\n" \
  "s_add_u32 s49, s49, 1\n" \
  "v_mfma_i32_16x16x64_i8 v[244:247], v[108:111], a[188:191], v[244:247]\n" \
  "s_waitcnt lgkmcnt(8)\n" \
  "v_mfma_i32_16x16x64_i8 v[152:155], v[112:115], a[160:163], v[152:155]\n" \
  "ds_read_b64_tr_b8 v[104:105], v81 offset:32768\n" \
  "v_mfma_i32_16x16x64_i8 v[152:155], v[116:119], a[176:179], v[152:155]\n" \
  "ds_read_b64_tr_b8 v[106:107], v81 offset:33792\n" \
  "s_add_u32 m0, s47, 32768\n" \
  "v_mfma_i32_16x16x64_i8 v[184:187], v[112:115], a[164:167], v[184:187]\n" \
  "buffer_load_dwordx4 v40, s[16:19], s82 offen nt lds\n" \
  "v_readlane_b32 s50, v46, s49\n" \
  "v_mfma_i32_16x16x64_i8 v[184:187], v[116:119], a[180:183], v[184:187]\n" \
  "v_mfma_i32_16x16x64_i8 v[216:219], v[112:115], a[168:171], v[216:219]\n" \
  "ds_read_b64_tr_b8 v[108:109], v81 offset:40960\n" \
  "v_mfma_i32_16x16x64_i8 v[216:219], v[116:119], a[184:187], v[216:219]\n" \
  "ds_read_b64_tr_b8 v[110:111], v81 offset:41984\n" \
  "s_add_u32 m0, s47, 33792\n" \
  "v_mfma_i32_16x16x64_i8 v[248:251], v[112:115], a[172:175], v[248:251]\n" \
  "buffer_load_dwordx4 v41, s[16:19], s82 offen nt lds\n" \
  "v_mfma_i32_16x16x64_i8 v[248:251], v[116:119], a[188:191], v[248:251]\n" \
  "s_waitcnt lgkmcnt(8)\n" \
  "v_mfma_i32_16x16x64_i8 v[156:159], v[120:123], a[160:163], v[156:159]\n" \
  "ds_read_b64_tr_b8 v[112:113], v82 offset:32768\n" \
  "v_mfma_i32_16x16x64_i8 v[156:159], v[124:127], a[176:179], v[156:159]\n" \
  "ds_read_b64_tr_b8 v[114:115], v82 offset:33792\n" \
  "s_add_u32 m0, s47, 34816\n" \
  "v_mfma_i32_16x16x64_i8 v[188:191], v[120:123], a[164:167], v[188:191]\n" \
  "buffer_load_dwordx4 v42, s[16:19], s82 offen nt lds\n" \
  "s_lshl_b32 s84, s50, 21\n" \
  "s_lshl_b32 s92, s50, 16\n" \
  "s_add_u32 s49, s49, 1\n" \
  "v_mfma_i32_16x16x64_i8 v[188:191], v[124:127], a[180:183], v[188:191]\n" \
  "v_mfma_i32_16x16x64_i8 v[220:223], v[120:123], a[168:171], v[220:223]\n" \
  "ds_read_b64_tr_b8 v[116:117], v82 offset:40960\n" \
  "v_mfma_i32_16x16x64_i8 v[220:223], v[124:127], a[184:187], v[220:223]\n" \
  "ds_read_b64_tr_b8 v[118:119], v82 offset:41984\n" \
  "s_add_u32 m0, s47, 35840\n" \
  "v_mfma_i32_16x16x64_i8 v[252:255], v[120:123], a[172:175], v[252:255]\n" \
  "buffer_load_dwordx4 v43, s[16:19], s82 offen nt lds\n" \
  "v_mfma_i32_16x16x64_i8 v[252:255], v[124:127], a[188:191], v[252:255]\n" \
  "; ---- step 6\n" \
  "s_waitcnt vmcnt(29)\n" \
  "s_waitcnt lgkmcnt(8)\n" \
  "v_mfma_i32_16x16x64_i8 v[128:131], v[96:99], a[192:195], v[128:131]\n" \
  "ds_read_b64_tr_b8 v[120:121], v83 offset:32768\n" \
  "s_add_u32 m0, s47, 49152\n" \
  "v_mfma_i32_16x16x64_i8 v[128:131], v[100:103], a[208:211], v[128:131]\n" \
  "ds_read_b64_tr_b8 v[122:123], v83 offset:33792\n" \
  "buffer_load_dwordx4 v40, s[16:19], s83 offen lds\n" \
  "s_add_u32 m0, s47, 50176\n" \
  "v_mfma_i32_16x16x64_i8 v[160:163], v[96:99], a[196:199], v[160:163]\n" \
  "buffer_load_dwordx4 v41, s[16:19], s83 offen lds\n" \
  "s_add_u32 m0, s47, 51200\n" \
  "v_mfma_i32_16x16x64_i8 v[160:163], v[100:103], a[212:215], v[160:163]\n" \
  "buffer_load_dwordx4 v42, s[16:19], s83 offen lds\n" \
  "s_add_u32 m0, s47, 52224\n" \
  "v_mfma_i32_16x16x64_i8 v[192:195], v[96:99], a[200:203], v[192:195]\n" \
  "ds_read_b64_tr_b8 v[124:125], v83 offset:40960\n" \
  "buffer_load_dwordx4 v43, s[16:19], s83 offen lds\n" \
  "s_add_u32 m0, s47, 65536\n" \
  "v_mfma_i32_16x16x64_i8 v[192:195], v[100:103], a[216:219], v[192:195]\n" \
  "ds_read_b64_tr_b8 v[126:127], v83 offset:41984\n" \
  "buffer_load_dwordx4 v40, s[16:19], s84 offen lds\n" \
  "s_add_u32 m0, s47, 66560\n" \
  "v_mfma_i32_16x16x64_i8 v[224:227], v[96:99], a[204:207], v[224:227]\n" \
  "buffer_load_dwordx4 v41, s[16:19], s84 offen lds\n" \
  "s_add_u32 m0, s47, 67584\n" \
  "v_mfma_i32_16x16x64_i8 v[224:227], v[100:103], a[220:223], v[224:227]\n" \
  "buffer_load_dwordx4 v42, s[16:19], s84 offen lds\n" \
  "s_add_u32 m0, s47, 68608\n" \
  "s_waitcnt lgkmcnt(8)\n" \
  "v_mfma_i32_16x16x64_i8 v[132:135], v[104:107], a[192:195], v[132:135]\n" \
  "ds_read_b64_tr_b8 v[96:97], v84 offset:32768\n" \
  "buffer_load_dwordx4 v43, s[16:19], s84 offen lds\n" \
  "v_mfma_i32_16x16x64_i8 v[132:135], v[108:111], a[208:211], v[132:135]\n" \
  "ds_read_b64_tr_b8 v[98:99], v84 offset:33792\n" \
  "v_mfma_i32_16x16x64_i8 v[164:167], v[104:107], a[196:199], v[164:167]\n" \
  "v_mfma_i32_16x16x64_i8 v[164:167], v[108:111], a[212:215], v[164:167]\n" \
  "v_mfma_i32_16x16x64_i8 v[196:199], v[104:107], a[200:203], v[196:199]\n" \
  "ds_read_b64_tr_b8 v[100:101], v84 offset:40960\n" \
  "v_mfma_i32_16x16x64_i8 v[196:199], v[108:111], a[216:219], v[196:199]\n" \
  "ds_read_b64_tr_b8 v[102:103], v84 offset:41984\n" \
  "v_mfma_i32_16x16x64_i8 v[228:231], v[104:107], a[204:207], v[228:231]\n" \
  "v_mfma_i32_16x16x64_i8 v[228:231], v[108:111], a[220:223], v[228:231]\n" \
  "s_waitcnt lgkmcnt(8)\n" \
  "v_mfma_i32_16x16x64_i8 v[136:139], v[112:115], a[192:195], v[136:139]\n" \
  "ds_read_b64_tr_b8 v[104:105], v85 offset:32768\n" \
  "v_mfma_i32_16x16x64_i8 v[136:139], v[116:119], a[208:211], v[136:139]\n" \
  "ds_read_b64_tr_b8 v[106:107], v85 offset:33792\n" \
  "v_mfma_i32_16x16x64_i8 v[168:171], v[112:115], a[196:199], v[168:171]\n" \
  "v_mfma_i32_16x16x64_i8 v[168:171], v[116:119], a[212:215], v[168:171]\n" \
  "v_mfma_i32_16x16x64_i8 v[200:203], v[112:115], a[200:203], v[200:203]\n" \
  "ds_read_b64_tr_b8 v[108:109], v85 offset:40960\n" \
  "v_mfma_i32_16x16x64_i8 v[200:203], v[116:119], a[216:219], v[200:203]\n" \
  "ds_read_b64_tr_b8 v[110:111], v85 offset:41984\n" \
  "v_mfma_i32_16x16x64_i8 v[232:235], v[112:115], a[204:207], v[232:235]\n" \
  "v_mfma_i32_16x16x64_i8 v[232:235], v[116:119], a[220:223], v[232:235]\n" \
  "s_waitcnt lgkmcnt(8)\n" \
  "v_mfma_i32_16x16x64_i8 v[140:143], v[120:123], a[192:195], v[140:143]\n" \
  "ds_read_b64_tr_b8 v[112:113], v86 offset:32768\n" \
  "v_mfma_i32_16x16x64_i8 v[140:143], v[124:127], a[208:211], v[140:143]\n" \
  "ds_read_b64_tr_b8 v[114:115], v86 offset:33792\n" \
  "v_mfma_i32_16x16x64_i8 v[172:175], v[120:123], a[196:199], v[172:175]\n" \
  "v_readlane_b32 s50, v46, s49\n" \
  "v_mfma_i32_16x16x64_i8 v[172:175], v[124:127], a[212:215], v[172:175]\n" \
  "s_waitcnt vmcnt(53)\n" \
  "s_barrier\n" \
  "v_mfma_i32_16x16x64_i8 v[204:207], v[120:123], a[200:203], v[204:207]\n" \
  "ds_read_b64_tr_b8 v[116:117], v86 offset:40960\n" \
  "v_mfma_i32_16x16x64_i8 v[204:207], v[124:127], a[216:219], v[204:207]\n" \
  "ds_read_b64_tr_b8 v[118:119], v86 offset:41984\n" \
  "v_mfma_i32_16x16x64_i8 v[236:239], v[120:123], a[204:207], v[236:239]\n" \
  "v_mfma_i32_16x16x64_i8 v[236:239], v[124:127], a[220:223], v[236:239]\n" \
  "s_waitcnt lgkmcnt(8)\n" \
  "v_mfma_i32_16x16x64_i8 v[144:147], v[96:99], a[192:195], v[144:147]\n" \
  "ds_read_b64_tr_b8 v[120:121], v87 offset:32768\n" \
  "v_mfma_i32_16x16x64_i8 v[144:147], v[100:103], a[208:211], v[144:147]\n" \
  "ds_read_b64_tr_b8 v[122:123], v87 offset:33792\n" \
  "v_mfma_i32_16x16x64_i8 v[176:179], v[96:99], a[196:199], v[176:179]\n" \
  "s_lshl_b32 s85, s50, 21\n" \
  "s_lshl_b32 s93, s50, 16\n" \
  "s_add_u32 s49, s49, 1\n" \
  "v_mfma_i32_16x16x64_i8 v[176:179], v[100:103], a[212:215], v[176:179]\n" \
  "v_mfma_i32_16x16x64_i8 v[208:211], v[96:99], a[200:203], v[208:211]\n" \
  "ds_read_b64_tr_b8 v[124:125], v87 offset:40960\n" \
  "v_mfma_i32_16x16x64_i8 v[208:211], v[100:103], a[216:219], v[208:211]\n" \
  "ds_read_b64_tr_b8 v[126:127], v87 offset:41984\n" \
  "v_mfma_i32_16x16x64_i8 v[240:243], v[96:99], a[204:207], v[240:243]\n" \
  "v_mfma_i32_16x16x64_i8 v[240:243], v[100:103], a[220:223], v[240:243]\n" \
  "s_waitcnt lgkmcnt(8)\n" \
  "v_mfma_i32_16x16x64_i8 v[148:151], v[104:107], a[192:195], v[148:151]\n" \
  "ds_read_b64_tr_b8 v[96:97], v80 offset:49152\n" \
  "s_add_u32 m0, s47, 81920\n" \
  "v_mfma_i32_16x16x64_i8 v[148:151], v[108:111], a[208:211], v[148:151]\n" \
  "ds_read_b64_tr_b8 v[98:99], v80 offset:50176\n" \
  "buffer_load_dwordx4 v40, s[16:19], s85 offen lds\n" \
  "s_add_u32 m0, s47, 82944\n" \
  "v_mfma_i32_16x16x64_i8 v[180:183], v[104:107], a[196:199], v[180:183]\n" \
  "buffer_load_dwordx4 v41, s[16:19], s85 offen lds\n" \
  "s_add_u32 m0, s47, 83968\n" \
  "v_mfma_i32_16x16x64_i8 v[180:183], v[108:111], a[212:215], v[180:183]\n" \
  "buffer_load_dwordx4 v42, s[16:19], s85 offen lds\n" \
  "s_add_u32 m0, s47, 84992\n" \
  "v_mfma_i32_16x16x64_i8 v[212:215], v[104:107], a[200:203], v[212:215]\n" \
  "ds_read_b64_tr_b8 v[100:101], v80 offset:57344\n" \
  "buffer_load_dwordx4 v43, s[16:19], s85 offen lds\n" \
  "v_mfma_i32_16x16x64_i8 v[212:215], v[108:111], a[216:219], v[212:215]\n" \
  "ds_read_b64_tr_b8 v[102:103], v80 offset:58368\n" \
  "v_mfma_i32_16x16x64_i8 v[244:247], v[104:107], a[204:207], v[244:247]\n" \
  "v_mfma_i32_16x16x64_i8 v[244:247], v[108:111], a[220:223], v[244:247]\n" \
  "s_waitcnt lgkmcnt(8)\n" \
  "v_mfma_i32_16x16x64_i8 v[152:155], v[112:115], a[192:195], v[152:155]\n" \
  "ds_read_b64_tr_b8 v[104:105], v81 offset:49152\n" \
  "v_mfma_i32_16x16x64_i8 v[152:155], v[116:119], a[208:211], v[152:155]\n" \
  "ds_read_b64_tr_b8 v[106:107], v81 offset:50176\n" \
  "v_mfma_i32_16x16x64_i8 v[184:187], v[112:115], a[196:199], v[184:187]\n" \
  "v_mfma_i32_16x16x64_i8 v[184:187], v[116:119], a[212:215], v[184:187]\n" \
  "v_mfma_i32_16x16x64_i8 v[216:219], v[112:115], a[200:203], v[216:219]\n" \
  "ds_read_b64_tr_b8 v[108:109], v81 offset:57344\n" \
  "v_mfma_i32_16x16x64_i8 v[216:219], v[116:119], a[216:219], v[216:219]\n" \
  "ds_read_b64_tr_b8 v[110:111], v81 offset:58368\n" \
  "v_mfma_i32_16x16x64_i8 v[248:251], v[112:115], a[204:207], v[248:251]\n" \
  "v_mfma_i32_16x16x64_i8 v[248:251], v[116:119], a[220:223], v[248:251]\n" \
  "s_waitcnt lgkmcnt(8)\n" \
  "v_mfma_i32_16x16x64_i8 v[156:159], v[120:123], a[192:195], v[156:159]\n" \
  "ds_read_b64_tr_b8 v[112:113], v82 offset:49152\n" \
  "v_mfma_i32_16x16x64_i8 v[156:159], v[124:127], a[208:211], v[156:159]\n" \
  "ds_read_b64_tr_b8 v[114:115], v82 offset:50176\n" \
  "v_mfma_i32_16x16x64_i8 v[188:191], v[120:123], a[196:199], v[188:191]\n" \
  "v_mfma_i32_16x16x64_i8 v[188:191], v[124:127], a[212:215], v[188:191]\n" \
  "v_mfma_i32_16x16x64_i8 v[220:223], v[120:123], a[200:203], v[220:223]\n" \
  "ds_read_b64_tr_b8 v[116:117], v82 offset:57344\n" \
  "v_mfma_i32_16x16x64_i8 v[220:223], v[124:127], a[216:219], v[220:223]\n" \
  "ds_read_b64_tr_b8 v[118:119], v82 offset:58368\n" \
  "v_mfma_i32_16x16x64_i8 v[252:255], v[120:123], a[204:207], v[252:255]\n" \
  "v_mfma_i32_16x16x64_i8 v[252:255], v[124:127], a[220:223], v[252:255]\n" \
  "; ---- step 7\n" \
  "s_waitcnt vmcnt(32)\n" \
  "s_waitcnt lgkmcnt(8)\n" \
  "v_mfma_i32_16x16x64_i8 v[128:131], v[96:99], a[224:227], v[128:131]\n" \
  "ds_read_b64_tr_b8 v[120:121], v83 offset:49152\n" \
  "v_mfma_i32_16x16x64_i8 v[128:131], v[100:103], a[240:243], v[128:131]\n" \
  "ds_read_b64_tr_b8 v[122:123], v83 offset:50176\n" \
  "v_mfma_i32_16x16x64_i8 v[160:163], v[96:99], a[228:231], v[160:163]\n" \
  "v_mfma_i32_16x16x64_i8 v[160:163], v[100:103], a[244:247], v[160:163]\n" \
  "v_mfma_i32_16x16x64_i8 v[192:195], v[96:99], a[232:235], v[192:195]\n" \
  "ds_read_b64_tr_b8 v[124:125], v83 offset:57344\n" \
  "v_mfma_i32_16x16x64_i8 v[192:195], v[100:103], a[248:251], v[192:195]\n" \
  "ds_read_b64_tr_b8 v[126:127], v83 offset:58368\n" \
  "v_mfma_i32_16x16x64_i8 v[224:227], v[96:99], a[236:239], v[224:227]\n" \
  "v_mfma_i32_16x16x64_i8 v[224:227], v[100:103], a[252:255], v[224:227]\n" \
  "s_waitcnt lgkmcnt(8)\n" \
  "v_mfma_i32_16x16x64_i8 v[132:135], v[104:107], a[224:227], v[132:135]\n" \
  "ds_read_b64_tr_b8 v[96:97], v84 offset:49152\n" \
  "v_mfma_i32_16x16x64_i8 v[132:135], v[108:111], a[240:243], v[132:135]\n" \
  "ds_read_b64_tr_b8 v[98:99], v84 offset:50176\n" \
  "v_mfma_i32_16x16x64_i8 v[164:167], v[104:107], a[228:231], v[164:167]\n" \
  "v_mfma_i32_16x16x64_i8 v[164:167], v[108:111], a[244:247], v[164:167]\n" \
  "v_mfma_i32_16x16x64_i8 v[196:199], v[104:107], a[232:235], v[196:199]\n" \
  "ds_read_b64_tr_b8 v[100:101], v84 offset:57344\n" \
  "v_mfma_i32_16x16x64_i8 v[196:199], v[108:111], a[248:251], v[196:199]\n" \
  "ds_read_b64_tr_b8 v[102:103], v84 offset:58368\n" \
  "v_mfma_i32_16x16x64_i8 v[228:231], v[104:107], a[236:239], v[228:231]\n" \
  "v_mfma_i32_16x16x64_i8 v[228:231], v[108:111], a[252:255], v[228:231]\n" \
  "s_waitcnt lgkmcnt(8)\n" \
  "v_mfma_i32_16x16x64_i8 v[136:139], v[112:115], a[224:227], v[136:139]\n" \
  "ds_read_b64_tr_b8 v[104:105], v85 offset:49152\n" \
  "v_mfma_i32_16x16x64_i8 v[136:139], v[116:119], a[240:243], v[136:139]\n" \
  "ds_read_b64_tr_b8 v[106:107], v85 offset:50176\n" \
  "v_mfma_i32_16x16x64_i8 v[168:171], v[112:115], a[228:231], v[168:171]\n" \
  "v_mfma_i32_16x16x64_i8 v[168:171], v[116:119], a[244:247], v[168:171]\n" \
  "v_mfma_i32_16x16x64_i8 v[200:203], v[112:115], a[232:235], v[200:203]\n" \
  "ds_read_b64_tr_b8 v[108:109], v85 offset:57344\n" \
  "v_mfma_i32_16x16x64_i8 v[200:203], v[116:119], a[248:251], v[200:203]\n" \
  "ds_read_b64_tr_b8 v[110:111], v85 offset:58368\n" \
  "buffer_load_dwordx4 a[48:51], v91, s[28:31], s89 offen offset:0 sc0 sc1\n" \
  "v_mfma_i32_16x16x64_i8 v[232:235], v[112:115], a[236:239], v[232:235]\n" \
  "v_mfma_i32_16x16x64_i8 v[232:235], v[116:119], a[252:255], v[232:235]\n" \
  "buffer_load_dwordx4 a[52:55], v91, s[28:31], s89 offen offset:1024 sc0 sc1\n" \
  "s_waitcnt lgkmcnt(8)\n" \
  "v_mfma_i32_16x16x64_i8 v[140:143], v[120:123], a[224:227], v[140:143]\n" \
  "ds_read_b64_tr_b8 v[112:113], v86 offset:49152\n" \
  "v_mfma_i32_16x16x64_i8 v[140:143], v[124:127], a[240:243], v[140:143]\n" \
  "ds_read_b64_tr_b8 v[114:115], v86 offset:50176\n" \
  "v_mfma_i32_16x16x64_i8 v[172:175], v[120:123], a[228:231], v[172:175]\n" \
  "v_mfma_i32_16x16x64_i8 v[172:175], v[124:127], a[244:247], v[172:175]\n" \
  "buffer_load_dwordx4 a[56:59], v91, s[28:31], s89 offen offset:2048 sc0 sc1\n" \
  "s_waitcnt vmcnt(56)\n" \
  "s_barrier\n" \
  "v_mfma_i32_16x16x64_i8 v[204:207], v[120:123], a[232:235], v[204:207]\n" \
  "ds_read_b64_tr_b8 v[116:117], v86 offset:57344\n" \
  "v_mfma_i32_16x16x64_i8 v[204:207], v[124:127], a[248:251], v[204:207]\n" \
  "ds_read_b64_tr_b8 v[118:119], v86 offset:58368\n" \
  "v_mfma_i32_16x16x64_i8 v[236:239], v[120:123], a[236:239], v[236:239]\n" \
  "v_mfma_i32_16x16x64_i8 v[236:239], v[124:127], a[252:255], v[236:239]\n" \
  "buffer_load_dwordx4 a[60:63], v91, s[28:31], s89 offen offset:3072 sc0 sc1\n" \
  "s_waitcnt lgkmcnt(8)\n" \
  "v_mfma_i32_16x16x64_i8 v[144:147], v[96:99], a[224:227], v[144:147]\n" \
  "ds_read_b64_tr_b8 v[120:121], v87 offset:49152\n" \
  "v_mfma_i32_16x16x64_i8 v[144:147], v[100:103], a[240:243], v[144:147]\n" \
  "ds_read_b64_tr_b8 v[122:123], v87 offset:50176\n" \
  "v_mfma_i32_16x16x64_i8 v[176:179], v[96:99], a[228:231], v[176:179]\n" \
  "v_mfma_i32_16x16x64_i8 v[176:179], v[100:103], a[244:247], v[176:179]\n" \
  "buffer_load_dwordx4 a[64:67], v44, s[28:31], s90 offen offset:0 sc0 sc1\n" \
  "v_mfma_i32_16x16x64_i8 v[208:211], v[96:99], a[232:235], v[208:211]\n" \
  "ds_read_b64_tr_b8 v[124:125], v87 offset:57344\n" \
  "v_mfma_i32_16x16x64_i8 v[208:211], v[100:103], a[248:251], v[208:211]\n" \
  "ds_read_b64_tr_b8 v[126:127], v87 offset:58368\n" \
  "v_mfma_i32_16x16x64_i8 v[240:243], v[96:99], a[236:239], v[240:243]\n" \
  "v_mfma_i32_16x16x64_i8 v[240:243], v[100:103], a[252:255], v[240:243]\n" \
  "buffer_load_dwordx4 a[68:71], v44, s[28:31], s90 offen offset:1024 sc0 sc1\n" \
  "s_waitcnt lgkmcnt(8)\n" \
  "v_mfma_i32_16x16x64_i8 v[148:151], v[104:107], a[224:227], v[148:151]\n" \
  "ds_read_b64_tr_b8 v[96:97], v32 offset:0\n" \
  "v_mfma_i32_16x16x64_i8 v[148:151], v[108:111], a[240:243], v[148:151]\n" \
  "ds_read_b64_tr_b8 v[98:99], v32 offset:1024\n" \
  "v_mfma_i32_16x16x64_i8 v[180:183], v[104:107], a[228:231], v[180:183]\n" \
  "v_mfma_i32_16x16x64_i8 v[180:183], v[108:111], a[244:247], v[180:183]\n" \
  "buffer_load_dwordx4 a[72:75], v44, s[28:31], s90 offen offset:2048 sc0 sc1\n" \
  "v_mfma_i32_16x16x64_i8 v[212:215], v[104:107], a[232:235], v[212:215]\n" \
  "ds_read_b64_tr_b8 v[100:101], v32 offset:8192\n" \
  "v_mfma_i32_16x16x64_i8 v[212:215], v[108:111], a[248:251], v[212:215]\n" \
  "ds_read_b64_tr_b8 v[102:103], v32 offset:9216\n" \
  "buffer_load_dwordx4 a[76:79], v44, s[28:31], s90 offen offset:3072 sc0 sc1\n" \
  "v_mfma_i32_16x16x64_i8 v[244:247], v[104:107], a[236:239], v[244:247]\n" \
  "v_mfma_i32_16x16x64_i8 v[244:247], v[108:111], a[252:255], v[244:247]\n" \
  "s_waitcnt lgkmcnt(8)\n" \
  "v_mfma_i32_16x16x64_i8 v[152:155], v[112:115], a[224:227], v[152:155]\n" \
  "ds_read_b64_tr_b8 v[104:105], v33 offset:0\n" \
  "v_mfma_i32_16x16x64_i8 v[152:155], v[116:119], a[240:243], v[152:155]\n" \
  "ds_read_b64_tr_b8 v[106:107], v33 offset:1024\n" \
  "buffer_load_dwordx4 a[80:83], v91, s[28:31], s90 offen offset:0 sc0 sc1\n" \
  "v_mfma_i32_16x16x64_i8 v[184:187], v[112:115], a[228:231], v[184:187]\n" \
  "v_mfma_i32_16x16x64_i8 v[184:187], v[116:119], a[244:247], v[184:187]\n" \
  "v_mfma_i32_16x16x64_i8 v[216:219], v[112:115], a[232:235], v[216:219]\n" \
  "ds_read_b64_tr_b8 v[108:109], v33 offset:8192\n" \
  "v_mfma_i32_16x16x64_i8 v[216:219], v[116:119], a[248:251], v[216:219]\n" \
  "ds_read_b64_tr_b8 v[110:111], v33 offset:9216\n" \
  "buffer_load_dwordx4 a[84:87], v91, s[28:31], s90 offen offset:1024 sc0 sc1\n" \
  "v_mfma_i32_16x16x64_i8 v[248:251], v[112:115], a[236:239], v[248:251]\n" \
  "v_mfma_i32_16x16x64_i8 v[248:251], v[116:119], a[252:255], v[248:251]\n" \
  "s_waitcnt lgkmcnt(8)\n" \
  "v_mfma_i32_16x16x64_i8 v[156:159], v[120:123], a[224:227], v[156:159]\n" \
  "ds_read_b64_tr_b8 v[112:113], v34 offset:0\n" \
  "v_mfma_i32_16x16x64_i8 v[156:159], v[124:127], a[240:243], v[156:159]\n" \
  "ds_read_b64_tr_b8 v[114:115], v34 offset:1024\n" \
  "buffer_load_dwordx4 a[88:91], v91, s[28:31], s90 offen offset:2048 sc0 sc1\n" \
  "v_mfma_i32_16x16x64_i8 v[188:191], v[120:123], a[228:231], v[188:191]\n" \
  "v_mfma_i32_16x16x64_i8 v[188:191], v[124:127], a[244:247], v[188:191]\n" \
  "v_mfma_i32_16x16x64_i8 v[220:223], v[120:123], a[232:235], v[220:223]\n" \
  "ds_read_b64_tr_b8 v[116:117], v34 offset:8192\n" \
  "v_mfma_i32_16x16x64_i8 v[220:223], v[124:127], a[248:251], v[220:223]\n" \
  "ds_read_b64_tr_b8 v[118:119], v34 offset:9216\n" \
  "buffer_load_dwordx4 a[92:95], v91, s[28:31], s90 offen offset:3072 sc0 sc1\n" \
  "v_mfma_i32_16x16x64_i8 v[252:255], v[120:123], a[236:239], v[252:255]\n" \
  "v_mfma_i32_16x16x64_i8 v[252:255], v[124:127], a[252:255], v[252:255]\n" \
  "s_add_u32 s53, s53, 1\n" \
  "s_add_u32 s54, s54, 1\n" \
  "; ---- step 0\n" \
  "global_load_dword v90, v51, s[12:13] sc1\n" \
  "s_waitcnt vmcnt(37)\n" \
  "s_waitcnt lgkmcnt(8)\n" \
  "v_mfma_i32_16x16x64_i8 v[128:131], v[96:99], a[0:3], v[128:131]\n" \
  "ds_read_b64_tr_b8 v[120:121], v35 offset:0\n" \
  "v_mfma_i32_16x16x64_i8 v[128:131], v[100:103], a[16:19], v[128:131]\n" \
  "ds_read_b64_tr_b8 v[122:123], v35 offset:1024\n" \
  "v_mfma_i32_16x16x64_i8 v[160:163], v[96:99], a[4:7], v[160:163]\n" \
  "v_mfma_i32_16x16x64_i8 v[160:163], v[100:103], a[20:23], v[160:163]\n" \
  "buffer_load_dwordx4 a[96:99], v44, s[28:31], s91 offen offset:0 sc0 sc1\n" \
  "v_mfma_i32_16x16x64_i8 v[192:195], v[96:99], a[8:11], v[192:195]\n" \
  "ds_read_b64_tr_b8 v[124:125], v35 offset:8192\n" \
  "v_mfma_i32_16x16x64_i8 v[192:195], v[100:103], a[24:27], v[192:195]\n" \
  "ds_read_b64_tr_b8 v[126:127], v35 offset:9216\n" \
  "v_mfma_i32_16x16x64_i8 v[224:227], v[96:99], a[12:15], v[224:227]\n" \
  "v_mfma_i32_16x16x64_i8 v[224:227], v[100:103], a[28:31], v[224:227]\n" \
  "buffer_load_dwordx4 a[100:103], v44, s[28:31], s91 offen offset:1024 sc0 sc1\n" \
  "s_waitcnt lgkmcnt(8)\n" \
  "v_mfma_i32_16x16x64_i8 v[132:135], v[104:107], a[0:3], v[132:135]\n" \
  "ds_read_b64_tr_b8 v[96:97], v36 offset:0\n" \
  "v_mfma_i32_16x16x64_i8 v[132:135], v[108:111], a[16:19], v[132:135]\n" \
  "ds_read_b64_tr_b8 v[98:99], v36 offset:1024\n" \
  "v_mfma_i32_16x16x64_i8 v[164:167], v[104:107], a[4:7], v[164:167]\n" \
  "v_mfma_i32_16x16x64_i8 v[164:167], v[108:111], a[20:23], v[164:167]\n" \
  "buffer_load_dwordx4 a[104:107], v44, s[28:31], s91 offen offset:2048 sc0 sc1\n" \
  "v_mfma_i32_16x16x64_i8 v[196:199], v[104:107], a[8:11], v[196:199]\n" \
  "ds_read_b64_tr_b8 v[100:101], v36 offset:8192\n" \
  "v_mfma_i32_16x16x64_i8 v[196:199], v[108:111], a[24:27], v[196:199]\n" \
  "ds_read_b64_tr_b8 v[102:103], v36 offset:9216\n" \
  "v_mfma_i32_16x16x64_i8 v[228:231], v[104:107], a[12:15], v[228:231]\n" \
  "v_mfma_i32_16x16x64_i8 v[228:231], v[108:111], a[28:31], v[228:231]\n" \
  "s_waitcnt lgkmcnt(8)\n" \
  "v_mfma_i32_16x16x64_i8 v[136:139], v[112:115], a[0:3], v[136:139]\n" \
  "ds_read_b64_tr_b8 v[104:105], v37 offset:0\n" \
  "v_mfma_i32_16x16x64_i8 v[136:139], v[116:119], a[16:19], v[136:139]\n" \
  "ds_read_b64_tr_b8 v[106:107], v37 offset:1024\n" \
  "v_mfma_i32_16x16x64_i8 v[168:171], v[112:115], a[4:7], v[168:171]\n" \
  "v_mfma_i32_16x16x64_i8 v[168:171], v[116:119], a[20:23], v[168:171]\n" \
  "buffer_load_dwordx4 a[108:111], v44, s[28:31], s91 offen offset:3072 sc0 sc1\n" \
  "v_mfma_i32_16x16x64_i8 v[200:203], v[112:115], a[8:11], v[200:203]\n" \
  "ds_read_b64_tr_b8 v[108:109], v37 offset:8192\n" \
  "v_mfma_i32_16x16x64_i8 v[200:203], v[116:119], a[24:27], v[200:203]\n" \
  "ds_read_b64_tr_b8 v[110:111], v37 offset:9216\n" \
  "v_mfma_i32_16x16x64_i8 v[232:235], v[112:115], a[12:15], v[232:235]\n" \
  "v_mfma_i32_16x16x64_i8 v[232:235], v[116:119], a[28:31], v[232:235]\n" \
  "buffer_load_dwordx4 a[112:115], v91, s[28:31], s91 offen offset:0 sc0 sc1\n" \
  "s_waitcnt lgkmcnt(8)\n" \
  "v_mfma_i32_16x16x64_i8 v[140:143], v[120:123], a[0:3], v[140:143]\n" \
  "ds_read_b64_tr_b8 v[112:113], v38 offset:0\n" \
  "v_mfma_i32_16x16x64_i8 v[140:143], v[124:127], a[16:19], v[140:143]\n" \
  "ds_read_b64_tr_b8 v[114:115], v38 offset:1024\n" \
  "v_mfma_i32_16x16x64_i8 v[172:175], v[120:123], a[4:7], v[172:175]\n" \
  "v_mfma_i32_16x16x64_i8 v[172:175], v[124:127], a[20:23], v[172:175]\n" \
  "buffer_load_dwordx4 a[116:119], v91, s[28:31], s91 offen offset:1024 sc0 sc1\n" \
  "s_waitcnt vmcnt(35)\n" \
  "s_barrier\n" \
  "v_mfma_i32_16x16x64_i8 v[204:207], v[120:123], a[8:11], v[204:207]\n" \
  "ds_read_b64_tr_b8 v[116:117], v38 offset:8192\n" \
  "v_mfma_i32_16x16x64_i8 v[204:207], v[124:127], a[24:27], v[204:207]\n" \
  "ds_read_b64_tr_b8 v[118:119], v38 offset:9216\n" \
  "v_mfma_i32_16x16x64_i8 v[236:239], v[120:123], a[12:15], v[236:239]\n" \
  "v_mfma_i32_16x16x64_i8 v[236:239], v[124:127], a[28:31], v[236:239]\n" \
  "s_waitcnt lgkmcnt(8)\n" \
  "v_mfma_i32_16x16x64_i8 v[144:147], v[96:99], a[0:3], v[144:147]\n" \
  "ds_read_b64_tr_b8 v[120:121], v39 offset:0\n" \
  "v_mfma_i32_16x16x64_i8 v[144:147], v[100:103], a[16:19], v[144:147]\n" \
  "ds_read_b64_tr_b8 v[122:123], v39 offset:1024\n" \
  "v_mfma_i32_16x16x64_i8 v[176:179], v[96:99], a[4:7], v[176:179]\n" \
  "v_mfma_i32_16x16x64_i8 v[176:179], v[100:103], a[20:23], v[176:179]\n" \
  "buffer_load_dwordx4 a[120:123], v91, s[28:31], s91 offen offset:2048 sc0 sc1\n" \
  "v_mfma_i32_16x16x64_i8 v[208:211], v[96:99], a[8:11], v[208:211]\n" \
  "ds_read_b64_tr_b8 v[124:125], v39 offset:8192\n" \
  "v_mfma_i32_16x16x64_i8 v[208:211], v[100:103], a[24:27], v[208:211]\n" \
  "ds_read_b64_tr_b8 v[126:127], v39 offset:9216\n" \
  "v_mfma_i32_16x16x64_i8 v[240:243], v[96:99], a[12:15], v[240:243]\n" \
  "v_mfma_i32_16x16x64_i8 v[240:243], v[100:103], a[28:31], v[240:243]\n" \
  "buffer_load_dwordx4 a[124:127], v91, s[28:31], s91 offen offset:3072 sc0 sc1\n" \
  "s_waitcnt lgkmcnt(8)\n" \
  "v_mfma_i32_16x16x64_i8 v[148:151], v[104:107], a[0:3], v[148:151]\n" \
  "ds_read_b64_tr_b8 v[96:97], v32 offset:16384\n" \
  "v_mfma_i32_16x16x64_i8 v[148:151], v[108:111], a[16:19], v[148:151]\n" \
  "ds_read_b64_tr_b8 v[98:99], v32 offset:17408\n" \
  "v_mfma_i32_16x16x64_i8 v[180:183], v[104:107], a[4:7], v[180:183]\n" \
  "v_mfma_i32_16x16x64_i8 v[180:183], v[108:111], a[20:23], v[180:183]\n" \
  "buffer_load_dwordx4 a[128:131], v44, s[28:31], s92 offen offset:0 sc0 sc1\n" \
  "v_mfma_i32_16x16x64_i8 v[212:215], v[104:107], a[8:11], v[212:215]\n" \
  "ds_read_b64_tr_b8 v[100:101], v32 offset:24576\n" \
  "v_mfma_i32_16x16x64_i8 v[212:215], v[108:111], a[24:27], v[212:215]\n" \
  "ds_read_b64_tr_b8 v[102:103], v32 offset:25600\n" \
  "v_mfma_i32_16x16x64_i8 v[244:247], v[104:107], a[12:15], v[244:247]\n" \
  "v_mfma_i32_16x16x64_i8 v[244:247], v[108:111], a[28:31], v[244:247]\n" \
  "s_waitcnt lgkmcnt(8)\n" \
  "v_mfma_i32_16x16x64_i8 v[152:155], v[112:115], a[0:3], v[152:155]\n" \
  "ds_read_b64_tr_b8 v[104:105], v33 offset:16384\n" \
  "v_mfma_i32_16x16x64_i8 v[152:155], v[116:119], a[16:19], v[152:155]\n" \
  "ds_read_b64_tr_b8 v[106:107], v33 offset:17408\n" \
  "v_mfma_i32_16x16x64_i8 v[184:187], v[112:115], a[4:7], v[184:187]\n" \
  "v_mfma_i32_16x16x64_i8 v[184:187], v[116:119], a[20:23], v[184:187]\n" \
  "buffer_load_dwordx4 a[132:135], v44, s[28:31], s92 offen offset:1024 sc0 sc1\n" \
  "v_mfma_i32_16x16x64_i8 v[216:219], v[112:115], a[8:11], v[216:219]\n" \
  "ds_read_b64_tr_b8 v[108:109], v33 offset:24576\n" \
  "v_mfma_i32_16x16x64_i8 v[216:219], v[116:119], a[24:27], v[216:219]\n" \
  "ds_read_b64_tr_b8 v[110:111], v33 offset:25600\n" \
  "v_mfma_i32_16x16x64_i8 v[248:251], v[112:115], a[12:15], v[248:251]\n" \
  "v_mfma_i32_16x16x64_i8 v[248:251], v[116:119], a[28:31], v[248:251]\n" \
  "buffer_load_dwordx4 a[136:139], v44, s[28:31], s92 offen offset:2048 sc0 sc1\n" \
  "s_waitcnt lgkmcnt(8)\n" \
  "v_mfma_i32_16x16x64_i8 v[156:159], v[120:123], a[0:3], v[156:159]\n" \
  "ds_read_b64_tr_b8 v[112:113], v34 offset:16384\n" \
  "v_mfma_i32_16x16x64_i8 v[156:159], v[124:127], a[16:19], v[156:159]\n" \
  "ds_read_b64_tr_b8 v[114:115], v34 offset:17408\n" \
  "v_mfma_i32_16x16x64_i8 v[188:191], v[120:123], a[4:7], v[188:191]\n" \
  "v_mfma_i32_16x16x64_i8 v[188:191], v[124:127], a[20:23], v[188:191]\n" \
  "buffer_load_dwordx4 a[140:143], v44, s[28:31], s92 offen offset:3072 sc0 sc1\n" \
  "v_mfma_i32_16x16x64_i8 v[220:223], v[120:123], a[8:11], v[220:223]\n" \
  "ds_read_b64_tr_b8 v[116:117], v34 offset:24576\n" \
  "v_mfma_i32_16x16x64_i8 v[220:223], v[124:127], a[24:27], v[220:223]\n" \
  "ds_read_b64_tr_b8 v[118:119], v34 offset:25600\n" \
  "v_mfma_i32_16x16x64_i8 v[252:255], v[120:123], a[12:15], v[252:255]\n" \
  "v_mfma_i32_16x16x64_i8 v[252:255], v[124:127], a[28:31], v[252:255]\n" \
  "; ---- step 1\n" \
  "s_waitcnt vmcnt(21)\n" \
  "s_waitcnt lgkmcnt(8)\n" \
  "v_mfma_i32_16x16x64_i8 v[128:131], v[96:99], a[32:35], v[128:131]\n" \
  "ds_read_b64_tr_b8 v[120:121], v35 offset:16384\n" \
  "v_mfma_i32_16x16x64_i8 v[128:131], v[100:103], a[48:51], v[128:131]\n" \
  "ds_read_b64_tr_b8 v[122:123], v35 offset:17408\n" \
  "v_mfma_i32_16x16x64_i8 v[160:163], v[96:99], a[36:39], v[160:163]\n" \
  "v_mfma_i32_16x16x64_i8 v[160:163], v[100:103], a[52:55], v[160:163]\n" \
  "buffer_load_dwordx4 a[144:147], v91, s[28:31], s92 offen offset:0 sc0 sc1\n" \
  "v_mfma_i32_16x16x64_i8 v[192:195], v[96:99], a[40:43], v[192:195]\n" \
  "ds_read_b64_tr_b8 v[124:125], v35 offset:24576\n" \
  "v_mfma_i32_16x16x64_i8 v[192:195], v[100:103], a[56:59], v[192:195]\n" \
  "ds_read_b64_tr_b8 v[126:127], v35 offset:25600\n" \
  "v_mfma_i32_16x16x64_i8 v[224:227], v[96:99], a[44:47], v[224:227]\n" \
  "v_mfma_i32_16x16x64_i8 v[224:227], v[100:103], a[60:63], v[224:227]\n" \
  "buffer_load_dwordx4 a[148:151], v91, s[28:31], s92 offen offset:1024 sc0 sc1\n" \
  "s_waitcnt lgkmcnt(8)\n" \
  "v_mfma_i32_16x16x64_i8 v[132:135], v[104:107], a[32:35], v[132:135]\n" \
  "ds_read_b64_tr_b8 v[96:97], v36 offset:16384\n" \
  "v_mfma_i32_16x16x64_i8 v[132:135], v[108:111], a[48:51], v[132:135]\n" \
  "ds_read_b64_tr_b8 v[98:99], v36 offset:17408\n" \
  "v_mfma_i32_16x16x64_i8 v[164:167], v[104:107], a[36:39], v[164:167]\n" \
  "v_mfma_i32_16x16x64_i8 v[164:167], v[108:111], a[52:55], v[164:167]\n" \
  "buffer_load_dwordx4 a[152:155], v91, s[28:31], s92 offen offset:2048 sc0 sc1\n" \
  "v_mfma_i32_16x16x64_i8 v[196:199], v[104:107], a[40:43], v[196:199]\n" \
  "ds_read_b64_tr_b8 v[100:101], v36 offset:24576\n" \
  "v_mfma_i32_16x16x64_i8 v[196:199], v[108:111], a[56:59], v[196:199]\n" \
  "ds_read_b64_tr_b8 v[102:103], v36 offset:25600\n" \
  "v_mfma_i32_16x16x64_i8 v[228:231], v[104:107], a[44:47], v[228:231]\n" \
  "v_mfma_i32_16x16x64_i8 v[228:231], v[108:111], a[60:63], v[228:231]\n" \
  "buffer_load_dwordx4 a[156:159], v91, s[28:31], s92 offen offset:3072 sc0 sc1\n" \
  "s_waitcnt lgkmcnt(8)\n" \
  "v_mfma_i32_16x16x64_i8 v[136:139], v[112:115], a[32:35], v[136:139]\n" \
  "ds_read_b64_tr_b8 v[104:105], v37 offset:16384\n" \
  "v_mfma_i32_16x16x64_i8 v[136:139], v[116:119], a[48:51], v[136:139]\n" \
  "ds_read_b64_tr_b8 v[106:107], v37 offset:17408\n" \
  "v_mfma_i32_16x16x64_i8 v[168:171], v[112:115], a[36:39], v[168:171]\n" \
  "v_mfma_i32_16x16x64_i8 v[168:171], v[116:119], a[52:55], v[168:171]\n" \
  "buffer_load_dwordx4 a[160:163], v44, s[28:31], s93 offen offset:0 sc0 sc1\n" \
  "v_mfma_i32_16x16x64_i8 v[200:203], v[112:115], a[40:43], v[200:203]\n" \
  "ds_read_b64_tr_b8 v[108:109], v37 offset:24576\n" \
  "v_mfma_i32_16x16x64_i8 v[200:203], v[116:119], a[56:59], v[200:203]\n" \
  "ds_read_b64_tr_b8 v[110:111], v37 offset:25600\n" \
  "v_mfma_i32_16x16x64_i8 v[232:235], v[112:115], a[44:47], v[232:235]\n" \
  "v_mfma_i32_16x16x64_i8 v[232:235], v[116:119], a[60:63], v[232:235]\n" \
  "buffer_load_dwordx4 a[164:167], v44, s[28:31], s93 offen offset:1024 sc0 sc1\n" \
  "s_waitcnt lgkmcnt(8)\n" \
  "v_mfma_i32_16x16x64_i8 v[140:143], v[120:123], a[32:35], v[140:143]\n" \
  "ds_read_b64_tr_b8 v[112:113], v38 offset:16384\n" \
  "v_mfma_i32_16x16x64_i8 v[140:143], v[124:127], a[48:51], v[140:143]\n" \
  "ds_read_b64_tr_b8 v[114:115], v38 offset:17408\n" \
  "v_mfma_i32_16x16x64_i8 v[172:175], v[120:123], a[36:39], v[172:175]\n" \
  "v_mfma_i32_16x16x64_i8 v[172:175], v[124:127], a[52:55], v[172:175]\n" \
  "buffer_load_dwordx4 a[168:171], v44, s[28:31], s93 offen offset:2048 sc0 sc1\n" \
  "s_waitcnt vmcnt(44)\n" \
  "s_barrier\n" \
  "v_mfma_i32_16x16x64_i8 v[204:207], v[120:123], a[40:43], v[204:207]\n" \
  "ds_read_b64_tr_b8 v[116:117], v38 offset:24576\n" \
  "v_mfma_i32_16x16x64_i8 v[204:207], v[124:127], a[56:59], v[204:207]\n" \
  "ds_read_b64_tr_b8 v[118:119], v38 offset:25600\n" \
  "v_mfma_i32_16x16x64_i8 v[236:239], v[120:123], a[44:47], v[236:239]\n" \
  "v_mfma_i32_16x16x64_i8 v[236:239], v[124:127], a[60:63], v[236:239]\n" \
  "buffer_load_dwordx4 a[172:175], v44, s[28:31], s93 offen offset:3072 sc0 sc1\n" \
  "s_waitcnt lgkmcnt(8)\n" \
  "v_mfma_i32_16x16x64_i8 v[144:147], v[96:99], a[32:35], v[144:147]\n" \
  "ds_read_b64_tr_b8 v[120:121], v39 offset:16384\n" \
  "v_mfma_i32_16x16x64_i8 v[144:147], v[100:103], a[48:51], v[144:147]\n" \
  "ds_read_b64_tr_b8 v[122:123], v39 offset:17408\n" \
  "v_mfma_i32_16x16x64_i8 v[176:179], v[96:99], a[36:39], v[176:179]\n" \
  "v_mfma_i32_16x16x64_i8 v[176:179], v[100:103], a[52:55], v[176:179]\n" \
  "v_mfma_i32_16x16x64_i8 v[208:211], v[96:99], a[40:43], v[208:211]\n" \
  "ds_read_b64_tr_b8 v[124:125], v39 offset:24576\n" \
  "v_mfma_i32_16x16x64_i8 v[208:211], v[100:103], a[56:59], v[208:211]\n" \
  "ds_read_b64_tr_b8 v[126:127], v39 offset:25600\n" \
  "v_mfma_i32_16x16x64_i8 v[240:243], v[96:99], a[44:47], v[240:243]\n" \
  "v_mfma_i32_16x16x64_i8 v[240:243], v[100:103], a[60:63], v[240:243]\n" \
  "s_waitcnt lgkmcnt(8)\n" \
  "v_mfma_i32_16x16x64_i8 v[148:151], v[104:107], a[32:35], v[148:151]\n" \
  "ds_read_b64_tr_b8 v[96:97], v32 offset:32768\n" \
  "v_mfma_i32_16x16x64_i8 v[148:151], v[108:111], a[48:51], v[148:151]\n" \
  "ds_read_b64_tr_b8 v[98:99], v32 offset:33792\n" \
  "v_mfma_i32_16x16x64_i8 v[180:183], v[104:107], a[36:39], v[180:183]\n" \
  "v_mfma_i32_16x16x64_i8 v[180:183], v[108:111], a[52:55], v[180:183]\n" \
  "v_mfma_i32_16x16x64_i8 v[212:215], v[104:107], a[40:43], v[212:215]\n" \
  "ds_read_b64_tr_b8 v[100:101], v32 offset:40960\n" \
  "v_mfma_i32_16x16x64_i8 v[212:215], v[108:111], a[56:59], v[212:215]\n" \
  "ds_read_b64_tr_b8 v[102:103], v32 offset:41984\n" \
  "v_mfma_i32_16x16x64_i8 v[244:247], v[104:107], a[44:47], v[244:247]\n" \
  "v_mfma_i32_16x16x64_i8 v[244:247], v[108:111], a[60:63], v[244:247]\n" \
  "s_waitcnt lgkmcnt(8)\n" \
  "v_mfma_i32_16x16x64_i8 v[152:155], v[112:115], a[32:35], v[152:155]\n" \
  "ds_read_b64_tr_b8 v[104:105], v33 offset:32768\n" \
  "v_mfma_i32_16x16x64_i8 v[152:155], v[116:119], a[48:51], v[152:155]\n" \
  "ds_read_b64_tr_b8 v[106:107], v33 offset:33792\n" \
  "v_mfma_i32_16x16x64_i8 v[184:187], v[112:115], a[36:39], v[184:187]\n" \
  "v_mfma_i32_16x16x64_i8 v[184:187], v[116:119], a[52:55], v[184:187]\n" \
  "v_mfma_i32_16x16x64_i8 v[216:219], v[112:115], a[40:43], v[216:219]\n" \
  "ds_read_b64_tr_b8 v[108:109], v33 offset:40960\n" \
  "v_mfma_i32_16x16x64_i8 v[216:219], v[116:119], a[56:59], v[216:219]\n" \
  "ds_read_b64_tr_b8 v[110:111], v33 offset:41984\n" \
  "v_mfma_i32_16x16x64_i8 v[248:251], v[112:115], a[44:47], v[248:251]\n" \
  "v_mfma_i32_16x16x64_i8 v[248:251], v[116:119], a[60:63], v[248:251]\n" \
  "s_waitcnt lgkmcnt(8)\n" \
  "v_mfma_i32_16x16x64_i8 v[156:159], v[120:123], a[32:35], v[156:159]\n" \
  "ds_read_b64_tr_b8 v[112:113], v34 offset:32768\n" \
  "v_mfma_i32_16x16x64_i8 v[156:159], v[124:127], a[48:51], v[156:159]\n" \
  "ds_read_b64_tr_b8 v[114:115], v34 offset:33792\n" \
  "v_mfma_i32_16x16x64_i8 v[188:191], v[120:123], a[36:39], v[188:191]\n" \
  "v_mfma_i32_16x16x64_i8 v[188:191], v[124:127], a[52:55], v[188:191]\n" \
  "v_mfma_i32_16x16x64_i8 v[220:223], v[120:123], a[40:43], v[220:223]\n" \
  "ds_read_b64_tr_b8 v[116:117], v34 offset:40960\n" \
  "v_mfma_i32_16x16x64_i8 v[220:223], v[124:127], a[56:59], v[220:223]\n" \
  "ds_read_b64_tr_b8 v[118:119], v34 offset:41984\n" \
  "v_mfma_i32_16x16x64_i8 v[252:255], v[120:123], a[44:47], v[252:255]\n" \
  "v_mfma_i32_16x16x64_i8 v[252:255], v[124:127], a[60:63], v[252:255]\n" \
  "; ---- step 2\n" \
  "s_waitcnt vmcnt(20)\n" \
  "v_mul_f32 v88, v77, v78\n" \
  "v_cmp_eq_u32 vcc, v90, v76\n" \
  "s_nop 1\n" \
  "s_cmp_lg_u32 s55, 0\n" \
  "s_cselect_b64 vcc, -1, vcc\n" \
  "v_mov_b32 v89, 0x7fc00000\n" \
  "s_nop 1\n" \
  "v_cndmask_b32 v88, v88, v89, vcc\n" \
  "v_mov_b32 v89, v88\n" \
  "s_waitcnt vmcnt(21)\n" \
  "s_waitcnt lgkmcnt(8)\n" \
  "v_mfma_i32_16x16x64_i8 v[128:131], v[96:99], a[64:67], v[128:131]\n" \
  "ds_read_b64_tr_b8 v[120:121], v35 offset:32768\n" \
  "v_mfma_i32_16x16x64_i8 v[128:131], v[100:103], a[80:83], v[128:131]\n" \
  "ds_read_b64_tr_b8 v[122:123], v35 offset:33792\n" \
  "v_mfma_i32_16x16x64_i8 v[160:163], v[96:99], a[68:71], v[160:163]\n" \
  "v_mfma_i32_16x16x64_i8 v[160:163], v[100:103], a[84:87], v[160:163]\n" \
  "v_mfma_i32_16x16x64_i8 v[192:195], v[96:99], a[72:75], v[192:195]\n" \
  "ds_read_b64_tr_b8 v[124:125], v35 offset:40960\n" \
  "v_mfma_i32_16x16x64_i8 v[192:195], v[100:103], a[88:91], v[192:195]\n" \
  "ds_read_b64_tr_b8 v[126:127], v35 offset:41984\n" \
  "v_mfma_i32_16x16x64_i8 v[224:227], v[96:99], a[76:79], v[224:227]\n" \
  "v_mfma_i32_16x16x64_i8 v[224:227], v[100:103], a[92:95], v[224:227]\n" \
  "s_waitcnt lgkmcnt(8)\n" \
  "v_mfma_i32_16x16x64_i8 v[132:135], v[104:107], a[64:67], v[132:135]\n" \
  "ds_read_b64_tr_b8 v[96:97], v36 offset:32768\n" \
  "v_mfma_i32_16x16x64_i8 v[132:135], v[108:111], a[80:83], v[132:135]\n" \
  "ds_read_b64_tr_b8 v[98:99], v36 offset:33792\n" \
  "v_mfma_i32_16x16x64_i8 v[164:167], v[104:107], a[68:71], v[164:167]\n" \
  "v_mfma_i32_16x16x64_i8 v[164:167], v[108:111], a[84:87], v[164:167]\n" \
  "v_mfma_i32_16x16x64_i8 v[196:199], v[104:107], a[72:75], v[196:199]\n" \
  "ds_read_b64_tr_b8 v[100:101], v36 offset:40960\n" \
  "v_mfma_i32_16x16x64_i8 v[196:199], v[108:111], a[88:91], v[196:199]\n" \
  "ds_read_b64_tr_b8 v[102:103], v36 offset:41984\n" \
  "v_mfma_i32_16x16x64_i8 v[228:231], v[104:107], a[76:79], v[228:231]\n" \
  "v_mfma_i32_16x16x64_i8 v[228:231], v[108:111], a[92:95], v[228:231]\n" \
  "s_waitcnt lgkmcnt(8)\n" \
  "v_mfma_i32_16x16x64_i8 v[136:139], v[112:115], a[64:67], v[136:139]\n" \
  "ds_read_b64_tr_b8 v[104:105], v37 offset:32768\n" \
  "v_mfma_i32_16x16x64_i8 v[136:139], v[116:119], a[80:83], v[136:139]\n" \
  "ds_read_b64_tr_b8 v[106:107], v37 offset:33792\n" \
  "v_mfma_i32_16x16x64_i8 v[168:171], v[112:115], a[68:71], v[168:171]\n" \
  "v_mfma_i32_16x16x64_i8 v[168:171], v[116:119], a[84:87], v[168:171]\n" \
  "v_mfma_i32_16x16x64_i8 v[200:203], v[112:115], a[72:75], v[200:203]\n" \
  "ds_read_b64_tr_b8 v[108:109], v37 offset:40960\n" \
  "v_mfma_i32_16x16x64_i8 v[200:203], v[116:119], a[88:91], v[200:203]\n" \
  "ds_read_b64_tr_b8 v[110:111], v37 offset:41984\n" \
  "v_mfma_i32_16x16x64_i8 v[232:235], v[112:115], a[76:79], v[232:235]\n" \
  "v_mfma_i32_16x16x64_i8 v[232:235], v[116:119], a[92:95], v[232:235]\n" \
  "s_waitcnt lgkmcnt(8)\n" \
  "v_mfma_i32_16x16x64_i8 v[140:143], v[120:123], a[64:67], v[140:143]\n" \
  "ds_read_b64_tr_b8 v[112:113], v38 offset:32768\n" \
  "v_mfma_i32_16x16x64_i8 v[140:143], v[124:127], a[80:83], v[140:143]\n" \
  "ds_read_b64_tr_b8 v[114:115], v38 offset:33792\n" \
  "v_mfma_i32_16x16x64_i8 v[172:175], v[120:123], a[68:71], v[172:175]\n" \
  "v_mfma_i32_16x16x64_i8 v[172:175], v[124:127], a[84:87], v[172:175]\n" \
  "s_waitcnt vmcnt(33)\n" \
  "s_barrier\n" \
  "v_mfma_i32_16x16x64_i8 v[204:207], v[120:123], a[72:75], v[204:207]\n" \
  "ds_read_b64_tr_b8 v[116:117], v38 offset:40960\n" \
  "v_mfma_i32_16x16x64_i8 v[204:207], v[124:127], a[88:91], v[204:207]\n" \
  "ds_read_b64_tr_b8 v[118:119], v38 offset:41984\n" \
  "v_mfma_i32_16x16x64_i8 v[236:239], v[120:123], a[76:79], v[236:239]\n" \
  "v_mfma_i32_16x16x64_i8 v[236:239], v[124:127], a[92:95], v[236:239]\n" \
  "s_waitcnt lgkmcnt(8)\n" \
  "v_mfma_i32_16x16x64_i8 v[144:147], v[96:99], a[64:67], v[144:147]\n" \
  "ds_read_b64_tr_b8 v[120:121], v39 offset:32768\n" \
  "v_mfma_i32_16x16x64_i8 v[144:147], v[100:103], a[80:83], v[144:147]\n" \
  "ds_read_b64_tr_b8 v[122:123], v39 offset:33792\n" \
  "v_mfma_i32_16x16x64_i8 v[176:179], v[96:99], a[68:71], v[176:179]\n" \
  "v_mfma_i32_16x16x64_i8 v[176:179], v[100:103], a[84:87], v[176:179]\n" \
  "v_mfma_i32_16x16x64_i8 v[208:211], v[96:99], a[72:75], v[208:211]\n" \
  "ds_read_b64_tr_b8 v[124:125], v39 offset:40960\n" \
  "v_mfma_i32_16x16x64_i8 v[208:211], v[100:103], a[88:91], v[208:211]\n" \
  "ds_read_b64_tr_b8 v[126:127], v39 offset:41984\n" \
  "v_mfma_i32_16x16x64_i8 v[240:243], v[96:99], a[76:79], v[240:243]\n" \
  "v_readlane_b32 s50, v46, s49\n" \
  "v_mfma_i32_16x16x64_i8 v[240:243], v[100:103], a[92:95], v[240:243]\n" \
  "s_waitcnt lgkmcnt(8)\n" \
  "v_mfma_i32_16x16x64_i8 v[148:151], v[104:107], a[64:67], v[148:151]\n" \
  "ds_read_b64_tr_b8 v[96:97], v32 offset:49152\n" \
  "v_mfma_i32_16x16x64_i8 v[148:151], v[108:111], a[80:83], v[148:151]\n" \
  "ds_read_b64_tr_b8 v[98:99], v32 offset:50176\n" \
  "v_mfma_i32_16x16x64_i8 v[180:183], v[104:107], a[68:71], v[180:183]\n" \
  "v_mfma_i32_16x16x64_i8 v[180:183], v[108:111], a[84:87], v[180:183]\n" \
  "v_mfma_i32_16x16x64_i8 v[212:215], v[104:107], a[72:75], v[212:215]\n" \
  "ds_read_b64_tr_b8 v[100:101], v32 offset:57344\n" \
  "v_mfma_i32_16x16x64_i8 v[212:215], v[108:111], a[88:91], v[212:215]\n" \
  "ds_read_b64_tr_b8 v[102:103], v32 offset:58368\n" \
  "v_mfma_i32_16x16x64_i8 v[244:247], v[104:107], a[76:79], v[244:247]\n" \
  "s_lshl_b32 s86, s50, 21\n" \
  "s_lshl_b32 s94, s50, 16\n" \
  "s_add_u32 s49, s49, 1\n" \
  "v_mfma_i32_16x16x64_i8 v[244:247], v[108:111], a[92:95], v[244:247]\n" \
  "s_waitcnt lgkmcnt(8)\n" \
  "v_mfma_i32_16x16x64_i8 v[152:155], v[112:115], a[64:67], v[152:155]\n" \
  "ds_read_b64_tr_b8 v[104:105], v33 offset:49152\n" \
  "v_mfma_i32_16x16x64_i8 v[152:155], v[116:119], a[80:83], v[152:155]\n" \
  "ds_read_b64_tr_b8 v[106:107], v33 offset:50176\n" \
  "v_mfma_i32_16x16x64_i8 v[184:187], v[112:115], a[68:71], v[184:187]\n" \
  "v_readlane_b32 s50, v46, s49\n" \
  "v_mfma_i32_16x16x64_i8 v[184:187], v[116:119], a[84:87], v[184:187]\n" \
  "v_mfma_i32_16x16x64_i8 v[216:219], v[112:115], a[72:75], v[216:219]\n" \
  "ds_read_b64_tr_b8 v[108:109], v33 offset:57344\n" \
  "v_mfma_i32_16x16x64_i8 v[216:219], v[116:119], a[88:91], v[216:219]\n" \
  "ds_read_b64_tr_b8 v[110:111], v33 offset:58368\n" \
  "v_mfma_i32_16x16x64_i8 v[248:251], v[112:115], a[76:79], v[248:251]\n" \
  "v_mfma_i32_16x16x64_i8 v[248:251], v[116:119], a[92:95], v[248:251]\n" \
  "s_waitcnt lgkmcnt(8)\n" \
  "v_mfma_i32_16x16x64_i8 v[156:159], v[120:123], a[64:67], v[156:159]\n" \
  "ds_read_b64_tr_b8 v[112:113], v80 offset:0\n" \
  "v_mfma_i32_16x16x64_i8 v[156:159], v[124:127], a[80:83], v[156:159]\n" \
  "ds_read_b64_tr_b8 v[114:115], v80 offset:1024\n" \
  "v_mfma_i32_16x16x64_i8 v[188:191], v[120:123], a[68:71], v[188:191]\n" \
  "s_lshl_b32 s87, s50, 21\n" \
  "s_lshl_b32 s95, s50, 16\n" \
  "s_add_u32 s49, s49, 1\n" \
  "v_mfma_i32_16x16x64_i8 v[188:191], v[124:127], a[84:87], v[188:191]\n" \
  "v_mfma_i32_16x16x64_i8 v[220:223], v[120:123], a[72:75], v[220:223]\n" \
  "ds_read_b64_tr_b8 v[116:117], v80 offset:8192\n" \
  "v_mfma_i32_16x16x64_i8 v[220:223], v[124:127], a[88:91], v[220:223]\n" \
  "ds_read_b64_tr_b8 v[118:119], v80 offset:9216\n" \
  "v_mfma_i32_16x16x64_i8 v[252:255], v[120:123], a[76:79], v[252:255]\n" \
  "v_mfma_i32_16x16x64_i8 v[252:255], v[124:127], a[92:95], v[252:255]\n" \
  "; ---- step 3\n" \
  "s_waitcnt vmcnt(12)\n" \
  "s_waitcnt lgkmcnt(8)\n" \
  "v_mfma_i32_16x16x64_i8 v[128:131], v[96:99], a[96:99], v[128:131]\n" \
  "ds_read_b64_tr_b8 v[120:121], v81 offset:0\n" \
  "v_mfma_i32_16x16x64_i8 v[128:131], v[100:103], a[112:115], v[128:131]\n" \
  "ds_read_b64_tr_b8 v[122:123], v81 offset:1024\n" \
  "buffer_load_dwordx4 a[176:179], v91, s[28:31], s93 offen offset:0 sc0 sc1\n" \
  "v_mfma_i32_16x16x64_i8 v[160:163], v[96:99], a[100:103], v[160:163]\n" \
  "v_mfma_i32_16x16x64_i8 v[160:163], v[100:103], a[116:119], v[160:163]\n" \
  "buffer_load_dwordx4 a[180:183], v91, s[28:31], s93 offen offset:1024 sc0 sc1\n" \
  "v_mfma_i32_16x16x64_i8 v[192:195], v[96:99], a[104:107], v[192:195]\n" \
  "ds_read_b64_tr_b8 v[124:125], v81 offset:8192\n" \
  "v_mfma_i32_16x16x64_i8 v[192:195], v[100:103], a[120:123], v[192:195]\n" \
  "ds_read_b64_tr_b8 v[126:127], v81 offset:9216\n" \
  "buffer_load_dwordx4 a[184:187], v91, s[28:31], s93 offen offset:2048 sc0 sc1\n" \
  "v_mfma_i32_16x16x64_i8 v[224:227], v[96:99], a[108:111], v[224:227]\n" \
  "v_mfma_i32_16x16x64_i8 v[224:227], v[100:103], a[124:127], v[224:227]\n" \
  "buffer_load_dwordx4 a[188:191], v91, s[28:31], s93 offen offset:3072 sc0 sc1\n" \
  "s_waitcnt lgkmcnt(8)\n" \
  "v_mfma_i32_16x16x64_i8 v[132:135], v[104:107], a[96:99], v[132:135]\n" \
  "ds_read_b64_tr_b8 v[96:97], v80 offset:16384\n" \
  "v_mfma_i32_16x16x64_i8 v[132:135], v[108:111], a[112:115], v[132:135]\n" \
  "ds_read_b64_tr_b8 v[98:99], v80 offset:17408\n" \
  "buffer_load_dwordx4 a[192:195], v44, s[28:31], s94 offen offset:0 sc0 sc1\n" \
  "v_mfma_i32_16x16x64_i8 v[164:167], v[104:107], a[100:103], v[164:167]\n" \
  "v_mfma_i32_16x16x64_i8 v[164:167], v[108:111], a[116:119], v[164:167]\n" \
  "buffer_load_dwordx4 a[196:199], v44, s[28:31], s94 offen offset:1024 sc0 sc1\n" \
  "v_mfma_i32_16x16x64_i8 v[196:199], v[104:107], a[104:107], v[196:199]\n" \
  "ds_read_b64_tr_b8 v[100:101], v80 offset:24576\n" \
  "v_mfma_i32_16x16x64_i8 v[196:199], v[108:111], a[120:123], v[196:199]\n" \
  "ds_read_b64_tr_b8 v[102:103], v80 offset:25600\n" \
  "buffer_load_dwordx4 a[200:203], v44, s[28:31], s94 offen offset:2048 sc0 sc1\n" \
  "v_mfma_i32_16x16x64_i8 v[228:231], v[104:107], a[108:111], v[228:231]\n" \
  "v_mfma_i32_16x16x64_i8 v[228:231], v[108:111], a[124:127], v[228:231]\n" \
  "buffer_load_dwordx4 a[204:207], v44, s[28:31], s94 offen offset:3072 sc0 sc1\n" \
  "s_waitcnt vmcnt(12)\n" \
  "s_waitcnt lgkmcnt(8)\n" \
  "v_mfma_i32_16x16x64_i8 v[128:131], v[112:115], a[128:131], v[128:131]\n" \
  "ds_read_b64_tr_b8 v[104:105], v81 offset:16384\n" \
  "v_mfma_i32_16x16x64_i8 v[128:131], v[116:119], a[144:147], v[128:131]\n" \
  "ds_read_b64_tr_b8 v[106:107], v81 offset:17408\n" \
  "buffer_load_dwordx4 a[208:211], v91, s[28:31], s94 offen offset:0 sc0 sc1\n" \
  "v_mfma_i32_16x16x64_i8 v[160:163], v[112:115], a[132:135], v[160:163]\n" \
  "v_mfma_i32_16x16x64_i8 v[160:163], v[116:119], a[148:151], v[160:163]\n" \
  "buffer_load_dwordx4 a[212:215], v91, s[28:31], s94 offen offset:1024 sc0 sc1\n" \
  "v_mfma_i32_16x16x64_i8 v[192:195], v[112:115], a[136:139], v[192:195]\n" \
  "ds_read_b64_tr_b8 v[108:109], v81 offset:24576\n" \
  "v_mfma_i32_16x16x64_i8 v[192:195], v[116:119], a[152:155], v[192:195]\n" \
  "ds_read_b64_tr_b8 v[110:111], v81 offset:25600\n" \
  "buffer_load_dwordx4 a[216:219], v91, s[28:31], s94 offen offset:2048 sc0 sc1\n" \
  "v_mfma_i32_16x16x64_i8 v[224:227], v[112:115], a[140:143], v[224:227]\n" \
  "v_mfma_i32_16x16x64_i8 v[224:227], v[116:119], a[156:159], v[224:227]\n" \
  "buffer_load_dwordx4 a[220:223], v91, s[28:31], s94 offen offset:3072 sc0 sc1\n" \
  "s_waitcnt lgkmcnt(8)\n" \
  "v_mfma_i32_16x16x64_i8 v[132:135], v[120:123], a[128:131], v[132:135]\n" \
  "ds_read_b64_tr_b8 v[112:113], v24 offset:0\n" \
  "v_mfma_i32_16x16x64_i8 v[132:135], v[124:127], a[144:147], v[132:135]\n" \
  "ds_read_b64_tr_b8 v[114:115], v24 offset:1024\n" \
  "buffer_load_dwordx4 a[224:227], v44, s[28:31], s95 offen offset:0 sc0 sc1\n" \
  "v_mfma_i32_16x16x64_i8 v[164:167], v[120:123], a[132:135], v[164:167]\n" \
  "v_mfma_i32_16x16x64_i8 v[164:167], v[124:127], a[148:151], v[164:167]\n" \
  "buffer_load_dwordx4 a[228:231], v44, s[28:31], s95 offen offset:1024 sc0 sc1\n" \
  "v_mfma_i32_16x16x64_i8 v[196:199], v[120:123], a[136:139], v[196:199]\n" \
  "ds_read_b64_tr_b8 v[116:117], v24 offset:8192\n" \
  "v_mfma_i32_16x16x64_i8 v[196:199], v[124:127], a[152:155], v[196:199]\n" \
  "ds_read_b64_tr_b8 v[118:119], v24 offset:9216\n" \
  "buffer_load_dwordx4 a[232:235], v44, s[28:31], s95 offen offset:2048 sc0 sc1\n" \
  "v_mfma_i32_16x16x64_i8 v[228:231], v[120:123], a[140:143], v[228:231]\n" \
  "v_mfma_i32_16x16x64_i8 v[228:231], v[124:127], a[156:159], v[228:231]\n" \
  "buffer_load_dwordx4 a[236:239], v44, s[28:31], s95 offen offset:3072 sc0 sc1\n" \
  "s_waitcnt vmcnt(12)\n" \
  "s_waitcnt lgkmcnt(8)\n" \
  "v_mfma_i32_16x16x64_i8 v[128:131], v[96:99], a[160:163], v[128:131]\n" \
  "ds_read_b64_tr_b8 v[120:121], v25 offset:0\n" \
  "v_mfma_i32_16x16x64_i8 v[128:131], v[100:103], a[176:179], v[128:131]\n" \
  "ds_read_b64_tr_b8 v[122:123], v25 offset:1024\n" \
  "buffer_load_dwordx4 a[240:243], v91, s[28:31], s95 offen offset:0 sc0 sc1\n" \
  "v_mfma_i32_16x16x64_i8 v[160:163], v[96:99], a[164:167], v[160:163]\n" \
  "v_mfma_i32_16x16x64_i8 v[160:163], v[100:103], a[180:183], v[160:163]\n" \
  "buffer_load_dwordx4 a[244:247], v91, s[28:31], s95 offen offset:1024 sc0 sc1\n" \
  "v_mfma_i32_16x16x64_i8 v[192:195], v[96:99], a[168:171], v[192:195]\n" \
  "ds_read_b64_tr_b8 v[124:125], v25 offset:8192\n" \
  "v_mfma_i32_16x16x64_i8 v[192:195], v[100:103], a[184:187], v[192:195]\n" \
  "ds_read_b64_tr_b8 v[126:127], v25 offset:9216\n" \
  "buffer_load_dwordx4 a[248:251], v91, s[28:31], s95 offen offset:2048 sc0 sc1\n" \
  "v_mfma_i32_16x16x64_i8 v[224:227], v[96:99], a[172:175], v[224:227]\n" \
  "v_mfma_i32_16x16x64_i8 v[224:227], v[100:103], a[188:191], v[224:227]\n" \
  "buffer_load_dwordx4 a[252:255], v91, s[28:31], s95 offen offset:3072 sc0 sc1\n" \
  "s_waitcnt lgkmcnt(8)\n" \
  "v_mfma_i32_16x16x64_i8 v[132:135], v[104:107], a[160:163], v[132:135]\n" \
  "ds_read_b64_tr_b8 v[96:97], v24 offset:16384\n" \
  "v_mfma_i32_16x16x64_i8 v[132:135], v[108:111], a[176:179], v[132:135]\n" \
  "ds_read_b64_tr_b8 v[98:99], v24 offset:17408\n" \
  "v_mfma_i32_16x16x64_i8 v[164:167], v[104:107], a[164:167], v[164:167]\n" \
  "v_mfma_i32_16x16x64_i8 v[164:167], v[108:111], a[180:183], v[164:167]\n" \
  "v_mfma_i32_16x16x64_i8 v[196:199], v[104:107], a[168:171], v[196:199]\n" \
  "ds_read_b64_tr_b8 v[100:101], v24 offset:24576\n" \
  "v_mfma_i32_16x16x64_i8 v[196:199], v[108:111], a[184:187], v[196:199]\n" \
  "ds_read_b64_tr_b8 v[102:103], v24 offset:25600\n" \
  "v_mfma_i32_16x16x64_i8 v[228:231], v[104:107], a[172:175], v[228:231]\n" \
  "v_mfma_i32_16x16x64_i8 v[228:231], v[108:111], a[188:191], v[228:231]\n" \
  "s_waitcnt vmcnt(8)\n" \
  "s_waitcnt lgkmcnt(8)\n" \
  "v_mfma_i32_16x16x64_i8 v[128:131], v[112:115], a[192:195], v[128:131]\n" \
  "ds_read_b64_tr_b8 v[104:105], v25 offset:16384\n" \
  "v_mfma_i32_16x16x64_i8 v[128:131], v[116:119], a[208:211], v[128:131]\n" \
  "ds_read_b64_tr_b8 v[106:107], v25 offset:17408\n" \
  "v_mfma_i32_16x16x64_i8 v[160:163], v[112:115], a[196:199], v[160:163]\n" \
  "v_mfma_i32_16x16x64_i8 v[160:163], v[116:119], a[212:215], v[160:163]\n" \
  "v_mfma_i32_16x16x64_i8 v[192:195], v[112:115], a[200:203], v[192:195]\n" \
  "ds_read_b64_tr_b8 v[108:109], v25 offset:24576\n" \
  "v_mfma_i32_16x16x64_i8 v[192:195], v[116:119], a[216:219], v[192:195]\n" \
  "ds_read_b64_tr_b8 v[110:111], v25 offset:25600\n" \
  "v_mfma_i32_16x16x64_i8 v[224:227], v[112:115], a[204:207], v[224:227]\n" \
  "v_mfma_i32_16x16x64_i8 v[224:227], v[116:119], a[220:223], v[224:227]\n" \
  "s_waitcnt lgkmcnt(8)\n" \
  "v_mfma_i32_16x16x64_i8 v[132:135], v[120:123], a[192:195], v[132:135]\n" \
  "ds_read_b64_tr_b8 v[112:113], v34 offset:49152\n" \
  "v_mfma_i32_16x16x64_i8 v[132:135], v[124:127], a[208:211], v[132:135]\n" \
  "ds_read_b64_tr_b8 v[114:115], v34 offset:50176\n" \
  "v_mfma_i32_16x16x64_i8 v[164:167], v[120:123], a[196:199], v[164:167]\n" \
  "v_mfma_i32_16x16x64_i8 v[164:167], v[124:127], a[212:215], v[164:167]\n" \
  "v_mfma_i32_16x16x64_i8 v[196:199], v[120:123], a[200:203], v[196:199]\n" \
  "ds_read_b64_tr_b8 v[116:117], v34 offset:57344\n" \
  "v_mfma_i32_16x16x64_i8 v[196:199], v[124:127], a[216:219], v[196:199]\n" \
  "ds_read_b64_tr_b8 v[118:119], v34 offset:58368\n" \
  "v_mfma_i32_16x16x64_i8 v[228:231], v[120:123], a[204:207], v[228:231]\n" \
  "v_mfma_i32_16x16x64_i8 v[228:231], v[124:127], a[220:223], v[228:231]\n" \
  "; ---- step 4\n" \
  "s_waitcnt vmcnt(0)\n" \
  "s_waitcnt lgkmcnt(8)\n" \
  "v_mfma_i32_16x16x64_i8 v[128:131], v[96:99], a[224:227], v[128:131]\n" \
  "ds_read_b64_tr_b8 v[120:121], v35 offset:49152\n" \
  "v_mfma_i32_16x16x64_i8 v[128:131], v[100:103], a[240:243], v[128:131]\n" \
  "ds_read_b64_tr_b8 v[122:123], v35 offset:50176\n" \
  "v_mfma_i32_16x16x64_i8 v[160:163], v[96:99], a[228:231], v[160:163]\n" \
  "v_mfma_i32_16x16x64_i8 v[160:163], v[100:103], a[244:247], v[160:163]\n" \
  "v_mfma_i32_16x16x64_i8 v[192:195], v[96:99], a[232:235], v[192:195]\n" \
  "ds_read_b64_tr_b8 v[124:125], v35 offset:57344\n" \
  "v_mfma_i32_16x16x64_i8 v[192:195], v[100:103], a[248:251], v[192:195]\n" \
  "ds_read_b64_tr_b8 v[126:127], v35 offset:58368\n" \
  "v_mfma_i32_16x16x64_i8 v[224:227], v[96:99], a[236:239], v[224:227]\n" \
  "v_mfma_i32_16x16x64_i8 v[224:227], v[100:103], a[252:255], v[224:227]\n" \
  "s_waitcnt lgkmcnt(8)\n" \
  "v_mfma_i32_16x16x64_i8 v[132:135], v[104:107], a[224:227], v[132:135]\n" \
  "ds_read_b64_tr_b8 v[96:97], v82 offset:0\n" \
  "v_mfma_i32_16x16x64_i8 v[132:135], v[108:111], a[240:243], v[132:135]\n" \
  "ds_read_b64_tr_b8 v[98:99], v82 offset:1024\n" \
  "v_mfma_i32_16x16x64_i8 v[164:167], v[104:107], a[228:231], v[164:167]\n" \
  "v_mfma_i32_16x16x64_i8 v[164:167], v[108:111], a[244:247], v[164:167]\n" \
  "v_mfma_i32_16x16x64_i8 v[196:199], v[104:107], a[232:235], v[196:199]\n" \
  "ds_read_b64_tr_b8 v[100:101], v82 offset:8192\n" \
  "v_mfma_i32_16x16x64_i8 v[196:199], v[108:111], a[248:251], v[196:199]\n" \
  "ds_read_b64_tr_b8 v[102:103], v82 offset:9216\n" \
  "v_mfma_i32_16x16x64_i8 v[228:231], v[104:107], a[236:239], v[228:231]\n" \
  "v_mfma_i32_16x16x64_i8 v[228:231], v[108:111], a[252:255], v[228:231]\n" \
  "s_waitcnt lgkmcnt(8)\n" \
  "v_mfma_i32_16x16x64_i8 v[136:139], v[112:115], a[96:99], v[136:139]\n" \
  "ds_read_b64_tr_b8 v[104:105], v83 offset:0\n" \
  "v_mfma_i32_16x16x64_i8 v[136:139], v[116:119], a[112:115], v[136:139]\n" \
  "ds_read_b64_tr_b8 v[106:107], v83 offset:1024\n" \
  "v_mfma_i32_16x16x64_i8 v[168:171], v[112:115], a[100:103], v[168:171]\n" \
  "v_mfma_i32_16x16x64_i8 v[168:171], v[116:119], a[116:119], v[168:171]\n" \
  "v_mfma_i32_16x16x64_i8 v[200:203], v[112:115], a[104:107], v[200:203]\n" \
  "ds_read_b64_tr_b8 v[108:109], v83 offset:8192\n" \
  "v_mfma_i32_16x16x64_i8 v[200:203], v[116:119], a[120:123], v[200:203]\n" \
  "ds_read_b64_tr_b8 v[110:111], v83 offset:9216\n" \
  "v_mfma_i32_16x16x64_i8 v[232:235], v[112:115], a[108:111], v[232:235]\n" \
  "v_mfma_i32_16x16x64_i8 v[232:235], v[116:119], a[124:127], v[232:235]\n" \
  "s_waitcnt lgkmcnt(8)\n" \
  "v_mfma_i32_16x16x64_i8 v[140:143], v[120:123], a[96:99], v[140:143]\n" \
  "ds_read_b64_tr_b8 v[112:113], v82 offset:16384\n" \
  "v_cvt_f32_i32 v128, v128\n" \
  "v_mfma_i32_16x16x64_i8 v[140:143], v[124:127], a[112:115], v[140:143]\n" \
  "ds_read_b64_tr_b8 v[114:115], v82 offset:17408\n" \
  "v_cvt_f32_i32 v129, v129\n" \
  "v_mfma_i32_16x16x64_i8 v[172:175], v[120:123], a[100:103], v[172:175]\n" \
  "v_cvt_f32_i32 v130, v130\n" \
  "v_mfma_i32_16x16x64_i8 v[172:175], v[124:127], a[116:119], v[172:175]\n" \
  "v_cvt_f32_i32 v131, v131\n" \
  "v_mfma_i32_16x16x64_i8 v[204:207], v[120:123], a[104:107], v[204:207]\n" \
  "ds_read_b64_tr_b8 v[116:117], v82 offset:24576\n" \
  "v_mfma_i32_16x16x64_i8 v[204:207], v[124:127], a[120:123], v[204:207]\n" \
  "ds_read_b64_tr_b8 v[118:119], v82 offset:25600\n" \
  "v_pk_mul_f32 v[128:129], v[128:129], v[88:89]\n" \
  "v_mfma_i32_16x16x64_i8 v[236:239], v[120:123], a[108:111], v[236:239]\n" \
  "v_pk_mul_f32 v[130:131], v[130:131], v[88:89]\n" \
  "v_mfma_i32_16x16x64_i8 v[236:239], v[124:127], a[124:127], v[236:239]\n" \
  "buffer_store_dwordx4 v[128:131], v47, s[20:23], s76 offen offset:0 sc0 sc1\n" \
  "s_waitcnt lgkmcnt(8)\n" \
  "v_mfma_i32_16x16x64_i8 v[136:139], v[96:99], a[128:131], v[136:139]\n" \
  "ds_read_b64_tr_b8 v[120:121], v83 offset:16384\n" \
  "v_mfma_i32_16x16x64_i8 v[136:139], v[100:103], a[144:147], v[136:139]\n" \
  "ds_read_b64_tr_b8 v[122:123], v83 offset:17408\n" \
  "v_cvt_f32_i32 v132, v132\n" \
  "v_mfma_i32_16x16x64_i8 v[168:171], v[96:99], a[132:135], v[168:171]\n" \
  "v_cvt_f32_i32 v133, v133\n" \
  "v_mfma_i32_16x16x64_i8 v[168:171], v[100:103], a[148:151], v[168:171]\n" \
  "v_cvt_f32_i32 v134, v134\n" \
  "v_mfma_i32_16x16x64_i8 v[200:203], v[96:99], a[136:139], v[200:203]\n" \
  "ds_read_b64_tr_b8 v[124:125], v83 offset:24576\n" \
  "v_cvt_f32_i32 v135, v135\n" \
  "v_mfma_i32_16x16x64_i8 v[200:203], v[100:103], a[152:155], v[200:203]\n" \
  "ds_read_b64_tr_b8 v[126:127], v83 offset:25600\n" \
  "v_mfma_i32_16x16x64_i8 v[232:235], v[96:99], a[140:143], v[232:235]\n" \
  "v_pk_mul_f32 v[132:133], v[132:133], v[88:89]\n" \
  "v_mfma_i32_16x16x64_i8 v[232:235], v[100:103], a[156:159], v[232:235]\n" \
  "v_pk_mul_f32 v[134:135], v[134:135], v[88:89]\n" \
  "s_waitcnt lgkmcnt(8)\n" \
  "v_mfma_i32_16x16x64_i8 v[140:143], v[104:107], a[128:131], v[140:143]\n" \
  "ds_read_b64_tr_b8 v[96:97], v26 offset:0\n" \
  "buffer_store_dwordx4 v[132:135], v47, s[20:23], s76 offen offset:64 sc0 sc1\n" \
  "v_mfma_i32_16x16x64_i8 v[140:143], v[108:111], a[144:147], v[140:143]\n" \
  "ds_read_b64_tr_b8 v[98:99], v26 offset:1024\n" \
  "v_mfma_i32_16x16x64_i8 v[172:175], v[104:107], a[132:135], v[172:175]\n" \
  "v_cvt_f32_i32 v160, v160\n" \
  "v_mfma_i32_16x16x64_i8 v[172:175], v[108:111], a[148:151], v[172:175]\n" \
  "v_cvt_f32_i32 v161, v161\n" \
  "v_mfma_i32_16x16x64_i8 v[204:207], v[104:107], a[136:139], v[204:207]\n" \
  "ds_read_b64_tr_b8 v[100:101], v26 offset:8192\n" \
  "v_cvt_f32_i32 v162, v162\n" \
  "v_mfma_i32_16x16x64_i8 v[204:207], v[108:111], a[152:155], v[204:207]\n" \
  "ds_read_b64_tr_b8 v[102:103], v26 offset:9216\n" \
  "v_cvt_f32_i32 v163, v163\n" \
  "v_mfma_i32_16x16x64_i8 v[236:239], v[104:107], a[140:143], v[236:239]\n" \
  "v_mfma_i32_16x16x64_i8 v[236:239], v[108:111], a[156:159], v[236:239]\n" \
  "v_pk_mul_f32 v[160:161], v[160:161], v[88:89]\n" \
  "s_waitcnt lgkmcnt(8)\n" \
  "v_mfma_i32_16x16x64_i8 v[136:139], v[112:115], a[160:163], v[136:139]\n" \
  "ds_read_b64_tr_b8 v[104:105], v27 offset:0\n" \
  "v_pk_mul_f32 v[162:163], v[162:163], v[88:89]\n" \
  "v_mfma_i32_16x16x64_i8 v[136:139], v[116:119], a[176:179], v[136:139]\n" \
  "ds_read_b64_tr_b8 v[106:107], v27 offset:1024\n" \
  "buffer_store_dwordx4 v[160:163], v47, s[20:23], s77 offen offset:0 sc0 sc1\n" \
  "v_mfma_i32_16x16x64_i8 v[168:171], v[112:115], a[164:167], v[168:171]\n" \
  "v_mfma_i32_16x16x64_i8 v[168:171], v[116:119], a[180:183], v[168:171]\n" \
  "v_cvt_f32_i32 v164, v164\n" \
  "v_mfma_i32_16x16x64_i8 v[200:203], v[112:115], a[168:171], v[200:203]\n" \
  "ds_read_b64_tr_b8 v[108:109], v27 offset:8192\n" \
  "v_cvt_f32_i32 v165, v165\n" \
  "v_mfma_i32_16x16x64_i8 v[200:203], v[116:119], a[184:187], v[200:203]\n" \
  "ds_read_b64_tr_b8 v[110:111], v27 offset:9216\n" \
  "v_cvt_f32_i32 v166, v166\n" \
  "v_mfma_i32_16x16x64_i8 v[232:235], v[112:115], a[172:175], v[232:235]\n" \
  "v_cvt_f32_i32 v167, v167\n" \
  "v_mfma_i32_16x16x64_i8 v[232:235], v[116:119], a[188:191], v[232:235]\n" \
  "s_waitcnt lgkmcnt(8)\n" \
  "v_mfma_i32_16x16x64_i8 v[140:143], v[120:123], a[160:163], v[140:143]\n" \
  "ds_read_b64_tr_b8 v[112:113], v26 offset:16384\n" \
  "v_pk_mul_f32 v[164:165], v[164:165], v[88:89]\n" \
  "v_mfma_i32_16x16x64_i8 v[140:143], v[124:127], a[176:179], v[140:143]\n" \
  "ds_read_b64_tr_b8 v[114:115], v26 offset:17408\n" \
  "v_pk_mul_f32 v[166:167], v[166:167], v[88:89]\n" \
  "v_mfma_i32_16x16x64_i8 v[172:175], v[120:123], a[164:167], v[172:175]\n" \
  "buffer_store_dwordx4 v[164:167], v47, s[20:23], s77 offen offset:64 sc0 sc1\n" \
  "v_mfma_i32_16x16x64_i8 v[172:175], v[124:127], a[180:183], v[172:175]\n" \
  "v_mfma_i32_16x16x64_i8 v[204:207], v[120:123], a[168:171], v[204:207]\n" \
  "ds_read_b64_tr_b8 v[116:117], v26 offset:24576\n" \
  "v_cvt_f32_i32 v192, v192\n" \
  "v_mfma_i32_16x16x64_i8 v[204:207], v[124:127], a[184:187], v[204:207]\n" \
  "ds_read_b64_tr_b8 v[118:119], v26 offset:25600\n" \
  "v_cvt_f32_i32 v193, v193\n" \
  "v_mfma_i32_16x16x64_i8 v[236:239], v[120:123], a[172:175], v[236:239]\n" \
  "v_cvt_f32_i32 v194, v194\n" \
  "v_mfma_i32_16x16x64_i8 v[236:239], v[124:127], a[188:191], v[236:239]\n" \
  "v_cvt_f32_i32 v195, v195\n" \
  "; ---- step 5\n" \
  "s_waitcnt lgkmcnt(8)\n" \
  "v_mfma_i32_16x16x64_i8 v[136:139], v[96:99], a[192:195], v[136:139]\n" \
  "ds_read_b64_tr_b8 v[120:121], v27 offset:16384\n" \
  "v_mfma_i32_16x16x64_i8 v[136:139], v[100:103], a[208:211], v[136:139]\n" \
  "ds_read_b64_tr_b8 v[122:123], v27 offset:17408\n" \
  "v_pk_mul_f32 v[192:193], v[192:193], v[88:89]\n" \
  "v_mfma_i32_16x16x64_i8 v[168:171], v[96:99], a[196:199], v[168:171]\n" \
  "v_pk_mul_f32 v[194:195], v[194:195], v[88:89]\n" \
  "v_mfma_i32_16x16x64_i8 v[168:171], v[100:103], a[212:215], v[168:171]\n" \
  "buffer_store_dwordx4 v[192:195], v47, s[20:23], s78 offen offset:0 sc0 sc1\n" \
  "v_mfma_i32_16x16x64_i8 v[200:203], v[96:99], a[200:203], v[200:203]\n" \
  "ds_read_b64_tr_b8 v[124:125], v27 offset:24576\n" \
  "v_mfma_i32_16x16x64_i8 v[200:203], v[100:103], a[216:219], v[200:203]\n" \
  "ds_read_b64_tr_b8 v[126:127], v27 offset:25600\n" \
  "v_cvt_f32_i32 v196, v196\n" \
  "v_mfma_i32_16x16x64_i8 v[232:235], v[96:99], a[204:207], v[232:235]\n" \
  "v_cvt_f32_i32 v197, v197\n" \
  "v_mfma_i32_16x16x64_i8 v[232:235], v[100:103], a[220:223], v[232:235]\n" \
  "v_cvt_f32_i32 v198, v198\n" \
  "s_waitcnt lgkmcnt(8)\n" \
  "v_mfma_i32_16x16x64_i8 v[140:143], v[104:107], a[192:195], v[140:143]\n" \
  "ds_read_b64_tr_b8 v[96:97], v36 offset:49152\n" \
  "v_cvt_f32_i32 v199, v199\n" \
  "v_mfma_i32_16x16x64_i8 v[140:143], v[108:111], a[208:211], v[140:143]\n" \
  "ds_read_b64_tr_b8 v[98:99], v36 offset:50176\n" \
  "v_mfma_i32_16x16x64_i8 v[172:175], v[104:107], a[196:199], v[172:175]\n" \
  "v_pk_mul_f32 v[196:197], v[196:197], v[88:89]\n" \
  "v_mfma_i32_16x16x64_i8 v[172:175], v[108:111], a[212:215], v[172:175]\n" \
  "v_pk_mul_f32 v[198:199], v[198:199], v[88:89]\n" \
  "v_mfma_i32_16x16x64_i8 v[204:207], v[104:107], a[200:203], v[204:207]\n" \
  "ds_read_b64_tr_b8 v[100:101], v36 offset:57344\n" \
  "buffer_store_dwordx4 v[196:199], v47, s[20:23], s78 offen offset:64 sc0 sc1\n" \
  "v_mfma_i32_16x16x64_i8 v[204:207], v[108:111], a[216:219], v[204:207]\n" \
  "ds_read_b64_tr_b8 v[102:103], v36 offset:58368\n" \
  "v_mfma_i32_16x16x64_i8 v[236:239], v[104:107], a[204:207], v[236:239]\n" \
  "v_cvt_f32_i32 v224, v224\n" \
  "v_mfma_i32_16x16x64_i8 v[236:239], v[108:111], a[220:223], v[236:239]\n" \
  "v_cvt_f32_i32 v225, v225\n" \
  "s_waitcnt lgkmcnt(8)\n" \
  "v_mfma_i32_16x16x64_i8 v[136:139], v[112:115], a[224:227], v[136:139]\n" \
  "ds_read_b64_tr_b8 v[104:105], v37 offset:49152\n" \
  "v_cvt_f32_i32 v226, v226\n" \
  "v_mfma_i32_16x16x64_i8 v[136:139], v[116:119], a[240:243], v[136:139]\n" \
  "ds_read_b64_tr_b8 v[106:107], v37 offset:50176\n" \
  "v_cvt_f32_i32 v227, v227\n" \
  "v_mfma_i32_16x16x64_i8 v[168:171], v[112:115], a[228:231], v[168:171]\n" \
  "v_mfma_i32_16x16x64_i8 v[168:171], v[116:119], a[244:247], v[168:171]\n" \
  "v_pk_mul_f32 v[224:225], v[224:225], v[88:89]\n" \
  "v_mfma_i32_16x16x64_i8 v[200:203], v[112:115], a[232:235], v[200:203]\n" \
  "ds_read_b64_tr_b8 v[108:109], v37 offset:57344\n" \
  "v_pk_mul_f32 v[226:227], v[226:227], v[88:89]\n" \
  "v_mfma_i32_16x16x64_i8 v[200:203], v[116:119], a[248:251], v[200:203]\n" \
  "ds_read_b64_tr_b8 v[110:111], v37 offset:58368\n" \
  "buffer_store_dwordx4 v[224:227], v47, s[20:23], s79 offen offset:0 sc0 sc1\n" \
  "v_mfma_i32_16x16x64_i8 v[232:235], v[112:115], a[236:239], v[232:235]\n" \
  "v_mfma_i32_16x16x64_i8 v[232:235], v[116:119], a[252:255], v[232:235]\n" \
  "v_cvt_f32_i32 v228, v228\n" \
  "s_waitcnt lgkmcnt(8)\n" \
  "v_mfma_i32_16x16x64_i8 v[140:143], v[120:123], a[224:227], v[140:143]\n" \
  "ds_read_b64_tr_b8 v[112:113], v84 offset:0\n" \
  "v_cvt_f32_i32 v229, v229\n" \
  "v_mfma_i32_16x16x64_i8 v[140:143], v[124:127], a[240:243], v[140:143]\n" \
  "ds_read_b64_tr_b8 v[114:115], v84 offset:1024\n" \
  "v_cvt_f32_i32 v230, v230\n" \
  "v_mfma_i32_16x16x64_i8 v[172:175], v[120:123], a[228:231], v[172:175]\n" \
  "v_cvt_f32_i32 v231, v231\n" \
  "v_mfma_i32_16x16x64_i8 v[172:175], v[124:127], a[244:247], v[172:175]\n" \
  "v_mfma_i32_16x16x64_i8 v[204:207], v[120:123], a[232:235], v[204:207]\n" \
  "ds_read_b64_tr_b8 v[116:117], v84 offset:8192\n" \
  "v_pk_mul_f32 v[228:229], v[228:229], v[88:89]\n" \
  "v_mfma_i32_16x16x64_i8 v[204:207], v[124:127], a[248:251], v[204:207]\n" \
  "ds_read_b64_tr_b8 v[118:119], v84 offset:9216\n" \
  "v_pk_mul_f32 v[230:231], v[230:231], v[88:89]\n" \
  "v_mfma_i32_16x16x64_i8 v[236:239], v[120:123], a[236:239], v[236:239]\n" \
  "buffer_store_dwordx4 v[228:231], v47, s[20:23], s79 offen offset:64 sc0 sc1\n" \
  "v_mfma_i32_16x16x64_i8 v[236:239], v[124:127], a[252:255], v[236:239]\n" \
  "s_waitcnt lgkmcnt(8)\n" \
  "v_mfma_i32_16x16x64_i8 v[144:147], v[96:99], a[96:99], v[144:147]\n" \
  "ds_read_b64_tr_b8 v[120:121], v85 offset:0\n" \
  "v_mfma_i32_16x16x64_i8 v[144:147], v[100:103], a[112:115], v[144:147]\n" \
  "ds_read_b64_tr_b8 v[122:123], v85 offset:1024\n" \
  "v_mfma_i32_16x16x64_i8 v[176:179], v[96:99], a[100:103], v[176:179]\n" \
  "v_mfma_i32_16x16x64_i8 v[176:179], v[100:103], a[116:119], v[176:179]\n" \
  "v_mfma_i32_16x16x64_i8 v[208:211], v[96:99], a[104:107], v[208:211]\n" \
  "ds_read_b64_tr_b8 v[124:125], v85 offset:8192\n" \
  "v_mfma_i32_16x16x64_i8 v[208:211], v[100:103], a[120:123], v[208:211]\n" \
  "ds_read_b64_tr_b8 v[126:127], v85 offset:9216\n" \
  "v_mfma_i32_16x16x64_i8 v[240:243], v[96:99], a[108:111], v[240:243]\n" \
  "v_mfma_i32_16x16x64_i8 v[240:243], v[100:103], a[124:127], v[240:243]\n" \
  "s_waitcnt lgkmcnt(8)\n" \
  "v_mfma_i32_16x16x64_i8 v[148:151], v[104:107], a[96:99], v[148:151]\n" \
  "ds_read_b64_tr_b8 v[96:97], v84 offset:16384\n" \
  "v_cvt_f32_i32 v136, v136\n" \
  "v_mfma_i32_16x16x64_i8 v[148:151], v[108:111], a[112:115], v[148:151]\n" \
  "ds_read_b64_tr_b8 v[98:99], v84 offset:17408\n" \
  "v_cvt_f32_i32 v137, v137\n" \
  "v_mfma_i32_16x16x64_i8 v[180:183], v[104:107], a[100:103], v[180:183]\n" \
  "v_cvt_f32_i32 v138, v138\n" \
  "v_mfma_i32_16x16x64_i8 v[180:183], v[108:111], a[116:119], v[180:183]\n" \
  "v_cvt_f32_i32 v139, v139\n" \
  "v_mfma_i32_16x16x64_i8 v[212:215], v[104:107], a[104:107], v[212:215]\n" \
  "ds_read_b64_tr_b8 v[100:101], v84 offset:24576\n" \
  "v_mfma_i32_16x16x64_i8 v[212:215], v[108:111], a[120:123], v[212:215]\n" \
  "ds_read_b64_tr_b8 v[102:103], v84 offset:25600\n" \
  "v_pk_mul_f32 v[136:137], v[136:137], v[88:89]\n" \
  "v_mfma_i32_16x16x64_i8 v[244:247], v[104:107], a[108:111], v[244:247]\n" \
  "v_pk_mul_f32 v[138:139], v[138:139], v[88:89]\n" \
  "v_mfma_i32_16x16x64_i8 v[244:247], v[108:111], a[124:127], v[244:247]\n" \
  "buffer_store_dwordx4 v[136:139], v47, s[20:23], s76 offen offset:128 sc0 sc1\n" \
  "s_waitcnt lgkmcnt(8)\n" \
  "v_mfma_i32_16x16x64_i8 v[144:147], v[112:115], a[128:131], v[144:147]\n" \
  "ds_read_b64_tr_b8 v[104:105], v85 offset:16384\n" \
  "v_mfma_i32_16x16x64_i8 v[144:147], v[116:119], a[144:147], v[144:147]\n" \
  "ds_read_b64_tr_b8 v[106:107], v85 offset:17408\n" \
  "v_cvt_f32_i32 v140, v140\n" \
  "v_mfma_i32_16x16x64_i8 v[176:179], v[112:115], a[132:135], v[176:179]\n" \
  "v_cvt_f32_i32 v141, v141\n" \
  "v_mfma_i32_16x16x64_i8 v[176:179], v[116:119], a[148:151], v[176:179]\n" \
  "v_cvt_f32_i32 v142, v142\n" \
  "v_mfma_i32_16x16x64_i8 v[208:211], v[112:115], a[136:139], v[208:211]\n" \
  "ds_read_b64_tr_b8 v[108:109], v85 offset:24576\n" \
  "v_cvt_f32_i32 v143, v143\n" \
  "v_mfma_i32_16x16x64_i8 v[208:211], v[116:119], a[152:155], v[208:211]\n" \
  "ds_read_b64_tr_b8 v[110:111], v85 offset:25600\n" \
  "v_mfma_i32_16x16x64_i8 v[240:243], v[112:115], a[140:143], v[240:243]\n" \
  "v_pk_mul_f32 v[140:141], v[140:141], v[88:89]\n" \
  "v_mfma_i32_16x16x64_i8 v[240:243], v[116:119], a[156:159], v[240:243]\n" \
  "v_pk_mul_f32 v[142:143], v[142:143], v[88:89]\n" \
  "s_waitcnt lgkmcnt(8)\n" \
  "v_mfma_i32_16x16x64_i8 v[148:151], v[120:123], a[128:131], v[148:151]\n" \
  "ds_read_b64_tr_b8 v[112:113], v28 offset:0\n" \
  "buffer_store_dwordx4 v[140:143], v47, s[20:23], s76 offen offset:192 sc0 sc1\n" \
  "v_mfma_i32_16x16x64_i8 v[148:151], v[124:127], a[144:147], v[148:151]\n" \
  "ds_read_b64_tr_b8 v[114:115], v28 offset:1024\n" \
  "v_mfma_i32_16x16x64_i8 v[180:183], v[120:123], a[132:135], v[180:183]\n" \
  "v_cvt_f32_i32 v168, v168\n" \
  "v_mfma_i32_16x16x64_i8 v[180:183], v[124:127], a[148:151], v[180:183]\n" \
  "v_cvt_f32_i32 v169, v169\n" \
  "v_mfma_i32_16x16x64_i8 v[212:215], v[120:123], a[136:139], v[212:215]\n" \
  "ds_read_b64_tr_b8 v[116:117], v28 offset:8192\n" \
  "v_cvt_f32_i32 v170, v170\n" \
  "v_mfma_i32_16x16x64_i8 v[212:215], v[124:127], a[152:155], v[212:215]\n" \
  "ds_read_b64_tr_b8 v[118:119], v28 offset:9216\n" \
  "v_cvt_f32_i32 v171, v171\n" \
  "v_mfma_i32_16x16x64_i8 v[244:247], v[120:123], a[140:143], v[244:247]\n" \
  "v_mfma_i32_16x16x64_i8 v[244:247], v[124:127], a[156:159], v[244:247]\n" \
  "v_pk_mul_f32 v[168:169], v[168:169], v[88:89]\n" \
  "; ---- step 6\n" \
  "s_waitcnt lgkmcnt(8)\n" \
  "v_mfma_i32_16x16x64_i8 v[144:147], v[96:99], a[160:163], v[144:147]\n" \
  "ds_read_b64_tr_b8 v[120:121], v29 offset:0\n" \
  "v_pk_mul_f32 v[170:171], v[170:171], v[88:89]\n" \
  "v_mfma_i32_16x16x64_i8 v[144:147], v[100:103], a[176:179], v[144:147]\n" \
  "ds_read_b64_tr_b8 v[122:123], v29 offset:1024\n" \
  "buffer_store_dwordx4 v[168:171], v47, s[20:23], s77 offen offset:128 sc0 sc1\n" \
  "v_mfma_i32_16x16x64_i8 v[176:179], v[96:99], a[164:167], v[176:179]\n" \
  "v_mfma_i32_16x16x64_i8 v[176:179], v[100:103], a[180:183], v[176:179]\n" \
  "v_cvt_f32_i32 v172, v172\n" \
  "v_mfma_i32_16x16x64_i8 v[208:211], v[96:99], a[168:171], v[208:211]\n" \
  "ds_read_b64_tr_b8 v[124:125], v29 offset:8192\n" \
  "v_cvt_f32_i32 v173, v173\n" \
  "v_mfma_i32_16x16x64_i8 v[208:211], v[100:103], a[184:187], v[208:211]\n" \
  "ds_read_b64_tr_b8 v[126:127], v29 offset:9216\n" \
  "v_cvt_f32_i32 v174, v174\n" \
  "v_mfma_i32_16x16x64_i8 v[240:243], v[96:99], a[172:175], v[240:243]\n" \
  "v_cvt_f32_i32 v175, v175\n" \
  "v_mfma_i32_16x16x64_i8 v[240:243], v[100:103], a[188:191], v[240:243]\n" \
  "s_waitcnt lgkmcnt(8)\n" \
  "v_mfma_i32_16x16x64_i8 v[148:151], v[104:107], a[160:163], v[148:151]\n" \
  "ds_read_b64_tr_b8 v[96:97], v28 offset:16384\n" \
  "v_pk_mul_f32 v[172:173], v[172:173], v[88:89]\n" \
  "v_mfma_i32_16x16x64_i8 v[148:151], v[108:111], a[176:179], v[148:151]\n" \
  "ds_read_b64_tr_b8 v[98:99], v28 offset:17408\n" \
  "v_pk_mul_f32 v[174:175], v[174:175], v[88:89]\n" \
  "v_mfma_i32_16x16x64_i8 v[180:183], v[104:107], a[164:167], v[180:183]\n" \
  "buffer_store_dwordx4 v[172:175], v47, s[20:23], s77 offen offset:192 sc0 sc1\n" \
  "v_mfma_i32_16x16x64_i8 v[180:183], v[108:111], a[180:183], v[180:183]\n" \
  "v_mfma_i32_16x16x64_i8 v[212:215], v[104:107], a[168:171], v[212:215]\n" \
  "ds_read_b64_tr_b8 v[100:101], v28 offset:24576\n" \
  "v_cvt_f32_i32 v200, v200\n" \
  "v_mfma_i32_16x16x64_i8 v[212:215], v[108:111], a[184:187], v[212:215]\n" \
  "ds_read_b64_tr_b8 v[102:103], v28 offset:25600\n" \
  "v_cvt_f32_i32 v201, v201\n" \
  "v_mfma_i32_16x16x64_i8 v[244:247], v[104:107], a[172:175], v[244:247]\n" \
  "v_cvt_f32_i32 v202, v202\n" \
  "v_mfma_i32_16x16x64_i8 v[244:247], v[108:111], a[188:191], v[244:247]\n" \
  "v_cvt_f32_i32 v203, v203\n" \
  "s_waitcnt lgkmcnt(8)\n" \
  "v_mfma_i32_16x16x64_i8 v[144:147], v[112:115], a[192:195], v[144:147]\n" \
  "ds_read_b64_tr_b8 v[104:105], v29 offset:16384\n" \
  "v_mfma_i32_16x16x64_i8 v[144:147], v[116:119], a[208:211], v[144:147]\n" \
  "ds_read_b64_tr_b8 v[106:107], v29 offset:17408\n" \
  "v_pk_mul_f32 v[200:201], v[200:201], v[88:89]\n" \
  "v_mfma_i32_16x16x64_i8 v[176:179], v[112:115], a[196:199], v[176:179]\n" \
  "v_pk_mul_f32 v[202:203], v[202:203], v[88:89]\n" \
  "v_mfma_i32_16x16x64_i8 v[176:179], v[116:119], a[212:215], v[176:179]\n" \
  "buffer_store_dwordx4 v[200:203], v47, s[20:23], s78 offen offset:128 sc0 sc1\n" \
  "v_mfma_i32_16x16x64_i8 v[208:211], v[112:115], a[200:203], v[208:211]\n" \
  "ds_read_b64_tr_b8 v[108:109], v29 offset:24576\n" \
  "v_mfma_i32_16x16x64_i8 v[208:211], v[116:119], a[216:219], v[208:211]\n" \
  "ds_read_b64_tr_b8 v[110:111], v29 offset:25600\n" \
  "v_cvt_f32_i32 v204, v204\n" \
  "v_mfma_i32_16x16x64_i8 v[240:243], v[112:115], a[204:207], v[240:243]\n" \
  "v_cvt_f32_i32 v205, v205\n" \
  "v_mfma_i32_16x16x64_i8 v[240:243], v[116:119], a[220:223], v[240:243]\n" \
  "v_cvt_f32_i32 v206, v206\n" \
  "s_waitcnt lgkmcnt(8)\n" \
  "v_mfma_i32_16x16x64_i8 v[148:151], v[120:123], a[192:195], v[148:151]\n" \
  "ds_read_b64_tr_b8 v[112:113], v38 offset:49152\n" \
  "v_cvt_f32_i32 v207, v207\n" \
  "v_mfma_i32_16x16x64_i8 v[148:151], v[124:127], a[208:211], v[148:151]\n" \
  "ds_read_b64_tr_b8 v[114:115], v38 offset:50176\n" \
  "v_mfma_i32_16x16x64_i8 v[180:183], v[120:123], a[196:199], v[180:183]\n" \
  "v_pk_mul_f32 v[204:205], v[204:205], v[88:89]\n" \
  "v_mfma_i32_16x16x64_i8 v[180:183], v[124:127], a[212:215], v[180:183]\n" \
  "v_pk_mul_f32 v[206:207], v[206:207], v[88:89]\n" \
  "v_mfma_i32_16x16x64_i8 v[212:215], v[120:123], a[200:203], v[212:215]\n" \
  "ds_read_b64_tr_b8 v[116:117], v38 offset:57344\n" \
  "buffer_store_dwordx4 v[204:207], v47, s[20:23], s78 offen offset:192 sc0 sc1\n" \
  "v_mfma_i32_16x16x64_i8 v[212:215], v[124:127], a[216:219], v[212:215]\n" \
  "ds_read_b64_tr_b8 v[118:119], v38 offset:58368\n" \
  "v_mfma_i32_16x16x64_i8 v[244:247], v[120:123], a[204:207], v[244:247]\n" \
  "v_cvt_f32_i32 v232, v232\n" \
  "v_mfma_i32_16x16x64_i8 v[244:247], v[124:127], a[220:223], v[244:247]\n" \
  "v_cvt_f32_i32 v233, v233\n" \
  "s_waitcnt lgkmcnt(8)\n" \
  "v_mfma_i32_16x16x64_i8 v[144:147], v[96:99], a[224:227], v[144:147]\n" \
  "ds_read_b64_tr_b8 v[120:121], v39 offset:49152\n" \
  "v_cvt_f32_i32 v234, v234\n" \
  "v_mfma_i32_16x16x64_i8 v[144:147], v[100:103], a[240:243], v[144:147]\n" \
  "ds_read_b64_tr_b8 v[122:123], v39 offset:50176\n" \
  "v_cvt_f32_i32 v235, v235\n" \
  "v_mfma_i32_16x16x64_i8 v[176:179], v[96:99], a[228:231], v[176:179]\n" \
  "v_mfma_i32_16x16x64_i8 v[176:179], v[100:103], a[244:247], v[176:179]\n" \
  "v_pk_mul_f32 v[232:233], v[232:233], v[88:89]\n" \
  "v_mfma_i32_16x16x64_i8 v[208:211], v[96:99], a[232:235], v[208:211]\n" \
  "ds_read_b64_tr_b8 v[124:125], v39 offset:57344\n" \
  "v_pk_mul_f32 v[234:235], v[234:235], v[88:89]\n" \
  "v_mfma_i32_16x16x64_i8 v[208:211], v[100:103], a[248:251], v[208:211]\n" \
  "ds_read_b64_tr_b8 v[126:127], v39 offset:58368\n" \
  "buffer_store_dwordx4 v[232:235], v47, s[20:23], s79 offen offset:128 sc0 sc1\n" \
  "v_mfma_i32_16x16x64_i8 v[240:243], v[96:99], a[236:239], v[240:243]\n" \
  "v_mfma_i32_16x16x64_i8 v[240:243], v[100:103], a[252:255], v[240:243]\n" \
  "v_cvt_f32_i32 v236, v236\n" \
  "s_waitcnt lgkmcnt(8)\n" \
  "v_mfma_i32_16x16x64_i8 v[148:151], v[104:107], a[224:227], v[148:151]\n" \
  "ds_read_b64_tr_b8 v[96:97], v86 offset:0\n" \
  "v_cvt_f32_i32 v237, v237\n" \
  "v_mfma_i32_16x16x64_i8 v[148:151], v[108:111], a[240:243], v[148:151]\n" \
  "ds_read_b64_tr_b8 v[98:99], v86 offset:1024\n" \
  "v_cvt_f32_i32 v238, v238\n" \
  "v_mfma_i32_16x16x64_i8 v[180:183], v[104:107], a[228:231], v[180:183]\n" \
  "v_cvt_f32_i32 v239, v239\n" \
  "v_mfma_i32_16x16x64_i8 v[180:183], v[108:111], a[244:247], v[180:183]\n" \
  "v_mfma_i32_16x16x64_i8 v[212:215], v[104:107], a[232:235], v[212:215]\n" \
  "ds_read_b64_tr_b8 v[100:101], v86 offset:8192\n" \
  "v_pk_mul_f32 v[236:237], v[236:237], v[88:89]\n" \
  "v_mfma_i32_16x16x64_i8 v[212:215], v[108:111], a[248:251], v[212:215]\n" \
  "ds_read_b64_tr_b8 v[102:103], v86 offset:9216\n" \
  "v_pk_mul_f32 v[238:239], v[238:239], v[88:89]\n" \
  "v_mfma_i32_16x16x64_i8 v[244:247], v[104:107], a[236:239], v[244:247]\n" \
  "buffer_store_dwordx4 v[236:239], v47, s[20:23], s79 offen offset:192 sc0 sc1\n" \
  "v_mfma_i32_16x16x64_i8 v[244:247], v[108:111], a[252:255], v[244:247]\n" \
  "s_waitcnt lgkmcnt(8)\n" \
  "v_mfma_i32_16x16x64_i8 v[152:155], v[112:115], a[96:99], v[152:155]\n" \
  "ds_read_b64_tr_b8 v[104:105], v87 offset:0\n" \
  "v_mfma_i32_16x16x64_i8 v[152:155], v[116:119], a[112:115], v[152:155]\n" \
  "ds_read_b64_tr_b8 v[106:107], v87 offset:1024\n" \
  "v_mfma_i32_16x16x64_i8 v[184:187], v[112:115], a[100:103], v[184:187]\n" \
  "v_mfma_i32_16x16x64_i8 v[184:187], v[116:119], a[116:119], v[184:187]\n" \
  "v_mfma_i32_16x16x64_i8 v[216:219], v[112:115], a[104:107], v[216:219]\n" \
  "ds_read_b64_tr_b8 v[108:109], v87 offset:8192\n" \
  "v_mfma_i32_16x16x64_i8 v[216:219], v[116:119], a[120:123], v[216:219]\n" \
  "ds_read_b64_tr_b8 v[110:111], v87 offset:9216\n" \
  "v_mfma_i32_16x16x64_i8 v[248:251], v[112:115], a[108:111], v[248:251]\n" \
  "v_mfma_i32_16x16x64_i8 v[248:251], v[116:119], a[124:127], v[248:251]\n" \
  "s_waitcnt lgkmcnt(8)\n" \
  "v_mfma_i32_16x16x64_i8 v[156:159], v[120:123], a[96:99], v[156:159]\n" \
  "ds_read_b64_tr_b8 v[112:113], v86 offset:16384\n" \
  "v_cvt_f32_i32 v144, v144\n" \
  "v_mfma_i32_16x16x64_i8 v[156:159], v[124:127], a[112:115], v[156:159]\n" \
  "ds_read_b64_tr_b8 v[114:115], v86 offset:17408\n" \
  "v_cvt_f32_i32 v145, v145\n" \
  "v_mfma_i32_16x16x64_i8 v[188:191], v[120:123], a[100:103], v[188:191]\n" \
  "v_cvt_f32_i32 v146, v146\n" \
  "v_mfma_i32_16x16x64_i8 v[188:191], v[124:127], a[116:119], v[188:191]\n" \
  "v_cvt_f32_i32 v147, v147\n" \
  "v_mfma_i32_16x16x64_i8 v[220:223], v[120:123], a[104:107], v[220:223]\n" \
  "ds_read_b64_tr_b8 v[116:117], v86 offset:24576\n" \
  "v_mfma_i32_16x16x64_i8 v[220:223], v[124:127], a[120:123], v[220:223]\n" \
  "ds_read_b64_tr_b8 v[118:119], v86 offset:25600\n" \
  "v_pk_mul_f32 v[144:145], v[144:145], v[88:89]\n" \
  "v_mfma_i32_16x16x64_i8 v[252:255], v[120:123], a[108:111], v[252:255]\n" \
  "v_pk_mul_f32 v[146:147], v[146:147], v[88:89]\n" \
  "v_mfma_i32_16x16x64_i8 v[252:255], v[124:127], a[124:127], v[252:255]\n" \
  "buffer_store_dwordx4 v[144:147], v47, s[20:23], s76 offen offset:256 sc0 sc1\n" \
  "; ---- step 7\n" \
  "s_waitcnt lgkmcnt(8)\n" \
  "v_mfma_i32_16x16x64_i8 v[152:155], v[96:99], a[128:131], v[152:155]\n" \
  "ds_read_b64_tr_b8 v[120:121], v87 offset:16384\n" \
  "v_mfma_i32_16x16x64_i8 v[152:155], v[100:103], a[144:147], v[152:155]\n" \
  "ds_read_b64_tr_b8 v[122:123], v87 offset:17408\n" \
  "v_cvt_f32_i32 v148, v148\n" \
  "v_mfma_i32_16x16x64_i8 v[184:187], v[96:99], a[132:135], v[184:187]\n" \
  "v_cvt_f32_i32 v149, v149\n" \
  "v_mfma_i32_16x16x64_i8 v[184:187], v[100:103], a[148:151], v[184:187]\n" \
  "v_cvt_f32_i32 v150, v150\n" \
  "v_mfma_i32_16x16x64_i8 v[216:219], v[96:99], a[136:139], v[216:219]\n" \
  "ds_read_b64_tr_b8 v[124:125], v87 offset:24576\n" \
  "v_cvt_f32_i32 v151, v151\n" \
  "v_mfma_i32_16x16x64_i8 v[216:219], v[100:103], a[152:155], v[216:219]\n" \
  "ds_read_b64_tr_b8 v[126:127], v87 offset:25600\n" \
  "v_mfma_i32_16x16x64_i8 v[248:251], v[96:99], a[140:143], v[248:251]\n" \
  "v_pk_mul_f32 v[148:149], v[148:149], v[88:89]\n" \
  "v_mfma_i32_16x16x64_i8 v[248:251], v[100:103], a[156:159], v[248:251]\n" \
  "v_pk_mul_f32 v[150:151], v[150:151], v[88:89]\n" \
  "s_waitcnt lgkmcnt(8)\n" \
  "v_mfma_i32_16x16x64_i8 v[156:159], v[104:107], a[128:131], v[156:159]\n" \
  "ds_read_b64_tr_b8 v[96:97], v30 offset:0\n" \
  "buffer_store_dwordx4 v[148:151], v47, s[20:23], s76 offen offset:320 sc0 sc1\n" \
  "v_mfma_i32_16x16x64_i8 v[156:159], v[108:111], a[144:147], v[156:159]\n" \
  "ds_read_b64_tr_b8 v[98:99], v30 offset:1024\n" \
  "v_mfma_i32_16x16x64_i8 v[188:191], v[104:107], a[132:135], v[188:191]\n" \
  "v_cvt_f32_i32 v176, v176\n" \
  "v_mfma_i32_16x16x64_i8 v[188:191], v[108:111], a[148:151], v[188:191]\n" \
  "v_cvt_f32_i32 v177, v177\n" \
  "v_mfma_i32_16x16x64_i8 v[220:223], v[104:107], a[136:139], v[220:223]\n" \
  "ds_read_b64_tr_b8 v[100:101], v30 offset:8192\n" \
  "v_cvt_f32_i32 v178, v178\n" \
  "v_mfma_i32_16x16x64_i8 v[220:223], v[108:111], a[152:155], v[220:223]\n" \
  "ds_read_b64_tr_b8 v[102:103], v30 offset:9216\n" \
  "v_cvt_f32_i32 v179, v179\n" \
  "v_mfma_i32_16x16x64_i8 v[252:255], v[104:107], a[140:143], v[252:255]\n" \
  "v_mfma_i32_16x16x64_i8 v[252:255], v[108:111], a[156:159], v[252:255]\n" \
  "v_pk_mul_f32 v[176:177], v[176:177], v[88:89]\n" \
  "s_waitcnt lgkmcnt(8)\n" \
  "v_mfma_i32_16x16x64_i8 v[152:155], v[112:115], a[160:163], v[152:155]\n" \
  "ds_read_b64_tr_b8 v[104:105], v31 offset:0\n" \
  "v_pk_mul_f32 v[178:179], v[178:179], v[88:89]\n" \
  "v_mfma_i32_16x16x64_i8 v[152:155], v[116:119], a[176:179], v[152:155]\n" \
  "ds_read_b64_tr_b8 v[106:107], v31 offset:1024\n" \
  "buffer_store_dwordx4 v[176:179], v47, s[20:23], s77 offen offset:256 sc0 sc1\n" \
  "v_mfma_i32_16x16x64_i8 v[184:187], v[112:115], a[164:167], v[184:187]\n" \
  "v_mfma_i32_16x16x64_i8 v[184:187], v[116:119], a[180:183], v[184:187]\n" \
  "v_cvt_f32_i32 v180, v180\n" \
  "v_mfma_i32_16x16x64_i8 v[216:219], v[112:115], a[168:171], v[216:219]\n" \
  "ds_read_b64_tr_b8 v[108:109], v31 offset:8192\n" \
  "v_cvt_f32_i32 v181, v181\n" \
  "v_mfma_i32_16x16x64_i8 v[216:219], v[116:119], a[184:187], v[216:219]\n" \
  "ds_read_b64_tr_b8 v[110:111], v31 offset:9216\n" \
  "v_cvt_f32_i32 v182, v182\n" \
  "v_mfma_i32_16x16x64_i8 v[248:251], v[112:115], a[172:175], v[248:251]\n" \
  "v_cvt_f32_i32 v183, v183\n" \
  "v_mfma_i32_16x16x64_i8 v[248:251], v[116:119], a[188:191], v[248:251]\n" \
  "s_waitcnt lgkmcnt(8)\n" \
  "v_mfma_i32_16x16x64_i8 v[156:159], v[120:123], a[160:163], v[156:159]\n" \
  "ds_read_b64_tr_b8 v[112:113], v30 offset:16384\n" \
  "v_pk_mul_f32 v[180:181], v[180:181], v[88:89]\n" \
  "v_mfma_i32_16x16x64_i8 v[156:159], v[124:127], a[176:179], v[156:159]\n" \
  "ds_read_b64_tr_b8 v[114:115], v30 offset:17408\n" \
  "v_pk_mul_f32 v[182:183], v[182:183], v[88:89]\n" \
  "v_mfma_i32_16x16x64_i8 v[188:191], v[120:123], a[164:167], v[188:191]\n" \
  "buffer_store_dwordx4 v[180:183], v47, s[20:23], s77 offen offset:320 sc0 sc1\n" \
  "v_mfma_i32_16x16x64_i8 v[188:191], v[124:127], a[180:183], v[188:191]\n" \
  "v_mfma_i32_16x16x64_i8 v[220:223], v[120:123], a[168:171], v[220:223]\n" \
  "ds_read_b64_tr_b8 v[116:117], v30 offset:24576\n" \
  "v_cvt_f32_i32 v208, v208\n" \
  "v_mfma_i32_16x16x64_i8 v[220:223], v[124:127], a[184:187], v[220:223]\n" \
  "ds_read_b64_tr_b8 v[118:119], v30 offset:25600\n" \
  "v_cvt_f32_i32 v209, v209\n" \
  "v_mfma_i32_16x16x64_i8 v[252:255], v[120:123], a[172:175], v[252:255]\n" \
  "v_cvt_f32_i32 v210, v210\n" \
  "v_mfma_i32_16x16x64_i8 v[252:255], v[124:127], a[188:191], v[252:255]\n" \
  "v_cvt_f32_i32 v211, v211\n" \
  "s_waitcnt lgkmcnt(8)\n" \
  "v_mfma_i32_16x16x64_i8 v[152:155], v[96:99], a[192:195], v[152:155]\n" \
  "ds_read_b64_tr_b8 v[120:121], v31 offset:16384\n" \
  "v_mfma_i32_16x16x64_i8 v[152:155], v[100:103], a[208:211], v[152:155]\n" \
  "ds_read_b64_tr_b8 v[122:123], v31 offset:17408\n" \
  "v_pk_mul_f32 v[208:209], v[208:209], v[88:89]\n" \
  "v_mfma_i32_16x16x64_i8 v[184:187], v[96:99], a[196:199], v[184:187]\n" \
  "v_pk_mul_f32 v[210:211], v[210:211], v[88:89]\n" \
  "v_mfma_i32_16x16x64_i8 v[184:187], v[100:103], a[212:215], v[184:187]\n" \
  "buffer_store_dwordx4 v[208:211], v47, s[20:23], s78 offen offset:256 sc0 sc1\n" \
  "v_mfma_i32_16x16x64_i8 v[216:219], v[96:99], a[200:203], v[216:219]\n" \
  "ds_read_b64_tr_b8 v[124:125], v31 offset:24576\n" \
  "v_mfma_i32_16x16x64_i8 v[216:219], v[100:103], a[216:219], v[216:219]\n" \
  "ds_read_b64_tr_b8 v[126:127], v31 offset:25600\n" \
  "v_cvt_f32_i32 v212, v212\n" \
  "v_mfma_i32_16x16x64_i8 v[248:251], v[96:99], a[204:207], v[248:251]\n" \
  "v_cvt_f32_i32 v213, v213\n" \
  "v_mfma_i32_16x16x64_i8 v[248:251], v[100:103], a[220:223], v[248:251]\n" \
  "v_cvt_f32_i32 v214, v214\n" \
  "s_waitcnt lgkmcnt(8)\n" \
  "v_mfma_i32_16x16x64_i8 v[156:159], v[104:107], a[192:195], v[156:159]\n" \
  "v_cvt_f32_i32 v215, v215\n" \
  "v_mfma_i32_16x16x64_i8 v[156:159], v[108:111], a[208:211], v[156:159]\n" \
  "v_mfma_i32_16x16x64_i8 v[188:191], v[104:107], a[196:199], v[188:191]\n" \
  "v_pk_mul_f32 v[212:213], v[212:213], v[88:89]\n" \
  "v_mfma_i32_16x16x64_i8 v[188:191], v[108:111], a[212:215], v[188:191]\n" \
  "v_pk_mul_f32 v[214:215], v[214:215], v[88:89]\n" \
  "v_mfma_i32_16x16x64_i8 v[220:223], v[104:107], a[200:203], v[220:223]\n" \
  "buffer_store_dwordx4 v[212:215], v47, s[20:23], s78 offen offset:320 sc0 sc1\n" \
  "v_mfma_i32_16x16x64_i8 v[220:223], v[108:111], a[216:219], v[220:223]\n" \
  "v_mfma_i32_16x16x64_i8 v[252:255], v[104:107], a[204:207], v[252:255]\n" \
  "v_cvt_f32_i32 v240, v240\n" \
  "v_mfma_i32_16x16x64_i8 v[252:255], v[108:111], a[220:223], v[252:255]\n" \
  "v_cvt_f32_i32 v241, v241\n" \
  "s_waitcnt lgkmcnt(4)\n" \
  "v_mfma_i32_16x16x64_i8 v[152:155], v[112:115], a[224:227], v[152:155]\n" \
  "v_cvt_f32_i32 v242, v242\n" \
  "v_mfma_i32_16x16x64_i8 v[152:155], v[116:119], a[240:243], v[152:155]\n" \
  "v_cvt_f32_i32 v243, v243\n" \
  "v_mfma_i32_16x16x64_i8 v[184:187], v[112:115], a[228:231], v[184:187]\n" \
  "v_mfma_i32_16x16x64_i8 v[184:187], v[116:119], a[244:247], v[184:187]\n" \
  "v_pk_mul_f32 v[240:241], v[240:241], v[88:89]\n" \
  "v_mfma_i32_16x16x64_i8 v[216:219], v[112:115], a[232:235], v[216:219]\n" \
  "v_pk_mul_f32 v[242:243], v[242:243], v[88:89]\n" \
  "v_mfma_i32_16x16x64_i8 v[216:219], v[116:119], a[248:251], v[216:219]\n" \
  "buffer_store_dwordx4 v[240:243], v47, s[20:23], s79 offen offset:256 sc0 sc1\n" \
  "v_mfma_i32_16x16x64_i8 v[248:251], v[112:115], a[236:239], v[248:251]\n" \
  "v_mfma_i32_16x16x64_i8 v[248:251], v[116:119], a[252:255], v[248:251]\n" \
  "v_cvt_f32_i32 v244, v244\n" \
  "s_waitcnt lgkmcnt(0)\n" \
  "v_mfma_i32_16x16x64_i8 v[156:159], v[120:123], a[224:227], v[156:159]\n" \
  "v_cvt_f32_i32 v245, v245\n" \
  "v_mfma_i32_16x16x64_i8 v[156:159], v[124:127], a[240:243], v[156:159]\n" \
  "v_cvt_f32_i32 v246, v246\n" \
  "v_mfma_i32_16x16x64_i8 v[188:191], v[120:123], a[228:231], v[188:191]\n" \
  "v_cvt_f32_i32 v247, v247\n" \
  "v_mfma_i32_16x16x64_i8 v[188:191], v[124:127], a[244:247], v[188:191]\n" \
  "v_mfma_i32_16x16x64_i8 v[220:223], v[120:123], a[232:235], v[220:223]\n" \
  "v_pk_mul_f32 v[244:245], v[244:245], v[88:89]\n" \
  "v_mfma_i32_16x16x64_i8 v[220:223], v[124:127], a[248:251], v[220:223]\n" \
  "v_pk_mul_f32 v[246:247], v[246:247], v[88:89]\n" \
  "v_mfma_i32_16x16x64_i8 v[252:255], v[120:123], a[236:239], v[252:255]\n" \
  "buffer_store_dwordx4 v[244:247], v47, s[20:23], s79 offen offset:320 sc0 sc1\n" \
  "v_mfma_i32_16x16x64_i8 v[252:255], v[124:127], a[252:255], v[252:255]\n" \
  "s_nop 15\n" \
  "v_cvt_f32_i32 v152, v152\n" \
  "v_cvt_f32_i32 v153, v153\n" \
  "v_cvt_f32_i32 v154, v154\n" \
  "v_cvt_f32_i32 v155, v155\n" \
  "v_pk_mul_f32 v[152:153], v[152:153], v[88:89]\n" \
  "v_pk_mul_f32 v[154:155], v[154:155], v[88:89]\n" \
  "buffer_store_dwordx4 v[152:155], v47, s[20:23], s76 offen offset:384 sc0 sc1\n" \
  "v_cvt_f32_i32 v156, v156\n" \
  "v_cvt_f32_i32 v157, v157\n" \
  "v_cvt_f32_i32 v158, v158\n" \
  "v_cvt_f32_i32 v159, v159\n" \
  "v_pk_mul_f32 v[156:157], v[156:157], v[88:89]\n" \
  "v_pk_mul_f32 v[158:159], v[158:159], v[88:89]\n" \
  "buffer_store_dwordx4 v[156:159], v47, s[20:23], s76 offen offset:448 sc0 sc1\n" \
  "v_cvt_f32_i32 v184, v184\n" \
  "v_cvt_f32_i32 v185, v185\n" \
  "v_cvt_f32_i32 v186, v186\n" \
  "v_cvt_f32_i32 v187, v187\n" \
  "v_pk_mul_f32 v[184:185], v[184:185], v[88:89]\n" \
  "v_pk_mul_f32 v[186:187], v[186:187], v[88:89]\n" \
  "buffer_store_dwordx4 v[184:187], v47, s[20:23], s77 offen offset:384 sc0 sc1\n" \
  "v_cvt_f32_i32 v188, v188\n" \
  "v_cvt_f32_i32 v189, v189\n" \
  "v_cvt_f32_i32 v190, v190\n" \
  "v_cvt_f32_i32 v191, v191\n" \
  "v_pk_mul_f32 v[188:189], v[188:189], v[88:89]\n" \
  "v_pk_mul_f32 v[190:191], v[190:191], v[88:89]\n" \
  "buffer_store_dwordx4 v[188:191], v47, s[20:23], s77 offen offset:448 sc0 sc1\n" \
  "v_cvt_f32_i32 v216, v216\n" \
  "v_cvt_f32_i32 v217, v217\n" \
  "v_cvt_f32_i32 v218, v218\n" \
  "v_cvt_f32_i32 v219, v219\n" \
  "v_pk_mul_f32 v[216:217], v[216:217], v[88:89]\n" \
  "v_pk_mul_f32 v[218:219], v[218:219], v[88:89]\n" \
  "buffer_store_dwordx4 v[216:219], v47, s[20:23], s78 offen offset:384 sc0 sc1\n" \
  "v_cvt_f32_i32 v220, v220\n" \
  "v_cvt_f32_i32 v221, v221\n" \
  "v_cvt_f32_i32 v222, v222\n" \
  "v_cvt_f32_i32 v223, v223\n" \
  "v_pk_mul_f32 v[220:221], v[220:221], v[88:89]\n" \
  "v_pk_mul_f32 v[222:223], v[222:223], v[88:89]\n" \
  "buffer_store_dwordx4 v[220:223], v47, s[20:23], s78 offen offset:448 sc0 sc1\n" \
  "v_cvt_f32_i32 v248, v248\n" \
  "v_cvt_f32_i32 v249, v249\n" \
  "v_cvt_f32_i32 v250, v250\n" \
  "v_cvt_f32_i32 v251, v251\n" \
  "v_pk_mul_f32 v[248:249], v[248:249], v[88:89]\n" \
  "v_pk_mul_f32 v[250:251], v[250:251], v[88:89]\n" \
  "buffer_store_dwordx4 v[248:251], v47, s[20:23], s79 offen offset:384 sc0 sc1\n" \
  "v_cvt_f32_i32 v252, v252\n" \
  "v_cvt_f32_i32 v253, v253\n" \
  "v_cvt_f32_i32 v254, v254\n" \
  "v_cvt_f32_i32 v255, v255\n" \
  "v_pk_mul_f32 v[252:253], v[252:253], v[88:89]\n" \
  "v_pk_mul_f32 v[254:255], v[254:255], v[88:89]\n" \
  "buffer_store_dwordx4 v[252:255], v47, s[20:23], s79 offen offset:448 sc0 sc1\n" \
  "s_mov_b64 exec, 1\n" \
  "global_store_dword v51, v76, s[32:33] sc0 sc1\n" \
  "s_mov_b64 exec, -1\n" \
  "s_waitcnt vmcnt(0)\n" \
  "s_branch L_endB_%=\n" \
  "Lslow12B_%=:\n" \
  "s_memrealtime s[96:97]\n" \
  "s_waitcnt lgkmcnt(0)\n" \
  "Lspin12B_%=:\n" \
  "s_sleep 4\n" \
  "global_load_dword v72, v50, s[38:39] sc0 sc1\n" \
  "global_load_dword v90, v51, s[12:13] sc1\n" \
  "s_waitcnt vmcnt(0)\n" \
  "v_sub_u32 v72, v72, v76\n" \
  "v_cmp_gt_i32 vcc, 0, v72\n" \
  "s_nop 1\n" \
  "s_andn2_b64 vcc, vcc, s[14:15]\n" \
  "s_cbranch_scc0 Lback13B_%=\n" \
  "v_cmp_eq_u32 vcc, v90, v76\n" \
  "s_nop 1\n" \
  "s_cmp_lg_u64 vcc, 0\n" \
  "s_cbranch_scc1 Ltmo12B_%=\n" \
  "s_memrealtime s[98:99]\n" \
  "s_waitcnt lgkmcnt(0)\n" \
  "s_sub_u32 s98, s98, s96\n" \
  "s_cmp_lt_u32 s98, 50000000\n" \
  "s_cbranch_scc1 Lspin12B_%=\n" \
  "Ltmo12B_%=:\n" \
  "global_store_dword v51, v76, s[12:13] sc1\n" \
  "s_mov_b32 s55, 1\n" \
  "s_branch Lback13B_%=\n" \
  "Lslow20B_%=:\n" \
  "s_memrealtime s[96:97]\n" \
  "s_waitcnt lgkmcnt(0)\n" \
  "Lspin20B_%=:\n" \
  "s_sleep 4\n" \
  "global_load_dword v72, v50, s[38:39] sc0 sc1\n" \
  "global_load_dword v90, v51, s[12:13] sc1\n" \
  "s_waitcnt vmcnt(0)\n" \
  "v_sub_u32 v72, v72, v76\n" \
  "v_cmp_gt_i32 vcc, 0, v72\n" \
  "s_nop 1\n" \
  "s_andn2_b64 vcc, vcc, s[14:15]\n" \
  "s_cbranch_scc0 Lback21B_%=\n" \
  "v_cmp_eq_u32 vcc, v90, v76\n" \
  "s_nop 1\n" \
  "s_cmp_lg_u64 vcc, 0\n" \
  "s_cbranch_scc1 Ltmo20B_%=\n" \
  "s_memrealtime s[98:99]\n" \
  "s_waitcnt lgkmcnt(0)\n" \
  "s_sub_u32 s98, s98, s96\n" \
  "s_cmp_lt_u32 s98, 50000000\n" \
  "s_cbranch_scc1 Lspin20B_%=\n" \
  "Ltmo20B_%=:\n" \
  "global_store_dword v51, v76, s[12:13] sc1\n" \
  "s_mov_b32 s55, 1\n" \
  "s_branch Lback21B_%=\n" \
  "Lslow28B_%=:\n" \
  "s_memrealtime s[96:97]\n" \
  "s_waitcnt lgkmcnt(0)\n" \
  "Lspin28B_%=:\n" \
  "s_sleep 4\n" \
  "global_load_dword v72, v50, s[38:39] sc0 sc1\n" \
  "global_load_dword v90, v51, s[12:13] sc1\n" \
  "s_waitcnt vmcnt(0)\n" \
  "v_sub_u32 v72, v72, v76\n" \
  "v_cmp_gt_i32 vcc, 0, v72\n" \
  "s_nop 1\n" \
  "s_andn2_b64 vcc, vcc, s[14:15]\n" \
  "s_cbranch_scc0 Lback29B_%=\n" \
  "v_cmp_eq_u32 vcc, v90, v76\n" \
  "s_nop 1\n" \
  "s_cmp_lg_u64 vcc, 0\n" \
  "s_cbranch_scc1 Ltmo28B_%=\n" \
  "s_memrealtime s[98:99]\n" \
  "s_waitcnt lgkmcnt(0)\n" \
  "s_sub_u32 s98, s98, s96\n" \
  "s_cmp_lt_u32 s98, 50000000\n" \
  "s_cbranch_scc1 Lspin28B_%=\n" \
  "Ltmo28B_%=:\n" \
  "global_store_dword v51, v76, s[12:13] sc1\n" \
  "s_mov_b32 s55, 1\n" \
  "s_branch Lback29B_%=\n" \
  "Lslow36B_%=:\n" \
  "s_memrealtime s[96:97]\n" \
  "s_waitcnt lgkmcnt(0)\n" \
  "Lspin36B_%=:\n" \
  "s_sleep 4\n" \
  "global_load_dword v72, v50, s[38:39] sc0 sc1\n" \
  "global_load_dword v90, v51, s[12:13] sc1\n" \
  "s_waitcnt vmcnt(0)\n" \
  "v_sub_u32 v72, v72, v76\n" \
  "v_cmp_gt_i32 vcc, 0, v72\n" \
  "s_nop 1\n" \
  "s_andn2_b64 vcc, vcc, s[14:15]\n" \
  "s_cbranch_scc0 Lback37B_%=\n" \
  "v_cmp_eq_u32 vcc, v90, v76\n" \
  "s_nop 1\n" \
  "s_cmp_lg_u64 vcc, 0\n" \
  "s_cbranch_scc1 Ltmo36B_%=\n" \
  "s_memrealtime s[98:99]\n" \
  "s_waitcnt lgkmcnt(0)\n" \
  "s_sub_u32 s98, s98, s96\n" \
  "s_cmp_lt_u32 s98, 50000000\n" \
  "s_cbranch_scc1 Lspin36B_%=\n" \
  "Ltmo36B_%=:\n" \
  "global_store_dword v51, v76, s[12:13] sc1\n" \
  "s_mov_b32 s55, 1\n" \
  "s_branch Lback37B_%=\n" \
  "L_endB_%=:\n" \
  "Lfin_%=:\n" \
  ""
#define A2A_GEMM_CLOB "v1","v2","v3","v4","v5","v6","v7","v8","v9","v10","v11","v12","v13","v14","v15","v16","v17","v18","v19","v20","v21","v22","v23","v24","v25","v26","v27","v28","v29","v30","v31","v32","v33","v34","v35","v36","v37","v38","v39","v40","v41","v42","v43","v44","v45","v46","v47","v48","v49","v50","v51","v52","v53","v54","v55","v56","v57","v58","v59","v60","v61","v62","v63","v64","v65","v66","v67","v68","v69","v70","v71","v72","v73","v74","v75","v76","v77","v78","v79","v80","v81","v82","v83","v84","v85","v86","v87","v88","v89","v90","v91","v92","v93","v94","v95","v96","v97","v98","v99","v100","v101","v102","v103","v104","v105","v106","v107","v108","v109","v110","v111","v112","v113","v114","v115","v116","v117","v118","v119","v120","v121","v122","v123","v124","v125","v126","v127","v128","v129","v130","v131","v132","v133","v134","v135","v136","v137","v138","v139","v140","v141","v142","v143","v144","v145","v146","v147","v148","v149","v150","v151","v152","v153","v154","v155","v156","v157","v158","v159","v160","v161","v162","v163","v164","v165","v166","v167","v168","v169","v170","v171","v172","v173","v174","v175","v176","v177","v178","v179","v180","v181","v182","v183","v184","v185","v186","v187","v188","v189","v190","v191","v192","v193","v194","v195","v196","v197","v198","v199","v200","v201","v202","v203","v204","v205","v206","v207","v208","v209","v210","v211","v212","v213","v214","v215","v216","v217","v218","v219","v220","v221","v222","v223","v224","v225","v226","v227","v228","v229","v230","v231","v232","v233","v234","v235","v236","v237","v238","v239","v240","v241","v242","v243","v244","v245","v246","v247","v248","v249","v250","v251","v252","v253","v254","v255","a0","a1","a2","a3","a4","a5","a6","a7","a8","a9","a10","a11","a12","a13","a14","a15","a16","a17","a18","a19","a20","a21","a22","a23","a24","a25","a26","a27","a28","a29","a30","a31","a32","a33","a34","a35","a36","a37","a38","a39","a40","a41","a42","a43","a44","a45","a46","a47","a48","a49","a50","a51","a52","a53","a54","a55","a56","a57","a58","a59","a60","a61","a62","a63","a64","a65","a66","a67","a68","a69","a70","a71","a72","a73","a74","a75","a76","a77","a78","a79","a80","a81","a82","a83","a84","a85","a86","a87","a88","a89","a90","a91","a92","a93","a94","a95","a96","a97","a98","a99","a100","a101","a102","a103","a104","a105","a106","a107","a108","a109","a110","a111","a112","a113","a114","a115","a116","a117","a118","a119","a120","a121","a122","a123","a124","a125","a126","a127","a128","a129","a130","a131","a132","a133","a134","a135","a136","a137","a138","a139","a140","a141","a142","a143","a144","a145","a146","a147","a148","a149","a150","a151","a152","a153","a154","a155","a156","a157","a158","a159","a160","a161","a162","a163","a164","a165","a166","a167","a168","a169","a170","a171","a172","a173","a174","a175","a176","a177","a178","a179","a180","a181","a182","a183","a184","a185","a186","a187","a188","a189","a190","a191","a192","a193","a194","a195","a196","a197","a198","a199","a200","a201","a202","a203","a204","a205","a206","a207","a208","a209","a210","a211","a212","a213","a214","a215","a216","a217","a218","a219","a220","a221","a222","a223","a224","a225","a226","a227","a228","a229","a230","a231","a232","a233","a234","a235","a236","a237","a238","a239","a240","a241","a242","a243","a244","a245","a246","a247","a248","a249","a250","a251","a252","a253","a254","a255","s3","s4","s5","s6","s7","s8","s9","s10","s11","s12","s13","s14","s15","s16","s17","s18","s19","s20","s21","s22","s23","s24","s25","s26","s27","s28","s29","s30","s31","s33","s34","s35","s36","s37","s38","s39","s40","s41","s42","s43","s44","s45","s46","s47","s48","s49","s50","s51","s52","s53","s54","s55","s56","s57","s58","s59","s60","s61","s62","s63","s64","s65","s66","s67","s68","s69","s70","s71","s72","s73","s74","s75","s76","s77","s78","s79","s80","s81","s82","s83","s84","s85","s86","s87","s88","s89","s90","s91","s92","s93","s94","s95","s96","s97","s98","s99","vcc","memory"


constexpr int M = 512, N = 16384, K = 4096, NRANK = 8;
constexpr int LDS_BYTES = 163840;
constexpr unsigned WS_CTR = 0;
constexpr unsigned WS_TMO = 16384;
constexpr unsigned WS_FLAGS = 32768;
constexpr unsigned WS_SLOT = 1u << 20;
constexpr unsigned SLOT_BYTES = 2u << 20;
constexpr size_t WS_NEED = (size_t)WS_SLOT + 2 * (size_t)SLOT_BYTES;

struct Params {
  const signed char* x; const signed char* W; const float* sx; const float* sw; float* out;
  char* myws; char* peer[7];
  void* unused; int rank; int pad;
};
static_assert(sizeof(Params) == 120, "argument offsets are hard-coded in the assembly");

__global__ __launch_bounds__(256) void a2a_gemm(Params p) {
  unsigned long long kp = (unsigned long long)__builtin_amdgcn_kernarg_segment_ptr();
  unsigned b = blockIdx.x, tid = threadIdx.x;
  asm volatile(A2A_GEMM_TEXT : "+{s[0:1]}"(kp), "+{s2}"(b), "+{v0}"(tid) : : A2A_GEMM_CLOB);
}

extern "C" void kernel_launch(int rank, int nranks, ncclComm_t comm, hipStream_t stream,
                              void* const* d_in, const int* in_sizes, int n_in,
                              void* d_out, int out_size, const int* in_dtypes, int out_dtype,
                              void* const* d_out_all, void* const* d_ws_all, size_t ws_bytes) {
  (void)comm; (void)in_dtypes; (void)out_dtype; (void)d_out_all;
  if (nranks != NRANK || (unsigned)rank >= (unsigned)NRANK || ws_bytes < WS_NEED || n_in != 4 || out_size != M * N ||
      in_sizes[0] != 4096 * 512 || in_sizes[1] != K * N) {
    fprintf(stderr, "a2a_gemm: unexpected configuration\n");
    return;
  }
  static bool init = false;
  if (!init) {
    if (hipFuncSetAttribute((const void*)a2a_gemm, hipFuncAttributeMaxDynamicSharedMemorySize, LDS_BYTES) != hipSuccess) {
      fprintf(stderr, "a2a_gemm: hipFuncSetAttribute failed\n");
      return;
    }
    init = true;
  }
  Params p;
  p.x = (const signed char*)d_in[0]; p.W = (const signed char*)d_in[1];
  p.sx = (const float*)d_in[2]; p.sw = (const float*)d_in[3];
  p.out = (float*)d_out;
  p.myws = (char*)d_ws_all[rank];
  for (int i = 0; i < 7; i++) p.peer[i] = (char*)d_ws_all[(rank + 1 + i) & 7];
  p.unused = nullptr; p.rank = rank; p.pad = 0;
  a2a_gemm<<<256, 256, LDS_BYTES, stream>>>(p);
}
```
